```python
import numpy as np

kernel_name = "gpu_gemm_ar_m4096_k4096_n16384_e4m3e4m3_dqsilu"


ABSMAX_RTOL = 0.03

COMPUTE_DTYPE = "float32"
EPILOGUE = "dequant_silu"
IN_QUANT = None
OUT_QUANT = None
IN_FP8 = "e4m3e4m3"
IN_DTYPES = {"x": "e4m3", "w_mat": "e4m3",
             "scale_x": "float32", "scale_w": "float32"}
WORLD_SIZE = 8

def _snap_fp8(x, mant_bits, max_abs, min_normal):
    a = np.abs(x.astype(np.float64))
    _m, _e = np.frexp(a)
    q = np.where(a >= min_normal, np.ldexp(1.0, _e - 1 - mant_bits),
                 min_normal * 2.0 ** -mant_bits)
    snapped = np.minimum(np.round(a / q) * q, max_abs)
    return (np.sign(x) * snapped).astype(np.float32)

def setup_inputs():

    x = _snap_fp8(np.random.randn(4096, 4096).astype(np.float32), 3, 448.0, 2.0 ** -6)
    w_mat = _snap_fp8(np.random.randn(4096, 16384).astype(np.float32), 3, 448.0, 2.0 ** -6)
    scale_x = (0.02 * (1.0 + np.random.rand(1))).astype(np.float32)
    scale_w = (0.02 * (1.0 + np.random.rand(1))).astype(np.float32)
    return {"x": x, "w_mat": w_mat, "scale_x": scale_x, "scale_w": scale_w}

def reference(x, w_mat, scale_x, scale_w):
    acc = (x @ w_mat).astype(np.float32)
    y = (acc * (scale_x * scale_w)).astype(np.float32)
    return (y / (1.0 + np.exp(-np.clip(y, -60.0, 60.0)))).astype(np.float32)
```

```cpp
#include <hip/hip_runtime.h>
#include <rccl/rccl.h>
#include <cstdio>

#define HIP_CHECK(expr) do { hipError_t _e = (expr); if (_e != hipSuccess) { \
    fprintf(stderr, "HIP error %s at %s:%d\n", hipGetErrorString(_e), __FILE__, __LINE__); \
    return; } } while (0)

#define GEMM_ASM \
  "s_and_b32 s53, %[call], 1\n\t" \
  "s_mul_i32 s53, s53, 83886080\n\t" \
  "s_add_u32 s53, s53, 1048576\n\t" \
  "s_add_u32 s36, %[ws_lo], s53\n\t" \
  "s_addc_u32 s37, %[ws_hi], 0\n\t" \
  "s_and_b32 s37, s37, 0xffff\n\t" \
  "s_mov_b32 s38, 83886080\n\t" \
  "s_mov_b32 s39, 0x20000\n\t" \
  "s_add_u32 s44, %[ws_lo], 4096\n\t" \
  "s_addc_u32 s45, %[ws_hi], 0\n\t" \
  "s_and_b32 s45, s45, 0xffff\n\t" \
  "s_mov_b32 s46, 77824\n\t" \
  "s_mov_b32 s47, 0x20000\n\t" \
  "s_mov_b32 s40, %[out_lo]\n\t" \
  "s_and_b32 s41, %[out_hi], 0xffff\n\t" \
  "s_mov_b32 s42, 0\n\t" \
  "s_mov_b32 s43, 0x20000\n\t" \
  "s_add_u32 s86, %[ws_lo], 128\n\t" \
  "s_addc_u32 s87, %[ws_hi], 0\n\t" \
  "s_mov_b32 s74, 268435456\n\t" \
  "s_lshl_b32 s48, %[wave], 12\n\t" \
  "s_mov_b32 s62, %[tl0]\n\t" \
  "s_mov_b32 s63, %[tl1]\n\t" \
  "s_mov_b32 s64, %[tl2]\n\t" \
  "s_mov_b32 s65, 0\n\t" \
  "s_load_dword s54, %[psx], 0x0\n\t" \
  "s_load_dword s55, %[psw], 0x0\n\t" \
  "s_waitcnt lgkmcnt(0)\n\t" \
  "v_mov_b32 v9, s54\n\t" \
  "v_mul_f32_e32 v9, s55, v9\n\t" \
  "v_mul_f32_e32 v9, 0xbfb8aa3b, v9\n\t" \
  "s_nop 1\n\t" \
  "v_readfirstlane_b32 s56, v9\n\t" \
  "v_readfirstlane_b32 s57, v9\n\t" \
  "s_mov_b32 s58, 0xbfb8aa3b\n\t" \
  "s_mov_b32 s59, 0xbfb8aa3b\n\t" \
  "s_mov_b32 s60, 0x42ad1f97\n\t" \
  "s_mov_b32 s70, 16777216\n\t" \
  "s_lshl_b32 s52, %[pm], 20\n\t" \
  "s_lshl_b32 s71, %[pm], 24\n\t" \
  "s_and_b32 s53, %[pm], 3\n\t" \
  "s_lshl_b32 s53, s53, 21\n\t" \
  "s_or_b32 s71, s71, s53\n\t" \
  "s_lshr_b32 s53, %[wave], 1\n\t" \
  "s_lshl_b32 s53, s53, 20\n\t" \
  "s_or_b32 s71, s71, s53\n\t" \
  "s_mov_b32 s88, %[call]\n\t" \
  "s_lshl_b32 s75, %[call], 7\n\t" \
  "s_mov_b32 s77, 0\n\t" \
  "s_mov_b32 s61, 0\n\t" \
  "s_getpc_b64 s[78:79]\n\t" \
  "L_PC_%=:\n\t" \
  "s_add_u32 s78, s78, L_POLL_%=-L_PC_%=\n\t" \
  "s_addc_u32 s79, s79, 0\n\t" \
  "v_mbcnt_lo_u32_b32 v9, -1, 0\n\t" \
  "v_mbcnt_hi_u32_b32 v9, -1, v9\n\t" \
  "v_lshlrev_b32_e32 v11, 4, v9\n\t" \
  "v_add_u32_e32 v0, s48, v11\n\t" \
  "s_lshr_b32 s53, %[wave], 1\n\t" \
  "s_lshl_b32 s53, s53, 12\n\t" \
  "s_and_b32 s54, %[wave], 1\n\t" \
  "s_lshl_b32 s54, s54, 12\n\t" \
  "v_add_u32_e32 v1, s53, v11\n\t" \
  "v_add_u32_e32 v2, 0x10000, v1\n\t" \
  "v_add_u32_e32 v3, s54, v11\n\t" \
  "v_add_u32_e32 v4, 0x10000, v3\n\t" \
  "s_lshl_b32 s55, %[wave], 13\n\t" \
  "s_add_u32 s55, s55, 131072\n\t" \
  "v_add_u32_e32 v6, s55, v11\n\t" \
  "s_mov_b32 s92, %[fbase]\n\t" \
  "s_cmp_lg_u32 s92, 0\n\t" \
  "s_cselect_b32 s55, 4096, 0\n\t" \
  "s_cselect_b32 s82, 1024, 0\n\t" \
  "v_and_b32_e32 v8, 7, v9\n\t" \
  "v_lshlrev_b32_e32 v8, 7, v8\n\t" \
  "v_and_b32_e32 v10, 8, v9\n\t" \
  "v_cmp_eq_u32_e32 vcc, 0, v10\n\t" \
  "v_mov_b32 v10, s92\n\t" \
  "v_mov_b32 v7, s55\n\t" \
  "s_nop 1\n\t" \
  "v_cndmask_b32_e32 v10, v7, v10, vcc\n\t" \
  "v_bfe_u32 v7, v9, 3, 2\n\t" \
  "v_mul_u32_u24_e32 v7, s82, v7\n\t" \
  "v_cmp_gt_u32_e32 vcc, 32, v9\n\t" \
  "s_nop 1\n\t" \
  "v_cndmask_b32_e32 v10, v7, v10, vcc\n\t" \
  "v_add_u32_e32 v8, v10, v8\n\t" \
  "v_lshrrev_b32_e32 v11, 4, v9\n\t" \
  "v_lshlrev_b32_e32 v11, 18, v11\n\t" \
  "v_and_b32_e32 v9, 15, v9\n\t" \
  "v_lshlrev_b32_e32 v9, 3, v9\n\t" \
  "v_add_u32_e32 v5, v9, v11\n\t" \
  "s_lshl_b32 s53, s53, 11\n\t" \
  "s_lshr_b32 s54, s54, 4\n\t" \
  "s_add_u32 s53, s53, s54\n\t" \
  "v_add_u32_e32 v5, s53, v5\n\t" \
  "v_mov_b32 v10, 0\n\t" \
  "s_cmp_lg_u32 %[wave], 0\n\t" \
  "s_cbranch_scc1 L_CPFE_%=\n\t" \
  "s_getpc_b64 s[82:83]\n\t" \
  "L_CPF0_%=:\n\t" \
  "v_mbcnt_lo_u32_b32 v9, -1, 0\n\t" \
  "v_mbcnt_hi_u32_b32 v9, -1, v9\n\t" \
  "v_lshlrev_b32_e32 v9, 7, v9\n\t" \
  "L_CPF_%=:\n\t" \
  "v_cmp_gt_u32_e32 vcc, L_END_%=-L_CPF0_%=, v9\n\t" \
  "s_and_saveexec_b64 s[84:85], vcc\n\t" \
  "global_load_dword v11, v9, s[82:83]\n\t" \
  "s_mov_b64 exec, s[84:85]\n\t" \
  "v_add_u32_e32 v9, 0x2000, v9\n\t" \
  "v_readfirstlane_b32 s54, v9\n\t" \
  "s_nop 3\n\t" \
  "s_cmp_lt_u32 s54, L_END_%=-L_CPF0_%=\n\t" \
  "s_cbranch_scc1 L_CPF_%=\n\t" \
  "s_waitcnt vmcnt(0)\n\t" \
  "L_CPFE_%=:\n\t" \
  "v_accvgpr_write_b32 a0, 0\n\t" \
  "v_accvgpr_write_b32 a1, 0\n\t" \
  "v_accvgpr_write_b32 a2, 0\n\t" \
  "v_accvgpr_write_b32 a3, 0\n\t" \
  "v_accvgpr_write_b32 a4, 0\n\t" \
  "v_accvgpr_write_b32 a5, 0\n\t" \
  "v_accvgpr_write_b32 a6, 0\n\t" \
  "v_accvgpr_write_b32 a7, 0\n\t" \
  "v_accvgpr_write_b32 a8, 0\n\t" \
  "v_accvgpr_write_b32 a9, 0\n\t" \
  "v_accvgpr_write_b32 a10, 0\n\t" \
  "v_accvgpr_write_b32 a11, 0\n\t" \
  "v_accvgpr_write_b32 a12, 0\n\t" \
  "v_accvgpr_write_b32 a13, 0\n\t" \
  "v_accvgpr_write_b32 a14, 0\n\t" \
  "v_accvgpr_write_b32 a15, 0\n\t" \
  "v_accvgpr_write_b32 a16, 0\n\t" \
  "v_accvgpr_write_b32 a17, 0\n\t" \
  "v_accvgpr_write_b32 a18, 0\n\t" \
  "v_accvgpr_write_b32 a19, 0\n\t" \
  "v_accvgpr_write_b32 a20, 0\n\t" \
  "v_accvgpr_write_b32 a21, 0\n\t" \
  "v_accvgpr_write_b32 a22, 0\n\t" \
  "v_accvgpr_write_b32 a23, 0\n\t" \
  "v_accvgpr_write_b32 a24, 0\n\t" \
  "v_accvgpr_write_b32 a25, 0\n\t" \
  "v_accvgpr_write_b32 a26, 0\n\t" \
  "v_accvgpr_write_b32 a27, 0\n\t" \
  "v_accvgpr_write_b32 a28, 0\n\t" \
  "v_accvgpr_write_b32 a29, 0\n\t" \
  "v_accvgpr_write_b32 a30, 0\n\t" \
  "v_accvgpr_write_b32 a31, 0\n\t" \
  "v_accvgpr_write_b32 a32, 0\n\t" \
  "v_accvgpr_write_b32 a33, 0\n\t" \
  "v_accvgpr_write_b32 a34, 0\n\t" \
  "v_accvgpr_write_b32 a35, 0\n\t" \
  "v_accvgpr_write_b32 a36, 0\n\t" \
  "v_accvgpr_write_b32 a37, 0\n\t" \
  "v_accvgpr_write_b32 a38, 0\n\t" \
  "v_accvgpr_write_b32 a39, 0\n\t" \
  "v_accvgpr_write_b32 a40, 0\n\t" \
  "v_accvgpr_write_b32 a41, 0\n\t" \
  "v_accvgpr_write_b32 a42, 0\n\t" \
  "v_accvgpr_write_b32 a43, 0\n\t" \
  "v_accvgpr_write_b32 a44, 0\n\t" \
  "v_accvgpr_write_b32 a45, 0\n\t" \
  "v_accvgpr_write_b32 a46, 0\n\t" \
  "v_accvgpr_write_b32 a47, 0\n\t" \
  "v_accvgpr_write_b32 a48, 0\n\t" \
  "v_accvgpr_write_b32 a49, 0\n\t" \
  "v_accvgpr_write_b32 a50, 0\n\t" \
  "v_accvgpr_write_b32 a51, 0\n\t" \
  "v_accvgpr_write_b32 a52, 0\n\t" \
  "v_accvgpr_write_b32 a53, 0\n\t" \
  "v_accvgpr_write_b32 a54, 0\n\t" \
  "v_accvgpr_write_b32 a55, 0\n\t" \
  "v_accvgpr_write_b32 a56, 0\n\t" \
  "v_accvgpr_write_b32 a57, 0\n\t" \
  "v_accvgpr_write_b32 a58, 0\n\t" \
  "v_accvgpr_write_b32 a59, 0\n\t" \
  "v_accvgpr_write_b32 a60, 0\n\t" \
  "v_accvgpr_write_b32 a61, 0\n\t" \
  "v_accvgpr_write_b32 a62, 0\n\t" \
  "v_accvgpr_write_b32 a63, 0\n\t" \
  "v_accvgpr_write_b32 a64, 0\n\t" \
  "v_accvgpr_write_b32 a65, 0\n\t" \
  "v_accvgpr_write_b32 a66, 0\n\t" \
  "v_accvgpr_write_b32 a67, 0\n\t" \
  "v_accvgpr_write_b32 a68, 0\n\t" \
  "v_accvgpr_write_b32 a69, 0\n\t" \
  "v_accvgpr_write_b32 a70, 0\n\t" \
  "v_accvgpr_write_b32 a71, 0\n\t" \
  "v_accvgpr_write_b32 a72, 0\n\t" \
  "v_accvgpr_write_b32 a73, 0\n\t" \
  "v_accvgpr_write_b32 a74, 0\n\t" \
  "v_accvgpr_write_b32 a75, 0\n\t" \
  "v_accvgpr_write_b32 a76, 0\n\t" \
  "v_accvgpr_write_b32 a77, 0\n\t" \
  "v_accvgpr_write_b32 a78, 0\n\t" \
  "v_accvgpr_write_b32 a79, 0\n\t" \
  "v_accvgpr_write_b32 a80, 0\n\t" \
  "v_accvgpr_write_b32 a81, 0\n\t" \
  "v_accvgpr_write_b32 a82, 0\n\t" \
  "v_accvgpr_write_b32 a83, 0\n\t" \
  "v_accvgpr_write_b32 a84, 0\n\t" \
  "v_accvgpr_write_b32 a85, 0\n\t" \
  "v_accvgpr_write_b32 a86, 0\n\t" \
  "v_accvgpr_write_b32 a87, 0\n\t" \
  "v_accvgpr_write_b32 a88, 0\n\t" \
  "v_accvgpr_write_b32 a89, 0\n\t" \
  "v_accvgpr_write_b32 a90, 0\n\t" \
  "v_accvgpr_write_b32 a91, 0\n\t" \
  "v_accvgpr_write_b32 a92, 0\n\t" \
  "v_accvgpr_write_b32 a93, 0\n\t" \
  "v_accvgpr_write_b32 a94, 0\n\t" \
  "v_accvgpr_write_b32 a95, 0\n\t" \
  "v_accvgpr_write_b32 a96, 0\n\t" \
  "v_accvgpr_write_b32 a97, 0\n\t" \
  "v_accvgpr_write_b32 a98, 0\n\t" \
  "v_accvgpr_write_b32 a99, 0\n\t" \
  "v_accvgpr_write_b32 a100, 0\n\t" \
  "v_accvgpr_write_b32 a101, 0\n\t" \
  "v_accvgpr_write_b32 a102, 0\n\t" \
  "v_accvgpr_write_b32 a103, 0\n\t" \
  "v_accvgpr_write_b32 a104, 0\n\t" \
  "v_accvgpr_write_b32 a105, 0\n\t" \
  "v_accvgpr_write_b32 a106, 0\n\t" \
  "v_accvgpr_write_b32 a107, 0\n\t" \
  "v_accvgpr_write_b32 a108, 0\n\t" \
  "v_accvgpr_write_b32 a109, 0\n\t" \
  "v_accvgpr_write_b32 a110, 0\n\t" \
  "v_accvgpr_write_b32 a111, 0\n\t" \
  "v_accvgpr_write_b32 a112, 0\n\t" \
  "v_accvgpr_write_b32 a113, 0\n\t" \
  "v_accvgpr_write_b32 a114, 0\n\t" \
  "v_accvgpr_write_b32 a115, 0\n\t" \
  "v_accvgpr_write_b32 a116, 0\n\t" \
  "v_accvgpr_write_b32 a117, 0\n\t" \
  "v_accvgpr_write_b32 a118, 0\n\t" \
  "v_accvgpr_write_b32 a119, 0\n\t" \
  "v_accvgpr_write_b32 a120, 0\n\t" \
  "v_accvgpr_write_b32 a121, 0\n\t" \
  "v_accvgpr_write_b32 a122, 0\n\t" \
  "v_accvgpr_write_b32 a123, 0\n\t" \
  "v_accvgpr_write_b32 a124, 0\n\t" \
  "v_accvgpr_write_b32 a125, 0\n\t" \
  "v_accvgpr_write_b32 a126, 0\n\t" \
  "v_accvgpr_write_b32 a127, 0\n\t" \
  "v_accvgpr_write_b32 a128, 0\n\t" \
  "v_accvgpr_write_b32 a129, 0\n\t" \
  "v_accvgpr_write_b32 a130, 0\n\t" \
  "v_accvgpr_write_b32 a131, 0\n\t" \
  "v_accvgpr_write_b32 a132, 0\n\t" \
  "v_accvgpr_write_b32 a133, 0\n\t" \
  "v_accvgpr_write_b32 a134, 0\n\t" \
  "v_accvgpr_write_b32 a135, 0\n\t" \
  "v_accvgpr_write_b32 a136, 0\n\t" \
  "v_accvgpr_write_b32 a137, 0\n\t" \
  "v_accvgpr_write_b32 a138, 0\n\t" \
  "v_accvgpr_write_b32 a139, 0\n\t" \
  "v_accvgpr_write_b32 a140, 0\n\t" \
  "v_accvgpr_write_b32 a141, 0\n\t" \
  "v_accvgpr_write_b32 a142, 0\n\t" \
  "v_accvgpr_write_b32 a143, 0\n\t" \
  "v_accvgpr_write_b32 a144, 0\n\t" \
  "v_accvgpr_write_b32 a145, 0\n\t" \
  "v_accvgpr_write_b32 a146, 0\n\t" \
  "v_accvgpr_write_b32 a147, 0\n\t" \
  "v_accvgpr_write_b32 a148, 0\n\t" \
  "v_accvgpr_write_b32 a149, 0\n\t" \
  "v_accvgpr_write_b32 a150, 0\n\t" \
  "v_accvgpr_write_b32 a151, 0\n\t" \
  "v_accvgpr_write_b32 a152, 0\n\t" \
  "v_accvgpr_write_b32 a153, 0\n\t" \
  "v_accvgpr_write_b32 a154, 0\n\t" \
  "v_accvgpr_write_b32 a155, 0\n\t" \
  "v_accvgpr_write_b32 a156, 0\n\t" \
  "v_accvgpr_write_b32 a157, 0\n\t" \
  "v_accvgpr_write_b32 a158, 0\n\t" \
  "v_accvgpr_write_b32 a159, 0\n\t" \
  "v_accvgpr_write_b32 a160, 0\n\t" \
  "v_accvgpr_write_b32 a161, 0\n\t" \
  "v_accvgpr_write_b32 a162, 0\n\t" \
  "v_accvgpr_write_b32 a163, 0\n\t" \
  "v_accvgpr_write_b32 a164, 0\n\t" \
  "v_accvgpr_write_b32 a165, 0\n\t" \
  "v_accvgpr_write_b32 a166, 0\n\t" \
  "v_accvgpr_write_b32 a167, 0\n\t" \
  "v_accvgpr_write_b32 a168, 0\n\t" \
  "v_accvgpr_write_b32 a169, 0\n\t" \
  "v_accvgpr_write_b32 a170, 0\n\t" \
  "v_accvgpr_write_b32 a171, 0\n\t" \
  "v_accvgpr_write_b32 a172, 0\n\t" \
  "v_accvgpr_write_b32 a173, 0\n\t" \
  "v_accvgpr_write_b32 a174, 0\n\t" \
  "v_accvgpr_write_b32 a175, 0\n\t" \
  "v_accvgpr_write_b32 a176, 0\n\t" \
  "v_accvgpr_write_b32 a177, 0\n\t" \
  "v_accvgpr_write_b32 a178, 0\n\t" \
  "v_accvgpr_write_b32 a179, 0\n\t" \
  "v_accvgpr_write_b32 a180, 0\n\t" \
  "v_accvgpr_write_b32 a181, 0\n\t" \
  "v_accvgpr_write_b32 a182, 0\n\t" \
  "v_accvgpr_write_b32 a183, 0\n\t" \
  "v_accvgpr_write_b32 a184, 0\n\t" \
  "v_accvgpr_write_b32 a185, 0\n\t" \
  "v_accvgpr_write_b32 a186, 0\n\t" \
  "v_accvgpr_write_b32 a187, 0\n\t" \
  "v_accvgpr_write_b32 a188, 0\n\t" \
  "v_accvgpr_write_b32 a189, 0\n\t" \
  "v_accvgpr_write_b32 a190, 0\n\t" \
  "v_accvgpr_write_b32 a191, 0\n\t" \
  "v_accvgpr_write_b32 a192, 0\n\t" \
  "v_accvgpr_write_b32 a193, 0\n\t" \
  "v_accvgpr_write_b32 a194, 0\n\t" \
  "v_accvgpr_write_b32 a195, 0\n\t" \
  "v_accvgpr_write_b32 a196, 0\n\t" \
  "v_accvgpr_write_b32 a197, 0\n\t" \
  "v_accvgpr_write_b32 a198, 0\n\t" \
  "v_accvgpr_write_b32 a199, 0\n\t" \
  "v_accvgpr_write_b32 a200, 0\n\t" \
  "v_accvgpr_write_b32 a201, 0\n\t" \
  "v_accvgpr_write_b32 a202, 0\n\t" \
  "v_accvgpr_write_b32 a203, 0\n\t" \
  "v_accvgpr_write_b32 a204, 0\n\t" \
  "v_accvgpr_write_b32 a205, 0\n\t" \
  "v_accvgpr_write_b32 a206, 0\n\t" \
  "v_accvgpr_write_b32 a207, 0\n\t" \
  "v_accvgpr_write_b32 a208, 0\n\t" \
  "v_accvgpr_write_b32 a209, 0\n\t" \
  "v_accvgpr_write_b32 a210, 0\n\t" \
  "v_accvgpr_write_b32 a211, 0\n\t" \
  "v_accvgpr_write_b32 a212, 0\n\t" \
  "v_accvgpr_write_b32 a213, 0\n\t" \
  "v_accvgpr_write_b32 a214, 0\n\t" \
  "v_accvgpr_write_b32 a215, 0\n\t" \
  "v_accvgpr_write_b32 a216, 0\n\t" \
  "v_accvgpr_write_b32 a217, 0\n\t" \
  "v_accvgpr_write_b32 a218, 0\n\t" \
  "v_accvgpr_write_b32 a219, 0\n\t" \
  "v_accvgpr_write_b32 a220, 0\n\t" \
  "v_accvgpr_write_b32 a221, 0\n\t" \
  "v_accvgpr_write_b32 a222, 0\n\t" \
  "v_accvgpr_write_b32 a223, 0\n\t" \
  "v_accvgpr_write_b32 a224, 0\n\t" \
  "v_accvgpr_write_b32 a225, 0\n\t" \
  "v_accvgpr_write_b32 a226, 0\n\t" \
  "v_accvgpr_write_b32 a227, 0\n\t" \
  "v_accvgpr_write_b32 a228, 0\n\t" \
  "v_accvgpr_write_b32 a229, 0\n\t" \
  "v_accvgpr_write_b32 a230, 0\n\t" \
  "v_accvgpr_write_b32 a231, 0\n\t" \
  "v_accvgpr_write_b32 a232, 0\n\t" \
  "v_accvgpr_write_b32 a233, 0\n\t" \
  "v_accvgpr_write_b32 a234, 0\n\t" \
  "v_accvgpr_write_b32 a235, 0\n\t" \
  "v_accvgpr_write_b32 a236, 0\n\t" \
  "v_accvgpr_write_b32 a237, 0\n\t" \
  "v_accvgpr_write_b32 a238, 0\n\t" \
  "v_accvgpr_write_b32 a239, 0\n\t" \
  "v_accvgpr_write_b32 a240, 0\n\t" \
  "v_accvgpr_write_b32 a241, 0\n\t" \
  "v_accvgpr_write_b32 a242, 0\n\t" \
  "v_accvgpr_write_b32 a243, 0\n\t" \
  "v_accvgpr_write_b32 a244, 0\n\t" \
  "v_accvgpr_write_b32 a245, 0\n\t" \
  "v_accvgpr_write_b32 a246, 0\n\t" \
  "v_accvgpr_write_b32 a247, 0\n\t" \
  "v_accvgpr_write_b32 a248, 0\n\t" \
  "v_accvgpr_write_b32 a249, 0\n\t" \
  "v_accvgpr_write_b32 a250, 0\n\t" \
  "v_accvgpr_write_b32 a251, 0\n\t" \
  "v_accvgpr_write_b32 a252, 0\n\t" \
  "v_accvgpr_write_b32 a253, 0\n\t" \
  "v_accvgpr_write_b32 a254, 0\n\t" \
  "v_accvgpr_write_b32 a255, 0\n\t" \
  "s_and_b32 s54, s62, 0xff\n\t" \
  "s_lshr_b64 s[62:63], s[62:63], 8\n\t" \
  "s_lshl_b32 s55, s64, 24\n\t" \
  "s_or_b32 s63, s63, s55\n\t" \
  "s_lshr_b64 s[64:65], s[64:65], 8\n\t" \
  "s_and_b32 s54, s54, 0x7f\n\t" \
  "s_lshl_b32 s69, s54, 9\n\t" \
  "s_add_u32 s69, s69, s71\n\t" \
  "s_lshl_b32 s54, s54, 19\n\t" \
  "s_add_u32 s54, s54, s70\n\t" \
  "s_mov_b32 s66, s69\n\t" \
  "s_mov_b32 s67, s69\n\t" \
  "s_lshl_b32 s53, %[start_odd], 18\n\t" \
  "s_add_u32 s50, s54, s53\n\t" \
  "s_mov_b32 s51, s50\n\t" \
  "s_lshl_b32 s53, %[start_odd], 19\n\t" \
  "s_add_u32 s49, s52, s53\n\t" \
  "s_mov_b32 s72, %[nwin]\n\t" \
  "s_mov_b32 s76, 0\n\t" \
  "s_swappc_b64 s[80:81], s[78:79]\n\t" \
  "s_add_u32 s76, s76, 1024\n\t" \
  "s_cmp_eq_u32 %[start_odd], 0\n\t" \
  "s_cbranch_scc1 L_NS_%=\n\t" \
  "s_branch L_NP_%=\n\t" \
  "L_WE_%=:\n\t" \
  "s_waitcnt lgkmcnt(0)\n\t" \
  "v_mfma_f32_16x16x128_f8f6f4 a[0:3], v[128:135], v[192:199], 0\n\t" \
  "s_waitcnt vmcnt(16)\n\t" \
  "s_barrier\n\t" \
  "v_mfma_f32_16x16x128_f8f6f4 a[4:7], v[128:135], v[200:207], 0\n\t" \
  "ds_read_b128 v[224:227], v3 offset:32768\n\t" \
  "ds_read_b128 v[228:231], v3 offset:40960\n\t" \
  "s_mov_b32 s68, s66\n\t" \
  "s_mov_b32 s66, s69\n\t" \
  "v_accvgpr_read_b32 v112, a160\n\t" \
  "v_mfma_f32_16x16x128_f8f6f4 a[8:11], v[128:135], v[208:215], 0\n\t" \
  "ds_read_b128 v[232:235], v3 offset:33792\n\t" \
  "ds_read_b128 v[236:239], v3 offset:41984\n\t" \
  "s_cmp_lt_u32 s77, 2\n\t" \
  "s_cselect_b32 s42, 0, s74\n\t" \
  "v_accvgpr_read_b32 v116, a161\n\t" \
  "v_mfma_f32_16x16x128_f8f6f4 a[12:15], v[128:135], v[216:223], 0\n\t" \
  "ds_read_b128 v[240:243], v3 offset:34816\n\t" \
  "ds_read_b128 v[244:247], v3 offset:43008\n\t" \
  "s_add_u32 s77, s77, 1\n\t" \
  "v_accvgpr_read_b32 v120, a162\n\t" \
  "v_accvgpr_read_b32 v124, a163\n\t" \
  "v_mfma_f32_16x16x128_f8f6f4 a[44:47], v[136:143], v[216:223], 0\n\t" \
  "ds_read_b128 v[248:251], v3 offset:35840\n\t" \
  "ds_read_b128 v[252:255], v3 offset:44032\n\t" \
  "v_accvgpr_read_b32 v113, a164\n\t" \
  "v_accvgpr_read_b32 v117, a165\n\t" \
  "v_accvgpr_read_b32 v121, a166\n\t" \
  "v_mfma_f32_16x16x128_f8f6f4 a[40:43], v[136:143], v[208:215], 0\n\t" \
  "v_accvgpr_read_b32 v125, a167\n\t" \
  "v_accvgpr_read_b32 v114, a168\n\t" \
  "v_accvgpr_read_b32 v118, a169\n\t" \
  "v_accvgpr_read_b32 v122, a170\n\t" \
  "v_mfma_f32_16x16x128_f8f6f4 a[36:39], v[136:143], v[200:207], 0\n\t" \
  "v_accvgpr_read_b32 v126, a171\n\t" \
  "v_accvgpr_read_b32 v115, a172\n\t" \
  "v_accvgpr_read_b32 v119, a173\n\t" \
  "v_accvgpr_read_b32 v123, a174\n\t" \
  "v_mfma_f32_16x16x128_f8f6f4 a[32:35], v[136:143], v[192:199], 0\n\t" \
  "s_mov_b32 s53, s49\n\t" \
  "s_mov_b32 m0, s48\n\t" \
  "v_accvgpr_read_b32 v127, a175\n\t" \
  "ds_write_b128 v6, a[192:195] offset:0\n\t" \
  "ds_write_b128 v6, a[196:199] offset:1024\n\t" \
  "v_mfma_f32_16x16x128_f8f6f4 a[64:67], v[144:151], v[192:199], 0\n\t" \
  "buffer_load_dwordx4 v0, s[36:39], s53 offen sc0 sc1 lds\n\t" \
  "ds_write_b128 v6, a[200:203] offset:2048\n\t" \
  "ds_write_b128 v6, a[204:207] offset:3072\n\t" \
  "ds_write_b128 v6, a[224:227] offset:4096\n\t" \
  "v_mfma_f32_16x16x128_f8f6f4 a[68:71], v[144:151], v[200:207], 0\n\t" \
  "ds_write_b128 v6, a[228:231] offset:5120\n\t" \
  "ds_write_b128 v6, a[232:235] offset:6144\n\t" \
  "ds_write_b128 v6, a[236:239] offset:7168\n\t" \
  "v_mfma_f32_16x16x128_f8f6f4 a[72:75], v[144:151], v[208:215], 0\n\t" \
  "buffer_load_dwordx4 v0, s[36:39], s53 offen offset:1024 sc0 sc1 lds\n\t" \
  "v_mfma_f32_16x16x128_f8f6f4 a[76:79], v[144:151], v[216:223], 0\n\t" \
  "v_mfma_f32_16x16x128_f8f6f4 a[108:111], v[152:159], v[216:223], 0\n\t" \
  "buffer_load_dwordx4 v0, s[36:39], s53 offen offset:2048 sc0 sc1 lds\n\t" \
  "v_mfma_f32_16x16x128_f8f6f4 a[104:107], v[152:159], v[208:215], 0\n\t" \
  "v_mfma_f32_16x16x128_f8f6f4 a[100:103], v[152:159], v[200:207], 0\n\t" \
  "buffer_load_dwordx4 v0, s[36:39], s53 offen offset:3072 sc0 sc1 lds\n\t" \
  "v_mfma_f32_16x16x128_f8f6f4 a[96:99], v[152:159], v[192:199], 0\n\t" \
  "s_waitcnt lgkmcnt(0)\n\t" \
  "v_mfma_f32_16x16x128_f8f6f4 a[16:19], v[128:135], v[224:231], a[16:19]\n\t" \
  "v_mfma_f32_16x16x128_f8f6f4 a[20:23], v[128:135], v[232:239], a[20:23]\n\t" \
  "ds_read_b128 v[160:163], v1 offset:49152\n\t" \
  "ds_read_b128 v[164:167], v1 offset:57344\n\t" \
  "v_mfma_f32_16x16x128_f8f6f4 a[24:27], v[128:135], v[240:247], a[24:27]\n\t" \
  "ds_read_b128 v[168:171], v1 offset:50176\n\t" \
  "ds_read_b128 v[172:175], v1 offset:58368\n\t" \
  "v_mfma_f32_16x16x128_f8f6f4 a[28:31], v[128:135], v[248:255], a[28:31]\n\t" \
  "ds_read_b128 v[176:179], v1 offset:51200\n\t" \
  "ds_read_b128 v[180:183], v1 offset:59392\n\t" \
  "v_mfma_f32_16x16x128_f8f6f4 a[60:63], v[136:143], v[248:255], a[60:63]\n\t" \
  "ds_read_b128 v[184:187], v1 offset:52224\n\t" \
  "ds_read_b128 v[188:191], v1 offset:60416\n\t" \
  "v_mfma_f32_16x16x128_f8f6f4 a[56:59], v[136:143], v[240:247], a[56:59]\n\t" \
  "v_mfma_f32_16x16x128_f8f6f4 a[52:55], v[136:143], v[232:239], a[52:55]\n\t" \
  "v_mfma_f32_16x16x128_f8f6f4 a[48:51], v[136:143], v[224:231], a[48:51]\n\t" \
  "s_mov_b32 s53, s50\n\t" \
  "s_add_u32 m0, s48, 16384\n\t" \
  "v_mfma_f32_16x16x128_f8f6f4 a[80:83], v[144:151], v[224:231], a[80:83]\n\t" \
  "buffer_load_dwordx4 v0, s[36:39], s53 offen sc0 sc1 lds\n\t" \
  "v_mfma_f32_16x16x128_f8f6f4 a[84:87], v[144:151], v[232:239], a[84:87]\n\t" \
  "v_mfma_f32_16x16x128_f8f6f4 a[88:91], v[144:151], v[240:247], a[88:91]\n\t" \
  "buffer_load_dwordx4 v0, s[36:39], s53 offen offset:1024 sc0 sc1 lds\n\t" \
  "v_mfma_f32_16x16x128_f8f6f4 a[92:95], v[144:151], v[248:255], a[92:95]\n\t" \
  "v_mfma_f32_16x16x128_f8f6f4 a[124:127], v[152:159], v[248:255], a[124:127]\n\t" \
  "buffer_load_dwordx4 v0, s[36:39], s53 offen offset:2048 sc0 sc1 lds\n\t" \
  "v_mfma_f32_16x16x128_f8f6f4 a[120:123], v[152:159], v[240:247], a[120:123]\n\t" \
  "v_mfma_f32_16x16x128_f8f6f4 a[116:119], v[152:159], v[232:239], a[116:119]\n\t" \
  "buffer_load_dwordx4 v0, s[36:39], s53 offen offset:3072 sc0 sc1 lds\n\t" \
  "v_mfma_f32_16x16x128_f8f6f4 a[112:115], v[152:159], v[224:231], a[112:115]\n\t" \
  "s_waitcnt lgkmcnt(0)\n\t" \
  "v_mfma_f32_16x16x128_f8f6f4 a[144:147], v[160:167], v[224:231], a[144:147]\n\t" \
  "s_waitcnt vmcnt(16)\n\t" \
  "s_barrier\n\t" \
  "v_mfma_f32_16x16x128_f8f6f4 a[148:151], v[160:167], v[232:239], a[148:151]\n\t" \
  "ds_read_b128 v[128:131], v2 offset:0\n\t" \
  "ds_read_b128 v[132:135], v2 offset:8192\n\t" \
  "v_mfma_f32_16x16x128_f8f6f4 a[152:155], v[160:167], v[240:247], a[152:155]\n\t" \
  "ds_read_b128 v[136:139], v2 offset:1024\n\t" \
  "ds_read_b128 v[140:143], v2 offset:9216\n\t" \
  "v_mfma_f32_16x16x128_f8f6f4 a[156:159], v[160:167], v[248:255], a[156:159]\n\t" \
  "ds_read_b128 v[144:147], v2 offset:2048\n\t" \
  "ds_read_b128 v[148:151], v2 offset:10240\n\t" \
  "v_mfma_f32_16x16x128_f8f6f4 a[188:191], v[168:175], v[248:255], a[188:191]\n\t" \
  "ds_read_b128 v[152:155], v2 offset:3072\n\t" \
  "ds_read_b128 v[156:159], v2 offset:11264\n\t" \
  "v_mfma_f32_16x16x128_f8f6f4 a[184:187], v[168:175], v[240:247], a[184:187]\n\t" \
  "v_mfma_f32_16x16x128_f8f6f4 a[180:183], v[168:175], v[232:239], a[180:183]\n\t" \
  "v_mfma_f32_16x16x128_f8f6f4 a[176:179], v[168:175], v[224:231], a[176:179]\n\t" \
  "s_mov_b32 s53, s51\n\t" \
  "s_add_u32 m0, s48, 32768\n\t" \
  "v_mfma_f32_16x16x128_f8f6f4 a[208:211], v[176:183], v[224:231], a[208:211]\n\t" \
  "buffer_load_dwordx4 v0, s[36:39], s53 offen sc0 sc1 lds\n\t" \
  "v_mfma_f32_16x16x128_f8f6f4 a[212:215], v[176:183], v[232:239], a[212:215]\n\t" \
  "v_mfma_f32_16x16x128_f8f6f4 a[216:219], v[176:183], v[240:247], a[216:219]\n\t" \
  "buffer_load_dwordx4 v0, s[36:39], s53 offen offset:1024 sc0 sc1 lds\n\t" \
  "v_mfma_f32_16x16x128_f8f6f4 a[220:223], v[176:183], v[248:255], a[220:223]\n\t" \
  "v_mfma_f32_16x16x128_f8f6f4 a[252:255], v[184:191], v[248:255], a[252:255]\n\t" \
  "buffer_load_dwordx4 v0, s[36:39], s53 offen offset:2048 sc0 sc1 lds\n\t" \
  "v_mfma_f32_16x16x128_f8f6f4 a[248:251], v[184:191], v[240:247], a[248:251]\n\t" \
  "v_mfma_f32_16x16x128_f8f6f4 a[244:247], v[184:191], v[232:239], a[244:247]\n\t" \
  "buffer_load_dwordx4 v0, s[36:39], s53 offen offset:3072 sc0 sc1 lds\n\t" \
  "v_mfma_f32_16x16x128_f8f6f4 a[240:243], v[184:191], v[224:231], a[240:243]\n\t" \
  "s_waitcnt lgkmcnt(0)\n\t" \
  "v_mfma_f32_16x16x128_f8f6f4 a[128:131], v[160:167], v[192:199], 0\n\t" \
  "v_mul_f32_e32 v32, s56, v32\n\t" \
  "v_mul_f32_e32 v33, s56, v33\n\t" \
  "v_mul_f32_e32 v34, s56, v34\n\t" \
  "v_mul_f32_e32 v35, s56, v35\n\t" \
  "v_mfma_f32_16x16x128_f8f6f4 a[132:135], v[160:167], v[200:207], 0\n\t" \
  "ds_read_b128 v[224:227], v4 offset:16384\n\t" \
  "ds_read_b128 v[228:231], v4 offset:24576\n\t" \
  "v_mfma_f32_16x16x128_f8f6f4 a[136:139], v[160:167], v[208:215], 0\n\t" \
  "ds_read_b128 v[232:235], v4 offset:17408\n\t" \
  "ds_read_b128 v[236:239], v4 offset:25600\n\t" \
  "v_mfma_f32_16x16x128_f8f6f4 a[140:143], v[160:167], v[216:223], 0\n\t" \
  "ds_read_b128 v[240:243], v4 offset:18432\n\t" \
  "ds_read_b128 v[244:247], v4 offset:26624\n\t" \
  "v_mfma_f32_16x16x128_f8f6f4 a[172:175], v[168:175], v[216:223], 0\n\t" \
  "ds_read_b128 v[248:251], v4 offset:19456\n\t" \
  "ds_read_b128 v[252:255], v4 offset:27648\n\t" \
  "v_mfma_f32_16x16x128_f8f6f4 a[168:171], v[168:175], v[208:215], 0\n\t" \
  "v_min_f32_e32 v12, s60, v32\n\t" \
  "v_min_f32_e32 v13, s60, v33\n\t" \
  "v_min_f32_e32 v14, s60, v34\n\t" \
  "v_min_f32_e32 v15, s60, v35\n\t" \
  "v_mfma_f32_16x16x128_f8f6f4 a[164:167], v[168:175], v[200:207], 0\n\t" \
  "v_exp_f32_e32 v12, v12\n\t" \
  "v_exp_f32_e32 v13, v13\n\t" \
  "v_mfma_f32_16x16x128_f8f6f4 a[160:163], v[168:175], v[192:199], 0\n\t" \
  "s_add_u32 s53, s49, 16384\n\t" \
  "s_add_u32 m0, s48, 49152\n\t" \
  "v_mfma_f32_16x16x128_f8f6f4 a[192:195], v[176:183], v[192:199], 0\n\t" \
  "buffer_load_dwordx4 v0, s[36:39], s53 offen sc0 sc1 lds\n\t" \
  "v_mfma_f32_16x16x128_f8f6f4 a[196:199], v[176:183], v[200:207], 0\n\t" \
  "v_exp_f32_e32 v14, v14\n\t" \
  "v_exp_f32_e32 v15, v15\n\t" \
  "v_mfma_f32_16x16x128_f8f6f4 a[200:203], v[176:183], v[208:215], 0\n\t" \
  "buffer_load_dwordx4 v0, s[36:39], s53 offen offset:1024 sc0 sc1 lds\n\t" \
  "v_mfma_f32_16x16x128_f8f6f4 a[204:207], v[176:183], v[216:223], 0\n\t" \
  "v_fma_f32 v12, v12, s58, s58\n\t" \
  "v_fma_f32 v13, v13, s58, s58\n\t" \
  "v_fma_f32 v14, v14, s58, s58\n\t" \
  "v_fma_f32 v15, v15, s58, s58\n\t" \
  "v_mfma_f32_16x16x128_f8f6f4 a[236:239], v[184:191], v[216:223], 0\n\t" \
  "buffer_load_dwordx4 v0, s[36:39], s53 offen offset:2048 sc0 sc1 lds\n\t" \
  "v_mfma_f32_16x16x128_f8f6f4 a[232:235], v[184:191], v[208:215], 0\n\t" \
  "v_rcp_f32_e32 v12, v12\n\t" \
  "v_rcp_f32_e32 v13, v13\n\t" \
  "v_mfma_f32_16x16x128_f8f6f4 a[228:231], v[184:191], v[200:207], 0\n\t" \
  "buffer_load_dwordx4 v0, s[36:39], s53 offen offset:3072 sc0 sc1 lds\n\t" \
  "v_mfma_f32_16x16x128_f8f6f4 a[224:227], v[184:191], v[192:199], 0\n\t" \
  "v_rcp_f32_e32 v14, v14\n\t" \
  "v_rcp_f32_e32 v15, v15\n\t" \
  "s_waitcnt lgkmcnt(0)\n\t" \
  "v_mfma_f32_16x16x128_f8f6f4 a[16:19], v[128:135], v[224:231], a[16:19]\n\t" \
  "s_waitcnt vmcnt(16)\n\t" \
  "s_barrier\n\t" \
  "v_mfma_f32_16x16x128_f8f6f4 a[20:23], v[128:135], v[232:239], a[20:23]\n\t" \
  "ds_read_b128 v[192:195], v4 offset:32768\n\t" \
  "ds_read_b128 v[196:199], v4 offset:40960\n\t" \
  "v_mfma_f32_16x16x128_f8f6f4 a[24:27], v[128:135], v[240:247], a[24:27]\n\t" \
  "ds_read_b128 v[200:203], v4 offset:33792\n\t" \
  "ds_read_b128 v[204:207], v4 offset:41984\n\t" \
  "v_mfma_f32_16x16x128_f8f6f4 a[28:31], v[128:135], v[248:255], a[28:31]\n\t" \
  "ds_read_b128 v[208:211], v4 offset:34816\n\t" \
  "ds_read_b128 v[212:215], v4 offset:43008\n\t" \
  "v_mfma_f32_16x16x128_f8f6f4 a[60:63], v[136:143], v[248:255], a[60:63]\n\t" \
  "ds_read_b128 v[216:219], v4 offset:35840\n\t" \
  "ds_read_b128 v[220:223], v4 offset:44032\n\t" \
  "v_mfma_f32_16x16x128_f8f6f4 a[56:59], v[136:143], v[240:247], a[56:59]\n\t" \
  "v_mul_f32_e32 v32, v32, v12\n\t" \
  "v_mul_f32_e32 v33, v33, v13\n\t" \
  "v_mul_f32_e32 v34, v34, v14\n\t" \
  "v_mul_f32_e32 v35, v35, v15\n\t" \
  "v_mfma_f32_16x16x128_f8f6f4 a[52:55], v[136:143], v[232:239], a[52:55]\n\t" \
  "s_xor_b32 s91, s68, 0\n\t" \
  "buffer_store_dwordx2 v[32:33], v5, s[40:43], s91 offen nt\n\t" \
  "v_mfma_f32_16x16x128_f8f6f4 a[48:51], v[136:143], v[224:231], a[48:51]\n\t" \
  "s_add_u32 s53, s49, 32768\n\t" \
  "s_add_u32 m0, s48, 65536\n\t" \
  "v_mfma_f32_16x16x128_f8f6f4 a[80:83], v[144:151], v[224:231], a[80:83]\n\t" \
  "buffer_load_dwordx4 v0, s[36:39], s53 offen sc0 sc1 lds\n\t" \
  "v_mfma_f32_16x16x128_f8f6f4 a[84:87], v[144:151], v[232:239], a[84:87]\n\t" \
  "buffer_load_dword v7, v8, s[44:47], s76 offen sc0 sc1\n\t" \
  "v_mfma_f32_16x16x128_f8f6f4 a[88:91], v[144:151], v[240:247], a[88:91]\n\t" \
  "buffer_load_dwordx4 v0, s[36:39], s53 offen offset:1024 sc0 sc1 lds\n\t" \
  "v_mfma_f32_16x16x128_f8f6f4 a[92:95], v[144:151], v[248:255], a[92:95]\n\t" \
  "buffer_store_dwordx2 v[34:35], v5, s[40:43], s91 offen offset:128 nt\n\t" \
  "v_mfma_f32_16x16x128_f8f6f4 a[124:127], v[152:159], v[248:255], a[124:127]\n\t" \
  "buffer_load_dwordx4 v0, s[36:39], s53 offen offset:2048 sc0 sc1 lds\n\t" \
  "v_mfma_f32_16x16x128_f8f6f4 a[120:123], v[152:159], v[240:247], a[120:123]\n\t" \
  "v_mul_f32_e32 v36, s56, v36\n\t" \
  "v_mul_f32_e32 v37, s56, v37\n\t" \
  "v_mul_f32_e32 v38, s56, v38\n\t" \
  "v_mul_f32_e32 v39, s56, v39\n\t" \
  "v_mfma_f32_16x16x128_f8f6f4 a[116:119], v[152:159], v[232:239], a[116:119]\n\t" \
  "buffer_load_dwordx4 v0, s[36:39], s53 offen offset:3072 sc0 sc1 lds\n\t" \
  "v_mfma_f32_16x16x128_f8f6f4 a[112:115], v[152:159], v[224:231], a[112:115]\n\t" \
  "v_min_f32_e32 v16, s60, v36\n\t" \
  "v_min_f32_e32 v17, s60, v37\n\t" \
  "v_min_f32_e32 v18, s60, v38\n\t" \
  "v_min_f32_e32 v19, s60, v39\n\t" \
  "s_waitcnt lgkmcnt(0)\n\t" \
  "v_mfma_f32_16x16x128_f8f6f4 a[0:3], v[128:135], v[192:199], a[0:3]\n\t" \
  "v_exp_f32_e32 v16, v16\n\t" \
  "v_exp_f32_e32 v17, v17\n\t" \
  "v_mfma_f32_16x16x128_f8f6f4 a[4:7], v[128:135], v[200:207], a[4:7]\n\t" \
  "ds_read_b128 v[160:163], v2 offset:49152\n\t" \
  "ds_read_b128 v[164:167], v2 offset:57344\n\t" \
  "v_mfma_f32_16x16x128_f8f6f4 a[8:11], v[128:135], v[208:215], a[8:11]\n\t" \
  "ds_read_b128 v[168:171], v2 offset:50176\n\t" \
  "ds_read_b128 v[172:175], v2 offset:58368\n\t" \
  "v_mfma_f32_16x16x128_f8f6f4 a[12:15], v[128:135], v[216:223], a[12:15]\n\t" \
  "ds_read_b128 v[176:179], v2 offset:51200\n\t" \
  "ds_read_b128 v[180:183], v2 offset:59392\n\t" \
  "v_mfma_f32_16x16x128_f8f6f4 a[44:47], v[136:143], v[216:223], a[44:47]\n\t" \
  "ds_read_b128 v[184:187], v2 offset:52224\n\t" \
  "ds_read_b128 v[188:191], v2 offset:60416\n\t" \
  "v_mfma_f32_16x16x128_f8f6f4 a[40:43], v[136:143], v[208:215], a[40:43]\n\t" \
  "v_exp_f32_e32 v18, v18\n\t" \
  "v_exp_f32_e32 v19, v19\n\t" \
  "v_mfma_f32_16x16x128_f8f6f4 a[36:39], v[136:143], v[200:207], a[36:39]\n\t" \
  "v_fma_f32 v16, v16, s58, s58\n\t" \
  "v_fma_f32 v17, v17, s58, s58\n\t" \
  "v_fma_f32 v18, v18, s58, s58\n\t" \
  "v_fma_f32 v19, v19, s58, s58\n\t" \
  "v_mfma_f32_16x16x128_f8f6f4 a[32:35], v[136:143], v[192:199], a[32:35]\n\t" \
  "s_add_u32 s53, s51, 16384\n\t" \
  "s_add_u32 m0, s48, 81920\n\t" \
  "v_mfma_f32_16x16x128_f8f6f4 a[64:67], v[144:151], v[192:199], a[64:67]\n\t" \
  "buffer_load_dwordx4 v0, s[36:39], s53 offen sc0 sc1 lds\n\t" \
  "v_mfma_f32_16x16x128_f8f6f4 a[68:71], v[144:151], v[200:207], a[68:71]\n\t" \
  "v_rcp_f32_e32 v16, v16\n\t" \
  "v_rcp_f32_e32 v17, v17\n\t" \
  "v_mfma_f32_16x16x128_f8f6f4 a[72:75], v[144:151], v[208:215], a[72:75]\n\t" \
  "buffer_load_dwordx4 v0, s[36:39], s53 offen offset:1024 sc0 sc1 lds\n\t" \
  "v_mfma_f32_16x16x128_f8f6f4 a[76:79], v[144:151], v[216:223], a[76:79]\n\t" \
  "v_rcp_f32_e32 v18, v18\n\t" \
  "v_rcp_f32_e32 v19, v19\n\t" \
  "v_mfma_f32_16x16x128_f8f6f4 a[108:111], v[152:159], v[216:223], a[108:111]\n\t" \
  "buffer_load_dwordx4 v0, s[36:39], s53 offen offset:2048 sc0 sc1 lds\n\t" \
  "v_mfma_f32_16x16x128_f8f6f4 a[104:107], v[152:159], v[208:215], a[104:107]\n\t" \
  "v_mul_f32_e32 v36, v36, v16\n\t" \
  "v_mul_f32_e32 v37, v37, v17\n\t" \
  "v_mul_f32_e32 v38, v38, v18\n\t" \
  "v_mul_f32_e32 v39, v39, v19\n\t" \
  "v_mfma_f32_16x16x128_f8f6f4 a[100:103], v[152:159], v[200:207], a[100:103]\n\t" \
  "buffer_load_dwordx4 v0, s[36:39], s53 offen offset:3072 sc0 sc1 lds\n\t" \
  "v_mfma_f32_16x16x128_f8f6f4 a[96:99], v[152:159], v[192:199], a[96:99]\n\t" \
  "s_xor_b32 s91, s68, 65536\n\t" \
  "buffer_store_dwordx2 v[36:37], v5, s[40:43], s91 offen nt\n\t" \
  "s_waitcnt lgkmcnt(0)\n\t" \
  "v_mfma_f32_16x16x128_f8f6f4 a[128:131], v[160:167], v[192:199], a[128:131]\n\t" \
  "s_waitcnt vmcnt(20)\n\t" \
  "s_barrier\n\t" \
  "v_mfma_f32_16x16x128_f8f6f4 a[132:135], v[160:167], v[200:207], a[132:135]\n\t" \
  "ds_read_b128 v[128:131], v1 offset:0\n\t" \
  "ds_read_b128 v[132:135], v1 offset:8192\n\t" \
  "v_mfma_f32_16x16x128_f8f6f4 a[136:139], v[160:167], v[208:215], a[136:139]\n\t" \
  "ds_read_b128 v[136:139], v1 offset:1024\n\t" \
  "ds_read_b128 v[140:143], v1 offset:9216\n\t" \
  "v_mfma_f32_16x16x128_f8f6f4 a[140:143], v[160:167], v[216:223], a[140:143]\n\t" \
  "ds_read_b128 v[144:147], v1 offset:2048\n\t" \
  "ds_read_b128 v[148:151], v1 offset:10240\n\t" \
  "v_mfma_f32_16x16x128_f8f6f4 a[172:175], v[168:175], v[216:223], a[172:175]\n\t" \
  "ds_read_b128 v[152:155], v1 offset:3072\n\t" \
  "ds_read_b128 v[156:159], v1 offset:11264\n\t" \
  "v_mfma_f32_16x16x128_f8f6f4 a[168:171], v[168:175], v[208:215], a[168:171]\n\t" \
  "buffer_store_dwordx2 v[38:39], v5, s[40:43], s91 offen offset:128 nt\n\t" \
  "v_mfma_f32_16x16x128_f8f6f4 a[164:167], v[168:175], v[200:207], a[164:167]\n\t" \
  "v_mul_f32_e32 v40, s56, v40\n\t" \
  "v_mul_f32_e32 v41, s56, v41\n\t" \
  "v_mul_f32_e32 v42, s56, v42\n\t" \
  "v_mul_f32_e32 v43, s56, v43\n\t" \
  "v_mfma_f32_16x16x128_f8f6f4 a[160:163], v[168:175], v[192:199], a[160:163]\n\t" \
  "s_add_u32 s53, s50, 16384\n\t" \
  "s_add_u32 m0, s48, 98304\n\t" \
  "v_mfma_f32_16x16x128_f8f6f4 a[192:195], v[176:183], v[192:199], a[192:195]\n\t" \
  "buffer_load_dwordx4 v0, s[36:39], s53 offen sc0 sc1 lds\n\t" \
  "v_mfma_f32_16x16x128_f8f6f4 a[196:199], v[176:183], v[200:207], a[196:199]\n\t" \
  "v_min_f32_e32 v20, s60, v40\n\t" \
  "v_min_f32_e32 v21, s60, v41\n\t" \
  "v_min_f32_e32 v22, s60, v42\n\t" \
  "v_min_f32_e32 v23, s60, v43\n\t" \
  "v_mfma_f32_16x16x128_f8f6f4 a[200:203], v[176:183], v[208:215], a[200:203]\n\t" \
  "buffer_load_dwordx4 v0, s[36:39], s53 offen offset:1024 sc0 sc1 lds\n\t" \
  "v_mfma_f32_16x16x128_f8f6f4 a[204:207], v[176:183], v[216:223], a[204:207]\n\t" \
  "v_mfma_f32_16x16x128_f8f6f4 a[236:239], v[184:191], v[216:223], a[236:239]\n\t" \
  "buffer_load_dwordx4 v0, s[36:39], s53 offen offset:2048 sc0 sc1 lds\n\t" \
  "v_mfma_f32_16x16x128_f8f6f4 a[232:235], v[184:191], v[208:215], a[232:235]\n\t" \
  "v_exp_f32_e32 v20, v20\n\t" \
  "v_exp_f32_e32 v21, v21\n\t" \
  "v_mfma_f32_16x16x128_f8f6f4 a[228:231], v[184:191], v[200:207], a[228:231]\n\t" \
  "buffer_load_dwordx4 v0, s[36:39], s53 offen offset:3072 sc0 sc1 lds\n\t" \
  "v_mfma_f32_16x16x128_f8f6f4 a[224:227], v[184:191], v[192:199], a[224:227]\n\t" \
  "v_exp_f32_e32 v22, v22\n\t" \
  "v_exp_f32_e32 v23, v23\n\t" \
  "s_waitcnt lgkmcnt(0)\n\t" \
  "v_mfma_f32_16x16x128_f8f6f4 a[144:147], v[160:167], v[224:231], a[144:147]\n\t" \
  "v_fma_f32 v20, v20, s58, s58\n\t" \
  "v_fma_f32 v21, v21, s58, s58\n\t" \
  "v_fma_f32 v22, v22, s58, s58\n\t" \
  "v_fma_f32 v23, v23, s58, s58\n\t" \
  "v_mfma_f32_16x16x128_f8f6f4 a[148:151], v[160:167], v[232:239], a[148:151]\n\t" \
  "ds_read_b128 v[192:195], v3 offset:16384\n\t" \
  "ds_read_b128 v[196:199], v3 offset:24576\n\t" \
  "v_mfma_f32_16x16x128_f8f6f4 a[152:155], v[160:167], v[240:247], a[152:155]\n\t" \
  "ds_read_b128 v[200:203], v3 offset:17408\n\t" \
  "ds_read_b128 v[204:207], v3 offset:25600\n\t" \
  "v_mfma_f32_16x16x128_f8f6f4 a[156:159], v[160:167], v[248:255], a[156:159]\n\t" \
  "ds_read_b128 v[208:211], v3 offset:18432\n\t" \
  "ds_read_b128 v[212:215], v3 offset:26624\n\t" \
  "v_mfma_f32_16x16x128_f8f6f4 a[188:191], v[168:175], v[248:255], a[188:191]\n\t" \
  "ds_read_b128 v[216:219], v3 offset:19456\n\t" \
  "ds_read_b128 v[220:223], v3 offset:27648\n\t" \
  "v_mfma_f32_16x16x128_f8f6f4 a[184:187], v[168:175], v[240:247], a[184:187]\n\t" \
  "v_rcp_f32_e32 v20, v20\n\t" \
  "v_rcp_f32_e32 v21, v21\n\t" \
  "v_mfma_f32_16x16x128_f8f6f4 a[180:183], v[168:175], v[232:239], a[180:183]\n\t" \
  "v_rcp_f32_e32 v22, v22\n\t" \
  "v_rcp_f32_e32 v23, v23\n\t" \
  "v_mfma_f32_16x16x128_f8f6f4 a[176:179], v[168:175], v[224:231], a[176:179]\n\t" \
  "s_add_u32 s53, s49, 49152\n\t" \
  "s_add_u32 m0, s48, 114688\n\t" \
  "v_mfma_f32_16x16x128_f8f6f4 a[208:211], v[176:183], v[224:231], a[208:211]\n\t" \
  "buffer_load_dwordx4 v0, s[36:39], s53 offen sc0 sc1 lds\n\t" \
  "v_mfma_f32_16x16x128_f8f6f4 a[212:215], v[176:183], v[232:239], a[212:215]\n\t" \
  "v_mul_f32_e32 v40, v40, v20\n\t" \
  "v_mul_f32_e32 v41, v41, v21\n\t" \
  "v_mul_f32_e32 v42, v42, v22\n\t" \
  "v_mul_f32_e32 v43, v43, v23\n\t" \
  "v_mfma_f32_16x16x128_f8f6f4 a[216:219], v[176:183], v[240:247], a[216:219]\n\t" \
  "buffer_load_dwordx4 v0, s[36:39], s53 offen offset:1024 sc0 sc1 lds\n\t" \
  "v_mfma_f32_16x16x128_f8f6f4 a[220:223], v[176:183], v[248:255], a[220:223]\n\t" \
  "s_xor_b32 s91, s68, 131072\n\t" \
  "buffer_store_dwordx2 v[40:41], v5, s[40:43], s91 offen nt\n\t" \
  "v_mfma_f32_16x16x128_f8f6f4 a[252:255], v[184:191], v[248:255], a[252:255]\n\t" \
  "buffer_load_dwordx4 v0, s[36:39], s53 offen offset:2048 sc0 sc1 lds\n\t" \
  "v_mfma_f32_16x16x128_f8f6f4 a[248:251], v[184:191], v[240:247], a[248:251]\n\t" \
  "buffer_store_dwordx2 v[42:43], v5, s[40:43], s91 offen offset:128 nt\n\t" \
  "v_mfma_f32_16x16x128_f8f6f4 a[244:247], v[184:191], v[232:239], a[244:247]\n\t" \
  "buffer_load_dwordx4 v0, s[36:39], s53 offen offset:3072 sc0 sc1 lds\n\t" \
  "v_mfma_f32_16x16x128_f8f6f4 a[240:243], v[184:191], v[224:231], a[240:243]\n\t" \
  "s_add_u32 s49, s49, 65536\n\t" \
  "s_add_u32 s50, s50, 32768\n\t" \
  "s_add_u32 s51, s51, 32768\n\t" \
  "s_waitcnt lgkmcnt(0)\n\t" \
  "v_mfma_f32_16x16x128_f8f6f4 a[0:3], v[128:135], v[192:199], a[0:3]\n\t" \
  "s_waitcnt vmcnt(23)\n\t" \
  "s_barrier\n\t" \
  "v_mfma_f32_16x16x128_f8f6f4 a[4:7], v[128:135], v[200:207], a[4:7]\n\t" \
  "ds_read_b128 v[224:227], v3 offset:32768\n\t" \
  "ds_read_b128 v[228:231], v3 offset:40960\n\t" \
  "v_mfma_f32_16x16x128_f8f6f4 a[8:11], v[128:135], v[208:215], a[8:11]\n\t" \
  "ds_read_b128 v[232:235], v3 offset:33792\n\t" \
  "ds_read_b128 v[236:239], v3 offset:41984\n\t" \
  "v_mfma_f32_16x16x128_f8f6f4 a[12:15], v[128:135], v[216:223], a[12:15]\n\t" \
  "ds_read_b128 v[240:243], v3 offset:34816\n\t" \
  "ds_read_b128 v[244:247], v3 offset:43008\n\t" \
  "v_mfma_f32_16x16x128_f8f6f4 a[44:47], v[136:143], v[216:223], a[44:47]\n\t" \
  "ds_read_b128 v[248:251], v3 offset:35840\n\t" \
  "ds_read_b128 v[252:255], v3 offset:44032\n\t" \
  "v_mfma_f32_16x16x128_f8f6f4 a[40:43], v[136:143], v[208:215], a[40:43]\n\t" \
  "s_waitcnt vmcnt(20)\n\t" \
  "v_cmp_le_u32_e32 vcc, s75, v7\n\t" \
  "v_mfma_f32_16x16x128_f8f6f4 a[36:39], v[136:143], v[200:207], a[36:39]\n\t" \
  "s_cmp_eq_u64 vcc, -1\n\t" \
  "s_cbranch_scc0 L_SLOW01_%=\n\t" \
  "L_OK01_%=:\n\t" \
  "s_add_u32 s76, s76, 1024\n\t" \
  "v_mfma_f32_16x16x128_f8f6f4 a[32:35], v[136:143], v[192:199], a[32:35]\n\t" \
  "s_mov_b32 s53, s49\n\t" \
  "s_mov_b32 m0, s48\n\t" \
  "v_mfma_f32_16x16x128_f8f6f4 a[64:67], v[144:151], v[192:199], a[64:67]\n\t" \
  "buffer_load_dwordx4 v0, s[36:39], s53 offen sc0 sc1 lds\n\t" \
  "v_mfma_f32_16x16x128_f8f6f4 a[68:71], v[144:151], v[200:207], a[68:71]\n\t" \
  "v_mul_f32_e32 v44, s56, v44\n\t" \
  "v_mul_f32_e32 v45, s56, v45\n\t" \
  "v_mul_f32_e32 v46, s56, v46\n\t" \
  "v_mul_f32_e32 v47, s56, v47\n\t" \
  "v_mfma_f32_16x16x128_f8f6f4 a[72:75], v[144:151], v[208:215], a[72:75]\n\t" \
  "buffer_load_dwordx4 v0, s[36:39], s53 offen offset:1024 sc0 sc1 lds\n\t" \
  "v_mfma_f32_16x16x128_f8f6f4 a[76:79], v[144:151], v[216:223], a[76:79]\n\t" \
  "v_min_f32_e32 v24, s60, v44\n\t" \
  "v_min_f32_e32 v25, s60, v45\n\t" \
  "v_min_f32_e32 v26, s60, v46\n\t" \
  "v_min_f32_e32 v27, s60, v47\n\t" \
  "v_mfma_f32_16x16x128_f8f6f4 a[108:111], v[152:159], v[216:223], a[108:111]\n\t" \
  "buffer_load_dwordx4 v0, s[36:39], s53 offen offset:2048 sc0 sc1 lds\n\t" \
  "v_mfma_f32_16x16x128_f8f6f4 a[104:107], v[152:159], v[208:215], a[104:107]\n\t" \
  "v_exp_f32_e32 v24, v24\n\t" \
  "v_exp_f32_e32 v25, v25\n\t" \
  "v_mfma_f32_16x16x128_f8f6f4 a[100:103], v[152:159], v[200:207], a[100:103]\n\t" \
  "buffer_load_dwordx4 v0, s[36:39], s53 offen offset:3072 sc0 sc1 lds\n\t" \
  "v_mfma_f32_16x16x128_f8f6f4 a[96:99], v[152:159], v[192:199], a[96:99]\n\t" \
  "v_exp_f32_e32 v26, v26\n\t" \
  "v_exp_f32_e32 v27, v27\n\t" \
  "s_waitcnt lgkmcnt(0)\n\t" \
  "v_mfma_f32_16x16x128_f8f6f4 a[16:19], v[128:135], v[224:231], a[16:19]\n\t" \
  "v_fma_f32 v24, v24, s58, s58\n\t" \
  "v_fma_f32 v25, v25, s58, s58\n\t" \
  "v_fma_f32 v26, v26, s58, s58\n\t" \
  "v_fma_f32 v27, v27, s58, s58\n\t" \
  "v_mfma_f32_16x16x128_f8f6f4 a[20:23], v[128:135], v[232:239], a[20:23]\n\t" \
  "ds_read_b128 v[160:163], v1 offset:49152\n\t" \
  "ds_read_b128 v[164:167], v1 offset:57344\n\t" \
  "v_mfma_f32_16x16x128_f8f6f4 a[24:27], v[128:135], v[240:247], a[24:27]\n\t" \
  "ds_read_b128 v[168:171], v1 offset:50176\n\t" \
  "ds_read_b128 v[172:175], v1 offset:58368\n\t" \
  "v_mfma_f32_16x16x128_f8f6f4 a[28:31], v[128:135], v[248:255], a[28:31]\n\t" \
  "ds_read_b128 v[176:179], v1 offset:51200\n\t" \
  "ds_read_b128 v[180:183], v1 offset:59392\n\t" \
  "v_mfma_f32_16x16x128_f8f6f4 a[60:63], v[136:143], v[248:255], a[60:63]\n\t" \
  "ds_read_b128 v[184:187], v1 offset:52224\n\t" \
  "ds_read_b128 v[188:191], v1 offset:60416\n\t" \
  "v_mfma_f32_16x16x128_f8f6f4 a[56:59], v[136:143], v[240:247], a[56:59]\n\t" \
  "v_rcp_f32_e32 v24, v24\n\t" \
  "v_rcp_f32_e32 v25, v25\n\t" \
  "v_mfma_f32_16x16x128_f8f6f4 a[52:55], v[136:143], v[232:239], a[52:55]\n\t" \
  "v_rcp_f32_e32 v26, v26\n\t" \
  "v_rcp_f32_e32 v27, v27\n\t" \
  "v_mfma_f32_16x16x128_f8f6f4 a[48:51], v[136:143], v[224:231], a[48:51]\n\t" \
  "s_mov_b32 s53, s50\n\t" \
  "s_add_u32 m0, s48, 16384\n\t" \
  "v_mfma_f32_16x16x128_f8f6f4 a[80:83], v[144:151], v[224:231], a[80:83]\n\t" \
  "buffer_load_dwordx4 v0, s[36:39], s53 offen sc0 sc1 lds\n\t" \
  "v_mfma_f32_16x16x128_f8f6f4 a[84:87], v[144:151], v[232:239], a[84:87]\n\t" \
  "v_mul_f32_e32 v44, v44, v24\n\t" \
  "v_mul_f32_e32 v45, v45, v25\n\t" \
  "v_mul_f32_e32 v46, v46, v26\n\t" \
  "v_mul_f32_e32 v47, v47, v27\n\t" \
  "v_mfma_f32_16x16x128_f8f6f4 a[88:91], v[144:151], v[240:247], a[88:91]\n\t" \
  "buffer_load_dwordx4 v0, s[36:39], s53 offen offset:1024 sc0 sc1 lds\n\t" \
  "v_mfma_f32_16x16x128_f8f6f4 a[92:95], v[144:151], v[248:255], a[92:95]\n\t" \
  "s_xor_b32 s91, s68, 196608\n\t" \
  "buffer_store_dwordx2 v[44:45], v5, s[40:43], s91 offen nt\n\t" \
  "v_mfma_f32_16x16x128_f8f6f4 a[124:127], v[152:159], v[248:255], a[124:127]\n\t" \
  "buffer_load_dwordx4 v0, s[36:39], s53 offen offset:2048 sc0 sc1 lds\n\t" \
  "v_mfma_f32_16x16x128_f8f6f4 a[120:123], v[152:159], v[240:247], a[120:123]\n\t" \
  "buffer_store_dwordx2 v[46:47], v5, s[40:43], s91 offen offset:128 nt\n\t" \
  "v_mfma_f32_16x16x128_f8f6f4 a[116:119], v[152:159], v[232:239], a[116:119]\n\t" \
  "buffer_load_dwordx4 v0, s[36:39], s53 offen offset:3072 sc0 sc1 lds\n\t" \
  "v_mfma_f32_16x16x128_f8f6f4 a[112:115], v[152:159], v[224:231], a[112:115]\n\t" \
  "v_mul_f32_e32 v48, s56, v48\n\t" \
  "v_mul_f32_e32 v49, s56, v49\n\t" \
  "v_mul_f32_e32 v50, s56, v50\n\t" \
  "v_mul_f32_e32 v51, s56, v51\n\t" \
  "s_waitcnt lgkmcnt(0)\n\t" \
  "v_mfma_f32_16x16x128_f8f6f4 a[144:147], v[160:167], v[224:231], a[144:147]\n\t" \
  "s_waitcnt vmcnt(22)\n\t" \
  "s_barrier\n\t" \
  "v_mfma_f32_16x16x128_f8f6f4 a[148:151], v[160:167], v[232:239], a[148:151]\n\t" \
  "ds_read_b128 v[128:131], v2 offset:0\n\t" \
  "ds_read_b128 v[132:135], v2 offset:8192\n\t" \
  "v_mfma_f32_16x16x128_f8f6f4 a[152:155], v[160:167], v[240:247], a[152:155]\n\t" \
  "ds_read_b128 v[136:139], v2 offset:1024\n\t" \
  "ds_read_b128 v[140:143], v2 offset:9216\n\t" \
  "v_mfma_f32_16x16x128_f8f6f4 a[156:159], v[160:167], v[248:255], a[156:159]\n\t" \
  "ds_read_b128 v[144:147], v2 offset:2048\n\t" \
  "ds_read_b128 v[148:151], v2 offset:10240\n\t" \
  "v_mfma_f32_16x16x128_f8f6f4 a[188:191], v[168:175], v[248:255], a[188:191]\n\t" \
  "ds_read_b128 v[152:155], v2 offset:3072\n\t" \
  "ds_read_b128 v[156:159], v2 offset:11264\n\t" \
  "v_mfma_f32_16x16x128_f8f6f4 a[184:187], v[168:175], v[240:247], a[184:187]\n\t" \
  "v_min_f32_e32 v28, s60, v48\n\t" \
  "v_min_f32_e32 v29, s60, v49\n\t" \
  "v_min_f32_e32 v30, s60, v50\n\t" \
  "v_min_f32_e32 v31, s60, v51\n\t" \
  "v_mfma_f32_16x16x128_f8f6f4 a[180:183], v[168:175], v[232:239], a[180:183]\n\t" \
  "v_exp_f32_e32 v28, v28\n\t" \
  "v_exp_f32_e32 v29, v29\n\t" \
  "v_mfma_f32_16x16x128_f8f6f4 a[176:179], v[168:175], v[224:231], a[176:179]\n\t" \
  "s_mov_b32 s53, s51\n\t" \
  "s_add_u32 m0, s48, 32768\n\t" \
  "v_mfma_f32_16x16x128_f8f6f4 a[208:211], v[176:183], v[224:231], a[208:211]\n\t" \
  "buffer_load_dwordx4 v0, s[36:39], s53 offen sc0 sc1 lds\n\t" \
  "v_mfma_f32_16x16x128_f8f6f4 a[212:215], v[176:183], v[232:239], a[212:215]\n\t" \
  "v_mfma_f32_16x16x128_f8f6f4 a[216:219], v[176:183], v[240:247], a[216:219]\n\t" \
  "buffer_load_dwordx4 v0, s[36:39], s53 offen offset:1024 sc0 sc1 lds\n\t" \
  "v_mfma_f32_16x16x128_f8f6f4 a[220:223], v[176:183], v[248:255], a[220:223]\n\t" \
  "v_exp_f32_e32 v30, v30\n\t" \
  "v_exp_f32_e32 v31, v31\n\t" \
  "v_mfma_f32_16x16x128_f8f6f4 a[252:255], v[184:191], v[248:255], a[252:255]\n\t" \
  "buffer_load_dwordx4 v0, s[36:39], s53 offen offset:2048 sc0 sc1 lds\n\t" \
  "v_mfma_f32_16x16x128_f8f6f4 a[248:251], v[184:191], v[240:247], a[248:251]\n\t" \
  "v_fma_f32 v28, v28, s58, s58\n\t" \
  "v_fma_f32 v29, v29, s58, s58\n\t" \
  "v_fma_f32 v30, v30, s58, s58\n\t" \
  "v_fma_f32 v31, v31, s58, s58\n\t" \
  "v_mfma_f32_16x16x128_f8f6f4 a[244:247], v[184:191], v[232:239], a[244:247]\n\t" \
  "buffer_load_dwordx4 v0, s[36:39], s53 offen offset:3072 sc0 sc1 lds\n\t" \
  "v_mfma_f32_16x16x128_f8f6f4 a[240:243], v[184:191], v[224:231], a[240:243]\n\t" \
  "v_rcp_f32_e32 v28, v28\n\t" \
  "v_rcp_f32_e32 v29, v29\n\t" \
  "s_waitcnt lgkmcnt(0)\n\t" \
  "v_mfma_f32_16x16x128_f8f6f4 a[128:131], v[160:167], v[192:199], a[128:131]\n\t" \
  "v_rcp_f32_e32 v30, v30\n\t" \
  "v_rcp_f32_e32 v31, v31\n\t" \
  "v_mfma_f32_16x16x128_f8f6f4 a[132:135], v[160:167], v[200:207], a[132:135]\n\t" \
  "ds_read_b128 v[224:227], v4 offset:16384\n\t" \
  "ds_read_b128 v[228:231], v4 offset:24576\n\t" \
  "v_mfma_f32_16x16x128_f8f6f4 a[136:139], v[160:167], v[208:215], a[136:139]\n\t" \
  "ds_read_b128 v[232:235], v4 offset:17408\n\t" \
  "ds_read_b128 v[236:239], v4 offset:25600\n\t" \
  "v_mfma_f32_16x16x128_f8f6f4 a[140:143], v[160:167], v[216:223], a[140:143]\n\t" \
  "ds_read_b128 v[240:243], v4 offset:18432\n\t" \
  "ds_read_b128 v[244:247], v4 offset:26624\n\t" \
  "v_mfma_f32_16x16x128_f8f6f4 a[172:175], v[168:175], v[216:223], a[172:175]\n\t" \
  "ds_read_b128 v[248:251], v4 offset:19456\n\t" \
  "ds_read_b128 v[252:255], v4 offset:27648\n\t" \
  "v_mfma_f32_16x16x128_f8f6f4 a[168:171], v[168:175], v[208:215], a[168:171]\n\t" \
  "v_mul_f32_e32 v48, v48, v28\n\t" \
  "v_mul_f32_e32 v49, v49, v29\n\t" \
  "v_mul_f32_e32 v50, v50, v30\n\t" \
  "v_mul_f32_e32 v51, v51, v31\n\t" \
  "v_mfma_f32_16x16x128_f8f6f4 a[164:167], v[168:175], v[200:207], a[164:167]\n\t" \
  "s_xor_b32 s91, s68, 1048576\n\t" \
  "buffer_store_dwordx2 v[48:49], v5, s[40:43], s91 offen nt\n\t" \
  "v_mfma_f32_16x16x128_f8f6f4 a[160:163], v[168:175], v[192:199], a[160:163]\n\t" \
  "s_add_u32 s53, s49, 16384\n\t" \
  "s_add_u32 m0, s48, 49152\n\t" \
  "v_mfma_f32_16x16x128_f8f6f4 a[192:195], v[176:183], v[192:199], a[192:195]\n\t" \
  "buffer_load_dwordx4 v0, s[36:39], s53 offen sc0 sc1 lds\n\t" \
  "v_mfma_f32_16x16x128_f8f6f4 a[196:199], v[176:183], v[200:207], a[196:199]\n\t" \
  "buffer_store_dwordx2 v[50:51], v5, s[40:43], s91 offen offset:128 nt\n\t" \
  "v_mfma_f32_16x16x128_f8f6f4 a[200:203], v[176:183], v[208:215], a[200:203]\n\t" \
  "buffer_load_dwordx4 v0, s[36:39], s53 offen offset:1024 sc0 sc1 lds\n\t" \
  "v_mfma_f32_16x16x128_f8f6f4 a[204:207], v[176:183], v[216:223], a[204:207]\n\t" \
  "v_mul_f32_e32 v52, s56, v52\n\t" \
  "v_mul_f32_e32 v53, s56, v53\n\t" \
  "v_mul_f32_e32 v54, s56, v54\n\t" \
  "v_mul_f32_e32 v55, s56, v55\n\t" \
  "v_mfma_f32_16x16x128_f8f6f4 a[236:239], v[184:191], v[216:223], a[236:239]\n\t" \
  "buffer_load_dwordx4 v0, s[36:39], s53 offen offset:2048 sc0 sc1 lds\n\t" \
  "v_mfma_f32_16x16x128_f8f6f4 a[232:235], v[184:191], v[208:215], a[232:235]\n\t" \
  "v_min_f32_e32 v12, s60, v52\n\t" \
  "v_min_f32_e32 v13, s60, v53\n\t" \
  "v_min_f32_e32 v14, s60, v54\n\t" \
  "v_min_f32_e32 v15, s60, v55\n\t" \
  "v_mfma_f32_16x16x128_f8f6f4 a[228:231], v[184:191], v[200:207], a[228:231]\n\t" \
  "buffer_load_dwordx4 v0, s[36:39], s53 offen offset:3072 sc0 sc1 lds\n\t" \
  "v_mfma_f32_16x16x128_f8f6f4 a[224:227], v[184:191], v[192:199], a[224:227]\n\t" \
  "v_exp_f32_e32 v12, v12\n\t" \
  "v_exp_f32_e32 v13, v13\n\t" \
  "s_waitcnt lgkmcnt(0)\n\t" \
  "v_mfma_f32_16x16x128_f8f6f4 a[16:19], v[128:135], v[224:231], a[16:19]\n\t" \
  "s_waitcnt vmcnt(20)\n\t" \
  "s_barrier\n\t" \
  "v_mfma_f32_16x16x128_f8f6f4 a[20:23], v[128:135], v[232:239], a[20:23]\n\t" \
  "ds_read_b128 v[192:195], v4 offset:32768\n\t" \
  "ds_read_b128 v[196:199], v4 offset:40960\n\t" \
  "v_mfma_f32_16x16x128_f8f6f4 a[24:27], v[128:135], v[240:247], a[24:27]\n\t" \
  "ds_read_b128 v[200:203], v4 offset:33792\n\t" \
  "ds_read_b128 v[204:207], v4 offset:41984\n\t" \
  "v_mfma_f32_16x16x128_f8f6f4 a[28:31], v[128:135], v[248:255], a[28:31]\n\t" \
  "ds_read_b128 v[208:211], v4 offset:34816\n\t" \
  "ds_read_b128 v[212:215], v4 offset:43008\n\t" \
  "v_mfma_f32_16x16x128_f8f6f4 a[60:63], v[136:143], v[248:255], a[60:63]\n\t" \
  "ds_read_b128 v[216:219], v4 offset:35840\n\t" \
  "ds_read_b128 v[220:223], v4 offset:44032\n\t" \
  "v_mfma_f32_16x16x128_f8f6f4 a[56:59], v[136:143], v[240:247], a[56:59]\n\t" \
  "v_exp_f32_e32 v14, v14\n\t" \
  "v_exp_f32_e32 v15, v15\n\t" \
  "v_mfma_f32_16x16x128_f8f6f4 a[52:55], v[136:143], v[232:239], a[52:55]\n\t" \
  "v_fma_f32 v12, v12, s58, s58\n\t" \
  "v_fma_f32 v13, v13, s58, s58\n\t" \
  "v_fma_f32 v14, v14, s58, s58\n\t" \
  "v_fma_f32 v15, v15, s58, s58\n\t" \
  "v_mfma_f32_16x16x128_f8f6f4 a[48:51], v[136:143], v[224:231], a[48:51]\n\t" \
  "s_add_u32 s53, s49, 32768\n\t" \
  "s_add_u32 m0, s48, 65536\n\t" \
  "v_mfma_f32_16x16x128_f8f6f4 a[80:83], v[144:151], v[224:231], a[80:83]\n\t" \
  "buffer_load_dwordx4 v0, s[36:39], s53 offen sc0 sc1 lds\n\t" \
  "v_mfma_f32_16x16x128_f8f6f4 a[84:87], v[144:151], v[232:239], a[84:87]\n\t" \
  "v_rcp_f32_e32 v12, v12\n\t" \
  "v_rcp_f32_e32 v13, v13\n\t" \
  "v_mfma_f32_16x16x128_f8f6f4 a[88:91], v[144:151], v[240:247], a[88:91]\n\t" \
  "buffer_load_dwordx4 v0, s[36:39], s53 offen offset:1024 sc0 sc1 lds\n\t" \
  "v_mfma_f32_16x16x128_f8f6f4 a[92:95], v[144:151], v[248:255], a[92:95]\n\t" \
  "v_rcp_f32_e32 v14, v14\n\t" \
  "v_rcp_f32_e32 v15, v15\n\t" \
  "v_mfma_f32_16x16x128_f8f6f4 a[124:127], v[152:159], v[248:255], a[124:127]\n\t" \
  "buffer_load_dwordx4 v0, s[36:39], s53 offen offset:2048 sc0 sc1 lds\n\t" \
  "v_mfma_f32_16x16x128_f8f6f4 a[120:123], v[152:159], v[240:247], a[120:123]\n\t" \
  "v_mul_f32_e32 v52, v52, v12\n\t" \
  "v_mul_f32_e32 v53, v53, v13\n\t" \
  "v_mul_f32_e32 v54, v54, v14\n\t" \
  "v_mul_f32_e32 v55, v55, v15\n\t" \
  "v_mfma_f32_16x16x128_f8f6f4 a[116:119], v[152:159], v[232:239], a[116:119]\n\t" \
  "buffer_load_dwordx4 v0, s[36:39], s53 offen offset:3072 sc0 sc1 lds\n\t" \
  "v_mfma_f32_16x16x128_f8f6f4 a[112:115], v[152:159], v[224:231], a[112:115]\n\t" \
  "s_xor_b32 s91, s68, 1114112\n\t" \
  "buffer_store_dwordx2 v[52:53], v5, s[40:43], s91 offen nt\n\t" \
  "s_waitcnt lgkmcnt(0)\n\t" \
  "v_mfma_f32_16x16x128_f8f6f4 a[0:3], v[128:135], v[192:199], a[0:3]\n\t" \
  "buffer_store_dwordx2 v[54:55], v5, s[40:43], s91 offen offset:128 nt\n\t" \
  "v_mfma_f32_16x16x128_f8f6f4 a[4:7], v[128:135], v[200:207], a[4:7]\n\t" \
  "ds_read_b128 v[160:163], v2 offset:49152\n\t" \
  "ds_read_b128 v[164:167], v2 offset:57344\n\t" \
  "v_mfma_f32_16x16x128_f8f6f4 a[8:11], v[128:135], v[208:215], a[8:11]\n\t" \
  "ds_read_b128 v[168:171], v2 offset:50176\n\t" \
  "ds_read_b128 v[172:175], v2 offset:58368\n\t" \
  "v_mfma_f32_16x16x128_f8f6f4 a[12:15], v[128:135], v[216:223], a[12:15]\n\t" \
  "ds_read_b128 v[176:179], v2 offset:51200\n\t" \
  "ds_read_b128 v[180:183], v2 offset:59392\n\t" \
  "v_mfma_f32_16x16x128_f8f6f4 a[44:47], v[136:143], v[216:223], a[44:47]\n\t" \
  "ds_read_b128 v[184:187], v2 offset:52224\n\t" \
  "ds_read_b128 v[188:191], v2 offset:60416\n\t" \
  "v_mfma_f32_16x16x128_f8f6f4 a[40:43], v[136:143], v[208:215], a[40:43]\n\t" \
  "v_mul_f32_e32 v56, s56, v56\n\t" \
  "v_mul_f32_e32 v57, s56, v57\n\t" \
  "v_mul_f32_e32 v58, s56, v58\n\t" \
  "v_mul_f32_e32 v59, s56, v59\n\t" \
  "v_mfma_f32_16x16x128_f8f6f4 a[36:39], v[136:143], v[200:207], a[36:39]\n\t" \
  "v_min_f32_e32 v16, s60, v56\n\t" \
  "v_min_f32_e32 v17, s60, v57\n\t" \
  "v_min_f32_e32 v18, s60, v58\n\t" \
  "v_min_f32_e32 v19, s60, v59\n\t" \
  "v_mfma_f32_16x16x128_f8f6f4 a[32:35], v[136:143], v[192:199], a[32:35]\n\t" \
  "s_add_u32 s53, s51, 16384\n\t" \
  "s_add_u32 m0, s48, 81920\n\t" \
  "v_mfma_f32_16x16x128_f8f6f4 a[64:67], v[144:151], v[192:199], a[64:67]\n\t" \
  "buffer_load_dwordx4 v0, s[36:39], s53 offen sc0 sc1 lds\n\t" \
  "v_mfma_f32_16x16x128_f8f6f4 a[68:71], v[144:151], v[200:207], a[68:71]\n\t" \
  "v_exp_f32_e32 v16, v16\n\t" \
  "v_exp_f32_e32 v17, v17\n\t" \
  "v_mfma_f32_16x16x128_f8f6f4 a[72:75], v[144:151], v[208:215], a[72:75]\n\t" \
  "buffer_load_dwordx4 v0, s[36:39], s53 offen offset:1024 sc0 sc1 lds\n\t" \
  "v_mfma_f32_16x16x128_f8f6f4 a[76:79], v[144:151], v[216:223], a[76:79]\n\t" \
  "v_exp_f32_e32 v18, v18\n\t" \
  "v_exp_f32_e32 v19, v19\n\t" \
  "v_mfma_f32_16x16x128_f8f6f4 a[108:111], v[152:159], v[216:223], a[108:111]\n\t" \
  "buffer_load_dwordx4 v0, s[36:39], s53 offen offset:2048 sc0 sc1 lds\n\t" \
  "v_mfma_f32_16x16x128_f8f6f4 a[104:107], v[152:159], v[208:215], a[104:107]\n\t" \
  "v_mfma_f32_16x16x128_f8f6f4 a[100:103], v[152:159], v[200:207], a[100:103]\n\t" \
  "buffer_load_dwordx4 v0, s[36:39], s53 offen offset:3072 sc0 sc1 lds\n\t" \
  "v_mfma_f32_16x16x128_f8f6f4 a[96:99], v[152:159], v[192:199], a[96:99]\n\t" \
  "v_fma_f32 v16, v16, s58, s58\n\t" \
  "v_fma_f32 v17, v17, s58, s58\n\t" \
  "v_fma_f32 v18, v18, s58, s58\n\t" \
  "v_fma_f32 v19, v19, s58, s58\n\t" \
  "s_waitcnt lgkmcnt(0)\n\t" \
  "v_mfma_f32_16x16x128_f8f6f4 a[128:131], v[160:167], v[192:199], a[128:131]\n\t" \
  "s_waitcnt vmcnt(20)\n\t" \
  "s_barrier\n\t" \
  "v_mfma_f32_16x16x128_f8f6f4 a[132:135], v[160:167], v[200:207], a[132:135]\n\t" \
  "ds_read_b128 v[128:131], v1 offset:0\n\t" \
  "ds_read_b128 v[132:135], v1 offset:8192\n\t" \
  "v_mfma_f32_16x16x128_f8f6f4 a[136:139], v[160:167], v[208:215], a[136:139]\n\t" \
  "ds_read_b128 v[136:139], v1 offset:1024\n\t" \
  "ds_read_b128 v[140:143], v1 offset:9216\n\t" \
  "v_mfma_f32_16x16x128_f8f6f4 a[140:143], v[160:167], v[216:223], a[140:143]\n\t" \
  "ds_read_b128 v[144:147], v1 offset:2048\n\t" \
  "ds_read_b128 v[148:151], v1 offset:10240\n\t" \
  "v_mfma_f32_16x16x128_f8f6f4 a[172:175], v[168:175], v[216:223], a[172:175]\n\t" \
  "ds_read_b128 v[152:155], v1 offset:3072\n\t" \
  "ds_read_b128 v[156:159], v1 offset:11264\n\t" \
  "v_mfma_f32_16x16x128_f8f6f4 a[168:171], v[168:175], v[208:215], a[168:171]\n\t" \
  "v_rcp_f32_e32 v16, v16\n\t" \
  "v_rcp_f32_e32 v17, v17\n\t" \
  "v_mfma_f32_16x16x128_f8f6f4 a[164:167], v[168:175], v[200:207], a[164:167]\n\t" \
  "v_rcp_f32_e32 v18, v18\n\t" \
  "v_rcp_f32_e32 v19, v19\n\t" \
  "v_mfma_f32_16x16x128_f8f6f4 a[160:163], v[168:175], v[192:199], a[160:163]\n\t" \
  "s_add_u32 s53, s50, 16384\n\t" \
  "s_add_u32 m0, s48, 98304\n\t" \
  "v_mfma_f32_16x16x128_f8f6f4 a[192:195], v[176:183], v[192:199], a[192:195]\n\t" \
  "buffer_load_dwordx4 v0, s[36:39], s53 offen sc0 sc1 lds\n\t" \
  "v_mfma_f32_16x16x128_f8f6f4 a[196:199], v[176:183], v[200:207], a[196:199]\n\t" \
  "v_mul_f32_e32 v56, v56, v16\n\t" \
  "v_mul_f32_e32 v57, v57, v17\n\t" \
  "v_mul_f32_e32 v58, v58, v18\n\t" \
  "v_mul_f32_e32 v59, v59, v19\n\t" \
  "v_mfma_f32_16x16x128_f8f6f4 a[200:203], v[176:183], v[208:215], a[200:203]\n\t" \
  "buffer_load_dwordx4 v0, s[36:39], s53 offen offset:1024 sc0 sc1 lds\n\t" \
  "v_mfma_f32_16x16x128_f8f6f4 a[204:207], v[176:183], v[216:223], a[204:207]\n\t" \
  "s_xor_b32 s91, s68, 1179648\n\t" \
  "buffer_store_dwordx2 v[56:57], v5, s[40:43], s91 offen nt\n\t" \
  "v_mfma_f32_16x16x128_f8f6f4 a[236:239], v[184:191], v[216:223], a[236:239]\n\t" \
  "buffer_load_dwordx4 v0, s[36:39], s53 offen offset:2048 sc0 sc1 lds\n\t" \
  "v_mfma_f32_16x16x128_f8f6f4 a[232:235], v[184:191], v[208:215], a[232:235]\n\t" \
  "buffer_store_dwordx2 v[58:59], v5, s[40:43], s91 offen offset:128 nt\n\t" \
  "v_mfma_f32_16x16x128_f8f6f4 a[228:231], v[184:191], v[200:207], a[228:231]\n\t" \
  "buffer_load_dwordx4 v0, s[36:39], s53 offen offset:3072 sc0 sc1 lds\n\t" \
  "v_mfma_f32_16x16x128_f8f6f4 a[224:227], v[184:191], v[192:199], a[224:227]\n\t" \
  "v_mul_f32_e32 v60, s56, v60\n\t" \
  "v_mul_f32_e32 v61, s56, v61\n\t" \
  "v_mul_f32_e32 v62, s56, v62\n\t" \
  "v_mul_f32_e32 v63, s56, v63\n\t" \
  "s_waitcnt lgkmcnt(0)\n\t" \
  "v_mfma_f32_16x16x128_f8f6f4 a[144:147], v[160:167], v[224:231], a[144:147]\n\t" \
  "v_min_f32_e32 v20, s60, v60\n\t" \
  "v_min_f32_e32 v21, s60, v61\n\t" \
  "v_min_f32_e32 v22, s60, v62\n\t" \
  "v_min_f32_e32 v23, s60, v63\n\t" \
  "v_mfma_f32_16x16x128_f8f6f4 a[148:151], v[160:167], v[232:239], a[148:151]\n\t" \
  "ds_read_b128 v[192:195], v3 offset:16384\n\t" \
  "ds_read_b128 v[196:199], v3 offset:24576\n\t" \
  "v_mfma_f32_16x16x128_f8f6f4 a[152:155], v[160:167], v[240:247], a[152:155]\n\t" \
  "ds_read_b128 v[200:203], v3 offset:17408\n\t" \
  "ds_read_b128 v[204:207], v3 offset:25600\n\t" \
  "v_mfma_f32_16x16x128_f8f6f4 a[156:159], v[160:167], v[248:255], a[156:159]\n\t" \
  "ds_read_b128 v[208:211], v3 offset:18432\n\t" \
  "ds_read_b128 v[212:215], v3 offset:26624\n\t" \
  "v_mfma_f32_16x16x128_f8f6f4 a[188:191], v[168:175], v[248:255], a[188:191]\n\t" \
  "ds_read_b128 v[216:219], v3 offset:19456\n\t" \
  "ds_read_b128 v[220:223], v3 offset:27648\n\t" \
  "v_mfma_f32_16x16x128_f8f6f4 a[184:187], v[168:175], v[240:247], a[184:187]\n\t" \
  "v_exp_f32_e32 v20, v20\n\t" \
  "v_exp_f32_e32 v21, v21\n\t" \
  "v_mfma_f32_16x16x128_f8f6f4 a[180:183], v[168:175], v[232:239], a[180:183]\n\t" \
  "v_exp_f32_e32 v22, v22\n\t" \
  "v_exp_f32_e32 v23, v23\n\t" \
  "v_mfma_f32_16x16x128_f8f6f4 a[176:179], v[168:175], v[224:231], a[176:179]\n\t" \
  "s_add_u32 s53, s49, 49152\n\t" \
  "s_add_u32 m0, s48, 114688\n\t" \
  "v_mfma_f32_16x16x128_f8f6f4 a[208:211], v[176:183], v[224:231], a[208:211]\n\t" \
  "buffer_load_dwordx4 v0, s[36:39], s53 offen sc0 sc1 lds\n\t" \
  "v_mfma_f32_16x16x128_f8f6f4 a[212:215], v[176:183], v[232:239], a[212:215]\n\t" \
  "v_fma_f32 v20, v20, s58, s58\n\t" \
  "v_fma_f32 v21, v21, s58, s58\n\t" \
  "v_fma_f32 v22, v22, s58, s58\n\t" \
  "v_fma_f32 v23, v23, s58, s58\n\t" \
  "v_mfma_f32_16x16x128_f8f6f4 a[216:219], v[176:183], v[240:247], a[216:219]\n\t" \
  "buffer_load_dwordx4 v0, s[36:39], s53 offen offset:1024 sc0 sc1 lds\n\t" \
  "v_mfma_f32_16x16x128_f8f6f4 a[220:223], v[176:183], v[248:255], a[220:223]\n\t" \
  "v_rcp_f32_e32 v20, v20\n\t" \
  "v_rcp_f32_e32 v21, v21\n\t" \
  "v_mfma_f32_16x16x128_f8f6f4 a[252:255], v[184:191], v[248:255], a[252:255]\n\t" \
  "buffer_load_dwordx4 v0, s[36:39], s53 offen offset:2048 sc0 sc1 lds\n\t" \
  "v_mfma_f32_16x16x128_f8f6f4 a[248:251], v[184:191], v[240:247], a[248:251]\n\t" \
  "v_rcp_f32_e32 v22, v22\n\t" \
  "v_rcp_f32_e32 v23, v23\n\t" \
  "v_mfma_f32_16x16x128_f8f6f4 a[244:247], v[184:191], v[232:239], a[244:247]\n\t" \
  "buffer_load_dwordx4 v0, s[36:39], s53 offen offset:3072 sc0 sc1 lds\n\t" \
  "v_mfma_f32_16x16x128_f8f6f4 a[240:243], v[184:191], v[224:231], a[240:243]\n\t" \
  "s_add_u32 s49, s49, 65536\n\t" \
  "s_add_u32 s50, s50, 32768\n\t" \
  "s_add_u32 s51, s51, 32768\n\t" \
  "s_waitcnt lgkmcnt(0)\n\t" \
  "v_mfma_f32_16x16x128_f8f6f4 a[0:3], v[128:135], v[192:199], a[0:3]\n\t" \
  "s_waitcnt vmcnt(20)\n\t" \
  "s_barrier\n\t" \
  "v_mfma_f32_16x16x128_f8f6f4 a[4:7], v[128:135], v[200:207], a[4:7]\n\t" \
  "ds_read_b128 v[224:227], v3 offset:32768\n\t" \
  "ds_read_b128 v[228:231], v3 offset:40960\n\t" \
  "v_mfma_f32_16x16x128_f8f6f4 a[8:11], v[128:135], v[208:215], a[8:11]\n\t" \
  "ds_read_b128 v[232:235], v3 offset:33792\n\t" \
  "ds_read_b128 v[236:239], v3 offset:41984\n\t" \
  "v_mfma_f32_16x16x128_f8f6f4 a[12:15], v[128:135], v[216:223], a[12:15]\n\t" \
  "ds_read_b128 v[240:243], v3 offset:34816\n\t" \
  "ds_read_b128 v[244:247], v3 offset:43008\n\t" \
  "v_mfma_f32_16x16x128_f8f6f4 a[44:47], v[136:143], v[216:223], a[44:47]\n\t" \
  "ds_read_b128 v[248:251], v3 offset:35840\n\t" \
  "ds_read_b128 v[252:255], v3 offset:44032\n\t" \
  "v_mfma_f32_16x16x128_f8f6f4 a[40:43], v[136:143], v[208:215], a[40:43]\n\t" \
  "v_mul_f32_e32 v60, v60, v20\n\t" \
  "v_mul_f32_e32 v61, v61, v21\n\t" \
  "v_mul_f32_e32 v62, v62, v22\n\t" \
  "v_mul_f32_e32 v63, v63, v23\n\t" \
  "v_mfma_f32_16x16x128_f8f6f4 a[36:39], v[136:143], v[200:207], a[36:39]\n\t" \
  "s_xor_b32 s91, s68, 1245184\n\t" \
  "buffer_store_dwordx2 v[60:61], v5, s[40:43], s91 offen nt\n\t" \
  "v_mfma_f32_16x16x128_f8f6f4 a[32:35], v[136:143], v[192:199], a[32:35]\n\t" \
  "s_mov_b32 s53, s49\n\t" \
  "s_mov_b32 m0, s48\n\t" \
  "v_mfma_f32_16x16x128_f8f6f4 a[64:67], v[144:151], v[192:199], a[64:67]\n\t" \
  "buffer_load_dwordx4 v0, s[36:39], s53 offen sc0 sc1 lds\n\t" \
  "v_mfma_f32_16x16x128_f8f6f4 a[68:71], v[144:151], v[200:207], a[68:71]\n\t" \
  "buffer_store_dwordx2 v[62:63], v5, s[40:43], s91 offen offset:128 nt\n\t" \
  "v_mfma_f32_16x16x128_f8f6f4 a[72:75], v[144:151], v[208:215], a[72:75]\n\t" \
  "buffer_load_dwordx4 v0, s[36:39], s53 offen offset:1024 sc0 sc1 lds\n\t" \
  "v_mfma_f32_16x16x128_f8f6f4 a[76:79], v[144:151], v[216:223], a[76:79]\n\t" \
  "v_mul_f32_e32 v64, s56, v64\n\t" \
  "v_mul_f32_e32 v65, s56, v65\n\t" \
  "v_mul_f32_e32 v66, s56, v66\n\t" \
  "v_mul_f32_e32 v67, s56, v67\n\t" \
  "v_mfma_f32_16x16x128_f8f6f4 a[108:111], v[152:159], v[216:223], a[108:111]\n\t" \
  "buffer_load_dwordx4 v0, s[36:39], s53 offen offset:2048 sc0 sc1 lds\n\t" \
  "v_mfma_f32_16x16x128_f8f6f4 a[104:107], v[152:159], v[208:215], a[104:107]\n\t" \
  "v_min_f32_e32 v24, s60, v64\n\t" \
  "v_min_f32_e32 v25, s60, v65\n\t" \
  "v_min_f32_e32 v26, s60, v66\n\t" \
  "v_min_f32_e32 v27, s60, v67\n\t" \
  "v_mfma_f32_16x16x128_f8f6f4 a[100:103], v[152:159], v[200:207], a[100:103]\n\t" \
  "buffer_load_dwordx4 v0, s[36:39], s53 offen offset:3072 sc0 sc1 lds\n\t" \
  "v_mfma_f32_16x16x128_f8f6f4 a[96:99], v[152:159], v[192:199], a[96:99]\n\t" \
  "v_exp_f32_e32 v24, v24\n\t" \
  "v_exp_f32_e32 v25, v25\n\t" \
  "s_waitcnt lgkmcnt(0)\n\t" \
  "v_mfma_f32_16x16x128_f8f6f4 a[16:19], v[128:135], v[224:231], a[16:19]\n\t" \
  "v_exp_f32_e32 v26, v26\n\t" \
  "v_exp_f32_e32 v27, v27\n\t" \
  "v_mfma_f32_16x16x128_f8f6f4 a[20:23], v[128:135], v[232:239], a[20:23]\n\t" \
  "ds_read_b128 v[160:163], v1 offset:49152\n\t" \
  "ds_read_b128 v[164:167], v1 offset:57344\n\t" \
  "v_mfma_f32_16x16x128_f8f6f4 a[24:27], v[128:135], v[240:247], a[24:27]\n\t" \
  "ds_read_b128 v[168:171], v1 offset:50176\n\t" \
  "ds_read_b128 v[172:175], v1 offset:58368\n\t" \
  "v_mfma_f32_16x16x128_f8f6f4 a[28:31], v[128:135], v[248:255], a[28:31]\n\t" \
  "ds_read_b128 v[176:179], v1 offset:51200\n\t" \
  "ds_read_b128 v[180:183], v1 offset:59392\n\t" \
  "v_mfma_f32_16x16x128_f8f6f4 a[60:63], v[136:143], v[248:255], a[60:63]\n\t" \
  "ds_read_b128 v[184:187], v1 offset:52224\n\t" \
  "ds_read_b128 v[188:191], v1 offset:60416\n\t" \
  "v_mfma_f32_16x16x128_f8f6f4 a[56:59], v[136:143], v[240:247], a[56:59]\n\t" \
  "v_fma_f32 v24, v24, s58, s58\n\t" \
  "v_fma_f32 v25, v25, s58, s58\n\t" \
  "v_fma_f32 v26, v26, s58, s58\n\t" \
  "v_fma_f32 v27, v27, s58, s58\n\t" \
  "v_mfma_f32_16x16x128_f8f6f4 a[52:55], v[136:143], v[232:239], a[52:55]\n\t" \
  "v_mfma_f32_16x16x128_f8f6f4 a[48:51], v[136:143], v[224:231], a[48:51]\n\t" \
  "s_mov_b32 s53, s50\n\t" \
  "s_add_u32 m0, s48, 16384\n\t" \
  "v_mfma_f32_16x16x128_f8f6f4 a[80:83], v[144:151], v[224:231], a[80:83]\n\t" \
  "buffer_load_dwordx4 v0, s[36:39], s53 offen sc0 sc1 lds\n\t" \
  "v_mfma_f32_16x16x128_f8f6f4 a[84:87], v[144:151], v[232:239], a[84:87]\n\t" \
  "v_rcp_f32_e32 v24, v24\n\t" \
  "v_rcp_f32_e32 v25, v25\n\t" \
  "v_mfma_f32_16x16x128_f8f6f4 a[88:91], v[144:151], v[240:247], a[88:91]\n\t" \
  "buffer_load_dwordx4 v0, s[36:39], s53 offen offset:1024 sc0 sc1 lds\n\t" \
  "v_mfma_f32_16x16x128_f8f6f4 a[92:95], v[144:151], v[248:255], a[92:95]\n\t" \
  "v_rcp_f32_e32 v26, v26\n\t" \
  "v_rcp_f32_e32 v27, v27\n\t" \
  "v_mfma_f32_16x16x128_f8f6f4 a[124:127], v[152:159], v[248:255], a[124:127]\n\t" \
  "buffer_load_dwordx4 v0, s[36:39], s53 offen offset:2048 sc0 sc1 lds\n\t" \
  "v_mfma_f32_16x16x128_f8f6f4 a[120:123], v[152:159], v[240:247], a[120:123]\n\t" \
  "v_mul_f32_e32 v64, v64, v24\n\t" \
  "v_mul_f32_e32 v65, v65, v25\n\t" \
  "v_mul_f32_e32 v66, v66, v26\n\t" \
  "v_mul_f32_e32 v67, v67, v27\n\t" \
  "v_mfma_f32_16x16x128_f8f6f4 a[116:119], v[152:159], v[232:239], a[116:119]\n\t" \
  "buffer_load_dwordx4 v0, s[36:39], s53 offen offset:3072 sc0 sc1 lds\n\t" \
  "v_mfma_f32_16x16x128_f8f6f4 a[112:115], v[152:159], v[224:231], a[112:115]\n\t" \
  "s_xor_b32 s91, s68, 2097152\n\t" \
  "buffer_store_dwordx2 v[64:65], v5, s[40:43], s91 offen nt\n\t" \
  "s_waitcnt lgkmcnt(0)\n\t" \
  "v_mfma_f32_16x16x128_f8f6f4 a[144:147], v[160:167], v[224:231], a[144:147]\n\t" \
  "s_waitcnt vmcnt(21)\n\t" \
  "s_barrier\n\t" \
  "v_mfma_f32_16x16x128_f8f6f4 a[148:151], v[160:167], v[232:239], a[148:151]\n\t" \
  "ds_read_b128 v[128:131], v2 offset:0\n\t" \
  "ds_read_b128 v[132:135], v2 offset:8192\n\t" \
  "v_mfma_f32_16x16x128_f8f6f4 a[152:155], v[160:167], v[240:247], a[152:155]\n\t" \
  "ds_read_b128 v[136:139], v2 offset:1024\n\t" \
  "ds_read_b128 v[140:143], v2 offset:9216\n\t" \
  "v_mfma_f32_16x16x128_f8f6f4 a[156:159], v[160:167], v[248:255], a[156:159]\n\t" \
  "ds_read_b128 v[144:147], v2 offset:2048\n\t" \
  "ds_read_b128 v[148:151], v2 offset:10240\n\t" \
  "v_mfma_f32_16x16x128_f8f6f4 a[188:191], v[168:175], v[248:255], a[188:191]\n\t" \
  "ds_read_b128 v[152:155], v2 offset:3072\n\t" \
  "ds_read_b128 v[156:159], v2 offset:11264\n\t" \
  "v_mfma_f32_16x16x128_f8f6f4 a[184:187], v[168:175], v[240:247], a[184:187]\n\t" \
  "buffer_store_dwordx2 v[66:67], v5, s[40:43], s91 offen offset:128 nt\n\t" \
  "v_mfma_f32_16x16x128_f8f6f4 a[180:183], v[168:175], v[232:239], a[180:183]\n\t" \
  "v_mul_f32_e32 v68, s56, v68\n\t" \
  "v_mul_f32_e32 v69, s56, v69\n\t" \
  "v_mul_f32_e32 v70, s56, v70\n\t" \
  "v_mul_f32_e32 v71, s56, v71\n\t" \
  "v_mfma_f32_16x16x128_f8f6f4 a[176:179], v[168:175], v[224:231], a[176:179]\n\t" \
  "s_mov_b32 s53, s51\n\t" \
  "s_add_u32 m0, s48, 32768\n\t" \
  "v_mfma_f32_16x16x128_f8f6f4 a[208:211], v[176:183], v[224:231], a[208:211]\n\t" \
  "buffer_load_dwordx4 v0, s[36:39], s53 offen sc0 sc1 lds\n\t" \
  "v_mfma_f32_16x16x128_f8f6f4 a[212:215], v[176:183], v[232:239], a[212:215]\n\t" \
  "v_min_f32_e32 v28, s60, v68\n\t" \
  "v_min_f32_e32 v29, s60, v69\n\t" \
  "v_min_f32_e32 v30, s60, v70\n\t" \
  "v_min_f32_e32 v31, s60, v71\n\t" \
  "v_mfma_f32_16x16x128_f8f6f4 a[216:219], v[176:183], v[240:247], a[216:219]\n\t" \
  "buffer_load_dwordx4 v0, s[36:39], s53 offen offset:1024 sc0 sc1 lds\n\t" \
  "v_mfma_f32_16x16x128_f8f6f4 a[220:223], v[176:183], v[248:255], a[220:223]\n\t" \
  "v_exp_f32_e32 v28, v28\n\t" \
  "v_exp_f32_e32 v29, v29\n\t" \
  "v_mfma_f32_16x16x128_f8f6f4 a[252:255], v[184:191], v[248:255], a[252:255]\n\t" \
  "buffer_load_dwordx4 v0, s[36:39], s53 offen offset:2048 sc0 sc1 lds\n\t" \
  "v_mfma_f32_16x16x128_f8f6f4 a[248:251], v[184:191], v[240:247], a[248:251]\n\t" \
  "v_exp_f32_e32 v30, v30\n\t" \
  "v_exp_f32_e32 v31, v31\n\t" \
  "v_mfma_f32_16x16x128_f8f6f4 a[244:247], v[184:191], v[232:239], a[244:247]\n\t" \
  "buffer_load_dwordx4 v0, s[36:39], s53 offen offset:3072 sc0 sc1 lds\n\t" \
  "v_mfma_f32_16x16x128_f8f6f4 a[240:243], v[184:191], v[224:231], a[240:243]\n\t" \
  "v_fma_f32 v28, v28, s58, s58\n\t" \
  "v_fma_f32 v29, v29, s58, s58\n\t" \
  "v_fma_f32 v30, v30, s58, s58\n\t" \
  "v_fma_f32 v31, v31, s58, s58\n\t" \
  "s_waitcnt lgkmcnt(0)\n\t" \
  "v_mfma_f32_16x16x128_f8f6f4 a[128:131], v[160:167], v[192:199], a[128:131]\n\t" \
  "v_rcp_f32_e32 v28, v28\n\t" \
  "v_rcp_f32_e32 v29, v29\n\t" \
  "v_mfma_f32_16x16x128_f8f6f4 a[132:135], v[160:167], v[200:207], a[132:135]\n\t" \
  "ds_read_b128 v[224:227], v4 offset:16384\n\t" \
  "ds_read_b128 v[228:231], v4 offset:24576\n\t" \
  "v_mfma_f32_16x16x128_f8f6f4 a[136:139], v[160:167], v[208:215], a[136:139]\n\t" \
  "ds_read_b128 v[232:235], v4 offset:17408\n\t" \
  "ds_read_b128 v[236:239], v4 offset:25600\n\t" \
  "v_mfma_f32_16x16x128_f8f6f4 a[140:143], v[160:167], v[216:223], a[140:143]\n\t" \
  "ds_read_b128 v[240:243], v4 offset:18432\n\t" \
  "ds_read_b128 v[244:247], v4 offset:26624\n\t" \
  "v_mfma_f32_16x16x128_f8f6f4 a[172:175], v[168:175], v[216:223], a[172:175]\n\t" \
  "ds_read_b128 v[248:251], v4 offset:19456\n\t" \
  "ds_read_b128 v[252:255], v4 offset:27648\n\t" \
  "v_mfma_f32_16x16x128_f8f6f4 a[168:171], v[168:175], v[208:215], a[168:171]\n\t" \
  "v_rcp_f32_e32 v30, v30\n\t" \
  "v_rcp_f32_e32 v31, v31\n\t" \
  "v_mfma_f32_16x16x128_f8f6f4 a[164:167], v[168:175], v[200:207], a[164:167]\n\t" \
  "v_mul_f32_e32 v68, v68, v28\n\t" \
  "v_mul_f32_e32 v69, v69, v29\n\t" \
  "v_mul_f32_e32 v70, v70, v30\n\t" \
  "v_mul_f32_e32 v71, v71, v31\n\t" \
  "v_mfma_f32_16x16x128_f8f6f4 a[160:163], v[168:175], v[192:199], a[160:163]\n\t" \
  "s_add_u32 s53, s49, 16384\n\t" \
  "s_add_u32 m0, s48, 49152\n\t" \
  "v_mfma_f32_16x16x128_f8f6f4 a[192:195], v[176:183], v[192:199], a[192:195]\n\t" \
  "buffer_load_dwordx4 v0, s[36:39], s53 offen sc0 sc1 lds\n\t" \
  "v_mfma_f32_16x16x128_f8f6f4 a[196:199], v[176:183], v[200:207], a[196:199]\n\t" \
  "s_xor_b32 s91, s68, 2162688\n\t" \
  "buffer_store_dwordx2 v[68:69], v5, s[40:43], s91 offen nt\n\t" \
  "v_mfma_f32_16x16x128_f8f6f4 a[200:203], v[176:183], v[208:215], a[200:203]\n\t" \
  "buffer_load_dwordx4 v0, s[36:39], s53 offen offset:1024 sc0 sc1 lds\n\t" \
  "v_mfma_f32_16x16x128_f8f6f4 a[204:207], v[176:183], v[216:223], a[204:207]\n\t" \
  "buffer_store_dwordx2 v[70:71], v5, s[40:43], s91 offen offset:128 nt\n\t" \
  "v_mfma_f32_16x16x128_f8f6f4 a[236:239], v[184:191], v[216:223], a[236:239]\n\t" \
  "buffer_load_dwordx4 v0, s[36:39], s53 offen offset:2048 sc0 sc1 lds\n\t" \
  "v_mfma_f32_16x16x128_f8f6f4 a[232:235], v[184:191], v[208:215], a[232:235]\n\t" \
  "v_mul_f32_e32 v72, s56, v72\n\t" \
  "v_mul_f32_e32 v73, s56, v73\n\t" \
  "v_mul_f32_e32 v74, s56, v74\n\t" \
  "v_mul_f32_e32 v75, s56, v75\n\t" \
  "v_mfma_f32_16x16x128_f8f6f4 a[228:231], v[184:191], v[200:207], a[228:231]\n\t" \
  "buffer_load_dwordx4 v0, s[36:39], s53 offen offset:3072 sc0 sc1 lds\n\t" \
  "v_mfma_f32_16x16x128_f8f6f4 a[224:227], v[184:191], v[192:199], a[224:227]\n\t" \
  "v_min_f32_e32 v12, s60, v72\n\t" \
  "v_min_f32_e32 v13, s60, v73\n\t" \
  "v_min_f32_e32 v14, s60, v74\n\t" \
  "v_min_f32_e32 v15, s60, v75\n\t" \
  "s_waitcnt lgkmcnt(0)\n\t" \
  "v_mfma_f32_16x16x128_f8f6f4 a[16:19], v[128:135], v[224:231], a[16:19]\n\t" \
  "s_waitcnt vmcnt(22)\n\t" \
  "s_barrier\n\t" \
  "v_mfma_f32_16x16x128_f8f6f4 a[20:23], v[128:135], v[232:239], a[20:23]\n\t" \
  "ds_read_b128 v[192:195], v4 offset:32768\n\t" \
  "ds_read_b128 v[196:199], v4 offset:40960\n\t" \
  "v_mfma_f32_16x16x128_f8f6f4 a[24:27], v[128:135], v[240:247], a[24:27]\n\t" \
  "ds_read_b128 v[200:203], v4 offset:33792\n\t" \
  "ds_read_b128 v[204:207], v4 offset:41984\n\t" \
  "v_mfma_f32_16x16x128_f8f6f4 a[28:31], v[128:135], v[248:255], a[28:31]\n\t" \
  "ds_read_b128 v[208:211], v4 offset:34816\n\t" \
  "ds_read_b128 v[212:215], v4 offset:43008\n\t" \
  "v_mfma_f32_16x16x128_f8f6f4 a[60:63], v[136:143], v[248:255], a[60:63]\n\t" \
  "ds_read_b128 v[216:219], v4 offset:35840\n\t" \
  "ds_read_b128 v[220:223], v4 offset:44032\n\t" \
  "v_mfma_f32_16x16x128_f8f6f4 a[56:59], v[136:143], v[240:247], a[56:59]\n\t" \
  "v_exp_f32_e32 v12, v12\n\t" \
  "v_exp_f32_e32 v13, v13\n\t" \
  "v_mfma_f32_16x16x128_f8f6f4 a[52:55], v[136:143], v[232:239], a[52:55]\n\t" \
  "v_exp_f32_e32 v14, v14\n\t" \
  "v_exp_f32_e32 v15, v15\n\t" \
  "v_mfma_f32_16x16x128_f8f6f4 a[48:51], v[136:143], v[224:231], a[48:51]\n\t" \
  "s_add_u32 s53, s49, 32768\n\t" \
  "s_add_u32 m0, s48, 65536\n\t" \
  "v_mfma_f32_16x16x128_f8f6f4 a[80:83], v[144:151], v[224:231], a[80:83]\n\t" \
  "buffer_load_dwordx4 v0, s[36:39], s53 offen sc0 sc1 lds\n\t" \
  "v_mfma_f32_16x16x128_f8f6f4 a[84:87], v[144:151], v[232:239], a[84:87]\n\t" \
  "buffer_load_dword v7, v8, s[44:47], s76 offen sc0 sc1\n\t" \
  "v_mfma_f32_16x16x128_f8f6f4 a[88:91], v[144:151], v[240:247], a[88:91]\n\t" \
  "buffer_load_dwordx4 v0, s[36:39], s53 offen offset:1024 sc0 sc1 lds\n\t" \
  "v_mfma_f32_16x16x128_f8f6f4 a[92:95], v[144:151], v[248:255], a[92:95]\n\t" \
  "v_fma_f32 v12, v12, s58, s58\n\t" \
  "v_fma_f32 v13, v13, s58, s58\n\t" \
  "v_fma_f32 v14, v14, s58, s58\n\t" \
  "v_fma_f32 v15, v15, s58, s58\n\t" \
  "v_mfma_f32_16x16x128_f8f6f4 a[124:127], v[152:159], v[248:255], a[124:127]\n\t" \
  "buffer_load_dwordx4 v0, s[36:39], s53 offen offset:2048 sc0 sc1 lds\n\t" \
  "v_mfma_f32_16x16x128_f8f6f4 a[120:123], v[152:159], v[240:247], a[120:123]\n\t" \
  "v_rcp_f32_e32 v12, v12\n\t" \
  "v_rcp_f32_e32 v13, v13\n\t" \
  "v_mfma_f32_16x16x128_f8f6f4 a[116:119], v[152:159], v[232:239], a[116:119]\n\t" \
  "buffer_load_dwordx4 v0, s[36:39], s53 offen offset:3072 sc0 sc1 lds\n\t" \
  "v_mfma_f32_16x16x128_f8f6f4 a[112:115], v[152:159], v[224:231], a[112:115]\n\t" \
  "s_waitcnt lgkmcnt(0)\n\t" \
  "v_mfma_f32_16x16x128_f8f6f4 a[0:3], v[128:135], v[192:199], a[0:3]\n\t" \
  "v_rcp_f32_e32 v14, v14\n\t" \
  "v_rcp_f32_e32 v15, v15\n\t" \
  "v_mfma_f32_16x16x128_f8f6f4 a[4:7], v[128:135], v[200:207], a[4:7]\n\t" \
  "ds_read_b128 v[160:163], v2 offset:49152\n\t" \
  "ds_read_b128 v[164:167], v2 offset:57344\n\t" \
  "v_mfma_f32_16x16x128_f8f6f4 a[8:11], v[128:135], v[208:215], a[8:11]\n\t" \
  "ds_read_b128 v[168:171], v2 offset:50176\n\t" \
  "ds_read_b128 v[172:175], v2 offset:58368\n\t" \
  "v_mfma_f32_16x16x128_f8f6f4 a[12:15], v[128:135], v[216:223], a[12:15]\n\t" \
  "ds_read_b128 v[176:179], v2 offset:51200\n\t" \
  "ds_read_b128 v[180:183], v2 offset:59392\n\t" \
  "v_mfma_f32_16x16x128_f8f6f4 a[44:47], v[136:143], v[216:223], a[44:47]\n\t" \
  "ds_read_b128 v[184:187], v2 offset:52224\n\t" \
  "ds_read_b128 v[188:191], v2 offset:60416\n\t" \
  "v_mfma_f32_16x16x128_f8f6f4 a[40:43], v[136:143], v[208:215], a[40:43]\n\t" \
  "v_mul_f32_e32 v72, v72, v12\n\t" \
  "v_mul_f32_e32 v73, v73, v13\n\t" \
  "v_mul_f32_e32 v74, v74, v14\n\t" \
  "v_mul_f32_e32 v75, v75, v15\n\t" \
  "v_mfma_f32_16x16x128_f8f6f4 a[36:39], v[136:143], v[200:207], a[36:39]\n\t" \
  "s_xor_b32 s91, s68, 2228224\n\t" \
  "buffer_store_dwordx2 v[72:73], v5, s[40:43], s91 offen nt\n\t" \
  "v_mfma_f32_16x16x128_f8f6f4 a[32:35], v[136:143], v[192:199], a[32:35]\n\t" \
  "s_add_u32 s53, s51, 16384\n\t" \
  "s_add_u32 m0, s48, 81920\n\t" \
  "v_mfma_f32_16x16x128_f8f6f4 a[64:67], v[144:151], v[192:199], a[64:67]\n\t" \
  "buffer_load_dwordx4 v0, s[36:39], s53 offen sc0 sc1 lds\n\t" \
  "v_mfma_f32_16x16x128_f8f6f4 a[68:71], v[144:151], v[200:207], a[68:71]\n\t" \
  "buffer_store_dwordx2 v[74:75], v5, s[40:43], s91 offen offset:128 nt\n\t" \
  "v_mfma_f32_16x16x128_f8f6f4 a[72:75], v[144:151], v[208:215], a[72:75]\n\t" \
  "buffer_load_dwordx4 v0, s[36:39], s53 offen offset:1024 sc0 sc1 lds\n\t" \
  "v_mfma_f32_16x16x128_f8f6f4 a[76:79], v[144:151], v[216:223], a[76:79]\n\t" \
  "v_mul_f32_e32 v76, s56, v76\n\t" \
  "v_mul_f32_e32 v77, s56, v77\n\t" \
  "v_mul_f32_e32 v78, s56, v78\n\t" \
  "v_mul_f32_e32 v79, s56, v79\n\t" \
  "v_mfma_f32_16x16x128_f8f6f4 a[108:111], v[152:159], v[216:223], a[108:111]\n\t" \
  "buffer_load_dwordx4 v0, s[36:39], s53 offen offset:2048 sc0 sc1 lds\n\t" \
  "v_mfma_f32_16x16x128_f8f6f4 a[104:107], v[152:159], v[208:215], a[104:107]\n\t" \
  "v_min_f32_e32 v16, s60, v76\n\t" \
  "v_min_f32_e32 v17, s60, v77\n\t" \
  "v_min_f32_e32 v18, s60, v78\n\t" \
  "v_min_f32_e32 v19, s60, v79\n\t" \
  "v_mfma_f32_16x16x128_f8f6f4 a[100:103], v[152:159], v[200:207], a[100:103]\n\t" \
  "buffer_load_dwordx4 v0, s[36:39], s53 offen offset:3072 sc0 sc1 lds\n\t" \
  "v_mfma_f32_16x16x128_f8f6f4 a[96:99], v[152:159], v[192:199], a[96:99]\n\t" \
  "v_exp_f32_e32 v16, v16\n\t" \
  "v_exp_f32_e32 v17, v17\n\t" \
  "s_waitcnt lgkmcnt(0)\n\t" \
  "v_mfma_f32_16x16x128_f8f6f4 a[128:131], v[160:167], v[192:199], a[128:131]\n\t" \
  "s_waitcnt vmcnt(23)\n\t" \
  "s_barrier\n\t" \
  "v_mfma_f32_16x16x128_f8f6f4 a[132:135], v[160:167], v[200:207], a[132:135]\n\t" \
  "ds_read_b128 v[128:131], v1 offset:0\n\t" \
  "ds_read_b128 v[132:135], v1 offset:8192\n\t" \
  "v_mfma_f32_16x16x128_f8f6f4 a[136:139], v[160:167], v[208:215], a[136:139]\n\t" \
  "ds_read_b128 v[136:139], v1 offset:1024\n\t" \
  "ds_read_b128 v[140:143], v1 offset:9216\n\t" \
  "v_mfma_f32_16x16x128_f8f6f4 a[140:143], v[160:167], v[216:223], a[140:143]\n\t" \
  "ds_read_b128 v[144:147], v1 offset:2048\n\t" \
  "ds_read_b128 v[148:151], v1 offset:10240\n\t" \
  "v_mfma_f32_16x16x128_f8f6f4 a[172:175], v[168:175], v[216:223], a[172:175]\n\t" \
  "ds_read_b128 v[152:155], v1 offset:3072\n\t" \
  "ds_read_b128 v[156:159], v1 offset:11264\n\t" \
  "v_mfma_f32_16x16x128_f8f6f4 a[168:171], v[168:175], v[208:215], a[168:171]\n\t" \
  "v_exp_f32_e32 v18, v18\n\t" \
  "v_exp_f32_e32 v19, v19\n\t" \
  "v_mfma_f32_16x16x128_f8f6f4 a[164:167], v[168:175], v[200:207], a[164:167]\n\t" \
  "v_fma_f32 v16, v16, s58, s58\n\t" \
  "v_fma_f32 v17, v17, s58, s58\n\t" \
  "v_fma_f32 v18, v18, s58, s58\n\t" \
  "v_fma_f32 v19, v19, s58, s58\n\t" \
  "v_mfma_f32_16x16x128_f8f6f4 a[160:163], v[168:175], v[192:199], a[160:163]\n\t" \
  "s_add_u32 s53, s50, 16384\n\t" \
  "s_add_u32 m0, s48, 98304\n\t" \
  "v_mfma_f32_16x16x128_f8f6f4 a[192:195], v[176:183], v[192:199], a[192:195]\n\t" \
  "buffer_load_dwordx4 v0, s[36:39], s53 offen sc0 sc1 lds\n\t" \
  "v_mfma_f32_16x16x128_f8f6f4 a[196:199], v[176:183], v[200:207], a[196:199]\n\t" \
  "v_rcp_f32_e32 v16, v16\n\t" \
  "v_rcp_f32_e32 v17, v17\n\t" \
  "v_mfma_f32_16x16x128_f8f6f4 a[200:203], v[176:183], v[208:215], a[200:203]\n\t" \
  "buffer_load_dwordx4 v0, s[36:39], s53 offen offset:1024 sc0 sc1 lds\n\t" \
  "v_mfma_f32_16x16x128_f8f6f4 a[204:207], v[176:183], v[216:223], a[204:207]\n\t" \
  "v_rcp_f32_e32 v18, v18\n\t" \
  "v_rcp_f32_e32 v19, v19\n\t" \
  "v_mfma_f32_16x16x128_f8f6f4 a[236:239], v[184:191], v[216:223], a[236:239]\n\t" \
  "buffer_load_dwordx4 v0, s[36:39], s53 offen offset:2048 sc0 sc1 lds\n\t" \
  "v_mfma_f32_16x16x128_f8f6f4 a[232:235], v[184:191], v[208:215], a[232:235]\n\t" \
  "v_mul_f32_e32 v76, v76, v16\n\t" \
  "v_mul_f32_e32 v77, v77, v17\n\t" \
  "v_mul_f32_e32 v78, v78, v18\n\t" \
  "v_mul_f32_e32 v79, v79, v19\n\t" \
  "v_mfma_f32_16x16x128_f8f6f4 a[228:231], v[184:191], v[200:207], a[228:231]\n\t" \
  "buffer_load_dwordx4 v0, s[36:39], s53 offen offset:3072 sc0 sc1 lds\n\t" \
  "v_mfma_f32_16x16x128_f8f6f4 a[224:227], v[184:191], v[192:199], a[224:227]\n\t" \
  "s_xor_b32 s91, s68, 2293760\n\t" \
  "buffer_store_dwordx2 v[76:77], v5, s[40:43], s91 offen nt\n\t" \
  "s_waitcnt lgkmcnt(0)\n\t" \
  "v_mfma_f32_16x16x128_f8f6f4 a[144:147], v[160:167], v[224:231], a[144:147]\n\t" \
  "buffer_store_dwordx2 v[78:79], v5, s[40:43], s91 offen offset:128 nt\n\t" \
  "v_mfma_f32_16x16x128_f8f6f4 a[148:151], v[160:167], v[232:239], a[148:151]\n\t" \
  "ds_read_b128 v[192:195], v3 offset:16384\n\t" \
  "ds_read_b128 v[196:199], v3 offset:24576\n\t" \
  "v_mfma_f32_16x16x128_f8f6f4 a[152:155], v[160:167], v[240:247], a[152:155]\n\t" \
  "ds_read_b128 v[200:203], v3 offset:17408\n\t" \
  "ds_read_b128 v[204:207], v3 offset:25600\n\t" \
  "v_mfma_f32_16x16x128_f8f6f4 a[156:159], v[160:167], v[248:255], a[156:159]\n\t" \
  "ds_read_b128 v[208:211], v3 offset:18432\n\t" \
  "ds_read_b128 v[212:215], v3 offset:26624\n\t" \
  "v_mfma_f32_16x16x128_f8f6f4 a[188:191], v[168:175], v[248:255], a[188:191]\n\t" \
  "ds_read_b128 v[216:219], v3 offset:19456\n\t" \
  "ds_read_b128 v[220:223], v3 offset:27648\n\t" \
  "v_mfma_f32_16x16x128_f8f6f4 a[184:187], v[168:175], v[240:247], a[184:187]\n\t" \
  "v_mul_f32_e32 v80, s56, v80\n\t" \
  "v_mul_f32_e32 v81, s56, v81\n\t" \
  "v_mul_f32_e32 v82, s56, v82\n\t" \
  "v_mul_f32_e32 v83, s56, v83\n\t" \
  "v_mfma_f32_16x16x128_f8f6f4 a[180:183], v[168:175], v[232:239], a[180:183]\n\t" \
  "v_min_f32_e32 v20, s60, v80\n\t" \
  "v_min_f32_e32 v21, s60, v81\n\t" \
  "v_min_f32_e32 v22, s60, v82\n\t" \
  "v_min_f32_e32 v23, s60, v83\n\t" \
  "v_mfma_f32_16x16x128_f8f6f4 a[176:179], v[168:175], v[224:231], a[176:179]\n\t" \
  "s_add_u32 s53, s49, 49152\n\t" \
  "s_add_u32 m0, s48, 114688\n\t" \
  "v_mfma_f32_16x16x128_f8f6f4 a[208:211], v[176:183], v[224:231], a[208:211]\n\t" \
  "buffer_load_dwordx4 v0, s[36:39], s53 offen sc0 sc1 lds\n\t" \
  "v_mfma_f32_16x16x128_f8f6f4 a[212:215], v[176:183], v[232:239], a[212:215]\n\t" \
  "v_exp_f32_e32 v20, v20\n\t" \
  "v_exp_f32_e32 v21, v21\n\t" \
  "v_mfma_f32_16x16x128_f8f6f4 a[216:219], v[176:183], v[240:247], a[216:219]\n\t" \
  "buffer_load_dwordx4 v0, s[36:39], s53 offen offset:1024 sc0 sc1 lds\n\t" \
  "v_mfma_f32_16x16x128_f8f6f4 a[220:223], v[176:183], v[248:255], a[220:223]\n\t" \
  "v_exp_f32_e32 v22, v22\n\t" \
  "v_exp_f32_e32 v23, v23\n\t" \
  "v_mfma_f32_16x16x128_f8f6f4 a[252:255], v[184:191], v[248:255], a[252:255]\n\t" \
  "buffer_load_dwordx4 v0, s[36:39], s53 offen offset:2048 sc0 sc1 lds\n\t" \
  "v_mfma_f32_16x16x128_f8f6f4 a[248:251], v[184:191], v[240:247], a[248:251]\n\t" \
  "v_fma_f32 v20, v20, s58, s58\n\t" \
  "v_fma_f32 v21, v21, s58, s58\n\t" \
  "v_fma_f32 v22, v22, s58, s58\n\t" \
  "v_fma_f32 v23, v23, s58, s58\n\t" \
  "v_mfma_f32_16x16x128_f8f6f4 a[244:247], v[184:191], v[232:239], a[244:247]\n\t" \
  "buffer_load_dwordx4 v0, s[36:39], s53 offen offset:3072 sc0 sc1 lds\n\t" \
  "v_mfma_f32_16x16x128_f8f6f4 a[240:243], v[184:191], v[224:231], a[240:243]\n\t" \
  "s_add_u32 s49, s49, 65536\n\t" \
  "s_add_u32 s50, s50, 32768\n\t" \
  "s_add_u32 s51, s51, 32768\n\t" \
  "s_waitcnt lgkmcnt(0)\n\t" \
  "v_mfma_f32_16x16x128_f8f6f4 a[0:3], v[128:135], v[192:199], a[0:3]\n\t" \
  "s_waitcnt vmcnt(21)\n\t" \
  "s_barrier\n\t" \
  "v_mfma_f32_16x16x128_f8f6f4 a[4:7], v[128:135], v[200:207], a[4:7]\n\t" \
  "ds_read_b128 v[224:227], v3 offset:32768\n\t" \
  "ds_read_b128 v[228:231], v3 offset:40960\n\t" \
  "v_mfma_f32_16x16x128_f8f6f4 a[8:11], v[128:135], v[208:215], a[8:11]\n\t" \
  "ds_read_b128 v[232:235], v3 offset:33792\n\t" \
  "ds_read_b128 v[236:239], v3 offset:41984\n\t" \
  "v_mfma_f32_16x16x128_f8f6f4 a[12:15], v[128:135], v[216:223], a[12:15]\n\t" \
  "ds_read_b128 v[240:243], v3 offset:34816\n\t" \
  "ds_read_b128 v[244:247], v3 offset:43008\n\t" \
  "v_mfma_f32_16x16x128_f8f6f4 a[44:47], v[136:143], v[216:223], a[44:47]\n\t" \
  "ds_read_b128 v[248:251], v3 offset:35840\n\t" \
  "ds_read_b128 v[252:255], v3 offset:44032\n\t" \
  "v_mfma_f32_16x16x128_f8f6f4 a[40:43], v[136:143], v[208:215], a[40:43]\n\t" \
  "s_waitcnt vmcnt(19)\n\t" \
  "v_cmp_le_u32_e32 vcc, s75, v7\n\t" \
  "v_mfma_f32_16x16x128_f8f6f4 a[36:39], v[136:143], v[200:207], a[36:39]\n\t" \
  "s_cmp_eq_u64 vcc, -1\n\t" \
  "s_cbranch_scc0 L_SLOW03_%=\n\t" \
  "L_OK03_%=:\n\t" \
  "s_add_u32 s76, s76, 1024\n\t" \
  "v_mfma_f32_16x16x128_f8f6f4 a[32:35], v[136:143], v[192:199], a[32:35]\n\t" \
  "s_mov_b32 s53, s49\n\t" \
  "s_mov_b32 m0, s48\n\t" \
  "v_mfma_f32_16x16x128_f8f6f4 a[64:67], v[144:151], v[192:199], a[64:67]\n\t" \
  "buffer_load_dwordx4 v0, s[36:39], s53 offen sc0 sc1 lds\n\t" \
  "v_mfma_f32_16x16x128_f8f6f4 a[68:71], v[144:151], v[200:207], a[68:71]\n\t" \
  "v_rcp_f32_e32 v20, v20\n\t" \
  "v_rcp_f32_e32 v21, v21\n\t" \
  "v_mfma_f32_16x16x128_f8f6f4 a[72:75], v[144:151], v[208:215], a[72:75]\n\t" \
  "buffer_load_dwordx4 v0, s[36:39], s53 offen offset:1024 sc0 sc1 lds\n\t" \
  "v_mfma_f32_16x16x128_f8f6f4 a[76:79], v[144:151], v[216:223], a[76:79]\n\t" \
  "v_rcp_f32_e32 v22, v22\n\t" \
  "v_rcp_f32_e32 v23, v23\n\t" \
  "v_mfma_f32_16x16x128_f8f6f4 a[108:111], v[152:159], v[216:223], a[108:111]\n\t" \
  "buffer_load_dwordx4 v0, s[36:39], s53 offen offset:2048 sc0 sc1 lds\n\t" \
  "v_mfma_f32_16x16x128_f8f6f4 a[104:107], v[152:159], v[208:215], a[104:107]\n\t" \
  "v_mfma_f32_16x16x128_f8f6f4 a[100:103], v[152:159], v[200:207], a[100:103]\n\t" \
  "buffer_load_dwordx4 v0, s[36:39], s53 offen offset:3072 sc0 sc1 lds\n\t" \
  "v_mfma_f32_16x16x128_f8f6f4 a[96:99], v[152:159], v[192:199], a[96:99]\n\t" \
  "v_mul_f32_e32 v80, v80, v20\n\t" \
  "v_mul_f32_e32 v81, v81, v21\n\t" \
  "v_mul_f32_e32 v82, v82, v22\n\t" \
  "v_mul_f32_e32 v83, v83, v23\n\t" \
  "s_waitcnt lgkmcnt(0)\n\t" \
  "v_mfma_f32_16x16x128_f8f6f4 a[16:19], v[128:135], v[224:231], a[16:19]\n\t" \
  "s_xor_b32 s91, s68, 3145728\n\t" \
  "buffer_store_dwordx2 v[80:81], v5, s[40:43], s91 offen nt\n\t" \
  "v_mfma_f32_16x16x128_f8f6f4 a[20:23], v[128:135], v[232:239], a[20:23]\n\t" \
  "ds_read_b128 v[160:163], v1 offset:49152\n\t" \
  "ds_read_b128 v[164:167], v1 offset:57344\n\t" \
  "v_mfma_f32_16x16x128_f8f6f4 a[24:27], v[128:135], v[240:247], a[24:27]\n\t" \
  "ds_read_b128 v[168:171], v1 offset:50176\n\t" \
  "ds_read_b128 v[172:175], v1 offset:58368\n\t" \
  "v_mfma_f32_16x16x128_f8f6f4 a[28:31], v[128:135], v[248:255], a[28:31]\n\t" \
  "ds_read_b128 v[176:179], v1 offset:51200\n\t" \
  "ds_read_b128 v[180:183], v1 offset:59392\n\t" \
  "v_mfma_f32_16x16x128_f8f6f4 a[60:63], v[136:143], v[248:255], a[60:63]\n\t" \
  "ds_read_b128 v[184:187], v1 offset:52224\n\t" \
  "ds_read_b128 v[188:191], v1 offset:60416\n\t" \
  "v_mfma_f32_16x16x128_f8f6f4 a[56:59], v[136:143], v[240:247], a[56:59]\n\t" \
  "buffer_store_dwordx2 v[82:83], v5, s[40:43], s91 offen offset:128 nt\n\t" \
  "v_mfma_f32_16x16x128_f8f6f4 a[52:55], v[136:143], v[232:239], a[52:55]\n\t" \
  "v_mul_f32_e32 v84, s56, v84\n\t" \
  "v_mul_f32_e32 v85, s56, v85\n\t" \
  "v_mul_f32_e32 v86, s56, v86\n\t" \
  "v_mul_f32_e32 v87, s56, v87\n\t" \
  "v_mfma_f32_16x16x128_f8f6f4 a[48:51], v[136:143], v[224:231], a[48:51]\n\t" \
  "s_mov_b32 s53, s50\n\t" \
  "s_add_u32 m0, s48, 16384\n\t" \
  "v_mfma_f32_16x16x128_f8f6f4 a[80:83], v[144:151], v[224:231], a[80:83]\n\t" \
  "buffer_load_dwordx4 v0, s[36:39], s53 offen sc0 sc1 lds\n\t" \
  "v_mfma_f32_16x16x128_f8f6f4 a[84:87], v[144:151], v[232:239], a[84:87]\n\t" \
  "v_min_f32_e32 v24, s60, v84\n\t" \
  "v_min_f32_e32 v25, s60, v85\n\t" \
  "v_min_f32_e32 v26, s60, v86\n\t" \
  "v_min_f32_e32 v27, s60, v87\n\t" \
  "v_mfma_f32_16x16x128_f8f6f4 a[88:91], v[144:151], v[240:247], a[88:91]\n\t" \
  "buffer_load_dwordx4 v0, s[36:39], s53 offen offset:1024 sc0 sc1 lds\n\t" \
  "v_mfma_f32_16x16x128_f8f6f4 a[92:95], v[144:151], v[248:255], a[92:95]\n\t" \
  "v_exp_f32_e32 v24, v24\n\t" \
  "v_exp_f32_e32 v25, v25\n\t" \
  "v_mfma_f32_16x16x128_f8f6f4 a[124:127], v[152:159], v[248:255], a[124:127]\n\t" \
  "buffer_load_dwordx4 v0, s[36:39], s53 offen offset:2048 sc0 sc1 lds\n\t" \
  "v_mfma_f32_16x16x128_f8f6f4 a[120:123], v[152:159], v[240:247], a[120:123]\n\t" \
  "v_exp_f32_e32 v26, v26\n\t" \
  "v_exp_f32_e32 v27, v27\n\t" \
  "v_mfma_f32_16x16x128_f8f6f4 a[116:119], v[152:159], v[232:239], a[116:119]\n\t" \
  "buffer_load_dwordx4 v0, s[36:39], s53 offen offset:3072 sc0 sc1 lds\n\t" \
  "v_mfma_f32_16x16x128_f8f6f4 a[112:115], v[152:159], v[224:231], a[112:115]\n\t" \
  "v_fma_f32 v24, v24, s58, s58\n\t" \
  "v_fma_f32 v25, v25, s58, s58\n\t" \
  "v_fma_f32 v26, v26, s58, s58\n\t" \
  "v_fma_f32 v27, v27, s58, s58\n\t" \
  "s_waitcnt lgkmcnt(0)\n\t" \
  "v_mfma_f32_16x16x128_f8f6f4 a[144:147], v[160:167], v[224:231], a[144:147]\n\t" \
  "s_waitcnt vmcnt(20)\n\t" \
  "s_barrier\n\t" \
  "v_mfma_f32_16x16x128_f8f6f4 a[148:151], v[160:167], v[232:239], a[148:151]\n\t" \
  "ds_read_b128 v[128:131], v2 offset:0\n\t" \
  "ds_read_b128 v[132:135], v2 offset:8192\n\t" \
  "v_mfma_f32_16x16x128_f8f6f4 a[152:155], v[160:167], v[240:247], a[152:155]\n\t" \
  "ds_read_b128 v[136:139], v2 offset:1024\n\t" \
  "ds_read_b128 v[140:143], v2 offset:9216\n\t" \
  "v_mfma_f32_16x16x128_f8f6f4 a[156:159], v[160:167], v[248:255], a[156:159]\n\t" \
  "ds_read_b128 v[144:147], v2 offset:2048\n\t" \
  "ds_read_b128 v[148:151], v2 offset:10240\n\t" \
  "v_mfma_f32_16x16x128_f8f6f4 a[188:191], v[168:175], v[248:255], a[188:191]\n\t" \
  "ds_read_b128 v[152:155], v2 offset:3072\n\t" \
  "ds_read_b128 v[156:159], v2 offset:11264\n\t" \
  "v_mfma_f32_16x16x128_f8f6f4 a[184:187], v[168:175], v[240:247], a[184:187]\n\t" \
  "v_rcp_f32_e32 v24, v24\n\t" \
  "v_rcp_f32_e32 v25, v25\n\t" \
  "v_mfma_f32_16x16x128_f8f6f4 a[180:183], v[168:175], v[232:239], a[180:183]\n\t" \
  "v_rcp_f32_e32 v26, v26\n\t" \
  "v_rcp_f32_e32 v27, v27\n\t" \
  "v_mfma_f32_16x16x128_f8f6f4 a[176:179], v[168:175], v[224:231], a[176:179]\n\t" \
  "s_mov_b32 s53, s51\n\t" \
  "s_add_u32 m0, s48, 32768\n\t" \
  "v_mfma_f32_16x16x128_f8f6f4 a[208:211], v[176:183], v[224:231], a[208:211]\n\t" \
  "buffer_load_dwordx4 v0, s[36:39], s53 offen sc0 sc1 lds\n\t" \
  "v_mfma_f32_16x16x128_f8f6f4 a[212:215], v[176:183], v[232:239], a[212:215]\n\t" \
  "v_mul_f32_e32 v84, v84, v24\n\t" \
  "v_mul_f32_e32 v85, v85, v25\n\t" \
  "v_mul_f32_e32 v86, v86, v26\n\t" \
  "v_mul_f32_e32 v87, v87, v27\n\t" \
  "v_mfma_f32_16x16x128_f8f6f4 a[216:219], v[176:183], v[240:247], a[216:219]\n\t" \
  "buffer_load_dwordx4 v0, s[36:39], s53 offen offset:1024 sc0 sc1 lds\n\t" \
  "v_mfma_f32_16x16x128_f8f6f4 a[220:223], v[176:183], v[248:255], a[220:223]\n\t" \
  "s_xor_b32 s91, s68, 3211264\n\t" \
  "buffer_store_dwordx2 v[84:85], v5, s[40:43], s91 offen nt\n\t" \
  "v_mfma_f32_16x16x128_f8f6f4 a[252:255], v[184:191], v[248:255], a[252:255]\n\t" \
  "buffer_load_dwordx4 v0, s[36:39], s53 offen offset:2048 sc0 sc1 lds\n\t" \
  "v_mfma_f32_16x16x128_f8f6f4 a[248:251], v[184:191], v[240:247], a[248:251]\n\t" \
  "buffer_store_dwordx2 v[86:87], v5, s[40:43], s91 offen offset:128 nt\n\t" \
  "v_mfma_f32_16x16x128_f8f6f4 a[244:247], v[184:191], v[232:239], a[244:247]\n\t" \
  "buffer_load_dwordx4 v0, s[36:39], s53 offen offset:3072 sc0 sc1 lds\n\t" \
  "v_mfma_f32_16x16x128_f8f6f4 a[240:243], v[184:191], v[224:231], a[240:243]\n\t" \
  "v_mul_f32_e32 v88, s56, v88\n\t" \
  "v_mul_f32_e32 v89, s56, v89\n\t" \
  "v_mul_f32_e32 v90, s56, v90\n\t" \
  "v_mul_f32_e32 v91, s56, v91\n\t" \
  "s_waitcnt lgkmcnt(0)\n\t" \
  "v_mfma_f32_16x16x128_f8f6f4 a[128:131], v[160:167], v[192:199], a[128:131]\n\t" \
  "v_min_f32_e32 v28, s60, v88\n\t" \
  "v_min_f32_e32 v29, s60, v89\n\t" \
  "v_min_f32_e32 v30, s60, v90\n\t" \
  "v_min_f32_e32 v31, s60, v91\n\t" \
  "v_mfma_f32_16x16x128_f8f6f4 a[132:135], v[160:167], v[200:207], a[132:135]\n\t" \
  "ds_read_b128 v[224:227], v4 offset:16384\n\t" \
  "ds_read_b128 v[228:231], v4 offset:24576\n\t" \
  "v_mfma_f32_16x16x128_f8f6f4 a[136:139], v[160:167], v[208:215], a[136:139]\n\t" \
  "ds_read_b128 v[232:235], v4 offset:17408\n\t" \
  "ds_read_b128 v[236:239], v4 offset:25600\n\t" \
  "v_mfma_f32_16x16x128_f8f6f4 a[140:143], v[160:167], v[216:223], a[140:143]\n\t" \
  "ds_read_b128 v[240:243], v4 offset:18432\n\t" \
  "ds_read_b128 v[244:247], v4 offset:26624\n\t" \
  "v_mfma_f32_16x16x128_f8f6f4 a[172:175], v[168:175], v[216:223], a[172:175]\n\t" \
  "ds_read_b128 v[248:251], v4 offset:19456\n\t" \
  "ds_read_b128 v[252:255], v4 offset:27648\n\t" \
  "v_mfma_f32_16x16x128_f8f6f4 a[168:171], v[168:175], v[208:215], a[168:171]\n\t" \
  "v_exp_f32_e32 v28, v28\n\t" \
  "v_exp_f32_e32 v29, v29\n\t" \
  "v_mfma_f32_16x16x128_f8f6f4 a[164:167], v[168:175], v[200:207], a[164:167]\n\t" \
  "v_exp_f32_e32 v30, v30\n\t" \
  "v_exp_f32_e32 v31, v31\n\t" \
  "v_mfma_f32_16x16x128_f8f6f4 a[160:163], v[168:175], v[192:199], a[160:163]\n\t" \
  "s_add_u32 s53, s49, 16384\n\t" \
  "s_add_u32 m0, s48, 49152\n\t" \
  "v_mfma_f32_16x16x128_f8f6f4 a[192:195], v[176:183], v[192:199], a[192:195]\n\t" \
  "buffer_load_dwordx4 v0, s[36:39], s53 offen sc0 sc1 lds\n\t" \
  "v_mfma_f32_16x16x128_f8f6f4 a[196:199], v[176:183], v[200:207], a[196:199]\n\t" \
  "v_fma_f32 v28, v28, s58, s58\n\t" \
  "v_fma_f32 v29, v29, s58, s58\n\t" \
  "v_fma_f32 v30, v30, s58, s58\n\t" \
  "v_fma_f32 v31, v31, s58, s58\n\t" \
  "v_mfma_f32_16x16x128_f8f6f4 a[200:203], v[176:183], v[208:215], a[200:203]\n\t" \
  "buffer_load_dwordx4 v0, s[36:39], s53 offen offset:1024 sc0 sc1 lds\n\t" \
  "v_mfma_f32_16x16x128_f8f6f4 a[204:207], v[176:183], v[216:223], a[204:207]\n\t" \
  "v_rcp_f32_e32 v28, v28\n\t" \
  "v_rcp_f32_e32 v29, v29\n\t" \
  "v_mfma_f32_16x16x128_f8f6f4 a[236:239], v[184:191], v[216:223], a[236:239]\n\t" \
  "buffer_load_dwordx4 v0, s[36:39], s53 offen offset:2048 sc0 sc1 lds\n\t" \
  "v_mfma_f32_16x16x128_f8f6f4 a[232:235], v[184:191], v[208:215], a[232:235]\n\t" \
  "v_rcp_f32_e32 v30, v30\n\t" \
  "v_rcp_f32_e32 v31, v31\n\t" \
  "v_mfma_f32_16x16x128_f8f6f4 a[228:231], v[184:191], v[200:207], a[228:231]\n\t" \
  "buffer_load_dwordx4 v0, s[36:39], s53 offen offset:3072 sc0 sc1 lds\n\t" \
  "v_mfma_f32_16x16x128_f8f6f4 a[224:227], v[184:191], v[192:199], a[224:227]\n\t" \
  "v_mul_f32_e32 v88, v88, v28\n\t" \
  "v_mul_f32_e32 v89, v89, v29\n\t" \
  "v_mul_f32_e32 v90, v90, v30\n\t" \
  "v_mul_f32_e32 v91, v91, v31\n\t" \
  "s_waitcnt lgkmcnt(0)\n\t" \
  "v_mfma_f32_16x16x128_f8f6f4 a[16:19], v[128:135], v[224:231], a[16:19]\n\t" \
  "s_waitcnt vmcnt(20)\n\t" \
  "s_barrier\n\t" \
  "v_mfma_f32_16x16x128_f8f6f4 a[20:23], v[128:135], v[232:239], a[20:23]\n\t" \
  "ds_read_b128 v[192:195], v4 offset:32768\n\t" \
  "ds_read_b128 v[196:199], v4 offset:40960\n\t" \
  "v_mfma_f32_16x16x128_f8f6f4 a[24:27], v[128:135], v[240:247], a[24:27]\n\t" \
  "ds_read_b128 v[200:203], v4 offset:33792\n\t" \
  "ds_read_b128 v[204:207], v4 offset:41984\n\t" \
  "v_mfma_f32_16x16x128_f8f6f4 a[28:31], v[128:135], v[248:255], a[28:31]\n\t" \
  "ds_read_b128 v[208:211], v4 offset:34816\n\t" \
  "ds_read_b128 v[212:215], v4 offset:43008\n\t" \
  "v_mfma_f32_16x16x128_f8f6f4 a[60:63], v[136:143], v[248:255], a[60:63]\n\t" \
  "ds_read_b128 v[216:219], v4 offset:35840\n\t" \
  "ds_read_b128 v[220:223], v4 offset:44032\n\t" \
  "v_mfma_f32_16x16x128_f8f6f4 a[56:59], v[136:143], v[240:247], a[56:59]\n\t" \
  "v_mfma_f32_16x16x128_f8f6f4 a[52:55], v[136:143], v[232:239], a[52:55]\n\t" \
  "s_xor_b32 s91, s68, 3276800\n\t" \
  "buffer_store_dwordx2 v[88:89], v5, s[40:43], s91 offen nt\n\t" \
  "v_mfma_f32_16x16x128_f8f6f4 a[48:51], v[136:143], v[224:231], a[48:51]\n\t" \
  "s_add_u32 s53, s49, 32768\n\t" \
  "s_add_u32 m0, s48, 65536\n\t" \
  "v_mfma_f32_16x16x128_f8f6f4 a[80:83], v[144:151], v[224:231], a[80:83]\n\t" \
  "buffer_load_dwordx4 v0, s[36:39], s53 offen sc0 sc1 lds\n\t" \
  "v_mfma_f32_16x16x128_f8f6f4 a[84:87], v[144:151], v[232:239], a[84:87]\n\t" \
  "buffer_store_dwordx2 v[90:91], v5, s[40:43], s91 offen offset:128 nt\n\t" \
  "v_mfma_f32_16x16x128_f8f6f4 a[88:91], v[144:151], v[240:247], a[88:91]\n\t" \
  "buffer_load_dwordx4 v0, s[36:39], s53 offen offset:1024 sc0 sc1 lds\n\t" \
  "v_mfma_f32_16x16x128_f8f6f4 a[92:95], v[144:151], v[248:255], a[92:95]\n\t" \
  "v_mul_f32_e32 v92, s56, v92\n\t" \
  "v_mul_f32_e32 v93, s56, v93\n\t" \
  "v_mul_f32_e32 v94, s56, v94\n\t" \
  "v_mul_f32_e32 v95, s56, v95\n\t" \
  "v_mfma_f32_16x16x128_f8f6f4 a[124:127], v[152:159], v[248:255], a[124:127]\n\t" \
  "buffer_load_dwordx4 v0, s[36:39], s53 offen offset:2048 sc0 sc1 lds\n\t" \
  "v_mfma_f32_16x16x128_f8f6f4 a[120:123], v[152:159], v[240:247], a[120:123]\n\t" \
  "v_min_f32_e32 v12, s60, v92\n\t" \
  "v_min_f32_e32 v13, s60, v93\n\t" \
  "v_min_f32_e32 v14, s60, v94\n\t" \
  "v_min_f32_e32 v15, s60, v95\n\t" \
  "v_mfma_f32_16x16x128_f8f6f4 a[116:119], v[152:159], v[232:239], a[116:119]\n\t" \
  "buffer_load_dwordx4 v0, s[36:39], s53 offen offset:3072 sc0 sc1 lds\n\t" \
  "v_mfma_f32_16x16x128_f8f6f4 a[112:115], v[152:159], v[224:231], a[112:115]\n\t" \
  "v_exp_f32_e32 v12, v12\n\t" \
  "v_exp_f32_e32 v13, v13\n\t" \
  "s_waitcnt lgkmcnt(0)\n\t" \
  "v_mfma_f32_16x16x128_f8f6f4 a[0:3], v[128:135], v[192:199], a[0:3]\n\t" \
  "v_exp_f32_e32 v14, v14\n\t" \
  "v_exp_f32_e32 v15, v15\n\t" \
  "v_mfma_f32_16x16x128_f8f6f4 a[4:7], v[128:135], v[200:207], a[4:7]\n\t" \
  "ds_read_b128 v[160:163], v2 offset:49152\n\t" \
  "ds_read_b128 v[164:167], v2 offset:57344\n\t" \
  "v_mfma_f32_16x16x128_f8f6f4 a[8:11], v[128:135], v[208:215], a[8:11]\n\t" \
  "ds_read_b128 v[168:171], v2 offset:50176\n\t" \
  "ds_read_b128 v[172:175], v2 offset:58368\n\t" \
  "v_mfma_f32_16x16x128_f8f6f4 a[12:15], v[128:135], v[216:223], a[12:15]\n\t" \
  "ds_read_b128 v[176:179], v2 offset:51200\n\t" \
  "ds_read_b128 v[180:183], v2 offset:59392\n\t" \
  "v_mfma_f32_16x16x128_f8f6f4 a[44:47], v[136:143], v[216:223], a[44:47]\n\t" \
  "ds_read_b128 v[184:187], v2 offset:52224\n\t" \
  "ds_read_b128 v[188:191], v2 offset:60416\n\t" \
  "v_mfma_f32_16x16x128_f8f6f4 a[40:43], v[136:143], v[208:215], a[40:43]\n\t" \
  "v_fma_f32 v12, v12, s58, s58\n\t" \
  "v_fma_f32 v13, v13, s58, s58\n\t" \
  "v_fma_f32 v14, v14, s58, s58\n\t" \
  "v_fma_f32 v15, v15, s58, s58\n\t" \
  "v_mfma_f32_16x16x128_f8f6f4 a[36:39], v[136:143], v[200:207], a[36:39]\n\t" \
  "v_rcp_f32_e32 v12, v12\n\t" \
  "v_rcp_f32_e32 v13, v13\n\t" \
  "v_mfma_f32_16x16x128_f8f6f4 a[32:35], v[136:143], v[192:199], a[32:35]\n\t" \
  "s_add_u32 s53, s51, 16384\n\t" \
  "s_add_u32 m0, s48, 81920\n\t" \
  "v_mfma_f32_16x16x128_f8f6f4 a[64:67], v[144:151], v[192:199], a[64:67]\n\t" \
  "buffer_load_dwordx4 v0, s[36:39], s53 offen sc0 sc1 lds\n\t" \
  "v_mfma_f32_16x16x128_f8f6f4 a[68:71], v[144:151], v[200:207], a[68:71]\n\t" \
  "v_rcp_f32_e32 v14, v14\n\t" \
  "v_rcp_f32_e32 v15, v15\n\t" \
  "v_mfma_f32_16x16x128_f8f6f4 a[72:75], v[144:151], v[208:215], a[72:75]\n\t" \
  "buffer_load_dwordx4 v0, s[36:39], s53 offen offset:1024 sc0 sc1 lds\n\t" \
  "v_mfma_f32_16x16x128_f8f6f4 a[76:79], v[144:151], v[216:223], a[76:79]\n\t" \
  "v_mul_f32_e32 v92, v92, v12\n\t" \
  "v_mul_f32_e32 v93, v93, v13\n\t" \
  "v_mul_f32_e32 v94, v94, v14\n\t" \
  "v_mul_f32_e32 v95, v95, v15\n\t" \
  "v_mfma_f32_16x16x128_f8f6f4 a[108:111], v[152:159], v[216:223], a[108:111]\n\t" \
  "buffer_load_dwordx4 v0, s[36:39], s53 offen offset:2048 sc0 sc1 lds\n\t" \
  "v_mfma_f32_16x16x128_f8f6f4 a[104:107], v[152:159], v[208:215], a[104:107]\n\t" \
  "s_xor_b32 s91, s68, 3342336\n\t" \
  "buffer_store_dwordx2 v[92:93], v5, s[40:43], s91 offen nt\n\t" \
  "v_mfma_f32_16x16x128_f8f6f4 a[100:103], v[152:159], v[200:207], a[100:103]\n\t" \
  "buffer_load_dwordx4 v0, s[36:39], s53 offen offset:3072 sc0 sc1 lds\n\t" \
  "v_mfma_f32_16x16x128_f8f6f4 a[96:99], v[152:159], v[192:199], a[96:99]\n\t" \
  "buffer_store_dwordx2 v[94:95], v5, s[40:43], s91 offen offset:128 nt\n\t" \
  "s_waitcnt lgkmcnt(0)\n\t" \
  "v_mfma_f32_16x16x128_f8f6f4 a[128:131], v[160:167], v[192:199], a[128:131]\n\t" \
  "s_waitcnt vmcnt(22)\n\t" \
  "s_barrier\n\t" \
  "v_mfma_f32_16x16x128_f8f6f4 a[132:135], v[160:167], v[200:207], a[132:135]\n\t" \
  "ds_read_b128 v[128:131], v1 offset:0\n\t" \
  "ds_read_b128 v[132:135], v1 offset:8192\n\t" \
  "v_mfma_f32_16x16x128_f8f6f4 a[136:139], v[160:167], v[208:215], a[136:139]\n\t" \
  "ds_read_b128 v[136:139], v1 offset:1024\n\t" \
  "ds_read_b128 v[140:143], v1 offset:9216\n\t" \
  "v_mfma_f32_16x16x128_f8f6f4 a[140:143], v[160:167], v[216:223], a[140:143]\n\t" \
  "ds_read_b128 v[144:147], v1 offset:2048\n\t" \
  "ds_read_b128 v[148:151], v1 offset:10240\n\t" \
  "v_mfma_f32_16x16x128_f8f6f4 a[172:175], v[168:175], v[216:223], a[172:175]\n\t" \
  "ds_read_b128 v[152:155], v1 offset:3072\n\t" \
  "ds_read_b128 v[156:159], v1 offset:11264\n\t" \
  "v_mfma_f32_16x16x128_f8f6f4 a[168:171], v[168:175], v[208:215], a[168:171]\n\t" \
  "v_mul_f32_e32 v96, s56, v96\n\t" \
  "v_mul_f32_e32 v97, s56, v97\n\t" \
  "v_mul_f32_e32 v98, s56, v98\n\t" \
  "v_mul_f32_e32 v99, s56, v99\n\t" \
  "v_mfma_f32_16x16x128_f8f6f4 a[164:167], v[168:175], v[200:207], a[164:167]\n\t" \
  "v_min_f32_e32 v16, s60, v96\n\t" \
  "v_min_f32_e32 v17, s60, v97\n\t" \
  "v_min_f32_e32 v18, s60, v98\n\t" \
  "v_min_f32_e32 v19, s60, v99\n\t" \
  "v_mfma_f32_16x16x128_f8f6f4 a[160:163], v[168:175], v[192:199], a[160:163]\n\t" \
  "s_add_u32 s53, s50, 16384\n\t" \
  "s_add_u32 m0, s48, 98304\n\t" \
  "v_mfma_f32_16x16x128_f8f6f4 a[192:195], v[176:183], v[192:199], a[192:195]\n\t" \
  "buffer_load_dwordx4 v0, s[36:39], s53 offen sc0 sc1 lds\n\t" \
  "v_mfma_f32_16x16x128_f8f6f4 a[196:199], v[176:183], v[200:207], a[196:199]\n\t" \
  "v_exp_f32_e32 v16, v16\n\t" \
  "v_exp_f32_e32 v17, v17\n\t" \
  "v_mfma_f32_16x16x128_f8f6f4 a[200:203], v[176:183], v[208:215], a[200:203]\n\t" \
  "buffer_load_dwordx4 v0, s[36:39], s53 offen offset:1024 sc0 sc1 lds\n\t" \
  "v_mfma_f32_16x16x128_f8f6f4 a[204:207], v[176:183], v[216:223], a[204:207]\n\t" \
  "v_exp_f32_e32 v18, v18\n\t" \
  "v_exp_f32_e32 v19, v19\n\t" \
  "v_mfma_f32_16x16x128_f8f6f4 a[236:239], v[184:191], v[216:223], a[236:239]\n\t" \
  "buffer_load_dwordx4 v0, s[36:39], s53 offen offset:2048 sc0 sc1 lds\n\t" \
  "v_mfma_f32_16x16x128_f8f6f4 a[232:235], v[184:191], v[208:215], a[232:235]\n\t" \
  "v_fma_f32 v16, v16, s58, s58\n\t" \
  "v_fma_f32 v17, v17, s58, s58\n\t" \
  "v_fma_f32 v18, v18, s58, s58\n\t" \
  "v_fma_f32 v19, v19, s58, s58\n\t" \
  "v_mfma_f32_16x16x128_f8f6f4 a[228:231], v[184:191], v[200:207], a[228:231]\n\t" \
  "buffer_load_dwordx4 v0, s[36:39], s53 offen offset:3072 sc0 sc1 lds\n\t" \
  "v_mfma_f32_16x16x128_f8f6f4 a[224:227], v[184:191], v[192:199], a[224:227]\n\t" \
  "v_rcp_f32_e32 v16, v16\n\t" \
  "v_rcp_f32_e32 v17, v17\n\t" \
  "s_waitcnt lgkmcnt(0)\n\t" \
  "v_mfma_f32_16x16x128_f8f6f4 a[144:147], v[160:167], v[224:231], a[144:147]\n\t" \
  "v_rcp_f32_e32 v18, v18\n\t" \
  "v_rcp_f32_e32 v19, v19\n\t" \
  "v_mfma_f32_16x16x128_f8f6f4 a[148:151], v[160:167], v[232:239], a[148:151]\n\t" \
  "ds_read_b128 v[192:195], v3 offset:16384\n\t" \
  "ds_read_b128 v[196:199], v3 offset:24576\n\t" \
  "v_mfma_f32_16x16x128_f8f6f4 a[152:155], v[160:167], v[240:247], a[152:155]\n\t" \
  "ds_read_b128 v[200:203], v3 offset:17408\n\t" \
  "ds_read_b128 v[204:207], v3 offset:25600\n\t" \
  "v_mfma_f32_16x16x128_f8f6f4 a[156:159], v[160:167], v[248:255], a[156:159]\n\t" \
  "ds_read_b128 v[208:211], v3 offset:18432\n\t" \
  "ds_read_b128 v[212:215], v3 offset:26624\n\t" \
  "v_mfma_f32_16x16x128_f8f6f4 a[188:191], v[168:175], v[248:255], a[188:191]\n\t" \
  "ds_read_b128 v[216:219], v3 offset:19456\n\t" \
  "ds_read_b128 v[220:223], v3 offset:27648\n\t" \
  "v_mfma_f32_16x16x128_f8f6f4 a[184:187], v[168:175], v[240:247], a[184:187]\n\t" \
  "v_mul_f32_e32 v96, v96, v16\n\t" \
  "v_mul_f32_e32 v97, v97, v17\n\t" \
  "v_mul_f32_e32 v98, v98, v18\n\t" \
  "v_mul_f32_e32 v99, v99, v19\n\t" \
  "v_mfma_f32_16x16x128_f8f6f4 a[180:183], v[168:175], v[232:239], a[180:183]\n\t" \
  "s_xor_b32 s91, s68, 4194304\n\t" \
  "buffer_store_dwordx2 v[96:97], v5, s[40:43], s91 offen nt\n\t" \
  "v_mfma_f32_16x16x128_f8f6f4 a[176:179], v[168:175], v[224:231], a[176:179]\n\t" \
  "s_add_u32 s53, s49, 49152\n\t" \
  "s_add_u32 m0, s48, 114688\n\t" \
  "v_mfma_f32_16x16x128_f8f6f4 a[208:211], v[176:183], v[224:231], a[208:211]\n\t" \
  "buffer_load_dwordx4 v0, s[36:39], s53 offen sc0 sc1 lds\n\t" \
  "v_mfma_f32_16x16x128_f8f6f4 a[212:215], v[176:183], v[232:239], a[212:215]\n\t" \
  "v_mfma_f32_16x16x128_f8f6f4 a[216:219], v[176:183], v[240:247], a[216:219]\n\t" \
  "buffer_load_dwordx4 v0, s[36:39], s53 offen offset:1024 sc0 sc1 lds\n\t" \
  "v_mfma_f32_16x16x128_f8f6f4 a[220:223], v[176:183], v[248:255], a[220:223]\n\t" \
  "buffer_store_dwordx2 v[98:99], v5, s[40:43], s91 offen offset:128 nt\n\t" \
  "v_mfma_f32_16x16x128_f8f6f4 a[252:255], v[184:191], v[248:255], a[252:255]\n\t" \
  "buffer_load_dwordx4 v0, s[36:39], s53 offen offset:2048 sc0 sc1 lds\n\t" \
  "v_mfma_f32_16x16x128_f8f6f4 a[248:251], v[184:191], v[240:247], a[248:251]\n\t" \
  "v_mul_f32_e32 v100, s56, v100\n\t" \
  "v_mul_f32_e32 v101, s56, v101\n\t" \
  "v_mul_f32_e32 v102, s56, v102\n\t" \
  "v_mul_f32_e32 v103, s56, v103\n\t" \
  "v_mfma_f32_16x16x128_f8f6f4 a[244:247], v[184:191], v[232:239], a[244:247]\n\t" \
  "buffer_load_dwordx4 v0, s[36:39], s53 offen offset:3072 sc0 sc1 lds\n\t" \
  "v_mfma_f32_16x16x128_f8f6f4 a[240:243], v[184:191], v[224:231], a[240:243]\n\t" \
  "s_add_u32 s49, s49, 65536\n\t" \
  "s_add_u32 s50, s50, 32768\n\t" \
  "s_add_u32 s51, s51, 32768\n\t" \
  "s_waitcnt lgkmcnt(0)\n\t" \
  "v_mfma_f32_16x16x128_f8f6f4 a[0:3], v[128:135], v[192:199], a[0:3]\n\t" \
  "s_waitcnt vmcnt(22)\n\t" \
  "s_barrier\n\t" \
  "v_mfma_f32_16x16x128_f8f6f4 a[4:7], v[128:135], v[200:207], a[4:7]\n\t" \
  "ds_read_b128 v[224:227], v3 offset:32768\n\t" \
  "ds_read_b128 v[228:231], v3 offset:40960\n\t" \
  "v_mfma_f32_16x16x128_f8f6f4 a[8:11], v[128:135], v[208:215], a[8:11]\n\t" \
  "ds_read_b128 v[232:235], v3 offset:33792\n\t" \
  "ds_read_b128 v[236:239], v3 offset:41984\n\t" \
  "v_mfma_f32_16x16x128_f8f6f4 a[12:15], v[128:135], v[216:223], a[12:15]\n\t" \
  "ds_read_b128 v[240:243], v3 offset:34816\n\t" \
  "ds_read_b128 v[244:247], v3 offset:43008\n\t" \
  "v_mfma_f32_16x16x128_f8f6f4 a[44:47], v[136:143], v[216:223], a[44:47]\n\t" \
  "ds_read_b128 v[248:251], v3 offset:35840\n\t" \
  "ds_read_b128 v[252:255], v3 offset:44032\n\t" \
  "v_mfma_f32_16x16x128_f8f6f4 a[40:43], v[136:143], v[208:215], a[40:43]\n\t" \
  "v_min_f32_e32 v20, s60, v100\n\t" \
  "v_min_f32_e32 v21, s60, v101\n\t" \
  "v_min_f32_e32 v22, s60, v102\n\t" \
  "v_min_f32_e32 v23, s60, v103\n\t" \
  "v_mfma_f32_16x16x128_f8f6f4 a[36:39], v[136:143], v[200:207], a[36:39]\n\t" \
  "v_exp_f32_e32 v20, v20\n\t" \
  "v_exp_f32_e32 v21, v21\n\t" \
  "v_mfma_f32_16x16x128_f8f6f4 a[32:35], v[136:143], v[192:199], a[32:35]\n\t" \
  "s_mov_b32 s53, s49\n\t" \
  "s_mov_b32 m0, s48\n\t" \
  "v_mfma_f32_16x16x128_f8f6f4 a[64:67], v[144:151], v[192:199], a[64:67]\n\t" \
  "buffer_load_dwordx4 v0, s[36:39], s53 offen sc0 sc1 lds\n\t" \
  "v_mfma_f32_16x16x128_f8f6f4 a[68:71], v[144:151], v[200:207], a[68:71]\n\t" \
  "v_exp_f32_e32 v22, v22\n\t" \
  "v_exp_f32_e32 v23, v23\n\t" \
  "v_mfma_f32_16x16x128_f8f6f4 a[72:75], v[144:151], v[208:215], a[72:75]\n\t" \
  "buffer_load_dwordx4 v0, s[36:39], s53 offen offset:1024 sc0 sc1 lds\n\t" \
  "v_mfma_f32_16x16x128_f8f6f4 a[76:79], v[144:151], v[216:223], a[76:79]\n\t" \
  "v_fma_f32 v20, v20, s58, s58\n\t" \
  "v_fma_f32 v21, v21, s58, s58\n\t" \
  "v_fma_f32 v22, v22, s58, s58\n\t" \
  "v_fma_f32 v23, v23, s58, s58\n\t" \
  "v_mfma_f32_16x16x128_f8f6f4 a[108:111], v[152:159], v[216:223], a[108:111]\n\t" \
  "buffer_load_dwordx4 v0, s[36:39], s53 offen offset:2048 sc0 sc1 lds\n\t" \
  "v_mfma_f32_16x16x128_f8f6f4 a[104:107], v[152:159], v[208:215], a[104:107]\n\t" \
  "v_rcp_f32_e32 v20, v20\n\t" \
  "v_rcp_f32_e32 v21, v21\n\t" \
  "v_mfma_f32_16x16x128_f8f6f4 a[100:103], v[152:159], v[200:207], a[100:103]\n\t" \
  "buffer_load_dwordx4 v0, s[36:39], s53 offen offset:3072 sc0 sc1 lds\n\t" \
  "v_mfma_f32_16x16x128_f8f6f4 a[96:99], v[152:159], v[192:199], a[96:99]\n\t" \
  "v_rcp_f32_e32 v22, v22\n\t" \
  "v_rcp_f32_e32 v23, v23\n\t" \
  "s_waitcnt lgkmcnt(0)\n\t" \
  "v_mfma_f32_16x16x128_f8f6f4 a[16:19], v[128:135], v[224:231], a[16:19]\n\t" \
  "v_mul_f32_e32 v100, v100, v20\n\t" \
  "v_mul_f32_e32 v101, v101, v21\n\t" \
  "v_mul_f32_e32 v102, v102, v22\n\t" \
  "v_mul_f32_e32 v103, v103, v23\n\t" \
  "v_mfma_f32_16x16x128_f8f6f4 a[20:23], v[128:135], v[232:239], a[20:23]\n\t" \
  "ds_read_b128 v[160:163], v1 offset:49152\n\t" \
  "ds_read_b128 v[164:167], v1 offset:57344\n\t" \
  "v_mfma_f32_16x16x128_f8f6f4 a[24:27], v[128:135], v[240:247], a[24:27]\n\t" \
  "ds_read_b128 v[168:171], v1 offset:50176\n\t" \
  "ds_read_b128 v[172:175], v1 offset:58368\n\t" \
  "v_mfma_f32_16x16x128_f8f6f4 a[28:31], v[128:135], v[248:255], a[28:31]\n\t" \
  "ds_read_b128 v[176:179], v1 offset:51200\n\t" \
  "ds_read_b128 v[180:183], v1 offset:59392\n\t" \
  "v_mfma_f32_16x16x128_f8f6f4 a[60:63], v[136:143], v[248:255], a[60:63]\n\t" \
  "ds_read_b128 v[184:187], v1 offset:52224\n\t" \
  "ds_read_b128 v[188:191], v1 offset:60416\n\t" \
  "v_mfma_f32_16x16x128_f8f6f4 a[56:59], v[136:143], v[240:247], a[56:59]\n\t" \
  "s_xor_b32 s91, s68, 4259840\n\t" \
  "buffer_store_dwordx2 v[100:101], v5, s[40:43], s91 offen nt\n\t" \
  "v_mfma_f32_16x16x128_f8f6f4 a[52:55], v[136:143], v[232:239], a[52:55]\n\t" \
  "buffer_store_dwordx2 v[102:103], v5, s[40:43], s91 offen offset:128 nt\n\t" \
  "v_mfma_f32_16x16x128_f8f6f4 a[48:51], v[136:143], v[224:231], a[48:51]\n\t" \
  "s_mov_b32 s53, s50\n\t" \
  "s_add_u32 m0, s48, 16384\n\t" \
  "v_mfma_f32_16x16x128_f8f6f4 a[80:83], v[144:151], v[224:231], a[80:83]\n\t" \
  "buffer_load_dwordx4 v0, s[36:39], s53 offen sc0 sc1 lds\n\t" \
  "v_mfma_f32_16x16x128_f8f6f4 a[84:87], v[144:151], v[232:239], a[84:87]\n\t" \
  "v_mul_f32_e32 v104, s56, v104\n\t" \
  "v_mul_f32_e32 v105, s56, v105\n\t" \
  "v_mul_f32_e32 v106, s56, v106\n\t" \
  "v_mul_f32_e32 v107, s56, v107\n\t" \
  "v_mfma_f32_16x16x128_f8f6f4 a[88:91], v[144:151], v[240:247], a[88:91]\n\t" \
  "buffer_load_dwordx4 v0, s[36:39], s53 offen offset:1024 sc0 sc1 lds\n\t" \
  "v_mfma_f32_16x16x128_f8f6f4 a[92:95], v[144:151], v[248:255], a[92:95]\n\t" \
  "v_min_f32_e32 v24, s60, v104\n\t" \
  "v_min_f32_e32 v25, s60, v105\n\t" \
  "v_min_f32_e32 v26, s60, v106\n\t" \
  "v_min_f32_e32 v27, s60, v107\n\t" \
  "v_mfma_f32_16x16x128_f8f6f4 a[124:127], v[152:159], v[248:255], a[124:127]\n\t" \
  "buffer_load_dwordx4 v0, s[36:39], s53 offen offset:2048 sc0 sc1 lds\n\t" \
  "v_mfma_f32_16x16x128_f8f6f4 a[120:123], v[152:159], v[240:247], a[120:123]\n\t" \
  "v_exp_f32_e32 v24, v24\n\t" \
  "v_exp_f32_e32 v25, v25\n\t" \
  "v_mfma_f32_16x16x128_f8f6f4 a[116:119], v[152:159], v[232:239], a[116:119]\n\t" \
  "buffer_load_dwordx4 v0, s[36:39], s53 offen offset:3072 sc0 sc1 lds\n\t" \
  "v_mfma_f32_16x16x128_f8f6f4 a[112:115], v[152:159], v[224:231], a[112:115]\n\t" \
  "v_exp_f32_e32 v26, v26\n\t" \
  "v_exp_f32_e32 v27, v27\n\t" \
  "s_waitcnt lgkmcnt(0)\n\t" \
  "v_mfma_f32_16x16x128_f8f6f4 a[144:147], v[160:167], v[224:231], a[144:147]\n\t" \
  "s_waitcnt vmcnt(21)\n\t" \
  "s_barrier\n\t" \
  "v_mfma_f32_16x16x128_f8f6f4 a[148:151], v[160:167], v[232:239], a[148:151]\n\t" \
  "ds_read_b128 v[128:131], v2 offset:0\n\t" \
  "ds_read_b128 v[132:135], v2 offset:8192\n\t" \
  "v_mfma_f32_16x16x128_f8f6f4 a[152:155], v[160:167], v[240:247], a[152:155]\n\t" \
  "ds_read_b128 v[136:139], v2 offset:1024\n\t" \
  "ds_read_b128 v[140:143], v2 offset:9216\n\t" \
  "v_mfma_f32_16x16x128_f8f6f4 a[156:159], v[160:167], v[248:255], a[156:159]\n\t" \
  "ds_read_b128 v[144:147], v2 offset:2048\n\t" \
  "ds_read_b128 v[148:151], v2 offset:10240\n\t" \
  "v_mfma_f32_16x16x128_f8f6f4 a[188:191], v[168:175], v[248:255], a[188:191]\n\t" \
  "ds_read_b128 v[152:155], v2 offset:3072\n\t" \
  "ds_read_b128 v[156:159], v2 offset:11264\n\t" \
  "v_mfma_f32_16x16x128_f8f6f4 a[184:187], v[168:175], v[240:247], a[184:187]\n\t" \
  "v_fma_f32 v24, v24, s58, s58\n\t" \
  "v_fma_f32 v25, v25, s58, s58\n\t" \
  "v_fma_f32 v26, v26, s58, s58\n\t" \
  "v_fma_f32 v27, v27, s58, s58\n\t" \
  "v_mfma_f32_16x16x128_f8f6f4 a[180:183], v[168:175], v[232:239], a[180:183]\n\t" \
  "v_rcp_f32_e32 v24, v24\n\t" \
  "v_rcp_f32_e32 v25, v25\n\t" \
  "v_mfma_f32_16x16x128_f8f6f4 a[176:179], v[168:175], v[224:231], a[176:179]\n\t" \
  "s_mov_b32 s53, s51\n\t" \
  "s_add_u32 m0, s48, 32768\n\t" \
  "v_mfma_f32_16x16x128_f8f6f4 a[208:211], v[176:183], v[224:231], a[208:211]\n\t" \
  "buffer_load_dwordx4 v0, s[36:39], s53 offen sc0 sc1 lds\n\t" \
  "v_mfma_f32_16x16x128_f8f6f4 a[212:215], v[176:183], v[232:239], a[212:215]\n\t" \
  "v_rcp_f32_e32 v26, v26\n\t" \
  "v_rcp_f32_e32 v27, v27\n\t" \
  "v_mfma_f32_16x16x128_f8f6f4 a[216:219], v[176:183], v[240:247], a[216:219]\n\t" \
  "buffer_load_dwordx4 v0, s[36:39], s53 offen offset:1024 sc0 sc1 lds\n\t" \
  "v_mfma_f32_16x16x128_f8f6f4 a[220:223], v[176:183], v[248:255], a[220:223]\n\t" \
  "v_mul_f32_e32 v104, v104, v24\n\t" \
  "v_mul_f32_e32 v105, v105, v25\n\t" \
  "v_mul_f32_e32 v106, v106, v26\n\t" \
  "v_mul_f32_e32 v107, v107, v27\n\t" \
  "v_mfma_f32_16x16x128_f8f6f4 a[252:255], v[184:191], v[248:255], a[252:255]\n\t" \
  "buffer_load_dwordx4 v0, s[36:39], s53 offen offset:2048 sc0 sc1 lds\n\t" \
  "v_mfma_f32_16x16x128_f8f6f4 a[248:251], v[184:191], v[240:247], a[248:251]\n\t" \
  "s_xor_b32 s91, s68, 4325376\n\t" \
  "buffer_store_dwordx2 v[104:105], v5, s[40:43], s91 offen nt\n\t" \
  "v_mfma_f32_16x16x128_f8f6f4 a[244:247], v[184:191], v[232:239], a[244:247]\n\t" \
  "buffer_load_dwordx4 v0, s[36:39], s53 offen offset:3072 sc0 sc1 lds\n\t" \
  "v_mfma_f32_16x16x128_f8f6f4 a[240:243], v[184:191], v[224:231], a[240:243]\n\t" \
  "buffer_store_dwordx2 v[106:107], v5, s[40:43], s91 offen offset:128 nt\n\t" \
  "s_waitcnt lgkmcnt(0)\n\t" \
  "v_mfma_f32_16x16x128_f8f6f4 a[128:131], v[160:167], v[192:199], a[128:131]\n\t" \
  "v_mul_f32_e32 v108, s56, v108\n\t" \
  "v_mul_f32_e32 v109, s56, v109\n\t" \
  "v_mul_f32_e32 v110, s56, v110\n\t" \
  "v_mul_f32_e32 v111, s56, v111\n\t" \
  "v_mfma_f32_16x16x128_f8f6f4 a[132:135], v[160:167], v[200:207], a[132:135]\n\t" \
  "ds_read_b128 v[224:227], v4 offset:16384\n\t" \
  "ds_read_b128 v[228:231], v4 offset:24576\n\t" \
  "v_mfma_f32_16x16x128_f8f6f4 a[136:139], v[160:167], v[208:215], a[136:139]\n\t" \
  "ds_read_b128 v[232:235], v4 offset:17408\n\t" \
  "ds_read_b128 v[236:239], v4 offset:25600\n\t" \
  "v_mfma_f32_16x16x128_f8f6f4 a[140:143], v[160:167], v[216:223], a[140:143]\n\t" \
  "ds_read_b128 v[240:243], v4 offset:18432\n\t" \
  "ds_read_b128 v[244:247], v4 offset:26624\n\t" \
  "v_mfma_f32_16x16x128_f8f6f4 a[172:175], v[168:175], v[216:223], a[172:175]\n\t" \
  "ds_read_b128 v[248:251], v4 offset:19456\n\t" \
  "ds_read_b128 v[252:255], v4 offset:27648\n\t" \
  "v_mfma_f32_16x16x128_f8f6f4 a[168:171], v[168:175], v[208:215], a[168:171]\n\t" \
  "v_mfma_f32_16x16x128_f8f6f4 a[164:167], v[168:175], v[200:207], a[164:167]\n\t" \
  "v_min_f32_e32 v28, s60, v108\n\t" \
  "v_min_f32_e32 v29, s60, v109\n\t" \
  "v_min_f32_e32 v30, s60, v110\n\t" \
  "v_min_f32_e32 v31, s60, v111\n\t" \
  "v_mfma_f32_16x16x128_f8f6f4 a[160:163], v[168:175], v[192:199], a[160:163]\n\t" \
  "s_add_u32 s53, s49, 16384\n\t" \
  "s_add_u32 m0, s48, 49152\n\t" \
  "v_mfma_f32_16x16x128_f8f6f4 a[192:195], v[176:183], v[192:199], a[192:195]\n\t" \
  "buffer_load_dwordx4 v0, s[36:39], s53 offen sc0 sc1 lds\n\t" \
  "v_mfma_f32_16x16x128_f8f6f4 a[196:199], v[176:183], v[200:207], a[196:199]\n\t" \
  "v_exp_f32_e32 v28, v28\n\t" \
  "v_exp_f32_e32 v29, v29\n\t" \
  "v_mfma_f32_16x16x128_f8f6f4 a[200:203], v[176:183], v[208:215], a[200:203]\n\t" \
  "buffer_load_dwordx4 v0, s[36:39], s53 offen offset:1024 sc0 sc1 lds\n\t" \
  "v_mfma_f32_16x16x128_f8f6f4 a[204:207], v[176:183], v[216:223], a[204:207]\n\t" \
  "v_exp_f32_e32 v30, v30\n\t" \
  "v_exp_f32_e32 v31, v31\n\t" \
  "v_mfma_f32_16x16x128_f8f6f4 a[236:239], v[184:191], v[216:223], a[236:239]\n\t" \
  "buffer_load_dwordx4 v0, s[36:39], s53 offen offset:2048 sc0 sc1 lds\n\t" \
  "v_mfma_f32_16x16x128_f8f6f4 a[232:235], v[184:191], v[208:215], a[232:235]\n\t" \
  "v_fma_f32 v28, v28, s58, s58\n\t" \
  "v_fma_f32 v29, v29, s58, s58\n\t" \
  "v_fma_f32 v30, v30, s58, s58\n\t" \
  "v_fma_f32 v31, v31, s58, s58\n\t" \
  "v_mfma_f32_16x16x128_f8f6f4 a[228:231], v[184:191], v[200:207], a[228:231]\n\t" \
  "buffer_load_dwordx4 v0, s[36:39], s53 offen offset:3072 sc0 sc1 lds\n\t" \
  "v_mfma_f32_16x16x128_f8f6f4 a[224:227], v[184:191], v[192:199], a[224:227]\n\t" \
  "v_rcp_f32_e32 v28, v28\n\t" \
  "v_rcp_f32_e32 v29, v29\n\t" \
  "s_waitcnt lgkmcnt(0)\n\t" \
  "v_mfma_f32_16x16x128_f8f6f4 a[16:19], v[128:135], v[224:231], a[16:19]\n\t" \
  "s_waitcnt vmcnt(20)\n\t" \
  "s_barrier\n\t" \
  "v_mfma_f32_16x16x128_f8f6f4 a[20:23], v[128:135], v[232:239], a[20:23]\n\t" \
  "ds_read_b128 v[192:195], v4 offset:32768\n\t" \
  "ds_read_b128 v[196:199], v4 offset:40960\n\t" \
  "v_mfma_f32_16x16x128_f8f6f4 a[24:27], v[128:135], v[240:247], a[24:27]\n\t" \
  "ds_read_b128 v[200:203], v4 offset:33792\n\t" \
  "ds_read_b128 v[204:207], v4 offset:41984\n\t" \
  "v_mfma_f32_16x16x128_f8f6f4 a[28:31], v[128:135], v[248:255], a[28:31]\n\t" \
  "ds_read_b128 v[208:211], v4 offset:34816\n\t" \
  "ds_read_b128 v[212:215], v4 offset:43008\n\t" \
  "v_mfma_f32_16x16x128_f8f6f4 a[60:63], v[136:143], v[248:255], a[60:63]\n\t" \
  "ds_read_b128 v[216:219], v4 offset:35840\n\t" \
  "ds_read_b128 v[220:223], v4 offset:44032\n\t" \
  "v_mfma_f32_16x16x128_f8f6f4 a[56:59], v[136:143], v[240:247], a[56:59]\n\t" \
  "v_rcp_f32_e32 v30, v30\n\t" \
  "v_rcp_f32_e32 v31, v31\n\t" \
  "v_mfma_f32_16x16x128_f8f6f4 a[52:55], v[136:143], v[232:239], a[52:55]\n\t" \
  "v_mul_f32_e32 v108, v108, v28\n\t" \
  "v_mul_f32_e32 v109, v109, v29\n\t" \
  "v_mul_f32_e32 v110, v110, v30\n\t" \
  "v_mul_f32_e32 v111, v111, v31\n\t" \
  "v_mfma_f32_16x16x128_f8f6f4 a[48:51], v[136:143], v[224:231], a[48:51]\n\t" \
  "s_add_u32 s53, s49, 32768\n\t" \
  "s_add_u32 m0, s48, 65536\n\t" \
  "v_mfma_f32_16x16x128_f8f6f4 a[80:83], v[144:151], v[224:231], a[80:83]\n\t" \
  "buffer_load_dwordx4 v0, s[36:39], s53 offen sc0 sc1 lds\n\t" \
  "v_mfma_f32_16x16x128_f8f6f4 a[84:87], v[144:151], v[232:239], a[84:87]\n\t" \
  "buffer_load_dword v7, v8, s[44:47], s76 offen sc0 sc1\n\t" \
  "v_mfma_f32_16x16x128_f8f6f4 a[88:91], v[144:151], v[240:247], a[88:91]\n\t" \
  "buffer_load_dwordx4 v0, s[36:39], s53 offen offset:1024 sc0 sc1 lds\n\t" \
  "v_mfma_f32_16x16x128_f8f6f4 a[92:95], v[144:151], v[248:255], a[92:95]\n\t" \
  "s_xor_b32 s91, s68, 4390912\n\t" \
  "buffer_store_dwordx2 v[108:109], v5, s[40:43], s91 offen nt\n\t" \
  "v_mfma_f32_16x16x128_f8f6f4 a[124:127], v[152:159], v[248:255], a[124:127]\n\t" \
  "buffer_load_dwordx4 v0, s[36:39], s53 offen offset:2048 sc0 sc1 lds\n\t" \
  "v_mfma_f32_16x16x128_f8f6f4 a[120:123], v[152:159], v[240:247], a[120:123]\n\t" \
  "buffer_store_dwordx2 v[110:111], v5, s[40:43], s91 offen offset:128 nt\n\t" \
  "v_mfma_f32_16x16x128_f8f6f4 a[116:119], v[152:159], v[232:239], a[116:119]\n\t" \
  "buffer_load_dwordx4 v0, s[36:39], s53 offen offset:3072 sc0 sc1 lds\n\t" \
  "v_mfma_f32_16x16x128_f8f6f4 a[112:115], v[152:159], v[224:231], a[112:115]\n\t" \
  "v_mul_f32_e32 v112, s56, v112\n\t" \
  "v_mul_f32_e32 v113, s56, v113\n\t" \
  "v_mul_f32_e32 v114, s56, v114\n\t" \
  "v_mul_f32_e32 v115, s56, v115\n\t" \
  "s_waitcnt lgkmcnt(0)\n\t" \
  "v_mfma_f32_16x16x128_f8f6f4 a[0:3], v[128:135], v[192:199], a[0:3]\n\t" \
  "v_min_f32_e32 v12, s60, v112\n\t" \
  "v_min_f32_e32 v13, s60, v113\n\t" \
  "v_min_f32_e32 v14, s60, v114\n\t" \
  "v_min_f32_e32 v15, s60, v115\n\t" \
  "v_mfma_f32_16x16x128_f8f6f4 a[4:7], v[128:135], v[200:207], a[4:7]\n\t" \
  "ds_read_b128 v[160:163], v2 offset:49152\n\t" \
  "ds_read_b128 v[164:167], v2 offset:57344\n\t" \
  "v_mfma_f32_16x16x128_f8f6f4 a[8:11], v[128:135], v[208:215], a[8:11]\n\t" \
  "ds_read_b128 v[168:171], v2 offset:50176\n\t" \
  "ds_read_b128 v[172:175], v2 offset:58368\n\t" \
  "v_mfma_f32_16x16x128_f8f6f4 a[12:15], v[128:135], v[216:223], a[12:15]\n\t" \
  "ds_read_b128 v[176:179], v2 offset:51200\n\t" \
  "ds_read_b128 v[180:183], v2 offset:59392\n\t" \
  "v_mfma_f32_16x16x128_f8f6f4 a[44:47], v[136:143], v[216:223], a[44:47]\n\t" \
  "ds_read_b128 v[184:187], v2 offset:52224\n\t" \
  "ds_read_b128 v[188:191], v2 offset:60416\n\t" \
  "v_mfma_f32_16x16x128_f8f6f4 a[40:43], v[136:143], v[208:215], a[40:43]\n\t" \
  "v_exp_f32_e32 v12, v12\n\t" \
  "v_exp_f32_e32 v13, v13\n\t" \
  "v_mfma_f32_16x16x128_f8f6f4 a[36:39], v[136:143], v[200:207], a[36:39]\n\t" \
  "v_exp_f32_e32 v14, v14\n\t" \
  "v_exp_f32_e32 v15, v15\n\t" \
  "v_mfma_f32_16x16x128_f8f6f4 a[32:35], v[136:143], v[192:199], a[32:35]\n\t" \
  "s_add_u32 s53, s51, 16384\n\t" \
  "s_add_u32 m0, s48, 81920\n\t" \
  "v_mfma_f32_16x16x128_f8f6f4 a[64:67], v[144:151], v[192:199], a[64:67]\n\t" \
  "buffer_load_dwordx4 v0, s[36:39], s53 offen sc0 sc1 lds\n\t" \
  "v_mfma_f32_16x16x128_f8f6f4 a[68:71], v[144:151], v[200:207], a[68:71]\n\t" \
  "v_fma_f32 v12, v12, s58, s58\n\t" \
  "v_fma_f32 v13, v13, s58, s58\n\t" \
  "v_fma_f32 v14, v14, s58, s58\n\t" \
  "v_fma_f32 v15, v15, s58, s58\n\t" \
  "v_mfma_f32_16x16x128_f8f6f4 a[72:75], v[144:151], v[208:215], a[72:75]\n\t" \
  "buffer_load_dwordx4 v0, s[36:39], s53 offen offset:1024 sc0 sc1 lds\n\t" \
  "v_mfma_f32_16x16x128_f8f6f4 a[76:79], v[144:151], v[216:223], a[76:79]\n\t" \
  "v_rcp_f32_e32 v12, v12\n\t" \
  "v_rcp_f32_e32 v13, v13\n\t" \
  "v_mfma_f32_16x16x128_f8f6f4 a[108:111], v[152:159], v[216:223], a[108:111]\n\t" \
  "buffer_load_dwordx4 v0, s[36:39], s53 offen offset:2048 sc0 sc1 lds\n\t" \
  "v_mfma_f32_16x16x128_f8f6f4 a[104:107], v[152:159], v[208:215], a[104:107]\n\t" \
  "v_rcp_f32_e32 v14, v14\n\t" \
  "v_rcp_f32_e32 v15, v15\n\t" \
  "v_mfma_f32_16x16x128_f8f6f4 a[100:103], v[152:159], v[200:207], a[100:103]\n\t" \
  "buffer_load_dwordx4 v0, s[36:39], s53 offen offset:3072 sc0 sc1 lds\n\t" \
  "v_mfma_f32_16x16x128_f8f6f4 a[96:99], v[152:159], v[192:199], a[96:99]\n\t" \
  "v_mul_f32_e32 v112, v112, v12\n\t" \
  "v_mul_f32_e32 v113, v113, v13\n\t" \
  "v_mul_f32_e32 v114, v114, v14\n\t" \
  "v_mul_f32_e32 v115, v115, v15\n\t" \
  "s_waitcnt lgkmcnt(0)\n\t" \
  "v_mfma_f32_16x16x128_f8f6f4 a[128:131], v[160:167], v[192:199], a[128:131]\n\t" \
  "s_waitcnt vmcnt(21)\n\t" \
  "s_barrier\n\t" \
  "v_mfma_f32_16x16x128_f8f6f4 a[132:135], v[160:167], v[200:207], a[132:135]\n\t" \
  "ds_read_b128 v[128:131], v1 offset:0\n\t" \
  "ds_read_b128 v[132:135], v1 offset:8192\n\t" \
  "v_mfma_f32_16x16x128_f8f6f4 a[136:139], v[160:167], v[208:215], a[136:139]\n\t" \
  "ds_read_b128 v[136:139], v1 offset:1024\n\t" \
  "ds_read_b128 v[140:143], v1 offset:9216\n\t" \
  "v_mfma_f32_16x16x128_f8f6f4 a[140:143], v[160:167], v[216:223], a[140:143]\n\t" \
  "ds_read_b128 v[144:147], v1 offset:2048\n\t" \
  "ds_read_b128 v[148:151], v1 offset:10240\n\t" \
  "v_mfma_f32_16x16x128_f8f6f4 a[172:175], v[168:175], v[216:223], a[172:175]\n\t" \
  "ds_read_b128 v[152:155], v1 offset:3072\n\t" \
  "ds_read_b128 v[156:159], v1 offset:11264\n\t" \
  "v_mfma_f32_16x16x128_f8f6f4 a[168:171], v[168:175], v[208:215], a[168:171]\n\t" \
  "s_xor_b32 s91, s68, 5242880\n\t" \
  "buffer_store_dwordx2 v[112:113], v5, s[40:43], s91 offen nt\n\t" \
  "v_mfma_f32_16x16x128_f8f6f4 a[164:167], v[168:175], v[200:207], a[164:167]\n\t" \
  "buffer_store_dwordx2 v[114:115], v5, s[40:43], s91 offen offset:128 nt\n\t" \
  "v_mfma_f32_16x16x128_f8f6f4 a[160:163], v[168:175], v[192:199], a[160:163]\n\t" \
  "s_add_u32 s53, s50, 16384\n\t" \
  "s_add_u32 m0, s48, 98304\n\t" \
  "v_mfma_f32_16x16x128_f8f6f4 a[192:195], v[176:183], v[192:199], a[192:195]\n\t" \
  "buffer_load_dwordx4 v0, s[36:39], s53 offen sc0 sc1 lds\n\t" \
  "v_mfma_f32_16x16x128_f8f6f4 a[196:199], v[176:183], v[200:207], a[196:199]\n\t" \
  "v_mul_f32_e32 v116, s56, v116\n\t" \
  "v_mul_f32_e32 v117, s56, v117\n\t" \
  "v_mul_f32_e32 v118, s56, v118\n\t" \
  "v_mul_f32_e32 v119, s56, v119\n\t" \
  "v_mfma_f32_16x16x128_f8f6f4 a[200:203], v[176:183], v[208:215], a[200:203]\n\t" \
  "buffer_load_dwordx4 v0, s[36:39], s53 offen offset:1024 sc0 sc1 lds\n\t" \
  "v_mfma_f32_16x16x128_f8f6f4 a[204:207], v[176:183], v[216:223], a[204:207]\n\t" \
  "v_min_f32_e32 v16, s60, v116\n\t" \
  "v_min_f32_e32 v17, s60, v117\n\t" \
  "v_min_f32_e32 v18, s60, v118\n\t" \
  "v_min_f32_e32 v19, s60, v119\n\t" \
  "v_mfma_f32_16x16x128_f8f6f4 a[236:239], v[184:191], v[216:223], a[236:239]\n\t" \
  "buffer_load_dwordx4 v0, s[36:39], s53 offen offset:2048 sc0 sc1 lds\n\t" \
  "v_mfma_f32_16x16x128_f8f6f4 a[232:235], v[184:191], v[208:215], a[232:235]\n\t" \
  "v_mfma_f32_16x16x128_f8f6f4 a[228:231], v[184:191], v[200:207], a[228:231]\n\t" \
  "buffer_load_dwordx4 v0, s[36:39], s53 offen offset:3072 sc0 sc1 lds\n\t" \
  "v_mfma_f32_16x16x128_f8f6f4 a[224:227], v[184:191], v[192:199], a[224:227]\n\t" \
  "v_exp_f32_e32 v16, v16\n\t" \
  "v_exp_f32_e32 v17, v17\n\t" \
  "s_waitcnt lgkmcnt(0)\n\t" \
  "v_mfma_f32_16x16x128_f8f6f4 a[144:147], v[160:167], v[224:231], a[144:147]\n\t" \
  "v_exp_f32_e32 v18, v18\n\t" \
  "v_exp_f32_e32 v19, v19\n\t" \
  "v_mfma_f32_16x16x128_f8f6f4 a[148:151], v[160:167], v[232:239], a[148:151]\n\t" \
  "ds_read_b128 v[192:195], v3 offset:16384\n\t" \
  "ds_read_b128 v[196:199], v3 offset:24576\n\t" \
  "v_mfma_f32_16x16x128_f8f6f4 a[152:155], v[160:167], v[240:247], a[152:155]\n\t" \
  "ds_read_b128 v[200:203], v3 offset:17408\n\t" \
  "ds_read_b128 v[204:207], v3 offset:25600\n\t" \
  "v_mfma_f32_16x16x128_f8f6f4 a[156:159], v[160:167], v[248:255], a[156:159]\n\t" \
  "ds_read_b128 v[208:211], v3 offset:18432\n\t" \
  "ds_read_b128 v[212:215], v3 offset:26624\n\t" \
  "v_mfma_f32_16x16x128_f8f6f4 a[188:191], v[168:175], v[248:255], a[188:191]\n\t" \
  "ds_read_b128 v[216:219], v3 offset:19456\n\t" \
  "ds_read_b128 v[220:223], v3 offset:27648\n\t" \
  "v_mfma_f32_16x16x128_f8f6f4 a[184:187], v[168:175], v[240:247], a[184:187]\n\t" \
  "v_fma_f32 v16, v16, s58, s58\n\t" \
  "v_fma_f32 v17, v17, s58, s58\n\t" \
  "v_fma_f32 v18, v18, s58, s58\n\t" \
  "v_fma_f32 v19, v19, s58, s58\n\t" \
  "v_mfma_f32_16x16x128_f8f6f4 a[180:183], v[168:175], v[232:239], a[180:183]\n\t" \
  "v_rcp_f32_e32 v16, v16\n\t" \
  "v_rcp_f32_e32 v17, v17\n\t" \
  "v_mfma_f32_16x16x128_f8f6f4 a[176:179], v[168:175], v[224:231], a[176:179]\n\t" \
  "s_add_u32 s53, s49, 49152\n\t" \
  "s_add_u32 m0, s48, 114688\n\t" \
  "v_mfma_f32_16x16x128_f8f6f4 a[208:211], v[176:183], v[224:231], a[208:211]\n\t" \
  "buffer_load_dwordx4 v0, s[36:39], s53 offen sc0 sc1 lds\n\t" \
  "v_mfma_f32_16x16x128_f8f6f4 a[212:215], v[176:183], v[232:239], a[212:215]\n\t" \
  "v_rcp_f32_e32 v18, v18\n\t" \
  "v_rcp_f32_e32 v19, v19\n\t" \
  "v_mfma_f32_16x16x128_f8f6f4 a[216:219], v[176:183], v[240:247], a[216:219]\n\t" \
  "buffer_load_dwordx4 v0, s[36:39], s53 offen offset:1024 sc0 sc1 lds\n\t" \
  "v_mfma_f32_16x16x128_f8f6f4 a[220:223], v[176:183], v[248:255], a[220:223]\n\t" \
  "v_mul_f32_e32 v116, v116, v16\n\t" \
  "v_mul_f32_e32 v117, v117, v17\n\t" \
  "v_mul_f32_e32 v118, v118, v18\n\t" \
  "v_mul_f32_e32 v119, v119, v19\n\t" \
  "v_mfma_f32_16x16x128_f8f6f4 a[252:255], v[184:191], v[248:255], a[252:255]\n\t" \
  "buffer_load_dwordx4 v0, s[36:39], s53 offen offset:2048 sc0 sc1 lds\n\t" \
  "v_mfma_f32_16x16x128_f8f6f4 a[248:251], v[184:191], v[240:247], a[248:251]\n\t" \
  "s_xor_b32 s91, s68, 5308416\n\t" \
  "buffer_store_dwordx2 v[116:117], v5, s[40:43], s91 offen nt\n\t" \
  "v_mfma_f32_16x16x128_f8f6f4 a[244:247], v[184:191], v[232:239], a[244:247]\n\t" \
  "buffer_load_dwordx4 v0, s[36:39], s53 offen offset:3072 sc0 sc1 lds\n\t" \
  "v_mfma_f32_16x16x128_f8f6f4 a[240:243], v[184:191], v[224:231], a[240:243]\n\t" \
  "s_add_u32 s49, s49, 65536\n\t" \
  "s_add_u32 s50, s50, 32768\n\t" \
  "s_add_u32 s51, s51, 32768\n\t" \
  "s_waitcnt lgkmcnt(0)\n\t" \
  "v_mfma_f32_16x16x128_f8f6f4 a[0:3], v[128:135], v[192:199], a[0:3]\n\t" \
  "s_waitcnt vmcnt(22)\n\t" \
  "s_barrier\n\t" \
  "v_mfma_f32_16x16x128_f8f6f4 a[4:7], v[128:135], v[200:207], a[4:7]\n\t" \
  "ds_read_b128 v[224:227], v3 offset:32768\n\t" \
  "ds_read_b128 v[228:231], v3 offset:40960\n\t" \
  "v_mfma_f32_16x16x128_f8f6f4 a[8:11], v[128:135], v[208:215], a[8:11]\n\t" \
  "ds_read_b128 v[232:235], v3 offset:33792\n\t" \
  "ds_read_b128 v[236:239], v3 offset:41984\n\t" \
  "v_mfma_f32_16x16x128_f8f6f4 a[12:15], v[128:135], v[216:223], a[12:15]\n\t" \
  "ds_read_b128 v[240:243], v3 offset:34816\n\t" \
  "ds_read_b128 v[244:247], v3 offset:43008\n\t" \
  "v_mfma_f32_16x16x128_f8f6f4 a[44:47], v[136:143], v[216:223], a[44:47]\n\t" \
  "ds_read_b128 v[248:251], v3 offset:35840\n\t" \
  "ds_read_b128 v[252:255], v3 offset:44032\n\t" \
  "v_mfma_f32_16x16x128_f8f6f4 a[40:43], v[136:143], v[208:215], a[40:43]\n\t" \
  "s_waitcnt vmcnt(20)\n\t" \
  "v_cmp_le_u32_e32 vcc, s75, v7\n\t" \
  "v_mfma_f32_16x16x128_f8f6f4 a[36:39], v[136:143], v[200:207], a[36:39]\n\t" \
  "s_cmp_eq_u64 vcc, -1\n\t" \
  "s_cbranch_scc0 L_SLOW05_%=\n\t" \
  "L_OK05_%=:\n\t" \
  "s_add_u32 s76, s76, 1024\n\t" \
  "v_mfma_f32_16x16x128_f8f6f4 a[32:35], v[136:143], v[192:199], a[32:35]\n\t" \
  "s_mov_b32 s53, s49\n\t" \
  "s_mov_b32 m0, s48\n\t" \
  "v_mfma_f32_16x16x128_f8f6f4 a[64:67], v[144:151], v[192:199], a[64:67]\n\t" \
  "buffer_load_dwordx4 v0, s[36:39], s53 offen sc0 sc1 lds\n\t" \
  "v_mfma_f32_16x16x128_f8f6f4 a[68:71], v[144:151], v[200:207], a[68:71]\n\t" \
  "buffer_store_dwordx2 v[118:119], v5, s[40:43], s91 offen offset:128 nt\n\t" \
  "v_mfma_f32_16x16x128_f8f6f4 a[72:75], v[144:151], v[208:215], a[72:75]\n\t" \
  "buffer_load_dwordx4 v0, s[36:39], s53 offen offset:1024 sc0 sc1 lds\n\t" \
  "v_mfma_f32_16x16x128_f8f6f4 a[76:79], v[144:151], v[216:223], a[76:79]\n\t" \
  "v_mul_f32_e32 v120, s56, v120\n\t" \
  "v_mul_f32_e32 v121, s56, v121\n\t" \
  "v_mul_f32_e32 v122, s56, v122\n\t" \
  "v_mul_f32_e32 v123, s56, v123\n\t" \
  "v_mfma_f32_16x16x128_f8f6f4 a[108:111], v[152:159], v[216:223], a[108:111]\n\t" \
  "buffer_load_dwordx4 v0, s[36:39], s53 offen offset:2048 sc0 sc1 lds\n\t" \
  "v_mfma_f32_16x16x128_f8f6f4 a[104:107], v[152:159], v[208:215], a[104:107]\n\t" \
  "v_min_f32_e32 v20, s60, v120\n\t" \
  "v_min_f32_e32 v21, s60, v121\n\t" \
  "v_min_f32_e32 v22, s60, v122\n\t" \
  "v_min_f32_e32 v23, s60, v123\n\t" \
  "v_mfma_f32_16x16x128_f8f6f4 a[100:103], v[152:159], v[200:207], a[100:103]\n\t" \
  "buffer_load_dwordx4 v0, s[36:39], s53 offen offset:3072 sc0 sc1 lds\n\t" \
  "v_mfma_f32_16x16x128_f8f6f4 a[96:99], v[152:159], v[192:199], a[96:99]\n\t" \
  "v_exp_f32_e32 v20, v20\n\t" \
  "v_exp_f32_e32 v21, v21\n\t" \
  "s_waitcnt lgkmcnt(0)\n\t" \
  "v_mfma_f32_16x16x128_f8f6f4 a[16:19], v[128:135], v[224:231], a[16:19]\n\t" \
  "v_exp_f32_e32 v22, v22\n\t" \
  "v_exp_f32_e32 v23, v23\n\t" \
  "v_mfma_f32_16x16x128_f8f6f4 a[20:23], v[128:135], v[232:239], a[20:23]\n\t" \
  "ds_read_b128 v[160:163], v1 offset:49152\n\t" \
  "ds_read_b128 v[164:167], v1 offset:57344\n\t" \
  "v_mfma_f32_16x16x128_f8f6f4 a[24:27], v[128:135], v[240:247], a[24:27]\n\t" \
  "ds_read_b128 v[168:171], v1 offset:50176\n\t" \
  "ds_read_b128 v[172:175], v1 offset:58368\n\t" \
  "v_mfma_f32_16x16x128_f8f6f4 a[28:31], v[128:135], v[248:255], a[28:31]\n\t" \
  "ds_read_b128 v[176:179], v1 offset:51200\n\t" \
  "ds_read_b128 v[180:183], v1 offset:59392\n\t" \
  "v_mfma_f32_16x16x128_f8f6f4 a[60:63], v[136:143], v[248:255], a[60:63]\n\t" \
  "ds_read_b128 v[184:187], v1 offset:52224\n\t" \
  "ds_read_b128 v[188:191], v1 offset:60416\n\t" \
  "v_mfma_f32_16x16x128_f8f6f4 a[56:59], v[136:143], v[240:247], a[56:59]\n\t" \
  "v_fma_f32 v20, v20, s58, s58\n\t" \
  "v_fma_f32 v21, v21, s58, s58\n\t" \
  "v_fma_f32 v22, v22, s58, s58\n\t" \
  "v_fma_f32 v23, v23, s58, s58\n\t" \
  "v_mfma_f32_16x16x128_f8f6f4 a[52:55], v[136:143], v[232:239], a[52:55]\n\t" \
  "v_rcp_f32_e32 v20, v20\n\t" \
  "v_rcp_f32_e32 v21, v21\n\t" \
  "v_mfma_f32_16x16x128_f8f6f4 a[48:51], v[136:143], v[224:231], a[48:51]\n\t" \
  "s_mov_b32 s53, s50\n\t" \
  "s_add_u32 m0, s48, 16384\n\t" \
  "v_mfma_f32_16x16x128_f8f6f4 a[80:83], v[144:151], v[224:231], a[80:83]\n\t" \
  "buffer_load_dwordx4 v0, s[36:39], s53 offen sc0 sc1 lds\n\t" \
  "v_mfma_f32_16x16x128_f8f6f4 a[84:87], v[144:151], v[232:239], a[84:87]\n\t" \
  "v_rcp_f32_e32 v22, v22\n\t" \
  "v_rcp_f32_e32 v23, v23\n\t" \
  "v_mfma_f32_16x16x128_f8f6f4 a[88:91], v[144:151], v[240:247], a[88:91]\n\t" \
  "buffer_load_dwordx4 v0, s[36:39], s53 offen offset:1024 sc0 sc1 lds\n\t" \
  "v_mfma_f32_16x16x128_f8f6f4 a[92:95], v[144:151], v[248:255], a[92:95]\n\t" \
  "v_mul_f32_e32 v120, v120, v20\n\t" \
  "v_mul_f32_e32 v121, v121, v21\n\t" \
  "v_mul_f32_e32 v122, v122, v22\n\t" \
  "v_mul_f32_e32 v123, v123, v23\n\t" \
  "v_mfma_f32_16x16x128_f8f6f4 a[124:127], v[152:159], v[248:255], a[124:127]\n\t" \
  "buffer_load_dwordx4 v0, s[36:39], s53 offen offset:2048 sc0 sc1 lds\n\t" \
  "v_mfma_f32_16x16x128_f8f6f4 a[120:123], v[152:159], v[240:247], a[120:123]\n\t" \
  "s_xor_b32 s91, s68, 5373952\n\t" \
  "buffer_store_dwordx2 v[120:121], v5, s[40:43], s91 offen nt\n\t" \
  "v_mfma_f32_16x16x128_f8f6f4 a[116:119], v[152:159], v[232:239], a[116:119]\n\t" \
  "buffer_load_dwordx4 v0, s[36:39], s53 offen offset:3072 sc0 sc1 lds\n\t" \
  "v_mfma_f32_16x16x128_f8f6f4 a[112:115], v[152:159], v[224:231], a[112:115]\n\t" \
  "buffer_store_dwordx2 v[122:123], v5, s[40:43], s91 offen offset:128 nt\n\t" \
  "s_waitcnt lgkmcnt(0)\n\t" \
  "v_mfma_f32_16x16x128_f8f6f4 a[144:147], v[160:167], v[224:231], a[144:147]\n\t" \
  "s_waitcnt vmcnt(22)\n\t" \
  "s_barrier\n\t" \
  "v_mfma_f32_16x16x128_f8f6f4 a[148:151], v[160:167], v[232:239], a[148:151]\n\t" \
  "ds_read_b128 v[128:131], v2 offset:0\n\t" \
  "ds_read_b128 v[132:135], v2 offset:8192\n\t" \
  "v_mfma_f32_16x16x128_f8f6f4 a[152:155], v[160:167], v[240:247], a[152:155]\n\t" \
  "ds_read_b128 v[136:139], v2 offset:1024\n\t" \
  "ds_read_b128 v[140:143], v2 offset:9216\n\t" \
  "v_mfma_f32_16x16x128_f8f6f4 a[156:159], v[160:167], v[248:255], a[156:159]\n\t" \
  "ds_read_b128 v[144:147], v2 offset:2048\n\t" \
  "ds_read_b128 v[148:151], v2 offset:10240\n\t" \
  "v_mfma_f32_16x16x128_f8f6f4 a[188:191], v[168:175], v[248:255], a[188:191]\n\t" \
  "ds_read_b128 v[152:155], v2 offset:3072\n\t" \
  "ds_read_b128 v[156:159], v2 offset:11264\n\t" \
  "v_mfma_f32_16x16x128_f8f6f4 a[184:187], v[168:175], v[240:247], a[184:187]\n\t" \
  "v_mul_f32_e32 v124, s56, v124\n\t" \
  "v_mul_f32_e32 v125, s56, v125\n\t" \
  "v_mul_f32_e32 v126, s56, v126\n\t" \
  "v_mul_f32_e32 v127, s56, v127\n\t" \
  "v_mfma_f32_16x16x128_f8f6f4 a[180:183], v[168:175], v[232:239], a[180:183]\n\t" \
  "v_min_f32_e32 v24, s60, v124\n\t" \
  "v_min_f32_e32 v25, s60, v125\n\t" \
  "v_min_f32_e32 v26, s60, v126\n\t" \
  "v_min_f32_e32 v27, s60, v127\n\t" \
  "v_mfma_f32_16x16x128_f8f6f4 a[176:179], v[168:175], v[224:231], a[176:179]\n\t" \
  "s_mov_b32 s53, s51\n\t" \
  "s_add_u32 m0, s48, 32768\n\t" \
  "v_mfma_f32_16x16x128_f8f6f4 a[208:211], v[176:183], v[224:231], a[208:211]\n\t" \
  "buffer_load_dwordx4 v0, s[36:39], s53 offen sc0 sc1 lds\n\t" \
  "v_mfma_f32_16x16x128_f8f6f4 a[212:215], v[176:183], v[232:239], a[212:215]\n\t" \
  "v_exp_f32_e32 v24, v24\n\t" \
  "v_exp_f32_e32 v25, v25\n\t" \
  "v_mfma_f32_16x16x128_f8f6f4 a[216:219], v[176:183], v[240:247], a[216:219]\n\t" \
  "buffer_load_dwordx4 v0, s[36:39], s53 offen offset:1024 sc0 sc1 lds\n\t" \
  "v_mfma_f32_16x16x128_f8f6f4 a[220:223], v[176:183], v[248:255], a[220:223]\n\t" \
  "v_mfma_f32_16x16x128_f8f6f4 a[252:255], v[184:191], v[248:255], a[252:255]\n\t" \
  "buffer_load_dwordx4 v0, s[36:39], s53 offen offset:2048 sc0 sc1 lds\n\t" \
  "v_mfma_f32_16x16x128_f8f6f4 a[248:251], v[184:191], v[240:247], a[248:251]\n\t" \
  "v_exp_f32_e32 v26, v26\n\t" \
  "v_exp_f32_e32 v27, v27\n\t" \
  "v_mfma_f32_16x16x128_f8f6f4 a[244:247], v[184:191], v[232:239], a[244:247]\n\t" \
  "buffer_load_dwordx4 v0, s[36:39], s53 offen offset:3072 sc0 sc1 lds\n\t" \
  "v_mfma_f32_16x16x128_f8f6f4 a[240:243], v[184:191], v[224:231], a[240:243]\n\t" \
  "v_fma_f32 v24, v24, s58, s58\n\t" \
  "v_fma_f32 v25, v25, s58, s58\n\t" \
  "v_fma_f32 v26, v26, s58, s58\n\t" \
  "v_fma_f32 v27, v27, s58, s58\n\t" \
  "s_waitcnt lgkmcnt(0)\n\t" \
  "v_mfma_f32_16x16x128_f8f6f4 a[128:131], v[160:167], v[192:199], a[128:131]\n\t" \
  "v_rcp_f32_e32 v24, v24\n\t" \
  "v_rcp_f32_e32 v25, v25\n\t" \
  "v_mfma_f32_16x16x128_f8f6f4 a[132:135], v[160:167], v[200:207], a[132:135]\n\t" \
  "ds_read_b128 v[224:227], v4 offset:16384\n\t" \
  "ds_read_b128 v[228:231], v4 offset:24576\n\t" \
  "v_mfma_f32_16x16x128_f8f6f4 a[136:139], v[160:167], v[208:215], a[136:139]\n\t" \
  "ds_read_b128 v[232:235], v4 offset:17408\n\t" \
  "ds_read_b128 v[236:239], v4 offset:25600\n\t" \
  "v_mfma_f32_16x16x128_f8f6f4 a[140:143], v[160:167], v[216:223], a[140:143]\n\t" \
  "ds_read_b128 v[240:243], v4 offset:18432\n\t" \
  "ds_read_b128 v[244:247], v4 offset:26624\n\t" \
  "v_mfma_f32_16x16x128_f8f6f4 a[172:175], v[168:175], v[216:223], a[172:175]\n\t" \
  "ds_read_b128 v[248:251], v4 offset:19456\n\t" \
  "ds_read_b128 v[252:255], v4 offset:27648\n\t" \
  "v_mfma_f32_16x16x128_f8f6f4 a[168:171], v[168:175], v[208:215], a[168:171]\n\t" \
  "v_rcp_f32_e32 v26, v26\n\t" \
  "v_rcp_f32_e32 v27, v27\n\t" \
  "v_mfma_f32_16x16x128_f8f6f4 a[164:167], v[168:175], v[200:207], a[164:167]\n\t" \
  "v_mul_f32_e32 v124, v124, v24\n\t" \
  "v_mul_f32_e32 v125, v125, v25\n\t" \
  "v_mul_f32_e32 v126, v126, v26\n\t" \
  "v_mul_f32_e32 v127, v127, v27\n\t" \
  "v_mfma_f32_16x16x128_f8f6f4 a[160:163], v[168:175], v[192:199], a[160:163]\n\t" \
  "s_add_u32 s53, s49, 16384\n\t" \
  "s_add_u32 m0, s48, 49152\n\t" \
  "v_mfma_f32_16x16x128_f8f6f4 a[192:195], v[176:183], v[192:199], a[192:195]\n\t" \
  "buffer_load_dwordx4 v0, s[36:39], s53 offen sc0 sc1 lds\n\t" \
  "v_mfma_f32_16x16x128_f8f6f4 a[196:199], v[176:183], v[200:207], a[196:199]\n\t" \
  "s_xor_b32 s91, s68, 5439488\n\t" \
  "buffer_store_dwordx2 v[124:125], v5, s[40:43], s91 offen nt\n\t" \
  "v_mfma_f32_16x16x128_f8f6f4 a[200:203], v[176:183], v[208:215], a[200:203]\n\t" \
  "buffer_load_dwordx4 v0, s[36:39], s53 offen offset:1024 sc0 sc1 lds\n\t" \
  "v_mfma_f32_16x16x128_f8f6f4 a[204:207], v[176:183], v[216:223], a[204:207]\n\t" \
  "buffer_store_dwordx2 v[126:127], v5, s[40:43], s91 offen offset:128 nt\n\t" \
  "v_mfma_f32_16x16x128_f8f6f4 a[236:239], v[184:191], v[216:223], a[236:239]\n\t" \
  "buffer_load_dwordx4 v0, s[36:39], s53 offen offset:2048 sc0 sc1 lds\n\t" \
  "v_mfma_f32_16x16x128_f8f6f4 a[232:235], v[184:191], v[208:215], a[232:235]\n\t" \
  "ds_read_b128 v[32:35], v6 offset:0\n\t" \
  "ds_read_b128 v[36:39], v6 offset:1024\n\t" \
  "ds_read_b128 v[40:43], v6 offset:2048\n\t" \
  "ds_read_b128 v[44:47], v6 offset:3072\n\t" \
  "v_mfma_f32_16x16x128_f8f6f4 a[228:231], v[184:191], v[200:207], a[228:231]\n\t" \
  "buffer_load_dwordx4 v0, s[36:39], s53 offen offset:3072 sc0 sc1 lds\n\t" \
  "v_mfma_f32_16x16x128_f8f6f4 a[224:227], v[184:191], v[192:199], a[224:227]\n\t" \
  "s_waitcnt lgkmcnt(0)\n\t" \
  "v_mfma_f32_16x16x128_f8f6f4 a[16:19], v[128:135], v[224:231], a[16:19]\n\t" \
  "s_waitcnt vmcnt(21)\n\t" \
  "s_barrier\n\t" \
  "v_mfma_f32_16x16x128_f8f6f4 a[20:23], v[128:135], v[232:239], a[20:23]\n\t" \
  "ds_read_b128 v[192:195], v4 offset:32768\n\t" \
  "ds_read_b128 v[196:199], v4 offset:40960\n\t" \
  "v_mfma_f32_16x16x128_f8f6f4 a[24:27], v[128:135], v[240:247], a[24:27]\n\t" \
  "ds_read_b128 v[200:203], v4 offset:33792\n\t" \
  "ds_read_b128 v[204:207], v4 offset:41984\n\t" \
  "v_mfma_f32_16x16x128_f8f6f4 a[28:31], v[128:135], v[248:255], a[28:31]\n\t" \
  "ds_read_b128 v[208:211], v4 offset:34816\n\t" \
  "ds_read_b128 v[212:215], v4 offset:43008\n\t" \
  "v_mfma_f32_16x16x128_f8f6f4 a[60:63], v[136:143], v[248:255], a[60:63]\n\t" \
  "ds_read_b128 v[216:219], v4 offset:35840\n\t" \
  "ds_read_b128 v[220:223], v4 offset:44032\n\t" \
  "v_mfma_f32_16x16x128_f8f6f4 a[56:59], v[136:143], v[240:247], a[56:59]\n\t" \
  "v_mul_f32_e32 v48, s56, v32\n\t" \
  "v_mul_f32_e32 v49, s56, v36\n\t" \
  "v_mul_f32_e32 v50, s56, v40\n\t" \
  "v_mul_f32_e32 v51, s56, v44\n\t" \
  "v_mfma_f32_16x16x128_f8f6f4 a[52:55], v[136:143], v[232:239], a[52:55]\n\t" \
  "v_mul_f32_e32 v52, s56, v33\n\t" \
  "v_mul_f32_e32 v53, s56, v37\n\t" \
  "v_mul_f32_e32 v54, s56, v41\n\t" \
  "v_mul_f32_e32 v55, s56, v45\n\t" \
  "v_mfma_f32_16x16x128_f8f6f4 a[48:51], v[136:143], v[224:231], a[48:51]\n\t" \
  "s_add_u32 s53, s49, 32768\n\t" \
  "s_add_u32 m0, s48, 65536\n\t" \
  "v_mfma_f32_16x16x128_f8f6f4 a[80:83], v[144:151], v[224:231], a[80:83]\n\t" \
  "buffer_load_dwordx4 v0, s[36:39], s53 offen sc0 sc1 lds\n\t" \
  "v_mfma_f32_16x16x128_f8f6f4 a[84:87], v[144:151], v[232:239], a[84:87]\n\t" \
  "v_mul_f32_e32 v56, s56, v34\n\t" \
  "v_mul_f32_e32 v57, s56, v38\n\t" \
  "v_mul_f32_e32 v58, s56, v42\n\t" \
  "v_mul_f32_e32 v59, s56, v46\n\t" \
  "v_mfma_f32_16x16x128_f8f6f4 a[88:91], v[144:151], v[240:247], a[88:91]\n\t" \
  "buffer_load_dwordx4 v0, s[36:39], s53 offen offset:1024 sc0 sc1 lds\n\t" \
  "v_mfma_f32_16x16x128_f8f6f4 a[92:95], v[144:151], v[248:255], a[92:95]\n\t" \
  "v_mul_f32_e32 v60, s56, v35\n\t" \
  "v_mul_f32_e32 v61, s56, v39\n\t" \
  "v_mul_f32_e32 v62, s56, v43\n\t" \
  "v_mul_f32_e32 v63, s56, v47\n\t" \
  "v_mfma_f32_16x16x128_f8f6f4 a[124:127], v[152:159], v[248:255], a[124:127]\n\t" \
  "buffer_load_dwordx4 v0, s[36:39], s53 offen offset:2048 sc0 sc1 lds\n\t" \
  "v_mfma_f32_16x16x128_f8f6f4 a[120:123], v[152:159], v[240:247], a[120:123]\n\t" \
  "v_min_f32_e32 v28, s60, v48\n\t" \
  "v_min_f32_e32 v29, s60, v49\n\t" \
  "v_min_f32_e32 v30, s60, v50\n\t" \
  "v_min_f32_e32 v31, s60, v51\n\t" \
  "v_mfma_f32_16x16x128_f8f6f4 a[116:119], v[152:159], v[232:239], a[116:119]\n\t" \
  "buffer_load_dwordx4 v0, s[36:39], s53 offen offset:3072 sc0 sc1 lds\n\t" \
  "v_mfma_f32_16x16x128_f8f6f4 a[112:115], v[152:159], v[224:231], a[112:115]\n\t" \
  "v_exp_f32_e32 v28, v28\n\t" \
  "v_exp_f32_e32 v29, v29\n\t" \
  "s_waitcnt lgkmcnt(0)\n\t" \
  "v_mfma_f32_16x16x128_f8f6f4 a[0:3], v[128:135], v[192:199], a[0:3]\n\t" \
  "v_exp_f32_e32 v30, v30\n\t" \
  "v_exp_f32_e32 v31, v31\n\t" \
  "v_mfma_f32_16x16x128_f8f6f4 a[4:7], v[128:135], v[200:207], a[4:7]\n\t" \
  "ds_read_b128 v[160:163], v2 offset:49152\n\t" \
  "ds_read_b128 v[164:167], v2 offset:57344\n\t" \
  "v_mfma_f32_16x16x128_f8f6f4 a[8:11], v[128:135], v[208:215], a[8:11]\n\t" \
  "ds_read_b128 v[168:171], v2 offset:50176\n\t" \
  "ds_read_b128 v[172:175], v2 offset:58368\n\t" \
  "v_mfma_f32_16x16x128_f8f6f4 a[12:15], v[128:135], v[216:223], a[12:15]\n\t" \
  "ds_read_b128 v[176:179], v2 offset:51200\n\t" \
  "ds_read_b128 v[180:183], v2 offset:59392\n\t" \
  "v_mfma_f32_16x16x128_f8f6f4 a[44:47], v[136:143], v[216:223], a[44:47]\n\t" \
  "ds_read_b128 v[184:187], v2 offset:52224\n\t" \
  "ds_read_b128 v[188:191], v2 offset:60416\n\t" \
  "v_mfma_f32_16x16x128_f8f6f4 a[40:43], v[136:143], v[208:215], a[40:43]\n\t" \
  "v_fma_f32 v28, v28, s58, s58\n\t" \
  "v_fma_f32 v29, v29, s58, s58\n\t" \
  "v_fma_f32 v30, v30, s58, s58\n\t" \
  "v_fma_f32 v31, v31, s58, s58\n\t" \
  "v_mfma_f32_16x16x128_f8f6f4 a[36:39], v[136:143], v[200:207], a[36:39]\n\t" \
  "v_rcp_f32_e32 v28, v28\n\t" \
  "v_rcp_f32_e32 v29, v29\n\t" \
  "v_mfma_f32_16x16x128_f8f6f4 a[32:35], v[136:143], v[192:199], a[32:35]\n\t" \
  "s_add_u32 s53, s51, 16384\n\t" \
  "s_add_u32 m0, s48, 81920\n\t" \
  "v_mfma_f32_16x16x128_f8f6f4 a[64:67], v[144:151], v[192:199], a[64:67]\n\t" \
  "buffer_load_dwordx4 v0, s[36:39], s53 offen sc0 sc1 lds\n\t" \
  "v_mfma_f32_16x16x128_f8f6f4 a[68:71], v[144:151], v[200:207], a[68:71]\n\t" \
  "v_rcp_f32_e32 v30, v30\n\t" \
  "v_rcp_f32_e32 v31, v31\n\t" \
  "v_mfma_f32_16x16x128_f8f6f4 a[72:75], v[144:151], v[208:215], a[72:75]\n\t" \
  "buffer_load_dwordx4 v0, s[36:39], s53 offen offset:1024 sc0 sc1 lds\n\t" \
  "v_mfma_f32_16x16x128_f8f6f4 a[76:79], v[144:151], v[216:223], a[76:79]\n\t" \
  "v_mul_f32_e32 v48, v48, v28\n\t" \
  "v_mul_f32_e32 v49, v49, v29\n\t" \
  "v_mul_f32_e32 v50, v50, v30\n\t" \
  "v_mul_f32_e32 v51, v51, v31\n\t" \
  "v_mfma_f32_16x16x128_f8f6f4 a[108:111], v[152:159], v[216:223], a[108:111]\n\t" \
  "buffer_load_dwordx4 v0, s[36:39], s53 offen offset:2048 sc0 sc1 lds\n\t" \
  "v_mfma_f32_16x16x128_f8f6f4 a[104:107], v[152:159], v[208:215], a[104:107]\n\t" \
  "s_xor_b32 s91, s68, 6291456\n\t" \
  "buffer_store_dwordx2 v[48:49], v5, s[40:43], s91 offen nt\n\t" \
  "v_mfma_f32_16x16x128_f8f6f4 a[100:103], v[152:159], v[200:207], a[100:103]\n\t" \
  "buffer_load_dwordx4 v0, s[36:39], s53 offen offset:3072 sc0 sc1 lds\n\t" \
  "v_mfma_f32_16x16x128_f8f6f4 a[96:99], v[152:159], v[192:199], a[96:99]\n\t" \
  "buffer_store_dwordx2 v[50:51], v5, s[40:43], s91 offen offset:128 nt\n\t" \
  "s_waitcnt lgkmcnt(0)\n\t" \
  "v_mfma_f32_16x16x128_f8f6f4 a[128:131], v[160:167], v[192:199], a[128:131]\n\t" \
  "s_waitcnt vmcnt(21)\n\t" \
  "s_barrier\n\t" \
  "v_mfma_f32_16x16x128_f8f6f4 a[132:135], v[160:167], v[200:207], a[132:135]\n\t" \
  "ds_read_b128 v[128:131], v1 offset:0\n\t" \
  "ds_read_b128 v[132:135], v1 offset:8192\n\t" \
  "v_mfma_f32_16x16x128_f8f6f4 a[136:139], v[160:167], v[208:215], a[136:139]\n\t" \
  "ds_read_b128 v[136:139], v1 offset:1024\n\t" \
  "ds_read_b128 v[140:143], v1 offset:9216\n\t" \
  "v_mfma_f32_16x16x128_f8f6f4 a[140:143], v[160:167], v[216:223], a[140:143]\n\t" \
  "ds_read_b128 v[144:147], v1 offset:2048\n\t" \
  "ds_read_b128 v[148:151], v1 offset:10240\n\t" \
  "v_mfma_f32_16x16x128_f8f6f4 a[172:175], v[168:175], v[216:223], a[172:175]\n\t" \
  "ds_read_b128 v[152:155], v1 offset:3072\n\t" \
  "ds_read_b128 v[156:159], v1 offset:11264\n\t" \
  "v_mfma_f32_16x16x128_f8f6f4 a[168:171], v[168:175], v[208:215], a[168:171]\n\t" \
  "v_min_f32_e32 v12, s60, v52\n\t" \
  "v_min_f32_e32 v13, s60, v53\n\t" \
  "v_min_f32_e32 v14, s60, v54\n\t" \
  "v_min_f32_e32 v15, s60, v55\n\t" \
  "v_mfma_f32_16x16x128_f8f6f4 a[164:167], v[168:175], v[200:207], a[164:167]\n\t" \
  "v_exp_f32_e32 v12, v12\n\t" \
  "v_exp_f32_e32 v13, v13\n\t" \
  "v_mfma_f32_16x16x128_f8f6f4 a[160:163], v[168:175], v[192:199], a[160:163]\n\t" \
  "s_add_u32 s53, s50, 16384\n\t" \
  "s_add_u32 m0, s48, 98304\n\t" \
  "v_mfma_f32_16x16x128_f8f6f4 a[192:195], v[176:183], v[192:199], a[192:195]\n\t" \
  "buffer_load_dwordx4 v0, s[36:39], s53 offen sc0 sc1 lds\n\t" \
  "v_mfma_f32_16x16x128_f8f6f4 a[196:199], v[176:183], v[200:207], a[196:199]\n\t" \
  "v_exp_f32_e32 v14, v14\n\t" \
  "v_exp_f32_e32 v15, v15\n\t" \
  "v_mfma_f32_16x16x128_f8f6f4 a[200:203], v[176:183], v[208:215], a[200:203]\n\t" \
  "buffer_load_dwordx4 v0, s[36:39], s53 offen offset:1024 sc0 sc1 lds\n\t" \
  "v_mfma_f32_16x16x128_f8f6f4 a[204:207], v[176:183], v[216:223], a[204:207]\n\t" \
  "v_fma_f32 v12, v12, s58, s58\n\t" \
  "v_fma_f32 v13, v13, s58, s58\n\t" \
  "v_fma_f32 v14, v14, s58, s58\n\t" \
  "v_fma_f32 v15, v15, s58, s58\n\t" \
  "v_mfma_f32_16x16x128_f8f6f4 a[236:239], v[184:191], v[216:223], a[236:239]\n\t" \
  "buffer_load_dwordx4 v0, s[36:39], s53 offen offset:2048 sc0 sc1 lds\n\t" \
  "v_mfma_f32_16x16x128_f8f6f4 a[232:235], v[184:191], v[208:215], a[232:235]\n\t" \
  "v_rcp_f32_e32 v12, v12\n\t" \
  "v_rcp_f32_e32 v13, v13\n\t" \
  "v_mfma_f32_16x16x128_f8f6f4 a[228:231], v[184:191], v[200:207], a[228:231]\n\t" \
  "buffer_load_dwordx4 v0, s[36:39], s53 offen offset:3072 sc0 sc1 lds\n\t" \
  "v_mfma_f32_16x16x128_f8f6f4 a[224:227], v[184:191], v[192:199], a[224:227]\n\t" \
  "v_rcp_f32_e32 v14, v14\n\t" \
  "v_rcp_f32_e32 v15, v15\n\t" \
  "s_waitcnt lgkmcnt(0)\n\t" \
  "v_mfma_f32_16x16x128_f8f6f4 a[144:147], v[160:167], v[224:231], a[144:147]\n\t" \
  "v_mul_f32_e32 v52, v52, v12\n\t" \
  "v_mul_f32_e32 v53, v53, v13\n\t" \
  "v_mul_f32_e32 v54, v54, v14\n\t" \
  "v_mul_f32_e32 v55, v55, v15\n\t" \
  "v_mfma_f32_16x16x128_f8f6f4 a[148:151], v[160:167], v[232:239], a[148:151]\n\t" \
  "ds_read_b128 v[192:195], v3 offset:16384\n\t" \
  "ds_read_b128 v[196:199], v3 offset:24576\n\t" \
  "v_mfma_f32_16x16x128_f8f6f4 a[152:155], v[160:167], v[240:247], a[152:155]\n\t" \
  "ds_read_b128 v[200:203], v3 offset:17408\n\t" \
  "ds_read_b128 v[204:207], v3 offset:25600\n\t" \
  "v_mfma_f32_16x16x128_f8f6f4 a[156:159], v[160:167], v[248:255], a[156:159]\n\t" \
  "ds_read_b128 v[208:211], v3 offset:18432\n\t" \
  "ds_read_b128 v[212:215], v3 offset:26624\n\t" \
  "v_mfma_f32_16x16x128_f8f6f4 a[188:191], v[168:175], v[248:255], a[188:191]\n\t" \
  "ds_read_b128 v[216:219], v3 offset:19456\n\t" \
  "ds_read_b128 v[220:223], v3 offset:27648\n\t" \
  "v_mfma_f32_16x16x128_f8f6f4 a[184:187], v[168:175], v[240:247], a[184:187]\n\t" \
  "s_xor_b32 s91, s68, 6356992\n\t" \
  "buffer_store_dwordx2 v[52:53], v5, s[40:43], s91 offen nt\n\t" \
  "v_mfma_f32_16x16x128_f8f6f4 a[180:183], v[168:175], v[232:239], a[180:183]\n\t" \
  "buffer_store_dwordx2 v[54:55], v5, s[40:43], s91 offen offset:128 nt\n\t" \
  "v_mfma_f32_16x16x128_f8f6f4 a[176:179], v[168:175], v[224:231], a[176:179]\n\t" \
  "s_add_u32 s53, s49, 49152\n\t" \
  "s_add_u32 m0, s48, 114688\n\t" \
  "v_mfma_f32_16x16x128_f8f6f4 a[208:211], v[176:183], v[224:231], a[208:211]\n\t" \
  "buffer_load_dwordx4 v0, s[36:39], s53 offen sc0 sc1 lds\n\t" \
  "v_mfma_f32_16x16x128_f8f6f4 a[212:215], v[176:183], v[232:239], a[212:215]\n\t" \
  "v_min_f32_e32 v16, s60, v56\n\t" \
  "v_min_f32_e32 v17, s60, v57\n\t" \
  "v_min_f32_e32 v18, s60, v58\n\t" \
  "v_min_f32_e32 v19, s60, v59\n\t" \
  "v_mfma_f32_16x16x128_f8f6f4 a[216:219], v[176:183], v[240:247], a[216:219]\n\t" \
  "buffer_load_dwordx4 v0, s[36:39], s53 offen offset:1024 sc0 sc1 lds\n\t" \
  "v_mfma_f32_16x16x128_f8f6f4 a[220:223], v[176:183], v[248:255], a[220:223]\n\t" \
  "v_exp_f32_e32 v16, v16\n\t" \
  "v_exp_f32_e32 v17, v17\n\t" \
  "v_mfma_f32_16x16x128_f8f6f4 a[252:255], v[184:191], v[248:255], a[252:255]\n\t" \
  "buffer_load_dwordx4 v0, s[36:39], s53 offen offset:2048 sc0 sc1 lds\n\t" \
  "v_mfma_f32_16x16x128_f8f6f4 a[248:251], v[184:191], v[240:247], a[248:251]\n\t" \
  "v_exp_f32_e32 v18, v18\n\t" \
  "v_exp_f32_e32 v19, v19\n\t" \
  "v_mfma_f32_16x16x128_f8f6f4 a[244:247], v[184:191], v[232:239], a[244:247]\n\t" \
  "buffer_load_dwordx4 v0, s[36:39], s53 offen offset:3072 sc0 sc1 lds\n\t" \
  "v_mfma_f32_16x16x128_f8f6f4 a[240:243], v[184:191], v[224:231], a[240:243]\n\t" \
  "s_add_u32 s49, s49, 65536\n\t" \
  "s_add_u32 s50, s50, 32768\n\t" \
  "s_add_u32 s51, s51, 32768\n\t" \
  "s_waitcnt lgkmcnt(0)\n\t" \
  "v_mfma_f32_16x16x128_f8f6f4 a[0:3], v[128:135], v[192:199], a[0:3]\n\t" \
  "s_waitcnt vmcnt(20)\n\t" \
  "s_barrier\n\t" \
  "v_mfma_f32_16x16x128_f8f6f4 a[4:7], v[128:135], v[200:207], a[4:7]\n\t" \
  "ds_read_b128 v[224:227], v3 offset:32768\n\t" \
  "ds_read_b128 v[228:231], v3 offset:40960\n\t" \
  "v_mfma_f32_16x16x128_f8f6f4 a[8:11], v[128:135], v[208:215], a[8:11]\n\t" \
  "ds_read_b128 v[232:235], v3 offset:33792\n\t" \
  "ds_read_b128 v[236:239], v3 offset:41984\n\t" \
  "v_mfma_f32_16x16x128_f8f6f4 a[12:15], v[128:135], v[216:223], a[12:15]\n\t" \
  "ds_read_b128 v[240:243], v3 offset:34816\n\t" \
  "ds_read_b128 v[244:247], v3 offset:43008\n\t" \
  "v_and_b32_e32 v8, 0x3ff, v8\n\t" \
  "v_add_u32_e32 v8, s92, v8\n\t" \
  "v_mfma_f32_16x16x128_f8f6f4 a[44:47], v[136:143], v[216:223], a[44:47]\n\t" \
  "ds_read_b128 v[248:251], v3 offset:35840\n\t" \
  "ds_read_b128 v[252:255], v3 offset:44032\n\t" \
  "v_mfma_f32_16x16x128_f8f6f4 a[40:43], v[136:143], v[208:215], a[40:43]\n\t" \
  "v_fma_f32 v16, v16, s58, s58\n\t" \
  "v_fma_f32 v17, v17, s58, s58\n\t" \
  "v_fma_f32 v18, v18, s58, s58\n\t" \
  "v_fma_f32 v19, v19, s58, s58\n\t" \
  "v_mfma_f32_16x16x128_f8f6f4 a[36:39], v[136:143], v[200:207], a[36:39]\n\t" \
  "v_rcp_f32_e32 v16, v16\n\t" \
  "v_rcp_f32_e32 v17, v17\n\t" \
  "v_mfma_f32_16x16x128_f8f6f4 a[32:35], v[136:143], v[192:199], a[32:35]\n\t" \
  "s_mov_b32 s53, s49\n\t" \
  "s_mov_b32 m0, s48\n\t" \
  "v_mfma_f32_16x16x128_f8f6f4 a[64:67], v[144:151], v[192:199], a[64:67]\n\t" \
  "buffer_load_dwordx4 v0, s[36:39], s53 offen sc0 sc1 lds\n\t" \
  "v_mfma_f32_16x16x128_f8f6f4 a[68:71], v[144:151], v[200:207], a[68:71]\n\t" \
  "v_rcp_f32_e32 v18, v18\n\t" \
  "v_rcp_f32_e32 v19, v19\n\t" \
  "v_mfma_f32_16x16x128_f8f6f4 a[72:75], v[144:151], v[208:215], a[72:75]\n\t" \
  "buffer_load_dwordx4 v0, s[36:39], s53 offen offset:1024 sc0 sc1 lds\n\t" \
  "v_mfma_f32_16x16x128_f8f6f4 a[76:79], v[144:151], v[216:223], a[76:79]\n\t" \
  "v_mul_f32_e32 v56, v56, v16\n\t" \
  "v_mul_f32_e32 v57, v57, v17\n\t" \
  "v_mul_f32_e32 v58, v58, v18\n\t" \
  "v_mul_f32_e32 v59, v59, v19\n\t" \
  "v_mfma_f32_16x16x128_f8f6f4 a[108:111], v[152:159], v[216:223], a[108:111]\n\t" \
  "buffer_load_dwordx4 v0, s[36:39], s53 offen offset:2048 sc0 sc1 lds\n\t" \
  "v_mfma_f32_16x16x128_f8f6f4 a[104:107], v[152:159], v[208:215], a[104:107]\n\t" \
  "s_xor_b32 s91, s68, 6422528\n\t" \
  "buffer_store_dwordx2 v[56:57], v5, s[40:43], s91 offen nt\n\t" \
  "v_mfma_f32_16x16x128_f8f6f4 a[100:103], v[152:159], v[200:207], a[100:103]\n\t" \
  "buffer_load_dwordx4 v0, s[36:39], s53 offen offset:3072 sc0 sc1 lds\n\t" \
  "v_mfma_f32_16x16x128_f8f6f4 a[96:99], v[152:159], v[192:199], a[96:99]\n\t" \
  "buffer_store_dwordx2 v[58:59], v5, s[40:43], s91 offen offset:128 nt\n\t" \
  "s_waitcnt lgkmcnt(0)\n\t" \
  "v_mfma_f32_16x16x128_f8f6f4 a[16:19], v[128:135], v[224:231], a[16:19]\n\t" \
  "v_min_f32_e32 v20, s60, v60\n\t" \
  "v_min_f32_e32 v21, s60, v61\n\t" \
  "v_min_f32_e32 v22, s60, v62\n\t" \
  "v_min_f32_e32 v23, s60, v63\n\t" \
  "v_mfma_f32_16x16x128_f8f6f4 a[20:23], v[128:135], v[232:239], a[20:23]\n\t" \
  "ds_read_b128 v[160:163], v1 offset:49152\n\t" \
  "ds_read_b128 v[164:167], v1 offset:57344\n\t" \
  "v_mfma_f32_16x16x128_f8f6f4 a[24:27], v[128:135], v[240:247], a[24:27]\n\t" \
  "ds_read_b128 v[168:171], v1 offset:50176\n\t" \
  "ds_read_b128 v[172:175], v1 offset:58368\n\t" \
  "v_mfma_f32_16x16x128_f8f6f4 a[28:31], v[128:135], v[248:255], a[28:31]\n\t" \
  "ds_read_b128 v[176:179], v1 offset:51200\n\t" \
  "ds_read_b128 v[180:183], v1 offset:59392\n\t" \
  "v_mfma_f32_16x16x128_f8f6f4 a[60:63], v[136:143], v[248:255], a[60:63]\n\t" \
  "ds_read_b128 v[184:187], v1 offset:52224\n\t" \
  "ds_read_b128 v[188:191], v1 offset:60416\n\t" \
  "v_mfma_f32_16x16x128_f8f6f4 a[56:59], v[136:143], v[240:247], a[56:59]\n\t" \
  "v_exp_f32_e32 v20, v20\n\t" \
  "v_exp_f32_e32 v21, v21\n\t" \
  "v_mfma_f32_16x16x128_f8f6f4 a[52:55], v[136:143], v[232:239], a[52:55]\n\t" \
  "v_exp_f32_e32 v22, v22\n\t" \
  "v_exp_f32_e32 v23, v23\n\t" \
  "v_mfma_f32_16x16x128_f8f6f4 a[48:51], v[136:143], v[224:231], a[48:51]\n\t" \
  "s_mov_b32 s53, s50\n\t" \
  "s_add_u32 m0, s48, 16384\n\t" \
  "v_mfma_f32_16x16x128_f8f6f4 a[80:83], v[144:151], v[224:231], a[80:83]\n\t" \
  "buffer_load_dwordx4 v0, s[36:39], s53 offen sc0 sc1 lds\n\t" \
  "v_mfma_f32_16x16x128_f8f6f4 a[84:87], v[144:151], v[232:239], a[84:87]\n\t" \
  "v_mfma_f32_16x16x128_f8f6f4 a[88:91], v[144:151], v[240:247], a[88:91]\n\t" \
  "buffer_load_dwordx4 v0, s[36:39], s53 offen offset:1024 sc0 sc1 lds\n\t" \
  "v_mfma_f32_16x16x128_f8f6f4 a[92:95], v[144:151], v[248:255], a[92:95]\n\t" \
  "v_fma_f32 v20, v20, s58, s58\n\t" \
  "v_fma_f32 v21, v21, s58, s58\n\t" \
  "v_fma_f32 v22, v22, s58, s58\n\t" \
  "v_fma_f32 v23, v23, s58, s58\n\t" \
  "v_mfma_f32_16x16x128_f8f6f4 a[124:127], v[152:159], v[248:255], a[124:127]\n\t" \
  "buffer_load_dwordx4 v0, s[36:39], s53 offen offset:2048 sc0 sc1 lds\n\t" \
  "v_mfma_f32_16x16x128_f8f6f4 a[120:123], v[152:159], v[240:247], a[120:123]\n\t" \
  "v_rcp_f32_e32 v20, v20\n\t" \
  "v_rcp_f32_e32 v21, v21\n\t" \
  "v_mfma_f32_16x16x128_f8f6f4 a[116:119], v[152:159], v[232:239], a[116:119]\n\t" \
  "buffer_load_dwordx4 v0, s[36:39], s53 offen offset:3072 sc0 sc1 lds\n\t" \
  "v_mfma_f32_16x16x128_f8f6f4 a[112:115], v[152:159], v[224:231], a[112:115]\n\t" \
  "v_rcp_f32_e32 v22, v22\n\t" \
  "v_rcp_f32_e32 v23, v23\n\t" \
  "s_waitcnt lgkmcnt(0)\n\t" \
  "v_mfma_f32_16x16x128_f8f6f4 a[144:147], v[160:167], v[224:231], a[144:147]\n\t" \
  "s_waitcnt vmcnt(21)\n\t" \
  "s_barrier\n\t" \
  "v_mfma_f32_16x16x128_f8f6f4 a[148:151], v[160:167], v[232:239], a[148:151]\n\t" \
  "ds_read_b128 v[128:131], v2 offset:0\n\t" \
  "ds_read_b128 v[132:135], v2 offset:8192\n\t" \
  "v_mfma_f32_16x16x128_f8f6f4 a[152:155], v[160:167], v[240:247], a[152:155]\n\t" \
  "ds_read_b128 v[136:139], v2 offset:1024\n\t" \
  "ds_read_b128 v[140:143], v2 offset:9216\n\t" \
  "v_mfma_f32_16x16x128_f8f6f4 a[156:159], v[160:167], v[248:255], a[156:159]\n\t" \
  "ds_read_b128 v[144:147], v2 offset:2048\n\t" \
  "ds_read_b128 v[148:151], v2 offset:10240\n\t" \
  "v_mfma_f32_16x16x128_f8f6f4 a[188:191], v[168:175], v[248:255], a[188:191]\n\t" \
  "ds_read_b128 v[152:155], v2 offset:3072\n\t" \
  "ds_read_b128 v[156:159], v2 offset:11264\n\t" \
  "v_mfma_f32_16x16x128_f8f6f4 a[184:187], v[168:175], v[240:247], a[184:187]\n\t" \
  "v_mul_f32_e32 v60, v60, v20\n\t" \
  "v_mul_f32_e32 v61, v61, v21\n\t" \
  "v_mul_f32_e32 v62, v62, v22\n\t" \
  "v_mul_f32_e32 v63, v63, v23\n\t" \
  "v_mfma_f32_16x16x128_f8f6f4 a[180:183], v[168:175], v[232:239], a[180:183]\n\t" \
  "s_xor_b32 s91, s68, 6488064\n\t" \
  "buffer_store_dwordx2 v[60:61], v5, s[40:43], s91 offen nt\n\t" \
  "v_mfma_f32_16x16x128_f8f6f4 a[176:179], v[168:175], v[224:231], a[176:179]\n\t" \
  "s_mov_b32 s53, s51\n\t" \
  "s_add_u32 m0, s48, 32768\n\t" \
  "v_mfma_f32_16x16x128_f8f6f4 a[208:211], v[176:183], v[224:231], a[208:211]\n\t" \
  "buffer_load_dwordx4 v0, s[36:39], s53 offen sc0 sc1 lds\n\t" \
  "v_mfma_f32_16x16x128_f8f6f4 a[212:215], v[176:183], v[232:239], a[212:215]\n\t" \
  "buffer_store_dwordx2 v[62:63], v5, s[40:43], s91 offen offset:128 nt\n\t" \
  "v_mfma_f32_16x16x128_f8f6f4 a[216:219], v[176:183], v[240:247], a[216:219]\n\t" \
  "buffer_load_dwordx4 v0, s[36:39], s53 offen offset:1024 sc0 sc1 lds\n\t" \
  "v_mfma_f32_16x16x128_f8f6f4 a[220:223], v[176:183], v[248:255], a[220:223]\n\t" \
  "ds_read_b128 v[64:67], v6 offset:4096\n\t" \
  "ds_read_b128 v[68:71], v6 offset:5120\n\t" \
  "ds_read_b128 v[72:75], v6 offset:6144\n\t" \
  "ds_read_b128 v[76:79], v6 offset:7168\n\t" \
  "v_mfma_f32_16x16x128_f8f6f4 a[252:255], v[184:191], v[248:255], a[252:255]\n\t" \
  "buffer_load_dwordx4 v0, s[36:39], s53 offen offset:2048 sc0 sc1 lds\n\t" \
  "v_mfma_f32_16x16x128_f8f6f4 a[248:251], v[184:191], v[240:247], a[248:251]\n\t" \
  "v_mfma_f32_16x16x128_f8f6f4 a[244:247], v[184:191], v[232:239], a[244:247]\n\t" \
  "buffer_load_dwordx4 v0, s[36:39], s53 offen offset:3072 sc0 sc1 lds\n\t" \
  "v_mfma_f32_16x16x128_f8f6f4 a[240:243], v[184:191], v[224:231], a[240:243]\n\t" \
  "s_waitcnt lgkmcnt(0)\n\t" \
  "v_mfma_f32_16x16x128_f8f6f4 a[128:131], v[160:167], v[192:199], a[128:131]\n\t" \
  "v_mul_f32_e32 v80, s56, v64\n\t" \
  "v_mul_f32_e32 v81, s56, v68\n\t" \
  "v_mul_f32_e32 v82, s56, v72\n\t" \
  "v_mul_f32_e32 v83, s56, v76\n\t" \
  "v_mfma_f32_16x16x128_f8f6f4 a[132:135], v[160:167], v[200:207], a[132:135]\n\t" \
  "ds_read_b128 v[224:227], v4 offset:16384\n\t" \
  "ds_read_b128 v[228:231], v4 offset:24576\n\t" \
  "v_mfma_f32_16x16x128_f8f6f4 a[136:139], v[160:167], v[208:215], a[136:139]\n\t" \
  "ds_read_b128 v[232:235], v4 offset:17408\n\t" \
  "ds_read_b128 v[236:239], v4 offset:25600\n\t" \
  "v_mfma_f32_16x16x128_f8f6f4 a[140:143], v[160:167], v[216:223], a[140:143]\n\t" \
  "ds_read_b128 v[240:243], v4 offset:18432\n\t" \
  "ds_read_b128 v[244:247], v4 offset:26624\n\t" \
  "v_mfma_f32_16x16x128_f8f6f4 a[172:175], v[168:175], v[216:223], a[172:175]\n\t" \
  "ds_read_b128 v[248:251], v4 offset:19456\n\t" \
  "ds_read_b128 v[252:255], v4 offset:27648\n\t" \
  "v_mfma_f32_16x16x128_f8f6f4 a[168:171], v[168:175], v[208:215], a[168:171]\n\t" \
  "v_mul_f32_e32 v84, s56, v65\n\t" \
  "v_mul_f32_e32 v85, s56, v69\n\t" \
  "v_mul_f32_e32 v86, s56, v73\n\t" \
  "v_mul_f32_e32 v87, s56, v77\n\t" \
  "v_mfma_f32_16x16x128_f8f6f4 a[164:167], v[168:175], v[200:207], a[164:167]\n\t" \
  "v_mul_f32_e32 v88, s56, v66\n\t" \
  "v_mul_f32_e32 v89, s56, v70\n\t" \
  "v_mul_f32_e32 v90, s56, v74\n\t" \
  "v_mul_f32_e32 v91, s56, v78\n\t" \
  "v_mfma_f32_16x16x128_f8f6f4 a[160:163], v[168:175], v[192:199], a[160:163]\n\t" \
  "s_add_u32 s53, s49, 16384\n\t" \
  "s_add_u32 m0, s48, 49152\n\t" \
  "v_mfma_f32_16x16x128_f8f6f4 a[192:195], v[176:183], v[192:199], a[192:195]\n\t" \
  "buffer_load_dwordx4 v0, s[36:39], s53 offen sc0 sc1 lds\n\t" \
  "v_mfma_f32_16x16x128_f8f6f4 a[196:199], v[176:183], v[200:207], a[196:199]\n\t" \
  "v_mul_f32_e32 v92, s56, v67\n\t" \
  "v_mul_f32_e32 v93, s56, v71\n\t" \
  "v_mul_f32_e32 v94, s56, v75\n\t" \
  "v_mul_f32_e32 v95, s56, v79\n\t" \
  "v_mfma_f32_16x16x128_f8f6f4 a[200:203], v[176:183], v[208:215], a[200:203]\n\t" \
  "buffer_load_dwordx4 v0, s[36:39], s53 offen offset:1024 sc0 sc1 lds\n\t" \
  "v_mfma_f32_16x16x128_f8f6f4 a[204:207], v[176:183], v[216:223], a[204:207]\n\t" \
  "v_min_f32_e32 v24, s60, v80\n\t" \
  "v_min_f32_e32 v25, s60, v81\n\t" \
  "v_min_f32_e32 v26, s60, v82\n\t" \
  "v_min_f32_e32 v27, s60, v83\n\t" \
  "v_mfma_f32_16x16x128_f8f6f4 a[236:239], v[184:191], v[216:223], a[236:239]\n\t" \
  "buffer_load_dwordx4 v0, s[36:39], s53 offen offset:2048 sc0 sc1 lds\n\t" \
  "v_mfma_f32_16x16x128_f8f6f4 a[232:235], v[184:191], v[208:215], a[232:235]\n\t" \
  "v_exp_f32_e32 v24, v24\n\t" \
  "v_exp_f32_e32 v25, v25\n\t" \
  "v_mfma_f32_16x16x128_f8f6f4 a[228:231], v[184:191], v[200:207], a[228:231]\n\t" \
  "buffer_load_dwordx4 v0, s[36:39], s53 offen offset:3072 sc0 sc1 lds\n\t" \
  "v_mfma_f32_16x16x128_f8f6f4 a[224:227], v[184:191], v[192:199], a[224:227]\n\t" \
  "v_exp_f32_e32 v26, v26\n\t" \
  "v_exp_f32_e32 v27, v27\n\t" \
  "s_waitcnt lgkmcnt(0)\n\t" \
  "v_mfma_f32_16x16x128_f8f6f4 a[16:19], v[128:135], v[224:231], a[16:19]\n\t" \
  "s_waitcnt vmcnt(20)\n\t" \
  "s_barrier\n\t" \
  "v_mfma_f32_16x16x128_f8f6f4 a[20:23], v[128:135], v[232:239], a[20:23]\n\t" \
  "ds_read_b128 v[192:195], v4 offset:32768\n\t" \
  "ds_read_b128 v[196:199], v4 offset:40960\n\t" \
  "v_mfma_f32_16x16x128_f8f6f4 a[24:27], v[128:135], v[240:247], a[24:27]\n\t" \
  "ds_read_b128 v[200:203], v4 offset:33792\n\t" \
  "ds_read_b128 v[204:207], v4 offset:41984\n\t" \
  "v_mfma_f32_16x16x128_f8f6f4 a[28:31], v[128:135], v[248:255], a[28:31]\n\t" \
  "ds_read_b128 v[208:211], v4 offset:34816\n\t" \
  "ds_read_b128 v[212:215], v4 offset:43008\n\t" \
  "v_mfma_f32_16x16x128_f8f6f4 a[60:63], v[136:143], v[248:255], a[60:63]\n\t" \
  "ds_read_b128 v[216:219], v4 offset:35840\n\t" \
  "ds_read_b128 v[220:223], v4 offset:44032\n\t" \
  "v_mfma_f32_16x16x128_f8f6f4 a[56:59], v[136:143], v[240:247], a[56:59]\n\t" \
  "v_fma_f32 v24, v24, s58, s58\n\t" \
  "v_fma_f32 v25, v25, s58, s58\n\t" \
  "v_fma_f32 v26, v26, s58, s58\n\t" \
  "v_fma_f32 v27, v27, s58, s58\n\t" \
  "v_mfma_f32_16x16x128_f8f6f4 a[52:55], v[136:143], v[232:239], a[52:55]\n\t" \
  "v_rcp_f32_e32 v24, v24\n\t" \
  "v_rcp_f32_e32 v25, v25\n\t" \
  "v_mfma_f32_16x16x128_f8f6f4 a[48:51], v[136:143], v[224:231], a[48:51]\n\t" \
  "s_add_u32 s53, s49, 32768\n\t" \
  "s_add_u32 m0, s48, 65536\n\t" \
  "v_mfma_f32_16x16x128_f8f6f4 a[80:83], v[144:151], v[224:231], a[80:83]\n\t" \
  "buffer_load_dwordx4 v0, s[36:39], s53 offen sc0 sc1 lds\n\t" \
  "v_mfma_f32_16x16x128_f8f6f4 a[84:87], v[144:151], v[232:239], a[84:87]\n\t" \
  "buffer_load_dword v7, v8, s[44:47], s76 offen sc0 sc1\n\t" \
  "v_mfma_f32_16x16x128_f8f6f4 a[88:91], v[144:151], v[240:247], a[88:91]\n\t" \
  "buffer_load_dwordx4 v0, s[36:39], s53 offen offset:1024 sc0 sc1 lds\n\t" \
  "v_mfma_f32_16x16x128_f8f6f4 a[92:95], v[144:151], v[248:255], a[92:95]\n\t" \
  "v_rcp_f32_e32 v26, v26\n\t" \
  "v_rcp_f32_e32 v27, v27\n\t" \
  "v_mfma_f32_16x16x128_f8f6f4 a[124:127], v[152:159], v[248:255], a[124:127]\n\t" \
  "buffer_load_dwordx4 v0, s[36:39], s53 offen offset:2048 sc0 sc1 lds\n\t" \
  "v_mfma_f32_16x16x128_f8f6f4 a[120:123], v[152:159], v[240:247], a[120:123]\n\t" \
  "v_mul_f32_e32 v80, v80, v24\n\t" \
  "v_mul_f32_e32 v81, v81, v25\n\t" \
  "v_mul_f32_e32 v82, v82, v26\n\t" \
  "v_mul_f32_e32 v83, v83, v27\n\t" \
  "v_mfma_f32_16x16x128_f8f6f4 a[116:119], v[152:159], v[232:239], a[116:119]\n\t" \
  "buffer_load_dwordx4 v0, s[36:39], s53 offen offset:3072 sc0 sc1 lds\n\t" \
  "v_mfma_f32_16x16x128_f8f6f4 a[112:115], v[152:159], v[224:231], a[112:115]\n\t" \
  "s_xor_b32 s91, s68, 7340032\n\t" \
  "buffer_store_dwordx2 v[80:81], v5, s[40:43], s91 offen nt\n\t" \
  "s_waitcnt lgkmcnt(0)\n\t" \
  "v_mfma_f32_16x16x128_f8f6f4 a[0:3], v[128:135], v[192:199], a[0:3]\n\t" \
  "buffer_store_dwordx2 v[82:83], v5, s[40:43], s91 offen offset:128 nt\n\t" \
  "v_mfma_f32_16x16x128_f8f6f4 a[4:7], v[128:135], v[200:207], a[4:7]\n\t" \
  "ds_read_b128 v[160:163], v2 offset:49152\n\t" \
  "ds_read_b128 v[164:167], v2 offset:57344\n\t" \
  "v_mfma_f32_16x16x128_f8f6f4 a[8:11], v[128:135], v[208:215], a[8:11]\n\t" \
  "ds_read_b128 v[168:171], v2 offset:50176\n\t" \
  "ds_read_b128 v[172:175], v2 offset:58368\n\t" \
  "v_mfma_f32_16x16x128_f8f6f4 a[12:15], v[128:135], v[216:223], a[12:15]\n\t" \
  "ds_read_b128 v[176:179], v2 offset:51200\n\t" \
  "ds_read_b128 v[180:183], v2 offset:59392\n\t" \
  "v_mfma_f32_16x16x128_f8f6f4 a[44:47], v[136:143], v[216:223], a[44:47]\n\t" \
  "ds_read_b128 v[184:187], v2 offset:52224\n\t" \
  "ds_read_b128 v[188:191], v2 offset:60416\n\t" \
  "v_mfma_f32_16x16x128_f8f6f4 a[40:43], v[136:143], v[208:215], a[40:43]\n\t" \
  "v_min_f32_e32 v28, s60, v84\n\t" \
  "v_min_f32_e32 v29, s60, v85\n\t" \
  "v_min_f32_e32 v30, s60, v86\n\t" \
  "v_min_f32_e32 v31, s60, v87\n\t" \
  "v_mfma_f32_16x16x128_f8f6f4 a[36:39], v[136:143], v[200:207], a[36:39]\n\t" \
  "v_exp_f32_e32 v28, v28\n\t" \
  "v_exp_f32_e32 v29, v29\n\t" \
  "v_mfma_f32_16x16x128_f8f6f4 a[32:35], v[136:143], v[192:199], a[32:35]\n\t" \
  "s_add_u32 s53, s51, 16384\n\t" \
  "s_add_u32 m0, s48, 81920\n\t" \
  "v_mfma_f32_16x16x128_f8f6f4 a[64:67], v[144:151], v[192:199], a[64:67]\n\t" \
  "buffer_load_dwordx4 v0, s[36:39], s53 offen sc0 sc1 lds\n\t" \
  "v_mfma_f32_16x16x128_f8f6f4 a[68:71], v[144:151], v[200:207], a[68:71]\n\t" \
  "v_exp_f32_e32 v30, v30\n\t" \
  "v_exp_f32_e32 v31, v31\n\t" \
  "v_mfma_f32_16x16x128_f8f6f4 a[72:75], v[144:151], v[208:215], a[72:75]\n\t" \
  "buffer_load_dwordx4 v0, s[36:39], s53 offen offset:1024 sc0 sc1 lds\n\t" \
  "v_mfma_f32_16x16x128_f8f6f4 a[76:79], v[144:151], v[216:223], a[76:79]\n\t" \
  "v_fma_f32 v28, v28, s58, s58\n\t" \
  "v_fma_f32 v29, v29, s58, s58\n\t" \
  "v_fma_f32 v30, v30, s58, s58\n\t" \
  "v_fma_f32 v31, v31, s58, s58\n\t" \
  "v_mfma_f32_16x16x128_f8f6f4 a[108:111], v[152:159], v[216:223], a[108:111]\n\t" \
  "buffer_load_dwordx4 v0, s[36:39], s53 offen offset:2048 sc0 sc1 lds\n\t" \
  "v_mfma_f32_16x16x128_f8f6f4 a[104:107], v[152:159], v[208:215], a[104:107]\n\t" \
  "v_rcp_f32_e32 v28, v28\n\t" \
  "v_rcp_f32_e32 v29, v29\n\t" \
  "v_mfma_f32_16x16x128_f8f6f4 a[100:103], v[152:159], v[200:207], a[100:103]\n\t" \
  "buffer_load_dwordx4 v0, s[36:39], s53 offen offset:3072 sc0 sc1 lds\n\t" \
  "v_mfma_f32_16x16x128_f8f6f4 a[96:99], v[152:159], v[192:199], a[96:99]\n\t" \
  "v_rcp_f32_e32 v30, v30\n\t" \
  "v_rcp_f32_e32 v31, v31\n\t" \
  "s_waitcnt lgkmcnt(0)\n\t" \
  "v_mfma_f32_16x16x128_f8f6f4 a[128:131], v[160:167], v[192:199], a[128:131]\n\t" \
  "s_waitcnt vmcnt(21)\n\t" \
  "s_barrier\n\t" \
  "v_mfma_f32_16x16x128_f8f6f4 a[132:135], v[160:167], v[200:207], a[132:135]\n\t" \
  "ds_read_b128 v[128:131], v1 offset:0\n\t" \
  "ds_read_b128 v[132:135], v1 offset:8192\n\t" \
  "v_mfma_f32_16x16x128_f8f6f4 a[136:139], v[160:167], v[208:215], a[136:139]\n\t" \
  "ds_read_b128 v[136:139], v1 offset:1024\n\t" \
  "ds_read_b128 v[140:143], v1 offset:9216\n\t" \
  "v_mfma_f32_16x16x128_f8f6f4 a[140:143], v[160:167], v[216:223], a[140:143]\n\t" \
  "ds_read_b128 v[144:147], v1 offset:2048\n\t" \
  "ds_read_b128 v[148:151], v1 offset:10240\n\t" \
  "v_mfma_f32_16x16x128_f8f6f4 a[172:175], v[168:175], v[216:223], a[172:175]\n\t" \
  "ds_read_b128 v[152:155], v1 offset:3072\n\t" \
  "ds_read_b128 v[156:159], v1 offset:11264\n\t" \
  "v_mfma_f32_16x16x128_f8f6f4 a[168:171], v[168:175], v[208:215], a[168:171]\n\t" \
  "v_mul_f32_e32 v84, v84, v28\n\t" \
  "v_mul_f32_e32 v85, v85, v29\n\t" \
  "v_mul_f32_e32 v86, v86, v30\n\t" \
  "v_mul_f32_e32 v87, v87, v31\n\t" \
  "v_mfma_f32_16x16x128_f8f6f4 a[164:167], v[168:175], v[200:207], a[164:167]\n\t" \
  "s_xor_b32 s91, s68, 7405568\n\t" \
  "buffer_store_dwordx2 v[84:85], v5, s[40:43], s91 offen nt\n\t" \
  "v_mfma_f32_16x16x128_f8f6f4 a[160:163], v[168:175], v[192:199], a[160:163]\n\t" \
  "s_add_u32 s53, s50, 16384\n\t" \
  "s_add_u32 m0, s48, 98304\n\t" \
  "buffer_store_dwordx2 v[86:87], v5, s[40:43], s91 offen offset:128 nt\n\t" \
  "v_mfma_f32_16x16x128_f8f6f4 a[192:195], v[176:183], v[192:199], a[192:195]\n\t" \
  "buffer_load_dwordx4 v0, s[36:39], s53 offen sc0 sc1 lds\n\t" \
  "v_mfma_f32_16x16x128_f8f6f4 a[196:199], v[176:183], v[200:207], a[196:199]\n\t" \
  "v_min_f32_e32 v12, s60, v88\n\t" \
  "v_min_f32_e32 v13, s60, v89\n\t" \
  "v_min_f32_e32 v14, s60, v90\n\t" \
  "v_min_f32_e32 v15, s60, v91\n\t" \
  "v_mfma_f32_16x16x128_f8f6f4 a[200:203], v[176:183], v[208:215], a[200:203]\n\t" \
  "buffer_load_dwordx4 v0, s[36:39], s53 offen offset:1024 sc0 sc1 lds\n\t" \
  "v_mfma_f32_16x16x128_f8f6f4 a[204:207], v[176:183], v[216:223], a[204:207]\n\t" \
  "v_exp_f32_e32 v12, v12\n\t" \
  "v_exp_f32_e32 v13, v13\n\t" \
  "v_mfma_f32_16x16x128_f8f6f4 a[236:239], v[184:191], v[216:223], a[236:239]\n\t" \
  "buffer_load_dwordx4 v0, s[36:39], s53 offen offset:2048 sc0 sc1 lds\n\t" \
  "v_mfma_f32_16x16x128_f8f6f4 a[232:235], v[184:191], v[208:215], a[232:235]\n\t" \
  "v_exp_f32_e32 v14, v14\n\t" \
  "v_exp_f32_e32 v15, v15\n\t" \
  "v_mfma_f32_16x16x128_f8f6f4 a[228:231], v[184:191], v[200:207], a[228:231]\n\t" \
  "buffer_load_dwordx4 v0, s[36:39], s53 offen offset:3072 sc0 sc1 lds\n\t" \
  "v_mfma_f32_16x16x128_f8f6f4 a[224:227], v[184:191], v[192:199], a[224:227]\n\t" \
  "v_fma_f32 v12, v12, s58, s58\n\t" \
  "v_fma_f32 v13, v13, s58, s58\n\t" \
  "v_fma_f32 v14, v14, s58, s58\n\t" \
  "v_fma_f32 v15, v15, s58, s58\n\t" \
  "s_waitcnt lgkmcnt(0)\n\t" \
  "v_mfma_f32_16x16x128_f8f6f4 a[144:147], v[160:167], v[224:231], a[144:147]\n\t" \
  "v_rcp_f32_e32 v12, v12\n\t" \
  "v_rcp_f32_e32 v13, v13\n\t" \
  "v_mfma_f32_16x16x128_f8f6f4 a[148:151], v[160:167], v[232:239], a[148:151]\n\t" \
  "ds_read_b128 v[192:195], v3 offset:16384\n\t" \
  "ds_read_b128 v[196:199], v3 offset:24576\n\t" \
  "v_mfma_f32_16x16x128_f8f6f4 a[152:155], v[160:167], v[240:247], a[152:155]\n\t" \
  "ds_read_b128 v[200:203], v3 offset:17408\n\t" \
  "ds_read_b128 v[204:207], v3 offset:25600\n\t" \
  "v_mfma_f32_16x16x128_f8f6f4 a[156:159], v[160:167], v[248:255], a[156:159]\n\t" \
  "ds_read_b128 v[208:211], v3 offset:18432\n\t" \
  "ds_read_b128 v[212:215], v3 offset:26624\n\t" \
  "v_mfma_f32_16x16x128_f8f6f4 a[188:191], v[168:175], v[248:255], a[188:191]\n\t" \
  "ds_read_b128 v[216:219], v3 offset:19456\n\t" \
  "ds_read_b128 v[220:223], v3 offset:27648\n\t" \
  "v_mfma_f32_16x16x128_f8f6f4 a[184:187], v[168:175], v[240:247], a[184:187]\n\t" \
  "v_rcp_f32_e32 v14, v14\n\t" \
  "v_rcp_f32_e32 v15, v15\n\t" \
  "v_mfma_f32_16x16x128_f8f6f4 a[180:183], v[168:175], v[232:239], a[180:183]\n\t" \
  "v_mul_f32_e32 v88, v88, v12\n\t" \
  "v_mul_f32_e32 v89, v89, v13\n\t" \
  "v_mul_f32_e32 v90, v90, v14\n\t" \
  "v_mul_f32_e32 v91, v91, v15\n\t" \
  "v_mfma_f32_16x16x128_f8f6f4 a[176:179], v[168:175], v[224:231], a[176:179]\n\t" \
  "s_add_u32 s53, s49, 49152\n\t" \
  "s_add_u32 m0, s48, 114688\n\t" \
  "v_mfma_f32_16x16x128_f8f6f4 a[208:211], v[176:183], v[224:231], a[208:211]\n\t" \
  "buffer_load_dwordx4 v0, s[36:39], s53 offen sc0 sc1 lds\n\t" \
  "v_mfma_f32_16x16x128_f8f6f4 a[212:215], v[176:183], v[232:239], a[212:215]\n\t" \
  "s_and_b32 s54, s62, 0xff\n\t" \
  "s_lshr_b64 s[62:63], s[62:63], 8\n\t" \
  "s_lshl_b32 s55, s64, 24\n\t" \
  "s_or_b32 s63, s63, s55\n\t" \
  "v_mfma_f32_16x16x128_f8f6f4 a[216:219], v[176:183], v[240:247], a[216:219]\n\t" \
  "buffer_load_dwordx4 v0, s[36:39], s53 offen offset:1024 sc0 sc1 lds\n\t" \
  "v_mfma_f32_16x16x128_f8f6f4 a[220:223], v[176:183], v[248:255], a[220:223]\n\t" \
  "s_lshr_b64 s[64:65], s[64:65], 8\n\t" \
  "s_and_b32 s54, s54, 0x7f\n\t" \
  "s_lshl_b32 s69, s54, 9\n\t" \
  "v_mfma_f32_16x16x128_f8f6f4 a[252:255], v[184:191], v[248:255], a[252:255]\n\t" \
  "buffer_load_dwordx4 v0, s[36:39], s53 offen offset:2048 sc0 sc1 lds\n\t" \
  "v_mfma_f32_16x16x128_f8f6f4 a[248:251], v[184:191], v[240:247], a[248:251]\n\t" \
  "s_add_u32 s69, s69, s71\n\t" \
  "s_lshl_b32 s54, s54, 19\n\t" \
  "s_add_u32 s54, s54, s70\n\t" \
  "v_mfma_f32_16x16x128_f8f6f4 a[244:247], v[184:191], v[232:239], a[244:247]\n\t" \
  "buffer_load_dwordx4 v0, s[36:39], s53 offen offset:3072 sc0 sc1 lds\n\t" \
  "v_mfma_f32_16x16x128_f8f6f4 a[240:243], v[184:191], v[224:231], a[240:243]\n\t" \
  "s_add_u32 s49, s49, 65536\n\t" \
  "s_add_u32 s50, s50, 32768\n\t" \
  "s_add_u32 s51, s51, 32768\n\t" \
  "s_waitcnt lgkmcnt(0)\n\t" \
  "v_mfma_f32_16x16x128_f8f6f4 a[0:3], v[128:135], v[192:199], a[0:3]\n\t" \
  "s_waitcnt vmcnt(21)\n\t" \
  "s_barrier\n\t" \
  "v_mfma_f32_16x16x128_f8f6f4 a[4:7], v[128:135], v[200:207], a[4:7]\n\t" \
  "ds_read_b128 v[224:227], v3 offset:32768\n\t" \
  "ds_read_b128 v[228:231], v3 offset:40960\n\t" \
  "s_add_u32 s51, s54, 262144\n\t" \
  "v_mfma_f32_16x16x128_f8f6f4 a[8:11], v[128:135], v[208:215], a[8:11]\n\t" \
  "ds_read_b128 v[232:235], v3 offset:33792\n\t" \
  "ds_read_b128 v[236:239], v3 offset:41984\n\t" \
  "v_mfma_f32_16x16x128_f8f6f4 a[12:15], v[128:135], v[216:223], a[12:15]\n\t" \
  "ds_read_b128 v[240:243], v3 offset:34816\n\t" \
  "ds_read_b128 v[244:247], v3 offset:43008\n\t" \
  "v_mfma_f32_16x16x128_f8f6f4 a[44:47], v[136:143], v[216:223], a[44:47]\n\t" \
  "ds_read_b128 v[248:251], v3 offset:35840\n\t" \
  "ds_read_b128 v[252:255], v3 offset:44032\n\t" \
  "v_mfma_f32_16x16x128_f8f6f4 a[40:43], v[136:143], v[208:215], a[40:43]\n\t" \
  "s_waitcnt vmcnt(19)\n\t" \
  "v_cmp_le_u32_e32 vcc, s75, v7\n\t" \
  "v_mfma_f32_16x16x128_f8f6f4 a[36:39], v[136:143], v[200:207], a[36:39]\n\t" \
  "s_cmp_eq_u64 vcc, -1\n\t" \
  "s_cbranch_scc0 L_SLOW07_%=\n\t" \
  "L_OK07_%=:\n\t" \
  "s_add_u32 s76, s76, 1024\n\t" \
  "v_mfma_f32_16x16x128_f8f6f4 a[32:35], v[136:143], v[192:199], a[32:35]\n\t" \
  "s_mov_b32 s53, s49\n\t" \
  "s_mov_b32 m0, s48\n\t" \
  "v_mfma_f32_16x16x128_f8f6f4 a[64:67], v[144:151], v[192:199], a[64:67]\n\t" \
  "buffer_load_dwordx4 v0, s[36:39], s53 offen sc0 sc1 lds\n\t" \
  "v_mfma_f32_16x16x128_f8f6f4 a[68:71], v[144:151], v[200:207], a[68:71]\n\t" \
  "s_xor_b32 s91, s68, 7471104\n\t" \
  "buffer_store_dwordx2 v[88:89], v5, s[40:43], s91 offen nt\n\t" \
  "v_mfma_f32_16x16x128_f8f6f4 a[72:75], v[144:151], v[208:215], a[72:75]\n\t" \
  "buffer_load_dwordx4 v0, s[36:39], s53 offen offset:1024 sc0 sc1 lds\n\t" \
  "v_mfma_f32_16x16x128_f8f6f4 a[76:79], v[144:151], v[216:223], a[76:79]\n\t" \
  "buffer_store_dwordx2 v[90:91], v5, s[40:43], s91 offen offset:128 nt\n\t" \
  "v_mfma_f32_16x16x128_f8f6f4 a[108:111], v[152:159], v[216:223], a[108:111]\n\t" \
  "buffer_load_dwordx4 v0, s[36:39], s53 offen offset:2048 sc0 sc1 lds\n\t" \
  "v_mfma_f32_16x16x128_f8f6f4 a[104:107], v[152:159], v[208:215], a[104:107]\n\t" \
  "v_min_f32_e32 v16, s60, v92\n\t" \
  "v_min_f32_e32 v17, s60, v93\n\t" \
  "v_min_f32_e32 v18, s60, v94\n\t" \
  "v_min_f32_e32 v19, s60, v95\n\t" \
  "v_mfma_f32_16x16x128_f8f6f4 a[100:103], v[152:159], v[200:207], a[100:103]\n\t" \
  "buffer_load_dwordx4 v0, s[36:39], s53 offen offset:3072 sc0 sc1 lds\n\t" \
  "v_mfma_f32_16x16x128_f8f6f4 a[96:99], v[152:159], v[192:199], a[96:99]\n\t" \
  "v_exp_f32_e32 v16, v16\n\t" \
  "v_exp_f32_e32 v17, v17\n\t" \
  "s_waitcnt lgkmcnt(0)\n\t" \
  "v_mfma_f32_16x16x128_f8f6f4 a[16:19], v[128:135], v[224:231], a[16:19]\n\t" \
  "v_exp_f32_e32 v18, v18\n\t" \
  "v_exp_f32_e32 v19, v19\n\t" \
  "v_mfma_f32_16x16x128_f8f6f4 a[20:23], v[128:135], v[232:239], a[20:23]\n\t" \
  "ds_read_b128 v[160:163], v1 offset:49152\n\t" \
  "ds_read_b128 v[164:167], v1 offset:57344\n\t" \
  "v_mfma_f32_16x16x128_f8f6f4 a[24:27], v[128:135], v[240:247], a[24:27]\n\t" \
  "ds_read_b128 v[168:171], v1 offset:50176\n\t" \
  "ds_read_b128 v[172:175], v1 offset:58368\n\t" \
  "v_mfma_f32_16x16x128_f8f6f4 a[28:31], v[128:135], v[248:255], a[28:31]\n\t" \
  "ds_read_b128 v[176:179], v1 offset:51200\n\t" \
  "ds_read_b128 v[180:183], v1 offset:59392\n\t" \
  "v_mfma_f32_16x16x128_f8f6f4 a[60:63], v[136:143], v[248:255], a[60:63]\n\t" \
  "ds_read_b128 v[184:187], v1 offset:52224\n\t" \
  "ds_read_b128 v[188:191], v1 offset:60416\n\t" \
  "v_mfma_f32_16x16x128_f8f6f4 a[56:59], v[136:143], v[240:247], a[56:59]\n\t" \
  "v_fma_f32 v16, v16, s58, s58\n\t" \
  "v_fma_f32 v17, v17, s58, s58\n\t" \
  "v_fma_f32 v18, v18, s58, s58\n\t" \
  "v_fma_f32 v19, v19, s58, s58\n\t" \
  "v_mfma_f32_16x16x128_f8f6f4 a[52:55], v[136:143], v[232:239], a[52:55]\n\t" \
  "v_mfma_f32_16x16x128_f8f6f4 a[48:51], v[136:143], v[224:231], a[48:51]\n\t" \
  "s_mov_b32 s53, s50\n\t" \
  "s_add_u32 m0, s48, 16384\n\t" \
  "v_mfma_f32_16x16x128_f8f6f4 a[80:83], v[144:151], v[224:231], a[80:83]\n\t" \
  "buffer_load_dwordx4 v0, s[36:39], s53 offen sc0 sc1 lds\n\t" \
  "v_mfma_f32_16x16x128_f8f6f4 a[84:87], v[144:151], v[232:239], a[84:87]\n\t" \
  "v_rcp_f32_e32 v16, v16\n\t" \
  "v_rcp_f32_e32 v17, v17\n\t" \
  "v_mfma_f32_16x16x128_f8f6f4 a[88:91], v[144:151], v[240:247], a[88:91]\n\t" \
  "buffer_load_dwordx4 v0, s[36:39], s53 offen offset:1024 sc0 sc1 lds\n\t" \
  "v_mfma_f32_16x16x128_f8f6f4 a[92:95], v[144:151], v[248:255], a[92:95]\n\t" \
  "v_rcp_f32_e32 v18, v18\n\t" \
  "v_rcp_f32_e32 v19, v19\n\t" \
  "v_mfma_f32_16x16x128_f8f6f4 a[124:127], v[152:159], v[248:255], a[124:127]\n\t" \
  "buffer_load_dwordx4 v0, s[36:39], s53 offen offset:2048 sc0 sc1 lds\n\t" \
  "v_mfma_f32_16x16x128_f8f6f4 a[120:123], v[152:159], v[240:247], a[120:123]\n\t" \
  "v_mul_f32_e32 v92, v92, v16\n\t" \
  "v_mul_f32_e32 v93, v93, v17\n\t" \
  "v_mul_f32_e32 v94, v94, v18\n\t" \
  "v_mul_f32_e32 v95, v95, v19\n\t" \
  "v_mfma_f32_16x16x128_f8f6f4 a[116:119], v[152:159], v[232:239], a[116:119]\n\t" \
  "buffer_load_dwordx4 v0, s[36:39], s53 offen offset:3072 sc0 sc1 lds\n\t" \
  "v_mfma_f32_16x16x128_f8f6f4 a[112:115], v[152:159], v[224:231], a[112:115]\n\t" \
  "s_xor_b32 s91, s68, 7536640\n\t" \
  "buffer_store_dwordx2 v[92:93], v5, s[40:43], s91 offen nt\n\t" \
  "s_waitcnt lgkmcnt(0)\n\t" \
  "v_mfma_f32_16x16x128_f8f6f4 a[144:147], v[160:167], v[224:231], a[144:147]\n\t" \
  "s_waitcnt vmcnt(21)\n\t" \
  "s_barrier\n\t" \
  "v_mfma_f32_16x16x128_f8f6f4 a[148:151], v[160:167], v[232:239], a[148:151]\n\t" \
  "ds_read_b128 v[128:131], v2 offset:0\n\t" \
  "ds_read_b128 v[132:135], v2 offset:8192\n\t" \
  "v_mfma_f32_16x16x128_f8f6f4 a[152:155], v[160:167], v[240:247], a[152:155]\n\t" \
  "ds_read_b128 v[136:139], v2 offset:1024\n\t" \
  "ds_read_b128 v[140:143], v2 offset:9216\n\t" \
  "v_mfma_f32_16x16x128_f8f6f4 a[156:159], v[160:167], v[248:255], a[156:159]\n\t" \
  "ds_read_b128 v[144:147], v2 offset:2048\n\t" \
  "ds_read_b128 v[148:151], v2 offset:10240\n\t" \
  "v_mfma_f32_16x16x128_f8f6f4 a[188:191], v[168:175], v[248:255], a[188:191]\n\t" \
  "ds_read_b128 v[152:155], v2 offset:3072\n\t" \
  "ds_read_b128 v[156:159], v2 offset:11264\n\t" \
  "v_mfma_f32_16x16x128_f8f6f4 a[184:187], v[168:175], v[240:247], a[184:187]\n\t" \
  "buffer_store_dwordx2 v[94:95], v5, s[40:43], s91 offen offset:128 nt\n\t" \
  "v_mfma_f32_16x16x128_f8f6f4 a[180:183], v[168:175], v[232:239], a[180:183]\n\t" \
  "v_mfma_f32_16x16x128_f8f6f4 a[176:179], v[168:175], v[224:231], a[176:179]\n\t" \
  "s_mov_b32 s53, s51\n\t" \
  "s_add_u32 m0, s48, 32768\n\t" \
  "v_mfma_f32_16x16x128_f8f6f4 a[208:211], v[176:183], v[224:231], a[208:211]\n\t" \
  "buffer_load_dwordx4 v0, s[36:39], s53 offen sc0 sc1 lds\n\t" \
  "v_mfma_f32_16x16x128_f8f6f4 a[212:215], v[176:183], v[232:239], a[212:215]\n\t" \
  "v_mfma_f32_16x16x128_f8f6f4 a[216:219], v[176:183], v[240:247], a[216:219]\n\t" \
  "buffer_load_dwordx4 v0, s[36:39], s53 offen offset:1024 sc0 sc1 lds\n\t" \
  "v_mfma_f32_16x16x128_f8f6f4 a[220:223], v[176:183], v[248:255], a[220:223]\n\t" \
  "v_mfma_f32_16x16x128_f8f6f4 a[252:255], v[184:191], v[248:255], a[252:255]\n\t" \
  "buffer_load_dwordx4 v0, s[36:39], s53 offen offset:2048 sc0 sc1 lds\n\t" \
  "v_mfma_f32_16x16x128_f8f6f4 a[248:251], v[184:191], v[240:247], a[248:251]\n\t" \
  "v_mfma_f32_16x16x128_f8f6f4 a[244:247], v[184:191], v[232:239], a[244:247]\n\t" \
  "buffer_load_dwordx4 v0, s[36:39], s53 offen offset:3072 sc0 sc1 lds\n\t" \
  "v_mfma_f32_16x16x128_f8f6f4 a[240:243], v[184:191], v[224:231], a[240:243]\n\t" \
  "s_waitcnt lgkmcnt(0)\n\t" \
  "v_mfma_f32_16x16x128_f8f6f4 a[128:131], v[160:167], v[192:199], a[128:131]\n\t" \
  "v_mfma_f32_16x16x128_f8f6f4 a[132:135], v[160:167], v[200:207], a[132:135]\n\t" \
  "ds_read_b128 v[224:227], v4 offset:16384\n\t" \
  "ds_read_b128 v[228:231], v4 offset:24576\n\t" \
  "v_mfma_f32_16x16x128_f8f6f4 a[136:139], v[160:167], v[208:215], a[136:139]\n\t" \
  "ds_read_b128 v[232:235], v4 offset:17408\n\t" \
  "ds_read_b128 v[236:239], v4 offset:25600\n\t" \
  "v_mfma_f32_16x16x128_f8f6f4 a[140:143], v[160:167], v[216:223], a[140:143]\n\t" \
  "ds_read_b128 v[240:243], v4 offset:18432\n\t" \
  "ds_read_b128 v[244:247], v4 offset:26624\n\t" \
  "v_mfma_f32_16x16x128_f8f6f4 a[172:175], v[168:175], v[216:223], a[172:175]\n\t" \
  "ds_read_b128 v[248:251], v4 offset:19456\n\t" \
  "ds_read_b128 v[252:255], v4 offset:27648\n\t" \
  "v_mfma_f32_16x16x128_f8f6f4 a[168:171], v[168:175], v[208:215], a[168:171]\n\t" \
  "v_mfma_f32_16x16x128_f8f6f4 a[164:167], v[168:175], v[200:207], a[164:167]\n\t" \
  "v_mfma_f32_16x16x128_f8f6f4 a[160:163], v[168:175], v[192:199], a[160:163]\n\t" \
  "s_add_u32 s53, s49, 16384\n\t" \
  "s_add_u32 m0, s48, 49152\n\t" \
  "v_mfma_f32_16x16x128_f8f6f4 a[192:195], v[176:183], v[192:199], a[192:195]\n\t" \
  "buffer_load_dwordx4 v0, s[36:39], s53 offen sc0 sc1 lds\n\t" \
  "v_mfma_f32_16x16x128_f8f6f4 a[196:199], v[176:183], v[200:207], a[196:199]\n\t" \
  "v_mfma_f32_16x16x128_f8f6f4 a[200:203], v[176:183], v[208:215], a[200:203]\n\t" \
  "buffer_load_dwordx4 v0, s[36:39], s53 offen offset:1024 sc0 sc1 lds\n\t" \
  "v_mfma_f32_16x16x128_f8f6f4 a[204:207], v[176:183], v[216:223], a[204:207]\n\t" \
  "v_mfma_f32_16x16x128_f8f6f4 a[236:239], v[184:191], v[216:223], a[236:239]\n\t" \
  "buffer_load_dwordx4 v0, s[36:39], s53 offen offset:2048 sc0 sc1 lds\n\t" \
  "v_mfma_f32_16x16x128_f8f6f4 a[232:235], v[184:191], v[208:215], a[232:235]\n\t" \
  "v_mfma_f32_16x16x128_f8f6f4 a[228:231], v[184:191], v[200:207], a[228:231]\n\t" \
  "buffer_load_dwordx4 v0, s[36:39], s53 offen offset:3072 sc0 sc1 lds\n\t" \
  "v_mfma_f32_16x16x128_f8f6f4 a[224:227], v[184:191], v[192:199], a[224:227]\n\t" \
  "s_waitcnt lgkmcnt(0)\n\t" \
  "v_mfma_f32_16x16x128_f8f6f4 a[16:19], v[128:135], v[224:231], a[16:19]\n\t" \
  "s_waitcnt vmcnt(20)\n\t" \
  "s_barrier\n\t" \
  "v_mfma_f32_16x16x128_f8f6f4 a[20:23], v[128:135], v[232:239], a[20:23]\n\t" \
  "ds_read_b128 v[192:195], v4 offset:32768\n\t" \
  "ds_read_b128 v[196:199], v4 offset:40960\n\t" \
  "v_mfma_f32_16x16x128_f8f6f4 a[24:27], v[128:135], v[240:247], a[24:27]\n\t" \
  "ds_read_b128 v[200:203], v4 offset:33792\n\t" \
  "ds_read_b128 v[204:207], v4 offset:41984\n\t" \
  "v_mfma_f32_16x16x128_f8f6f4 a[28:31], v[128:135], v[248:255], a[28:31]\n\t" \
  "ds_read_b128 v[208:211], v4 offset:34816\n\t" \
  "ds_read_b128 v[212:215], v4 offset:43008\n\t" \
  "v_mfma_f32_16x16x128_f8f6f4 a[60:63], v[136:143], v[248:255], a[60:63]\n\t" \
  "ds_read_b128 v[216:219], v4 offset:35840\n\t" \
  "ds_read_b128 v[220:223], v4 offset:44032\n\t" \
  "v_mfma_f32_16x16x128_f8f6f4 a[56:59], v[136:143], v[240:247], a[56:59]\n\t" \
  "v_mfma_f32_16x16x128_f8f6f4 a[52:55], v[136:143], v[232:239], a[52:55]\n\t" \
  "v_mfma_f32_16x16x128_f8f6f4 a[48:51], v[136:143], v[224:231], a[48:51]\n\t" \
  "s_add_u32 s53, s49, 32768\n\t" \
  "s_add_u32 m0, s48, 65536\n\t" \
  "v_mfma_f32_16x16x128_f8f6f4 a[80:83], v[144:151], v[224:231], a[80:83]\n\t" \
  "buffer_load_dwordx4 v0, s[36:39], s53 offen sc0 sc1 lds\n\t" \
  "v_mfma_f32_16x16x128_f8f6f4 a[84:87], v[144:151], v[232:239], a[84:87]\n\t" \
  "v_mfma_f32_16x16x128_f8f6f4 a[88:91], v[144:151], v[240:247], a[88:91]\n\t" \
  "buffer_load_dwordx4 v0, s[36:39], s53 offen offset:1024 sc0 sc1 lds\n\t" \
  "v_mfma_f32_16x16x128_f8f6f4 a[92:95], v[144:151], v[248:255], a[92:95]\n\t" \
  "v_mfma_f32_16x16x128_f8f6f4 a[124:127], v[152:159], v[248:255], a[124:127]\n\t" \
  "buffer_load_dwordx4 v0, s[36:39], s53 offen offset:2048 sc0 sc1 lds\n\t" \
  "v_mfma_f32_16x16x128_f8f6f4 a[120:123], v[152:159], v[240:247], a[120:123]\n\t" \
  "v_mfma_f32_16x16x128_f8f6f4 a[116:119], v[152:159], v[232:239], a[116:119]\n\t" \
  "buffer_load_dwordx4 v0, s[36:39], s53 offen offset:3072 sc0 sc1 lds\n\t" \
  "v_mfma_f32_16x16x128_f8f6f4 a[112:115], v[152:159], v[224:231], a[112:115]\n\t" \
  "s_waitcnt lgkmcnt(0)\n\t" \
  "v_mfma_f32_16x16x128_f8f6f4 a[0:3], v[128:135], v[192:199], a[0:3]\n\t" \
  "v_mfma_f32_16x16x128_f8f6f4 a[4:7], v[128:135], v[200:207], a[4:7]\n\t" \
  "ds_read_b128 v[160:163], v2 offset:49152\n\t" \
  "ds_read_b128 v[164:167], v2 offset:57344\n\t" \
  "v_accvgpr_read_b32 v32, a16\n\t" \
  "v_accvgpr_read_b32 v36, a17\n\t" \
  "v_accvgpr_read_b32 v40, a18\n\t" \
  "v_mfma_f32_16x16x128_f8f6f4 a[8:11], v[128:135], v[208:215], a[8:11]\n\t" \
  "ds_read_b128 v[168:171], v2 offset:50176\n\t" \
  "ds_read_b128 v[172:175], v2 offset:58368\n\t" \
  "v_accvgpr_read_b32 v44, a19\n\t" \
  "v_accvgpr_read_b32 v33, a20\n\t" \
  "v_accvgpr_read_b32 v37, a21\n\t" \
  "v_mfma_f32_16x16x128_f8f6f4 a[12:15], v[128:135], v[216:223], a[12:15]\n\t" \
  "ds_read_b128 v[176:179], v2 offset:51200\n\t" \
  "ds_read_b128 v[180:183], v2 offset:59392\n\t" \
  "v_accvgpr_read_b32 v41, a22\n\t" \
  "v_accvgpr_read_b32 v45, a23\n\t" \
  "v_accvgpr_read_b32 v34, a24\n\t" \
  "v_mfma_f32_16x16x128_f8f6f4 a[44:47], v[136:143], v[216:223], a[44:47]\n\t" \
  "ds_read_b128 v[184:187], v2 offset:52224\n\t" \
  "ds_read_b128 v[188:191], v2 offset:60416\n\t" \
  "v_accvgpr_read_b32 v38, a25\n\t" \
  "v_accvgpr_read_b32 v42, a26\n\t" \
  "v_accvgpr_read_b32 v46, a27\n\t" \
  "v_mfma_f32_16x16x128_f8f6f4 a[40:43], v[136:143], v[208:215], a[40:43]\n\t" \
  "v_accvgpr_read_b32 v35, a28\n\t" \
  "v_accvgpr_read_b32 v39, a29\n\t" \
  "v_accvgpr_read_b32 v43, a30\n\t" \
  "v_accvgpr_read_b32 v47, a31\n\t" \
  "v_mfma_f32_16x16x128_f8f6f4 a[36:39], v[136:143], v[200:207], a[36:39]\n\t" \
  "v_accvgpr_read_b32 v48, a48\n\t" \
  "v_accvgpr_read_b32 v52, a49\n\t" \
  "v_accvgpr_read_b32 v56, a50\n\t" \
  "v_accvgpr_read_b32 v60, a51\n\t" \
  "v_mfma_f32_16x16x128_f8f6f4 a[32:35], v[136:143], v[192:199], a[32:35]\n\t" \
  "s_add_u32 s53, s51, 16384\n\t" \
  "s_add_u32 m0, s48, 81920\n\t" \
  "v_accvgpr_read_b32 v49, a52\n\t" \
  "v_accvgpr_read_b32 v53, a53\n\t" \
  "v_accvgpr_read_b32 v57, a54\n\t" \
  "v_mfma_f32_16x16x128_f8f6f4 a[64:67], v[144:151], v[192:199], a[64:67]\n\t" \
  "buffer_load_dwordx4 v0, s[36:39], s53 offen sc0 sc1 lds\n\t" \
  "v_accvgpr_read_b32 v61, a55\n\t" \
  "v_accvgpr_read_b32 v50, a56\n\t" \
  "v_accvgpr_read_b32 v54, a57\n\t" \
  "v_mfma_f32_16x16x128_f8f6f4 a[68:71], v[144:151], v[200:207], a[68:71]\n\t" \
  "v_accvgpr_read_b32 v58, a58\n\t" \
  "v_accvgpr_read_b32 v62, a59\n\t" \
  "v_accvgpr_read_b32 v51, a60\n\t" \
  "v_accvgpr_read_b32 v55, a61\n\t" \
  "v_mfma_f32_16x16x128_f8f6f4 a[72:75], v[144:151], v[208:215], a[72:75]\n\t" \
  "buffer_load_dwordx4 v0, s[36:39], s53 offen offset:1024 sc0 sc1 lds\n\t" \
  "v_accvgpr_read_b32 v59, a62\n\t" \
  "v_accvgpr_read_b32 v63, a63\n\t" \
  "v_accvgpr_read_b32 v64, a80\n\t" \
  "v_mfma_f32_16x16x128_f8f6f4 a[76:79], v[144:151], v[216:223], a[76:79]\n\t" \
  "v_accvgpr_read_b32 v68, a81\n\t" \
  "v_accvgpr_read_b32 v72, a82\n\t" \
  "v_accvgpr_read_b32 v76, a83\n\t" \
  "v_accvgpr_read_b32 v65, a84\n\t" \
  "v_mfma_f32_16x16x128_f8f6f4 a[108:111], v[152:159], v[216:223], a[108:111]\n\t" \
  "buffer_load_dwordx4 v0, s[36:39], s53 offen offset:2048 sc0 sc1 lds\n\t" \
  "v_accvgpr_read_b32 v69, a85\n\t" \
  "v_accvgpr_read_b32 v73, a86\n\t" \
  "v_accvgpr_read_b32 v77, a87\n\t" \
  "v_mfma_f32_16x16x128_f8f6f4 a[104:107], v[152:159], v[208:215], a[104:107]\n\t" \
  "v_accvgpr_read_b32 v66, a88\n\t" \
  "v_accvgpr_read_b32 v70, a89\n\t" \
  "v_accvgpr_read_b32 v74, a90\n\t" \
  "v_accvgpr_read_b32 v78, a91\n\t" \
  "v_mfma_f32_16x16x128_f8f6f4 a[100:103], v[152:159], v[200:207], a[100:103]\n\t" \
  "buffer_load_dwordx4 v0, s[36:39], s53 offen offset:3072 sc0 sc1 lds\n\t" \
  "v_accvgpr_read_b32 v67, a92\n\t" \
  "v_accvgpr_read_b32 v71, a93\n\t" \
  "v_accvgpr_read_b32 v75, a94\n\t" \
  "v_mfma_f32_16x16x128_f8f6f4 a[96:99], v[152:159], v[192:199], a[96:99]\n\t" \
  "v_accvgpr_read_b32 v79, a95\n\t" \
  "v_accvgpr_read_b32 v80, a112\n\t" \
  "v_accvgpr_read_b32 v84, a113\n\t" \
  "v_accvgpr_read_b32 v88, a114\n\t" \
  "s_waitcnt lgkmcnt(0)\n\t" \
  "v_mfma_f32_16x16x128_f8f6f4 a[128:131], v[160:167], v[192:199], a[128:131]\n\t" \
  "s_waitcnt vmcnt(18)\n\t" \
  "s_barrier\n\t" \
  "v_mfma_f32_16x16x128_f8f6f4 a[132:135], v[160:167], v[200:207], a[132:135]\n\t" \
  "ds_read_b128 v[128:131], v1 offset:0\n\t" \
  "ds_read_b128 v[132:135], v1 offset:8192\n\t" \
  "v_accvgpr_read_b32 v92, a115\n\t" \
  "v_accvgpr_read_b32 v81, a116\n\t" \
  "v_accvgpr_read_b32 v85, a117\n\t" \
  "v_mfma_f32_16x16x128_f8f6f4 a[136:139], v[160:167], v[208:215], a[136:139]\n\t" \
  "ds_read_b128 v[136:139], v1 offset:1024\n\t" \
  "ds_read_b128 v[140:143], v1 offset:9216\n\t" \
  "v_accvgpr_read_b32 v89, a118\n\t" \
  "v_accvgpr_read_b32 v93, a119\n\t" \
  "v_accvgpr_read_b32 v82, a120\n\t" \
  "v_mfma_f32_16x16x128_f8f6f4 a[140:143], v[160:167], v[216:223], a[140:143]\n\t" \
  "ds_read_b128 v[144:147], v1 offset:2048\n\t" \
  "ds_read_b128 v[148:151], v1 offset:10240\n\t" \
  "v_accvgpr_read_b32 v86, a121\n\t" \
  "v_accvgpr_read_b32 v90, a122\n\t" \
  "v_accvgpr_read_b32 v94, a123\n\t" \
  "v_mfma_f32_16x16x128_f8f6f4 a[172:175], v[168:175], v[216:223], a[172:175]\n\t" \
  "ds_read_b128 v[152:155], v1 offset:3072\n\t" \
  "ds_read_b128 v[156:159], v1 offset:11264\n\t" \
  "v_accvgpr_read_b32 v83, a124\n\t" \
  "v_accvgpr_read_b32 v87, a125\n\t" \
  "v_accvgpr_read_b32 v91, a126\n\t" \
  "v_mfma_f32_16x16x128_f8f6f4 a[168:171], v[168:175], v[208:215], a[168:171]\n\t" \
  "v_accvgpr_read_b32 v95, a127\n\t" \
  "v_mfma_f32_16x16x128_f8f6f4 a[164:167], v[168:175], v[200:207], a[164:167]\n\t" \
  "v_mfma_f32_16x16x128_f8f6f4 a[160:163], v[168:175], v[192:199], a[160:163]\n\t" \
  "s_add_u32 s53, s50, 16384\n\t" \
  "s_add_u32 m0, s48, 98304\n\t" \
  "v_mfma_f32_16x16x128_f8f6f4 a[192:195], v[176:183], v[192:199], a[192:195]\n\t" \
  "buffer_load_dwordx4 v0, s[36:39], s53 offen sc0 sc1 lds\n\t" \
  "v_mfma_f32_16x16x128_f8f6f4 a[196:199], v[176:183], v[200:207], a[196:199]\n\t" \
  "v_mfma_f32_16x16x128_f8f6f4 a[200:203], v[176:183], v[208:215], a[200:203]\n\t" \
  "buffer_load_dwordx4 v0, s[36:39], s53 offen offset:1024 sc0 sc1 lds\n\t" \
  "v_mfma_f32_16x16x128_f8f6f4 a[204:207], v[176:183], v[216:223], a[204:207]\n\t" \
  "v_mfma_f32_16x16x128_f8f6f4 a[236:239], v[184:191], v[216:223], a[236:239]\n\t" \
  "buffer_load_dwordx4 v0, s[36:39], s53 offen offset:2048 sc0 sc1 lds\n\t" \
  "v_mfma_f32_16x16x128_f8f6f4 a[232:235], v[184:191], v[208:215], a[232:235]\n\t" \
  "v_mfma_f32_16x16x128_f8f6f4 a[228:231], v[184:191], v[200:207], a[228:231]\n\t" \
  "buffer_load_dwordx4 v0, s[36:39], s53 offen offset:3072 sc0 sc1 lds\n\t" \
  "v_mfma_f32_16x16x128_f8f6f4 a[224:227], v[184:191], v[192:199], a[224:227]\n\t" \
  "s_waitcnt lgkmcnt(0)\n\t" \
  "v_mfma_f32_16x16x128_f8f6f4 a[144:147], v[160:167], v[224:231], a[144:147]\n\t" \
  "v_mfma_f32_16x16x128_f8f6f4 a[148:151], v[160:167], v[232:239], a[148:151]\n\t" \
  "ds_read_b128 v[192:195], v3 offset:16384\n\t" \
  "ds_read_b128 v[196:199], v3 offset:24576\n\t" \
  "v_mfma_f32_16x16x128_f8f6f4 a[152:155], v[160:167], v[240:247], a[152:155]\n\t" \
  "ds_read_b128 v[200:203], v3 offset:17408\n\t" \
  "ds_read_b128 v[204:207], v3 offset:25600\n\t" \
  "v_mfma_f32_16x16x128_f8f6f4 a[156:159], v[160:167], v[248:255], a[156:159]\n\t" \
  "ds_read_b128 v[208:211], v3 offset:18432\n\t" \
  "ds_read_b128 v[212:215], v3 offset:26624\n\t" \
  "v_mfma_f32_16x16x128_f8f6f4 a[188:191], v[168:175], v[248:255], a[188:191]\n\t" \
  "ds_read_b128 v[216:219], v3 offset:19456\n\t" \
  "ds_read_b128 v[220:223], v3 offset:27648\n\t" \
  "v_mfma_f32_16x16x128_f8f6f4 a[184:187], v[168:175], v[240:247], a[184:187]\n\t" \
  "v_mfma_f32_16x16x128_f8f6f4 a[180:183], v[168:175], v[232:239], a[180:183]\n\t" \
  "v_mfma_f32_16x16x128_f8f6f4 a[176:179], v[168:175], v[224:231], a[176:179]\n\t" \
  "s_add_u32 s53, s49, 49152\n\t" \
  "s_add_u32 m0, s48, 114688\n\t" \
  "v_mfma_f32_16x16x128_f8f6f4 a[208:211], v[176:183], v[224:231], a[208:211]\n\t" \
  "buffer_load_dwordx4 v0, s[36:39], s53 offen sc0 sc1 lds\n\t" \
  "v_mfma_f32_16x16x128_f8f6f4 a[212:215], v[176:183], v[232:239], a[212:215]\n\t" \
  "v_mfma_f32_16x16x128_f8f6f4 a[216:219], v[176:183], v[240:247], a[216:219]\n\t" \
  "buffer_load_dwordx4 v0, s[36:39], s53 offen offset:1024 sc0 sc1 lds\n\t" \
  "v_mfma_f32_16x16x128_f8f6f4 a[220:223], v[176:183], v[248:255], a[220:223]\n\t" \
  "v_mfma_f32_16x16x128_f8f6f4 a[252:255], v[184:191], v[248:255], a[252:255]\n\t" \
  "buffer_load_dwordx4 v0, s[36:39], s53 offen offset:2048 sc0 sc1 lds\n\t" \
  "v_mfma_f32_16x16x128_f8f6f4 a[248:251], v[184:191], v[240:247], a[248:251]\n\t" \
  "v_mfma_f32_16x16x128_f8f6f4 a[244:247], v[184:191], v[232:239], a[244:247]\n\t" \
  "buffer_load_dwordx4 v0, s[36:39], s53 offen offset:3072 sc0 sc1 lds\n\t" \
  "v_mfma_f32_16x16x128_f8f6f4 a[240:243], v[184:191], v[224:231], a[240:243]\n\t" \
  "s_add_u32 s49, s49, 65536\n\t" \
  "s_add_u32 s50, s50, 32768\n\t" \
  "s_add_u32 s51, s51, 32768\n\t" \
  "s_sub_u32 s72, s72, 1\n\t" \
  "L_WO_%=:\n\t" \
  "s_waitcnt lgkmcnt(0)\n\t" \
  "v_mfma_f32_16x16x128_f8f6f4 a[0:3], v[128:135], v[192:199], a[0:3]\n\t" \
  "s_waitcnt vmcnt(16)\n\t" \
  "s_barrier\n\t" \
  "v_mfma_f32_16x16x128_f8f6f4 a[4:7], v[128:135], v[200:207], a[4:7]\n\t" \
  "ds_read_b128 v[224:227], v3 offset:32768\n\t" \
  "ds_read_b128 v[228:231], v3 offset:40960\n\t" \
  "s_mov_b32 s68, s67\n\t" \
  "s_mov_b32 s67, s69\n\t" \
  "v_mfma_f32_16x16x128_f8f6f4 a[8:11], v[128:135], v[208:215], a[8:11]\n\t" \
  "ds_read_b128 v[232:235], v3 offset:33792\n\t" \
  "ds_read_b128 v[236:239], v3 offset:41984\n\t" \
  "s_cmp_lt_u32 s77, 2\n\t" \
  "s_cselect_b32 s42, 0, s74\n\t" \
  "v_accvgpr_read_b32 v96, a144\n\t" \
  "v_mfma_f32_16x16x128_f8f6f4 a[12:15], v[128:135], v[216:223], a[12:15]\n\t" \
  "ds_read_b128 v[240:243], v3 offset:34816\n\t" \
  "ds_read_b128 v[244:247], v3 offset:43008\n\t" \
  "s_add_u32 s77, s77, 1\n\t" \
  "v_accvgpr_read_b32 v100, a145\n\t" \
  "v_accvgpr_read_b32 v104, a146\n\t" \
  "v_mfma_f32_16x16x128_f8f6f4 a[44:47], v[136:143], v[216:223], a[44:47]\n\t" \
  "ds_read_b128 v[248:251], v3 offset:35840\n\t" \
  "ds_read_b128 v[252:255], v3 offset:44032\n\t" \
  "s_cmp_eq_u32 s72, 1\n\t" \
  "s_cselect_b32 s61, 1024, 0\n\t" \
  "v_accvgpr_read_b32 v108, a147\n\t" \
  "v_mfma_f32_16x16x128_f8f6f4 a[40:43], v[136:143], v[208:215], a[40:43]\n\t" \
  "v_accvgpr_read_b32 v97, a148\n\t" \
  "v_accvgpr_read_b32 v101, a149\n\t" \
  "v_accvgpr_read_b32 v105, a150\n\t" \
  "v_accvgpr_read_b32 v109, a151\n\t" \
  "v_mfma_f32_16x16x128_f8f6f4 a[36:39], v[136:143], v[200:207], a[36:39]\n\t" \
  "v_accvgpr_read_b32 v98, a152\n\t" \
  "v_accvgpr_read_b32 v102, a153\n\t" \
  "v_accvgpr_read_b32 v106, a154\n\t" \
  "v_accvgpr_read_b32 v110, a155\n\t" \
  "v_mfma_f32_16x16x128_f8f6f4 a[32:35], v[136:143], v[192:199], a[32:35]\n\t" \
  "s_mov_b32 s53, s49\n\t" \
  "s_mov_b32 m0, s48\n\t" \
  "v_accvgpr_read_b32 v99, a156\n\t" \
  "v_accvgpr_read_b32 v103, a157\n\t" \
  "v_accvgpr_read_b32 v107, a158\n\t" \
  "v_mfma_f32_16x16x128_f8f6f4 a[64:67], v[144:151], v[192:199], a[64:67]\n\t" \
  "buffer_load_dwordx4 v0, s[36:39], s53 offen sc0 sc1 lds\n\t" \
  "v_accvgpr_read_b32 v111, a159\n\t" \
  "v_accvgpr_read_b32 v112, a176\n\t" \
  "v_accvgpr_read_b32 v116, a177\n\t" \
  "v_mfma_f32_16x16x128_f8f6f4 a[68:71], v[144:151], v[200:207], a[68:71]\n\t" \
  "v_accvgpr_read_b32 v120, a178\n\t" \
  "v_accvgpr_read_b32 v124, a179\n\t" \
  "v_accvgpr_read_b32 v113, a180\n\t" \
  "v_accvgpr_read_b32 v117, a181\n\t" \
  "v_mfma_f32_16x16x128_f8f6f4 a[72:75], v[144:151], v[208:215], a[72:75]\n\t" \
  "buffer_load_dwordx4 v0, s[36:39], s53 offen offset:1024 sc0 sc1 lds\n\t" \
  "v_accvgpr_read_b32 v121, a182\n\t" \
  "v_accvgpr_read_b32 v125, a183\n\t" \
  "v_accvgpr_read_b32 v114, a184\n\t" \
  "v_mfma_f32_16x16x128_f8f6f4 a[76:79], v[144:151], v[216:223], a[76:79]\n\t" \
  "v_accvgpr_read_b32 v118, a185\n\t" \
  "v_accvgpr_read_b32 v122, a186\n\t" \
  "v_accvgpr_read_b32 v126, a187\n\t" \
  "v_accvgpr_read_b32 v115, a188\n\t" \
  "v_mfma_f32_16x16x128_f8f6f4 a[108:111], v[152:159], v[216:223], a[108:111]\n\t" \
  "buffer_load_dwordx4 v0, s[36:39], s53 offen offset:2048 sc0 sc1 lds\n\t" \
  "v_accvgpr_read_b32 v119, a189\n\t" \
  "v_accvgpr_read_b32 v123, a190\n\t" \
  "v_accvgpr_read_b32 v127, a191\n\t" \
  "v_mfma_f32_16x16x128_f8f6f4 a[104:107], v[152:159], v[208:215], a[104:107]\n\t" \
  "ds_write_b128 v6, a[208:211] offset:0\n\t" \
  "ds_write_b128 v6, a[212:215] offset:1024\n\t" \
  "ds_write_b128 v6, a[216:219] offset:2048\n\t" \
  "ds_write_b128 v6, a[220:223] offset:3072\n\t" \
  "v_mfma_f32_16x16x128_f8f6f4 a[100:103], v[152:159], v[200:207], a[100:103]\n\t" \
  "buffer_load_dwordx4 v0, s[36:39], s53 offen offset:3072 sc0 sc1 lds\n\t" \
  "ds_write_b128 v6, a[240:243] offset:4096\n\t" \
  "ds_write_b128 v6, a[244:247] offset:5120\n\t" \
  "ds_write_b128 v6, a[248:251] offset:6144\n\t" \
  "v_mfma_f32_16x16x128_f8f6f4 a[96:99], v[152:159], v[192:199], a[96:99]\n\t" \
  "ds_write_b128 v6, a[252:255] offset:7168\n\t" \
  "s_waitcnt lgkmcnt(0)\n\t" \
  "v_mfma_f32_16x16x128_f8f6f4 a[16:19], v[128:135], v[224:231], 0\n\t" \
  "v_mfma_f32_16x16x128_f8f6f4 a[20:23], v[128:135], v[232:239], 0\n\t" \
  "ds_read_b128 v[160:163], v1 offset:49152\n\t" \
  "ds_read_b128 v[164:167], v1 offset:57344\n\t" \
  "v_mfma_f32_16x16x128_f8f6f4 a[24:27], v[128:135], v[240:247], 0\n\t" \
  "ds_read_b128 v[168:171], v1 offset:50176\n\t" \
  "ds_read_b128 v[172:175], v1 offset:58368\n\t" \
  "v_mfma_f32_16x16x128_f8f6f4 a[28:31], v[128:135], v[248:255], 0\n\t" \
  "ds_read_b128 v[176:179], v1 offset:51200\n\t" \
  "ds_read_b128 v[180:183], v1 offset:59392\n\t" \
  "v_mfma_f32_16x16x128_f8f6f4 a[60:63], v[136:143], v[248:255], 0\n\t" \
  "ds_read_b128 v[184:187], v1 offset:52224\n\t" \
  "ds_read_b128 v[188:191], v1 offset:60416\n\t" \
  "v_mfma_f32_16x16x128_f8f6f4 a[56:59], v[136:143], v[240:247], 0\n\t" \
  "v_mfma_f32_16x16x128_f8f6f4 a[52:55], v[136:143], v[232:239], 0\n\t" \
  "v_mfma_f32_16x16x128_f8f6f4 a[48:51], v[136:143], v[224:231], 0\n\t" \
  "s_mov_b32 s53, s50\n\t" \
  "s_add_u32 m0, s48, 16384\n\t" \
  "v_mfma_f32_16x16x128_f8f6f4 a[80:83], v[144:151], v[224:231], 0\n\t" \
  "buffer_load_dwordx4 v0, s[36:39], s53 offen sc0 sc1 lds\n\t" \
  "v_mfma_f32_16x16x128_f8f6f4 a[84:87], v[144:151], v[232:239], 0\n\t" \
  "v_mfma_f32_16x16x128_f8f6f4 a[88:91], v[144:151], v[240:247], 0\n\t" \
  "buffer_load_dwordx4 v0, s[36:39], s53 offen offset:1024 sc0 sc1 lds\n\t" \
  "v_mfma_f32_16x16x128_f8f6f4 a[92:95], v[144:151], v[248:255], 0\n\t" \
  "v_mfma_f32_16x16x128_f8f6f4 a[124:127], v[152:159], v[248:255], 0\n\t" \
  "buffer_load_dwordx4 v0, s[36:39], s53 offen offset:2048 sc0 sc1 lds\n\t" \
  "v_mfma_f32_16x16x128_f8f6f4 a[120:123], v[152:159], v[240:247], 0\n\t" \
  "v_mfma_f32_16x16x128_f8f6f4 a[116:119], v[152:159], v[232:239], 0\n\t" \
  "buffer_load_dwordx4 v0, s[36:39], s53 offen offset:3072 sc0 sc1 lds\n\t" \
  "v_mfma_f32_16x16x128_f8f6f4 a[112:115], v[152:159], v[224:231], 0\n\t" \
  "s_waitcnt lgkmcnt(0)\n\t" \
  "v_mfma_f32_16x16x128_f8f6f4 a[144:147], v[160:167], v[224:231], 0\n\t" \
  "s_waitcnt vmcnt(16)\n\t" \
  "s_barrier\n\t" \
  "v_mfma_f32_16x16x128_f8f6f4 a[148:151], v[160:167], v[232:239], 0\n\t" \
  "ds_read_b128 v[128:131], v2 offset:0\n\t" \
  "ds_read_b128 v[132:135], v2 offset:8192\n\t" \
  "v_mfma_f32_16x16x128_f8f6f4 a[152:155], v[160:167], v[240:247], 0\n\t" \
  "ds_read_b128 v[136:139], v2 offset:1024\n\t" \
  "ds_read_b128 v[140:143], v2 offset:9216\n\t" \
  "v_mfma_f32_16x16x128_f8f6f4 a[156:159], v[160:167], v[248:255], 0\n\t" \
  "ds_read_b128 v[144:147], v2 offset:2048\n\t" \
  "ds_read_b128 v[148:151], v2 offset:10240\n\t" \
  "v_mfma_f32_16x16x128_f8f6f4 a[188:191], v[168:175], v[248:255], 0\n\t" \
  "ds_read_b128 v[152:155], v2 offset:3072\n\t" \
  "ds_read_b128 v[156:159], v2 offset:11264\n\t" \
  "v_mfma_f32_16x16x128_f8f6f4 a[184:187], v[168:175], v[240:247], 0\n\t" \
  "v_mfma_f32_16x16x128_f8f6f4 a[180:183], v[168:175], v[232:239], 0\n\t" \
  "v_mfma_f32_16x16x128_f8f6f4 a[176:179], v[168:175], v[224:231], 0\n\t" \
  "s_mov_b32 s53, s51\n\t" \
  "s_add_u32 m0, s48, 32768\n\t" \
  "v_mfma_f32_16x16x128_f8f6f4 a[208:211], v[176:183], v[224:231], 0\n\t" \
  "buffer_load_dwordx4 v0, s[36:39], s53 offen sc0 sc1 lds\n\t" \
  "v_mfma_f32_16x16x128_f8f6f4 a[212:215], v[176:183], v[232:239], 0\n\t" \
  "v_mfma_f32_16x16x128_f8f6f4 a[216:219], v[176:183], v[240:247], 0\n\t" \
  "buffer_load_dwordx4 v0, s[36:39], s53 offen offset:1024 sc0 sc1 lds\n\t" \
  "v_mfma_f32_16x16x128_f8f6f4 a[220:223], v[176:183], v[248:255], 0\n\t" \
  "v_mfma_f32_16x16x128_f8f6f4 a[252:255], v[184:191], v[248:255], 0\n\t" \
  "buffer_load_dwordx4 v0, s[36:39], s53 offen offset:2048 sc0 sc1 lds\n\t" \
  "v_mfma_f32_16x16x128_f8f6f4 a[248:251], v[184:191], v[240:247], 0\n\t" \
  "v_mfma_f32_16x16x128_f8f6f4 a[244:247], v[184:191], v[232:239], 0\n\t" \
  "buffer_load_dwordx4 v0, s[36:39], s53 offen offset:3072 sc0 sc1 lds\n\t" \
  "v_mfma_f32_16x16x128_f8f6f4 a[240:243], v[184:191], v[224:231], 0\n\t" \
  "s_waitcnt lgkmcnt(0)\n\t" \
  "v_mfma_f32_16x16x128_f8f6f4 a[128:131], v[160:167], v[192:199], a[128:131]\n\t" \
  "v_mul_f32_e32 v32, s56, v32\n\t" \
  "v_mul_f32_e32 v33, s56, v33\n\t" \
  "v_mul_f32_e32 v34, s56, v34\n\t" \
  "v_mul_f32_e32 v35, s56, v35\n\t" \
  "v_mfma_f32_16x16x128_f8f6f4 a[132:135], v[160:167], v[200:207], a[132:135]\n\t" \
  "ds_read_b128 v[224:227], v4 offset:16384\n\t" \
  "ds_read_b128 v[228:231], v4 offset:24576\n\t" \
  "v_mfma_f32_16x16x128_f8f6f4 a[136:139], v[160:167], v[208:215], a[136:139]\n\t" \
  "ds_read_b128 v[232:235], v4 offset:17408\n\t" \
  "ds_read_b128 v[236:239], v4 offset:25600\n\t" \
  "v_mfma_f32_16x16x128_f8f6f4 a[140:143], v[160:167], v[216:223], a[140:143]\n\t" \
  "ds_read_b128 v[240:243], v4 offset:18432\n\t" \
  "ds_read_b128 v[244:247], v4 offset:26624\n\t" \
  "v_mfma_f32_16x16x128_f8f6f4 a[172:175], v[168:175], v[216:223], a[172:175]\n\t" \
  "ds_read_b128 v[248:251], v4 offset:19456\n\t" \
  "ds_read_b128 v[252:255], v4 offset:27648\n\t" \
  "v_mfma_f32_16x16x128_f8f6f4 a[168:171], v[168:175], v[208:215], a[168:171]\n\t" \
  "v_min_f32_e32 v12, s60, v32\n\t" \
  "v_min_f32_e32 v13, s60, v33\n\t" \
  "v_min_f32_e32 v14, s60, v34\n\t" \
  "v_min_f32_e32 v15, s60, v35\n\t" \
  "v_mfma_f32_16x16x128_f8f6f4 a[164:167], v[168:175], v[200:207], a[164:167]\n\t" \
  "v_exp_f32_e32 v12, v12\n\t" \
  "v_exp_f32_e32 v13, v13\n\t" \
  "v_mfma_f32_16x16x128_f8f6f4 a[160:163], v[168:175], v[192:199], a[160:163]\n\t" \
  "s_add_u32 s53, s49, 16384\n\t" \
  "s_add_u32 m0, s48, 49152\n\t" \
  "v_mfma_f32_16x16x128_f8f6f4 a[192:195], v[176:183], v[192:199], a[192:195]\n\t" \
  "buffer_load_dwordx4 v0, s[36:39], s53 offen sc0 sc1 lds\n\t" \
  "v_mfma_f32_16x16x128_f8f6f4 a[196:199], v[176:183], v[200:207], a[196:199]\n\t" \
  "v_exp_f32_e32 v14, v14\n\t" \
  "v_exp_f32_e32 v15, v15\n\t" \
  "v_mfma_f32_16x16x128_f8f6f4 a[200:203], v[176:183], v[208:215], a[200:203]\n\t" \
  "buffer_load_dwordx4 v0, s[36:39], s53 offen offset:1024 sc0 sc1 lds\n\t" \
  "v_mfma_f32_16x16x128_f8f6f4 a[204:207], v[176:183], v[216:223], a[204:207]\n\t" \
  "v_fma_f32 v12, v12, s58, s58\n\t" \
  "v_fma_f32 v13, v13, s58, s58\n\t" \
  "v_fma_f32 v14, v14, s58, s58\n\t" \
  "v_fma_f32 v15, v15, s58, s58\n\t" \
  "v_mfma_f32_16x16x128_f8f6f4 a[236:239], v[184:191], v[216:223], a[236:239]\n\t" \
  "buffer_load_dwordx4 v0, s[36:39], s53 offen offset:2048 sc0 sc1 lds\n\t" \
  "v_mfma_f32_16x16x128_f8f6f4 a[232:235], v[184:191], v[208:215], a[232:235]\n\t" \
  "v_rcp_f32_e32 v12, v12\n\t" \
  "v_rcp_f32_e32 v13, v13\n\t" \
  "v_mfma_f32_16x16x128_f8f6f4 a[228:231], v[184:191], v[200:207], a[228:231]\n\t" \
  "buffer_load_dwordx4 v0, s[36:39], s53 offen offset:3072 sc0 sc1 lds\n\t" \
  "v_mfma_f32_16x16x128_f8f6f4 a[224:227], v[184:191], v[192:199], a[224:227]\n\t" \
  "v_rcp_f32_e32 v14, v14\n\t" \
  "v_rcp_f32_e32 v15, v15\n\t" \
  "s_waitcnt lgkmcnt(0)\n\t" \
  "v_mfma_f32_16x16x128_f8f6f4 a[16:19], v[128:135], v[224:231], a[16:19]\n\t" \
  "s_waitcnt vmcnt(16)\n\t" \
  "s_barrier\n\t" \
  "v_mfma_f32_16x16x128_f8f6f4 a[20:23], v[128:135], v[232:239], a[20:23]\n\t" \
  "ds_read_b128 v[192:195], v4 offset:32768\n\t" \
  "ds_read_b128 v[196:199], v4 offset:40960\n\t" \
  "v_mfma_f32_16x16x128_f8f6f4 a[24:27], v[128:135], v[240:247], a[24:27]\n\t" \
  "ds_read_b128 v[200:203], v4 offset:33792\n\t" \
  "ds_read_b128 v[204:207], v4 offset:41984\n\t" \
  "v_mfma_f32_16x16x128_f8f6f4 a[28:31], v[128:135], v[248:255], a[28:31]\n\t" \
  "ds_read_b128 v[208:211], v4 offset:34816\n\t" \
  "ds_read_b128 v[212:215], v4 offset:43008\n\t" \
  "v_mfma_f32_16x16x128_f8f6f4 a[60:63], v[136:143], v[248:255], a[60:63]\n\t" \
  "ds_read_b128 v[216:219], v4 offset:35840\n\t" \
  "ds_read_b128 v[220:223], v4 offset:44032\n\t" \
  "v_mfma_f32_16x16x128_f8f6f4 a[56:59], v[136:143], v[240:247], a[56:59]\n\t" \
  "v_mul_f32_e32 v32, v32, v12\n\t" \
  "v_mul_f32_e32 v33, v33, v13\n\t" \
  "v_mul_f32_e32 v34, v34, v14\n\t" \
  "v_mul_f32_e32 v35, v35, v15\n\t" \
  "v_mfma_f32_16x16x128_f8f6f4 a[52:55], v[136:143], v[232:239], a[52:55]\n\t" \
  "s_xor_b32 s91, s68, 0\n\t" \
  "buffer_store_dwordx2 v[32:33], v5, s[40:43], s91 offen nt\n\t" \
  "v_mfma_f32_16x16x128_f8f6f4 a[48:51], v[136:143], v[224:231], a[48:51]\n\t" \
  "s_add_u32 s53, s49, 32768\n\t" \
  "s_add_u32 m0, s48, 65536\n\t" \
  "v_mfma_f32_16x16x128_f8f6f4 a[80:83], v[144:151], v[224:231], a[80:83]\n\t" \
  "buffer_load_dwordx4 v0, s[36:39], s53 offen sc0 sc1 lds\n\t" \
  "v_mfma_f32_16x16x128_f8f6f4 a[84:87], v[144:151], v[232:239], a[84:87]\n\t" \
  "buffer_load_dword v7, v8, s[44:47], s76 offen sc0 sc1\n\t" \
  "v_mfma_f32_16x16x128_f8f6f4 a[88:91], v[144:151], v[240:247], a[88:91]\n\t" \
  "buffer_load_dwordx4 v0, s[36:39], s53 offen offset:1024 sc0 sc1 lds\n\t" \
  "v_mfma_f32_16x16x128_f8f6f4 a[92:95], v[144:151], v[248:255], a[92:95]\n\t" \
  "buffer_store_dwordx2 v[34:35], v5, s[40:43], s91 offen offset:128 nt\n\t" \
  "v_mfma_f32_16x16x128_f8f6f4 a[124:127], v[152:159], v[248:255], a[124:127]\n\t" \
  "buffer_load_dwordx4 v0, s[36:39], s53 offen offset:2048 sc0 sc1 lds\n\t" \
  "v_mfma_f32_16x16x128_f8f6f4 a[120:123], v[152:159], v[240:247], a[120:123]\n\t" \
  "v_mul_f32_e32 v36, s56, v36\n\t" \
  "v_mul_f32_e32 v37, s56, v37\n\t" \
  "v_mul_f32_e32 v38, s56, v38\n\t" \
  "v_mul_f32_e32 v39, s56, v39\n\t" \
  "v_mfma_f32_16x16x128_f8f6f4 a[116:119], v[152:159], v[232:239], a[116:119]\n\t" \
  "buffer_load_dwordx4 v0, s[36:39], s53 offen offset:3072 sc0 sc1 lds\n\t" \
  "v_mfma_f32_16x16x128_f8f6f4 a[112:115], v[152:159], v[224:231], a[112:115]\n\t" \
  "v_min_f32_e32 v16, s60, v36\n\t" \
  "v_min_f32_e32 v17, s60, v37\n\t" \
  "v_min_f32_e32 v18, s60, v38\n\t" \
  "v_min_f32_e32 v19, s60, v39\n\t" \
  "s_waitcnt lgkmcnt(0)\n\t" \
  "v_mfma_f32_16x16x128_f8f6f4 a[0:3], v[128:135], v[192:199], a[0:3]\n\t" \
  "v_exp_f32_e32 v16, v16\n\t" \
  "v_exp_f32_e32 v17, v17\n\t" \
  "v_mfma_f32_16x16x128_f8f6f4 a[4:7], v[128:135], v[200:207], a[4:7]\n\t" \
  "ds_read_b128 v[160:163], v2 offset:49152\n\t" \
  "ds_read_b128 v[164:167], v2 offset:57344\n\t" \
  "v_mfma_f32_16x16x128_f8f6f4 a[8:11], v[128:135], v[208:215], a[8:11]\n\t" \
  "ds_read_b128 v[168:171], v2 offset:50176\n\t" \
  "ds_read_b128 v[172:175], v2 offset:58368\n\t" \
  "v_mfma_f32_16x16x128_f8f6f4 a[12:15], v[128:135], v[216:223], a[12:15]\n\t" \
  "ds_read_b128 v[176:179], v2 offset:51200\n\t" \
  "ds_read_b128 v[180:183], v2 offset:59392\n\t" \
  "v_mfma_f32_16x16x128_f8f6f4 a[44:47], v[136:143], v[216:223], a[44:47]\n\t" \
  "ds_read_b128 v[184:187], v2 offset:52224\n\t" \
  "ds_read_b128 v[188:191], v2 offset:60416\n\t" \
  "v_mfma_f32_16x16x128_f8f6f4 a[40:43], v[136:143], v[208:215], a[40:43]\n\t" \
  "v_exp_f32_e32 v18, v18\n\t" \
  "v_exp_f32_e32 v19, v19\n\t" \
  "v_mfma_f32_16x16x128_f8f6f4 a[36:39], v[136:143], v[200:207], a[36:39]\n\t" \
  "v_fma_f32 v16, v16, s58, s58\n\t" \
  "v_fma_f32 v17, v17, s58, s58\n\t" \
  "v_fma_f32 v18, v18, s58, s58\n\t" \
  "v_fma_f32 v19, v19, s58, s58\n\t" \
  "v_mfma_f32_16x16x128_f8f6f4 a[32:35], v[136:143], v[192:199], a[32:35]\n\t" \
  "s_add_u32 s53, s51, 16384\n\t" \
  "s_add_u32 m0, s48, 81920\n\t" \
  "v_mfma_f32_16x16x128_f8f6f4 a[64:67], v[144:151], v[192:199], a[64:67]\n\t" \
  "buffer_load_dwordx4 v0, s[36:39], s53 offen sc0 sc1 lds\n\t" \
  "v_mfma_f32_16x16x128_f8f6f4 a[68:71], v[144:151], v[200:207], a[68:71]\n\t" \
  "v_rcp_f32_e32 v16, v16\n\t" \
  "v_rcp_f32_e32 v17, v17\n\t" \
  "v_mfma_f32_16x16x128_f8f6f4 a[72:75], v[144:151], v[208:215], a[72:75]\n\t" \
  "buffer_load_dwordx4 v0, s[36:39], s53 offen offset:1024 sc0 sc1 lds\n\t" \
  "v_mfma_f32_16x16x128_f8f6f4 a[76:79], v[144:151], v[216:223], a[76:79]\n\t" \
  "v_rcp_f32_e32 v18, v18\n\t" \
  "v_rcp_f32_e32 v19, v19\n\t" \
  "v_mfma_f32_16x16x128_f8f6f4 a[108:111], v[152:159], v[216:223], a[108:111]\n\t" \
  "buffer_load_dwordx4 v0, s[36:39], s53 offen offset:2048 sc0 sc1 lds\n\t" \
  "v_mfma_f32_16x16x128_f8f6f4 a[104:107], v[152:159], v[208:215], a[104:107]\n\t" \
  "v_mul_f32_e32 v36, v36, v16\n\t" \
  "v_mul_f32_e32 v37, v37, v17\n\t" \
  "v_mul_f32_e32 v38, v38, v18\n\t" \
  "v_mul_f32_e32 v39, v39, v19\n\t" \
  "v_mfma_f32_16x16x128_f8f6f4 a[100:103], v[152:159], v[200:207], a[100:103]\n\t" \
  "buffer_load_dwordx4 v0, s[36:39], s53 offen offset:3072 sc0 sc1 lds\n\t" \
  "v_mfma_f32_16x16x128_f8f6f4 a[96:99], v[152:159], v[192:199], a[96:99]\n\t" \
  "s_xor_b32 s91, s68, 65536\n\t" \
  "buffer_store_dwordx2 v[36:37], v5, s[40:43], s91 offen nt\n\t" \
  "s_waitcnt lgkmcnt(0)\n\t" \
  "v_mfma_f32_16x16x128_f8f6f4 a[128:131], v[160:167], v[192:199], a[128:131]\n\t" \
  "s_waitcnt vmcnt(20)\n\t" \
  "s_barrier\n\t" \
  "v_mfma_f32_16x16x128_f8f6f4 a[132:135], v[160:167], v[200:207], a[132:135]\n\t" \
  "ds_read_b128 v[128:131], v1 offset:0\n\t" \
  "ds_read_b128 v[132:135], v1 offset:8192\n\t" \
  "v_mfma_f32_16x16x128_f8f6f4 a[136:139], v[160:167], v[208:215], a[136:139]\n\t" \
  "ds_read_b128 v[136:139], v1 offset:1024\n\t" \
  "ds_read_b128 v[140:143], v1 offset:9216\n\t" \
  "v_mfma_f32_16x16x128_f8f6f4 a[140:143], v[160:167], v[216:223], a[140:143]\n\t" \
  "ds_read_b128 v[144:147], v1 offset:2048\n\t" \
  "ds_read_b128 v[148:151], v1 offset:10240\n\t" \
  "v_mfma_f32_16x16x128_f8f6f4 a[172:175], v[168:175], v[216:223], a[172:175]\n\t" \
  "ds_read_b128 v[152:155], v1 offset:3072\n\t" \
  "ds_read_b128 v[156:159], v1 offset:11264\n\t" \
  "v_mfma_f32_16x16x128_f8f6f4 a[168:171], v[168:175], v[208:215], a[168:171]\n\t" \
  "buffer_store_dwordx2 v[38:39], v5, s[40:43], s91 offen offset:128 nt\n\t" \
  "v_mfma_f32_16x16x128_f8f6f4 a[164:167], v[168:175], v[200:207], a[164:167]\n\t" \
  "v_mul_f32_e32 v40, s56, v40\n\t" \
  "v_mul_f32_e32 v41, s56, v41\n\t" \
  "v_mul_f32_e32 v42, s56, v42\n\t" \
  "v_mul_f32_e32 v43, s56, v43\n\t" \
  "v_mfma_f32_16x16x128_f8f6f4 a[160:163], v[168:175], v[192:199], a[160:163]\n\t" \
  "s_add_u32 s53, s50, 16384\n\t" \
  "s_add_u32 m0, s48, 98304\n\t" \
  "v_mfma_f32_16x16x128_f8f6f4 a[192:195], v[176:183], v[192:199], a[192:195]\n\t" \
  "buffer_load_dwordx4 v0, s[36:39], s53 offen sc0 sc1 lds\n\t" \
  "v_mfma_f32_16x16x128_f8f6f4 a[196:199], v[176:183], v[200:207], a[196:199]\n\t" \
  "v_min_f32_e32 v20, s60, v40\n\t" \
  "v_min_f32_e32 v21, s60, v41\n\t" \
  "v_min_f32_e32 v22, s60, v42\n\t" \
  "v_min_f32_e32 v23, s60, v43\n\t" \
  "v_mfma_f32_16x16x128_f8f6f4 a[200:203], v[176:183], v[208:215], a[200:203]\n\t" \
  "buffer_load_dwordx4 v0, s[36:39], s53 offen offset:1024 sc0 sc1 lds\n\t" \
  "v_mfma_f32_16x16x128_f8f6f4 a[204:207], v[176:183], v[216:223], a[204:207]\n\t" \
  "v_exp_f32_e32 v20, v20\n\t" \
  "v_exp_f32_e32 v21, v21\n\t" \
  "v_mfma_f32_16x16x128_f8f6f4 a[236:239], v[184:191], v[216:223], a[236:239]\n\t" \
  "buffer_load_dwordx4 v0, s[36:39], s53 offen offset:2048 sc0 sc1 lds\n\t" \
  "v_mfma_f32_16x16x128_f8f6f4 a[232:235], v[184:191], v[208:215], a[232:235]\n\t" \
  "v_exp_f32_e32 v22, v22\n\t" \
  "v_exp_f32_e32 v23, v23\n\t" \
  "v_mfma_f32_16x16x128_f8f6f4 a[228:231], v[184:191], v[200:207], a[228:231]\n\t" \
  "buffer_load_dwordx4 v0, s[36:39], s53 offen offset:3072 sc0 sc1 lds\n\t" \
  "v_mfma_f32_16x16x128_f8f6f4 a[224:227], v[184:191], v[192:199], a[224:227]\n\t" \
  "v_fma_f32 v20, v20, s58, s58\n\t" \
  "v_fma_f32 v21, v21, s58, s58\n\t" \
  "v_fma_f32 v22, v22, s58, s58\n\t" \
  "v_fma_f32 v23, v23, s58, s58\n\t" \
  "s_waitcnt lgkmcnt(0)\n\t" \
  "v_mfma_f32_16x16x128_f8f6f4 a[144:147], v[160:167], v[224:231], a[144:147]\n\t" \
  "v_mfma_f32_16x16x128_f8f6f4 a[148:151], v[160:167], v[232:239], a[148:151]\n\t" \
  "ds_read_b128 v[192:195], v3 offset:16384\n\t" \
  "ds_read_b128 v[196:199], v3 offset:24576\n\t" \
  "v_mfma_f32_16x16x128_f8f6f4 a[152:155], v[160:167], v[240:247], a[152:155]\n\t" \
  "ds_read_b128 v[200:203], v3 offset:17408\n\t" \
  "ds_read_b128 v[204:207], v3 offset:25600\n\t" \
  "v_mfma_f32_16x16x128_f8f6f4 a[156:159], v[160:167], v[248:255], a[156:159]\n\t" \
  "ds_read_b128 v[208:211], v3 offset:18432\n\t" \
  "ds_read_b128 v[212:215], v3 offset:26624\n\t" \
  "v_mfma_f32_16x16x128_f8f6f4 a[188:191], v[168:175], v[248:255], a[188:191]\n\t" \
  "ds_read_b128 v[216:219], v3 offset:19456\n\t" \
  "ds_read_b128 v[220:223], v3 offset:27648\n\t" \
  "v_mfma_f32_16x16x128_f8f6f4 a[184:187], v[168:175], v[240:247], a[184:187]\n\t" \
  "v_rcp_f32_e32 v20, v20\n\t" \
  "v_rcp_f32_e32 v21, v21\n\t" \
  "v_mfma_f32_16x16x128_f8f6f4 a[180:183], v[168:175], v[232:239], a[180:183]\n\t" \
  "v_rcp_f32_e32 v22, v22\n\t" \
  "v_rcp_f32_e32 v23, v23\n\t" \
  "v_mfma_f32_16x16x128_f8f6f4 a[176:179], v[168:175], v[224:231], a[176:179]\n\t" \
  "s_add_u32 s53, s49, 49152\n\t" \
  "s_add_u32 m0, s48, 114688\n\t" \
  "v_mfma_f32_16x16x128_f8f6f4 a[208:211], v[176:183], v[224:231], a[208:211]\n\t" \
  "buffer_load_dwordx4 v0, s[36:39], s53 offen sc0 sc1 lds\n\t" \
  "v_mfma_f32_16x16x128_f8f6f4 a[212:215], v[176:183], v[232:239], a[212:215]\n\t" \
  "v_mul_f32_e32 v40, v40, v20\n\t" \
  "v_mul_f32_e32 v41, v41, v21\n\t" \
  "v_mul_f32_e32 v42, v42, v22\n\t" \
  "v_mul_f32_e32 v43, v43, v23\n\t" \
  "v_mfma_f32_16x16x128_f8f6f4 a[216:219], v[176:183], v[240:247], a[216:219]\n\t" \
  "buffer_load_dwordx4 v0, s[36:39], s53 offen offset:1024 sc0 sc1 lds\n\t" \
  "v_mfma_f32_16x16x128_f8f6f4 a[220:223], v[176:183], v[248:255], a[220:223]\n\t" \
  "s_xor_b32 s91, s68, 131072\n\t" \
  "buffer_store_dwordx2 v[40:41], v5, s[40:43], s91 offen nt\n\t" \
  "v_mfma_f32_16x16x128_f8f6f4 a[252:255], v[184:191], v[248:255], a[252:255]\n\t" \
  "buffer_load_dwordx4 v0, s[36:39], s53 offen offset:2048 sc0 sc1 lds\n\t" \
  "v_mfma_f32_16x16x128_f8f6f4 a[248:251], v[184:191], v[240:247], a[248:251]\n\t" \
  "buffer_store_dwordx2 v[42:43], v5, s[40:43], s91 offen offset:128 nt\n\t" \
  "v_mfma_f32_16x16x128_f8f6f4 a[244:247], v[184:191], v[232:239], a[244:247]\n\t" \
  "buffer_load_dwordx4 v0, s[36:39], s53 offen offset:3072 sc0 sc1 lds\n\t" \
  "v_mfma_f32_16x16x128_f8f6f4 a[240:243], v[184:191], v[224:231], a[240:243]\n\t" \
  "s_add_u32 s49, s49, 65536\n\t" \
  "s_add_u32 s50, s50, 32768\n\t" \
  "s_add_u32 s51, s51, 32768\n\t" \
  "s_waitcnt lgkmcnt(0)\n\t" \
  "v_mfma_f32_16x16x128_f8f6f4 a[0:3], v[128:135], v[192:199], a[0:3]\n\t" \
  "s_waitcnt vmcnt(23)\n\t" \
  "s_barrier\n\t" \
  "v_mfma_f32_16x16x128_f8f6f4 a[4:7], v[128:135], v[200:207], a[4:7]\n\t" \
  "ds_read_b128 v[224:227], v3 offset:32768\n\t" \
  "ds_read_b128 v[228:231], v3 offset:40960\n\t" \
  "v_mfma_f32_16x16x128_f8f6f4 a[8:11], v[128:135], v[208:215], a[8:11]\n\t" \
  "ds_read_b128 v[232:235], v3 offset:33792\n\t" \
  "ds_read_b128 v[236:239], v3 offset:41984\n\t" \
  "v_mfma_f32_16x16x128_f8f6f4 a[12:15], v[128:135], v[216:223], a[12:15]\n\t" \
  "ds_read_b128 v[240:243], v3 offset:34816\n\t" \
  "ds_read_b128 v[244:247], v3 offset:43008\n\t" \
  "v_mfma_f32_16x16x128_f8f6f4 a[44:47], v[136:143], v[216:223], a[44:47]\n\t" \
  "ds_read_b128 v[248:251], v3 offset:35840\n\t" \
  "ds_read_b128 v[252:255], v3 offset:44032\n\t" \
  "v_mfma_f32_16x16x128_f8f6f4 a[40:43], v[136:143], v[208:215], a[40:43]\n\t" \
  "s_waitcnt vmcnt(20)\n\t" \
  "v_cmp_le_u32_e32 vcc, s75, v7\n\t" \
  "v_mfma_f32_16x16x128_f8f6f4 a[36:39], v[136:143], v[200:207], a[36:39]\n\t" \
  "s_cmp_eq_u64 vcc, -1\n\t" \
  "s_cbranch_scc0 L_SLOW11_%=\n\t" \
  "L_OK11_%=:\n\t" \
  "s_add_u32 s76, s76, 1024\n\t" \
  "v_mfma_f32_16x16x128_f8f6f4 a[32:35], v[136:143], v[192:199], a[32:35]\n\t" \
  "s_mov_b32 s53, s49\n\t" \
  "s_mov_b32 m0, s48\n\t" \
  "v_mfma_f32_16x16x128_f8f6f4 a[64:67], v[144:151], v[192:199], a[64:67]\n\t" \
  "buffer_load_dwordx4 v0, s[36:39], s53 offen sc0 sc1 lds\n\t" \
  "v_mfma_f32_16x16x128_f8f6f4 a[68:71], v[144:151], v[200:207], a[68:71]\n\t" \
  "v_mul_f32_e32 v44, s56, v44\n\t" \
  "v_mul_f32_e32 v45, s56, v45\n\t" \
  "v_mul_f32_e32 v46, s56, v46\n\t" \
  "v_mul_f32_e32 v47, s56, v47\n\t" \
  "v_mfma_f32_16x16x128_f8f6f4 a[72:75], v[144:151], v[208:215], a[72:75]\n\t" \
  "buffer_load_dwordx4 v0, s[36:39], s53 offen offset:1024 sc0 sc1 lds\n\t" \
  "v_mfma_f32_16x16x128_f8f6f4 a[76:79], v[144:151], v[216:223], a[76:79]\n\t" \
  "v_min_f32_e32 v24, s60, v44\n\t" \
  "v_min_f32_e32 v25, s60, v45\n\t" \
  "v_min_f32_e32 v26, s60, v46\n\t" \
  "v_min_f32_e32 v27, s60, v47\n\t" \
  "v_mfma_f32_16x16x128_f8f6f4 a[108:111], v[152:159], v[216:223], a[108:111]\n\t" \
  "buffer_load_dwordx4 v0, s[36:39], s53 offen offset:2048 sc0 sc1 lds\n\t" \
  "v_mfma_f32_16x16x128_f8f6f4 a[104:107], v[152:159], v[208:215], a[104:107]\n\t" \
  "v_exp_f32_e32 v24, v24\n\t" \
  "v_exp_f32_e32 v25, v25\n\t" \
  "v_mfma_f32_16x16x128_f8f6f4 a[100:103], v[152:159], v[200:207], a[100:103]\n\t" \
  "buffer_load_dwordx4 v0, s[36:39], s53 offen offset:3072 sc0 sc1 lds\n\t" \
  "v_mfma_f32_16x16x128_f8f6f4 a[96:99], v[152:159], v[192:199], a[96:99]\n\t" \
  "v_exp_f32_e32 v26, v26\n\t" \
  "v_exp_f32_e32 v27, v27\n\t" \
  "s_waitcnt lgkmcnt(0)\n\t" \
  "v_mfma_f32_16x16x128_f8f6f4 a[16:19], v[128:135], v[224:231], a[16:19]\n\t" \
  "v_fma_f32 v24, v24, s58, s58\n\t" \
  "v_fma_f32 v25, v25, s58, s58\n\t" \
  "v_fma_f32 v26, v26, s58, s58\n\t" \
  "v_fma_f32 v27, v27, s58, s58\n\t" \
  "v_mfma_f32_16x16x128_f8f6f4 a[20:23], v[128:135], v[232:239], a[20:23]\n\t" \
  "ds_read_b128 v[160:163], v1 offset:49152\n\t" \
  "ds_read_b128 v[164:167], v1 offset:57344\n\t" \
  "v_mfma_f32_16x16x128_f8f6f4 a[24:27], v[128:135], v[240:247], a[24:27]\n\t" \
  "ds_read_b128 v[168:171], v1 offset:50176\n\t" \
  "ds_read_b128 v[172:175], v1 offset:58368\n\t" \
  "v_mfma_f32_16x16x128_f8f6f4 a[28:31], v[128:135], v[248:255], a[28:31]\n\t" \
  "ds_read_b128 v[176:179], v1 offset:51200\n\t" \
  "ds_read_b128 v[180:183], v1 offset:59392\n\t" \
  "v_mfma_f32_16x16x128_f8f6f4 a[60:63], v[136:143], v[248:255], a[60:63]\n\t" \
  "ds_read_b128 v[184:187], v1 offset:52224\n\t" \
  "ds_read_b128 v[188:191], v1 offset:60416\n\t" \
  "v_mfma_f32_16x16x128_f8f6f4 a[56:59], v[136:143], v[240:247], a[56:59]\n\t" \
  "v_rcp_f32_e32 v24, v24\n\t" \
  "v_rcp_f32_e32 v25, v25\n\t" \
  "v_mfma_f32_16x16x128_f8f6f4 a[52:55], v[136:143], v[232:239], a[52:55]\n\t" \
  "v_rcp_f32_e32 v26, v26\n\t" \
  "v_rcp_f32_e32 v27, v27\n\t" \
  "v_mfma_f32_16x16x128_f8f6f4 a[48:51], v[136:143], v[224:231], a[48:51]\n\t" \
  "s_mov_b32 s53, s50\n\t" \
  "s_add_u32 m0, s48, 16384\n\t" \
  "v_mfma_f32_16x16x128_f8f6f4 a[80:83], v[144:151], v[224:231], a[80:83]\n\t" \
  "buffer_load_dwordx4 v0, s[36:39], s53 offen sc0 sc1 lds\n\t" \
  "v_mfma_f32_16x16x128_f8f6f4 a[84:87], v[144:151], v[232:239], a[84:87]\n\t" \
  "v_mul_f32_e32 v44, v44, v24\n\t" \
  "v_mul_f32_e32 v45, v45, v25\n\t" \
  "v_mul_f32_e32 v46, v46, v26\n\t" \
  "v_mul_f32_e32 v47, v47, v27\n\t" \
  "v_mfma_f32_16x16x128_f8f6f4 a[88:91], v[144:151], v[240:247], a[88:91]\n\t" \
  "buffer_load_dwordx4 v0, s[36:39], s53 offen offset:1024 sc0 sc1 lds\n\t" \
  "v_mfma_f32_16x16x128_f8f6f4 a[92:95], v[144:151], v[248:255], a[92:95]\n\t" \
  "s_xor_b32 s91, s68, 196608\n\t" \
  "buffer_store_dwordx2 v[44:45], v5, s[40:43], s91 offen nt\n\t" \
  "v_mfma_f32_16x16x128_f8f6f4 a[124:127], v[152:159], v[248:255], a[124:127]\n\t" \
  "buffer_load_dwordx4 v0, s[36:39], s53 offen offset:2048 sc0 sc1 lds\n\t" \
  "v_mfma_f32_16x16x128_f8f6f4 a[120:123], v[152:159], v[240:247], a[120:123]\n\t" \
  "buffer_store_dwordx2 v[46:47], v5, s[40:43], s91 offen offset:128 nt\n\t" \
  "v_mfma_f32_16x16x128_f8f6f4 a[116:119], v[152:159], v[232:239], a[116:119]\n\t" \
  "buffer_load_dwordx4 v0, s[36:39], s53 offen offset:3072 sc0 sc1 lds\n\t" \
  "v_mfma_f32_16x16x128_f8f6f4 a[112:115], v[152:159], v[224:231], a[112:115]\n\t" \
  "v_mul_f32_e32 v48, s56, v48\n\t" \
  "v_mul_f32_e32 v49, s56, v49\n\t" \
  "v_mul_f32_e32 v50, s56, v50\n\t" \
  "v_mul_f32_e32 v51, s56, v51\n\t" \
  "s_waitcnt lgkmcnt(0)\n\t" \
  "v_mfma_f32_16x16x128_f8f6f4 a[144:147], v[160:167], v[224:231], a[144:147]\n\t" \
  "s_waitcnt vmcnt(22)\n\t" \
  "s_barrier\n\t" \
  "v_mfma_f32_16x16x128_f8f6f4 a[148:151], v[160:167], v[232:239], a[148:151]\n\t" \
  "ds_read_b128 v[128:131], v2 offset:0\n\t" \
  "ds_read_b128 v[132:135], v2 offset:8192\n\t" \
  "v_mfma_f32_16x16x128_f8f6f4 a[152:155], v[160:167], v[240:247], a[152:155]\n\t" \
  "ds_read_b128 v[136:139], v2 offset:1024\n\t" \
  "ds_read_b128 v[140:143], v2 offset:9216\n\t" \
  "v_mfma_f32_16x16x128_f8f6f4 a[156:159], v[160:167], v[248:255], a[156:159]\n\t" \
  "ds_read_b128 v[144:147], v2 offset:2048\n\t" \
  "ds_read_b128 v[148:151], v2 offset:10240\n\t" \
  "v_mfma_f32_16x16x128_f8f6f4 a[188:191], v[168:175], v[248:255], a[188:191]\n\t" \
  "ds_read_b128 v[152:155], v2 offset:3072\n\t" \
  "ds_read_b128 v[156:159], v2 offset:11264\n\t" \
  "v_mfma_f32_16x16x128_f8f6f4 a[184:187], v[168:175], v[240:247], a[184:187]\n\t" \
  "v_min_f32_e32 v28, s60, v48\n\t" \
  "v_min_f32_e32 v29, s60, v49\n\t" \
  "v_min_f32_e32 v30, s60, v50\n\t" \
  "v_min_f32_e32 v31, s60, v51\n\t" \
  "v_mfma_f32_16x16x128_f8f6f4 a[180:183], v[168:175], v[232:239], a[180:183]\n\t" \
  "v_exp_f32_e32 v28, v28\n\t" \
  "v_exp_f32_e32 v29, v29\n\t" \
  "v_mfma_f32_16x16x128_f8f6f4 a[176:179], v[168:175], v[224:231], a[176:179]\n\t" \
  "s_mov_b32 s53, s51\n\t" \
  "s_add_u32 m0, s48, 32768\n\t" \
  "v_mfma_f32_16x16x128_f8f6f4 a[208:211], v[176:183], v[224:231], a[208:211]\n\t" \
  "buffer_load_dwordx4 v0, s[36:39], s53 offen sc0 sc1 lds\n\t" \
  "v_mfma_f32_16x16x128_f8f6f4 a[212:215], v[176:183], v[232:239], a[212:215]\n\t" \
  "v_exp_f32_e32 v30, v30\n\t" \
  "v_exp_f32_e32 v31, v31\n\t" \
  "v_mfma_f32_16x16x128_f8f6f4 a[216:219], v[176:183], v[240:247], a[216:219]\n\t" \
  "buffer_load_dwordx4 v0, s[36:39], s53 offen offset:1024 sc0 sc1 lds\n\t" \
  "v_mfma_f32_16x16x128_f8f6f4 a[220:223], v[176:183], v[248:255], a[220:223]\n\t" \
  "v_fma_f32 v28, v28, s58, s58\n\t" \
  "v_fma_f32 v29, v29, s58, s58\n\t" \
  "v_fma_f32 v30, v30, s58, s58\n\t" \
  "v_fma_f32 v31, v31, s58, s58\n\t" \
  "v_mfma_f32_16x16x128_f8f6f4 a[252:255], v[184:191], v[248:255], a[252:255]\n\t" \
  "buffer_load_dwordx4 v0, s[36:39], s53 offen offset:2048 sc0 sc1 lds\n\t" \
  "v_mfma_f32_16x16x128_f8f6f4 a[248:251], v[184:191], v[240:247], a[248:251]\n\t" \
  "v_rcp_f32_e32 v28, v28\n\t" \
  "v_rcp_f32_e32 v29, v29\n\t" \
  "v_mfma_f32_16x16x128_f8f6f4 a[244:247], v[184:191], v[232:239], a[244:247]\n\t" \
  "buffer_load_dwordx4 v0, s[36:39], s53 offen offset:3072 sc0 sc1 lds\n\t" \
  "v_mfma_f32_16x16x128_f8f6f4 a[240:243], v[184:191], v[224:231], a[240:243]\n\t" \
  "v_rcp_f32_e32 v30, v30\n\t" \
  "v_rcp_f32_e32 v31, v31\n\t" \
  "s_waitcnt lgkmcnt(0)\n\t" \
  "v_mfma_f32_16x16x128_f8f6f4 a[128:131], v[160:167], v[192:199], a[128:131]\n\t" \
  "v_mul_f32_e32 v48, v48, v28\n\t" \
  "v_mul_f32_e32 v49, v49, v29\n\t" \
  "v_mul_f32_e32 v50, v50, v30\n\t" \
  "v_mul_f32_e32 v51, v51, v31\n\t" \
  "v_mfma_f32_16x16x128_f8f6f4 a[132:135], v[160:167], v[200:207], a[132:135]\n\t" \
  "ds_read_b128 v[224:227], v4 offset:16384\n\t" \
  "ds_read_b128 v[228:231], v4 offset:24576\n\t" \
  "v_mfma_f32_16x16x128_f8f6f4 a[136:139], v[160:167], v[208:215], a[136:139]\n\t" \
  "ds_read_b128 v[232:235], v4 offset:17408\n\t" \
  "ds_read_b128 v[236:239], v4 offset:25600\n\t" \
  "v_mfma_f32_16x16x128_f8f6f4 a[140:143], v[160:167], v[216:223], a[140:143]\n\t" \
  "ds_read_b128 v[240:243], v4 offset:18432\n\t" \
  "ds_read_b128 v[244:247], v4 offset:26624\n\t" \
  "v_mfma_f32_16x16x128_f8f6f4 a[172:175], v[168:175], v[216:223], a[172:175]\n\t" \
  "ds_read_b128 v[248:251], v4 offset:19456\n\t" \
  "ds_read_b128 v[252:255], v4 offset:27648\n\t" \
  "v_mfma_f32_16x16x128_f8f6f4 a[168:171], v[168:175], v[208:215], a[168:171]\n\t" \
  "s_xor_b32 s91, s68, 1048576\n\t" \
  "buffer_store_dwordx2 v[48:49], v5, s[40:43], s91 offen nt\n\t" \
  "v_mfma_f32_16x16x128_f8f6f4 a[164:167], v[168:175], v[200:207], a[164:167]\n\t" \
  "v_mfma_f32_16x16x128_f8f6f4 a[160:163], v[168:175], v[192:199], a[160:163]\n\t" \
  "s_add_u32 s53, s49, 16384\n\t" \
  "s_add_u32 m0, s48, 49152\n\t" \
  "v_mfma_f32_16x16x128_f8f6f4 a[192:195], v[176:183], v[192:199], a[192:195]\n\t" \
  "buffer_load_dwordx4 v0, s[36:39], s53 offen sc0 sc1 lds\n\t" \
  "v_mfma_f32_16x16x128_f8f6f4 a[196:199], v[176:183], v[200:207], a[196:199]\n\t" \
  "buffer_store_dwordx2 v[50:51], v5, s[40:43], s91 offen offset:128 nt\n\t" \
  "v_mfma_f32_16x16x128_f8f6f4 a[200:203], v[176:183], v[208:215], a[200:203]\n\t" \
  "buffer_load_dwordx4 v0, s[36:39], s53 offen offset:1024 sc0 sc1 lds\n\t" \
  "v_mfma_f32_16x16x128_f8f6f4 a[204:207], v[176:183], v[216:223], a[204:207]\n\t" \
  "v_mul_f32_e32 v52, s56, v52\n\t" \
  "v_mul_f32_e32 v53, s56, v53\n\t" \
  "v_mul_f32_e32 v54, s56, v54\n\t" \
  "v_mul_f32_e32 v55, s56, v55\n\t" \
  "v_mfma_f32_16x16x128_f8f6f4 a[236:239], v[184:191], v[216:223], a[236:239]\n\t" \
  "buffer_load_dwordx4 v0, s[36:39], s53 offen offset:2048 sc0 sc1 lds\n\t" \
  "v_mfma_f32_16x16x128_f8f6f4 a[232:235], v[184:191], v[208:215], a[232:235]\n\t" \
  "v_min_f32_e32 v12, s60, v52\n\t" \
  "v_min_f32_e32 v13, s60, v53\n\t" \
  "v_min_f32_e32 v14, s60, v54\n\t" \
  "v_min_f32_e32 v15, s60, v55\n\t" \
  "v_mfma_f32_16x16x128_f8f6f4 a[228:231], v[184:191], v[200:207], a[228:231]\n\t" \
  "buffer_load_dwordx4 v0, s[36:39], s53 offen offset:3072 sc0 sc1 lds\n\t" \
  "v_mfma_f32_16x16x128_f8f6f4 a[224:227], v[184:191], v[192:199], a[224:227]\n\t" \
  "v_exp_f32_e32 v12, v12\n\t" \
  "v_exp_f32_e32 v13, v13\n\t" \
  "s_waitcnt lgkmcnt(0)\n\t" \
  "v_mfma_f32_16x16x128_f8f6f4 a[16:19], v[128:135], v[224:231], a[16:19]\n\t" \
  "s_waitcnt vmcnt(20)\n\t" \
  "s_barrier\n\t" \
  "v_mfma_f32_16x16x128_f8f6f4 a[20:23], v[128:135], v[232:239], a[20:23]\n\t" \
  "ds_read_b128 v[192:195], v4 offset:32768\n\t" \
  "ds_read_b128 v[196:199], v4 offset:40960\n\t" \
  "v_mfma_f32_16x16x128_f8f6f4 a[24:27], v[128:135], v[240:247], a[24:27]\n\t" \
  "ds_read_b128 v[200:203], v4 offset:33792\n\t" \
  "ds_read_b128 v[204:207], v4 offset:41984\n\t" \
  "v_mfma_f32_16x16x128_f8f6f4 a[28:31], v[128:135], v[248:255], a[28:31]\n\t" \
  "ds_read_b128 v[208:211], v4 offset:34816\n\t" \
  "ds_read_b128 v[212:215], v4 offset:43008\n\t" \
  "v_mfma_f32_16x16x128_f8f6f4 a[60:63], v[136:143], v[248:255], a[60:63]\n\t" \
  "ds_read_b128 v[216:219], v4 offset:35840\n\t" \
  "ds_read_b128 v[220:223], v4 offset:44032\n\t" \
  "v_mfma_f32_16x16x128_f8f6f4 a[56:59], v[136:143], v[240:247], a[56:59]\n\t" \
  "v_exp_f32_e32 v14, v14\n\t" \
  "v_exp_f32_e32 v15, v15\n\t" \
  "v_mfma_f32_16x16x128_f8f6f4 a[52:55], v[136:143], v[232:239], a[52:55]\n\t" \
  "v_fma_f32 v12, v12, s58, s58\n\t" \
  "v_fma_f32 v13, v13, s58, s58\n\t" \
  "v_fma_f32 v14, v14, s58, s58\n\t" \
  "v_fma_f32 v15, v15, s58, s58\n\t" \
  "v_mfma_f32_16x16x128_f8f6f4 a[48:51], v[136:143], v[224:231], a[48:51]\n\t" \
  "s_add_u32 s53, s49, 32768\n\t" \
  "s_add_u32 m0, s48, 65536\n\t" \
  "v_mfma_f32_16x16x128_f8f6f4 a[80:83], v[144:151], v[224:231], a[80:83]\n\t" \
  "buffer_load_dwordx4 v0, s[36:39], s53 offen sc0 sc1 lds\n\t" \
  "v_mfma_f32_16x16x128_f8f6f4 a[84:87], v[144:151], v[232:239], a[84:87]\n\t" \
  "v_rcp_f32_e32 v12, v12\n\t" \
  "v_rcp_f32_e32 v13, v13\n\t" \
  "v_mfma_f32_16x16x128_f8f6f4 a[88:91], v[144:151], v[240:247], a[88:91]\n\t" \
  "buffer_load_dwordx4 v0, s[36:39], s53 offen offset:1024 sc0 sc1 lds\n\t" \
  "v_mfma_f32_16x16x128_f8f6f4 a[92:95], v[144:151], v[248:255], a[92:95]\n\t" \
  "v_rcp_f32_e32 v14, v14\n\t" \
  "v_rcp_f32_e32 v15, v15\n\t" \
  "v_mfma_f32_16x16x128_f8f6f4 a[124:127], v[152:159], v[248:255], a[124:127]\n\t" \
  "buffer_load_dwordx4 v0, s[36:39], s53 offen offset:2048 sc0 sc1 lds\n\t" \
  "v_mfma_f32_16x16x128_f8f6f4 a[120:123], v[152:159], v[240:247], a[120:123]\n\t" \
  "v_mul_f32_e32 v52, v52, v12\n\t" \
  "v_mul_f32_e32 v53, v53, v13\n\t" \
  "v_mul_f32_e32 v54, v54, v14\n\t" \
  "v_mul_f32_e32 v55, v55, v15\n\t" \
  "v_mfma_f32_16x16x128_f8f6f4 a[116:119], v[152:159], v[232:239], a[116:119]\n\t" \
  "buffer_load_dwordx4 v0, s[36:39], s53 offen offset:3072 sc0 sc1 lds\n\t" \
  "v_mfma_f32_16x16x128_f8f6f4 a[112:115], v[152:159], v[224:231], a[112:115]\n\t" \
  "s_xor_b32 s91, s68, 1114112\n\t" \
  "buffer_store_dwordx2 v[52:53], v5, s[40:43], s91 offen nt\n\t" \
  "s_waitcnt lgkmcnt(0)\n\t" \
  "v_mfma_f32_16x16x128_f8f6f4 a[0:3], v[128:135], v[192:199], a[0:3]\n\t" \
  "buffer_store_dwordx2 v[54:55], v5, s[40:43], s91 offen offset:128 nt\n\t" \
  "v_mfma_f32_16x16x128_f8f6f4 a[4:7], v[128:135], v[200:207], a[4:7]\n\t" \
  "ds_read_b128 v[160:163], v2 offset:49152\n\t" \
  "ds_read_b128 v[164:167], v2 offset:57344\n\t" \
  "v_mfma_f32_16x16x128_f8f6f4 a[8:11], v[128:135], v[208:215], a[8:11]\n\t" \
  "ds_read_b128 v[168:171], v2 offset:50176\n\t" \
  "ds_read_b128 v[172:175], v2 offset:58368\n\t" \
  "v_mfma_f32_16x16x128_f8f6f4 a[12:15], v[128:135], v[216:223], a[12:15]\n\t" \
  "ds_read_b128 v[176:179], v2 offset:51200\n\t" \
  "ds_read_b128 v[180:183], v2 offset:59392\n\t" \
  "v_mfma_f32_16x16x128_f8f6f4 a[44:47], v[136:143], v[216:223], a[44:47]\n\t" \
  "ds_read_b128 v[184:187], v2 offset:52224\n\t" \
  "ds_read_b128 v[188:191], v2 offset:60416\n\t" \
  "v_mfma_f32_16x16x128_f8f6f4 a[40:43], v[136:143], v[208:215], a[40:43]\n\t" \
  "v_mul_f32_e32 v56, s56, v56\n\t" \
  "v_mul_f32_e32 v57, s56, v57\n\t" \
  "v_mul_f32_e32 v58, s56, v58\n\t" \
  "v_mul_f32_e32 v59, s56, v59\n\t" \
  "v_mfma_f32_16x16x128_f8f6f4 a[36:39], v[136:143], v[200:207], a[36:39]\n\t" \
  "v_min_f32_e32 v16, s60, v56\n\t" \
  "v_min_f32_e32 v17, s60, v57\n\t" \
  "v_min_f32_e32 v18, s60, v58\n\t" \
  "v_min_f32_e32 v19, s60, v59\n\t" \
  "v_mfma_f32_16x16x128_f8f6f4 a[32:35], v[136:143], v[192:199], a[32:35]\n\t" \
  "s_add_u32 s53, s51, 16384\n\t" \
  "s_add_u32 m0, s48, 81920\n\t" \
  "v_mfma_f32_16x16x128_f8f6f4 a[64:67], v[144:151], v[192:199], a[64:67]\n\t" \
  "buffer_load_dwordx4 v0, s[36:39], s53 offen sc0 sc1 lds\n\t" \
  "v_mfma_f32_16x16x128_f8f6f4 a[68:71], v[144:151], v[200:207], a[68:71]\n\t" \
  "v_exp_f32_e32 v16, v16\n\t" \
  "v_exp_f32_e32 v17, v17\n\t" \
  "v_mfma_f32_16x16x128_f8f6f4 a[72:75], v[144:151], v[208:215], a[72:75]\n\t" \
  "buffer_load_dwordx4 v0, s[36:39], s53 offen offset:1024 sc0 sc1 lds\n\t" \
  "v_mfma_f32_16x16x128_f8f6f4 a[76:79], v[144:151], v[216:223], a[76:79]\n\t" \
  "v_exp_f32_e32 v18, v18\n\t" \
  "v_exp_f32_e32 v19, v19\n\t" \
  "v_mfma_f32_16x16x128_f8f6f4 a[108:111], v[152:159], v[216:223], a[108:111]\n\t" \
  "buffer_load_dwordx4 v0, s[36:39], s53 offen offset:2048 sc0 sc1 lds\n\t" \
  "v_mfma_f32_16x16x128_f8f6f4 a[104:107], v[152:159], v[208:215], a[104:107]\n\t" \
  "v_fma_f32 v16, v16, s58, s58\n\t" \
  "v_fma_f32 v17, v17, s58, s58\n\t" \
  "v_fma_f32 v18, v18, s58, s58\n\t" \
  "v_fma_f32 v19, v19, s58, s58\n\t" \
  "v_mfma_f32_16x16x128_f8f6f4 a[100:103], v[152:159], v[200:207], a[100:103]\n\t" \
  "buffer_load_dwordx4 v0, s[36:39], s53 offen offset:3072 sc0 sc1 lds\n\t" \
  "v_mfma_f32_16x16x128_f8f6f4 a[96:99], v[152:159], v[192:199], a[96:99]\n\t" \
  "v_rcp_f32_e32 v16, v16\n\t" \
  "v_rcp_f32_e32 v17, v17\n\t" \
  "s_waitcnt lgkmcnt(0)\n\t" \
  "v_mfma_f32_16x16x128_f8f6f4 a[128:131], v[160:167], v[192:199], a[128:131]\n\t" \
  "s_waitcnt vmcnt(20)\n\t" \
  "s_barrier\n\t" \
  "v_mfma_f32_16x16x128_f8f6f4 a[132:135], v[160:167], v[200:207], a[132:135]\n\t" \
  "ds_read_b128 v[128:131], v1 offset:0\n\t" \
  "ds_read_b128 v[132:135], v1 offset:8192\n\t" \
  "v_mfma_f32_16x16x128_f8f6f4 a[136:139], v[160:167], v[208:215], a[136:139]\n\t" \
  "ds_read_b128 v[136:139], v1 offset:1024\n\t" \
  "ds_read_b128 v[140:143], v1 offset:9216\n\t" \
  "v_mfma_f32_16x16x128_f8f6f4 a[140:143], v[160:167], v[216:223], a[140:143]\n\t" \
  "ds_read_b128 v[144:147], v1 offset:2048\n\t" \
  "ds_read_b128 v[148:151], v1 offset:10240\n\t" \
  "v_mfma_f32_16x16x128_f8f6f4 a[172:175], v[168:175], v[216:223], a[172:175]\n\t" \
  "ds_read_b128 v[152:155], v1 offset:3072\n\t" \
  "ds_read_b128 v[156:159], v1 offset:11264\n\t" \
  "v_mfma_f32_16x16x128_f8f6f4 a[168:171], v[168:175], v[208:215], a[168:171]\n\t" \
  "v_rcp_f32_e32 v18, v18\n\t" \
  "v_rcp_f32_e32 v19, v19\n\t" \
  "v_mfma_f32_16x16x128_f8f6f4 a[164:167], v[168:175], v[200:207], a[164:167]\n\t" \
  "v_mul_f32_e32 v56, v56, v16\n\t" \
  "v_mul_f32_e32 v57, v57, v17\n\t" \
  "v_mul_f32_e32 v58, v58, v18\n\t" \
  "v_mul_f32_e32 v59, v59, v19\n\t" \
  "v_mfma_f32_16x16x128_f8f6f4 a[160:163], v[168:175], v[192:199], a[160:163]\n\t" \
  "s_add_u32 s53, s50, 16384\n\t" \
  "s_add_u32 m0, s48, 98304\n\t" \
  "v_mfma_f32_16x16x128_f8f6f4 a[192:195], v[176:183], v[192:199], a[192:195]\n\t" \
  "buffer_load_dwordx4 v0, s[36:39], s53 offen sc0 sc1 lds\n\t" \
  "v_mfma_f32_16x16x128_f8f6f4 a[196:199], v[176:183], v[200:207], a[196:199]\n\t" \
  "s_xor_b32 s91, s68, 1179648\n\t" \
  "buffer_store_dwordx2 v[56:57], v5, s[40:43], s91 offen nt\n\t" \
  "v_mfma_f32_16x16x128_f8f6f4 a[200:203], v[176:183], v[208:215], a[200:203]\n\t" \
  "buffer_load_dwordx4 v0, s[36:39], s53 offen offset:1024 sc0 sc1 lds\n\t" \
  "v_mfma_f32_16x16x128_f8f6f4 a[204:207], v[176:183], v[216:223], a[204:207]\n\t" \
  "buffer_store_dwordx2 v[58:59], v5, s[40:43], s91 offen offset:128 nt\n\t" \
  "v_mfma_f32_16x16x128_f8f6f4 a[236:239], v[184:191], v[216:223], a[236:239]\n\t" \
  "buffer_load_dwordx4 v0, s[36:39], s53 offen offset:2048 sc0 sc1 lds\n\t" \
  "v_mfma_f32_16x16x128_f8f6f4 a[232:235], v[184:191], v[208:215], a[232:235]\n\t" \
  "v_mul_f32_e32 v60, s56, v60\n\t" \
  "v_mul_f32_e32 v61, s56, v61\n\t" \
  "v_mul_f32_e32 v62, s56, v62\n\t" \
  "v_mul_f32_e32 v63, s56, v63\n\t" \
  "v_mfma_f32_16x16x128_f8f6f4 a[228:231], v[184:191], v[200:207], a[228:231]\n\t" \
  "buffer_load_dwordx4 v0, s[36:39], s53 offen offset:3072 sc0 sc1 lds\n\t" \
  "v_mfma_f32_16x16x128_f8f6f4 a[224:227], v[184:191], v[192:199], a[224:227]\n\t" \
  "v_min_f32_e32 v20, s60, v60\n\t" \
  "v_min_f32_e32 v21, s60, v61\n\t" \
  "v_min_f32_e32 v22, s60, v62\n\t" \
  "v_min_f32_e32 v23, s60, v63\n\t" \
  "s_waitcnt lgkmcnt(0)\n\t" \
  "v_mfma_f32_16x16x128_f8f6f4 a[144:147], v[160:167], v[224:231], a[144:147]\n\t" \
  "v_exp_f32_e32 v20, v20\n\t" \
  "v_exp_f32_e32 v21, v21\n\t" \
  "v_mfma_f32_16x16x128_f8f6f4 a[148:151], v[160:167], v[232:239], a[148:151]\n\t" \
  "ds_read_b128 v[192:195], v3 offset:16384\n\t" \
  "ds_read_b128 v[196:199], v3 offset:24576\n\t" \
  "v_mfma_f32_16x16x128_f8f6f4 a[152:155], v[160:167], v[240:247], a[152:155]\n\t" \
  "ds_read_b128 v[200:203], v3 offset:17408\n\t" \
  "ds_read_b128 v[204:207], v3 offset:25600\n\t" \
  "v_mfma_f32_16x16x128_f8f6f4 a[156:159], v[160:167], v[248:255], a[156:159]\n\t" \
  "ds_read_b128 v[208:211], v3 offset:18432\n\t" \
  "ds_read_b128 v[212:215], v3 offset:26624\n\t" \
  "v_mfma_f32_16x16x128_f8f6f4 a[188:191], v[168:175], v[248:255], a[188:191]\n\t" \
  "ds_read_b128 v[216:219], v3 offset:19456\n\t" \
  "ds_read_b128 v[220:223], v3 offset:27648\n\t" \
  "v_mfma_f32_16x16x128_f8f6f4 a[184:187], v[168:175], v[240:247], a[184:187]\n\t" \
  "v_mfma_f32_16x16x128_f8f6f4 a[180:183], v[168:175], v[232:239], a[180:183]\n\t" \
  "v_exp_f32_e32 v22, v22\n\t" \
  "v_exp_f32_e32 v23, v23\n\t" \
  "v_mfma_f32_16x16x128_f8f6f4 a[176:179], v[168:175], v[224:231], a[176:179]\n\t" \
  "s_add_u32 s53, s49, 49152\n\t" \
  "s_add_u32 m0, s48, 114688\n\t" \
  "v_mfma_f32_16x16x128_f8f6f4 a[208:211], v[176:183], v[224:231], a[208:211]\n\t" \
  "buffer_load_dwordx4 v0, s[36:39], s53 offen sc0 sc1 lds\n\t" \
  "v_mfma_f32_16x16x128_f8f6f4 a[212:215], v[176:183], v[232:239], a[212:215]\n\t" \
  "v_fma_f32 v20, v20, s58, s58\n\t" \
  "v_fma_f32 v21, v21, s58, s58\n\t" \
  "v_fma_f32 v22, v22, s58, s58\n\t" \
  "v_fma_f32 v23, v23, s58, s58\n\t" \
  "v_mfma_f32_16x16x128_f8f6f4 a[216:219], v[176:183], v[240:247], a[216:219]\n\t" \
  "buffer_load_dwordx4 v0, s[36:39], s53 offen offset:1024 sc0 sc1 lds\n\t" \
  "v_mfma_f32_16x16x128_f8f6f4 a[220:223], v[176:183], v[248:255], a[220:223]\n\t" \
  "v_rcp_f32_e32 v20, v20\n\t" \
  "v_rcp_f32_e32 v21, v21\n\t" \
  "v_mfma_f32_16x16x128_f8f6f4 a[252:255], v[184:191], v[248:255], a[252:255]\n\t" \
  "buffer_load_dwordx4 v0, s[36:39], s53 offen offset:2048 sc0 sc1 lds\n\t" \
  "v_mfma_f32_16x16x128_f8f6f4 a[248:251], v[184:191], v[240:247], a[248:251]\n\t" \
  "v_rcp_f32_e32 v22, v22\n\t" \
  "v_rcp_f32_e32 v23, v23\n\t" \
  "v_mfma_f32_16x16x128_f8f6f4 a[244:247], v[184:191], v[232:239], a[244:247]\n\t" \
  "buffer_load_dwordx4 v0, s[36:39], s53 offen offset:3072 sc0 sc1 lds\n\t" \
  "v_mfma_f32_16x16x128_f8f6f4 a[240:243], v[184:191], v[224:231], a[240:243]\n\t" \
  "s_add_u32 s49, s49, 65536\n\t" \
  "s_add_u32 s50, s50, 32768\n\t" \
  "s_add_u32 s51, s51, 32768\n\t" \
  "s_waitcnt lgkmcnt(0)\n\t" \
  "v_mfma_f32_16x16x128_f8f6f4 a[0:3], v[128:135], v[192:199], a[0:3]\n\t" \
  "s_waitcnt vmcnt(20)\n\t" \
  "s_barrier\n\t" \
  "v_mfma_f32_16x16x128_f8f6f4 a[4:7], v[128:135], v[200:207], a[4:7]\n\t" \
  "ds_read_b128 v[224:227], v3 offset:32768\n\t" \
  "ds_read_b128 v[228:231], v3 offset:40960\n\t" \
  "v_mfma_f32_16x16x128_f8f6f4 a[8:11], v[128:135], v[208:215], a[8:11]\n\t" \
  "ds_read_b128 v[232:235], v3 offset:33792\n\t" \
  "ds_read_b128 v[236:239], v3 offset:41984\n\t" \
  "v_mfma_f32_16x16x128_f8f6f4 a[12:15], v[128:135], v[216:223], a[12:15]\n\t" \
  "ds_read_b128 v[240:243], v3 offset:34816\n\t" \
  "ds_read_b128 v[244:247], v3 offset:43008\n\t" \
  "v_mfma_f32_16x16x128_f8f6f4 a[44:47], v[136:143], v[216:223], a[44:47]\n\t" \
  "ds_read_b128 v[248:251], v3 offset:35840\n\t" \
  "ds_read_b128 v[252:255], v3 offset:44032\n\t" \
  "v_mfma_f32_16x16x128_f8f6f4 a[40:43], v[136:143], v[208:215], a[40:43]\n\t" \
  "v_mul_f32_e32 v60, v60, v20\n\t" \
  "v_mul_f32_e32 v61, v61, v21\n\t" \
  "v_mul_f32_e32 v62, v62, v22\n\t" \
  "v_mul_f32_e32 v63, v63, v23\n\t" \
  "v_mfma_f32_16x16x128_f8f6f4 a[36:39], v[136:143], v[200:207], a[36:39]\n\t" \
  "s_xor_b32 s91, s68, 1245184\n\t" \
  "buffer_store_dwordx2 v[60:61], v5, s[40:43], s91 offen nt\n\t" \
  "v_mfma_f32_16x16x128_f8f6f4 a[32:35], v[136:143], v[192:199], a[32:35]\n\t" \
  "s_mov_b32 s53, s49\n\t" \
  "s_mov_b32 m0, s48\n\t" \
  "v_mfma_f32_16x16x128_f8f6f4 a[64:67], v[144:151], v[192:199], a[64:67]\n\t" \
  "buffer_load_dwordx4 v0, s[36:39], s53 offen sc0 sc1 lds\n\t" \
  "v_mfma_f32_16x16x128_f8f6f4 a[68:71], v[144:151], v[200:207], a[68:71]\n\t" \
  "buffer_store_dwordx2 v[62:63], v5, s[40:43], s91 offen offset:128 nt\n\t" \
  "v_mfma_f32_16x16x128_f8f6f4 a[72:75], v[144:151], v[208:215], a[72:75]\n\t" \
  "buffer_load_dwordx4 v0, s[36:39], s53 offen offset:1024 sc0 sc1 lds\n\t" \
  "v_mfma_f32_16x16x128_f8f6f4 a[76:79], v[144:151], v[216:223], a[76:79]\n\t" \
  "v_mul_f32_e32 v64, s56, v64\n\t" \
  "v_mul_f32_e32 v65, s56, v65\n\t" \
  "v_mul_f32_e32 v66, s56, v66\n\t" \
  "v_mul_f32_e32 v67, s56, v67\n\t" \
  "v_mfma_f32_16x16x128_f8f6f4 a[108:111], v[152:159], v[216:223], a[108:111]\n\t" \
  "buffer_load_dwordx4 v0, s[36:39], s53 offen offset:2048 sc0 sc1 lds\n\t" \
  "v_mfma_f32_16x16x128_f8f6f4 a[104:107], v[152:159], v[208:215], a[104:107]\n\t" \
  "v_min_f32_e32 v24, s60, v64\n\t" \
  "v_min_f32_e32 v25, s60, v65\n\t" \
  "v_min_f32_e32 v26, s60, v66\n\t" \
  "v_min_f32_e32 v27, s60, v67\n\t" \
  "v_mfma_f32_16x16x128_f8f6f4 a[100:103], v[152:159], v[200:207], a[100:103]\n\t" \
  "buffer_load_dwordx4 v0, s[36:39], s53 offen offset:3072 sc0 sc1 lds\n\t" \
  "v_mfma_f32_16x16x128_f8f6f4 a[96:99], v[152:159], v[192:199], a[96:99]\n\t" \
  "v_exp_f32_e32 v24, v24\n\t" \
  "v_exp_f32_e32 v25, v25\n\t" \
  "s_waitcnt lgkmcnt(0)\n\t" \
  "v_mfma_f32_16x16x128_f8f6f4 a[16:19], v[128:135], v[224:231], a[16:19]\n\t" \
  "v_exp_f32_e32 v26, v26\n\t" \
  "v_exp_f32_e32 v27, v27\n\t" \
  "v_mfma_f32_16x16x128_f8f6f4 a[20:23], v[128:135], v[232:239], a[20:23]\n\t" \
  "ds_read_b128 v[160:163], v1 offset:49152\n\t" \
  "ds_read_b128 v[164:167], v1 offset:57344\n\t" \
  "v_mfma_f32_16x16x128_f8f6f4 a[24:27], v[128:135], v[240:247], a[24:27]\n\t" \
  "ds_read_b128 v[168:171], v1 offset:50176\n\t" \
  "ds_read_b128 v[172:175], v1 offset:58368\n\t" \
  "v_mfma_f32_16x16x128_f8f6f4 a[28:31], v[128:135], v[248:255], a[28:31]\n\t" \
  "ds_read_b128 v[176:179], v1 offset:51200\n\t" \
  "ds_read_b128 v[180:183], v1 offset:59392\n\t" \
  "v_mfma_f32_16x16x128_f8f6f4 a[60:63], v[136:143], v[248:255], a[60:63]\n\t" \
  "ds_read_b128 v[184:187], v1 offset:52224\n\t" \
  "ds_read_b128 v[188:191], v1 offset:60416\n\t" \
  "v_mfma_f32_16x16x128_f8f6f4 a[56:59], v[136:143], v[240:247], a[56:59]\n\t" \
  "v_fma_f32 v24, v24, s58, s58\n\t" \
  "v_fma_f32 v25, v25, s58, s58\n\t" \
  "v_fma_f32 v26, v26, s58, s58\n\t" \
  "v_fma_f32 v27, v27, s58, s58\n\t" \
  "v_mfma_f32_16x16x128_f8f6f4 a[52:55], v[136:143], v[232:239], a[52:55]\n\t" \
  "v_rcp_f32_e32 v24, v24\n\t" \
  "v_rcp_f32_e32 v25, v25\n\t" \
  "v_mfma_f32_16x16x128_f8f6f4 a[48:51], v[136:143], v[224:231], a[48:51]\n\t" \
  "s_mov_b32 s53, s50\n\t" \
  "s_add_u32 m0, s48, 16384\n\t" \
  "v_mfma_f32_16x16x128_f8f6f4 a[80:83], v[144:151], v[224:231], a[80:83]\n\t" \
  "buffer_load_dwordx4 v0, s[36:39], s53 offen sc0 sc1 lds\n\t" \
  "v_mfma_f32_16x16x128_f8f6f4 a[84:87], v[144:151], v[232:239], a[84:87]\n\t" \
  "v_rcp_f32_e32 v26, v26\n\t" \
  "v_rcp_f32_e32 v27, v27\n\t" \
  "v_mfma_f32_16x16x128_f8f6f4 a[88:91], v[144:151], v[240:247], a[88:91]\n\t" \
  "buffer_load_dwordx4 v0, s[36:39], s53 offen offset:1024 sc0 sc1 lds\n\t" \
  "v_mfma_f32_16x16x128_f8f6f4 a[92:95], v[144:151], v[248:255], a[92:95]\n\t" \
  "v_mul_f32_e32 v64, v64, v24\n\t" \
  "v_mul_f32_e32 v65, v65, v25\n\t" \
  "v_mul_f32_e32 v66, v66, v26\n\t" \
  "v_mul_f32_e32 v67, v67, v27\n\t" \
  "v_mfma_f32_16x16x128_f8f6f4 a[124:127], v[152:159], v[248:255], a[124:127]\n\t" \
  "buffer_load_dwordx4 v0, s[36:39], s53 offen offset:2048 sc0 sc1 lds\n\t" \
  "v_mfma_f32_16x16x128_f8f6f4 a[120:123], v[152:159], v[240:247], a[120:123]\n\t" \
  "s_xor_b32 s91, s68, 2097152\n\t" \
  "buffer_store_dwordx2 v[64:65], v5, s[40:43], s91 offen nt\n\t" \
  "v_mfma_f32_16x16x128_f8f6f4 a[116:119], v[152:159], v[232:239], a[116:119]\n\t" \
  "buffer_load_dwordx4 v0, s[36:39], s53 offen offset:3072 sc0 sc1 lds\n\t" \
  "v_mfma_f32_16x16x128_f8f6f4 a[112:115], v[152:159], v[224:231], a[112:115]\n\t" \
  "buffer_store_dwordx2 v[66:67], v5, s[40:43], s91 offen offset:128 nt\n\t" \
  "s_waitcnt lgkmcnt(0)\n\t" \
  "v_mfma_f32_16x16x128_f8f6f4 a[144:147], v[160:167], v[224:231], a[144:147]\n\t" \
  "s_waitcnt vmcnt(22)\n\t" \
  "s_barrier\n\t" \
  "v_mfma_f32_16x16x128_f8f6f4 a[148:151], v[160:167], v[232:239], a[148:151]\n\t" \
  "ds_read_b128 v[128:131], v2 offset:0\n\t" \
  "ds_read_b128 v[132:135], v2 offset:8192\n\t" \
  "v_mfma_f32_16x16x128_f8f6f4 a[152:155], v[160:167], v[240:247], a[152:155]\n\t" \
  "ds_read_b128 v[136:139], v2 offset:1024\n\t" \
  "ds_read_b128 v[140:143], v2 offset:9216\n\t" \
  "v_mfma_f32_16x16x128_f8f6f4 a[156:159], v[160:167], v[248:255], a[156:159]\n\t" \
  "ds_read_b128 v[144:147], v2 offset:2048\n\t" \
  "ds_read_b128 v[148:151], v2 offset:10240\n\t" \
  "v_mfma_f32_16x16x128_f8f6f4 a[188:191], v[168:175], v[248:255], a[188:191]\n\t" \
  "ds_read_b128 v[152:155], v2 offset:3072\n\t" \
  "ds_read_b128 v[156:159], v2 offset:11264\n\t" \
  "v_mfma_f32_16x16x128_f8f6f4 a[184:187], v[168:175], v[240:247], a[184:187]\n\t" \
  "v_mul_f32_e32 v68, s56, v68\n\t" \
  "v_mul_f32_e32 v69, s56, v69\n\t" \
  "v_mul_f32_e32 v70, s56, v70\n\t" \
  "v_mul_f32_e32 v71, s56, v71\n\t" \
  "v_mfma_f32_16x16x128_f8f6f4 a[180:183], v[168:175], v[232:239], a[180:183]\n\t" \
  "v_min_f32_e32 v28, s60, v68\n\t" \
  "v_min_f32_e32 v29, s60, v69\n\t" \
  "v_min_f32_e32 v30, s60, v70\n\t" \
  "v_min_f32_e32 v31, s60, v71\n\t" \
  "v_mfma_f32_16x16x128_f8f6f4 a[176:179], v[168:175], v[224:231], a[176:179]\n\t" \
  "s_mov_b32 s53, s51\n\t" \
  "s_add_u32 m0, s48, 32768\n\t" \
  "v_mfma_f32_16x16x128_f8f6f4 a[208:211], v[176:183], v[224:231], a[208:211]\n\t" \
  "buffer_load_dwordx4 v0, s[36:39], s53 offen sc0 sc1 lds\n\t" \
  "v_mfma_f32_16x16x128_f8f6f4 a[212:215], v[176:183], v[232:239], a[212:215]\n\t" \
  "v_exp_f32_e32 v28, v28\n\t" \
  "v_exp_f32_e32 v29, v29\n\t" \
  "v_mfma_f32_16x16x128_f8f6f4 a[216:219], v[176:183], v[240:247], a[216:219]\n\t" \
  "buffer_load_dwordx4 v0, s[36:39], s53 offen offset:1024 sc0 sc1 lds\n\t" \
  "v_mfma_f32_16x16x128_f8f6f4 a[220:223], v[176:183], v[248:255], a[220:223]\n\t" \
  "v_exp_f32_e32 v30, v30\n\t" \
  "v_exp_f32_e32 v31, v31\n\t" \
  "v_mfma_f32_16x16x128_f8f6f4 a[252:255], v[184:191], v[248:255], a[252:255]\n\t" \
  "buffer_load_dwordx4 v0, s[36:39], s53 offen offset:2048 sc0 sc1 lds\n\t" \
  "v_mfma_f32_16x16x128_f8f6f4 a[248:251], v[184:191], v[240:247], a[248:251]\n\t" \
  "v_fma_f32 v28, v28, s58, s58\n\t" \
  "v_fma_f32 v29, v29, s58, s58\n\t" \
  "v_fma_f32 v30, v30, s58, s58\n\t" \
  "v_fma_f32 v31, v31, s58, s58\n\t" \
  "v_mfma_f32_16x16x128_f8f6f4 a[244:247], v[184:191], v[232:239], a[244:247]\n\t" \
  "buffer_load_dwordx4 v0, s[36:39], s53 offen offset:3072 sc0 sc1 lds\n\t" \
  "v_mfma_f32_16x16x128_f8f6f4 a[240:243], v[184:191], v[224:231], a[240:243]\n\t" \
  "v_rcp_f32_e32 v28, v28\n\t" \
  "v_rcp_f32_e32 v29, v29\n\t" \
  "s_waitcnt lgkmcnt(0)\n\t" \
  "v_mfma_f32_16x16x128_f8f6f4 a[128:131], v[160:167], v[192:199], a[128:131]\n\t" \
  "v_rcp_f32_e32 v30, v30\n\t" \
  "v_rcp_f32_e32 v31, v31\n\t" \
  "v_mfma_f32_16x16x128_f8f6f4 a[132:135], v[160:167], v[200:207], a[132:135]\n\t" \
  "ds_read_b128 v[224:227], v4 offset:16384\n\t" \
  "ds_read_b128 v[228:231], v4 offset:24576\n\t" \
  "v_mfma_f32_16x16x128_f8f6f4 a[136:139], v[160:167], v[208:215], a[136:139]\n\t" \
  "ds_read_b128 v[232:235], v4 offset:17408\n\t" \
  "ds_read_b128 v[236:239], v4 offset:25600\n\t" \
  "v_mfma_f32_16x16x128_f8f6f4 a[140:143], v[160:167], v[216:223], a[140:143]\n\t" \
  "ds_read_b128 v[240:243], v4 offset:18432\n\t" \
  "ds_read_b128 v[244:247], v4 offset:26624\n\t" \
  "v_mfma_f32_16x16x128_f8f6f4 a[172:175], v[168:175], v[216:223], a[172:175]\n\t" \
  "ds_read_b128 v[248:251], v4 offset:19456\n\t" \
  "ds_read_b128 v[252:255], v4 offset:27648\n\t" \
  "v_mfma_f32_16x16x128_f8f6f4 a[168:171], v[168:175], v[208:215], a[168:171]\n\t" \
  "v_mfma_f32_16x16x128_f8f6f4 a[164:167], v[168:175], v[200:207], a[164:167]\n\t" \
  "v_mul_f32_e32 v68, v68, v28\n\t" \
  "v_mul_f32_e32 v69, v69, v29\n\t" \
  "v_mul_f32_e32 v70, v70, v30\n\t" \
  "v_mul_f32_e32 v71, v71, v31\n\t" \
  "v_mfma_f32_16x16x128_f8f6f4 a[160:163], v[168:175], v[192:199], a[160:163]\n\t" \
  "s_add_u32 s53, s49, 16384\n\t" \
  "s_add_u32 m0, s48, 49152\n\t" \
  "v_mfma_f32_16x16x128_f8f6f4 a[192:195], v[176:183], v[192:199], a[192:195]\n\t" \
  "buffer_load_dwordx4 v0, s[36:39], s53 offen sc0 sc1 lds\n\t" \
  "v_mfma_f32_16x16x128_f8f6f4 a[196:199], v[176:183], v[200:207], a[196:199]\n\t" \
  "s_xor_b32 s91, s68, 2162688\n\t" \
  "buffer_store_dwordx2 v[68:69], v5, s[40:43], s91 offen nt\n\t" \
  "v_mfma_f32_16x16x128_f8f6f4 a[200:203], v[176:183], v[208:215], a[200:203]\n\t" \
  "buffer_load_dwordx4 v0, s[36:39], s53 offen offset:1024 sc0 sc1 lds\n\t" \
  "v_mfma_f32_16x16x128_f8f6f4 a[204:207], v[176:183], v[216:223], a[204:207]\n\t" \
  "buffer_store_dwordx2 v[70:71], v5, s[40:43], s91 offen offset:128 nt\n\t" \
  "v_mfma_f32_16x16x128_f8f6f4 a[236:239], v[184:191], v[216:223], a[236:239]\n\t" \
  "buffer_load_dwordx4 v0, s[36:39], s53 offen offset:2048 sc0 sc1 lds\n\t" \
  "v_mfma_f32_16x16x128_f8f6f4 a[232:235], v[184:191], v[208:215], a[232:235]\n\t" \
  "v_mul_f32_e32 v72, s56, v72\n\t" \
  "v_mul_f32_e32 v73, s56, v73\n\t" \
  "v_mul_f32_e32 v74, s56, v74\n\t" \
  "v_mul_f32_e32 v75, s56, v75\n\t" \
  "v_mfma_f32_16x16x128_f8f6f4 a[228:231], v[184:191], v[200:207], a[228:231]\n\t" \
  "buffer_load_dwordx4 v0, s[36:39], s53 offen offset:3072 sc0 sc1 lds\n\t" \
  "v_mfma_f32_16x16x128_f8f6f4 a[224:227], v[184:191], v[192:199], a[224:227]\n\t" \
  "v_min_f32_e32 v12, s60, v72\n\t" \
  "v_min_f32_e32 v13, s60, v73\n\t" \
  "v_min_f32_e32 v14, s60, v74\n\t" \
  "v_min_f32_e32 v15, s60, v75\n\t" \
  "s_waitcnt lgkmcnt(0)\n\t" \
  "v_mfma_f32_16x16x128_f8f6f4 a[16:19], v[128:135], v[224:231], a[16:19]\n\t" \
  "s_waitcnt vmcnt(22)\n\t" \
  "s_barrier\n\t" \
  "v_mfma_f32_16x16x128_f8f6f4 a[20:23], v[128:135], v[232:239], a[20:23]\n\t" \
  "ds_read_b128 v[192:195], v4 offset:32768\n\t" \
  "ds_read_b128 v[196:199], v4 offset:40960\n\t" \
  "v_mfma_f32_16x16x128_f8f6f4 a[24:27], v[128:135], v[240:247], a[24:27]\n\t" \
  "ds_read_b128 v[200:203], v4 offset:33792\n\t" \
  "ds_read_b128 v[204:207], v4 offset:41984\n\t" \
  "v_mfma_f32_16x16x128_f8f6f4 a[28:31], v[128:135], v[248:255], a[28:31]\n\t" \
  "ds_read_b128 v[208:211], v4 offset:34816\n\t" \
  "ds_read_b128 v[212:215], v4 offset:43008\n\t" \
  "v_mfma_f32_16x16x128_f8f6f4 a[60:63], v[136:143], v[248:255], a[60:63]\n\t" \
  "ds_read_b128 v[216:219], v4 offset:35840\n\t" \
  "ds_read_b128 v[220:223], v4 offset:44032\n\t" \
  "v_mfma_f32_16x16x128_f8f6f4 a[56:59], v[136:143], v[240:247], a[56:59]\n\t" \
  "v_exp_f32_e32 v12, v12\n\t" \
  "v_exp_f32_e32 v13, v13\n\t" \
  "v_mfma_f32_16x16x128_f8f6f4 a[52:55], v[136:143], v[232:239], a[52:55]\n\t" \
  "v_exp_f32_e32 v14, v14\n\t" \
  "v_exp_f32_e32 v15, v15\n\t" \
  "v_mfma_f32_16x16x128_f8f6f4 a[48:51], v[136:143], v[224:231], a[48:51]\n\t" \
  "s_add_u32 s53, s49, 32768\n\t" \
  "s_add_u32 m0, s48, 65536\n\t" \
  "v_mfma_f32_16x16x128_f8f6f4 a[80:83], v[144:151], v[224:231], a[80:83]\n\t" \
  "buffer_load_dwordx4 v0, s[36:39], s53 offen sc0 sc1 lds\n\t" \
  "v_mfma_f32_16x16x128_f8f6f4 a[84:87], v[144:151], v[232:239], a[84:87]\n\t" \
  "buffer_load_dword v7, v8, s[44:47], s76 offen sc0 sc1\n\t" \
  "v_mfma_f32_16x16x128_f8f6f4 a[88:91], v[144:151], v[240:247], a[88:91]\n\t" \
  "buffer_load_dwordx4 v0, s[36:39], s53 offen offset:1024 sc0 sc1 lds\n\t" \
  "v_mfma_f32_16x16x128_f8f6f4 a[92:95], v[144:151], v[248:255], a[92:95]\n\t" \
  "v_fma_f32 v12, v12, s58, s58\n\t" \
  "v_fma_f32 v13, v13, s58, s58\n\t" \
  "v_fma_f32 v14, v14, s58, s58\n\t" \
  "v_fma_f32 v15, v15, s58, s58\n\t" \
  "v_mfma_f32_16x16x128_f8f6f4 a[124:127], v[152:159], v[248:255], a[124:127]\n\t" \
  "buffer_load_dwordx4 v0, s[36:39], s53 offen offset:2048 sc0 sc1 lds\n\t" \
  "v_mfma_f32_16x16x128_f8f6f4 a[120:123], v[152:159], v[240:247], a[120:123]\n\t" \
  "v_rcp_f32_e32 v12, v12\n\t" \
  "v_rcp_f32_e32 v13, v13\n\t" \
  "v_mfma_f32_16x16x128_f8f6f4 a[116:119], v[152:159], v[232:239], a[116:119]\n\t" \
  "buffer_load_dwordx4 v0, s[36:39], s53 offen offset:3072 sc0 sc1 lds\n\t" \
  "v_mfma_f32_16x16x128_f8f6f4 a[112:115], v[152:159], v[224:231], a[112:115]\n\t" \
  "v_rcp_f32_e32 v14, v14\n\t" \
  "v_rcp_f32_e32 v15, v15\n\t" \
  "s_waitcnt lgkmcnt(0)\n\t" \
  "v_mfma_f32_16x16x128_f8f6f4 a[0:3], v[128:135], v[192:199], a[0:3]\n\t" \
  "v_mul_f32_e32 v72, v72, v12\n\t" \
  "v_mul_f32_e32 v73, v73, v13\n\t" \
  "v_mul_f32_e32 v74, v74, v14\n\t" \
  "v_mul_f32_e32 v75, v75, v15\n\t" \
  "v_mfma_f32_16x16x128_f8f6f4 a[4:7], v[128:135], v[200:207], a[4:7]\n\t" \
  "ds_read_b128 v[160:163], v2 offset:49152\n\t" \
  "ds_read_b128 v[164:167], v2 offset:57344\n\t" \
  "v_mfma_f32_16x16x128_f8f6f4 a[8:11], v[128:135], v[208:215], a[8:11]\n\t" \
  "ds_read_b128 v[168:171], v2 offset:50176\n\t" \
  "ds_read_b128 v[172:175], v2 offset:58368\n\t" \
  "v_mfma_f32_16x16x128_f8f6f4 a[12:15], v[128:135], v[216:223], a[12:15]\n\t" \
  "ds_read_b128 v[176:179], v2 offset:51200\n\t" \
  "ds_read_b128 v[180:183], v2 offset:59392\n\t" \
  "v_mfma_f32_16x16x128_f8f6f4 a[44:47], v[136:143], v[216:223], a[44:47]\n\t" \
  "ds_read_b128 v[184:187], v2 offset:52224\n\t" \
  "ds_read_b128 v[188:191], v2 offset:60416\n\t" \
  "v_mfma_f32_16x16x128_f8f6f4 a[40:43], v[136:143], v[208:215], a[40:43]\n\t" \
  "s_xor_b32 s91, s68, 2228224\n\t" \
  "buffer_store_dwordx2 v[72:73], v5, s[40:43], s91 offen nt\n\t" \
  "v_mfma_f32_16x16x128_f8f6f4 a[36:39], v[136:143], v[200:207], a[36:39]\n\t" \
  "buffer_store_dwordx2 v[74:75], v5, s[40:43], s91 offen offset:128 nt\n\t" \
  "v_mfma_f32_16x16x128_f8f6f4 a[32:35], v[136:143], v[192:199], a[32:35]\n\t" \
  "s_add_u32 s53, s51, 16384\n\t" \
  "s_add_u32 m0, s48, 81920\n\t" \
  "v_mfma_f32_16x16x128_f8f6f4 a[64:67], v[144:151], v[192:199], a[64:67]\n\t" \
  "buffer_load_dwordx4 v0, s[36:39], s53 offen sc0 sc1 lds\n\t" \
  "v_mfma_f32_16x16x128_f8f6f4 a[68:71], v[144:151], v[200:207], a[68:71]\n\t" \
  "v_mul_f32_e32 v76, s56, v76\n\t" \
  "v_mul_f32_e32 v77, s56, v77\n\t" \
  "v_mul_f32_e32 v78, s56, v78\n\t" \
  "v_mul_f32_e32 v79, s56, v79\n\t" \
  "v_mfma_f32_16x16x128_f8f6f4 a[72:75], v[144:151], v[208:215], a[72:75]\n\t" \
  "buffer_load_dwordx4 v0, s[36:39], s53 offen offset:1024 sc0 sc1 lds\n\t" \
  "v_mfma_f32_16x16x128_f8f6f4 a[76:79], v[144:151], v[216:223], a[76:79]\n\t" \
  "v_min_f32_e32 v16, s60, v76\n\t" \
  "v_min_f32_e32 v17, s60, v77\n\t" \
  "v_min_f32_e32 v18, s60, v78\n\t" \
  "v_min_f32_e32 v19, s60, v79\n\t" \
  "v_mfma_f32_16x16x128_f8f6f4 a[108:111], v[152:159], v[216:223], a[108:111]\n\t" \
  "buffer_load_dwordx4 v0, s[36:39], s53 offen offset:2048 sc0 sc1 lds\n\t" \
  "v_mfma_f32_16x16x128_f8f6f4 a[104:107], v[152:159], v[208:215], a[104:107]\n\t" \
  "v_exp_f32_e32 v16, v16\n\t" \
  "v_exp_f32_e32 v17, v17\n\t" \
  "v_mfma_f32_16x16x128_f8f6f4 a[100:103], v[152:159], v[200:207], a[100:103]\n\t" \
  "buffer_load_dwordx4 v0, s[36:39], s53 offen offset:3072 sc0 sc1 lds\n\t" \
  "v_mfma_f32_16x16x128_f8f6f4 a[96:99], v[152:159], v[192:199], a[96:99]\n\t" \
  "v_exp_f32_e32 v18, v18\n\t" \
  "v_exp_f32_e32 v19, v19\n\t" \
  "s_waitcnt lgkmcnt(0)\n\t" \
  "v_mfma_f32_16x16x128_f8f6f4 a[128:131], v[160:167], v[192:199], a[128:131]\n\t" \
  "s_waitcnt vmcnt(22)\n\t" \
  "s_barrier\n\t" \
  "v_mfma_f32_16x16x128_f8f6f4 a[132:135], v[160:167], v[200:207], a[132:135]\n\t" \
  "ds_read_b128 v[128:131], v1 offset:0\n\t" \
  "ds_read_b128 v[132:135], v1 offset:8192\n\t" \
  "v_mfma_f32_16x16x128_f8f6f4 a[136:139], v[160:167], v[208:215], a[136:139]\n\t" \
  "ds_read_b128 v[136:139], v1 offset:1024\n\t" \
  "ds_read_b128 v[140:143], v1 offset:9216\n\t" \
  "v_mfma_f32_16x16x128_f8f6f4 a[140:143], v[160:167], v[216:223], a[140:143]\n\t" \
  "ds_read_b128 v[144:147], v1 offset:2048\n\t" \
  "ds_read_b128 v[148:151], v1 offset:10240\n\t" \
  "v_mfma_f32_16x16x128_f8f6f4 a[172:175], v[168:175], v[216:223], a[172:175]\n\t" \
  "ds_read_b128 v[152:155], v1 offset:3072\n\t" \
  "ds_read_b128 v[156:159], v1 offset:11264\n\t" \
  "v_mfma_f32_16x16x128_f8f6f4 a[168:171], v[168:175], v[208:215], a[168:171]\n\t" \
  "v_fma_f32 v16, v16, s58, s58\n\t" \
  "v_fma_f32 v17, v17, s58, s58\n\t" \
  "v_fma_f32 v18, v18, s58, s58\n\t" \
  "v_fma_f32 v19, v19, s58, s58\n\t" \
  "v_mfma_f32_16x16x128_f8f6f4 a[164:167], v[168:175], v[200:207], a[164:167]\n\t" \
  "v_rcp_f32_e32 v16, v16\n\t" \
  "v_rcp_f32_e32 v17, v17\n\t" \
  "v_mfma_f32_16x16x128_f8f6f4 a[160:163], v[168:175], v[192:199], a[160:163]\n\t" \
  "s_add_u32 s53, s50, 16384\n\t" \
  "s_add_u32 m0, s48, 98304\n\t" \
  "v_mfma_f32_16x16x128_f8f6f4 a[192:195], v[176:183], v[192:199], a[192:195]\n\t" \
  "buffer_load_dwordx4 v0, s[36:39], s53 offen sc0 sc1 lds\n\t" \
  "v_mfma_f32_16x16x128_f8f6f4 a[196:199], v[176:183], v[200:207], a[196:199]\n\t" \
  "v_rcp_f32_e32 v18, v18\n\t" \
  "v_rcp_f32_e32 v19, v19\n\t" \
  "v_mfma_f32_16x16x128_f8f6f4 a[200:203], v[176:183], v[208:215], a[200:203]\n\t" \
  "buffer_load_dwordx4 v0, s[36:39], s53 offen offset:1024 sc0 sc1 lds\n\t" \
  "v_mfma_f32_16x16x128_f8f6f4 a[204:207], v[176:183], v[216:223], a[204:207]\n\t" \
  "v_mul_f32_e32 v76, v76, v16\n\t" \
  "v_mul_f32_e32 v77, v77, v17\n\t" \
  "v_mul_f32_e32 v78, v78, v18\n\t" \
  "v_mul_f32_e32 v79, v79, v19\n\t" \
  "v_mfma_f32_16x16x128_f8f6f4 a[236:239], v[184:191], v[216:223], a[236:239]\n\t" \
  "buffer_load_dwordx4 v0, s[36:39], s53 offen offset:2048 sc0 sc1 lds\n\t" \
  "v_mfma_f32_16x16x128_f8f6f4 a[232:235], v[184:191], v[208:215], a[232:235]\n\t" \
  "s_xor_b32 s91, s68, 2293760\n\t" \
  "buffer_store_dwordx2 v[76:77], v5, s[40:43], s91 offen nt\n\t" \
  "v_mfma_f32_16x16x128_f8f6f4 a[228:231], v[184:191], v[200:207], a[228:231]\n\t" \
  "buffer_load_dwordx4 v0, s[36:39], s53 offen offset:3072 sc0 sc1 lds\n\t" \
  "v_mfma_f32_16x16x128_f8f6f4 a[224:227], v[184:191], v[192:199], a[224:227]\n\t" \
  "buffer_store_dwordx2 v[78:79], v5, s[40:43], s91 offen offset:128 nt\n\t" \
  "s_waitcnt lgkmcnt(0)\n\t" \
  "v_mfma_f32_16x16x128_f8f6f4 a[144:147], v[160:167], v[224:231], a[144:147]\n\t" \
  "v_mul_f32_e32 v80, s56, v80\n\t" \
  "v_mul_f32_e32 v81, s56, v81\n\t" \
  "v_mul_f32_e32 v82, s56, v82\n\t" \
  "v_mul_f32_e32 v83, s56, v83\n\t" \
  "v_mfma_f32_16x16x128_f8f6f4 a[148:151], v[160:167], v[232:239], a[148:151]\n\t" \
  "ds_read_b128 v[192:195], v3 offset:16384\n\t" \
  "ds_read_b128 v[196:199], v3 offset:24576\n\t" \
  "v_mfma_f32_16x16x128_f8f6f4 a[152:155], v[160:167], v[240:247], a[152:155]\n\t" \
  "ds_read_b128 v[200:203], v3 offset:17408\n\t" \
  "ds_read_b128 v[204:207], v3 offset:25600\n\t" \
  "v_mfma_f32_16x16x128_f8f6f4 a[156:159], v[160:167], v[248:255], a[156:159]\n\t" \
  "ds_read_b128 v[208:211], v3 offset:18432\n\t" \
  "ds_read_b128 v[212:215], v3 offset:26624\n\t" \
  "v_mfma_f32_16x16x128_f8f6f4 a[188:191], v[168:175], v[248:255], a[188:191]\n\t" \
  "ds_read_b128 v[216:219], v3 offset:19456\n\t" \
  "ds_read_b128 v[220:223], v3 offset:27648\n\t" \
  "v_mfma_f32_16x16x128_f8f6f4 a[184:187], v[168:175], v[240:247], a[184:187]\n\t" \
  "v_mfma_f32_16x16x128_f8f6f4 a[180:183], v[168:175], v[232:239], a[180:183]\n\t" \
  "v_min_f32_e32 v20, s60, v80\n\t" \
  "v_min_f32_e32 v21, s60, v81\n\t" \
  "v_min_f32_e32 v22, s60, v82\n\t" \
  "v_min_f32_e32 v23, s60, v83\n\t" \
  "v_mfma_f32_16x16x128_f8f6f4 a[176:179], v[168:175], v[224:231], a[176:179]\n\t" \
  "s_add_u32 s53, s49, 49152\n\t" \
  "s_add_u32 m0, s48, 114688\n\t" \
  "v_mfma_f32_16x16x128_f8f6f4 a[208:211], v[176:183], v[224:231], a[208:211]\n\t" \
  "buffer_load_dwordx4 v0, s[36:39], s53 offen sc0 sc1 lds\n\t" \
  "v_mfma_f32_16x16x128_f8f6f4 a[212:215], v[176:183], v[232:239], a[212:215]\n\t" \
  "v_exp_f32_e32 v20, v20\n\t" \
  "v_exp_f32_e32 v21, v21\n\t" \
  "v_mfma_f32_16x16x128_f8f6f4 a[216:219], v[176:183], v[240:247], a[216:219]\n\t" \
  "buffer_load_dwordx4 v0, s[36:39], s53 offen offset:1024 sc0 sc1 lds\n\t" \
  "v_mfma_f32_16x16x128_f8f6f4 a[220:223], v[176:183], v[248:255], a[220:223]\n\t" \
  "v_exp_f32_e32 v22, v22\n\t" \
  "v_exp_f32_e32 v23, v23\n\t" \
  "v_mfma_f32_16x16x128_f8f6f4 a[252:255], v[184:191], v[248:255], a[252:255]\n\t" \
  "buffer_load_dwordx4 v0, s[36:39], s53 offen offset:2048 sc0 sc1 lds\n\t" \
  "v_mfma_f32_16x16x128_f8f6f4 a[248:251], v[184:191], v[240:247], a[248:251]\n\t" \
  "v_fma_f32 v20, v20, s58, s58\n\t" \
  "v_fma_f32 v21, v21, s58, s58\n\t" \
  "v_fma_f32 v22, v22, s58, s58\n\t" \
  "v_fma_f32 v23, v23, s58, s58\n\t" \
  "v_mfma_f32_16x16x128_f8f6f4 a[244:247], v[184:191], v[232:239], a[244:247]\n\t" \
  "buffer_load_dwordx4 v0, s[36:39], s53 offen offset:3072 sc0 sc1 lds\n\t" \
  "v_mfma_f32_16x16x128_f8f6f4 a[240:243], v[184:191], v[224:231], a[240:243]\n\t" \
  "s_add_u32 s49, s49, 65536\n\t" \
  "s_add_u32 s50, s50, 32768\n\t" \
  "s_add_u32 s51, s51, 32768\n\t" \
  "s_waitcnt lgkmcnt(0)\n\t" \
  "v_mfma_f32_16x16x128_f8f6f4 a[0:3], v[128:135], v[192:199], a[0:3]\n\t" \
  "s_waitcnt vmcnt(21)\n\t" \
  "s_barrier\n\t" \
  "v_mfma_f32_16x16x128_f8f6f4 a[4:7], v[128:135], v[200:207], a[4:7]\n\t" \
  "ds_read_b128 v[224:227], v3 offset:32768\n\t" \
  "ds_read_b128 v[228:231], v3 offset:40960\n\t" \
  "v_mfma_f32_16x16x128_f8f6f4 a[8:11], v[128:135], v[208:215], a[8:11]\n\t" \
  "ds_read_b128 v[232:235], v3 offset:33792\n\t" \
  "ds_read_b128 v[236:239], v3 offset:41984\n\t" \
  "v_mfma_f32_16x16x128_f8f6f4 a[12:15], v[128:135], v[216:223], a[12:15]\n\t" \
  "ds_read_b128 v[240:243], v3 offset:34816\n\t" \
  "ds_read_b128 v[244:247], v3 offset:43008\n\t" \
  "v_mfma_f32_16x16x128_f8f6f4 a[44:47], v[136:143], v[216:223], a[44:47]\n\t" \
  "ds_read_b128 v[248:251], v3 offset:35840\n\t" \
  "ds_read_b128 v[252:255], v3 offset:44032\n\t" \
  "v_mfma_f32_16x16x128_f8f6f4 a[40:43], v[136:143], v[208:215], a[40:43]\n\t" \
  "s_waitcnt vmcnt(19)\n\t" \
  "v_cmp_le_u32_e32 vcc, s75, v7\n\t" \
  "v_mfma_f32_16x16x128_f8f6f4 a[36:39], v[136:143], v[200:207], a[36:39]\n\t" \
  "s_cmp_eq_u64 vcc, -1\n\t" \
  "s_cbranch_scc0 L_SLOW13_%=\n\t" \
  "L_OK13_%=:\n\t" \
  "s_add_u32 s76, s76, 1024\n\t" \
  "v_mfma_f32_16x16x128_f8f6f4 a[32:35], v[136:143], v[192:199], a[32:35]\n\t" \
  "s_mov_b32 s53, s49\n\t" \
  "s_mov_b32 m0, s48\n\t" \
  "v_mfma_f32_16x16x128_f8f6f4 a[64:67], v[144:151], v[192:199], a[64:67]\n\t" \
  "buffer_load_dwordx4 v0, s[36:39], s53 offen sc0 sc1 lds\n\t" \
  "v_mfma_f32_16x16x128_f8f6f4 a[68:71], v[144:151], v[200:207], a[68:71]\n\t" \
  "v_rcp_f32_e32 v20, v20\n\t" \
  "v_rcp_f32_e32 v21, v21\n\t" \
  "v_mfma_f32_16x16x128_f8f6f4 a[72:75], v[144:151], v[208:215], a[72:75]\n\t" \
  "buffer_load_dwordx4 v0, s[36:39], s53 offen offset:1024 sc0 sc1 lds\n\t" \
  "v_mfma_f32_16x16x128_f8f6f4 a[76:79], v[144:151], v[216:223], a[76:79]\n\t" \
  "v_rcp_f32_e32 v22, v22\n\t" \
  "v_rcp_f32_e32 v23, v23\n\t" \
  "v_mfma_f32_16x16x128_f8f6f4 a[108:111], v[152:159], v[216:223], a[108:111]\n\t" \
  "buffer_load_dwordx4 v0, s[36:39], s53 offen offset:2048 sc0 sc1 lds\n\t" \
  "v_mfma_f32_16x16x128_f8f6f4 a[104:107], v[152:159], v[208:215], a[104:107]\n\t" \
  "v_mul_f32_e32 v80, v80, v20\n\t" \
  "v_mul_f32_e32 v81, v81, v21\n\t" \
  "v_mul_f32_e32 v82, v82, v22\n\t" \
  "v_mul_f32_e32 v83, v83, v23\n\t" \
  "v_mfma_f32_16x16x128_f8f6f4 a[100:103], v[152:159], v[200:207], a[100:103]\n\t" \
  "buffer_load_dwordx4 v0, s[36:39], s53 offen offset:3072 sc0 sc1 lds\n\t" \
  "v_mfma_f32_16x16x128_f8f6f4 a[96:99], v[152:159], v[192:199], a[96:99]\n\t" \
  "s_xor_b32 s91, s68, 3145728\n\t" \
  "buffer_store_dwordx2 v[80:81], v5, s[40:43], s91 offen nt\n\t" \
  "s_waitcnt lgkmcnt(0)\n\t" \
  "v_mfma_f32_16x16x128_f8f6f4 a[16:19], v[128:135], v[224:231], a[16:19]\n\t" \
  "buffer_store_dwordx2 v[82:83], v5, s[40:43], s91 offen offset:128 nt\n\t" \
  "v_mfma_f32_16x16x128_f8f6f4 a[20:23], v[128:135], v[232:239], a[20:23]\n\t" \
  "ds_read_b128 v[160:163], v1 offset:49152\n\t" \
  "ds_read_b128 v[164:167], v1 offset:57344\n\t" \
  "v_mfma_f32_16x16x128_f8f6f4 a[24:27], v[128:135], v[240:247], a[24:27]\n\t" \
  "ds_read_b128 v[168:171], v1 offset:50176\n\t" \
  "ds_read_b128 v[172:175], v1 offset:58368\n\t" \
  "v_mfma_f32_16x16x128_f8f6f4 a[28:31], v[128:135], v[248:255], a[28:31]\n\t" \
  "ds_read_b128 v[176:179], v1 offset:51200\n\t" \
  "ds_read_b128 v[180:183], v1 offset:59392\n\t" \
  "v_mfma_f32_16x16x128_f8f6f4 a[60:63], v[136:143], v[248:255], a[60:63]\n\t" \
  "ds_read_b128 v[184:187], v1 offset:52224\n\t" \
  "ds_read_b128 v[188:191], v1 offset:60416\n\t" \
  "v_mfma_f32_16x16x128_f8f6f4 a[56:59], v[136:143], v[240:247], a[56:59]\n\t" \
  "v_mul_f32_e32 v84, s56, v84\n\t" \
  "v_mul_f32_e32 v85, s56, v85\n\t" \
  "v_mul_f32_e32 v86, s56, v86\n\t" \
  "v_mul_f32_e32 v87, s56, v87\n\t" \
  "v_mfma_f32_16x16x128_f8f6f4 a[52:55], v[136:143], v[232:239], a[52:55]\n\t" \
  "v_min_f32_e32 v24, s60, v84\n\t" \
  "v_min_f32_e32 v25, s60, v85\n\t" \
  "v_min_f32_e32 v26, s60, v86\n\t" \
  "v_min_f32_e32 v27, s60, v87\n\t" \
  "v_mfma_f32_16x16x128_f8f6f4 a[48:51], v[136:143], v[224:231], a[48:51]\n\t" \
  "s_mov_b32 s53, s50\n\t" \
  "s_add_u32 m0, s48, 16384\n\t" \
  "v_mfma_f32_16x16x128_f8f6f4 a[80:83], v[144:151], v[224:231], a[80:83]\n\t" \
  "buffer_load_dwordx4 v0, s[36:39], s53 offen sc0 sc1 lds\n\t" \
  "v_mfma_f32_16x16x128_f8f6f4 a[84:87], v[144:151], v[232:239], a[84:87]\n\t" \
  "v_exp_f32_e32 v24, v24\n\t" \
  "v_exp_f32_e32 v25, v25\n\t" \
  "v_mfma_f32_16x16x128_f8f6f4 a[88:91], v[144:151], v[240:247], a[88:91]\n\t" \
  "buffer_load_dwordx4 v0, s[36:39], s53 offen offset:1024 sc0 sc1 lds\n\t" \
  "v_mfma_f32_16x16x128_f8f6f4 a[92:95], v[144:151], v[248:255], a[92:95]\n\t" \
  "v_exp_f32_e32 v26, v26\n\t" \
  "v_exp_f32_e32 v27, v27\n\t" \
  "v_mfma_f32_16x16x128_f8f6f4 a[124:127], v[152:159], v[248:255], a[124:127]\n\t" \
  "buffer_load_dwordx4 v0, s[36:39], s53 offen offset:2048 sc0 sc1 lds\n\t" \
  "v_mfma_f32_16x16x128_f8f6f4 a[120:123], v[152:159], v[240:247], a[120:123]\n\t" \
  "v_fma_f32 v24, v24, s58, s58\n\t" \
  "v_fma_f32 v25, v25, s58, s58\n\t" \
  "v_fma_f32 v26, v26, s58, s58\n\t" \
  "v_fma_f32 v27, v27, s58, s58\n\t" \
  "v_mfma_f32_16x16x128_f8f6f4 a[116:119], v[152:159], v[232:239], a[116:119]\n\t" \
  "buffer_load_dwordx4 v0, s[36:39], s53 offen offset:3072 sc0 sc1 lds\n\t" \
  "v_mfma_f32_16x16x128_f8f6f4 a[112:115], v[152:159], v[224:231], a[112:115]\n\t" \
  "v_rcp_f32_e32 v24, v24\n\t" \
  "v_rcp_f32_e32 v25, v25\n\t" \
  "s_waitcnt lgkmcnt(0)\n\t" \
  "v_mfma_f32_16x16x128_f8f6f4 a[144:147], v[160:167], v[224:231], a[144:147]\n\t" \
  "s_waitcnt vmcnt(20)\n\t" \
  "s_barrier\n\t" \
  "v_mfma_f32_16x16x128_f8f6f4 a[148:151], v[160:167], v[232:239], a[148:151]\n\t" \
  "ds_read_b128 v[128:131], v2 offset:0\n\t" \
  "ds_read_b128 v[132:135], v2 offset:8192\n\t" \
  "v_mfma_f32_16x16x128_f8f6f4 a[152:155], v[160:167], v[240:247], a[152:155]\n\t" \
  "ds_read_b128 v[136:139], v2 offset:1024\n\t" \
  "ds_read_b128 v[140:143], v2 offset:9216\n\t" \
  "v_mfma_f32_16x16x128_f8f6f4 a[156:159], v[160:167], v[248:255], a[156:159]\n\t" \
  "ds_read_b128 v[144:147], v2 offset:2048\n\t" \
  "ds_read_b128 v[148:151], v2 offset:10240\n\t" \
  "v_mfma_f32_16x16x128_f8f6f4 a[188:191], v[168:175], v[248:255], a[188:191]\n\t" \
  "ds_read_b128 v[152:155], v2 offset:3072\n\t" \
  "ds_read_b128 v[156:159], v2 offset:11264\n\t" \
  "v_mfma_f32_16x16x128_f8f6f4 a[184:187], v[168:175], v[240:247], a[184:187]\n\t" \
  "v_rcp_f32_e32 v26, v26\n\t" \
  "v_rcp_f32_e32 v27, v27\n\t" \
  "v_mfma_f32_16x16x128_f8f6f4 a[180:183], v[168:175], v[232:239], a[180:183]\n\t" \
  "v_mul_f32_e32 v84, v84, v24\n\t" \
  "v_mul_f32_e32 v85, v85, v25\n\t" \
  "v_mul_f32_e32 v86, v86, v26\n\t" \
  "v_mul_f32_e32 v87, v87, v27\n\t" \
  "v_mfma_f32_16x16x128_f8f6f4 a[176:179], v[168:175], v[224:231], a[176:179]\n\t" \
  "s_mov_b32 s53, s51\n\t" \
  "s_add_u32 m0, s48, 32768\n\t" \
  "v_mfma_f32_16x16x128_f8f6f4 a[208:211], v[176:183], v[224:231], a[208:211]\n\t" \
  "buffer_load_dwordx4 v0, s[36:39], s53 offen sc0 sc1 lds\n\t" \
  "v_mfma_f32_16x16x128_f8f6f4 a[212:215], v[176:183], v[232:239], a[212:215]\n\t" \
  "s_xor_b32 s91, s68, 3211264\n\t" \
  "buffer_store_dwordx2 v[84:85], v5, s[40:43], s91 offen nt\n\t" \
  "v_mfma_f32_16x16x128_f8f6f4 a[216:219], v[176:183], v[240:247], a[216:219]\n\t" \
  "buffer_load_dwordx4 v0, s[36:39], s53 offen offset:1024 sc0 sc1 lds\n\t" \
  "v_mfma_f32_16x16x128_f8f6f4 a[220:223], v[176:183], v[248:255], a[220:223]\n\t" \
  "buffer_store_dwordx2 v[86:87], v5, s[40:43], s91 offen offset:128 nt\n\t" \
  "v_mfma_f32_16x16x128_f8f6f4 a[252:255], v[184:191], v[248:255], a[252:255]\n\t" \
  "buffer_load_dwordx4 v0, s[36:39], s53 offen offset:2048 sc0 sc1 lds\n\t" \
  "v_mfma_f32_16x16x128_f8f6f4 a[248:251], v[184:191], v[240:247], a[248:251]\n\t" \
  "v_mul_f32_e32 v88, s56, v88\n\t" \
  "v_mul_f32_e32 v89, s56, v89\n\t" \
  "v_mul_f32_e32 v90, s56, v90\n\t" \
  "v_mul_f32_e32 v91, s56, v91\n\t" \
  "v_mfma_f32_16x16x128_f8f6f4 a[244:247], v[184:191], v[232:239], a[244:247]\n\t" \
  "buffer_load_dwordx4 v0, s[36:39], s53 offen offset:3072 sc0 sc1 lds\n\t" \
  "v_mfma_f32_16x16x128_f8f6f4 a[240:243], v[184:191], v[224:231], a[240:243]\n\t" \
  "v_min_f32_e32 v28, s60, v88\n\t" \
  "v_min_f32_e32 v29, s60, v89\n\t" \
  "v_min_f32_e32 v30, s60, v90\n\t" \
  "v_min_f32_e32 v31, s60, v91\n\t" \
  "s_waitcnt lgkmcnt(0)\n\t" \
  "v_mfma_f32_16x16x128_f8f6f4 a[128:131], v[160:167], v[192:199], a[128:131]\n\t" \
  "v_exp_f32_e32 v28, v28\n\t" \
  "v_exp_f32_e32 v29, v29\n\t" \
  "v_mfma_f32_16x16x128_f8f6f4 a[132:135], v[160:167], v[200:207], a[132:135]\n\t" \
  "ds_read_b128 v[224:227], v4 offset:16384\n\t" \
  "ds_read_b128 v[228:231], v4 offset:24576\n\t" \
  "v_mfma_f32_16x16x128_f8f6f4 a[136:139], v[160:167], v[208:215], a[136:139]\n\t" \
  "ds_read_b128 v[232:235], v4 offset:17408\n\t" \
  "ds_read_b128 v[236:239], v4 offset:25600\n\t" \
  "v_mfma_f32_16x16x128_f8f6f4 a[140:143], v[160:167], v[216:223], a[140:143]\n\t" \
  "ds_read_b128 v[240:243], v4 offset:18432\n\t" \
  "ds_read_b128 v[244:247], v4 offset:26624\n\t" \
  "v_mfma_f32_16x16x128_f8f6f4 a[172:175], v[168:175], v[216:223], a[172:175]\n\t" \
  "ds_read_b128 v[248:251], v4 offset:19456\n\t" \
  "ds_read_b128 v[252:255], v4 offset:27648\n\t" \
  "v_mfma_f32_16x16x128_f8f6f4 a[168:171], v[168:175], v[208:215], a[168:171]\n\t" \
  "v_exp_f32_e32 v30, v30\n\t" \
  "v_exp_f32_e32 v31, v31\n\t" \
  "v_mfma_f32_16x16x128_f8f6f4 a[164:167], v[168:175], v[200:207], a[164:167]\n\t" \
  "v_fma_f32 v28, v28, s58, s58\n\t" \
  "v_fma_f32 v29, v29, s58, s58\n\t" \
  "v_fma_f32 v30, v30, s58, s58\n\t" \
  "v_fma_f32 v31, v31, s58, s58\n\t" \
  "v_mfma_f32_16x16x128_f8f6f4 a[160:163], v[168:175], v[192:199], a[160:163]\n\t" \
  "s_add_u32 s53, s49, 16384\n\t" \
  "s_add_u32 m0, s48, 49152\n\t" \
  "v_mfma_f32_16x16x128_f8f6f4 a[192:195], v[176:183], v[192:199], a[192:195]\n\t" \
  "buffer_load_dwordx4 v0, s[36:39], s53 offen sc0 sc1 lds\n\t" \
  "v_mfma_f32_16x16x128_f8f6f4 a[196:199], v[176:183], v[200:207], a[196:199]\n\t" \
  "v_mfma_f32_16x16x128_f8f6f4 a[200:203], v[176:183], v[208:215], a[200:203]\n\t" \
  "buffer_load_dwordx4 v0, s[36:39], s53 offen offset:1024 sc0 sc1 lds\n\t" \
  "v_mfma_f32_16x16x128_f8f6f4 a[204:207], v[176:183], v[216:223], a[204:207]\n\t" \
  "v_rcp_f32_e32 v28, v28\n\t" \
  "v_rcp_f32_e32 v29, v29\n\t" \
  "v_mfma_f32_16x16x128_f8f6f4 a[236:239], v[184:191], v[216:223], a[236:239]\n\t" \
  "buffer_load_dwordx4 v0, s[36:39], s53 offen offset:2048 sc0 sc1 lds\n\t" \
  "v_mfma_f32_16x16x128_f8f6f4 a[232:235], v[184:191], v[208:215], a[232:235]\n\t" \
  "v_rcp_f32_e32 v30, v30\n\t" \
  "v_rcp_f32_e32 v31, v31\n\t" \
  "v_mfma_f32_16x16x128_f8f6f4 a[228:231], v[184:191], v[200:207], a[228:231]\n\t" \
  "buffer_load_dwordx4 v0, s[36:39], s53 offen offset:3072 sc0 sc1 lds\n\t" \
  "v_mfma_f32_16x16x128_f8f6f4 a[224:227], v[184:191], v[192:199], a[224:227]\n\t" \
  "v_mul_f32_e32 v88, v88, v28\n\t" \
  "v_mul_f32_e32 v89, v89, v29\n\t" \
  "v_mul_f32_e32 v90, v90, v30\n\t" \
  "v_mul_f32_e32 v91, v91, v31\n\t" \
  "s_waitcnt lgkmcnt(0)\n\t" \
  "v_mfma_f32_16x16x128_f8f6f4 a[16:19], v[128:135], v[224:231], a[16:19]\n\t" \
  "s_waitcnt vmcnt(20)\n\t" \
  "s_barrier\n\t" \
  "v_mfma_f32_16x16x128_f8f6f4 a[20:23], v[128:135], v[232:239], a[20:23]\n\t" \
  "ds_read_b128 v[192:195], v4 offset:32768\n\t" \
  "ds_read_b128 v[196:199], v4 offset:40960\n\t" \
  "v_mfma_f32_16x16x128_f8f6f4 a[24:27], v[128:135], v[240:247], a[24:27]\n\t" \
  "ds_read_b128 v[200:203], v4 offset:33792\n\t" \
  "ds_read_b128 v[204:207], v4 offset:41984\n\t" \
  "v_mfma_f32_16x16x128_f8f6f4 a[28:31], v[128:135], v[248:255], a[28:31]\n\t" \
  "ds_read_b128 v[208:211], v4 offset:34816\n\t" \
  "ds_read_b128 v[212:215], v4 offset:43008\n\t" \
  "v_mfma_f32_16x16x128_f8f6f4 a[60:63], v[136:143], v[248:255], a[60:63]\n\t" \
  "ds_read_b128 v[216:219], v4 offset:35840\n\t" \
  "ds_read_b128 v[220:223], v4 offset:44032\n\t" \
  "v_mfma_f32_16x16x128_f8f6f4 a[56:59], v[136:143], v[240:247], a[56:59]\n\t" \
  "s_xor_b32 s91, s68, 3276800\n\t" \
  "buffer_store_dwordx2 v[88:89], v5, s[40:43], s91 offen nt\n\t" \
  "v_mfma_f32_16x16x128_f8f6f4 a[52:55], v[136:143], v[232:239], a[52:55]\n\t" \
  "buffer_store_dwordx2 v[90:91], v5, s[40:43], s91 offen offset:128 nt\n\t" \
  "v_mfma_f32_16x16x128_f8f6f4 a[48:51], v[136:143], v[224:231], a[48:51]\n\t" \
  "s_add_u32 s53, s49, 32768\n\t" \
  "s_add_u32 m0, s48, 65536\n\t" \
  "v_mfma_f32_16x16x128_f8f6f4 a[80:83], v[144:151], v[224:231], a[80:83]\n\t" \
  "buffer_load_dwordx4 v0, s[36:39], s53 offen sc0 sc1 lds\n\t" \
  "v_mfma_f32_16x16x128_f8f6f4 a[84:87], v[144:151], v[232:239], a[84:87]\n\t" \
  "v_mul_f32_e32 v92, s56, v92\n\t" \
  "v_mul_f32_e32 v93, s56, v93\n\t" \
  "v_mul_f32_e32 v94, s56, v94\n\t" \
  "v_mul_f32_e32 v95, s56, v95\n\t" \
  "v_mfma_f32_16x16x128_f8f6f4 a[88:91], v[144:151], v[240:247], a[88:91]\n\t" \
  "buffer_load_dwordx4 v0, s[36:39], s53 offen offset:1024 sc0 sc1 lds\n\t" \
  "v_mfma_f32_16x16x128_f8f6f4 a[92:95], v[144:151], v[248:255], a[92:95]\n\t" \
  "v_min_f32_e32 v12, s60, v92\n\t" \
  "v_min_f32_e32 v13, s60, v93\n\t" \
  "v_min_f32_e32 v14, s60, v94\n\t" \
  "v_min_f32_e32 v15, s60, v95\n\t" \
  "v_mfma_f32_16x16x128_f8f6f4 a[124:127], v[152:159], v[248:255], a[124:127]\n\t" \
  "buffer_load_dwordx4 v0, s[36:39], s53 offen offset:2048 sc0 sc1 lds\n\t" \
  "v_mfma_f32_16x16x128_f8f6f4 a[120:123], v[152:159], v[240:247], a[120:123]\n\t" \
  "v_exp_f32_e32 v12, v12\n\t" \
  "v_exp_f32_e32 v13, v13\n\t" \
  "v_mfma_f32_16x16x128_f8f6f4 a[116:119], v[152:159], v[232:239], a[116:119]\n\t" \
  "buffer_load_dwordx4 v0, s[36:39], s53 offen offset:3072 sc0 sc1 lds\n\t" \
  "v_mfma_f32_16x16x128_f8f6f4 a[112:115], v[152:159], v[224:231], a[112:115]\n\t" \
  "v_exp_f32_e32 v14, v14\n\t" \
  "v_exp_f32_e32 v15, v15\n\t" \
  "s_waitcnt lgkmcnt(0)\n\t" \
  "v_mfma_f32_16x16x128_f8f6f4 a[0:3], v[128:135], v[192:199], a[0:3]\n\t" \
  "v_fma_f32 v12, v12, s58, s58\n\t" \
  "v_fma_f32 v13, v13, s58, s58\n\t" \
  "v_fma_f32 v14, v14, s58, s58\n\t" \
  "v_fma_f32 v15, v15, s58, s58\n\t" \
  "v_mfma_f32_16x16x128_f8f6f4 a[4:7], v[128:135], v[200:207], a[4:7]\n\t" \
  "ds_read_b128 v[160:163], v2 offset:49152\n\t" \
  "ds_read_b128 v[164:167], v2 offset:57344\n\t" \
  "v_mfma_f32_16x16x128_f8f6f4 a[8:11], v[128:135], v[208:215], a[8:11]\n\t" \
  "ds_read_b128 v[168:171], v2 offset:50176\n\t" \
  "ds_read_b128 v[172:175], v2 offset:58368\n\t" \
  "v_mfma_f32_16x16x128_f8f6f4 a[12:15], v[128:135], v[216:223], a[12:15]\n\t" \
  "ds_read_b128 v[176:179], v2 offset:51200\n\t" \
  "ds_read_b128 v[180:183], v2 offset:59392\n\t" \
  "v_mfma_f32_16x16x128_f8f6f4 a[44:47], v[136:143], v[216:223], a[44:47]\n\t" \
  "ds_read_b128 v[184:187], v2 offset:52224\n\t" \
  "ds_read_b128 v[188:191], v2 offset:60416\n\t" \
  "v_mfma_f32_16x16x128_f8f6f4 a[40:43], v[136:143], v[208:215], a[40:43]\n\t" \
  "v_rcp_f32_e32 v12, v12\n\t" \
  "v_rcp_f32_e32 v13, v13\n\t" \
  "v_mfma_f32_16x16x128_f8f6f4 a[36:39], v[136:143], v[200:207], a[36:39]\n\t" \
  "v_rcp_f32_e32 v14, v14\n\t" \
  "v_rcp_f32_e32 v15, v15\n\t" \
  "v_mfma_f32_16x16x128_f8f6f4 a[32:35], v[136:143], v[192:199], a[32:35]\n\t" \
  "s_add_u32 s53, s51, 16384\n\t" \
  "s_add_u32 m0, s48, 81920\n\t" \
  "v_mfma_f32_16x16x128_f8f6f4 a[64:67], v[144:151], v[192:199], a[64:67]\n\t" \
  "buffer_load_dwordx4 v0, s[36:39], s53 offen sc0 sc1 lds\n\t" \
  "v_mfma_f32_16x16x128_f8f6f4 a[68:71], v[144:151], v[200:207], a[68:71]\n\t" \
  "v_mul_f32_e32 v92, v92, v12\n\t" \
  "v_mul_f32_e32 v93, v93, v13\n\t" \
  "v_mul_f32_e32 v94, v94, v14\n\t" \
  "v_mul_f32_e32 v95, v95, v15\n\t" \
  "v_mfma_f32_16x16x128_f8f6f4 a[72:75], v[144:151], v[208:215], a[72:75]\n\t" \
  "buffer_load_dwordx4 v0, s[36:39], s53 offen offset:1024 sc0 sc1 lds\n\t" \
  "v_mfma_f32_16x16x128_f8f6f4 a[76:79], v[144:151], v[216:223], a[76:79]\n\t" \
  "s_xor_b32 s91, s68, 3342336\n\t" \
  "buffer_store_dwordx2 v[92:93], v5, s[40:43], s91 offen nt\n\t" \
  "v_mfma_f32_16x16x128_f8f6f4 a[108:111], v[152:159], v[216:223], a[108:111]\n\t" \
  "buffer_load_dwordx4 v0, s[36:39], s53 offen offset:2048 sc0 sc1 lds\n\t" \
  "v_mfma_f32_16x16x128_f8f6f4 a[104:107], v[152:159], v[208:215], a[104:107]\n\t" \
  "buffer_store_dwordx2 v[94:95], v5, s[40:43], s91 offen offset:128 nt\n\t" \
  "v_mfma_f32_16x16x128_f8f6f4 a[100:103], v[152:159], v[200:207], a[100:103]\n\t" \
  "buffer_load_dwordx4 v0, s[36:39], s53 offen offset:3072 sc0 sc1 lds\n\t" \
  "v_mfma_f32_16x16x128_f8f6f4 a[96:99], v[152:159], v[192:199], a[96:99]\n\t" \
  "v_mul_f32_e32 v96, s56, v96\n\t" \
  "v_mul_f32_e32 v97, s56, v97\n\t" \
  "v_mul_f32_e32 v98, s56, v98\n\t" \
  "v_mul_f32_e32 v99, s56, v99\n\t" \
  "s_waitcnt lgkmcnt(0)\n\t" \
  "v_mfma_f32_16x16x128_f8f6f4 a[128:131], v[160:167], v[192:199], a[128:131]\n\t" \
  "s_waitcnt vmcnt(22)\n\t" \
  "s_barrier\n\t" \
  "v_mfma_f32_16x16x128_f8f6f4 a[132:135], v[160:167], v[200:207], a[132:135]\n\t" \
  "ds_read_b128 v[128:131], v1 offset:0\n\t" \
  "ds_read_b128 v[132:135], v1 offset:8192\n\t" \
  "v_mfma_f32_16x16x128_f8f6f4 a[136:139], v[160:167], v[208:215], a[136:139]\n\t" \
  "ds_read_b128 v[136:139], v1 offset:1024\n\t" \
  "ds_read_b128 v[140:143], v1 offset:9216\n\t" \
  "v_mfma_f32_16x16x128_f8f6f4 a[140:143], v[160:167], v[216:223], a[140:143]\n\t" \
  "ds_read_b128 v[144:147], v1 offset:2048\n\t" \
  "ds_read_b128 v[148:151], v1 offset:10240\n\t" \
  "v_mfma_f32_16x16x128_f8f6f4 a[172:175], v[168:175], v[216:223], a[172:175]\n\t" \
  "ds_read_b128 v[152:155], v1 offset:3072\n\t" \
  "ds_read_b128 v[156:159], v1 offset:11264\n\t" \
  "v_mfma_f32_16x16x128_f8f6f4 a[168:171], v[168:175], v[208:215], a[168:171]\n\t" \
  "v_min_f32_e32 v16, s60, v96\n\t" \
  "v_min_f32_e32 v17, s60, v97\n\t" \
  "v_min_f32_e32 v18, s60, v98\n\t" \
  "v_min_f32_e32 v19, s60, v99\n\t" \
  "v_mfma_f32_16x16x128_f8f6f4 a[164:167], v[168:175], v[200:207], a[164:167]\n\t" \
  "v_exp_f32_e32 v16, v16\n\t" \
  "v_exp_f32_e32 v17, v17\n\t" \
  "v_mfma_f32_16x16x128_f8f6f4 a[160:163], v[168:175], v[192:199], a[160:163]\n\t" \
  "s_add_u32 s53, s50, 16384\n\t" \
  "s_add_u32 m0, s48, 98304\n\t" \
  "v_mfma_f32_16x16x128_f8f6f4 a[192:195], v[176:183], v[192:199], a[192:195]\n\t" \
  "buffer_load_dwordx4 v0, s[36:39], s53 offen sc0 sc1 lds\n\t" \
  "v_mfma_f32_16x16x128_f8f6f4 a[196:199], v[176:183], v[200:207], a[196:199]\n\t" \
  "v_exp_f32_e32 v18, v18\n\t" \
  "v_exp_f32_e32 v19, v19\n\t" \
  "v_mfma_f32_16x16x128_f8f6f4 a[200:203], v[176:183], v[208:215], a[200:203]\n\t" \
  "buffer_load_dwordx4 v0, s[36:39], s53 offen offset:1024 sc0 sc1 lds\n\t" \
  "v_mfma_f32_16x16x128_f8f6f4 a[204:207], v[176:183], v[216:223], a[204:207]\n\t" \
  "v_fma_f32 v16, v16, s58, s58\n\t" \
  "v_fma_f32 v17, v17, s58, s58\n\t" \
  "v_fma_f32 v18, v18, s58, s58\n\t" \
  "v_fma_f32 v19, v19, s58, s58\n\t" \
  "v_mfma_f32_16x16x128_f8f6f4 a[236:239], v[184:191], v[216:223], a[236:239]\n\t" \
  "buffer_load_dwordx4 v0, s[36:39], s53 offen offset:2048 sc0 sc1 lds\n\t" \
  "v_mfma_f32_16x16x128_f8f6f4 a[232:235], v[184:191], v[208:215], a[232:235]\n\t" \
  "v_rcp_f32_e32 v16, v16\n\t" \
  "v_rcp_f32_e32 v17, v17\n\t" \
  "v_mfma_f32_16x16x128_f8f6f4 a[228:231], v[184:191], v[200:207], a[228:231]\n\t" \
  "buffer_load_dwordx4 v0, s[36:39], s53 offen offset:3072 sc0 sc1 lds\n\t" \
  "v_mfma_f32_16x16x128_f8f6f4 a[224:227], v[184:191], v[192:199], a[224:227]\n\t" \
  "v_rcp_f32_e32 v18, v18\n\t" \
  "v_rcp_f32_e32 v19, v19\n\t" \
  "s_waitcnt lgkmcnt(0)\n\t" \
  "v_mfma_f32_16x16x128_f8f6f4 a[144:147], v[160:167], v[224:231], a[144:147]\n\t" \
  "v_mul_f32_e32 v96, v96, v16\n\t" \
  "v_mul_f32_e32 v97, v97, v17\n\t" \
  "v_mul_f32_e32 v98, v98, v18\n\t" \
  "v_mul_f32_e32 v99, v99, v19\n\t" \
  "v_mfma_f32_16x16x128_f8f6f4 a[148:151], v[160:167], v[232:239], a[148:151]\n\t" \
  "ds_read_b128 v[192:195], v3 offset:16384\n\t" \
  "ds_read_b128 v[196:199], v3 offset:24576\n\t" \
  "v_mfma_f32_16x16x128_f8f6f4 a[152:155], v[160:167], v[240:247], a[152:155]\n\t" \
  "ds_read_b128 v[200:203], v3 offset:17408\n\t" \
  "ds_read_b128 v[204:207], v3 offset:25600\n\t" \
  "v_mfma_f32_16x16x128_f8f6f4 a[156:159], v[160:167], v[248:255], a[156:159]\n\t" \
  "ds_read_b128 v[208:211], v3 offset:18432\n\t" \
  "ds_read_b128 v[212:215], v3 offset:26624\n\t" \
  "v_mfma_f32_16x16x128_f8f6f4 a[188:191], v[168:175], v[248:255], a[188:191]\n\t" \
  "ds_read_b128 v[216:219], v3 offset:19456\n\t" \
  "ds_read_b128 v[220:223], v3 offset:27648\n\t" \
  "v_mfma_f32_16x16x128_f8f6f4 a[184:187], v[168:175], v[240:247], a[184:187]\n\t" \
  "s_xor_b32 s91, s68, 4194304\n\t" \
  "buffer_store_dwordx2 v[96:97], v5, s[40:43], s91 offen nt\n\t" \
  "v_mfma_f32_16x16x128_f8f6f4 a[180:183], v[168:175], v[232:239], a[180:183]\n\t" \
  "v_mfma_f32_16x16x128_f8f6f4 a[176:179], v[168:175], v[224:231], a[176:179]\n\t" \
  "s_add_u32 s53, s49, 49152\n\t" \
  "s_add_u32 m0, s48, 114688\n\t" \
  "v_mfma_f32_16x16x128_f8f6f4 a[208:211], v[176:183], v[224:231], a[208:211]\n\t" \
  "buffer_load_dwordx4 v0, s[36:39], s53 offen sc0 sc1 lds\n\t" \
  "v_mfma_f32_16x16x128_f8f6f4 a[212:215], v[176:183], v[232:239], a[212:215]\n\t" \
  "buffer_store_dwordx2 v[98:99], v5, s[40:43], s91 offen offset:128 nt\n\t" \
  "v_mfma_f32_16x16x128_f8f6f4 a[216:219], v[176:183], v[240:247], a[216:219]\n\t" \
  "buffer_load_dwordx4 v0, s[36:39], s53 offen offset:1024 sc0 sc1 lds\n\t" \
  "v_mfma_f32_16x16x128_f8f6f4 a[220:223], v[176:183], v[248:255], a[220:223]\n\t" \
  "v_mul_f32_e32 v100, s56, v100\n\t" \
  "v_mul_f32_e32 v101, s56, v101\n\t" \
  "v_mul_f32_e32 v102, s56, v102\n\t" \
  "v_mul_f32_e32 v103, s56, v103\n\t" \
  "v_mfma_f32_16x16x128_f8f6f4 a[252:255], v[184:191], v[248:255], a[252:255]\n\t" \
  "buffer_load_dwordx4 v0, s[36:39], s53 offen offset:2048 sc0 sc1 lds\n\t" \
  "v_mfma_f32_16x16x128_f8f6f4 a[248:251], v[184:191], v[240:247], a[248:251]\n\t" \
  "v_min_f32_e32 v20, s60, v100\n\t" \
  "v_min_f32_e32 v21, s60, v101\n\t" \
  "v_min_f32_e32 v22, s60, v102\n\t" \
  "v_min_f32_e32 v23, s60, v103\n\t" \
  "v_mfma_f32_16x16x128_f8f6f4 a[244:247], v[184:191], v[232:239], a[244:247]\n\t" \
  "buffer_load_dwordx4 v0, s[36:39], s53 offen offset:3072 sc0 sc1 lds\n\t" \
  "v_mfma_f32_16x16x128_f8f6f4 a[240:243], v[184:191], v[224:231], a[240:243]\n\t" \
  "s_add_u32 s49, s49, 65536\n\t" \
  "s_add_u32 s50, s50, 32768\n\t" \
  "s_add_u32 s51, s51, 32768\n\t" \
  "s_waitcnt lgkmcnt(0)\n\t" \
  "v_mfma_f32_16x16x128_f8f6f4 a[0:3], v[128:135], v[192:199], a[0:3]\n\t" \
  "s_waitcnt vmcnt(22)\n\t" \
  "s_barrier\n\t" \
  "v_mfma_f32_16x16x128_f8f6f4 a[4:7], v[128:135], v[200:207], a[4:7]\n\t" \
  "ds_read_b128 v[224:227], v3 offset:32768\n\t" \
  "ds_read_b128 v[228:231], v3 offset:40960\n\t" \
  "v_mfma_f32_16x16x128_f8f6f4 a[8:11], v[128:135], v[208:215], a[8:11]\n\t" \
  "ds_read_b128 v[232:235], v3 offset:33792\n\t" \
  "ds_read_b128 v[236:239], v3 offset:41984\n\t" \
  "v_mfma_f32_16x16x128_f8f6f4 a[12:15], v[128:135], v[216:223], a[12:15]\n\t" \
  "ds_read_b128 v[240:243], v3 offset:34816\n\t" \
  "ds_read_b128 v[244:247], v3 offset:43008\n\t" \
  "v_mfma_f32_16x16x128_f8f6f4 a[44:47], v[136:143], v[216:223], a[44:47]\n\t" \
  "ds_read_b128 v[248:251], v3 offset:35840\n\t" \
  "ds_read_b128 v[252:255], v3 offset:44032\n\t" \
  "v_mfma_f32_16x16x128_f8f6f4 a[40:43], v[136:143], v[208:215], a[40:43]\n\t" \
  "v_exp_f32_e32 v20, v20\n\t" \
  "v_exp_f32_e32 v21, v21\n\t" \
  "v_mfma_f32_16x16x128_f8f6f4 a[36:39], v[136:143], v[200:207], a[36:39]\n\t" \
  "v_exp_f32_e32 v22, v22\n\t" \
  "v_exp_f32_e32 v23, v23\n\t" \
  "v_mfma_f32_16x16x128_f8f6f4 a[32:35], v[136:143], v[192:199], a[32:35]\n\t" \
  "s_mov_b32 s53, s49\n\t" \
  "s_mov_b32 m0, s48\n\t" \
  "v_mfma_f32_16x16x128_f8f6f4 a[64:67], v[144:151], v[192:199], a[64:67]\n\t" \
  "buffer_load_dwordx4 v0, s[36:39], s53 offen sc0 sc1 lds\n\t" \
  "v_mfma_f32_16x16x128_f8f6f4 a[68:71], v[144:151], v[200:207], a[68:71]\n\t" \
  "v_fma_f32 v20, v20, s58, s58\n\t" \
  "v_fma_f32 v21, v21, s58, s58\n\t" \
  "v_fma_f32 v22, v22, s58, s58\n\t" \
  "v_fma_f32 v23, v23, s58, s58\n\t" \
  "v_mfma_f32_16x16x128_f8f6f4 a[72:75], v[144:151], v[208:215], a[72:75]\n\t" \
  "buffer_load_dwordx4 v0, s[36:39], s53 offen offset:1024 sc0 sc1 lds\n\t" \
  "v_mfma_f32_16x16x128_f8f6f4 a[76:79], v[144:151], v[216:223], a[76:79]\n\t" \
  "v_rcp_f32_e32 v20, v20\n\t" \
  "v_rcp_f32_e32 v21, v21\n\t" \
  "v_mfma_f32_16x16x128_f8f6f4 a[108:111], v[152:159], v[216:223], a[108:111]\n\t" \
  "buffer_load_dwordx4 v0, s[36:39], s53 offen offset:2048 sc0 sc1 lds\n\t" \
  "v_mfma_f32_16x16x128_f8f6f4 a[104:107], v[152:159], v[208:215], a[104:107]\n\t" \
  "v_rcp_f32_e32 v22, v22\n\t" \
  "v_rcp_f32_e32 v23, v23\n\t" \
  "v_mfma_f32_16x16x128_f8f6f4 a[100:103], v[152:159], v[200:207], a[100:103]\n\t" \
  "buffer_load_dwordx4 v0, s[36:39], s53 offen offset:3072 sc0 sc1 lds\n\t" \
  "v_mfma_f32_16x16x128_f8f6f4 a[96:99], v[152:159], v[192:199], a[96:99]\n\t" \
  "v_mul_f32_e32 v100, v100, v20\n\t" \
  "v_mul_f32_e32 v101, v101, v21\n\t" \
  "v_mul_f32_e32 v102, v102, v22\n\t" \
  "v_mul_f32_e32 v103, v103, v23\n\t" \
  "s_waitcnt lgkmcnt(0)\n\t" \
  "v_mfma_f32_16x16x128_f8f6f4 a[16:19], v[128:135], v[224:231], a[16:19]\n\t" \
  "s_xor_b32 s91, s68, 4259840\n\t" \
  "buffer_store_dwordx2 v[100:101], v5, s[40:43], s91 offen nt\n\t" \
  "v_mfma_f32_16x16x128_f8f6f4 a[20:23], v[128:135], v[232:239], a[20:23]\n\t" \
  "ds_read_b128 v[160:163], v1 offset:49152\n\t" \
  "ds_read_b128 v[164:167], v1 offset:57344\n\t" \
  "v_mfma_f32_16x16x128_f8f6f4 a[24:27], v[128:135], v[240:247], a[24:27]\n\t" \
  "ds_read_b128 v[168:171], v1 offset:50176\n\t" \
  "ds_read_b128 v[172:175], v1 offset:58368\n\t" \
  "v_mfma_f32_16x16x128_f8f6f4 a[28:31], v[128:135], v[248:255], a[28:31]\n\t" \
  "ds_read_b128 v[176:179], v1 offset:51200\n\t" \
  "ds_read_b128 v[180:183], v1 offset:59392\n\t" \
  "v_mfma_f32_16x16x128_f8f6f4 a[60:63], v[136:143], v[248:255], a[60:63]\n\t" \
  "ds_read_b128 v[184:187], v1 offset:52224\n\t" \
  "ds_read_b128 v[188:191], v1 offset:60416\n\t" \
  "v_mfma_f32_16x16x128_f8f6f4 a[56:59], v[136:143], v[240:247], a[56:59]\n\t" \
  "buffer_store_dwordx2 v[102:103], v5, s[40:43], s91 offen offset:128 nt\n\t" \
  "v_mfma_f32_16x16x128_f8f6f4 a[52:55], v[136:143], v[232:239], a[52:55]\n\t" \
  "v_mul_f32_e32 v104, s56, v104\n\t" \
  "v_mul_f32_e32 v105, s56, v105\n\t" \
  "v_mul_f32_e32 v106, s56, v106\n\t" \
  "v_mul_f32_e32 v107, s56, v107\n\t" \
  "v_mfma_f32_16x16x128_f8f6f4 a[48:51], v[136:143], v[224:231], a[48:51]\n\t" \
  "s_mov_b32 s53, s50\n\t" \
  "s_add_u32 m0, s48, 16384\n\t" \
  "v_mfma_f32_16x16x128_f8f6f4 a[80:83], v[144:151], v[224:231], a[80:83]\n\t" \
  "buffer_load_dwordx4 v0, s[36:39], s53 offen sc0 sc1 lds\n\t" \
  "v_mfma_f32_16x16x128_f8f6f4 a[84:87], v[144:151], v[232:239], a[84:87]\n\t" \
  "v_min_f32_e32 v24, s60, v104\n\t" \
  "v_min_f32_e32 v25, s60, v105\n\t" \
  "v_min_f32_e32 v26, s60, v106\n\t" \
  "v_min_f32_e32 v27, s60, v107\n\t" \
  "v_mfma_f32_16x16x128_f8f6f4 a[88:91], v[144:151], v[240:247], a[88:91]\n\t" \
  "buffer_load_dwordx4 v0, s[36:39], s53 offen offset:1024 sc0 sc1 lds\n\t" \
  "v_mfma_f32_16x16x128_f8f6f4 a[92:95], v[144:151], v[248:255], a[92:95]\n\t" \
  "v_exp_f32_e32 v24, v24\n\t" \
  "v_exp_f32_e32 v25, v25\n\t" \
  "v_mfma_f32_16x16x128_f8f6f4 a[124:127], v[152:159], v[248:255], a[124:127]\n\t" \
  "buffer_load_dwordx4 v0, s[36:39], s53 offen offset:2048 sc0 sc1 lds\n\t" \
  "v_mfma_f32_16x16x128_f8f6f4 a[120:123], v[152:159], v[240:247], a[120:123]\n\t" \
  "v_exp_f32_e32 v26, v26\n\t" \
  "v_exp_f32_e32 v27, v27\n\t" \
  "v_mfma_f32_16x16x128_f8f6f4 a[116:119], v[152:159], v[232:239], a[116:119]\n\t" \
  "buffer_load_dwordx4 v0, s[36:39], s53 offen offset:3072 sc0 sc1 lds\n\t" \
  "v_mfma_f32_16x16x128_f8f6f4 a[112:115], v[152:159], v[224:231], a[112:115]\n\t" \
  "v_fma_f32 v24, v24, s58, s58\n\t" \
  "v_fma_f32 v25, v25, s58, s58\n\t" \
  "v_fma_f32 v26, v26, s58, s58\n\t" \
  "v_fma_f32 v27, v27, s58, s58\n\t" \
  "s_waitcnt lgkmcnt(0)\n\t" \
  "v_mfma_f32_16x16x128_f8f6f4 a[144:147], v[160:167], v[224:231], a[144:147]\n\t" \
  "s_waitcnt vmcnt(20)\n\t" \
  "s_barrier\n\t" \
  "v_mfma_f32_16x16x128_f8f6f4 a[148:151], v[160:167], v[232:239], a[148:151]\n\t" \
  "ds_read_b128 v[128:131], v2 offset:0\n\t" \
  "ds_read_b128 v[132:135], v2 offset:8192\n\t" \
  "v_mfma_f32_16x16x128_f8f6f4 a[152:155], v[160:167], v[240:247], a[152:155]\n\t" \
  "ds_read_b128 v[136:139], v2 offset:1024\n\t" \
  "ds_read_b128 v[140:143], v2 offset:9216\n\t" \
  "v_mfma_f32_16x16x128_f8f6f4 a[156:159], v[160:167], v[248:255], a[156:159]\n\t" \
  "ds_read_b128 v[144:147], v2 offset:2048\n\t" \
  "ds_read_b128 v[148:151], v2 offset:10240\n\t" \
  "v_mfma_f32_16x16x128_f8f6f4 a[188:191], v[168:175], v[248:255], a[188:191]\n\t" \
  "ds_read_b128 v[152:155], v2 offset:3072\n\t" \
  "ds_read_b128 v[156:159], v2 offset:11264\n\t" \
  "v_mfma_f32_16x16x128_f8f6f4 a[184:187], v[168:175], v[240:247], a[184:187]\n\t" \
  "v_rcp_f32_e32 v24, v24\n\t" \
  "v_rcp_f32_e32 v25, v25\n\t" \
  "v_mfma_f32_16x16x128_f8f6f4 a[180:183], v[168:175], v[232:239], a[180:183]\n\t" \
  "v_rcp_f32_e32 v26, v26\n\t" \
  "v_rcp_f32_e32 v27, v27\n\t" \
  "v_mfma_f32_16x16x128_f8f6f4 a[176:179], v[168:175], v[224:231], a[176:179]\n\t" \
  "s_mov_b32 s53, s51\n\t" \
  "s_add_u32 m0, s48, 32768\n\t" \
  "v_mfma_f32_16x16x128_f8f6f4 a[208:211], v[176:183], v[224:231], a[208:211]\n\t" \
  "buffer_load_dwordx4 v0, s[36:39], s53 offen sc0 sc1 lds\n\t" \
  "v_mfma_f32_16x16x128_f8f6f4 a[212:215], v[176:183], v[232:239], a[212:215]\n\t" \
  "v_mul_f32_e32 v104, v104, v24\n\t" \
  "v_mul_f32_e32 v105, v105, v25\n\t" \
  "v_mul_f32_e32 v106, v106, v26\n\t" \
  "v_mul_f32_e32 v107, v107, v27\n\t" \
  "v_mfma_f32_16x16x128_f8f6f4 a[216:219], v[176:183], v[240:247], a[216:219]\n\t" \
  "buffer_load_dwordx4 v0, s[36:39], s53 offen offset:1024 sc0 sc1 lds\n\t" \
  "v_mfma_f32_16x16x128_f8f6f4 a[220:223], v[176:183], v[248:255], a[220:223]\n\t" \
  "s_xor_b32 s91, s68, 4325376\n\t" \
  "buffer_store_dwordx2 v[104:105], v5, s[40:43], s91 offen nt\n\t" \
  "v_mfma_f32_16x16x128_f8f6f4 a[252:255], v[184:191], v[248:255], a[252:255]\n\t" \
  "buffer_load_dwordx4 v0, s[36:39], s53 offen offset:2048 sc0 sc1 lds\n\t" \
  "v_mfma_f32_16x16x128_f8f6f4 a[248:251], v[184:191], v[240:247], a[248:251]\n\t" \
  "buffer_store_dwordx2 v[106:107], v5, s[40:43], s91 offen offset:128 nt\n\t" \
  "v_mfma_f32_16x16x128_f8f6f4 a[244:247], v[184:191], v[232:239], a[244:247]\n\t" \
  "buffer_load_dwordx4 v0, s[36:39], s53 offen offset:3072 sc0 sc1 lds\n\t" \
  "v_mfma_f32_16x16x128_f8f6f4 a[240:243], v[184:191], v[224:231], a[240:243]\n\t" \
  "v_mul_f32_e32 v108, s56, v108\n\t" \
  "v_mul_f32_e32 v109, s56, v109\n\t" \
  "v_mul_f32_e32 v110, s56, v110\n\t" \
  "v_mul_f32_e32 v111, s56, v111\n\t" \
  "s_waitcnt lgkmcnt(0)\n\t" \
  "v_mfma_f32_16x16x128_f8f6f4 a[128:131], v[160:167], v[192:199], a[128:131]\n\t" \
  "v_min_f32_e32 v28, s60, v108\n\t" \
  "v_min_f32_e32 v29, s60, v109\n\t" \
  "v_min_f32_e32 v30, s60, v110\n\t" \
  "v_min_f32_e32 v31, s60, v111\n\t" \
  "v_mfma_f32_16x16x128_f8f6f4 a[132:135], v[160:167], v[200:207], a[132:135]\n\t" \
  "ds_read_b128 v[224:227], v4 offset:16384\n\t" \
  "ds_read_b128 v[228:231], v4 offset:24576\n\t" \
  "v_mfma_f32_16x16x128_f8f6f4 a[136:139], v[160:167], v[208:215], a[136:139]\n\t" \
  "ds_read_b128 v[232:235], v4 offset:17408\n\t" \
  "ds_read_b128 v[236:239], v4 offset:25600\n\t" \
  "v_mfma_f32_16x16x128_f8f6f4 a[140:143], v[160:167], v[216:223], a[140:143]\n\t" \
  "ds_read_b128 v[240:243], v4 offset:18432\n\t" \
  "ds_read_b128 v[244:247], v4 offset:26624\n\t" \
  "v_mfma_f32_16x16x128_f8f6f4 a[172:175], v[168:175], v[216:223], a[172:175]\n\t" \
  "ds_read_b128 v[248:251], v4 offset:19456\n\t" \
  "ds_read_b128 v[252:255], v4 offset:27648\n\t" \
  "v_mfma_f32_16x16x128_f8f6f4 a[168:171], v[168:175], v[208:215], a[168:171]\n\t" \
  "v_exp_f32_e32 v28, v28\n\t" \
  "v_exp_f32_e32 v29, v29\n\t" \
  "v_mfma_f32_16x16x128_f8f6f4 a[164:167], v[168:175], v[200:207], a[164:167]\n\t" \
  "v_mfma_f32_16x16x128_f8f6f4 a[160:163], v[168:175], v[192:199], a[160:163]\n\t" \
  "s_add_u32 s53, s49, 16384\n\t" \
  "s_add_u32 m0, s48, 49152\n\t" \
  "v_mfma_f32_16x16x128_f8f6f4 a[192:195], v[176:183], v[192:199], a[192:195]\n\t" \
  "buffer_load_dwordx4 v0, s[36:39], s53 offen sc0 sc1 lds\n\t" \
  "v_mfma_f32_16x16x128_f8f6f4 a[196:199], v[176:183], v[200:207], a[196:199]\n\t" \
  "v_exp_f32_e32 v30, v30\n\t" \
  "v_exp_f32_e32 v31, v31\n\t" \
  "v_mfma_f32_16x16x128_f8f6f4 a[200:203], v[176:183], v[208:215], a[200:203]\n\t" \
  "buffer_load_dwordx4 v0, s[36:39], s53 offen offset:1024 sc0 sc1 lds\n\t" \
  "v_mfma_f32_16x16x128_f8f6f4 a[204:207], v[176:183], v[216:223], a[204:207]\n\t" \
  "v_fma_f32 v28, v28, s58, s58\n\t" \
  "v_fma_f32 v29, v29, s58, s58\n\t" \
  "v_fma_f32 v30, v30, s58, s58\n\t" \
  "v_fma_f32 v31, v31, s58, s58\n\t" \
  "v_mfma_f32_16x16x128_f8f6f4 a[236:239], v[184:191], v[216:223], a[236:239]\n\t" \
  "buffer_load_dwordx4 v0, s[36:39], s53 offen offset:2048 sc0 sc1 lds\n\t" \
  "v_mfma_f32_16x16x128_f8f6f4 a[232:235], v[184:191], v[208:215], a[232:235]\n\t" \
  "v_rcp_f32_e32 v28, v28\n\t" \
  "v_rcp_f32_e32 v29, v29\n\t" \
  "v_mfma_f32_16x16x128_f8f6f4 a[228:231], v[184:191], v[200:207], a[228:231]\n\t" \
  "buffer_load_dwordx4 v0, s[36:39], s53 offen offset:3072 sc0 sc1 lds\n\t" \
  "v_mfma_f32_16x16x128_f8f6f4 a[224:227], v[184:191], v[192:199], a[224:227]\n\t" \
  "v_rcp_f32_e32 v30, v30\n\t" \
  "v_rcp_f32_e32 v31, v31\n\t" \
  "s_waitcnt lgkmcnt(0)\n\t" \
  "v_mfma_f32_16x16x128_f8f6f4 a[16:19], v[128:135], v[224:231], a[16:19]\n\t" \
  "s_waitcnt vmcnt(20)\n\t" \
  "s_barrier\n\t" \
  "v_mfma_f32_16x16x128_f8f6f4 a[20:23], v[128:135], v[232:239], a[20:23]\n\t" \
  "ds_read_b128 v[192:195], v4 offset:32768\n\t" \
  "ds_read_b128 v[196:199], v4 offset:40960\n\t" \
  "v_mfma_f32_16x16x128_f8f6f4 a[24:27], v[128:135], v[240:247], a[24:27]\n\t" \
  "ds_read_b128 v[200:203], v4 offset:33792\n\t" \
  "ds_read_b128 v[204:207], v4 offset:41984\n\t" \
  "v_mfma_f32_16x16x128_f8f6f4 a[28:31], v[128:135], v[248:255], a[28:31]\n\t" \
  "ds_read_b128 v[208:211], v4 offset:34816\n\t" \
  "ds_read_b128 v[212:215], v4 offset:43008\n\t" \
  "v_mfma_f32_16x16x128_f8f6f4 a[60:63], v[136:143], v[248:255], a[60:63]\n\t" \
  "ds_read_b128 v[216:219], v4 offset:35840\n\t" \
  "ds_read_b128 v[220:223], v4 offset:44032\n\t" \
  "v_mfma_f32_16x16x128_f8f6f4 a[56:59], v[136:143], v[240:247], a[56:59]\n\t" \
  "v_mul_f32_e32 v108, v108, v28\n\t" \
  "v_mul_f32_e32 v109, v109, v29\n\t" \
  "v_mul_f32_e32 v110, v110, v30\n\t" \
  "v_mul_f32_e32 v111, v111, v31\n\t" \
  "v_mfma_f32_16x16x128_f8f6f4 a[52:55], v[136:143], v[232:239], a[52:55]\n\t" \
  "s_xor_b32 s91, s68, 4390912\n\t" \
  "buffer_store_dwordx2 v[108:109], v5, s[40:43], s91 offen nt\n\t" \
  "v_mfma_f32_16x16x128_f8f6f4 a[48:51], v[136:143], v[224:231], a[48:51]\n\t" \
  "s_add_u32 s53, s49, 32768\n\t" \
  "s_add_u32 m0, s48, 65536\n\t" \
  "v_mfma_f32_16x16x128_f8f6f4 a[80:83], v[144:151], v[224:231], a[80:83]\n\t" \
  "buffer_load_dwordx4 v0, s[36:39], s53 offen sc0 sc1 lds\n\t" \
  "v_mfma_f32_16x16x128_f8f6f4 a[84:87], v[144:151], v[232:239], a[84:87]\n\t" \
  "buffer_load_dword v7, v8, s[44:47], s76 offen sc0 sc1\n\t" \
  "v_mfma_f32_16x16x128_f8f6f4 a[88:91], v[144:151], v[240:247], a[88:91]\n\t" \
  "buffer_load_dwordx4 v0, s[36:39], s53 offen offset:1024 sc0 sc1 lds\n\t" \
  "v_mfma_f32_16x16x128_f8f6f4 a[92:95], v[144:151], v[248:255], a[92:95]\n\t" \
  "buffer_store_dwordx2 v[110:111], v5, s[40:43], s91 offen offset:128 nt\n\t" \
  "v_mfma_f32_16x16x128_f8f6f4 a[124:127], v[152:159], v[248:255], a[124:127]\n\t" \
  "buffer_load_dwordx4 v0, s[36:39], s53 offen offset:2048 sc0 sc1 lds\n\t" \
  "v_mfma_f32_16x16x128_f8f6f4 a[120:123], v[152:159], v[240:247], a[120:123]\n\t" \
  "v_mul_f32_e32 v112, s56, v112\n\t" \
  "v_mul_f32_e32 v113, s56, v113\n\t" \
  "v_mul_f32_e32 v114, s56, v114\n\t" \
  "v_mul_f32_e32 v115, s56, v115\n\t" \
  "v_mfma_f32_16x16x128_f8f6f4 a[116:119], v[152:159], v[232:239], a[116:119]\n\t" \
  "buffer_load_dwordx4 v0, s[36:39], s53 offen offset:3072 sc0 sc1 lds\n\t" \
  "v_mfma_f32_16x16x128_f8f6f4 a[112:115], v[152:159], v[224:231], a[112:115]\n\t" \
  "v_min_f32_e32 v12, s60, v112\n\t" \
  "v_min_f32_e32 v13, s60, v113\n\t" \
  "v_min_f32_e32 v14, s60, v114\n\t" \
  "v_min_f32_e32 v15, s60, v115\n\t" \
  "s_waitcnt lgkmcnt(0)\n\t" \
  "v_mfma_f32_16x16x128_f8f6f4 a[0:3], v[128:135], v[192:199], a[0:3]\n\t" \
  "v_exp_f32_e32 v12, v12\n\t" \
  "v_exp_f32_e32 v13, v13\n\t" \
  "v_mfma_f32_16x16x128_f8f6f4 a[4:7], v[128:135], v[200:207], a[4:7]\n\t" \
  "ds_read_b128 v[160:163], v2 offset:49152\n\t" \
  "ds_read_b128 v[164:167], v2 offset:57344\n\t" \
  "v_mfma_f32_16x16x128_f8f6f4 a[8:11], v[128:135], v[208:215], a[8:11]\n\t" \
  "ds_read_b128 v[168:171], v2 offset:50176\n\t" \
  "ds_read_b128 v[172:175], v2 offset:58368\n\t" \
  "v_mfma_f32_16x16x128_f8f6f4 a[12:15], v[128:135], v[216:223], a[12:15]\n\t" \
  "ds_read_b128 v[176:179], v2 offset:51200\n\t" \
  "ds_read_b128 v[180:183], v2 offset:59392\n\t" \
  "v_mfma_f32_16x16x128_f8f6f4 a[44:47], v[136:143], v[216:223], a[44:47]\n\t" \
  "ds_read_b128 v[184:187], v2 offset:52224\n\t" \
  "ds_read_b128 v[188:191], v2 offset:60416\n\t" \
  "v_mfma_f32_16x16x128_f8f6f4 a[40:43], v[136:143], v[208:215], a[40:43]\n\t" \
  "v_exp_f32_e32 v14, v14\n\t" \
  "v_exp_f32_e32 v15, v15\n\t" \
  "v_mfma_f32_16x16x128_f8f6f4 a[36:39], v[136:143], v[200:207], a[36:39]\n\t" \
  "v_fma_f32 v12, v12, s58, s58\n\t" \
  "v_fma_f32 v13, v13, s58, s58\n\t" \
  "v_fma_f32 v14, v14, s58, s58\n\t" \
  "v_fma_f32 v15, v15, s58, s58\n\t" \
  "v_mfma_f32_16x16x128_f8f6f4 a[32:35], v[136:143], v[192:199], a[32:35]\n\t" \
  "s_add_u32 s53, s51, 16384\n\t" \
  "s_add_u32 m0, s48, 81920\n\t" \
  "v_mfma_f32_16x16x128_f8f6f4 a[64:67], v[144:151], v[192:199], a[64:67]\n\t" \
  "buffer_load_dwordx4 v0, s[36:39], s53 offen sc0 sc1 lds\n\t" \
  "v_mfma_f32_16x16x128_f8f6f4 a[68:71], v[144:151], v[200:207], a[68:71]\n\t" \
  "v_rcp_f32_e32 v12, v12\n\t" \
  "v_rcp_f32_e32 v13, v13\n\t" \
  "v_mfma_f32_16x16x128_f8f6f4 a[72:75], v[144:151], v[208:215], a[72:75]\n\t" \
  "buffer_load_dwordx4 v0, s[36:39], s53 offen offset:1024 sc0 sc1 lds\n\t" \
  "v_mfma_f32_16x16x128_f8f6f4 a[76:79], v[144:151], v[216:223], a[76:79]\n\t" \
  "v_rcp_f32_e32 v14, v14\n\t" \
  "v_rcp_f32_e32 v15, v15\n\t" \
  "v_mfma_f32_16x16x128_f8f6f4 a[108:111], v[152:159], v[216:223], a[108:111]\n\t" \
  "buffer_load_dwordx4 v0, s[36:39], s53 offen offset:2048 sc0 sc1 lds\n\t" \
  "v_mfma_f32_16x16x128_f8f6f4 a[104:107], v[152:159], v[208:215], a[104:107]\n\t" \
  "v_mul_f32_e32 v112, v112, v12\n\t" \
  "v_mul_f32_e32 v113, v113, v13\n\t" \
  "v_mul_f32_e32 v114, v114, v14\n\t" \
  "v_mul_f32_e32 v115, v115, v15\n\t" \
  "v_mfma_f32_16x16x128_f8f6f4 a[100:103], v[152:159], v[200:207], a[100:103]\n\t" \
  "buffer_load_dwordx4 v0, s[36:39], s53 offen offset:3072 sc0 sc1 lds\n\t" \
  "v_mfma_f32_16x16x128_f8f6f4 a[96:99], v[152:159], v[192:199], a[96:99]\n\t" \
  "s_xor_b32 s91, s68, 5242880\n\t" \
  "buffer_store_dwordx2 v[112:113], v5, s[40:43], s91 offen nt\n\t" \
  "s_waitcnt lgkmcnt(0)\n\t" \
  "v_mfma_f32_16x16x128_f8f6f4 a[128:131], v[160:167], v[192:199], a[128:131]\n\t" \
  "s_waitcnt vmcnt(22)\n\t" \
  "s_barrier\n\t" \
  "v_mfma_f32_16x16x128_f8f6f4 a[132:135], v[160:167], v[200:207], a[132:135]\n\t" \
  "ds_read_b128 v[128:131], v1 offset:0\n\t" \
  "ds_read_b128 v[132:135], v1 offset:8192\n\t" \
  "v_mfma_f32_16x16x128_f8f6f4 a[136:139], v[160:167], v[208:215], a[136:139]\n\t" \
  "ds_read_b128 v[136:139], v1 offset:1024\n\t" \
  "ds_read_b128 v[140:143], v1 offset:9216\n\t" \
  "v_mfma_f32_16x16x128_f8f6f4 a[140:143], v[160:167], v[216:223], a[140:143]\n\t" \
  "ds_read_b128 v[144:147], v1 offset:2048\n\t" \
  "ds_read_b128 v[148:151], v1 offset:10240\n\t" \
  "v_mfma_f32_16x16x128_f8f6f4 a[172:175], v[168:175], v[216:223], a[172:175]\n\t" \
  "ds_read_b128 v[152:155], v1 offset:3072\n\t" \
  "ds_read_b128 v[156:159], v1 offset:11264\n\t" \
  "v_mfma_f32_16x16x128_f8f6f4 a[168:171], v[168:175], v[208:215], a[168:171]\n\t" \
  "buffer_store_dwordx2 v[114:115], v5, s[40:43], s91 offen offset:128 nt\n\t" \
  "v_mfma_f32_16x16x128_f8f6f4 a[164:167], v[168:175], v[200:207], a[164:167]\n\t" \
  "v_mul_f32_e32 v116, s56, v116\n\t" \
  "v_mul_f32_e32 v117, s56, v117\n\t" \
  "v_mul_f32_e32 v118, s56, v118\n\t" \
  "v_mul_f32_e32 v119, s56, v119\n\t" \
  "v_mfma_f32_16x16x128_f8f6f4 a[160:163], v[168:175], v[192:199], a[160:163]\n\t" \
  "s_add_u32 s53, s50, 16384\n\t" \
  "s_add_u32 m0, s48, 98304\n\t" \
  "v_mfma_f32_16x16x128_f8f6f4 a[192:195], v[176:183], v[192:199], a[192:195]\n\t" \
  "buffer_load_dwordx4 v0, s[36:39], s53 offen sc0 sc1 lds\n\t" \
  "v_mfma_f32_16x16x128_f8f6f4 a[196:199], v[176:183], v[200:207], a[196:199]\n\t" \
  "v_min_f32_e32 v16, s60, v116\n\t" \
  "v_min_f32_e32 v17, s60, v117\n\t" \
  "v_min_f32_e32 v18, s60, v118\n\t" \
  "v_min_f32_e32 v19, s60, v119\n\t" \
  "v_mfma_f32_16x16x128_f8f6f4 a[200:203], v[176:183], v[208:215], a[200:203]\n\t" \
  "buffer_load_dwordx4 v0, s[36:39], s53 offen offset:1024 sc0 sc1 lds\n\t" \
  "v_mfma_f32_16x16x128_f8f6f4 a[204:207], v[176:183], v[216:223], a[204:207]\n\t" \
  "v_exp_f32_e32 v16, v16\n\t" \
  "v_exp_f32_e32 v17, v17\n\t" \
  "v_mfma_f32_16x16x128_f8f6f4 a[236:239], v[184:191], v[216:223], a[236:239]\n\t" \
  "buffer_load_dwordx4 v0, s[36:39], s53 offen offset:2048 sc0 sc1 lds\n\t" \
  "v_mfma_f32_16x16x128_f8f6f4 a[232:235], v[184:191], v[208:215], a[232:235]\n\t" \
  "v_exp_f32_e32 v18, v18\n\t" \
  "v_exp_f32_e32 v19, v19\n\t" \
  "v_mfma_f32_16x16x128_f8f6f4 a[228:231], v[184:191], v[200:207], a[228:231]\n\t" \
  "buffer_load_dwordx4 v0, s[36:39], s53 offen offset:3072 sc0 sc1 lds\n\t" \
  "v_mfma_f32_16x16x128_f8f6f4 a[224:227], v[184:191], v[192:199], a[224:227]\n\t" \
  "v_fma_f32 v16, v16, s58, s58\n\t" \
  "v_fma_f32 v17, v17, s58, s58\n\t" \
  "v_fma_f32 v18, v18, s58, s58\n\t" \
  "v_fma_f32 v19, v19, s58, s58\n\t" \
  "s_waitcnt lgkmcnt(0)\n\t" \
  "v_mfma_f32_16x16x128_f8f6f4 a[144:147], v[160:167], v[224:231], a[144:147]\n\t" \
  "v_rcp_f32_e32 v16, v16\n\t" \
  "v_rcp_f32_e32 v17, v17\n\t" \
  "v_mfma_f32_16x16x128_f8f6f4 a[148:151], v[160:167], v[232:239], a[148:151]\n\t" \
  "ds_read_b128 v[192:195], v3 offset:16384\n\t" \
  "ds_read_b128 v[196:199], v3 offset:24576\n\t" \
  "v_mfma_f32_16x16x128_f8f6f4 a[152:155], v[160:167], v[240:247], a[152:155]\n\t" \
  "ds_read_b128 v[200:203], v3 offset:17408\n\t" \
  "ds_read_b128 v[204:207], v3 offset:25600\n\t" \
  "v_mfma_f32_16x16x128_f8f6f4 a[156:159], v[160:167], v[248:255], a[156:159]\n\t" \
  "ds_read_b128 v[208:211], v3 offset:18432\n\t" \
  "ds_read_b128 v[212:215], v3 offset:26624\n\t" \
  "v_mfma_f32_16x16x128_f8f6f4 a[188:191], v[168:175], v[248:255], a[188:191]\n\t" \
  "ds_read_b128 v[216:219], v3 offset:19456\n\t" \
  "ds_read_b128 v[220:223], v3 offset:27648\n\t" \
  "v_mfma_f32_16x16x128_f8f6f4 a[184:187], v[168:175], v[240:247], a[184:187]\n\t" \
  "v_rcp_f32_e32 v18, v18\n\t" \
  "v_rcp_f32_e32 v19, v19\n\t" \
  "v_mfma_f32_16x16x128_f8f6f4 a[180:183], v[168:175], v[232:239], a[180:183]\n\t" \
  "v_mul_f32_e32 v116, v116, v16\n\t" \
  "v_mul_f32_e32 v117, v117, v17\n\t" \
  "v_mul_f32_e32 v118, v118, v18\n\t" \
  "v_mul_f32_e32 v119, v119, v19\n\t" \
  "v_mfma_f32_16x16x128_f8f6f4 a[176:179], v[168:175], v[224:231], a[176:179]\n\t" \
  "s_add_u32 s53, s49, 49152\n\t" \
  "s_add_u32 m0, s48, 114688\n\t" \
  "v_mfma_f32_16x16x128_f8f6f4 a[208:211], v[176:183], v[224:231], a[208:211]\n\t" \
  "buffer_load_dwordx4 v0, s[36:39], s53 offen sc0 sc1 lds\n\t" \
  "v_mfma_f32_16x16x128_f8f6f4 a[212:215], v[176:183], v[232:239], a[212:215]\n\t" \
  "v_mfma_f32_16x16x128_f8f6f4 a[216:219], v[176:183], v[240:247], a[216:219]\n\t" \
  "buffer_load_dwordx4 v0, s[36:39], s53 offen offset:1024 sc0 sc1 lds\n\t" \
  "v_mfma_f32_16x16x128_f8f6f4 a[220:223], v[176:183], v[248:255], a[220:223]\n\t" \
  "s_xor_b32 s91, s68, 5308416\n\t" \
  "buffer_store_dwordx2 v[116:117], v5, s[40:43], s91 offen nt\n\t" \
  "v_mfma_f32_16x16x128_f8f6f4 a[252:255], v[184:191], v[248:255], a[252:255]\n\t" \
  "buffer_load_dwordx4 v0, s[36:39], s53 offen offset:2048 sc0 sc1 lds\n\t" \
  "v_mfma_f32_16x16x128_f8f6f4 a[248:251], v[184:191], v[240:247], a[248:251]\n\t" \
  "buffer_store_dwordx2 v[118:119], v5, s[40:43], s91 offen offset:128 nt\n\t" \
  "v_mfma_f32_16x16x128_f8f6f4 a[244:247], v[184:191], v[232:239], a[244:247]\n\t" \
  "buffer_load_dwordx4 v0, s[36:39], s53 offen offset:3072 sc0 sc1 lds\n\t" \
  "v_mfma_f32_16x16x128_f8f6f4 a[240:243], v[184:191], v[224:231], a[240:243]\n\t" \
  "s_add_u32 s49, s49, 65536\n\t" \
  "s_add_u32 s50, s50, 32768\n\t" \
  "s_add_u32 s51, s51, 32768\n\t" \
  "s_waitcnt lgkmcnt(0)\n\t" \
  "v_mfma_f32_16x16x128_f8f6f4 a[0:3], v[128:135], v[192:199], a[0:3]\n\t" \
  "s_waitcnt vmcnt(23)\n\t" \
  "s_barrier\n\t" \
  "v_mfma_f32_16x16x128_f8f6f4 a[4:7], v[128:135], v[200:207], a[4:7]\n\t" \
  "ds_read_b128 v[224:227], v3 offset:32768\n\t" \
  "ds_read_b128 v[228:231], v3 offset:40960\n\t" \
  "v_mfma_f32_16x16x128_f8f6f4 a[8:11], v[128:135], v[208:215], a[8:11]\n\t" \
  "ds_read_b128 v[232:235], v3 offset:33792\n\t" \
  "ds_read_b128 v[236:239], v3 offset:41984\n\t" \
  "v_mfma_f32_16x16x128_f8f6f4 a[12:15], v[128:135], v[216:223], a[12:15]\n\t" \
  "ds_read_b128 v[240:243], v3 offset:34816\n\t" \
  "ds_read_b128 v[244:247], v3 offset:43008\n\t" \
  "v_mfma_f32_16x16x128_f8f6f4 a[44:47], v[136:143], v[216:223], a[44:47]\n\t" \
  "ds_read_b128 v[248:251], v3 offset:35840\n\t" \
  "ds_read_b128 v[252:255], v3 offset:44032\n\t" \
  "v_mfma_f32_16x16x128_f8f6f4 a[40:43], v[136:143], v[208:215], a[40:43]\n\t" \
  "s_waitcnt vmcnt(20)\n\t" \
  "v_cmp_le_u32_e32 vcc, s75, v7\n\t" \
  "v_mfma_f32_16x16x128_f8f6f4 a[36:39], v[136:143], v[200:207], a[36:39]\n\t" \
  "s_cmp_eq_u64 vcc, -1\n\t" \
  "s_cbranch_scc0 L_SLOW15_%=\n\t" \
  "L_OK15_%=:\n\t" \
  "s_add_u32 s76, s76, 1024\n\t" \
  "v_mfma_f32_16x16x128_f8f6f4 a[32:35], v[136:143], v[192:199], a[32:35]\n\t" \
  "s_mov_b32 s53, s49\n\t" \
  "s_mov_b32 m0, s48\n\t" \
  "v_mfma_f32_16x16x128_f8f6f4 a[64:67], v[144:151], v[192:199], a[64:67]\n\t" \
  "buffer_load_dwordx4 v0, s[36:39], s53 offen sc0 sc1 lds\n\t" \
  "v_mfma_f32_16x16x128_f8f6f4 a[68:71], v[144:151], v[200:207], a[68:71]\n\t" \
  "v_mul_f32_e32 v120, s56, v120\n\t" \
  "v_mul_f32_e32 v121, s56, v121\n\t" \
  "v_mul_f32_e32 v122, s56, v122\n\t" \
  "v_mul_f32_e32 v123, s56, v123\n\t" \
  "v_mfma_f32_16x16x128_f8f6f4 a[72:75], v[144:151], v[208:215], a[72:75]\n\t" \
  "buffer_load_dwordx4 v0, s[36:39], s53 offen offset:1024 sc0 sc1 lds\n\t" \
  "v_mfma_f32_16x16x128_f8f6f4 a[76:79], v[144:151], v[216:223], a[76:79]\n\t" \
  "v_min_f32_e32 v20, s60, v120\n\t" \
  "v_min_f32_e32 v21, s60, v121\n\t" \
  "v_min_f32_e32 v22, s60, v122\n\t" \
  "v_min_f32_e32 v23, s60, v123\n\t" \
  "v_mfma_f32_16x16x128_f8f6f4 a[108:111], v[152:159], v[216:223], a[108:111]\n\t" \
  "buffer_load_dwordx4 v0, s[36:39], s53 offen offset:2048 sc0 sc1 lds\n\t" \
  "v_mfma_f32_16x16x128_f8f6f4 a[104:107], v[152:159], v[208:215], a[104:107]\n\t" \
  "v_exp_f32_e32 v20, v20\n\t" \
  "v_exp_f32_e32 v21, v21\n\t" \
  "v_mfma_f32_16x16x128_f8f6f4 a[100:103], v[152:159], v[200:207], a[100:103]\n\t" \
  "buffer_load_dwordx4 v0, s[36:39], s53 offen offset:3072 sc0 sc1 lds\n\t" \
  "v_mfma_f32_16x16x128_f8f6f4 a[96:99], v[152:159], v[192:199], a[96:99]\n\t" \
  "v_exp_f32_e32 v22, v22\n\t" \
  "v_exp_f32_e32 v23, v23\n\t" \
  "s_waitcnt lgkmcnt(0)\n\t" \
  "v_mfma_f32_16x16x128_f8f6f4 a[16:19], v[128:135], v[224:231], a[16:19]\n\t" \
  "v_fma_f32 v20, v20, s58, s58\n\t" \
  "v_fma_f32 v21, v21, s58, s58\n\t" \
  "v_fma_f32 v22, v22, s58, s58\n\t" \
  "v_fma_f32 v23, v23, s58, s58\n\t" \
  "v_mfma_f32_16x16x128_f8f6f4 a[20:23], v[128:135], v[232:239], a[20:23]\n\t" \
  "ds_read_b128 v[160:163], v1 offset:49152\n\t" \
  "ds_read_b128 v[164:167], v1 offset:57344\n\t" \
  "v_mfma_f32_16x16x128_f8f6f4 a[24:27], v[128:135], v[240:247], a[24:27]\n\t" \
  "ds_read_b128 v[168:171], v1 offset:50176\n\t" \
  "ds_read_b128 v[172:175], v1 offset:58368\n\t" \
  "v_mfma_f32_16x16x128_f8f6f4 a[28:31], v[128:135], v[248:255], a[28:31]\n\t" \
  "ds_read_b128 v[176:179], v1 offset:51200\n\t" \
  "ds_read_b128 v[180:183], v1 offset:59392\n\t" \
  "v_mfma_f32_16x16x128_f8f6f4 a[60:63], v[136:143], v[248:255], a[60:63]\n\t" \
  "ds_read_b128 v[184:187], v1 offset:52224\n\t" \
  "ds_read_b128 v[188:191], v1 offset:60416\n\t" \
  "v_mfma_f32_16x16x128_f8f6f4 a[56:59], v[136:143], v[240:247], a[56:59]\n\t" \
  "v_rcp_f32_e32 v20, v20\n\t" \
  "v_rcp_f32_e32 v21, v21\n\t" \
  "v_mfma_f32_16x16x128_f8f6f4 a[52:55], v[136:143], v[232:239], a[52:55]\n\t" \
  "v_rcp_f32_e32 v22, v22\n\t" \
  "v_rcp_f32_e32 v23, v23\n\t" \
  "v_mfma_f32_16x16x128_f8f6f4 a[48:51], v[136:143], v[224:231], a[48:51]\n\t" \
  "s_mov_b32 s53, s50\n\t" \
  "s_add_u32 m0, s48, 16384\n\t" \
  "v_mfma_f32_16x16x128_f8f6f4 a[80:83], v[144:151], v[224:231], a[80:83]\n\t" \
  "buffer_load_dwordx4 v0, s[36:39], s53 offen sc0 sc1 lds\n\t" \
  "v_mfma_f32_16x16x128_f8f6f4 a[84:87], v[144:151], v[232:239], a[84:87]\n\t" \
  "v_mul_f32_e32 v120, v120, v20\n\t" \
  "v_mul_f32_e32 v121, v121, v21\n\t" \
  "v_mul_f32_e32 v122, v122, v22\n\t" \
  "v_mul_f32_e32 v123, v123, v23\n\t" \
  "v_mfma_f32_16x16x128_f8f6f4 a[88:91], v[144:151], v[240:247], a[88:91]\n\t" \
  "buffer_load_dwordx4 v0, s[36:39], s53 offen offset:1024 sc0 sc1 lds\n\t" \
  "v_mfma_f32_16x16x128_f8f6f4 a[92:95], v[144:151], v[248:255], a[92:95]\n\t" \
  "s_xor_b32 s91, s68, 5373952\n\t" \
  "buffer_store_dwordx2 v[120:121], v5, s[40:43], s91 offen nt\n\t" \
  "v_mfma_f32_16x16x128_f8f6f4 a[124:127], v[152:159], v[248:255], a[124:127]\n\t" \
  "buffer_load_dwordx4 v0, s[36:39], s53 offen offset:2048 sc0 sc1 lds\n\t" \
  "v_mfma_f32_16x16x128_f8f6f4 a[120:123], v[152:159], v[240:247], a[120:123]\n\t" \
  "buffer_store_dwordx2 v[122:123], v5, s[40:43], s91 offen offset:128 nt\n\t" \
  "v_mfma_f32_16x16x128_f8f6f4 a[116:119], v[152:159], v[232:239], a[116:119]\n\t" \
  "buffer_load_dwordx4 v0, s[36:39], s53 offen offset:3072 sc0 sc1 lds\n\t" \
  "v_mfma_f32_16x16x128_f8f6f4 a[112:115], v[152:159], v[224:231], a[112:115]\n\t" \
  "v_mul_f32_e32 v124, s56, v124\n\t" \
  "v_mul_f32_e32 v125, s56, v125\n\t" \
  "v_mul_f32_e32 v126, s56, v126\n\t" \
  "v_mul_f32_e32 v127, s56, v127\n\t" \
  "s_waitcnt lgkmcnt(0)\n\t" \
  "v_mfma_f32_16x16x128_f8f6f4 a[144:147], v[160:167], v[224:231], a[144:147]\n\t" \
  "s_waitcnt vmcnt(22)\n\t" \
  "s_barrier\n\t" \
  "v_mfma_f32_16x16x128_f8f6f4 a[148:151], v[160:167], v[232:239], a[148:151]\n\t" \
  "ds_read_b128 v[128:131], v2 offset:0\n\t" \
  "ds_read_b128 v[132:135], v2 offset:8192\n\t" \
  "v_mfma_f32_16x16x128_f8f6f4 a[152:155], v[160:167], v[240:247], a[152:155]\n\t" \
  "ds_read_b128 v[136:139], v2 offset:1024\n\t" \
  "ds_read_b128 v[140:143], v2 offset:9216\n\t" \
  "v_mfma_f32_16x16x128_f8f6f4 a[156:159], v[160:167], v[248:255], a[156:159]\n\t" \
  "ds_read_b128 v[144:147], v2 offset:2048\n\t" \
  "ds_read_b128 v[148:151], v2 offset:10240\n\t" \
  "v_mfma_f32_16x16x128_f8f6f4 a[188:191], v[168:175], v[248:255], a[188:191]\n\t" \
  "ds_read_b128 v[152:155], v2 offset:3072\n\t" \
  "ds_read_b128 v[156:159], v2 offset:11264\n\t" \
  "v_mfma_f32_16x16x128_f8f6f4 a[184:187], v[168:175], v[240:247], a[184:187]\n\t" \
  "v_min_f32_e32 v24, s60, v124\n\t" \
  "v_min_f32_e32 v25, s60, v125\n\t" \
  "v_min_f32_e32 v26, s60, v126\n\t" \
  "v_min_f32_e32 v27, s60, v127\n\t" \
  "v_mfma_f32_16x16x128_f8f6f4 a[180:183], v[168:175], v[232:239], a[180:183]\n\t" \
  "v_exp_f32_e32 v24, v24\n\t" \
  "v_exp_f32_e32 v25, v25\n\t" \
  "v_mfma_f32_16x16x128_f8f6f4 a[176:179], v[168:175], v[224:231], a[176:179]\n\t" \
  "s_mov_b32 s53, s51\n\t" \
  "s_add_u32 m0, s48, 32768\n\t" \
  "v_mfma_f32_16x16x128_f8f6f4 a[208:211], v[176:183], v[224:231], a[208:211]\n\t" \
  "buffer_load_dwordx4 v0, s[36:39], s53 offen sc0 sc1 lds\n\t" \
  "v_mfma_f32_16x16x128_f8f6f4 a[212:215], v[176:183], v[232:239], a[212:215]\n\t" \
  "v_exp_f32_e32 v26, v26\n\t" \
  "v_exp_f32_e32 v27, v27\n\t" \
  "v_mfma_f32_16x16x128_f8f6f4 a[216:219], v[176:183], v[240:247], a[216:219]\n\t" \
  "buffer_load_dwordx4 v0, s[36:39], s53 offen offset:1024 sc0 sc1 lds\n\t" \
  "v_mfma_f32_16x16x128_f8f6f4 a[220:223], v[176:183], v[248:255], a[220:223]\n\t" \
  "v_fma_f32 v24, v24, s58, s58\n\t" \
  "v_fma_f32 v25, v25, s58, s58\n\t" \
  "v_fma_f32 v26, v26, s58, s58\n\t" \
  "v_fma_f32 v27, v27, s58, s58\n\t" \
  "v_mfma_f32_16x16x128_f8f6f4 a[252:255], v[184:191], v[248:255], a[252:255]\n\t" \
  "buffer_load_dwordx4 v0, s[36:39], s53 offen offset:2048 sc0 sc1 lds\n\t" \
  "v_mfma_f32_16x16x128_f8f6f4 a[248:251], v[184:191], v[240:247], a[248:251]\n\t" \
  "v_rcp_f32_e32 v24, v24\n\t" \
  "v_rcp_f32_e32 v25, v25\n\t" \
  "v_mfma_f32_16x16x128_f8f6f4 a[244:247], v[184:191], v[232:239], a[244:247]\n\t" \
  "buffer_load_dwordx4 v0, s[36:39], s53 offen offset:3072 sc0 sc1 lds\n\t" \
  "v_mfma_f32_16x16x128_f8f6f4 a[240:243], v[184:191], v[224:231], a[240:243]\n\t" \
  "v_rcp_f32_e32 v26, v26\n\t" \
  "v_rcp_f32_e32 v27, v27\n\t" \
  "s_waitcnt lgkmcnt(0)\n\t" \
  "v_mfma_f32_16x16x128_f8f6f4 a[128:131], v[160:167], v[192:199], a[128:131]\n\t" \
  "v_mul_f32_e32 v124, v124, v24\n\t" \
  "v_mul_f32_e32 v125, v125, v25\n\t" \
  "v_mul_f32_e32 v126, v126, v26\n\t" \
  "v_mul_f32_e32 v127, v127, v27\n\t" \
  "v_mfma_f32_16x16x128_f8f6f4 a[132:135], v[160:167], v[200:207], a[132:135]\n\t" \
  "ds_read_b128 v[224:227], v4 offset:16384\n\t" \
  "ds_read_b128 v[228:231], v4 offset:24576\n\t" \
  "v_mfma_f32_16x16x128_f8f6f4 a[136:139], v[160:167], v[208:215], a[136:139]\n\t" \
  "ds_read_b128 v[232:235], v4 offset:17408\n\t" \
  "ds_read_b128 v[236:239], v4 offset:25600\n\t" \
  "v_mfma_f32_16x16x128_f8f6f4 a[140:143], v[160:167], v[216:223], a[140:143]\n\t" \
  "ds_read_b128 v[240:243], v4 offset:18432\n\t" \
  "ds_read_b128 v[244:247], v4 offset:26624\n\t" \
  "v_mfma_f32_16x16x128_f8f6f4 a[172:175], v[168:175], v[216:223], a[172:175]\n\t" \
  "ds_read_b128 v[248:251], v4 offset:19456\n\t" \
  "ds_read_b128 v[252:255], v4 offset:27648\n\t" \
  "v_mfma_f32_16x16x128_f8f6f4 a[168:171], v[168:175], v[208:215], a[168:171]\n\t" \
  "s_xor_b32 s91, s68, 5439488\n\t" \
  "buffer_store_dwordx2 v[124:125], v5, s[40:43], s91 offen nt\n\t" \
  "v_mfma_f32_16x16x128_f8f6f4 a[164:167], v[168:175], v[200:207], a[164:167]\n\t" \
  "buffer_store_dwordx2 v[126:127], v5, s[40:43], s91 offen offset:128 nt\n\t" \
  "v_mfma_f32_16x16x128_f8f6f4 a[160:163], v[168:175], v[192:199], a[160:163]\n\t" \
  "s_add_u32 s53, s49, 16384\n\t" \
  "s_add_u32 m0, s48, 49152\n\t" \
  "v_mfma_f32_16x16x128_f8f6f4 a[192:195], v[176:183], v[192:199], a[192:195]\n\t" \
  "buffer_load_dwordx4 v0, s[36:39], s53 offen sc0 sc1 lds\n\t" \
  "v_mfma_f32_16x16x128_f8f6f4 a[196:199], v[176:183], v[200:207], a[196:199]\n\t" \
  "ds_read_b128 v[32:35], v6 offset:0\n\t" \
  "ds_read_b128 v[36:39], v6 offset:1024\n\t" \
  "ds_read_b128 v[40:43], v6 offset:2048\n\t" \
  "ds_read_b128 v[44:47], v6 offset:3072\n\t" \
  "v_mfma_f32_16x16x128_f8f6f4 a[200:203], v[176:183], v[208:215], a[200:203]\n\t" \
  "buffer_load_dwordx4 v0, s[36:39], s53 offen offset:1024 sc0 sc1 lds\n\t" \
  "v_mfma_f32_16x16x128_f8f6f4 a[204:207], v[176:183], v[216:223], a[204:207]\n\t" \
  "v_mfma_f32_16x16x128_f8f6f4 a[236:239], v[184:191], v[216:223], a[236:239]\n\t" \
  "buffer_load_dwordx4 v0, s[36:39], s53 offen offset:2048 sc0 sc1 lds\n\t" \
  "v_mfma_f32_16x16x128_f8f6f4 a[232:235], v[184:191], v[208:215], a[232:235]\n\t" \
  "v_mfma_f32_16x16x128_f8f6f4 a[228:231], v[184:191], v[200:207], a[228:231]\n\t" \
  "buffer_load_dwordx4 v0, s[36:39], s53 offen offset:3072 sc0 sc1 lds\n\t" \
  "v_mfma_f32_16x16x128_f8f6f4 a[224:227], v[184:191], v[192:199], a[224:227]\n\t" \
  "s_waitcnt lgkmcnt(0)\n\t" \
  "v_mfma_f32_16x16x128_f8f6f4 a[16:19], v[128:135], v[224:231], a[16:19]\n\t" \
  "s_waitcnt vmcnt(20)\n\t" \
  "s_barrier\n\t" \
  "v_mfma_f32_16x16x128_f8f6f4 a[20:23], v[128:135], v[232:239], a[20:23]\n\t" \
  "ds_read_b128 v[192:195], v4 offset:32768\n\t" \
  "ds_read_b128 v[196:199], v4 offset:40960\n\t" \
  "v_mfma_f32_16x16x128_f8f6f4 a[24:27], v[128:135], v[240:247], a[24:27]\n\t" \
  "ds_read_b128 v[200:203], v4 offset:33792\n\t" \
  "ds_read_b128 v[204:207], v4 offset:41984\n\t" \
  "v_mfma_f32_16x16x128_f8f6f4 a[28:31], v[128:135], v[248:255], a[28:31]\n\t" \
  "ds_read_b128 v[208:211], v4 offset:34816\n\t" \
  "ds_read_b128 v[212:215], v4 offset:43008\n\t" \
  "v_mfma_f32_16x16x128_f8f6f4 a[60:63], v[136:143], v[248:255], a[60:63]\n\t" \
  "ds_read_b128 v[216:219], v4 offset:35840\n\t" \
  "ds_read_b128 v[220:223], v4 offset:44032\n\t" \
  "v_mfma_f32_16x16x128_f8f6f4 a[56:59], v[136:143], v[240:247], a[56:59]\n\t" \
  "v_mul_f32_e32 v48, s56, v32\n\t" \
  "v_mul_f32_e32 v49, s56, v36\n\t" \
  "v_mul_f32_e32 v50, s56, v40\n\t" \
  "v_mul_f32_e32 v51, s56, v44\n\t" \
  "v_mfma_f32_16x16x128_f8f6f4 a[52:55], v[136:143], v[232:239], a[52:55]\n\t" \
  "v_mul_f32_e32 v52, s56, v33\n\t" \
  "v_mul_f32_e32 v53, s56, v37\n\t" \
  "v_mul_f32_e32 v54, s56, v41\n\t" \
  "v_mul_f32_e32 v55, s56, v45\n\t" \
  "v_mfma_f32_16x16x128_f8f6f4 a[48:51], v[136:143], v[224:231], a[48:51]\n\t" \
  "s_add_u32 s53, s49, 32768\n\t" \
  "s_add_u32 m0, s48, 65536\n\t" \
  "v_mfma_f32_16x16x128_f8f6f4 a[80:83], v[144:151], v[224:231], a[80:83]\n\t" \
  "buffer_load_dwordx4 v0, s[36:39], s53 offen sc0 sc1 lds\n\t" \
  "v_mfma_f32_16x16x128_f8f6f4 a[84:87], v[144:151], v[232:239], a[84:87]\n\t" \
  "v_mul_f32_e32 v56, s56, v34\n\t" \
  "v_mul_f32_e32 v57, s56, v38\n\t" \
  "v_mul_f32_e32 v58, s56, v42\n\t" \
  "v_mul_f32_e32 v59, s56, v46\n\t" \
  "v_mfma_f32_16x16x128_f8f6f4 a[88:91], v[144:151], v[240:247], a[88:91]\n\t" \
  "buffer_load_dwordx4 v0, s[36:39], s53 offen offset:1024 sc0 sc1 lds\n\t" \
  "v_mfma_f32_16x16x128_f8f6f4 a[92:95], v[144:151], v[248:255], a[92:95]\n\t" \
  "v_mul_f32_e32 v60, s56, v35\n\t" \
  "v_mul_f32_e32 v61, s56, v39\n\t" \
  "v_mul_f32_e32 v62, s56, v43\n\t" \
  "v_mul_f32_e32 v63, s56, v47\n\t" \
  "v_mfma_f32_16x16x128_f8f6f4 a[124:127], v[152:159], v[248:255], a[124:127]\n\t" \
  "buffer_load_dwordx4 v0, s[36:39], s53 offen offset:2048 sc0 sc1 lds\n\t" \
  "v_mfma_f32_16x16x128_f8f6f4 a[120:123], v[152:159], v[240:247], a[120:123]\n\t" \
  "v_min_f32_e32 v28, s60, v48\n\t" \
  "v_min_f32_e32 v29, s60, v49\n\t" \
  "v_min_f32_e32 v30, s60, v50\n\t" \
  "v_min_f32_e32 v31, s60, v51\n\t" \
  "v_mfma_f32_16x16x128_f8f6f4 a[116:119], v[152:159], v[232:239], a[116:119]\n\t" \
  "buffer_load_dwordx4 v0, s[36:39], s53 offen offset:3072 sc0 sc1 lds\n\t" \
  "v_mfma_f32_16x16x128_f8f6f4 a[112:115], v[152:159], v[224:231], a[112:115]\n\t" \
  "v_exp_f32_e32 v28, v28\n\t" \
  "v_exp_f32_e32 v29, v29\n\t" \
  "s_waitcnt lgkmcnt(0)\n\t" \
  "v_mfma_f32_16x16x128_f8f6f4 a[0:3], v[128:135], v[192:199], a[0:3]\n\t" \
  "v_exp_f32_e32 v30, v30\n\t" \
  "v_exp_f32_e32 v31, v31\n\t" \
  "v_mfma_f32_16x16x128_f8f6f4 a[4:7], v[128:135], v[200:207], a[4:7]\n\t" \
  "ds_read_b128 v[160:163], v2 offset:49152\n\t" \
  "ds_read_b128 v[164:167], v2 offset:57344\n\t" \
  "v_mfma_f32_16x16x128_f8f6f4 a[8:11], v[128:135], v[208:215], a[8:11]\n\t" \
  "ds_read_b128 v[168:171], v2 offset:50176\n\t" \
  "ds_read_b128 v[172:175], v2 offset:58368\n\t" \
  "v_mfma_f32_16x16x128_f8f6f4 a[12:15], v[128:135], v[216:223], a[12:15]\n\t" \
  "ds_read_b128 v[176:179], v2 offset:51200\n\t" \
  "ds_read_b128 v[180:183], v2 offset:59392\n\t" \
  "v_mfma_f32_16x16x128_f8f6f4 a[44:47], v[136:143], v[216:223], a[44:47]\n\t" \
  "ds_read_b128 v[184:187], v2 offset:52224\n\t" \
  "ds_read_b128 v[188:191], v2 offset:60416\n\t" \
  "v_mfma_f32_16x16x128_f8f6f4 a[40:43], v[136:143], v[208:215], a[40:43]\n\t" \
  "v_fma_f32 v28, v28, s58, s58\n\t" \
  "v_fma_f32 v29, v29, s58, s58\n\t" \
  "v_fma_f32 v30, v30, s58, s58\n\t" \
  "v_fma_f32 v31, v31, s58, s58\n\t" \
  "v_mfma_f32_16x16x128_f8f6f4 a[36:39], v[136:143], v[200:207], a[36:39]\n\t" \
  "v_rcp_f32_e32 v28, v28\n\t" \
  "v_rcp_f32_e32 v29, v29\n\t" \
  "v_mfma_f32_16x16x128_f8f6f4 a[32:35], v[136:143], v[192:199], a[32:35]\n\t" \
  "s_add_u32 s53, s51, 16384\n\t" \
  "s_add_u32 m0, s48, 81920\n\t" \
  "v_mfma_f32_16x16x128_f8f6f4 a[64:67], v[144:151], v[192:199], a[64:67]\n\t" \
  "buffer_load_dwordx4 v0, s[36:39], s53 offen sc0 sc1 lds\n\t" \
  "v_mfma_f32_16x16x128_f8f6f4 a[68:71], v[144:151], v[200:207], a[68:71]\n\t" \
  "v_rcp_f32_e32 v30, v30\n\t" \
  "v_rcp_f32_e32 v31, v31\n\t" \
  "v_mfma_f32_16x16x128_f8f6f4 a[72:75], v[144:151], v[208:215], a[72:75]\n\t" \
  "buffer_load_dwordx4 v0, s[36:39], s53 offen offset:1024 sc0 sc1 lds\n\t" \
  "v_mfma_f32_16x16x128_f8f6f4 a[76:79], v[144:151], v[216:223], a[76:79]\n\t" \
  "v_mul_f32_e32 v48, v48, v28\n\t" \
  "v_mul_f32_e32 v49, v49, v29\n\t" \
  "v_mul_f32_e32 v50, v50, v30\n\t" \
  "v_mul_f32_e32 v51, v51, v31\n\t" \
  "v_mfma_f32_16x16x128_f8f6f4 a[108:111], v[152:159], v[216:223], a[108:111]\n\t" \
  "buffer_load_dwordx4 v0, s[36:39], s53 offen offset:2048 sc0 sc1 lds\n\t" \
  "v_mfma_f32_16x16x128_f8f6f4 a[104:107], v[152:159], v[208:215], a[104:107]\n\t" \
  "s_xor_b32 s91, s68, 6291456\n\t" \
  "buffer_store_dwordx2 v[48:49], v5, s[40:43], s91 offen nt\n\t" \
  "v_mfma_f32_16x16x128_f8f6f4 a[100:103], v[152:159], v[200:207], a[100:103]\n\t" \
  "buffer_load_dwordx4 v0, s[36:39], s53 offen offset:3072 sc0 sc1 lds\n\t" \
  "v_mfma_f32_16x16x128_f8f6f4 a[96:99], v[152:159], v[192:199], a[96:99]\n\t" \
  "buffer_store_dwordx2 v[50:51], v5, s[40:43], s91 offen offset:128 nt\n\t" \
  "s_waitcnt lgkmcnt(0)\n\t" \
  "v_mfma_f32_16x16x128_f8f6f4 a[128:131], v[160:167], v[192:199], a[128:131]\n\t" \
  "s_waitcnt vmcnt(20)\n\t" \
  "s_barrier\n\t" \
  "v_mfma_f32_16x16x128_f8f6f4 a[132:135], v[160:167], v[200:207], a[132:135]\n\t" \
  "ds_read_b128 v[128:131], v1 offset:0\n\t" \
  "ds_read_b128 v[132:135], v1 offset:8192\n\t" \
  "v_mfma_f32_16x16x128_f8f6f4 a[136:139], v[160:167], v[208:215], a[136:139]\n\t" \
  "ds_read_b128 v[136:139], v1 offset:1024\n\t" \
  "ds_read_b128 v[140:143], v1 offset:9216\n\t" \
  "v_mfma_f32_16x16x128_f8f6f4 a[140:143], v[160:167], v[216:223], a[140:143]\n\t" \
  "ds_read_b128 v[144:147], v1 offset:2048\n\t" \
  "ds_read_b128 v[148:151], v1 offset:10240\n\t" \
  "v_mfma_f32_16x16x128_f8f6f4 a[172:175], v[168:175], v[216:223], a[172:175]\n\t" \
  "ds_read_b128 v[152:155], v1 offset:3072\n\t" \
  "ds_read_b128 v[156:159], v1 offset:11264\n\t" \
  "v_mfma_f32_16x16x128_f8f6f4 a[168:171], v[168:175], v[208:215], a[168:171]\n\t" \
  "v_min_f32_e32 v12, s60, v52\n\t" \
  "v_min_f32_e32 v13, s60, v53\n\t" \
  "v_min_f32_e32 v14, s60, v54\n\t" \
  "v_min_f32_e32 v15, s60, v55\n\t" \
  "v_mfma_f32_16x16x128_f8f6f4 a[164:167], v[168:175], v[200:207], a[164:167]\n\t" \
  "v_exp_f32_e32 v12, v12\n\t" \
  "v_exp_f32_e32 v13, v13\n\t" \
  "v_mfma_f32_16x16x128_f8f6f4 a[160:163], v[168:175], v[192:199], a[160:163]\n\t" \
  "s_add_u32 s53, s50, 16384\n\t" \
  "s_add_u32 m0, s48, 98304\n\t" \
  "v_mfma_f32_16x16x128_f8f6f4 a[192:195], v[176:183], v[192:199], a[192:195]\n\t" \
  "buffer_load_dwordx4 v0, s[36:39], s53 offen sc0 sc1 lds\n\t" \
  "v_mfma_f32_16x16x128_f8f6f4 a[196:199], v[176:183], v[200:207], a[196:199]\n\t" \
  "v_exp_f32_e32 v14, v14\n\t" \
  "v_exp_f32_e32 v15, v15\n\t" \
  "v_mfma_f32_16x16x128_f8f6f4 a[200:203], v[176:183], v[208:215], a[200:203]\n\t" \
  "buffer_load_dwordx4 v0, s[36:39], s53 offen offset:1024 sc0 sc1 lds\n\t" \
  "v_mfma_f32_16x16x128_f8f6f4 a[204:207], v[176:183], v[216:223], a[204:207]\n\t" \
  "v_fma_f32 v12, v12, s58, s58\n\t" \
  "v_fma_f32 v13, v13, s58, s58\n\t" \
  "v_fma_f32 v14, v14, s58, s58\n\t" \
  "v_fma_f32 v15, v15, s58, s58\n\t" \
  "v_mfma_f32_16x16x128_f8f6f4 a[236:239], v[184:191], v[216:223], a[236:239]\n\t" \
  "buffer_load_dwordx4 v0, s[36:39], s53 offen offset:2048 sc0 sc1 lds\n\t" \
  "v_mfma_f32_16x16x128_f8f6f4 a[232:235], v[184:191], v[208:215], a[232:235]\n\t" \
  "v_rcp_f32_e32 v12, v12\n\t" \
  "v_rcp_f32_e32 v13, v13\n\t" \
  "v_mfma_f32_16x16x128_f8f6f4 a[228:231], v[184:191], v[200:207], a[228:231]\n\t" \
  "buffer_load_dwordx4 v0, s[36:39], s53 offen offset:3072 sc0 sc1 lds\n\t" \
  "v_mfma_f32_16x16x128_f8f6f4 a[224:227], v[184:191], v[192:199], a[224:227]\n\t" \
  "v_rcp_f32_e32 v14, v14\n\t" \
  "v_rcp_f32_e32 v15, v15\n\t" \
  "s_waitcnt lgkmcnt(0)\n\t" \
  "v_mfma_f32_16x16x128_f8f6f4 a[144:147], v[160:167], v[224:231], a[144:147]\n\t" \
  "v_mul_f32_e32 v52, v52, v12\n\t" \
  "v_mul_f32_e32 v53, v53, v13\n\t" \
  "v_mul_f32_e32 v54, v54, v14\n\t" \
  "v_mul_f32_e32 v55, v55, v15\n\t" \
  "v_mfma_f32_16x16x128_f8f6f4 a[148:151], v[160:167], v[232:239], a[148:151]\n\t" \
  "ds_read_b128 v[192:195], v3 offset:16384\n\t" \
  "ds_read_b128 v[196:199], v3 offset:24576\n\t" \
  "v_mfma_f32_16x16x128_f8f6f4 a[152:155], v[160:167], v[240:247], a[152:155]\n\t" \
  "ds_read_b128 v[200:203], v3 offset:17408\n\t" \
  "ds_read_b128 v[204:207], v3 offset:25600\n\t" \
  "v_mfma_f32_16x16x128_f8f6f4 a[156:159], v[160:167], v[248:255], a[156:159]\n\t" \
  "ds_read_b128 v[208:211], v3 offset:18432\n\t" \
  "ds_read_b128 v[212:215], v3 offset:26624\n\t" \
  "v_mfma_f32_16x16x128_f8f6f4 a[188:191], v[168:175], v[248:255], a[188:191]\n\t" \
  "ds_read_b128 v[216:219], v3 offset:19456\n\t" \
  "ds_read_b128 v[220:223], v3 offset:27648\n\t" \
  "v_mfma_f32_16x16x128_f8f6f4 a[184:187], v[168:175], v[240:247], a[184:187]\n\t" \
  "s_xor_b32 s91, s68, 6356992\n\t" \
  "buffer_store_dwordx2 v[52:53], v5, s[40:43], s91 offen nt\n\t" \
  "v_mfma_f32_16x16x128_f8f6f4 a[180:183], v[168:175], v[232:239], a[180:183]\n\t" \
  "buffer_store_dwordx2 v[54:55], v5, s[40:43], s91 offen offset:128 nt\n\t" \
  "v_mfma_f32_16x16x128_f8f6f4 a[176:179], v[168:175], v[224:231], a[176:179]\n\t" \
  "s_add_u32 s53, s49, 49152\n\t" \
  "s_add_u32 m0, s48, 114688\n\t" \
  "v_mfma_f32_16x16x128_f8f6f4 a[208:211], v[176:183], v[224:231], a[208:211]\n\t" \
  "buffer_load_dwordx4 v0, s[36:39], s53 offen sc0 sc1 lds\n\t" \
  "v_mfma_f32_16x16x128_f8f6f4 a[212:215], v[176:183], v[232:239], a[212:215]\n\t" \
  "v_min_f32_e32 v16, s60, v56\n\t" \
  "v_min_f32_e32 v17, s60, v57\n\t" \
  "v_min_f32_e32 v18, s60, v58\n\t" \
  "v_min_f32_e32 v19, s60, v59\n\t" \
  "v_mfma_f32_16x16x128_f8f6f4 a[216:219], v[176:183], v[240:247], a[216:219]\n\t" \
  "buffer_load_dwordx4 v0, s[36:39], s53 offen offset:1024 sc0 sc1 lds\n\t" \
  "v_mfma_f32_16x16x128_f8f6f4 a[220:223], v[176:183], v[248:255], a[220:223]\n\t" \
  "v_exp_f32_e32 v16, v16\n\t" \
  "v_exp_f32_e32 v17, v17\n\t" \
  "v_mfma_f32_16x16x128_f8f6f4 a[252:255], v[184:191], v[248:255], a[252:255]\n\t" \
  "buffer_load_dwordx4 v0, s[36:39], s53 offen offset:2048 sc0 sc1 lds\n\t" \
  "v_mfma_f32_16x16x128_f8f6f4 a[248:251], v[184:191], v[240:247], a[248:251]\n\t" \
  "v_exp_f32_e32 v18, v18\n\t" \
  "v_exp_f32_e32 v19, v19\n\t" \
  "v_mfma_f32_16x16x128_f8f6f4 a[244:247], v[184:191], v[232:239], a[244:247]\n\t" \
  "buffer_load_dwordx4 v0, s[36:39], s53 offen offset:3072 sc0 sc1 lds\n\t" \
  "v_mfma_f32_16x16x128_f8f6f4 a[240:243], v[184:191], v[224:231], a[240:243]\n\t" \
  "s_add_u32 s49, s49, 65536\n\t" \
  "s_add_u32 s50, s50, 32768\n\t" \
  "s_add_u32 s51, s51, 32768\n\t" \
  "s_waitcnt lgkmcnt(0)\n\t" \
  "v_mfma_f32_16x16x128_f8f6f4 a[0:3], v[128:135], v[192:199], a[0:3]\n\t" \
  "s_waitcnt vmcnt(20)\n\t" \
  "s_barrier\n\t" \
  "v_mfma_f32_16x16x128_f8f6f4 a[4:7], v[128:135], v[200:207], a[4:7]\n\t" \
  "ds_read_b128 v[224:227], v3 offset:32768\n\t" \
  "ds_read_b128 v[228:231], v3 offset:40960\n\t" \
  "v_mfma_f32_16x16x128_f8f6f4 a[8:11], v[128:135], v[208:215], a[8:11]\n\t" \
  "ds_read_b128 v[232:235], v3 offset:33792\n\t" \
  "ds_read_b128 v[236:239], v3 offset:41984\n\t" \
  "v_mfma_f32_16x16x128_f8f6f4 a[12:15], v[128:135], v[216:223], a[12:15]\n\t" \
  "ds_read_b128 v[240:243], v3 offset:34816\n\t" \
  "ds_read_b128 v[244:247], v3 offset:43008\n\t" \
  "v_and_b32_e32 v8, 0x3ff, v8\n\t" \
  "v_add_u32_e32 v8, s92, v8\n\t" \
  "v_mfma_f32_16x16x128_f8f6f4 a[44:47], v[136:143], v[216:223], a[44:47]\n\t" \
  "ds_read_b128 v[248:251], v3 offset:35840\n\t" \
  "ds_read_b128 v[252:255], v3 offset:44032\n\t" \
  "v_mfma_f32_16x16x128_f8f6f4 a[40:43], v[136:143], v[208:215], a[40:43]\n\t" \
  "v_mbcnt_lo_u32_b32 v9, -1, 0\n\t" \
  "v_mbcnt_hi_u32_b32 v9, -1, v9\n\t" \
  "v_bfe_u32 v9, v9, 3, 1\n\t" \
  "v_mfma_f32_16x16x128_f8f6f4 a[36:39], v[136:143], v[200:207], a[36:39]\n\t" \
  "v_mul_u32_u24_e32 v9, s61, v9\n\t" \
  "v_add_u32_e32 v8, v9, v8\n\t" \
  "v_mfma_f32_16x16x128_f8f6f4 a[32:35], v[136:143], v[192:199], a[32:35]\n\t" \
  "s_mov_b32 s53, s49\n\t" \
  "s_mov_b32 m0, s48\n\t" \
  "v_mfma_f32_16x16x128_f8f6f4 a[64:67], v[144:151], v[192:199], a[64:67]\n\t" \
  "buffer_load_dwordx4 v0, s[36:39], s53 offen sc0 sc1 lds\n\t" \
  "v_mfma_f32_16x16x128_f8f6f4 a[68:71], v[144:151], v[200:207], a[68:71]\n\t" \
  "v_fma_f32 v16, v16, s58, s58\n\t" \
  "v_fma_f32 v17, v17, s58, s58\n\t" \
  "v_fma_f32 v18, v18, s58, s58\n\t" \
  "v_fma_f32 v19, v19, s58, s58\n\t" \
  "v_mfma_f32_16x16x128_f8f6f4 a[72:75], v[144:151], v[208:215], a[72:75]\n\t" \
  "buffer_load_dwordx4 v0, s[36:39], s53 offen offset:1024 sc0 sc1 lds\n\t" \
  "v_mfma_f32_16x16x128_f8f6f4 a[76:79], v[144:151], v[216:223], a[76:79]\n\t" \
  "v_rcp_f32_e32 v16, v16\n\t" \
  "v_rcp_f32_e32 v17, v17\n\t" \
  "v_mfma_f32_16x16x128_f8f6f4 a[108:111], v[152:159], v[216:223], a[108:111]\n\t" \
  "buffer_load_dwordx4 v0, s[36:39], s53 offen offset:2048 sc0 sc1 lds\n\t" \
  "v_mfma_f32_16x16x128_f8f6f4 a[104:107], v[152:159], v[208:215], a[104:107]\n\t" \
  "v_rcp_f32_e32 v18, v18\n\t" \
  "v_rcp_f32_e32 v19, v19\n\t" \
  "v_mfma_f32_16x16x128_f8f6f4 a[100:103], v[152:159], v[200:207], a[100:103]\n\t" \
  "buffer_load_dwordx4 v0, s[36:39], s53 offen offset:3072 sc0 sc1 lds\n\t" \
  "v_mfma_f32_16x16x128_f8f6f4 a[96:99], v[152:159], v[192:199], a[96:99]\n\t" \
  "v_mul_f32_e32 v56, v56, v16\n\t" \
  "v_mul_f32_e32 v57, v57, v17\n\t" \
  "v_mul_f32_e32 v58, v58, v18\n\t" \
  "v_mul_f32_e32 v59, v59, v19\n\t" \
  "s_waitcnt lgkmcnt(0)\n\t" \
  "v_mfma_f32_16x16x128_f8f6f4 a[16:19], v[128:135], v[224:231], a[16:19]\n\t" \
  "s_xor_b32 s91, s68, 6422528\n\t" \
  "buffer_store_dwordx2 v[56:57], v5, s[40:43], s91 offen nt\n\t" \
  "v_mfma_f32_16x16x128_f8f6f4 a[20:23], v[128:135], v[232:239], a[20:23]\n\t" \
  "ds_read_b128 v[160:163], v1 offset:49152\n\t" \
  "ds_read_b128 v[164:167], v1 offset:57344\n\t" \
  "v_mfma_f32_16x16x128_f8f6f4 a[24:27], v[128:135], v[240:247], a[24:27]\n\t" \
  "ds_read_b128 v[168:171], v1 offset:50176\n\t" \
  "ds_read_b128 v[172:175], v1 offset:58368\n\t" \
  "v_mfma_f32_16x16x128_f8f6f4 a[28:31], v[128:135], v[248:255], a[28:31]\n\t" \
  "ds_read_b128 v[176:179], v1 offset:51200\n\t" \
  "ds_read_b128 v[180:183], v1 offset:59392\n\t" \
  "v_mfma_f32_16x16x128_f8f6f4 a[60:63], v[136:143], v[248:255], a[60:63]\n\t" \
  "ds_read_b128 v[184:187], v1 offset:52224\n\t" \
  "ds_read_b128 v[188:191], v1 offset:60416\n\t" \
  "v_mfma_f32_16x16x128_f8f6f4 a[56:59], v[136:143], v[240:247], a[56:59]\n\t" \
  "buffer_store_dwordx2 v[58:59], v5, s[40:43], s91 offen offset:128 nt\n\t" \
  "v_mfma_f32_16x16x128_f8f6f4 a[52:55], v[136:143], v[232:239], a[52:55]\n\t" \
  "v_min_f32_e32 v20, s60, v60\n\t" \
  "v_min_f32_e32 v21, s60, v61\n\t" \
  "v_min_f32_e32 v22, s60, v62\n\t" \
  "v_min_f32_e32 v23, s60, v63\n\t" \
  "v_mfma_f32_16x16x128_f8f6f4 a[48:51], v[136:143], v[224:231], a[48:51]\n\t" \
  "s_mov_b32 s53, s50\n\t" \
  "s_add_u32 m0, s48, 16384\n\t" \
  "v_mfma_f32_16x16x128_f8f6f4 a[80:83], v[144:151], v[224:231], a[80:83]\n\t" \
  "buffer_load_dwordx4 v0, s[36:39], s53 offen sc0 sc1 lds\n\t" \
  "v_mfma_f32_16x16x128_f8f6f4 a[84:87], v[144:151], v[232:239], a[84:87]\n\t" \
  "v_exp_f32_e32 v20, v20\n\t" \
  "v_exp_f32_e32 v21, v21\n\t" \
  "v_mfma_f32_16x16x128_f8f6f4 a[88:91], v[144:151], v[240:247], a[88:91]\n\t" \
  "buffer_load_dwordx4 v0, s[36:39], s53 offen offset:1024 sc0 sc1 lds\n\t" \
  "v_mfma_f32_16x16x128_f8f6f4 a[92:95], v[144:151], v[248:255], a[92:95]\n\t" \
  "v_exp_f32_e32 v22, v22\n\t" \
  "v_exp_f32_e32 v23, v23\n\t" \
  "v_mfma_f32_16x16x128_f8f6f4 a[124:127], v[152:159], v[248:255], a[124:127]\n\t" \
  "buffer_load_dwordx4 v0, s[36:39], s53 offen offset:2048 sc0 sc1 lds\n\t" \
  "v_mfma_f32_16x16x128_f8f6f4 a[120:123], v[152:159], v[240:247], a[120:123]\n\t" \
  "v_fma_f32 v20, v20, s58, s58\n\t" \
  "v_fma_f32 v21, v21, s58, s58\n\t" \
  "v_fma_f32 v22, v22, s58, s58\n\t" \
  "v_fma_f32 v23, v23, s58, s58\n\t" \
  "v_mfma_f32_16x16x128_f8f6f4 a[116:119], v[152:159], v[232:239], a[116:119]\n\t" \
  "buffer_load_dwordx4 v0, s[36:39], s53 offen offset:3072 sc0 sc1 lds\n\t" \
  "v_mfma_f32_16x16x128_f8f6f4 a[112:115], v[152:159], v[224:231], a[112:115]\n\t" \
  "v_rcp_f32_e32 v20, v20\n\t" \
  "v_rcp_f32_e32 v21, v21\n\t" \
  "s_waitcnt lgkmcnt(0)\n\t" \
  "v_mfma_f32_16x16x128_f8f6f4 a[144:147], v[160:167], v[224:231], a[144:147]\n\t" \
  "s_waitcnt vmcnt(21)\n\t" \
  "s_barrier\n\t" \
  "v_mfma_f32_16x16x128_f8f6f4 a[148:151], v[160:167], v[232:239], a[148:151]\n\t" \
  "ds_read_b128 v[128:131], v2 offset:0\n\t" \
  "ds_read_b128 v[132:135], v2 offset:8192\n\t" \
  "v_mfma_f32_16x16x128_f8f6f4 a[152:155], v[160:167], v[240:247], a[152:155]\n\t" \
  "ds_read_b128 v[136:139], v2 offset:1024\n\t" \
  "ds_read_b128 v[140:143], v2 offset:9216\n\t" \
  "v_mfma_f32_16x16x128_f8f6f4 a[156:159], v[160:167], v[248:255], a[156:159]\n\t" \
  "ds_read_b128 v[144:147], v2 offset:2048\n\t" \
  "ds_read_b128 v[148:151], v2 offset:10240\n\t" \
  "v_mfma_f32_16x16x128_f8f6f4 a[188:191], v[168:175], v[248:255], a[188:191]\n\t" \
  "ds_read_b128 v[152:155], v2 offset:3072\n\t" \
  "ds_read_b128 v[156:159], v2 offset:11264\n\t" \
  "v_mfma_f32_16x16x128_f8f6f4 a[184:187], v[168:175], v[240:247], a[184:187]\n\t" \
  "v_rcp_f32_e32 v22, v22\n\t" \
  "v_rcp_f32_e32 v23, v23\n\t" \
  "v_mfma_f32_16x16x128_f8f6f4 a[180:183], v[168:175], v[232:239], a[180:183]\n\t" \
  "v_mul_f32_e32 v60, v60, v20\n\t" \
  "v_mul_f32_e32 v61, v61, v21\n\t" \
  "v_mul_f32_e32 v62, v62, v22\n\t" \
  "v_mul_f32_e32 v63, v63, v23\n\t" \
  "v_mfma_f32_16x16x128_f8f6f4 a[176:179], v[168:175], v[224:231], a[176:179]\n\t" \
  "s_mov_b32 s53, s51\n\t" \
  "s_add_u32 m0, s48, 32768\n\t" \
  "s_xor_b32 s91, s68, 6488064\n\t" \
  "buffer_store_dwordx2 v[60:61], v5, s[40:43], s91 offen nt\n\t" \
  "v_mfma_f32_16x16x128_f8f6f4 a[208:211], v[176:183], v[224:231], a[208:211]\n\t" \
  "buffer_load_dwordx4 v0, s[36:39], s53 offen sc0 sc1 lds\n\t" \
  "v_mfma_f32_16x16x128_f8f6f4 a[212:215], v[176:183], v[232:239], a[212:215]\n\t" \
  "buffer_store_dwordx2 v[62:63], v5, s[40:43], s91 offen offset:128 nt\n\t" \
  "v_mfma_f32_16x16x128_f8f6f4 a[216:219], v[176:183], v[240:247], a[216:219]\n\t" \
  "buffer_load_dwordx4 v0, s[36:39], s53 offen offset:1024 sc0 sc1 lds\n\t" \
  "v_mfma_f32_16x16x128_f8f6f4 a[220:223], v[176:183], v[248:255], a[220:223]\n\t" \
  "ds_read_b128 v[64:67], v6 offset:4096\n\t" \
  "ds_read_b128 v[68:71], v6 offset:5120\n\t" \
  "ds_read_b128 v[72:75], v6 offset:6144\n\t" \
  "ds_read_b128 v[76:79], v6 offset:7168\n\t" \
  "v_mfma_f32_16x16x128_f8f6f4 a[252:255], v[184:191], v[248:255], a[252:255]\n\t" \
  "buffer_load_dwordx4 v0, s[36:39], s53 offen offset:2048 sc0 sc1 lds\n\t" \
  "v_mfma_f32_16x16x128_f8f6f4 a[248:251], v[184:191], v[240:247], a[248:251]\n\t" \
  "v_mfma_f32_16x16x128_f8f6f4 a[244:247], v[184:191], v[232:239], a[244:247]\n\t" \
  "buffer_load_dwordx4 v0, s[36:39], s53 offen offset:3072 sc0 sc1 lds\n\t" \
  "v_mfma_f32_16x16x128_f8f6f4 a[240:243], v[184:191], v[224:231], a[240:243]\n\t" \
  "s_waitcnt lgkmcnt(0)\n\t" \
  "v_mfma_f32_16x16x128_f8f6f4 a[128:131], v[160:167], v[192:199], a[128:131]\n\t" \
  "v_mul_f32_e32 v80, s56, v64\n\t" \
  "v_mul_f32_e32 v81, s56, v68\n\t" \
  "v_mul_f32_e32 v82, s56, v72\n\t" \
  "v_mul_f32_e32 v83, s56, v76\n\t" \
  "v_mfma_f32_16x16x128_f8f6f4 a[132:135], v[160:167], v[200:207], a[132:135]\n\t" \
  "ds_read_b128 v[224:227], v4 offset:16384\n\t" \
  "ds_read_b128 v[228:231], v4 offset:24576\n\t" \
  "v_mfma_f32_16x16x128_f8f6f4 a[136:139], v[160:167], v[208:215], a[136:139]\n\t" \
  "ds_read_b128 v[232:235], v4 offset:17408\n\t" \
  "ds_read_b128 v[236:239], v4 offset:25600\n\t" \
  "v_mfma_f32_16x16x128_f8f6f4 a[140:143], v[160:167], v[216:223], a[140:143]\n\t" \
  "ds_read_b128 v[240:243], v4 offset:18432\n\t" \
  "ds_read_b128 v[244:247], v4 offset:26624\n\t" \
  "v_mfma_f32_16x16x128_f8f6f4 a[172:175], v[168:175], v[216:223], a[172:175]\n\t" \
  "ds_read_b128 v[248:251], v4 offset:19456\n\t" \
  "ds_read_b128 v[252:255], v4 offset:27648\n\t" \
  "v_mfma_f32_16x16x128_f8f6f4 a[168:171], v[168:175], v[208:215], a[168:171]\n\t" \
  "v_mul_f32_e32 v84, s56, v65\n\t" \
  "v_mul_f32_e32 v85, s56, v69\n\t" \
  "v_mul_f32_e32 v86, s56, v73\n\t" \
  "v_mul_f32_e32 v87, s56, v77\n\t" \
  "v_mfma_f32_16x16x128_f8f6f4 a[164:167], v[168:175], v[200:207], a[164:167]\n\t" \
  "v_mul_f32_e32 v88, s56, v66\n\t" \
  "v_mul_f32_e32 v89, s56, v70\n\t" \
  "v_mul_f32_e32 v90, s56, v74\n\t" \
  "v_mul_f32_e32 v91, s56, v78\n\t" \
  "v_mfma_f32_16x16x128_f8f6f4 a[160:163], v[168:175], v[192:199], a[160:163]\n\t" \
  "s_add_u32 s53, s49, 16384\n\t" \
  "s_add_u32 m0, s48, 49152\n\t" \
  "v_mfma_f32_16x16x128_f8f6f4 a[192:195], v[176:183], v[192:199], a[192:195]\n\t" \
  "buffer_load_dwordx4 v0, s[36:39], s53 offen sc0 sc1 lds\n\t" \
  "v_mfma_f32_16x16x128_f8f6f4 a[196:199], v[176:183], v[200:207], a[196:199]\n\t" \
  "v_mul_f32_e32 v92, s56, v67\n\t" \
  "v_mul_f32_e32 v93, s56, v71\n\t" \
  "v_mul_f32_e32 v94, s56, v75\n\t" \
  "v_mul_f32_e32 v95, s56, v79\n\t" \
  "v_mfma_f32_16x16x128_f8f6f4 a[200:203], v[176:183], v[208:215], a[200:203]\n\t" \
  "buffer_load_dwordx4 v0, s[36:39], s53 offen offset:1024 sc0 sc1 lds\n\t" \
  "v_mfma_f32_16x16x128_f8f6f4 a[204:207], v[176:183], v[216:223], a[204:207]\n\t" \
  "v_min_f32_e32 v24, s60, v80\n\t" \
  "v_min_f32_e32 v25, s60, v81\n\t" \
  "v_min_f32_e32 v26, s60, v82\n\t" \
  "v_min_f32_e32 v27, s60, v83\n\t" \
  "v_mfma_f32_16x16x128_f8f6f4 a[236:239], v[184:191], v[216:223], a[236:239]\n\t" \
  "buffer_load_dwordx4 v0, s[36:39], s53 offen offset:2048 sc0 sc1 lds\n\t" \
  "v_mfma_f32_16x16x128_f8f6f4 a[232:235], v[184:191], v[208:215], a[232:235]\n\t" \
  "v_exp_f32_e32 v24, v24\n\t" \
  "v_exp_f32_e32 v25, v25\n\t" \
  "v_mfma_f32_16x16x128_f8f6f4 a[228:231], v[184:191], v[200:207], a[228:231]\n\t" \
  "buffer_load_dwordx4 v0, s[36:39], s53 offen offset:3072 sc0 sc1 lds\n\t" \
  "v_mfma_f32_16x16x128_f8f6f4 a[224:227], v[184:191], v[192:199], a[224:227]\n\t" \
  "v_exp_f32_e32 v26, v26\n\t" \
  "v_exp_f32_e32 v27, v27\n\t" \
  "s_waitcnt lgkmcnt(0)\n\t" \
  "v_mfma_f32_16x16x128_f8f6f4 a[16:19], v[128:135], v[224:231], a[16:19]\n\t" \
  "s_waitcnt vmcnt(20)\n\t" \
  "s_barrier\n\t" \
  "v_mfma_f32_16x16x128_f8f6f4 a[20:23], v[128:135], v[232:239], a[20:23]\n\t" \
  "ds_read_b128 v[192:195], v4 offset:32768\n\t" \
  "ds_read_b128 v[196:199], v4 offset:40960\n\t" \
  "v_mfma_f32_16x16x128_f8f6f4 a[24:27], v[128:135], v[240:247], a[24:27]\n\t" \
  "ds_read_b128 v[200:203], v4 offset:33792\n\t" \
  "ds_read_b128 v[204:207], v4 offset:41984\n\t" \
  "v_mfma_f32_16x16x128_f8f6f4 a[28:31], v[128:135], v[248:255], a[28:31]\n\t" \
  "ds_read_b128 v[208:211], v4 offset:34816\n\t" \
  "ds_read_b128 v[212:215], v4 offset:43008\n\t" \
  "v_mfma_f32_16x16x128_f8f6f4 a[60:63], v[136:143], v[248:255], a[60:63]\n\t" \
  "ds_read_b128 v[216:219], v4 offset:35840\n\t" \
  "ds_read_b128 v[220:223], v4 offset:44032\n\t" \
  "v_mfma_f32_16x16x128_f8f6f4 a[56:59], v[136:143], v[240:247], a[56:59]\n\t" \
  "v_fma_f32 v24, v24, s58, s58\n\t" \
  "v_fma_f32 v25, v25, s58, s58\n\t" \
  "v_fma_f32 v26, v26, s58, s58\n\t" \
  "v_fma_f32 v27, v27, s58, s58\n\t" \
  "v_mfma_f32_16x16x128_f8f6f4 a[52:55], v[136:143], v[232:239], a[52:55]\n\t" \
  "v_rcp_f32_e32 v24, v24\n\t" \
  "v_rcp_f32_e32 v25, v25\n\t" \
  "v_mfma_f32_16x16x128_f8f6f4 a[48:51], v[136:143], v[224:231], a[48:51]\n\t" \
  "s_add_u32 s53, s49, 32768\n\t" \
  "s_add_u32 m0, s48, 65536\n\t" \
  "v_mfma_f32_16x16x128_f8f6f4 a[80:83], v[144:151], v[224:231], a[80:83]\n\t" \
  "buffer_load_dwordx4 v0, s[36:39], s53 offen sc0 sc1 lds\n\t" \
  "v_mfma_f32_16x16x128_f8f6f4 a[84:87], v[144:151], v[232:239], a[84:87]\n\t" \
  "buffer_load_dword v7, v8, s[44:47], s76 offen sc0 sc1\n\t" \
  "v_mfma_f32_16x16x128_f8f6f4 a[88:91], v[144:151], v[240:247], a[88:91]\n\t" \
  "buffer_load_dwordx4 v0, s[36:39], s53 offen offset:1024 sc0 sc1 lds\n\t" \
  "v_mfma_f32_16x16x128_f8f6f4 a[92:95], v[144:151], v[248:255], a[92:95]\n\t" \
  "v_rcp_f32_e32 v26, v26\n\t" \
  "v_rcp_f32_e32 v27, v27\n\t" \
  "v_mfma_f32_16x16x128_f8f6f4 a[124:127], v[152:159], v[248:255], a[124:127]\n\t" \
  "buffer_load_dwordx4 v0, s[36:39], s53 offen offset:2048 sc0 sc1 lds\n\t" \
  "v_mfma_f32_16x16x128_f8f6f4 a[120:123], v[152:159], v[240:247], a[120:123]\n\t" \
  "v_mul_f32_e32 v80, v80, v24\n\t" \
  "v_mul_f32_e32 v81, v81, v25\n\t" \
  "v_mul_f32_e32 v82, v82, v26\n\t" \
  "v_mul_f32_e32 v83, v83, v27\n\t" \
  "v_mfma_f32_16x16x128_f8f6f4 a[116:119], v[152:159], v[232:239], a[116:119]\n\t" \
  "buffer_load_dwordx4 v0, s[36:39], s53 offen offset:3072 sc0 sc1 lds\n\t" \
  "v_mfma_f32_16x16x128_f8f6f4 a[112:115], v[152:159], v[224:231], a[112:115]\n\t" \
  "s_xor_b32 s91, s68, 7340032\n\t" \
  "buffer_store_dwordx2 v[80:81], v5, s[40:43], s91 offen nt\n\t" \
  "s_waitcnt lgkmcnt(0)\n\t" \
  "v_mfma_f32_16x16x128_f8f6f4 a[0:3], v[128:135], v[192:199], a[0:3]\n\t" \
  "buffer_store_dwordx2 v[82:83], v5, s[40:43], s91 offen offset:128 nt\n\t" \
  "v_mfma_f32_16x16x128_f8f6f4 a[4:7], v[128:135], v[200:207], a[4:7]\n\t" \
  "ds_read_b128 v[160:163], v2 offset:49152\n\t" \
  "ds_read_b128 v[164:167], v2 offset:57344\n\t" \
  "v_mfma_f32_16x16x128_f8f6f4 a[8:11], v[128:135], v[208:215], a[8:11]\n\t" \
  "ds_read_b128 v[168:171], v2 offset:50176\n\t" \
  "ds_read_b128 v[172:175], v2 offset:58368\n\t" \
  "v_mfma_f32_16x16x128_f8f6f4 a[12:15], v[128:135], v[216:223], a[12:15]\n\t" \
  "ds_read_b128 v[176:179], v2 offset:51200\n\t" \
  "ds_read_b128 v[180:183], v2 offset:59392\n\t" \
  "v_mfma_f32_16x16x128_f8f6f4 a[44:47], v[136:143], v[216:223], a[44:47]\n\t" \
  "ds_read_b128 v[184:187], v2 offset:52224\n\t" \
  "ds_read_b128 v[188:191], v2 offset:60416\n\t" \
  "v_mfma_f32_16x16x128_f8f6f4 a[40:43], v[136:143], v[208:215], a[40:43]\n\t" \
  "v_min_f32_e32 v28, s60, v84\n\t" \
  "v_min_f32_e32 v29, s60, v85\n\t" \
  "v_min_f32_e32 v30, s60, v86\n\t" \
  "v_min_f32_e32 v31, s60, v87\n\t" \
  "v_mfma_f32_16x16x128_f8f6f4 a[36:39], v[136:143], v[200:207], a[36:39]\n\t" \
  "v_exp_f32_e32 v28, v28\n\t" \
  "v_exp_f32_e32 v29, v29\n\t" \
  "v_mfma_f32_16x16x128_f8f6f4 a[32:35], v[136:143], v[192:199], a[32:35]\n\t" \
  "s_add_u32 s53, s51, 16384\n\t" \
  "s_add_u32 m0, s48, 81920\n\t" \
  "v_mfma_f32_16x16x128_f8f6f4 a[64:67], v[144:151], v[192:199], a[64:67]\n\t" \
  "buffer_load_dwordx4 v0, s[36:39], s53 offen sc0 sc1 lds\n\t" \
  "v_mfma_f32_16x16x128_f8f6f4 a[68:71], v[144:151], v[200:207], a[68:71]\n\t" \
  "v_exp_f32_e32 v30, v30\n\t" \
  "v_exp_f32_e32 v31, v31\n\t" \
  "v_mfma_f32_16x16x128_f8f6f4 a[72:75], v[144:151], v[208:215], a[72:75]\n\t" \
  "buffer_load_dwordx4 v0, s[36:39], s53 offen offset:1024 sc0 sc1 lds\n\t" \
  "v_mfma_f32_16x16x128_f8f6f4 a[76:79], v[144:151], v[216:223], a[76:79]\n\t" \
  "v_fma_f32 v28, v28, s58, s58\n\t" \
  "v_fma_f32 v29, v29, s58, s58\n\t" \
  "v_fma_f32 v30, v30, s58, s58\n\t" \
  "v_fma_f32 v31, v31, s58, s58\n\t" \
  "v_mfma_f32_16x16x128_f8f6f4 a[108:111], v[152:159], v[216:223], a[108:111]\n\t" \
  "buffer_load_dwordx4 v0, s[36:39], s53 offen offset:2048 sc0 sc1 lds\n\t" \
  "v_mfma_f32_16x16x128_f8f6f4 a[104:107], v[152:159], v[208:215], a[104:107]\n\t" \
  "v_rcp_f32_e32 v28, v28\n\t" \
  "v_rcp_f32_e32 v29, v29\n\t" \
  "v_mfma_f32_16x16x128_f8f6f4 a[100:103], v[152:159], v[200:207], a[100:103]\n\t" \
  "buffer_load_dwordx4 v0, s[36:39], s53 offen offset:3072 sc0 sc1 lds\n\t" \
  "v_mfma_f32_16x16x128_f8f6f4 a[96:99], v[152:159], v[192:199], a[96:99]\n\t" \
  "v_rcp_f32_e32 v30, v30\n\t" \
  "v_rcp_f32_e32 v31, v31\n\t" \
  "s_waitcnt lgkmcnt(0)\n\t" \
  "v_mfma_f32_16x16x128_f8f6f4 a[128:131], v[160:167], v[192:199], a[128:131]\n\t" \
  "s_waitcnt vmcnt(21)\n\t" \
  "s_barrier\n\t" \
  "v_mfma_f32_16x16x128_f8f6f4 a[132:135], v[160:167], v[200:207], a[132:135]\n\t" \
  "ds_read_b128 v[128:131], v1 offset:0\n\t" \
  "ds_read_b128 v[132:135], v1 offset:8192\n\t" \
  "v_mfma_f32_16x16x128_f8f6f4 a[136:139], v[160:167], v[208:215], a[136:139]\n\t" \
  "ds_read_b128 v[136:139], v1 offset:1024\n\t" \
  "ds_read_b128 v[140:143], v1 offset:9216\n\t" \
  "v_mfma_f32_16x16x128_f8f6f4 a[140:143], v[160:167], v[216:223], a[140:143]\n\t" \
  "ds_read_b128 v[144:147], v1 offset:2048\n\t" \
  "ds_read_b128 v[148:151], v1 offset:10240\n\t" \
  "v_mfma_f32_16x16x128_f8f6f4 a[172:175], v[168:175], v[216:223], a[172:175]\n\t" \
  "ds_read_b128 v[152:155], v1 offset:3072\n\t" \
  "ds_read_b128 v[156:159], v1 offset:11264\n\t" \
  "v_mfma_f32_16x16x128_f8f6f4 a[168:171], v[168:175], v[208:215], a[168:171]\n\t" \
  "v_mul_f32_e32 v84, v84, v28\n\t" \
  "v_mul_f32_e32 v85, v85, v29\n\t" \
  "v_mul_f32_e32 v86, v86, v30\n\t" \
  "v_mul_f32_e32 v87, v87, v31\n\t" \
  "v_mfma_f32_16x16x128_f8f6f4 a[164:167], v[168:175], v[200:207], a[164:167]\n\t" \
  "s_xor_b32 s91, s68, 7405568\n\t" \
  "buffer_store_dwordx2 v[84:85], v5, s[40:43], s91 offen nt\n\t" \
  "v_mfma_f32_16x16x128_f8f6f4 a[160:163], v[168:175], v[192:199], a[160:163]\n\t" \
  "s_add_u32 s53, s50, 16384\n\t" \
  "s_add_u32 m0, s48, 98304\n\t" \
  "buffer_store_dwordx2 v[86:87], v5, s[40:43], s91 offen offset:128 nt\n\t" \
  "v_mfma_f32_16x16x128_f8f6f4 a[192:195], v[176:183], v[192:199], a[192:195]\n\t" \
  "buffer_load_dwordx4 v0, s[36:39], s53 offen sc0 sc1 lds\n\t" \
  "v_mfma_f32_16x16x128_f8f6f4 a[196:199], v[176:183], v[200:207], a[196:199]\n\t" \
  "v_min_f32_e32 v12, s60, v88\n\t" \
  "v_min_f32_e32 v13, s60, v89\n\t" \
  "v_min_f32_e32 v14, s60, v90\n\t" \
  "v_min_f32_e32 v15, s60, v91\n\t" \
  "v_mfma_f32_16x16x128_f8f6f4 a[200:203], v[176:183], v[208:215], a[200:203]\n\t" \
  "buffer_load_dwordx4 v0, s[36:39], s53 offen offset:1024 sc0 sc1 lds\n\t" \
  "v_mfma_f32_16x16x128_f8f6f4 a[204:207], v[176:183], v[216:223], a[204:207]\n\t" \
  "v_exp_f32_e32 v12, v12\n\t" \
  "v_exp_f32_e32 v13, v13\n\t" \
  "v_mfma_f32_16x16x128_f8f6f4 a[236:239], v[184:191], v[216:223], a[236:239]\n\t" \
  "buffer_load_dwordx4 v0, s[36:39], s53 offen offset:2048 sc0 sc1 lds\n\t" \
  "v_mfma_f32_16x16x128_f8f6f4 a[232:235], v[184:191], v[208:215], a[232:235]\n\t" \
  "v_exp_f32_e32 v14, v14\n\t" \
  "v_exp_f32_e32 v15, v15\n\t" \
  "v_mfma_f32_16x16x128_f8f6f4 a[228:231], v[184:191], v[200:207], a[228:231]\n\t" \
  "buffer_load_dwordx4 v0, s[36:39], s53 offen offset:3072 sc0 sc1 lds\n\t" \
  "v_mfma_f32_16x16x128_f8f6f4 a[224:227], v[184:191], v[192:199], a[224:227]\n\t" \
  "v_fma_f32 v12, v12, s58, s58\n\t" \
  "v_fma_f32 v13, v13, s58, s58\n\t" \
  "v_fma_f32 v14, v14, s58, s58\n\t" \
  "v_fma_f32 v15, v15, s58, s58\n\t" \
  "s_waitcnt lgkmcnt(0)\n\t" \
  "v_mfma_f32_16x16x128_f8f6f4 a[144:147], v[160:167], v[224:231], a[144:147]\n\t" \
  "v_rcp_f32_e32 v12, v12\n\t" \
  "v_rcp_f32_e32 v13, v13\n\t" \
  "v_mfma_f32_16x16x128_f8f6f4 a[148:151], v[160:167], v[232:239], a[148:151]\n\t" \
  "ds_read_b128 v[192:195], v3 offset:16384\n\t" \
  "ds_read_b128 v[196:199], v3 offset:24576\n\t" \
  "v_mfma_f32_16x16x128_f8f6f4 a[152:155], v[160:167], v[240:247], a[152:155]\n\t" \
  "ds_read_b128 v[200:203], v3 offset:17408\n\t" \
  "ds_read_b128 v[204:207], v3 offset:25600\n\t" \
  "v_mfma_f32_16x16x128_f8f6f4 a[156:159], v[160:167], v[248:255], a[156:159]\n\t" \
  "ds_read_b128 v[208:211], v3 offset:18432\n\t" \
  "ds_read_b128 v[212:215], v3 offset:26624\n\t" \
  "v_mfma_f32_16x16x128_f8f6f4 a[188:191], v[168:175], v[248:255], a[188:191]\n\t" \
  "ds_read_b128 v[216:219], v3 offset:19456\n\t" \
  "ds_read_b128 v[220:223], v3 offset:27648\n\t" \
  "v_mfma_f32_16x16x128_f8f6f4 a[184:187], v[168:175], v[240:247], a[184:187]\n\t" \
  "v_rcp_f32_e32 v14, v14\n\t" \
  "v_rcp_f32_e32 v15, v15\n\t" \
  "v_mfma_f32_16x16x128_f8f6f4 a[180:183], v[168:175], v[232:239], a[180:183]\n\t" \
  "v_mul_f32_e32 v88, v88, v12\n\t" \
  "v_mul_f32_e32 v89, v89, v13\n\t" \
  "v_mul_f32_e32 v90, v90, v14\n\t" \
  "v_mul_f32_e32 v91, v91, v15\n\t" \
  "v_mfma_f32_16x16x128_f8f6f4 a[176:179], v[168:175], v[224:231], a[176:179]\n\t" \
  "s_add_u32 s53, s49, 49152\n\t" \
  "s_add_u32 m0, s48, 114688\n\t" \
  "v_mfma_f32_16x16x128_f8f6f4 a[208:211], v[176:183], v[224:231], a[208:211]\n\t" \
  "buffer_load_dwordx4 v0, s[36:39], s53 offen sc0 sc1 lds\n\t" \
  "v_mfma_f32_16x16x128_f8f6f4 a[212:215], v[176:183], v[232:239], a[212:215]\n\t" \
  "s_and_b32 s54, s62, 0xff\n\t" \
  "s_lshr_b64 s[62:63], s[62:63], 8\n\t" \
  "s_lshl_b32 s55, s64, 24\n\t" \
  "s_or_b32 s63, s63, s55\n\t" \
  "v_mfma_f32_16x16x128_f8f6f4 a[216:219], v[176:183], v[240:247], a[216:219]\n\t" \
  "buffer_load_dwordx4 v0, s[36:39], s53 offen offset:1024 sc0 sc1 lds\n\t" \
  "v_mfma_f32_16x16x128_f8f6f4 a[220:223], v[176:183], v[248:255], a[220:223]\n\t" \
  "s_lshr_b64 s[64:65], s[64:65], 8\n\t" \
  "s_and_b32 s54, s54, 0x7f\n\t" \
  "s_lshl_b32 s69, s54, 9\n\t" \
  "v_mfma_f32_16x16x128_f8f6f4 a[252:255], v[184:191], v[248:255], a[252:255]\n\t" \
  "buffer_load_dwordx4 v0, s[36:39], s53 offen offset:2048 sc0 sc1 lds\n\t" \
  "v_mfma_f32_16x16x128_f8f6f4 a[248:251], v[184:191], v[240:247], a[248:251]\n\t" \
  "s_add_u32 s69, s69, s71\n\t" \
  "s_lshl_b32 s54, s54, 19\n\t" \
  "s_add_u32 s54, s54, s70\n\t" \
  "v_mfma_f32_16x16x128_f8f6f4 a[244:247], v[184:191], v[232:239], a[244:247]\n\t" \
  "buffer_load_dwordx4 v0, s[36:39], s53 offen offset:3072 sc0 sc1 lds\n\t" \
  "v_mfma_f32_16x16x128_f8f6f4 a[240:243], v[184:191], v[224:231], a[240:243]\n\t" \
  "s_add_u32 s49, s49, 65536\n\t" \
  "s_add_u32 s50, s50, 32768\n\t" \
  "s_add_u32 s51, s51, 32768\n\t" \
  "s_waitcnt lgkmcnt(0)\n\t" \
  "v_mfma_f32_16x16x128_f8f6f4 a[0:3], v[128:135], v[192:199], a[0:3]\n\t" \
  "s_waitcnt vmcnt(21)\n\t" \
  "s_barrier\n\t" \
  "v_mfma_f32_16x16x128_f8f6f4 a[4:7], v[128:135], v[200:207], a[4:7]\n\t" \
  "ds_read_b128 v[224:227], v3 offset:32768\n\t" \
  "ds_read_b128 v[228:231], v3 offset:40960\n\t" \
  "s_mov_b32 s49, s52\n\t" \
  "s_sub_u32 s51, s51, 524288\n\t" \
  "s_add_u32 s55, s51, 16384\n\t" \
  "v_mfma_f32_16x16x128_f8f6f4 a[8:11], v[128:135], v[208:215], a[8:11]\n\t" \
  "ds_read_b128 v[232:235], v3 offset:33792\n\t" \
  "ds_read_b128 v[236:239], v3 offset:41984\n\t" \
  "s_cmp_eq_u32 s72, 1\n\t" \
  "s_cselect_b32 s50, s51, s54\n\t" \
  "s_cselect_b32 s51, s55, s51\n\t" \
  "v_mfma_f32_16x16x128_f8f6f4 a[12:15], v[128:135], v[216:223], a[12:15]\n\t" \
  "ds_read_b128 v[240:243], v3 offset:34816\n\t" \
  "ds_read_b128 v[244:247], v3 offset:43008\n\t" \
  "v_mfma_f32_16x16x128_f8f6f4 a[44:47], v[136:143], v[216:223], a[44:47]\n\t" \
  "ds_read_b128 v[248:251], v3 offset:35840\n\t" \
  "ds_read_b128 v[252:255], v3 offset:44032\n\t" \
  "v_mfma_f32_16x16x128_f8f6f4 a[40:43], v[136:143], v[208:215], a[40:43]\n\t" \
  "s_waitcnt vmcnt(19)\n\t" \
  "v_cmp_le_u32_e32 vcc, s75, v7\n\t" \
  "v_mfma_f32_16x16x128_f8f6f4 a[36:39], v[136:143], v[200:207], a[36:39]\n\t" \
  "s_cmp_eq_u64 vcc, -1\n\t" \
  "s_cbranch_scc0 L_SLOW17_%=\n\t" \
  "L_OK17_%=:\n\t" \
  "s_add_u32 s76, s76, 1024\n\t" \
  "v_mfma_f32_16x16x128_f8f6f4 a[32:35], v[136:143], v[192:199], a[32:35]\n\t" \
  "s_mov_b32 s53, s49\n\t" \
  "s_mov_b32 m0, s48\n\t" \
  "v_mfma_f32_16x16x128_f8f6f4 a[64:67], v[144:151], v[192:199], a[64:67]\n\t" \
  "buffer_load_dwordx4 v0, s[36:39], s53 offen sc0 sc1 lds\n\t" \
  "v_mfma_f32_16x16x128_f8f6f4 a[68:71], v[144:151], v[200:207], a[68:71]\n\t" \
  "s_xor_b32 s91, s68, 7471104\n\t" \
  "buffer_store_dwordx2 v[88:89], v5, s[40:43], s91 offen nt\n\t" \
  "v_mfma_f32_16x16x128_f8f6f4 a[72:75], v[144:151], v[208:215], a[72:75]\n\t" \
  "buffer_load_dwordx4 v0, s[36:39], s53 offen offset:1024 sc0 sc1 lds\n\t" \
  "v_mfma_f32_16x16x128_f8f6f4 a[76:79], v[144:151], v[216:223], a[76:79]\n\t" \
  "buffer_store_dwordx2 v[90:91], v5, s[40:43], s91 offen offset:128 nt\n\t" \
  "v_mfma_f32_16x16x128_f8f6f4 a[108:111], v[152:159], v[216:223], a[108:111]\n\t" \
  "buffer_load_dwordx4 v0, s[36:39], s53 offen offset:2048 sc0 sc1 lds\n\t" \
  "v_mfma_f32_16x16x128_f8f6f4 a[104:107], v[152:159], v[208:215], a[104:107]\n\t" \
  "v_min_f32_e32 v16, s60, v92\n\t" \
  "v_min_f32_e32 v17, s60, v93\n\t" \
  "v_min_f32_e32 v18, s60, v94\n\t" \
  "v_min_f32_e32 v19, s60, v95\n\t" \
  "v_mfma_f32_16x16x128_f8f6f4 a[100:103], v[152:159], v[200:207], a[100:103]\n\t" \
  "buffer_load_dwordx4 v0, s[36:39], s53 offen offset:3072 sc0 sc1 lds\n\t" \
  "v_mfma_f32_16x16x128_f8f6f4 a[96:99], v[152:159], v[192:199], a[96:99]\n\t" \
  "s_waitcnt lgkmcnt(0)\n\t" \
  "v_mfma_f32_16x16x128_f8f6f4 a[16:19], v[128:135], v[224:231], a[16:19]\n\t" \
  "v_exp_f32_e32 v16, v16\n\t" \
  "v_exp_f32_e32 v17, v17\n\t" \
  "v_mfma_f32_16x16x128_f8f6f4 a[20:23], v[128:135], v[232:239], a[20:23]\n\t" \
  "ds_read_b128 v[160:163], v1 offset:49152\n\t" \
  "ds_read_b128 v[164:167], v1 offset:57344\n\t" \
  "v_mfma_f32_16x16x128_f8f6f4 a[24:27], v[128:135], v[240:247], a[24:27]\n\t" \
  "ds_read_b128 v[168:171], v1 offset:50176\n\t" \
  "ds_read_b128 v[172:175], v1 offset:58368\n\t" \
  "v_mfma_f32_16x16x128_f8f6f4 a[28:31], v[128:135], v[248:255], a[28:31]\n\t" \
  "ds_read_b128 v[176:179], v1 offset:51200\n\t" \
  "ds_read_b128 v[180:183], v1 offset:59392\n\t" \
  "v_mfma_f32_16x16x128_f8f6f4 a[60:63], v[136:143], v[248:255], a[60:63]\n\t" \
  "ds_read_b128 v[184:187], v1 offset:52224\n\t" \
  "ds_read_b128 v[188:191], v1 offset:60416\n\t" \
  "v_mfma_f32_16x16x128_f8f6f4 a[56:59], v[136:143], v[240:247], a[56:59]\n\t" \
  "v_exp_f32_e32 v18, v18\n\t" \
  "v_exp_f32_e32 v19, v19\n\t" \
  "v_mfma_f32_16x16x128_f8f6f4 a[52:55], v[136:143], v[232:239], a[52:55]\n\t" \
  "v_fma_f32 v16, v16, s58, s58\n\t" \
  "v_fma_f32 v17, v17, s58, s58\n\t" \
  "v_fma_f32 v18, v18, s58, s58\n\t" \
  "v_fma_f32 v19, v19, s58, s58\n\t" \
  "v_mfma_f32_16x16x128_f8f6f4 a[48:51], v[136:143], v[224:231], a[48:51]\n\t" \
  "s_mov_b32 s53, s50\n\t" \
  "s_add_u32 m0, s48, 16384\n\t" \
  "v_mfma_f32_16x16x128_f8f6f4 a[80:83], v[144:151], v[224:231], a[80:83]\n\t" \
  "buffer_load_dwordx4 v0, s[36:39], s53 offen sc0 sc1 lds\n\t" \
  "v_mfma_f32_16x16x128_f8f6f4 a[84:87], v[144:151], v[232:239], a[84:87]\n\t" \
  "v_rcp_f32_e32 v16, v16\n\t" \
  "v_rcp_f32_e32 v17, v17\n\t" \
  "v_mfma_f32_16x16x128_f8f6f4 a[88:91], v[144:151], v[240:247], a[88:91]\n\t" \
  "buffer_load_dwordx4 v0, s[36:39], s53 offen offset:1024 sc0 sc1 lds\n\t" \
  "v_mfma_f32_16x16x128_f8f6f4 a[92:95], v[144:151], v[248:255], a[92:95]\n\t" \
  "v_rcp_f32_e32 v18, v18\n\t" \
  "v_rcp_f32_e32 v19, v19\n\t" \
  "v_mfma_f32_16x16x128_f8f6f4 a[124:127], v[152:159], v[248:255], a[124:127]\n\t" \
  "buffer_load_dwordx4 v0, s[36:39], s53 offen offset:2048 sc0 sc1 lds\n\t" \
  "v_mfma_f32_16x16x128_f8f6f4 a[120:123], v[152:159], v[240:247], a[120:123]\n\t" \
  "v_mul_f32_e32 v92, v92, v16\n\t" \
  "v_mul_f32_e32 v93, v93, v17\n\t" \
  "v_mul_f32_e32 v94, v94, v18\n\t" \
  "v_mul_f32_e32 v95, v95, v19\n\t" \
  "v_mfma_f32_16x16x128_f8f6f4 a[116:119], v[152:159], v[232:239], a[116:119]\n\t" \
  "buffer_load_dwordx4 v0, s[36:39], s53 offen offset:3072 sc0 sc1 lds\n\t" \
  "v_mfma_f32_16x16x128_f8f6f4 a[112:115], v[152:159], v[224:231], a[112:115]\n\t" \
  "s_xor_b32 s91, s68, 7536640\n\t" \
  "buffer_store_dwordx2 v[92:93], v5, s[40:43], s91 offen nt\n\t" \
  "s_waitcnt lgkmcnt(0)\n\t" \
  "v_mfma_f32_16x16x128_f8f6f4 a[144:147], v[160:167], v[224:231], a[144:147]\n\t" \
  "s_waitcnt vmcnt(21)\n\t" \
  "s_barrier\n\t" \
  "v_mfma_f32_16x16x128_f8f6f4 a[148:151], v[160:167], v[232:239], a[148:151]\n\t" \
  "ds_read_b128 v[128:131], v2 offset:0\n\t" \
  "ds_read_b128 v[132:135], v2 offset:8192\n\t" \
  "v_mfma_f32_16x16x128_f8f6f4 a[152:155], v[160:167], v[240:247], a[152:155]\n\t" \
  "ds_read_b128 v[136:139], v2 offset:1024\n\t" \
  "ds_read_b128 v[140:143], v2 offset:9216\n\t" \
  "v_mfma_f32_16x16x128_f8f6f4 a[156:159], v[160:167], v[248:255], a[156:159]\n\t" \
  "ds_read_b128 v[144:147], v2 offset:2048\n\t" \
  "ds_read_b128 v[148:151], v2 offset:10240\n\t" \
  "v_mfma_f32_16x16x128_f8f6f4 a[188:191], v[168:175], v[248:255], a[188:191]\n\t" \
  "ds_read_b128 v[152:155], v2 offset:3072\n\t" \
  "ds_read_b128 v[156:159], v2 offset:11264\n\t" \
  "v_mfma_f32_16x16x128_f8f6f4 a[184:187], v[168:175], v[240:247], a[184:187]\n\t" \
  "buffer_store_dwordx2 v[94:95], v5, s[40:43], s91 offen offset:128 nt\n\t" \
  "v_mfma_f32_16x16x128_f8f6f4 a[180:183], v[168:175], v[232:239], a[180:183]\n\t" \
  "v_mfma_f32_16x16x128_f8f6f4 a[176:179], v[168:175], v[224:231], a[176:179]\n\t" \
  "s_mov_b32 s53, s51\n\t" \
  "s_add_u32 m0, s48, 32768\n\t" \
  "v_mfma_f32_16x16x128_f8f6f4 a[208:211], v[176:183], v[224:231], a[208:211]\n\t" \
  "buffer_load_dwordx4 v0, s[36:39], s53 offen sc0 sc1 lds\n\t" \
  "v_mfma_f32_16x16x128_f8f6f4 a[212:215], v[176:183], v[232:239], a[212:215]\n\t" \
  "v_mfma_f32_16x16x128_f8f6f4 a[216:219], v[176:183], v[240:247], a[216:219]\n\t" \
  "buffer_load_dwordx4 v0, s[36:39], s53 offen offset:1024 sc0 sc1 lds\n\t" \
  "v_mfma_f32_16x16x128_f8f6f4 a[220:223], v[176:183], v[248:255], a[220:223]\n\t" \
  "v_mfma_f32_16x16x128_f8f6f4 a[252:255], v[184:191], v[248:255], a[252:255]\n\t" \
  "buffer_load_dwordx4 v0, s[36:39], s53 offen offset:2048 sc0 sc1 lds\n\t" \
  "v_mfma_f32_16x16x128_f8f6f4 a[248:251], v[184:191], v[240:247], a[248:251]\n\t" \
  "v_mfma_f32_16x16x128_f8f6f4 a[244:247], v[184:191], v[232:239], a[244:247]\n\t" \
  "buffer_load_dwordx4 v0, s[36:39], s53 offen offset:3072 sc0 sc1 lds\n\t" \
  "v_mfma_f32_16x16x128_f8f6f4 a[240:243], v[184:191], v[224:231], a[240:243]\n\t" \
  "s_waitcnt lgkmcnt(0)\n\t" \
  "v_mfma_f32_16x16x128_f8f6f4 a[128:131], v[160:167], v[192:199], a[128:131]\n\t" \
  "v_mfma_f32_16x16x128_f8f6f4 a[132:135], v[160:167], v[200:207], a[132:135]\n\t" \
  "ds_read_b128 v[224:227], v4 offset:16384\n\t" \
  "ds_read_b128 v[228:231], v4 offset:24576\n\t" \
  "v_mfma_f32_16x16x128_f8f6f4 a[136:139], v[160:167], v[208:215], a[136:139]\n\t" \
  "ds_read_b128 v[232:235], v4 offset:17408\n\t" \
  "ds_read_b128 v[236:239], v4 offset:25600\n\t" \
  "v_mfma_f32_16x16x128_f8f6f4 a[140:143], v[160:167], v[216:223], a[140:143]\n\t" \
  "ds_read_b128 v[240:243], v4 offset:18432\n\t" \
  "ds_read_b128 v[244:247], v4 offset:26624\n\t" \
  "v_mfma_f32_16x16x128_f8f6f4 a[172:175], v[168:175], v[216:223], a[172:175]\n\t" \
  "ds_read_b128 v[248:251], v4 offset:19456\n\t" \
  "ds_read_b128 v[252:255], v4 offset:27648\n\t" \
  "v_mfma_f32_16x16x128_f8f6f4 a[168:171], v[168:175], v[208:215], a[168:171]\n\t" \
  "v_mfma_f32_16x16x128_f8f6f4 a[164:167], v[168:175], v[200:207], a[164:167]\n\t" \
  "v_mfma_f32_16x16x128_f8f6f4 a[160:163], v[168:175], v[192:199], a[160:163]\n\t" \
  "s_add_u32 s53, s49, 16384\n\t" \
  "s_add_u32 m0, s48, 49152\n\t" \
  "v_mfma_f32_16x16x128_f8f6f4 a[192:195], v[176:183], v[192:199], a[192:195]\n\t" \
  "buffer_load_dwordx4 v0, s[36:39], s53 offen sc0 sc1 lds\n\t" \
  "v_mfma_f32_16x16x128_f8f6f4 a[196:199], v[176:183], v[200:207], a[196:199]\n\t" \
  "v_mfma_f32_16x16x128_f8f6f4 a[200:203], v[176:183], v[208:215], a[200:203]\n\t" \
  "buffer_load_dwordx4 v0, s[36:39], s53 offen offset:1024 sc0 sc1 lds\n\t" \
  "v_mfma_f32_16x16x128_f8f6f4 a[204:207], v[176:183], v[216:223], a[204:207]\n\t" \
  "v_mfma_f32_16x16x128_f8f6f4 a[236:239], v[184:191], v[216:223], a[236:239]\n\t" \
  "buffer_load_dwordx4 v0, s[36:39], s53 offen offset:2048 sc0 sc1 lds\n\t" \
  "v_mfma_f32_16x16x128_f8f6f4 a[232:235], v[184:191], v[208:215], a[232:235]\n\t" \
  "v_mfma_f32_16x16x128_f8f6f4 a[228:231], v[184:191], v[200:207], a[228:231]\n\t" \
  "buffer_load_dwordx4 v0, s[36:39], s53 offen offset:3072 sc0 sc1 lds\n\t" \
  "v_mfma_f32_16x16x128_f8f6f4 a[224:227], v[184:191], v[192:199], a[224:227]\n\t" \
  "s_waitcnt lgkmcnt(0)\n\t" \
  "v_mfma_f32_16x16x128_f8f6f4 a[16:19], v[128:135], v[224:231], a[16:19]\n\t" \
  "s_waitcnt vmcnt(20)\n\t" \
  "s_barrier\n\t" \
  "v_mfma_f32_16x16x128_f8f6f4 a[20:23], v[128:135], v[232:239], a[20:23]\n\t" \
  "ds_read_b128 v[192:195], v4 offset:32768\n\t" \
  "ds_read_b128 v[196:199], v4 offset:40960\n\t" \
  "s_lshl_b32 s55, s61, 5\n\t" \
  "s_add_u32 s50, s50, s55\n\t" \
  "v_mfma_f32_16x16x128_f8f6f4 a[24:27], v[128:135], v[240:247], a[24:27]\n\t" \
  "ds_read_b128 v[200:203], v4 offset:33792\n\t" \
  "ds_read_b128 v[204:207], v4 offset:41984\n\t" \
  "v_mfma_f32_16x16x128_f8f6f4 a[28:31], v[128:135], v[248:255], a[28:31]\n\t" \
  "ds_read_b128 v[208:211], v4 offset:34816\n\t" \
  "ds_read_b128 v[212:215], v4 offset:43008\n\t" \
  "v_mfma_f32_16x16x128_f8f6f4 a[60:63], v[136:143], v[248:255], a[60:63]\n\t" \
  "ds_read_b128 v[216:219], v4 offset:35840\n\t" \
  "ds_read_b128 v[220:223], v4 offset:44032\n\t" \
  "v_mfma_f32_16x16x128_f8f6f4 a[56:59], v[136:143], v[240:247], a[56:59]\n\t" \
  "v_mfma_f32_16x16x128_f8f6f4 a[52:55], v[136:143], v[232:239], a[52:55]\n\t" \
  "v_mfma_f32_16x16x128_f8f6f4 a[48:51], v[136:143], v[224:231], a[48:51]\n\t" \
  "s_add_u32 s53, s49, 32768\n\t" \
  "s_add_u32 m0, s48, 65536\n\t" \
  "v_mfma_f32_16x16x128_f8f6f4 a[80:83], v[144:151], v[224:231], a[80:83]\n\t" \
  "buffer_load_dwordx4 v0, s[36:39], s53 offen sc0 sc1 lds\n\t" \
  "v_mfma_f32_16x16x128_f8f6f4 a[84:87], v[144:151], v[232:239], a[84:87]\n\t" \
  "v_mfma_f32_16x16x128_f8f6f4 a[88:91], v[144:151], v[240:247], a[88:91]\n\t" \
  "buffer_load_dwordx4 v0, s[36:39], s53 offen offset:1024 sc0 sc1 lds\n\t" \
  "v_mfma_f32_16x16x128_f8f6f4 a[92:95], v[144:151], v[248:255], a[92:95]\n\t" \
  "v_mfma_f32_16x16x128_f8f6f4 a[124:127], v[152:159], v[248:255], a[124:127]\n\t" \
  "buffer_load_dwordx4 v0, s[36:39], s53 offen offset:2048 sc0 sc1 lds\n\t" \
  "v_mfma_f32_16x16x128_f8f6f4 a[120:123], v[152:159], v[240:247], a[120:123]\n\t" \
  "v_mfma_f32_16x16x128_f8f6f4 a[116:119], v[152:159], v[232:239], a[116:119]\n\t" \
  "buffer_load_dwordx4 v0, s[36:39], s53 offen offset:3072 sc0 sc1 lds\n\t" \
  "v_mfma_f32_16x16x128_f8f6f4 a[112:115], v[152:159], v[224:231], a[112:115]\n\t" \
  "s_waitcnt lgkmcnt(0)\n\t" \
  "v_mfma_f32_16x16x128_f8f6f4 a[0:3], v[128:135], v[192:199], a[0:3]\n\t" \
  "v_mfma_f32_16x16x128_f8f6f4 a[4:7], v[128:135], v[200:207], a[4:7]\n\t" \
  "ds_read_b128 v[160:163], v2 offset:49152\n\t" \
  "ds_read_b128 v[164:167], v2 offset:57344\n\t" \
  "v_mfma_f32_16x16x128_f8f6f4 a[8:11], v[128:135], v[208:215], a[8:11]\n\t" \
  "ds_read_b128 v[168:171], v2 offset:50176\n\t" \
  "ds_read_b128 v[172:175], v2 offset:58368\n\t" \
  "v_mfma_f32_16x16x128_f8f6f4 a[12:15], v[128:135], v[216:223], a[12:15]\n\t" \
  "ds_read_b128 v[176:179], v2 offset:51200\n\t" \
  "ds_read_b128 v[180:183], v2 offset:59392\n\t" \
  "v_mfma_f32_16x16x128_f8f6f4 a[44:47], v[136:143], v[216:223], a[44:47]\n\t" \
  "ds_read_b128 v[184:187], v2 offset:52224\n\t" \
  "ds_read_b128 v[188:191], v2 offset:60416\n\t" \
  "v_mfma_f32_16x16x128_f8f6f4 a[40:43], v[136:143], v[208:215], a[40:43]\n\t" \
  "v_mfma_f32_16x16x128_f8f6f4 a[36:39], v[136:143], v[200:207], a[36:39]\n\t" \
  "v_mfma_f32_16x16x128_f8f6f4 a[32:35], v[136:143], v[192:199], a[32:35]\n\t" \
  "s_add_u32 s53, s51, 16384\n\t" \
  "s_add_u32 m0, s48, 81920\n\t" \
  "v_mfma_f32_16x16x128_f8f6f4 a[64:67], v[144:151], v[192:199], a[64:67]\n\t" \
  "buffer_load_dwordx4 v0, s[36:39], s53 offen sc0 sc1 lds\n\t" \
  "v_mfma_f32_16x16x128_f8f6f4 a[68:71], v[144:151], v[200:207], a[68:71]\n\t" \
  "v_mfma_f32_16x16x128_f8f6f4 a[72:75], v[144:151], v[208:215], a[72:75]\n\t" \
  "buffer_load_dwordx4 v0, s[36:39], s53 offen offset:1024 sc0 sc1 lds\n\t" \
  "v_mfma_f32_16x16x128_f8f6f4 a[76:79], v[144:151], v[216:223], a[76:79]\n\t" \
  "v_mfma_f32_16x16x128_f8f6f4 a[108:111], v[152:159], v[216:223], a[108:111]\n\t" \
  "buffer_load_dwordx4 v0, s[36:39], s53 offen offset:2048 sc0 sc1 lds\n\t" \
  "v_mfma_f32_16x16x128_f8f6f4 a[104:107], v[152:159], v[208:215], a[104:107]\n\t" \
  "v_mfma_f32_16x16x128_f8f6f4 a[100:103], v[152:159], v[200:207], a[100:103]\n\t" \
  "buffer_load_dwordx4 v0, s[36:39], s53 offen offset:3072 sc0 sc1 lds\n\t" \
  "v_mfma_f32_16x16x128_f8f6f4 a[96:99], v[152:159], v[192:199], a[96:99]\n\t" \
  "s_waitcnt lgkmcnt(0)\n\t" \
  "v_mfma_f32_16x16x128_f8f6f4 a[128:131], v[160:167], v[192:199], a[128:131]\n\t" \
  "s_waitcnt vmcnt(18)\n\t" \
  "s_barrier\n\t" \
  "v_mfma_f32_16x16x128_f8f6f4 a[132:135], v[160:167], v[200:207], a[132:135]\n\t" \
  "ds_read_b128 v[128:131], v1 offset:0\n\t" \
  "ds_read_b128 v[132:135], v1 offset:8192\n\t" \
  "v_accvgpr_read_b32 v32, a0\n\t" \
  "v_accvgpr_read_b32 v36, a1\n\t" \
  "v_accvgpr_read_b32 v40, a2\n\t" \
  "v_mfma_f32_16x16x128_f8f6f4 a[136:139], v[160:167], v[208:215], a[136:139]\n\t" \
  "ds_read_b128 v[136:139], v1 offset:1024\n\t" \
  "ds_read_b128 v[140:143], v1 offset:9216\n\t" \
  "v_accvgpr_read_b32 v44, a3\n\t" \
  "v_accvgpr_read_b32 v33, a4\n\t" \
  "v_accvgpr_read_b32 v37, a5\n\t" \
  "v_mfma_f32_16x16x128_f8f6f4 a[140:143], v[160:167], v[216:223], a[140:143]\n\t" \
  "ds_read_b128 v[144:147], v1 offset:2048\n\t" \
  "ds_read_b128 v[148:151], v1 offset:10240\n\t" \
  "v_accvgpr_read_b32 v41, a6\n\t" \
  "v_accvgpr_read_b32 v45, a7\n\t" \
  "v_accvgpr_read_b32 v34, a8\n\t" \
  "v_mfma_f32_16x16x128_f8f6f4 a[172:175], v[168:175], v[216:223], a[172:175]\n\t" \
  "ds_read_b128 v[152:155], v1 offset:3072\n\t" \
  "ds_read_b128 v[156:159], v1 offset:11264\n\t" \
  "v_accvgpr_read_b32 v38, a9\n\t" \
  "v_accvgpr_read_b32 v42, a10\n\t" \
  "v_accvgpr_read_b32 v46, a11\n\t" \
  "v_mfma_f32_16x16x128_f8f6f4 a[168:171], v[168:175], v[208:215], a[168:171]\n\t" \
  "v_accvgpr_read_b32 v35, a12\n\t" \
  "v_accvgpr_read_b32 v39, a13\n\t" \
  "v_accvgpr_read_b32 v43, a14\n\t" \
  "v_accvgpr_read_b32 v47, a15\n\t" \
  "v_mfma_f32_16x16x128_f8f6f4 a[164:167], v[168:175], v[200:207], a[164:167]\n\t" \
  "v_accvgpr_read_b32 v48, a32\n\t" \
  "v_accvgpr_read_b32 v52, a33\n\t" \
  "v_accvgpr_read_b32 v56, a34\n\t" \
  "v_accvgpr_read_b32 v60, a35\n\t" \
  "v_mfma_f32_16x16x128_f8f6f4 a[160:163], v[168:175], v[192:199], a[160:163]\n\t" \
  "s_add_u32 s53, s50, 16384\n\t" \
  "s_add_u32 m0, s48, 98304\n\t" \
  "v_accvgpr_read_b32 v49, a36\n\t" \
  "v_accvgpr_read_b32 v53, a37\n\t" \
  "v_accvgpr_read_b32 v57, a38\n\t" \
  "v_mfma_f32_16x16x128_f8f6f4 a[192:195], v[176:183], v[192:199], a[192:195]\n\t" \
  "buffer_load_dwordx4 v0, s[36:39], s53 offen sc0 sc1 lds\n\t" \
  "v_accvgpr_read_b32 v61, a39\n\t" \
  "v_accvgpr_read_b32 v50, a40\n\t" \
  "v_accvgpr_read_b32 v54, a41\n\t" \
  "v_mfma_f32_16x16x128_f8f6f4 a[196:199], v[176:183], v[200:207], a[196:199]\n\t" \
  "v_accvgpr_read_b32 v58, a42\n\t" \
  "v_accvgpr_read_b32 v62, a43\n\t" \
  "v_accvgpr_read_b32 v51, a44\n\t" \
  "v_accvgpr_read_b32 v55, a45\n\t" \
  "v_mfma_f32_16x16x128_f8f6f4 a[200:203], v[176:183], v[208:215], a[200:203]\n\t" \
  "buffer_load_dwordx4 v0, s[36:39], s53 offen offset:1024 sc0 sc1 lds\n\t" \
  "v_accvgpr_read_b32 v59, a46\n\t" \
  "v_accvgpr_read_b32 v63, a47\n\t" \
  "v_accvgpr_read_b32 v64, a64\n\t" \
  "v_mfma_f32_16x16x128_f8f6f4 a[204:207], v[176:183], v[216:223], a[204:207]\n\t" \
  "v_accvgpr_read_b32 v68, a65\n\t" \
  "v_accvgpr_read_b32 v72, a66\n\t" \
  "v_accvgpr_read_b32 v76, a67\n\t" \
  "v_accvgpr_read_b32 v65, a68\n\t" \
  "v_mfma_f32_16x16x128_f8f6f4 a[236:239], v[184:191], v[216:223], a[236:239]\n\t" \
  "buffer_load_dwordx4 v0, s[36:39], s53 offen offset:2048 sc0 sc1 lds\n\t" \
  "v_accvgpr_read_b32 v69, a69\n\t" \
  "v_accvgpr_read_b32 v73, a70\n\t" \
  "v_accvgpr_read_b32 v77, a71\n\t" \
  "v_mfma_f32_16x16x128_f8f6f4 a[232:235], v[184:191], v[208:215], a[232:235]\n\t" \
  "v_accvgpr_read_b32 v66, a72\n\t" \
  "v_accvgpr_read_b32 v70, a73\n\t" \
  "v_accvgpr_read_b32 v74, a74\n\t" \
  "v_accvgpr_read_b32 v78, a75\n\t" \
  "v_mfma_f32_16x16x128_f8f6f4 a[228:231], v[184:191], v[200:207], a[228:231]\n\t" \
  "buffer_load_dwordx4 v0, s[36:39], s53 offen offset:3072 sc0 sc1 lds\n\t" \
  "v_accvgpr_read_b32 v67, a76\n\t" \
  "v_accvgpr_read_b32 v71, a77\n\t" \
  "v_accvgpr_read_b32 v75, a78\n\t" \
  "v_mfma_f32_16x16x128_f8f6f4 a[224:227], v[184:191], v[192:199], a[224:227]\n\t" \
  "v_accvgpr_read_b32 v79, a79\n\t" \
  "v_accvgpr_read_b32 v80, a96\n\t" \
  "v_accvgpr_read_b32 v84, a97\n\t" \
  "v_accvgpr_read_b32 v88, a98\n\t" \
  "s_waitcnt lgkmcnt(0)\n\t" \
  "v_mfma_f32_16x16x128_f8f6f4 a[144:147], v[160:167], v[224:231], a[144:147]\n\t" \
  "v_accvgpr_read_b32 v92, a99\n\t" \
  "v_accvgpr_read_b32 v81, a100\n\t" \
  "v_accvgpr_read_b32 v85, a101\n\t" \
  "v_accvgpr_read_b32 v89, a102\n\t" \
  "v_mfma_f32_16x16x128_f8f6f4 a[148:151], v[160:167], v[232:239], a[148:151]\n\t" \
  "ds_read_b128 v[192:195], v3 offset:16384\n\t" \
  "ds_read_b128 v[196:199], v3 offset:24576\n\t" \
  "v_accvgpr_read_b32 v93, a103\n\t" \
  "v_accvgpr_read_b32 v82, a104\n\t" \
  "v_accvgpr_read_b32 v86, a105\n\t" \
  "v_mfma_f32_16x16x128_f8f6f4 a[152:155], v[160:167], v[240:247], a[152:155]\n\t" \
  "ds_read_b128 v[200:203], v3 offset:17408\n\t" \
  "ds_read_b128 v[204:207], v3 offset:25600\n\t" \
  "v_accvgpr_read_b32 v90, a106\n\t" \
  "v_accvgpr_read_b32 v94, a107\n\t" \
  "v_accvgpr_read_b32 v83, a108\n\t" \
  "v_mfma_f32_16x16x128_f8f6f4 a[156:159], v[160:167], v[248:255], a[156:159]\n\t" \
  "ds_read_b128 v[208:211], v3 offset:18432\n\t" \
  "ds_read_b128 v[212:215], v3 offset:26624\n\t" \
  "v_accvgpr_read_b32 v87, a109\n\t" \
  "v_accvgpr_read_b32 v91, a110\n\t" \
  "v_accvgpr_read_b32 v95, a111\n\t" \
  "v_mfma_f32_16x16x128_f8f6f4 a[188:191], v[168:175], v[248:255], a[188:191]\n\t" \
  "ds_read_b128 v[216:219], v3 offset:19456\n\t" \
  "ds_read_b128 v[220:223], v3 offset:27648\n\t" \
  "v_accvgpr_read_b32 v96, a128\n\t" \
  "v_accvgpr_read_b32 v100, a129\n\t" \
  "v_accvgpr_read_b32 v104, a130\n\t" \
  "v_mfma_f32_16x16x128_f8f6f4 a[184:187], v[168:175], v[240:247], a[184:187]\n\t" \
  "v_accvgpr_read_b32 v108, a131\n\t" \
  "v_accvgpr_read_b32 v97, a132\n\t" \
  "v_accvgpr_read_b32 v101, a133\n\t" \
  "v_accvgpr_read_b32 v105, a134\n\t" \
  "v_mfma_f32_16x16x128_f8f6f4 a[180:183], v[168:175], v[232:239], a[180:183]\n\t" \
  "v_accvgpr_read_b32 v109, a135\n\t" \
  "v_accvgpr_read_b32 v98, a136\n\t" \
  "v_accvgpr_read_b32 v102, a137\n\t" \
  "v_accvgpr_read_b32 v106, a138\n\t" \
  "v_mfma_f32_16x16x128_f8f6f4 a[176:179], v[168:175], v[224:231], a[176:179]\n\t" \
  "s_add_u32 s53, s49, 49152\n\t" \
  "s_add_u32 m0, s48, 114688\n\t" \
  "v_accvgpr_read_b32 v110, a139\n\t" \
  "v_accvgpr_read_b32 v99, a140\n\t" \
  "v_accvgpr_read_b32 v103, a141\n\t" \
  "v_mfma_f32_16x16x128_f8f6f4 a[208:211], v[176:183], v[224:231], a[208:211]\n\t" \
  "buffer_load_dwordx4 v0, s[36:39], s53 offen sc0 sc1 lds\n\t" \
  "v_accvgpr_read_b32 v107, a142\n\t" \
  "v_accvgpr_read_b32 v111, a143\n\t" \
  "v_mfma_f32_16x16x128_f8f6f4 a[212:215], v[176:183], v[232:239], a[212:215]\n\t" \
  "v_mfma_f32_16x16x128_f8f6f4 a[216:219], v[176:183], v[240:247], a[216:219]\n\t" \
  "buffer_load_dwordx4 v0, s[36:39], s53 offen offset:1024 sc0 sc1 lds\n\t" \
  "v_mfma_f32_16x16x128_f8f6f4 a[220:223], v[176:183], v[248:255], a[220:223]\n\t" \
  "v_mfma_f32_16x16x128_f8f6f4 a[252:255], v[184:191], v[248:255], a[252:255]\n\t" \
  "buffer_load_dwordx4 v0, s[36:39], s53 offen offset:2048 sc0 sc1 lds\n\t" \
  "v_mfma_f32_16x16x128_f8f6f4 a[248:251], v[184:191], v[240:247], a[248:251]\n\t" \
  "v_mfma_f32_16x16x128_f8f6f4 a[244:247], v[184:191], v[232:239], a[244:247]\n\t" \
  "buffer_load_dwordx4 v0, s[36:39], s53 offen offset:3072 sc0 sc1 lds\n\t" \
  "v_mfma_f32_16x16x128_f8f6f4 a[240:243], v[184:191], v[224:231], a[240:243]\n\t" \
  "s_add_u32 s49, s49, 65536\n\t" \
  "s_add_u32 s50, s50, 32768\n\t" \
  "s_add_u32 s51, s51, 32768\n\t" \
  "s_sub_u32 s72, s72, 1\n\t" \
  "s_cmp_eq_u32 s72, 0\n\t" \
  "s_cbranch_scc0 L_WE_%=\n\t" \
  "s_mov_b32 s68, s66\n\t" \
  "s_mov_b32 s42, s74\n\t" \
  "v_and_b32_e32 v8, 0x3ff, v8\n\t" \
  "v_add_u32_e32 v8, s92, v8\n\t" \
  "s_add_u32 s76, s76, 1024\n\t" \
  "v_add_u32_e32 v6, 0x20000, v1\n\t" \
  "v_add_u32_e32 v127, 0x20000, v3\n\t" \
  "s_waitcnt lgkmcnt(0)\n\t" \
  "v_mfma_f32_16x16x128_f8f6f4 a[16:19], v[128:135], v[192:199], a[16:19]\n\t" \
  "s_waitcnt vmcnt(16)\n\t" \
  "s_barrier\n\t" \
  "v_mfma_f32_16x16x128_f8f6f4 a[20:23], v[128:135], v[200:207], a[20:23]\n\t" \
  "ds_read_b128 v[160:163], v1 offset:49152\n\t" \
  "ds_read_b128 v[164:167], v1 offset:57344\n\t" \
  "v_mul_f32_e32 v32, s56, v32\n\t" \
  "v_mul_f32_e32 v33, s56, v33\n\t" \
  "v_mul_f32_e32 v34, s56, v34\n\t" \
  "v_mul_f32_e32 v35, s56, v35\n\t" \
  "v_mfma_f32_16x16x128_f8f6f4 a[24:27], v[128:135], v[208:215], a[24:27]\n\t" \
  "ds_read_b128 v[168:171], v1 offset:50176\n\t" \
  "ds_read_b128 v[172:175], v1 offset:58368\n\t" \
  "v_min_f32_e32 v12, s60, v32\n\t" \
  "v_min_f32_e32 v13, s60, v33\n\t" \
  "v_min_f32_e32 v14, s60, v34\n\t" \
  "v_min_f32_e32 v15, s60, v35\n\t" \
  "v_mfma_f32_16x16x128_f8f6f4 a[28:31], v[128:135], v[216:223], a[28:31]\n\t" \
  "ds_read_b128 v[176:179], v1 offset:51200\n\t" \
  "ds_read_b128 v[180:183], v1 offset:59392\n\t" \
  "v_exp_f32_e32 v12, v12\n\t" \
  "v_exp_f32_e32 v13, v13\n\t" \
  "v_mfma_f32_16x16x128_f8f6f4 a[60:63], v[136:143], v[216:223], a[60:63]\n\t" \
  "ds_read_b128 v[184:187], v1 offset:52224\n\t" \
  "ds_read_b128 v[188:191], v1 offset:60416\n\t" \
  "v_mfma_f32_16x16x128_f8f6f4 a[56:59], v[136:143], v[208:215], a[56:59]\n\t" \
  "ds_read_b128 v[224:227], v3 offset:32768\n\t" \
  "ds_read_b128 v[228:231], v3 offset:40960\n\t" \
  "v_exp_f32_e32 v14, v14\n\t" \
  "v_exp_f32_e32 v15, v15\n\t" \
  "v_mfma_f32_16x16x128_f8f6f4 a[52:55], v[136:143], v[200:207], a[52:55]\n\t" \
  "ds_read_b128 v[232:235], v3 offset:33792\n\t" \
  "ds_read_b128 v[236:239], v3 offset:41984\n\t" \
  "s_add_u32 s53, s52, 65536\n\t" \
  "s_mov_b32 m0, s48\n\t" \
  "v_fma_f32 v12, v12, s58, s58\n\t" \
  "v_fma_f32 v13, v13, s58, s58\n\t" \
  "v_fma_f32 v14, v14, s58, s58\n\t" \
  "v_fma_f32 v15, v15, s58, s58\n\t" \
  "v_mfma_f32_16x16x128_f8f6f4 a[48:51], v[136:143], v[192:199], a[48:51]\n\t" \
  "buffer_load_dwordx4 v0, s[36:39], s53 offen sc0 sc1 lds\n\t" \
  "v_rcp_f32_e32 v12, v12\n\t" \
  "v_rcp_f32_e32 v13, v13\n\t" \
  "v_mfma_f32_16x16x128_f8f6f4 a[80:83], v[144:151], v[192:199], a[80:83]\n\t" \
  "buffer_load_dwordx4 v0, s[36:39], s53 offen offset:1024 sc0 sc1 lds\n\t" \
  "v_mfma_f32_16x16x128_f8f6f4 a[84:87], v[144:151], v[200:207], a[84:87]\n\t" \
  "buffer_load_dwordx4 v0, s[36:39], s53 offen offset:2048 sc0 sc1 lds\n\t" \
  "v_rcp_f32_e32 v14, v14\n\t" \
  "v_rcp_f32_e32 v15, v15\n\t" \
  "v_mfma_f32_16x16x128_f8f6f4 a[88:91], v[144:151], v[208:215], a[88:91]\n\t" \
  "buffer_load_dwordx4 v0, s[36:39], s53 offen offset:3072 sc0 sc1 lds\n\t" \
  "s_add_u32 s53, s52, 81920\n\t" \
  "s_add_u32 m0, s48, 16384\n\t" \
  "v_mul_f32_e32 v32, v32, v12\n\t" \
  "v_mul_f32_e32 v33, v33, v13\n\t" \
  "v_mul_f32_e32 v34, v34, v14\n\t" \
  "v_mul_f32_e32 v35, v35, v15\n\t" \
  "v_mfma_f32_16x16x128_f8f6f4 a[92:95], v[144:151], v[216:223], a[92:95]\n\t" \
  "buffer_load_dwordx4 v0, s[36:39], s53 offen sc0 sc1 lds\n\t" \
  "s_xor_b32 s91, s68, 0\n\t" \
  "buffer_store_dwordx2 v[32:33], v5, s[40:43], s91 offen nt\n\t" \
  "v_mfma_f32_16x16x128_f8f6f4 a[124:127], v[152:159], v[216:223], a[124:127]\n\t" \
  "buffer_load_dwordx4 v0, s[36:39], s53 offen offset:1024 sc0 sc1 lds\n\t" \
  "v_mfma_f32_16x16x128_f8f6f4 a[120:123], v[152:159], v[208:215], a[120:123]\n\t" \
  "buffer_load_dwordx4 v0, s[36:39], s53 offen offset:2048 sc0 sc1 lds\n\t" \
  "buffer_store_dwordx2 v[34:35], v5, s[40:43], s91 offen offset:128 nt\n\t" \
  "v_mfma_f32_16x16x128_f8f6f4 a[116:119], v[152:159], v[200:207], a[116:119]\n\t" \
  "buffer_load_dwordx4 v0, s[36:39], s53 offen offset:3072 sc0 sc1 lds\n\t" \
  "v_mul_f32_e32 v36, s56, v36\n\t" \
  "v_mul_f32_e32 v37, s56, v37\n\t" \
  "v_mul_f32_e32 v38, s56, v38\n\t" \
  "v_mul_f32_e32 v39, s56, v39\n\t" \
  "v_mfma_f32_16x16x128_f8f6f4 a[112:115], v[152:159], v[192:199], a[112:115]\n\t" \
  "s_waitcnt lgkmcnt(0)\n\t" \
  "v_mfma_f32_16x16x128_f8f6f4 a[144:147], v[160:167], v[192:199], a[144:147]\n\t" \
  "s_waitcnt vmcnt(22)\n\t" \
  "s_barrier\n\t" \
  "v_mfma_f32_16x16x128_f8f6f4 a[148:151], v[160:167], v[200:207], a[148:151]\n\t" \
  "ds_read_b128 v[128:131], v2 offset:0\n\t" \
  "ds_read_b128 v[132:135], v2 offset:8192\n\t" \
  "v_min_f32_e32 v16, s60, v36\n\t" \
  "v_min_f32_e32 v17, s60, v37\n\t" \
  "v_min_f32_e32 v18, s60, v38\n\t" \
  "v_min_f32_e32 v19, s60, v39\n\t" \
  "v_mfma_f32_16x16x128_f8f6f4 a[152:155], v[160:167], v[208:215], a[152:155]\n\t" \
  "ds_read_b128 v[136:139], v2 offset:1024\n\t" \
  "ds_read_b128 v[140:143], v2 offset:9216\n\t" \
  "v_exp_f32_e32 v16, v16\n\t" \
  "v_exp_f32_e32 v17, v17\n\t" \
  "v_mfma_f32_16x16x128_f8f6f4 a[156:159], v[160:167], v[216:223], a[156:159]\n\t" \
  "ds_read_b128 v[144:147], v2 offset:2048\n\t" \
  "ds_read_b128 v[148:151], v2 offset:10240\n\t" \
  "v_exp_f32_e32 v18, v18\n\t" \
  "v_exp_f32_e32 v19, v19\n\t" \
  "v_mfma_f32_16x16x128_f8f6f4 a[188:191], v[168:175], v[216:223], a[188:191]\n\t" \
  "ds_read_b128 v[152:155], v2 offset:3072\n\t" \
  "ds_read_b128 v[156:159], v2 offset:11264\n\t" \
  "v_mfma_f32_16x16x128_f8f6f4 a[184:187], v[168:175], v[208:215], a[184:187]\n\t" \
  "ds_read_b128 v[240:243], v3 offset:34816\n\t" \
  "ds_read_b128 v[244:247], v3 offset:43008\n\t" \
  "v_fma_f32 v16, v16, s58, s58\n\t" \
  "v_fma_f32 v17, v17, s58, s58\n\t" \
  "v_fma_f32 v18, v18, s58, s58\n\t" \
  "v_fma_f32 v19, v19, s58, s58\n\t" \
  "v_mfma_f32_16x16x128_f8f6f4 a[180:183], v[168:175], v[200:207], a[180:183]\n\t" \
  "ds_read_b128 v[248:251], v3 offset:35840\n\t" \
  "ds_read_b128 v[252:255], v3 offset:44032\n\t" \
  "s_add_u32 s53, s52, 98304\n\t" \
  "s_add_u32 m0, s48, 131072\n\t" \
  "v_rcp_f32_e32 v16, v16\n\t" \
  "v_rcp_f32_e32 v17, v17\n\t" \
  "v_mfma_f32_16x16x128_f8f6f4 a[176:179], v[168:175], v[192:199], a[176:179]\n\t" \
  "buffer_load_dwordx4 v0, s[36:39], s53 offen sc0 sc1 lds\n\t" \
  "v_rcp_f32_e32 v18, v18\n\t" \
  "v_rcp_f32_e32 v19, v19\n\t" \
  "v_mfma_f32_16x16x128_f8f6f4 a[208:211], v[176:183], v[192:199], a[208:211]\n\t" \
  "buffer_load_dwordx4 v0, s[36:39], s53 offen offset:1024 sc0 sc1 lds\n\t" \
  "v_mfma_f32_16x16x128_f8f6f4 a[212:215], v[176:183], v[200:207], a[212:215]\n\t" \
  "buffer_load_dwordx4 v0, s[36:39], s53 offen offset:2048 sc0 sc1 lds\n\t" \
  "v_mul_f32_e32 v36, v36, v16\n\t" \
  "v_mul_f32_e32 v37, v37, v17\n\t" \
  "v_mul_f32_e32 v38, v38, v18\n\t" \
  "v_mul_f32_e32 v39, v39, v19\n\t" \
  "v_mfma_f32_16x16x128_f8f6f4 a[216:219], v[176:183], v[208:215], a[216:219]\n\t" \
  "buffer_load_dwordx4 v0, s[36:39], s53 offen offset:3072 sc0 sc1 lds\n\t" \
  "s_add_u32 s53, s52, 114688\n\t" \
  "s_add_u32 m0, s48, 147456\n\t" \
  "s_xor_b32 s91, s68, 65536\n\t" \
  "buffer_store_dwordx2 v[36:37], v5, s[40:43], s91 offen nt\n\t" \
  "v_mfma_f32_16x16x128_f8f6f4 a[220:223], v[176:183], v[216:223], a[220:223]\n\t" \
  "buffer_load_dwordx4 v0, s[36:39], s53 offen sc0 sc1 lds\n\t" \
  "buffer_store_dwordx2 v[38:39], v5, s[40:43], s91 offen offset:128 nt\n\t" \
  "v_mfma_f32_16x16x128_f8f6f4 a[252:255], v[184:191], v[216:223], a[252:255]\n\t" \
  "buffer_load_dwordx4 v0, s[36:39], s53 offen offset:1024 sc0 sc1 lds\n\t" \
  "v_mfma_f32_16x16x128_f8f6f4 a[248:251], v[184:191], v[208:215], a[248:251]\n\t" \
  "buffer_load_dwordx4 v0, s[36:39], s53 offen offset:2048 sc0 sc1 lds\n\t" \
  "v_mul_f32_e32 v40, s56, v40\n\t" \
  "v_mul_f32_e32 v41, s56, v41\n\t" \
  "v_mul_f32_e32 v42, s56, v42\n\t" \
  "v_mul_f32_e32 v43, s56, v43\n\t" \
  "v_mfma_f32_16x16x128_f8f6f4 a[244:247], v[184:191], v[200:207], a[244:247]\n\t" \
  "buffer_load_dwordx4 v0, s[36:39], s53 offen offset:3072 sc0 sc1 lds\n\t" \
  "v_min_f32_e32 v20, s60, v40\n\t" \
  "v_min_f32_e32 v21, s60, v41\n\t" \
  "v_min_f32_e32 v22, s60, v42\n\t" \
  "v_min_f32_e32 v23, s60, v43\n\t" \
  "v_mfma_f32_16x16x128_f8f6f4 a[240:243], v[184:191], v[192:199], a[240:243]\n\t" \
  "s_waitcnt lgkmcnt(0)\n\t" \
  "v_mfma_f32_16x16x128_f8f6f4 a[16:19], v[128:135], v[224:231], a[16:19]\n\t" \
  "s_waitcnt vmcnt(20)\n\t" \
  "s_barrier\n\t" \
  "v_mfma_f32_16x16x128_f8f6f4 a[20:23], v[128:135], v[232:239], a[20:23]\n\t" \
  "ds_read_b128 v[160:163], v2 offset:49152\n\t" \
  "ds_read_b128 v[164:167], v2 offset:57344\n\t" \
  "v_exp_f32_e32 v20, v20\n\t" \
  "v_exp_f32_e32 v21, v21\n\t" \
  "v_mfma_f32_16x16x128_f8f6f4 a[24:27], v[128:135], v[240:247], a[24:27]\n\t" \
  "ds_read_b128 v[168:171], v2 offset:50176\n\t" \
  "ds_read_b128 v[172:175], v2 offset:58368\n\t" \
  "v_exp_f32_e32 v22, v22\n\t" \
  "v_exp_f32_e32 v23, v23\n\t" \
  "v_mfma_f32_16x16x128_f8f6f4 a[28:31], v[128:135], v[248:255], a[28:31]\n\t" \
  "ds_read_b128 v[176:179], v2 offset:51200\n\t" \
  "ds_read_b128 v[180:183], v2 offset:59392\n\t" \
  "v_fma_f32 v20, v20, s58, s58\n\t" \
  "v_fma_f32 v21, v21, s58, s58\n\t" \
  "v_fma_f32 v22, v22, s58, s58\n\t" \
  "v_fma_f32 v23, v23, s58, s58\n\t" \
  "v_mfma_f32_16x16x128_f8f6f4 a[60:63], v[136:143], v[248:255], a[60:63]\n\t" \
  "ds_read_b128 v[184:187], v2 offset:52224\n\t" \
  "ds_read_b128 v[188:191], v2 offset:60416\n\t" \
  "v_mfma_f32_16x16x128_f8f6f4 a[56:59], v[136:143], v[240:247], a[56:59]\n\t" \
  "ds_read_b128 v[192:195], v4 offset:16384\n\t" \
  "ds_read_b128 v[196:199], v4 offset:24576\n\t" \
  "v_rcp_f32_e32 v20, v20\n\t" \
  "v_rcp_f32_e32 v21, v21\n\t" \
  "v_mfma_f32_16x16x128_f8f6f4 a[52:55], v[136:143], v[232:239], a[52:55]\n\t" \
  "ds_read_b128 v[200:203], v4 offset:17408\n\t" \
  "ds_read_b128 v[204:207], v4 offset:25600\n\t" \
  "s_mov_b32 s53, s50\n\t" \
  "s_add_u32 m0, s48, 49152\n\t" \
  "v_rcp_f32_e32 v22, v22\n\t" \
  "v_rcp_f32_e32 v23, v23\n\t" \
  "v_mfma_f32_16x16x128_f8f6f4 a[48:51], v[136:143], v[224:231], a[48:51]\n\t" \
  "buffer_load_dwordx4 v0, s[36:39], s53 offen sc0 sc1 lds\n\t" \
  "v_mul_f32_e32 v40, v40, v20\n\t" \
  "v_mul_f32_e32 v41, v41, v21\n\t" \
  "v_mul_f32_e32 v42, v42, v22\n\t" \
  "v_mul_f32_e32 v43, v43, v23\n\t" \
  "v_mfma_f32_16x16x128_f8f6f4 a[80:83], v[144:151], v[224:231], a[80:83]\n\t" \
  "buffer_load_dwordx4 v0, s[36:39], s53 offen offset:1024 sc0 sc1 lds\n\t" \
  "v_mfma_f32_16x16x128_f8f6f4 a[84:87], v[144:151], v[232:239], a[84:87]\n\t" \
  "buffer_load_dwordx4 v0, s[36:39], s53 offen offset:2048 sc0 sc1 lds\n\t" \
  "s_xor_b32 s91, s68, 131072\n\t" \
  "buffer_store_dwordx2 v[40:41], v5, s[40:43], s91 offen nt\n\t" \
  "v_mfma_f32_16x16x128_f8f6f4 a[88:91], v[144:151], v[240:247], a[88:91]\n\t" \
  "buffer_load_dwordx4 v0, s[36:39], s53 offen offset:3072 sc0 sc1 lds\n\t" \
  "s_add_u32 s53, s52, 131072\n\t" \
  "s_add_u32 m0, s48, 32768\n\t" \
  "buffer_store_dwordx2 v[42:43], v5, s[40:43], s91 offen offset:128 nt\n\t" \
  "v_mfma_f32_16x16x128_f8f6f4 a[92:95], v[144:151], v[248:255], a[92:95]\n\t" \
  "buffer_load_dwordx4 v0, s[36:39], s53 offen sc0 sc1 lds\n\t" \
  "v_mfma_f32_16x16x128_f8f6f4 a[124:127], v[152:159], v[248:255], a[124:127]\n\t" \
  "buffer_load_dwordx4 v0, s[36:39], s53 offen offset:1024 sc0 sc1 lds\n\t" \
  "v_mul_f32_e32 v44, s56, v44\n\t" \
  "v_mul_f32_e32 v45, s56, v45\n\t" \
  "v_mul_f32_e32 v46, s56, v46\n\t" \
  "v_mul_f32_e32 v47, s56, v47\n\t" \
  "v_mfma_f32_16x16x128_f8f6f4 a[120:123], v[152:159], v[240:247], a[120:123]\n\t" \
  "buffer_load_dwordx4 v0, s[36:39], s53 offen offset:2048 sc0 sc1 lds\n\t" \
  "v_min_f32_e32 v24, s60, v44\n\t" \
  "v_min_f32_e32 v25, s60, v45\n\t" \
  "v_min_f32_e32 v26, s60, v46\n\t" \
  "v_min_f32_e32 v27, s60, v47\n\t" \
  "v_mfma_f32_16x16x128_f8f6f4 a[116:119], v[152:159], v[232:239], a[116:119]\n\t" \
  "buffer_load_dwordx4 v0, s[36:39], s53 offen offset:3072 sc0 sc1 lds\n\t" \
  "v_exp_f32_e32 v24, v24\n\t" \
  "v_exp_f32_e32 v25, v25\n\t" \
  "v_mfma_f32_16x16x128_f8f6f4 a[112:115], v[152:159], v[224:231], a[112:115]\n\t" \
  "s_waitcnt lgkmcnt(0)\n\t" \
  "v_mfma_f32_16x16x128_f8f6f4 a[144:147], v[160:167], v[224:231], a[144:147]\n\t" \
  "s_waitcnt vmcnt(26)\n\t" \
  "s_barrier\n\t" \
  "v_mfma_f32_16x16x128_f8f6f4 a[148:151], v[160:167], v[232:239], a[148:151]\n\t" \
  "ds_read_b128 v[128:131], v1 offset:0\n\t" \
  "ds_read_b128 v[132:135], v1 offset:8192\n\t" \
  "v_exp_f32_e32 v26, v26\n\t" \
  "v_exp_f32_e32 v27, v27\n\t" \
  "v_mfma_f32_16x16x128_f8f6f4 a[152:155], v[160:167], v[240:247], a[152:155]\n\t" \
  "ds_read_b128 v[136:139], v1 offset:1024\n\t" \
  "ds_read_b128 v[140:143], v1 offset:9216\n\t" \
  "v_fma_f32 v24, v24, s58, s58\n\t" \
  "v_fma_f32 v25, v25, s58, s58\n\t" \
  "v_fma_f32 v26, v26, s58, s58\n\t" \
  "v_fma_f32 v27, v27, s58, s58\n\t" \
  "v_mfma_f32_16x16x128_f8f6f4 a[156:159], v[160:167], v[248:255], a[156:159]\n\t" \
  "ds_read_b128 v[144:147], v1 offset:2048\n\t" \
  "ds_read_b128 v[148:151], v1 offset:10240\n\t" \
  "v_rcp_f32_e32 v24, v24\n\t" \
  "v_rcp_f32_e32 v25, v25\n\t" \
  "v_mfma_f32_16x16x128_f8f6f4 a[188:191], v[168:175], v[248:255], a[188:191]\n\t" \
  "ds_read_b128 v[152:155], v1 offset:3072\n\t" \
  "ds_read_b128 v[156:159], v1 offset:11264\n\t" \
  "v_mfma_f32_16x16x128_f8f6f4 a[184:187], v[168:175], v[240:247], a[184:187]\n\t" \
  "ds_read_b128 v[208:211], v4 offset:18432\n\t" \
  "ds_read_b128 v[212:215], v4 offset:26624\n\t" \
  "v_rcp_f32_e32 v26, v26\n\t" \
  "v_rcp_f32_e32 v27, v27\n\t" \
  "v_mfma_f32_16x16x128_f8f6f4 a[180:183], v[168:175], v[232:239], a[180:183]\n\t" \
  "ds_read_b128 v[216:219], v4 offset:19456\n\t" \
  "ds_read_b128 v[220:223], v4 offset:27648\n\t" \
  "s_add_u32 s53, s52, 147456\n\t" \
  "s_add_u32 m0, s48, 65536\n\t" \
  "v_mul_f32_e32 v44, v44, v24\n\t" \
  "v_mul_f32_e32 v45, v45, v25\n\t" \
  "v_mul_f32_e32 v46, v46, v26\n\t" \
  "v_mul_f32_e32 v47, v47, v27\n\t" \
  "v_mfma_f32_16x16x128_f8f6f4 a[176:179], v[168:175], v[224:231], a[176:179]\n\t" \
  "buffer_load_dwordx4 v0, s[36:39], s53 offen sc0 sc1 lds\n\t" \
  "s_xor_b32 s91, s68, 196608\n\t" \
  "buffer_store_dwordx2 v[44:45], v5, s[40:43], s91 offen nt\n\t" \
  "v_mfma_f32_16x16x128_f8f6f4 a[208:211], v[176:183], v[224:231], a[208:211]\n\t" \
  "buffer_load_dwordx4 v0, s[36:39], s53 offen offset:1024 sc0 sc1 lds\n\t" \
  "v_mfma_f32_16x16x128_f8f6f4 a[212:215], v[176:183], v[232:239], a[212:215]\n\t" \
  "buffer_load_dwordx4 v0, s[36:39], s53 offen offset:2048 sc0 sc1 lds\n\t" \
  "buffer_store_dwordx2 v[46:47], v5, s[40:43], s91 offen offset:128 nt\n\t" \
  "v_mfma_f32_16x16x128_f8f6f4 a[216:219], v[176:183], v[240:247], a[216:219]\n\t" \
  "buffer_load_dwordx4 v0, s[36:39], s53 offen offset:3072 sc0 sc1 lds\n\t" \
  "s_add_u32 s53, s50, 16384\n\t" \
  "s_add_u32 m0, s48, 114688\n\t" \
  "v_mul_f32_e32 v48, s56, v48\n\t" \
  "v_mul_f32_e32 v49, s56, v49\n\t" \
  "v_mul_f32_e32 v50, s56, v50\n\t" \
  "v_mul_f32_e32 v51, s56, v51\n\t" \
  "v_mfma_f32_16x16x128_f8f6f4 a[220:223], v[176:183], v[248:255], a[220:223]\n\t" \
  "buffer_load_dwordx4 v0, s[36:39], s53 offen sc0 sc1 lds\n\t" \
  "v_mfma_f32_16x16x128_f8f6f4 a[252:255], v[184:191], v[248:255], a[252:255]\n\t" \
  "buffer_load_dwordx4 v0, s[36:39], s53 offen offset:1024 sc0 sc1 lds\n\t" \
  "v_min_f32_e32 v28, s60, v48\n\t" \
  "v_min_f32_e32 v29, s60, v49\n\t" \
  "v_min_f32_e32 v30, s60, v50\n\t" \
  "v_min_f32_e32 v31, s60, v51\n\t" \
  "v_mfma_f32_16x16x128_f8f6f4 a[248:251], v[184:191], v[240:247], a[248:251]\n\t" \
  "buffer_load_dwordx4 v0, s[36:39], s53 offen offset:2048 sc0 sc1 lds\n\t" \
  "v_exp_f32_e32 v28, v28\n\t" \
  "v_exp_f32_e32 v29, v29\n\t" \
  "v_mfma_f32_16x16x128_f8f6f4 a[244:247], v[184:191], v[232:239], a[244:247]\n\t" \
  "buffer_load_dwordx4 v0, s[36:39], s53 offen offset:3072 sc0 sc1 lds\n\t" \
  "v_exp_f32_e32 v30, v30\n\t" \
  "v_exp_f32_e32 v31, v31\n\t" \
  "v_mfma_f32_16x16x128_f8f6f4 a[240:243], v[184:191], v[224:231], a[240:243]\n\t" \
  "s_waitcnt lgkmcnt(0)\n\t" \
  "v_mfma_f32_16x16x128_f8f6f4 a[16:19], v[128:135], v[192:199], a[16:19]\n\t" \
  "s_waitcnt vmcnt(30)\n\t" \
  "s_barrier\n\t" \
  "v_mfma_f32_16x16x128_f8f6f4 a[20:23], v[128:135], v[200:207], a[20:23]\n\t" \
  "ds_read_b128 v[160:163], v1 offset:16384\n\t" \
  "ds_read_b128 v[164:167], v1 offset:24576\n\t" \
  "v_fma_f32 v28, v28, s58, s58\n\t" \
  "v_fma_f32 v29, v29, s58, s58\n\t" \
  "v_fma_f32 v30, v30, s58, s58\n\t" \
  "v_fma_f32 v31, v31, s58, s58\n\t" \
  "v_mfma_f32_16x16x128_f8f6f4 a[24:27], v[128:135], v[208:215], a[24:27]\n\t" \
  "ds_read_b128 v[168:171], v1 offset:17408\n\t" \
  "ds_read_b128 v[172:175], v1 offset:25600\n\t" \
  "v_rcp_f32_e32 v28, v28\n\t" \
  "v_rcp_f32_e32 v29, v29\n\t" \
  "v_mfma_f32_16x16x128_f8f6f4 a[28:31], v[128:135], v[216:223], a[28:31]\n\t" \
  "ds_read_b128 v[176:179], v1 offset:18432\n\t" \
  "ds_read_b128 v[180:183], v1 offset:26624\n\t" \
  "v_rcp_f32_e32 v30, v30\n\t" \
  "v_rcp_f32_e32 v31, v31\n\t" \
  "v_mfma_f32_16x16x128_f8f6f4 a[60:63], v[136:143], v[216:223], a[60:63]\n\t" \
  "ds_read_b128 v[184:187], v1 offset:19456\n\t" \
  "ds_read_b128 v[188:191], v1 offset:27648\n\t" \
  "v_mfma_f32_16x16x128_f8f6f4 a[56:59], v[136:143], v[208:215], a[56:59]\n\t" \
  "ds_read_b128 v[224:227], v4 offset:32768\n\t" \
  "ds_read_b128 v[228:231], v4 offset:40960\n\t" \
  "v_mul_f32_e32 v48, v48, v28\n\t" \
  "v_mul_f32_e32 v49, v49, v29\n\t" \
  "v_mul_f32_e32 v50, v50, v30\n\t" \
  "v_mul_f32_e32 v51, v51, v31\n\t" \
  "v_mfma_f32_16x16x128_f8f6f4 a[52:55], v[136:143], v[200:207], a[52:55]\n\t" \
  "ds_read_b128 v[232:235], v4 offset:33792\n\t" \
  "ds_read_b128 v[236:239], v4 offset:41984\n\t" \
  "s_add_u32 s53, s52, 163840\n\t" \
  "s_mov_b32 m0, s48\n\t" \
  "s_xor_b32 s91, s68, 1048576\n\t" \
  "buffer_store_dwordx2 v[48:49], v5, s[40:43], s91 offen nt\n\t" \
  "v_mfma_f32_16x16x128_f8f6f4 a[48:51], v[136:143], v[192:199], a[48:51]\n\t" \
  "buffer_load_dwordx4 v0, s[36:39], s53 offen sc0 sc1 lds\n\t" \
  "v_mfma_f32_16x16x128_f8f6f4 a[80:83], v[144:151], v[192:199], a[80:83]\n\t" \
  "buffer_load_dwordx4 v0, s[36:39], s53 offen offset:1024 sc0 sc1 lds\n\t" \
  "buffer_store_dwordx2 v[50:51], v5, s[40:43], s91 offen offset:128 nt\n\t" \
  "v_mfma_f32_16x16x128_f8f6f4 a[84:87], v[144:151], v[200:207], a[84:87]\n\t" \
  "buffer_load_dwordx4 v0, s[36:39], s53 offen offset:2048 sc0 sc1 lds\n\t" \
  "v_mul_f32_e32 v52, s56, v52\n\t" \
  "v_mul_f32_e32 v53, s56, v53\n\t" \
  "v_mul_f32_e32 v54, s56, v54\n\t" \
  "v_mul_f32_e32 v55, s56, v55\n\t" \
  "v_mfma_f32_16x16x128_f8f6f4 a[88:91], v[144:151], v[208:215], a[88:91]\n\t" \
  "buffer_load_dwordx4 v0, s[36:39], s53 offen offset:3072 sc0 sc1 lds\n\t" \
  "s_add_u32 s53, s52, 180224\n\t" \
  "s_add_u32 m0, s48, 81920\n\t" \
  "v_min_f32_e32 v12, s60, v52\n\t" \
  "v_min_f32_e32 v13, s60, v53\n\t" \
  "v_min_f32_e32 v14, s60, v54\n\t" \
  "v_min_f32_e32 v15, s60, v55\n\t" \
  "v_mfma_f32_16x16x128_f8f6f4 a[92:95], v[144:151], v[216:223], a[92:95]\n\t" \
  "buffer_load_dwordx4 v0, s[36:39], s53 offen sc0 sc1 lds\n\t" \
  "v_mfma_f32_16x16x128_f8f6f4 a[124:127], v[152:159], v[216:223], a[124:127]\n\t" \
  "buffer_load_dwordx4 v0, s[36:39], s53 offen offset:1024 sc0 sc1 lds\n\t" \
  "v_exp_f32_e32 v12, v12\n\t" \
  "v_exp_f32_e32 v13, v13\n\t" \
  "v_mfma_f32_16x16x128_f8f6f4 a[120:123], v[152:159], v[208:215], a[120:123]\n\t" \
  "buffer_load_dwordx4 v0, s[36:39], s53 offen offset:2048 sc0 sc1 lds\n\t" \
  "v_exp_f32_e32 v14, v14\n\t" \
  "v_exp_f32_e32 v15, v15\n\t" \
  "v_mfma_f32_16x16x128_f8f6f4 a[116:119], v[152:159], v[200:207], a[116:119]\n\t" \
  "buffer_load_dwordx4 v0, s[36:39], s53 offen offset:3072 sc0 sc1 lds\n\t" \
  "v_fma_f32 v12, v12, s58, s58\n\t" \
  "v_fma_f32 v13, v13, s58, s58\n\t" \
  "v_fma_f32 v14, v14, s58, s58\n\t" \
  "v_fma_f32 v15, v15, s58, s58\n\t" \
  "v_mfma_f32_16x16x128_f8f6f4 a[112:115], v[152:159], v[192:199], a[112:115]\n\t" \
  "buffer_load_dword v7, v8, s[44:47], s76 offen sc0 sc1\n\t" \
  "s_waitcnt lgkmcnt(0)\n\t" \
  "v_mfma_f32_16x16x128_f8f6f4 a[144:147], v[160:167], v[192:199], a[144:147]\n\t" \
  "s_waitcnt vmcnt(37)\n\t" \
  "s_barrier\n\t" \
  "v_mfma_f32_16x16x128_f8f6f4 a[148:151], v[160:167], v[200:207], a[148:151]\n\t" \
  "ds_read_b128 v[128:131], v6 offset:0\n\t" \
  "ds_read_b128 v[132:135], v6 offset:8192\n\t" \
  "v_rcp_f32_e32 v12, v12\n\t" \
  "v_rcp_f32_e32 v13, v13\n\t" \
  "v_mfma_f32_16x16x128_f8f6f4 a[152:155], v[160:167], v[208:215], a[152:155]\n\t" \
  "ds_read_b128 v[136:139], v6 offset:1024\n\t" \
  "ds_read_b128 v[140:143], v6 offset:9216\n\t" \
  "v_rcp_f32_e32 v14, v14\n\t" \
  "v_rcp_f32_e32 v15, v15\n\t" \
  "v_mfma_f32_16x16x128_f8f6f4 a[156:159], v[160:167], v[216:223], a[156:159]\n\t" \
  "ds_read_b128 v[144:147], v6 offset:2048\n\t" \
  "ds_read_b128 v[148:151], v6 offset:10240\n\t" \
  "v_mul_f32_e32 v52, v52, v12\n\t" \
  "v_mul_f32_e32 v53, v53, v13\n\t" \
  "v_mul_f32_e32 v54, v54, v14\n\t" \
  "v_mul_f32_e32 v55, v55, v15\n\t" \
  "v_mfma_f32_16x16x128_f8f6f4 a[188:191], v[168:175], v[216:223], a[188:191]\n\t" \
  "ds_read_b128 v[152:155], v6 offset:3072\n\t" \
  "ds_read_b128 v[156:159], v6 offset:11264\n\t" \
  "v_mfma_f32_16x16x128_f8f6f4 a[184:187], v[168:175], v[208:215], a[184:187]\n\t" \
  "ds_read_b128 v[240:243], v4 offset:34816\n\t" \
  "ds_read_b128 v[244:247], v4 offset:43008\n\t" \
  "s_xor_b32 s91, s68, 1114112\n\t" \
  "buffer_store_dwordx2 v[52:53], v5, s[40:43], s91 offen nt\n\t" \
  "v_mfma_f32_16x16x128_f8f6f4 a[180:183], v[168:175], v[200:207], a[180:183]\n\t" \
  "ds_read_b128 v[248:251], v4 offset:35840\n\t" \
  "ds_read_b128 v[252:255], v4 offset:44032\n\t" \
  "buffer_store_dwordx2 v[54:55], v5, s[40:43], s91 offen offset:128 nt\n\t" \
  "v_mfma_f32_16x16x128_f8f6f4 a[176:179], v[168:175], v[192:199], a[176:179]\n\t" \
  "s_add_u32 s53, s50, 32768\n\t" \
  "s_add_u32 m0, s48, 16384\n\t" \
  "v_mfma_f32_16x16x128_f8f6f4 a[208:211], v[176:183], v[192:199], a[208:211]\n\t" \
  "buffer_load_dwordx4 v0, s[36:39], s53 offen sc0 sc1 lds\n\t" \
  "v_mul_f32_e32 v56, s56, v56\n\t" \
  "v_mul_f32_e32 v57, s56, v57\n\t" \
  "v_mul_f32_e32 v58, s56, v58\n\t" \
  "v_mul_f32_e32 v59, s56, v59\n\t" \
  "v_mfma_f32_16x16x128_f8f6f4 a[212:215], v[176:183], v[200:207], a[212:215]\n\t" \
  "v_min_f32_e32 v16, s60, v56\n\t" \
  "v_min_f32_e32 v17, s60, v57\n\t" \
  "v_min_f32_e32 v18, s60, v58\n\t" \
  "v_min_f32_e32 v19, s60, v59\n\t" \
  "v_mfma_f32_16x16x128_f8f6f4 a[216:219], v[176:183], v[208:215], a[216:219]\n\t" \
  "buffer_load_dwordx4 v0, s[36:39], s53 offen offset:1024 sc0 sc1 lds\n\t" \
  "v_exp_f32_e32 v16, v16\n\t" \
  "v_exp_f32_e32 v17, v17\n\t" \
  "v_mfma_f32_16x16x128_f8f6f4 a[220:223], v[176:183], v[216:223], a[220:223]\n\t" \
  "v_mfma_f32_16x16x128_f8f6f4 a[252:255], v[184:191], v[216:223], a[252:255]\n\t" \
  "buffer_load_dwordx4 v0, s[36:39], s53 offen offset:2048 sc0 sc1 lds\n\t" \
  "v_exp_f32_e32 v18, v18\n\t" \
  "v_exp_f32_e32 v19, v19\n\t" \
  "v_mfma_f32_16x16x128_f8f6f4 a[248:251], v[184:191], v[208:215], a[248:251]\n\t" \
  "v_fma_f32 v16, v16, s58, s58\n\t" \
  "v_fma_f32 v17, v17, s58, s58\n\t" \
  "v_fma_f32 v18, v18, s58, s58\n\t" \
  "v_fma_f32 v19, v19, s58, s58\n\t" \
  "v_mfma_f32_16x16x128_f8f6f4 a[244:247], v[184:191], v[200:207], a[244:247]\n\t" \
  "buffer_load_dwordx4 v0, s[36:39], s53 offen offset:3072 sc0 sc1 lds\n\t" \
  "v_rcp_f32_e32 v16, v16\n\t" \
  "v_rcp_f32_e32 v17, v17\n\t" \
  "v_mfma_f32_16x16x128_f8f6f4 a[240:243], v[184:191], v[192:199], a[240:243]\n\t" \
  "s_waitcnt lgkmcnt(0)\n\t" \
  "v_mfma_f32_16x16x128_f8f6f4 a[16:19], v[128:135], v[224:231], a[16:19]\n\t" \
  "s_waitcnt vmcnt(32)\n\t" \
  "s_barrier\n\t" \
  "v_mfma_f32_16x16x128_f8f6f4 a[20:23], v[128:135], v[232:239], a[20:23]\n\t" \
  "ds_read_b128 v[160:163], v6 offset:16384\n\t" \
  "ds_read_b128 v[164:167], v6 offset:24576\n\t" \
  "v_rcp_f32_e32 v18, v18\n\t" \
  "v_rcp_f32_e32 v19, v19\n\t" \
  "v_mfma_f32_16x16x128_f8f6f4 a[24:27], v[128:135], v[240:247], a[24:27]\n\t" \
  "ds_read_b128 v[168:171], v6 offset:17408\n\t" \
  "ds_read_b128 v[172:175], v6 offset:25600\n\t" \
  "v_mul_f32_e32 v56, v56, v16\n\t" \
  "v_mul_f32_e32 v57, v57, v17\n\t" \
  "v_mul_f32_e32 v58, v58, v18\n\t" \
  "v_mul_f32_e32 v59, v59, v19\n\t" \
  "v_mfma_f32_16x16x128_f8f6f4 a[28:31], v[128:135], v[248:255], a[28:31]\n\t" \
  "ds_read_b128 v[176:179], v6 offset:18432\n\t" \
  "ds_read_b128 v[180:183], v6 offset:26624\n\t" \
  "v_mfma_f32_16x16x128_f8f6f4 a[60:63], v[136:143], v[248:255], a[60:63]\n\t" \
  "ds_read_b128 v[184:187], v6 offset:19456\n\t" \
  "ds_read_b128 v[188:191], v6 offset:27648\n\t" \
  "s_xor_b32 s91, s68, 1179648\n\t" \
  "buffer_store_dwordx2 v[56:57], v5, s[40:43], s91 offen nt\n\t" \
  "v_mfma_f32_16x16x128_f8f6f4 a[56:59], v[136:143], v[240:247], a[56:59]\n\t" \
  "ds_read_b128 v[192:195], v3 offset:49152\n\t" \
  "ds_read_b128 v[196:199], v3 offset:57344\n\t" \
  "buffer_store_dwordx2 v[58:59], v5, s[40:43], s91 offen offset:128 nt\n\t" \
  "v_mfma_f32_16x16x128_f8f6f4 a[52:55], v[136:143], v[232:239], a[52:55]\n\t" \
  "ds_read_b128 v[200:203], v3 offset:50176\n\t" \
  "ds_read_b128 v[204:207], v3 offset:58368\n\t" \
  "s_add_u32 s53, s52, 196608\n\t" \
  "s_add_u32 m0, s48, 131072\n\t" \
  "v_mul_f32_e32 v60, s56, v60\n\t" \
  "v_mul_f32_e32 v61, s56, v61\n\t" \
  "v_mul_f32_e32 v62, s56, v62\n\t" \
  "v_mul_f32_e32 v63, s56, v63\n\t" \
  "v_mfma_f32_16x16x128_f8f6f4 a[48:51], v[136:143], v[224:231], a[48:51]\n\t" \
  "buffer_load_dwordx4 v0, s[36:39], s53 offen sc0 sc1 lds\n\t" \
  "v_mfma_f32_16x16x128_f8f6f4 a[80:83], v[144:151], v[224:231], a[80:83]\n\t" \
  "buffer_load_dwordx4 v0, s[36:39], s53 offen offset:1024 sc0 sc1 lds\n\t" \
  "v_min_f32_e32 v20, s60, v60\n\t" \
  "v_min_f32_e32 v21, s60, v61\n\t" \
  "v_min_f32_e32 v22, s60, v62\n\t" \
  "v_min_f32_e32 v23, s60, v63\n\t" \
  "v_mfma_f32_16x16x128_f8f6f4 a[84:87], v[144:151], v[232:239], a[84:87]\n\t" \
  "buffer_load_dwordx4 v0, s[36:39], s53 offen offset:2048 sc0 sc1 lds\n\t" \
  "v_exp_f32_e32 v20, v20\n\t" \
  "v_exp_f32_e32 v21, v21\n\t" \
  "v_mfma_f32_16x16x128_f8f6f4 a[88:91], v[144:151], v[240:247], a[88:91]\n\t" \
  "buffer_load_dwordx4 v0, s[36:39], s53 offen offset:3072 sc0 sc1 lds\n\t" \
  "s_add_u32 s53, s52, 212992\n\t" \
  "s_add_u32 m0, s48, 98304\n\t" \
  "v_exp_f32_e32 v22, v22\n\t" \
  "v_exp_f32_e32 v23, v23\n\t" \
  "v_mfma_f32_16x16x128_f8f6f4 a[92:95], v[144:151], v[248:255], a[92:95]\n\t" \
  "buffer_load_dwordx4 v0, s[36:39], s53 offen sc0 sc1 lds\n\t" \
  "v_mfma_f32_16x16x128_f8f6f4 a[124:127], v[152:159], v[248:255], a[124:127]\n\t" \
  "buffer_load_dwordx4 v0, s[36:39], s53 offen offset:1024 sc0 sc1 lds\n\t" \
  "v_fma_f32 v20, v20, s58, s58\n\t" \
  "v_fma_f32 v21, v21, s58, s58\n\t" \
  "v_fma_f32 v22, v22, s58, s58\n\t" \
  "v_fma_f32 v23, v23, s58, s58\n\t" \
  "v_mfma_f32_16x16x128_f8f6f4 a[120:123], v[152:159], v[240:247], a[120:123]\n\t" \
  "buffer_load_dwordx4 v0, s[36:39], s53 offen offset:2048 sc0 sc1 lds\n\t" \
  "v_rcp_f32_e32 v20, v20\n\t" \
  "v_rcp_f32_e32 v21, v21\n\t" \
  "v_mfma_f32_16x16x128_f8f6f4 a[116:119], v[152:159], v[232:239], a[116:119]\n\t" \
  "buffer_load_dwordx4 v0, s[36:39], s53 offen offset:3072 sc0 sc1 lds\n\t" \
  "v_rcp_f32_e32 v22, v22\n\t" \
  "v_rcp_f32_e32 v23, v23\n\t" \
  "v_mfma_f32_16x16x128_f8f6f4 a[112:115], v[152:159], v[224:231], a[112:115]\n\t" \
  "s_waitcnt lgkmcnt(0)\n\t" \
  "v_mfma_f32_16x16x128_f8f6f4 a[144:147], v[160:167], v[224:231], a[144:147]\n\t" \
  "s_waitcnt vmcnt(37)\n\t" \
  "s_barrier\n\t" \
  "v_mfma_f32_16x16x128_f8f6f4 a[148:151], v[160:167], v[232:239], a[148:151]\n\t" \
  "ds_read_b128 v[128:131], v1 offset:32768\n\t" \
  "ds_read_b128 v[132:135], v1 offset:40960\n\t" \
  "v_mul_f32_e32 v60, v60, v20\n\t" \
  "v_mul_f32_e32 v61, v61, v21\n\t" \
  "v_mul_f32_e32 v62, v62, v22\n\t" \
  "v_mul_f32_e32 v63, v63, v23\n\t" \
  "v_mfma_f32_16x16x128_f8f6f4 a[152:155], v[160:167], v[240:247], a[152:155]\n\t" \
  "ds_read_b128 v[136:139], v1 offset:33792\n\t" \
  "ds_read_b128 v[140:143], v1 offset:41984\n\t" \
  "s_xor_b32 s91, s68, 1245184\n\t" \
  "buffer_store_dwordx2 v[60:61], v5, s[40:43], s91 offen nt\n\t" \
  "v_mfma_f32_16x16x128_f8f6f4 a[156:159], v[160:167], v[248:255], a[156:159]\n\t" \
  "ds_read_b128 v[144:147], v1 offset:34816\n\t" \
  "ds_read_b128 v[148:151], v1 offset:43008\n\t" \
  "v_mfma_f32_16x16x128_f8f6f4 a[188:191], v[168:175], v[248:255], a[188:191]\n\t" \
  "ds_read_b128 v[152:155], v1 offset:35840\n\t" \
  "ds_read_b128 v[156:159], v1 offset:44032\n\t" \
  "buffer_store_dwordx2 v[62:63], v5, s[40:43], s91 offen offset:128 nt\n\t" \
  "v_mfma_f32_16x16x128_f8f6f4 a[184:187], v[168:175], v[240:247], a[184:187]\n\t" \
  "ds_read_b128 v[208:211], v3 offset:51200\n\t" \
  "ds_read_b128 v[212:215], v3 offset:59392\n\t" \
  "v_mul_f32_e32 v64, s56, v64\n\t" \
  "v_mul_f32_e32 v65, s56, v65\n\t" \
  "v_mul_f32_e32 v66, s56, v66\n\t" \
  "v_mul_f32_e32 v67, s56, v67\n\t" \
  "v_mfma_f32_16x16x128_f8f6f4 a[180:183], v[168:175], v[232:239], a[180:183]\n\t" \
  "ds_read_b128 v[216:219], v3 offset:52224\n\t" \
  "ds_read_b128 v[220:223], v3 offset:60416\n\t" \
  "v_min_f32_e32 v24, s60, v64\n\t" \
  "v_min_f32_e32 v25, s60, v65\n\t" \
  "v_min_f32_e32 v26, s60, v66\n\t" \
  "v_min_f32_e32 v27, s60, v67\n\t" \
  "v_mfma_f32_16x16x128_f8f6f4 a[176:179], v[168:175], v[224:231], a[176:179]\n\t" \
  "s_add_u32 s53, s50, 49152\n\t" \
  "s_add_u32 m0, s48, 147456\n\t" \
  "v_mfma_f32_16x16x128_f8f6f4 a[208:211], v[176:183], v[224:231], a[208:211]\n\t" \
  "buffer_load_dwordx4 v0, s[36:39], s53 offen sc0 sc1 lds\n\t" \
  "v_exp_f32_e32 v24, v24\n\t" \
  "v_exp_f32_e32 v25, v25\n\t" \
  "v_mfma_f32_16x16x128_f8f6f4 a[212:215], v[176:183], v[232:239], a[212:215]\n\t" \
  "v_exp_f32_e32 v26, v26\n\t" \
  "v_exp_f32_e32 v27, v27\n\t" \
  "v_mfma_f32_16x16x128_f8f6f4 a[216:219], v[176:183], v[240:247], a[216:219]\n\t" \
  "buffer_load_dwordx4 v0, s[36:39], s53 offen offset:1024 sc0 sc1 lds\n\t" \
  "v_fma_f32 v24, v24, s58, s58\n\t" \
  "v_fma_f32 v25, v25, s58, s58\n\t" \
  "v_fma_f32 v26, v26, s58, s58\n\t" \
  "v_fma_f32 v27, v27, s58, s58\n\t" \
  "v_mfma_f32_16x16x128_f8f6f4 a[220:223], v[176:183], v[248:255], a[220:223]\n\t" \
  "v_mfma_f32_16x16x128_f8f6f4 a[252:255], v[184:191], v[248:255], a[252:255]\n\t" \
  "buffer_load_dwordx4 v0, s[36:39], s53 offen offset:2048 sc0 sc1 lds\n\t" \
  "v_rcp_f32_e32 v24, v24\n\t" \
  "v_rcp_f32_e32 v25, v25\n\t" \
  "v_mfma_f32_16x16x128_f8f6f4 a[248:251], v[184:191], v[240:247], a[248:251]\n\t" \
  "v_rcp_f32_e32 v26, v26\n\t" \
  "v_rcp_f32_e32 v27, v27\n\t" \
  "v_mfma_f32_16x16x128_f8f6f4 a[244:247], v[184:191], v[232:239], a[244:247]\n\t" \
  "buffer_load_dwordx4 v0, s[36:39], s53 offen offset:3072 sc0 sc1 lds\n\t" \
  "v_mul_f32_e32 v64, v64, v24\n\t" \
  "v_mul_f32_e32 v65, v65, v25\n\t" \
  "v_mul_f32_e32 v66, v66, v26\n\t" \
  "v_mul_f32_e32 v67, v67, v27\n\t" \
  "v_mfma_f32_16x16x128_f8f6f4 a[240:243], v[184:191], v[224:231], a[240:243]\n\t" \
  "s_waitcnt lgkmcnt(0)\n\t" \
  "v_mfma_f32_16x16x128_f8f6f4 a[16:19], v[128:135], v[192:199], a[16:19]\n\t" \
  "s_waitcnt vmcnt(33)\n\t" \
  "s_barrier\n\t" \
  "v_mfma_f32_16x16x128_f8f6f4 a[20:23], v[128:135], v[200:207], a[20:23]\n\t" \
  "ds_read_b128 v[160:163], v2 offset:0\n\t" \
  "ds_read_b128 v[164:167], v2 offset:8192\n\t" \
  "s_xor_b32 s91, s68, 2097152\n\t" \
  "buffer_store_dwordx2 v[64:65], v5, s[40:43], s91 offen nt\n\t" \
  "v_mfma_f32_16x16x128_f8f6f4 a[24:27], v[128:135], v[208:215], a[24:27]\n\t" \
  "ds_read_b128 v[168:171], v2 offset:1024\n\t" \
  "ds_read_b128 v[172:175], v2 offset:9216\n\t" \
  "buffer_store_dwordx2 v[66:67], v5, s[40:43], s91 offen offset:128 nt\n\t" \
  "v_mfma_f32_16x16x128_f8f6f4 a[28:31], v[128:135], v[216:223], a[28:31]\n\t" \
  "ds_read_b128 v[176:179], v2 offset:2048\n\t" \
  "ds_read_b128 v[180:183], v2 offset:10240\n\t" \
  "v_mfma_f32_16x16x128_f8f6f4 a[60:63], v[136:143], v[216:223], a[60:63]\n\t" \
  "ds_read_b128 v[184:187], v2 offset:3072\n\t" \
  "ds_read_b128 v[188:191], v2 offset:11264\n\t" \
  "v_mul_f32_e32 v68, s56, v68\n\t" \
  "v_mul_f32_e32 v69, s56, v69\n\t" \
  "v_mul_f32_e32 v70, s56, v70\n\t" \
  "v_mul_f32_e32 v71, s56, v71\n\t" \
  "v_mfma_f32_16x16x128_f8f6f4 a[56:59], v[136:143], v[208:215], a[56:59]\n\t" \
  "ds_read_b128 v[224:227], v4 offset:49152\n\t" \
  "ds_read_b128 v[228:231], v4 offset:57344\n\t" \
  "v_min_f32_e32 v28, s60, v68\n\t" \
  "v_min_f32_e32 v29, s60, v69\n\t" \
  "v_min_f32_e32 v30, s60, v70\n\t" \
  "v_min_f32_e32 v31, s60, v71\n\t" \
  "v_mfma_f32_16x16x128_f8f6f4 a[52:55], v[136:143], v[200:207], a[52:55]\n\t" \
  "ds_read_b128 v[232:235], v4 offset:50176\n\t" \
  "ds_read_b128 v[236:239], v4 offset:58368\n\t" \
  "s_add_u32 s53, s52, 229376\n\t" \
  "s_add_u32 m0, s48, 32768\n\t" \
  "v_exp_f32_e32 v28, v28\n\t" \
  "v_exp_f32_e32 v29, v29\n\t" \
  "v_mfma_f32_16x16x128_f8f6f4 a[48:51], v[136:143], v[192:199], a[48:51]\n\t" \
  "buffer_load_dwordx4 v0, s[36:39], s53 offen sc0 sc1 lds\n\t" \
  "v_mfma_f32_16x16x128_f8f6f4 a[80:83], v[144:151], v[192:199], a[80:83]\n\t" \
  "buffer_load_dwordx4 v0, s[36:39], s53 offen offset:1024 sc0 sc1 lds\n\t" \
  "v_exp_f32_e32 v30, v30\n\t" \
  "v_exp_f32_e32 v31, v31\n\t" \
  "v_mfma_f32_16x16x128_f8f6f4 a[84:87], v[144:151], v[200:207], a[84:87]\n\t" \
  "buffer_load_dwordx4 v0, s[36:39], s53 offen offset:2048 sc0 sc1 lds\n\t" \
  "v_fma_f32 v28, v28, s58, s58\n\t" \
  "v_fma_f32 v29, v29, s58, s58\n\t" \
  "v_fma_f32 v30, v30, s58, s58\n\t" \
  "v_fma_f32 v31, v31, s58, s58\n\t" \
  "v_mfma_f32_16x16x128_f8f6f4 a[88:91], v[144:151], v[208:215], a[88:91]\n\t" \
  "buffer_load_dwordx4 v0, s[36:39], s53 offen offset:3072 sc0 sc1 lds\n\t" \
  "s_add_u32 s53, s52, 245760\n\t" \
  "s_add_u32 m0, s48, 49152\n\t" \
  "v_rcp_f32_e32 v28, v28\n\t" \
  "v_rcp_f32_e32 v29, v29\n\t" \
  "v_mfma_f32_16x16x128_f8f6f4 a[92:95], v[144:151], v[216:223], a[92:95]\n\t" \
  "buffer_load_dwordx4 v0, s[36:39], s53 offen sc0 sc1 lds\n\t" \
  "v_mfma_f32_16x16x128_f8f6f4 a[124:127], v[152:159], v[216:223], a[124:127]\n\t" \
  "buffer_load_dwordx4 v0, s[36:39], s53 offen offset:1024 sc0 sc1 lds\n\t" \
  "v_rcp_f32_e32 v30, v30\n\t" \
  "v_rcp_f32_e32 v31, v31\n\t" \
  "v_mfma_f32_16x16x128_f8f6f4 a[120:123], v[152:159], v[208:215], a[120:123]\n\t" \
  "buffer_load_dwordx4 v0, s[36:39], s53 offen offset:2048 sc0 sc1 lds\n\t" \
  "v_mul_f32_e32 v68, v68, v28\n\t" \
  "v_mul_f32_e32 v69, v69, v29\n\t" \
  "v_mul_f32_e32 v70, v70, v30\n\t" \
  "v_mul_f32_e32 v71, v71, v31\n\t" \
  "v_mfma_f32_16x16x128_f8f6f4 a[116:119], v[152:159], v[200:207], a[116:119]\n\t" \
  "buffer_load_dwordx4 v0, s[36:39], s53 offen offset:3072 sc0 sc1 lds\n\t" \
  "v_mfma_f32_16x16x128_f8f6f4 a[112:115], v[152:159], v[192:199], a[112:115]\n\t" \
  "s_xor_b32 s91, s68, 2162688\n\t" \
  "buffer_store_dwordx2 v[68:69], v5, s[40:43], s91 offen nt\n\t" \
  "s_waitcnt lgkmcnt(0)\n\t" \
  "v_mfma_f32_16x16x128_f8f6f4 a[144:147], v[160:167], v[192:199], a[144:147]\n\t" \
  "s_waitcnt vmcnt(38)\n\t" \
  "s_barrier\n\t" \
  "v_mfma_f32_16x16x128_f8f6f4 a[148:151], v[160:167], v[200:207], a[148:151]\n\t" \
  "ds_read_b128 v[128:131], v1 offset:0\n\t" \
  "ds_read_b128 v[132:135], v1 offset:8192\n\t" \
  "buffer_store_dwordx2 v[70:71], v5, s[40:43], s91 offen offset:128 nt\n\t" \
  "v_mfma_f32_16x16x128_f8f6f4 a[152:155], v[160:167], v[208:215], a[152:155]\n\t" \
  "ds_read_b128 v[136:139], v1 offset:1024\n\t" \
  "ds_read_b128 v[140:143], v1 offset:9216\n\t" \
  "v_mul_f32_e32 v72, s56, v72\n\t" \
  "v_mul_f32_e32 v73, s56, v73\n\t" \
  "v_mul_f32_e32 v74, s56, v74\n\t" \
  "v_mul_f32_e32 v75, s56, v75\n\t" \
  "v_mfma_f32_16x16x128_f8f6f4 a[156:159], v[160:167], v[216:223], a[156:159]\n\t" \
  "ds_read_b128 v[144:147], v1 offset:2048\n\t" \
  "ds_read_b128 v[148:151], v1 offset:10240\n\t" \
  "s_waitcnt vmcnt(34)\n\t" \
  "v_cmp_le_u32_e32 vcc, s75, v7\n\t" \
  "v_mfma_f32_16x16x128_f8f6f4 a[188:191], v[168:175], v[216:223], a[188:191]\n\t" \
  "ds_read_b128 v[152:155], v1 offset:3072\n\t" \
  "ds_read_b128 v[156:159], v1 offset:11264\n\t" \
  "v_mfma_f32_16x16x128_f8f6f4 a[184:187], v[168:175], v[208:215], a[184:187]\n\t" \
  "ds_read_b128 v[240:243], v4 offset:51200\n\t" \
  "ds_read_b128 v[244:247], v4 offset:59392\n\t" \
  "s_cmp_eq_u64 vcc, -1\n\t" \
  "s_cbranch_scc0 L_SLOWH0_%=\n\t" \
  "L_OKH0_%=:\n\t" \
  "s_add_u32 s76, s76, 1024\n\t" \
  "v_min_f32_e32 v12, s60, v72\n\t" \
  "v_min_f32_e32 v13, s60, v73\n\t" \
  "v_min_f32_e32 v14, s60, v74\n\t" \
  "v_min_f32_e32 v15, s60, v75\n\t" \
  "v_mfma_f32_16x16x128_f8f6f4 a[180:183], v[168:175], v[200:207], a[180:183]\n\t" \
  "ds_read_b128 v[248:251], v4 offset:52224\n\t" \
  "ds_read_b128 v[252:255], v4 offset:60416\n\t" \
  "v_exp_f32_e32 v12, v12\n\t" \
  "v_exp_f32_e32 v13, v13\n\t" \
  "v_mfma_f32_16x16x128_f8f6f4 a[176:179], v[168:175], v[192:199], a[176:179]\n\t" \
  "s_add_u32 s53, s50, 65536\n\t" \
  "s_add_u32 m0, s48, 65536\n\t" \
  "v_exp_f32_e32 v14, v14\n\t" \
  "v_exp_f32_e32 v15, v15\n\t" \
  "v_mfma_f32_16x16x128_f8f6f4 a[208:211], v[176:183], v[192:199], a[208:211]\n\t" \
  "buffer_load_dwordx4 v0, s[36:39], s53 offen sc0 sc1 lds\n\t" \
  "v_mfma_f32_16x16x128_f8f6f4 a[212:215], v[176:183], v[200:207], a[212:215]\n\t" \
  "v_fma_f32 v12, v12, s58, s58\n\t" \
  "v_fma_f32 v13, v13, s58, s58\n\t" \
  "v_fma_f32 v14, v14, s58, s58\n\t" \
  "v_fma_f32 v15, v15, s58, s58\n\t" \
  "v_mfma_f32_16x16x128_f8f6f4 a[216:219], v[176:183], v[208:215], a[216:219]\n\t" \
  "buffer_load_dwordx4 v0, s[36:39], s53 offen offset:1024 sc0 sc1 lds\n\t" \
  "v_rcp_f32_e32 v12, v12\n\t" \
  "v_rcp_f32_e32 v13, v13\n\t" \
  "v_mfma_f32_16x16x128_f8f6f4 a[220:223], v[176:183], v[216:223], a[220:223]\n\t" \
  "v_rcp_f32_e32 v14, v14\n\t" \
  "v_rcp_f32_e32 v15, v15\n\t" \
  "v_mfma_f32_16x16x128_f8f6f4 a[252:255], v[184:191], v[216:223], a[252:255]\n\t" \
  "buffer_load_dwordx4 v0, s[36:39], s53 offen offset:2048 sc0 sc1 lds\n\t" \
  "v_mfma_f32_16x16x128_f8f6f4 a[248:251], v[184:191], v[208:215], a[248:251]\n\t" \
  "v_mul_f32_e32 v72, v72, v12\n\t" \
  "v_mul_f32_e32 v73, v73, v13\n\t" \
  "v_mul_f32_e32 v74, v74, v14\n\t" \
  "v_mul_f32_e32 v75, v75, v15\n\t" \
  "v_mfma_f32_16x16x128_f8f6f4 a[244:247], v[184:191], v[200:207], a[244:247]\n\t" \
  "buffer_load_dwordx4 v0, s[36:39], s53 offen offset:3072 sc0 sc1 lds\n\t" \
  "s_xor_b32 s91, s68, 2228224\n\t" \
  "buffer_store_dwordx2 v[72:73], v5, s[40:43], s91 offen nt\n\t" \
  "v_mfma_f32_16x16x128_f8f6f4 a[240:243], v[184:191], v[192:199], a[240:243]\n\t" \
  "s_waitcnt lgkmcnt(0)\n\t" \
  "v_mfma_f32_16x16x128_f8f6f4 a[16:19], v[128:135], v[224:231], a[16:19]\n\t" \
  "s_waitcnt vmcnt(33)\n\t" \
  "s_barrier\n\t" \
  "v_mfma_f32_16x16x128_f8f6f4 a[20:23], v[128:135], v[232:239], a[20:23]\n\t" \
  "ds_read_b128 v[160:163], v2 offset:16384\n\t" \
  "ds_read_b128 v[164:167], v2 offset:24576\n\t" \
  "buffer_store_dwordx2 v[74:75], v5, s[40:43], s91 offen offset:128 nt\n\t" \
  "v_mfma_f32_16x16x128_f8f6f4 a[24:27], v[128:135], v[240:247], a[24:27]\n\t" \
  "ds_read_b128 v[168:171], v2 offset:17408\n\t" \
  "ds_read_b128 v[172:175], v2 offset:25600\n\t" \
  "v_mul_f32_e32 v76, s56, v76\n\t" \
  "v_mul_f32_e32 v77, s56, v77\n\t" \
  "v_mul_f32_e32 v78, s56, v78\n\t" \
  "v_mul_f32_e32 v79, s56, v79\n\t" \
  "v_mfma_f32_16x16x128_f8f6f4 a[28:31], v[128:135], v[248:255], a[28:31]\n\t" \
  "ds_read_b128 v[176:179], v2 offset:18432\n\t" \
  "ds_read_b128 v[180:183], v2 offset:26624\n\t" \
  "v_min_f32_e32 v16, s60, v76\n\t" \
  "v_min_f32_e32 v17, s60, v77\n\t" \
  "v_min_f32_e32 v18, s60, v78\n\t" \
  "v_min_f32_e32 v19, s60, v79\n\t" \
  "v_mfma_f32_16x16x128_f8f6f4 a[60:63], v[136:143], v[248:255], a[60:63]\n\t" \
  "ds_read_b128 v[184:187], v2 offset:19456\n\t" \
  "ds_read_b128 v[188:191], v2 offset:27648\n\t" \
  "v_mfma_f32_16x16x128_f8f6f4 a[56:59], v[136:143], v[240:247], a[56:59]\n\t" \
  "ds_read_b128 v[192:195], v3 offset:16384\n\t" \
  "ds_read_b128 v[196:199], v3 offset:24576\n\t" \
  "v_exp_f32_e32 v16, v16\n\t" \
  "v_exp_f32_e32 v17, v17\n\t" \
  "v_mfma_f32_16x16x128_f8f6f4 a[52:55], v[136:143], v[232:239], a[52:55]\n\t" \
  "ds_read_b128 v[200:203], v3 offset:17408\n\t" \
  "ds_read_b128 v[204:207], v3 offset:25600\n\t" \
  "s_add_u32 s53, s52, 262144\n\t" \
  "s_mov_b32 m0, s48\n\t" \
  "v_exp_f32_e32 v18, v18\n\t" \
  "v_exp_f32_e32 v19, v19\n\t" \
  "v_mfma_f32_16x16x128_f8f6f4 a[48:51], v[136:143], v[224:231], a[48:51]\n\t" \
  "buffer_load_dwordx4 v0, s[36:39], s53 offen sc0 sc1 lds\n\t" \
  "v_fma_f32 v16, v16, s58, s58\n\t" \
  "v_fma_f32 v17, v17, s58, s58\n\t" \
  "v_fma_f32 v18, v18, s58, s58\n\t" \
  "v_fma_f32 v19, v19, s58, s58\n\t" \
  "v_mfma_f32_16x16x128_f8f6f4 a[80:83], v[144:151], v[224:231], a[80:83]\n\t" \
  "buffer_load_dwordx4 v0, s[36:39], s53 offen offset:1024 sc0 sc1 lds\n\t" \
  "v_mfma_f32_16x16x128_f8f6f4 a[84:87], v[144:151], v[232:239], a[84:87]\n\t" \
  "buffer_load_dwordx4 v0, s[36:39], s53 offen offset:2048 sc0 sc1 lds\n\t" \
  "v_rcp_f32_e32 v16, v16\n\t" \
  "v_rcp_f32_e32 v17, v17\n\t" \
  "v_mfma_f32_16x16x128_f8f6f4 a[88:91], v[144:151], v[240:247], a[88:91]\n\t" \
  "buffer_load_dwordx4 v0, s[36:39], s53 offen offset:3072 sc0 sc1 lds\n\t" \
  "s_add_u32 s53, s52, 278528\n\t" \
  "s_add_u32 m0, s48, 114688\n\t" \
  "v_rcp_f32_e32 v18, v18\n\t" \
  "v_rcp_f32_e32 v19, v19\n\t" \
  "v_mfma_f32_16x16x128_f8f6f4 a[92:95], v[144:151], v[248:255], a[92:95]\n\t" \
  "buffer_load_dwordx4 v0, s[36:39], s53 offen sc0 sc1 lds\n\t" \
  "v_mfma_f32_16x16x128_f8f6f4 a[124:127], v[152:159], v[248:255], a[124:127]\n\t" \
  "buffer_load_dwordx4 v0, s[36:39], s53 offen offset:1024 sc0 sc1 lds\n\t" \
  "v_mul_f32_e32 v76, v76, v16\n\t" \
  "v_mul_f32_e32 v77, v77, v17\n\t" \
  "v_mul_f32_e32 v78, v78, v18\n\t" \
  "v_mul_f32_e32 v79, v79, v19\n\t" \
  "v_mfma_f32_16x16x128_f8f6f4 a[120:123], v[152:159], v[240:247], a[120:123]\n\t" \
  "buffer_load_dwordx4 v0, s[36:39], s53 offen offset:2048 sc0 sc1 lds\n\t" \
  "s_xor_b32 s91, s68, 2293760\n\t" \
  "buffer_store_dwordx2 v[76:77], v5, s[40:43], s91 offen nt\n\t" \
  "v_mfma_f32_16x16x128_f8f6f4 a[116:119], v[152:159], v[232:239], a[116:119]\n\t" \
  "buffer_load_dwordx4 v0, s[36:39], s53 offen offset:3072 sc0 sc1 lds\n\t" \
  "buffer_store_dwordx2 v[78:79], v5, s[40:43], s91 offen offset:128 nt\n\t" \
  "v_mfma_f32_16x16x128_f8f6f4 a[112:115], v[152:159], v[224:231], a[112:115]\n\t" \
  "s_waitcnt lgkmcnt(0)\n\t" \
  "v_mfma_f32_16x16x128_f8f6f4 a[144:147], v[160:167], v[224:231], a[144:147]\n\t" \
  "s_waitcnt vmcnt(38)\n\t" \
  "s_barrier\n\t" \
  "v_mfma_f32_16x16x128_f8f6f4 a[148:151], v[160:167], v[232:239], a[148:151]\n\t" \
  "ds_read_b128 v[128:131], v6 offset:0\n\t" \
  "ds_read_b128 v[132:135], v6 offset:8192\n\t" \
  "v_mul_f32_e32 v80, s56, v80\n\t" \
  "v_mul_f32_e32 v81, s56, v81\n\t" \
  "v_mul_f32_e32 v82, s56, v82\n\t" \
  "v_mul_f32_e32 v83, s56, v83\n\t" \
  "v_mfma_f32_16x16x128_f8f6f4 a[152:155], v[160:167], v[240:247], a[152:155]\n\t" \
  "ds_read_b128 v[136:139], v6 offset:1024\n\t" \
  "ds_read_b128 v[140:143], v6 offset:9216\n\t" \
  "v_min_f32_e32 v20, s60, v80\n\t" \
  "v_min_f32_e32 v21, s60, v81\n\t" \
  "v_min_f32_e32 v22, s60, v82\n\t" \
  "v_min_f32_e32 v23, s60, v83\n\t" \
  "v_mfma_f32_16x16x128_f8f6f4 a[156:159], v[160:167], v[248:255], a[156:159]\n\t" \
  "ds_read_b128 v[144:147], v6 offset:2048\n\t" \
  "ds_read_b128 v[148:151], v6 offset:10240\n\t" \
  "v_exp_f32_e32 v20, v20\n\t" \
  "v_exp_f32_e32 v21, v21\n\t" \
  "v_mfma_f32_16x16x128_f8f6f4 a[188:191], v[168:175], v[248:255], a[188:191]\n\t" \
  "ds_read_b128 v[152:155], v6 offset:3072\n\t" \
  "ds_read_b128 v[156:159], v6 offset:11264\n\t" \
  "v_mfma_f32_16x16x128_f8f6f4 a[184:187], v[168:175], v[240:247], a[184:187]\n\t" \
  "ds_read_b128 v[208:211], v3 offset:18432\n\t" \
  "ds_read_b128 v[212:215], v3 offset:26624\n\t" \
  "v_exp_f32_e32 v22, v22\n\t" \
  "v_exp_f32_e32 v23, v23\n\t" \
  "v_mfma_f32_16x16x128_f8f6f4 a[180:183], v[168:175], v[232:239], a[180:183]\n\t" \
  "ds_read_b128 v[216:219], v3 offset:19456\n\t" \
  "ds_read_b128 v[220:223], v3 offset:27648\n\t" \
  "v_fma_f32 v20, v20, s58, s58\n\t" \
  "v_fma_f32 v21, v21, s58, s58\n\t" \
  "v_fma_f32 v22, v22, s58, s58\n\t" \
  "v_fma_f32 v23, v23, s58, s58\n\t" \
  "v_mfma_f32_16x16x128_f8f6f4 a[176:179], v[168:175], v[224:231], a[176:179]\n\t" \
  "s_add_u32 s53, s50, 81920\n\t" \
  "s_add_u32 m0, s48, 81920\n\t" \
  "v_rcp_f32_e32 v20, v20\n\t" \
  "v_rcp_f32_e32 v21, v21\n\t" \
  "v_mfma_f32_16x16x128_f8f6f4 a[208:211], v[176:183], v[224:231], a[208:211]\n\t" \
  "buffer_load_dwordx4 v0, s[36:39], s53 offen sc0 sc1 lds\n\t" \
  "v_mfma_f32_16x16x128_f8f6f4 a[212:215], v[176:183], v[232:239], a[212:215]\n\t" \
  "v_rcp_f32_e32 v22, v22\n\t" \
  "v_rcp_f32_e32 v23, v23\n\t" \
  "v_mfma_f32_16x16x128_f8f6f4 a[216:219], v[176:183], v[240:247], a[216:219]\n\t" \
  "buffer_load_dwordx4 v0, s[36:39], s53 offen offset:1024 sc0 sc1 lds\n\t" \
  "v_mul_f32_e32 v80, v80, v20\n\t" \
  "v_mul_f32_e32 v81, v81, v21\n\t" \
  "v_mul_f32_e32 v82, v82, v22\n\t" \
  "v_mul_f32_e32 v83, v83, v23\n\t" \
  "v_mfma_f32_16x16x128_f8f6f4 a[220:223], v[176:183], v[248:255], a[220:223]\n\t" \
  "v_mfma_f32_16x16x128_f8f6f4 a[252:255], v[184:191], v[248:255], a[252:255]\n\t" \
  "buffer_load_dwordx4 v0, s[36:39], s53 offen offset:2048 sc0 sc1 lds\n\t" \
  "s_xor_b32 s91, s68, 3145728\n\t" \
  "buffer_store_dwordx2 v[80:81], v5, s[40:43], s91 offen nt\n\t" \
  "v_mfma_f32_16x16x128_f8f6f4 a[248:251], v[184:191], v[240:247], a[248:251]\n\t" \
  "buffer_store_dwordx2 v[82:83], v5, s[40:43], s91 offen offset:128 nt\n\t" \
  "v_mfma_f32_16x16x128_f8f6f4 a[244:247], v[184:191], v[232:239], a[244:247]\n\t" \
  "buffer_load_dwordx4 v0, s[36:39], s53 offen offset:3072 sc0 sc1 lds\n\t" \
  "v_mul_f32_e32 v84, s56, v84\n\t" \
  "v_mul_f32_e32 v85, s56, v85\n\t" \
  "v_mul_f32_e32 v86, s56, v86\n\t" \
  "v_mul_f32_e32 v87, s56, v87\n\t" \
  "v_mfma_f32_16x16x128_f8f6f4 a[240:243], v[184:191], v[224:231], a[240:243]\n\t" \
  "s_waitcnt lgkmcnt(0)\n\t" \
  "v_mfma_f32_16x16x128_f8f6f4 a[16:19], v[128:135], v[192:199], a[16:19]\n\t" \
  "s_waitcnt vmcnt(34)\n\t" \
  "s_barrier\n\t" \
  "v_mfma_f32_16x16x128_f8f6f4 a[20:23], v[128:135], v[200:207], a[20:23]\n\t" \
  "ds_read_b128 v[160:163], v2 offset:32768\n\t" \
  "ds_read_b128 v[164:167], v2 offset:40960\n\t" \
  "v_min_f32_e32 v24, s60, v84\n\t" \
  "v_min_f32_e32 v25, s60, v85\n\t" \
  "v_min_f32_e32 v26, s60, v86\n\t" \
  "v_min_f32_e32 v27, s60, v87\n\t" \
  "v_mfma_f32_16x16x128_f8f6f4 a[24:27], v[128:135], v[208:215], a[24:27]\n\t" \
  "ds_read_b128 v[168:171], v2 offset:33792\n\t" \
  "ds_read_b128 v[172:175], v2 offset:41984\n\t" \
  "v_exp_f32_e32 v24, v24\n\t" \
  "v_exp_f32_e32 v25, v25\n\t" \
  "v_mfma_f32_16x16x128_f8f6f4 a[28:31], v[128:135], v[216:223], a[28:31]\n\t" \
  "ds_read_b128 v[176:179], v2 offset:34816\n\t" \
  "ds_read_b128 v[180:183], v2 offset:43008\n\t" \
  "v_exp_f32_e32 v26, v26\n\t" \
  "v_exp_f32_e32 v27, v27\n\t" \
  "v_mfma_f32_16x16x128_f8f6f4 a[60:63], v[136:143], v[216:223], a[60:63]\n\t" \
  "ds_read_b128 v[184:187], v2 offset:35840\n\t" \
  "ds_read_b128 v[188:191], v2 offset:44032\n\t" \
  "v_mfma_f32_16x16x128_f8f6f4 a[56:59], v[136:143], v[208:215], a[56:59]\n\t" \
  "ds_read_b128 v[224:227], v127 offset:16384\n\t" \
  "ds_read_b128 v[228:231], v127 offset:24576\n\t" \
  "v_fma_f32 v24, v24, s58, s58\n\t" \
  "v_fma_f32 v25, v25, s58, s58\n\t" \
  "v_fma_f32 v26, v26, s58, s58\n\t" \
  "v_fma_f32 v27, v27, s58, s58\n\t" \
  "v_mfma_f32_16x16x128_f8f6f4 a[52:55], v[136:143], v[200:207], a[52:55]\n\t" \
  "ds_read_b128 v[232:235], v127 offset:17408\n\t" \
  "ds_read_b128 v[236:239], v127 offset:25600\n\t" \
  "s_add_u32 s53, s52, 294912\n\t" \
  "s_add_u32 m0, s48, 16384\n\t" \
  "v_rcp_f32_e32 v24, v24\n\t" \
  "v_rcp_f32_e32 v25, v25\n\t" \
  "v_mfma_f32_16x16x128_f8f6f4 a[48:51], v[136:143], v[192:199], a[48:51]\n\t" \
  "buffer_load_dwordx4 v0, s[36:39], s53 offen sc0 sc1 lds\n\t" \
  "v_mfma_f32_16x16x128_f8f6f4 a[80:83], v[144:151], v[192:199], a[80:83]\n\t" \
  "buffer_load_dwordx4 v0, s[36:39], s53 offen offset:1024 sc0 sc1 lds\n\t" \
  "v_rcp_f32_e32 v26, v26\n\t" \
  "v_rcp_f32_e32 v27, v27\n\t" \
  "v_mfma_f32_16x16x128_f8f6f4 a[84:87], v[144:151], v[200:207], a[84:87]\n\t" \
  "buffer_load_dwordx4 v0, s[36:39], s53 offen offset:2048 sc0 sc1 lds\n\t" \
  "v_mul_f32_e32 v84, v84, v24\n\t" \
  "v_mul_f32_e32 v85, v85, v25\n\t" \
  "v_mul_f32_e32 v86, v86, v26\n\t" \
  "v_mul_f32_e32 v87, v87, v27\n\t" \
  "v_mfma_f32_16x16x128_f8f6f4 a[88:91], v[144:151], v[208:215], a[88:91]\n\t" \
  "buffer_load_dwordx4 v0, s[36:39], s53 offen offset:3072 sc0 sc1 lds\n\t" \
  "s_add_u32 s53, s52, 311296\n\t" \
  "s_add_u32 m0, s48, 131072\n\t" \
  "s_xor_b32 s91, s68, 3211264\n\t" \
  "buffer_store_dwordx2 v[84:85], v5, s[40:43], s91 offen nt\n\t" \
  "v_mfma_f32_16x16x128_f8f6f4 a[92:95], v[144:151], v[216:223], a[92:95]\n\t" \
  "buffer_load_dwordx4 v0, s[36:39], s53 offen sc0 sc1 lds\n\t" \
  "v_mfma_f32_16x16x128_f8f6f4 a[124:127], v[152:159], v[216:223], a[124:127]\n\t" \
  "buffer_load_dwordx4 v0, s[36:39], s53 offen offset:1024 sc0 sc1 lds\n\t" \
  "buffer_store_dwordx2 v[86:87], v5, s[40:43], s91 offen offset:128 nt\n\t" \
  "v_mfma_f32_16x16x128_f8f6f4 a[120:123], v[152:159], v[208:215], a[120:123]\n\t" \
  "buffer_load_dwordx4 v0, s[36:39], s53 offen offset:2048 sc0 sc1 lds\n\t" \
  "v_mul_f32_e32 v88, s56, v88\n\t" \
  "v_mul_f32_e32 v89, s56, v89\n\t" \
  "v_mul_f32_e32 v90, s56, v90\n\t" \
  "v_mul_f32_e32 v91, s56, v91\n\t" \
  "v_mfma_f32_16x16x128_f8f6f4 a[116:119], v[152:159], v[200:207], a[116:119]\n\t" \
  "buffer_load_dwordx4 v0, s[36:39], s53 offen offset:3072 sc0 sc1 lds\n\t" \
  "v_min_f32_e32 v28, s60, v88\n\t" \
  "v_min_f32_e32 v29, s60, v89\n\t" \
  "v_min_f32_e32 v30, s60, v90\n\t" \
  "v_min_f32_e32 v31, s60, v91\n\t" \
  "v_mfma_f32_16x16x128_f8f6f4 a[112:115], v[152:159], v[192:199], a[112:115]\n\t" \
  "buffer_load_dword v7, v8, s[44:47], s76 offen sc0 sc1\n\t" \
  "s_waitcnt lgkmcnt(0)\n\t" \
  "v_mfma_f32_16x16x128_f8f6f4 a[144:147], v[160:167], v[192:199], a[144:147]\n\t" \
  "s_waitcnt vmcnt(39)\n\t" \
  "s_barrier\n\t" \
  "v_mfma_f32_16x16x128_f8f6f4 a[148:151], v[160:167], v[200:207], a[148:151]\n\t" \
  "ds_read_b128 v[128:131], v1 offset:32768\n\t" \
  "ds_read_b128 v[132:135], v1 offset:40960\n\t" \
  "v_exp_f32_e32 v28, v28\n\t" \
  "v_exp_f32_e32 v29, v29\n\t" \
  "v_mfma_f32_16x16x128_f8f6f4 a[152:155], v[160:167], v[208:215], a[152:155]\n\t" \
  "ds_read_b128 v[136:139], v1 offset:33792\n\t" \
  "ds_read_b128 v[140:143], v1 offset:41984\n\t" \
  "v_exp_f32_e32 v30, v30\n\t" \
  "v_exp_f32_e32 v31, v31\n\t" \
  "v_mfma_f32_16x16x128_f8f6f4 a[156:159], v[160:167], v[216:223], a[156:159]\n\t" \
  "ds_read_b128 v[144:147], v1 offset:34816\n\t" \
  "ds_read_b128 v[148:151], v1 offset:43008\n\t" \
  "v_fma_f32 v28, v28, s58, s58\n\t" \
  "v_fma_f32 v29, v29, s58, s58\n\t" \
  "v_fma_f32 v30, v30, s58, s58\n\t" \
  "v_fma_f32 v31, v31, s58, s58\n\t" \
  "v_mfma_f32_16x16x128_f8f6f4 a[188:191], v[168:175], v[216:223], a[188:191]\n\t" \
  "ds_read_b128 v[152:155], v1 offset:35840\n\t" \
  "ds_read_b128 v[156:159], v1 offset:44032\n\t" \
  "v_mfma_f32_16x16x128_f8f6f4 a[184:187], v[168:175], v[208:215], a[184:187]\n\t" \
  "ds_read_b128 v[240:243], v127 offset:18432\n\t" \
  "ds_read_b128 v[244:247], v127 offset:26624\n\t" \
  "v_rcp_f32_e32 v28, v28\n\t" \
  "v_rcp_f32_e32 v29, v29\n\t" \
  "v_mfma_f32_16x16x128_f8f6f4 a[180:183], v[168:175], v[200:207], a[180:183]\n\t" \
  "ds_read_b128 v[248:251], v127 offset:19456\n\t" \
  "ds_read_b128 v[252:255], v127 offset:27648\n\t" \
  "v_rcp_f32_e32 v30, v30\n\t" \
  "v_rcp_f32_e32 v31, v31\n\t" \
  "v_mfma_f32_16x16x128_f8f6f4 a[176:179], v[168:175], v[192:199], a[176:179]\n\t" \
  "s_add_u32 s53, s50, 98304\n\t" \
  "s_add_u32 m0, s48, 98304\n\t" \
  "v_mfma_f32_16x16x128_f8f6f4 a[208:211], v[176:183], v[192:199], a[208:211]\n\t" \
  "buffer_load_dwordx4 v0, s[36:39], s53 offen sc0 sc1 lds\n\t" \
  "v_mul_f32_e32 v88, v88, v28\n\t" \
  "v_mul_f32_e32 v89, v89, v29\n\t" \
  "v_mul_f32_e32 v90, v90, v30\n\t" \
  "v_mul_f32_e32 v91, v91, v31\n\t" \
  "v_mfma_f32_16x16x128_f8f6f4 a[212:215], v[176:183], v[200:207], a[212:215]\n\t" \
  "s_xor_b32 s91, s68, 3276800\n\t" \
  "buffer_store_dwordx2 v[88:89], v5, s[40:43], s91 offen nt\n\t" \
  "v_mfma_f32_16x16x128_f8f6f4 a[216:219], v[176:183], v[208:215], a[216:219]\n\t" \
  "buffer_load_dwordx4 v0, s[36:39], s53 offen offset:1024 sc0 sc1 lds\n\t" \
  "buffer_store_dwordx2 v[90:91], v5, s[40:43], s91 offen offset:128 nt\n\t" \
  "v_mfma_f32_16x16x128_f8f6f4 a[220:223], v[176:183], v[216:223], a[220:223]\n\t" \
  "v_mfma_f32_16x16x128_f8f6f4 a[252:255], v[184:191], v[216:223], a[252:255]\n\t" \
  "buffer_load_dwordx4 v0, s[36:39], s53 offen offset:2048 sc0 sc1 lds\n\t" \
  "v_mul_f32_e32 v92, s56, v92\n\t" \
  "v_mul_f32_e32 v93, s56, v93\n\t" \
  "v_mul_f32_e32 v94, s56, v94\n\t" \
  "v_mul_f32_e32 v95, s56, v95\n\t" \
  "v_mfma_f32_16x16x128_f8f6f4 a[248:251], v[184:191], v[208:215], a[248:251]\n\t" \
  "v_min_f32_e32 v12, s60, v92\n\t" \
  "v_min_f32_e32 v13, s60, v93\n\t" \
  "v_min_f32_e32 v14, s60, v94\n\t" \
  "v_min_f32_e32 v15, s60, v95\n\t" \
  "v_mfma_f32_16x16x128_f8f6f4 a[244:247], v[184:191], v[200:207], a[244:247]\n\t" \
  "buffer_load_dwordx4 v0, s[36:39], s53 offen offset:3072 sc0 sc1 lds\n\t" \
  "v_exp_f32_e32 v12, v12\n\t" \
  "v_exp_f32_e32 v13, v13\n\t" \
  "v_mfma_f32_16x16x128_f8f6f4 a[240:243], v[184:191], v[192:199], a[240:243]\n\t" \
  "s_waitcnt lgkmcnt(0)\n\t" \
  "v_mfma_f32_16x16x128_f8f6f4 a[16:19], v[128:135], v[224:231], a[16:19]\n\t" \
  "s_waitcnt vmcnt(35)\n\t" \
  "s_barrier\n\t" \
  "v_mfma_f32_16x16x128_f8f6f4 a[20:23], v[128:135], v[232:239], a[20:23]\n\t" \
  "ds_read_b128 v[160:163], v1 offset:49152\n\t" \
  "ds_read_b128 v[164:167], v1 offset:57344\n\t" \
  "v_exp_f32_e32 v14, v14\n\t" \
  "v_exp_f32_e32 v15, v15\n\t" \
  "v_mfma_f32_16x16x128_f8f6f4 a[24:27], v[128:135], v[240:247], a[24:27]\n\t" \
  "ds_read_b128 v[168:171], v1 offset:50176\n\t" \
  "ds_read_b128 v[172:175], v1 offset:58368\n\t" \
  "v_fma_f32 v12, v12, s58, s58\n\t" \
  "v_fma_f32 v13, v13, s58, s58\n\t" \
  "v_fma_f32 v14, v14, s58, s58\n\t" \
  "v_fma_f32 v15, v15, s58, s58\n\t" \
  "v_mfma_f32_16x16x128_f8f6f4 a[28:31], v[128:135], v[248:255], a[28:31]\n\t" \
  "ds_read_b128 v[176:179], v1 offset:51200\n\t" \
  "ds_read_b128 v[180:183], v1 offset:59392\n\t" \
  "v_mfma_f32_16x16x128_f8f6f4 a[60:63], v[136:143], v[248:255], a[60:63]\n\t" \
  "ds_read_b128 v[184:187], v1 offset:52224\n\t" \
  "ds_read_b128 v[188:191], v1 offset:60416\n\t" \
  "v_rcp_f32_e32 v12, v12\n\t" \
  "v_rcp_f32_e32 v13, v13\n\t" \
  "v_mfma_f32_16x16x128_f8f6f4 a[56:59], v[136:143], v[240:247], a[56:59]\n\t" \
  "ds_read_b128 v[192:195], v4 offset:0\n\t" \
  "ds_read_b128 v[196:199], v4 offset:8192\n\t" \
  "v_rcp_f32_e32 v14, v14\n\t" \
  "v_rcp_f32_e32 v15, v15\n\t" \
  "v_mfma_f32_16x16x128_f8f6f4 a[52:55], v[136:143], v[232:239], a[52:55]\n\t" \
  "ds_read_b128 v[200:203], v4 offset:1024\n\t" \
  "ds_read_b128 v[204:207], v4 offset:9216\n\t" \
  "s_add_u32 s53, s52, 327680\n\t" \
  "s_add_u32 m0, s48, 147456\n\t" \
  "v_mul_f32_e32 v92, v92, v12\n\t" \
  "v_mul_f32_e32 v93, v93, v13\n\t" \
  "v_mul_f32_e32 v94, v94, v14\n\t" \
  "v_mul_f32_e32 v95, v95, v15\n\t" \
  "v_mfma_f32_16x16x128_f8f6f4 a[48:51], v[136:143], v[224:231], a[48:51]\n\t" \
  "buffer_load_dwordx4 v0, s[36:39], s53 offen sc0 sc1 lds\n\t" \
  "v_mfma_f32_16x16x128_f8f6f4 a[80:83], v[144:151], v[224:231], a[80:83]\n\t" \
  "buffer_load_dwordx4 v0, s[36:39], s53 offen offset:1024 sc0 sc1 lds\n\t" \
  "s_xor_b32 s91, s68, 3342336\n\t" \
  "buffer_store_dwordx2 v[92:93], v5, s[40:43], s91 offen nt\n\t" \
  "v_mfma_f32_16x16x128_f8f6f4 a[84:87], v[144:151], v[232:239], a[84:87]\n\t" \
  "buffer_load_dwordx4 v0, s[36:39], s53 offen offset:2048 sc0 sc1 lds\n\t" \
  "buffer_store_dwordx2 v[94:95], v5, s[40:43], s91 offen offset:128 nt\n\t" \
  "v_mfma_f32_16x16x128_f8f6f4 a[88:91], v[144:151], v[240:247], a[88:91]\n\t" \
  "buffer_load_dwordx4 v0, s[36:39], s53 offen offset:3072 sc0 sc1 lds\n\t" \
  "s_add_u32 s53, s52, 344064\n\t" \
  "s_add_u32 m0, s48, 32768\n\t" \
  "v_mul_f32_e32 v96, s56, v96\n\t" \
  "v_mul_f32_e32 v97, s56, v97\n\t" \
  "v_mul_f32_e32 v98, s56, v98\n\t" \
  "v_mul_f32_e32 v99, s56, v99\n\t" \
  "v_mfma_f32_16x16x128_f8f6f4 a[92:95], v[144:151], v[248:255], a[92:95]\n\t" \
  "buffer_load_dwordx4 v0, s[36:39], s53 offen sc0 sc1 lds\n\t" \
  "v_mfma_f32_16x16x128_f8f6f4 a[124:127], v[152:159], v[248:255], a[124:127]\n\t" \
  "buffer_load_dwordx4 v0, s[36:39], s53 offen offset:1024 sc0 sc1 lds\n\t" \
  "v_min_f32_e32 v16, s60, v96\n\t" \
  "v_min_f32_e32 v17, s60, v97\n\t" \
  "v_min_f32_e32 v18, s60, v98\n\t" \
  "v_min_f32_e32 v19, s60, v99\n\t" \
  "v_mfma_f32_16x16x128_f8f6f4 a[120:123], v[152:159], v[240:247], a[120:123]\n\t" \
  "buffer_load_dwordx4 v0, s[36:39], s53 offen offset:2048 sc0 sc1 lds\n\t" \
  "v_exp_f32_e32 v16, v16\n\t" \
  "v_exp_f32_e32 v17, v17\n\t" \
  "v_mfma_f32_16x16x128_f8f6f4 a[116:119], v[152:159], v[232:239], a[116:119]\n\t" \
  "buffer_load_dwordx4 v0, s[36:39], s53 offen offset:3072 sc0 sc1 lds\n\t" \
  "v_exp_f32_e32 v18, v18\n\t" \
  "v_exp_f32_e32 v19, v19\n\t" \
  "v_mfma_f32_16x16x128_f8f6f4 a[112:115], v[152:159], v[224:231], a[112:115]\n\t" \
  "s_waitcnt lgkmcnt(0)\n\t" \
  "v_mfma_f32_16x16x128_f8f6f4 a[144:147], v[160:167], v[224:231], a[144:147]\n\t" \
  "s_waitcnt vmcnt(39)\n\t" \
  "s_barrier\n\t" \
  "v_mfma_f32_16x16x128_f8f6f4 a[148:151], v[160:167], v[232:239], a[148:151]\n\t" \
  "ds_read_b128 v[128:131], v1 offset:0\n\t" \
  "ds_read_b128 v[132:135], v1 offset:8192\n\t" \
  "v_fma_f32 v16, v16, s58, s58\n\t" \
  "v_fma_f32 v17, v17, s58, s58\n\t" \
  "v_fma_f32 v18, v18, s58, s58\n\t" \
  "v_fma_f32 v19, v19, s58, s58\n\t" \
  "v_mfma_f32_16x16x128_f8f6f4 a[152:155], v[160:167], v[240:247], a[152:155]\n\t" \
  "ds_read_b128 v[136:139], v1 offset:1024\n\t" \
  "ds_read_b128 v[140:143], v1 offset:9216\n\t" \
  "v_rcp_f32_e32 v16, v16\n\t" \
  "v_rcp_f32_e32 v17, v17\n\t" \
  "v_mfma_f32_16x16x128_f8f6f4 a[156:159], v[160:167], v[248:255], a[156:159]\n\t" \
  "ds_read_b128 v[144:147], v1 offset:2048\n\t" \
  "ds_read_b128 v[148:151], v1 offset:10240\n\t" \
  "v_mfma_f32_16x16x128_f8f6f4 a[188:191], v[168:175], v[248:255], a[188:191]\n\t" \
  "ds_read_b128 v[152:155], v1 offset:3072\n\t" \
  "ds_read_b128 v[156:159], v1 offset:11264\n\t" \
  "v_rcp_f32_e32 v18, v18\n\t" \
  "v_rcp_f32_e32 v19, v19\n\t" \
  "v_mfma_f32_16x16x128_f8f6f4 a[184:187], v[168:175], v[240:247], a[184:187]\n\t" \
  "ds_read_b128 v[208:211], v4 offset:2048\n\t" \
  "ds_read_b128 v[212:215], v4 offset:10240\n\t" \
  "v_mul_f32_e32 v96, v96, v16\n\t" \
  "v_mul_f32_e32 v97, v97, v17\n\t" \
  "v_mul_f32_e32 v98, v98, v18\n\t" \
  "v_mul_f32_e32 v99, v99, v19\n\t" \
  "v_mfma_f32_16x16x128_f8f6f4 a[180:183], v[168:175], v[232:239], a[180:183]\n\t" \
  "ds_read_b128 v[216:219], v4 offset:3072\n\t" \
  "ds_read_b128 v[220:223], v4 offset:11264\n\t" \
  "s_xor_b32 s91, s68, 4194304\n\t" \
  "buffer_store_dwordx2 v[96:97], v5, s[40:43], s91 offen nt\n\t" \
  "v_mfma_f32_16x16x128_f8f6f4 a[176:179], v[168:175], v[224:231], a[176:179]\n\t" \
  "s_add_u32 s53, s50, 114688\n\t" \
  "s_add_u32 m0, s48, 49152\n\t" \
  "v_mfma_f32_16x16x128_f8f6f4 a[208:211], v[176:183], v[224:231], a[208:211]\n\t" \
  "buffer_load_dwordx4 v0, s[36:39], s53 offen sc0 sc1 lds\n\t" \
  "buffer_store_dwordx2 v[98:99], v5, s[40:43], s91 offen offset:128 nt\n\t" \
  "v_mfma_f32_16x16x128_f8f6f4 a[212:215], v[176:183], v[232:239], a[212:215]\n\t" \
  "v_mul_f32_e32 v100, s56, v100\n\t" \
  "v_mul_f32_e32 v101, s56, v101\n\t" \
  "v_mul_f32_e32 v102, s56, v102\n\t" \
  "v_mul_f32_e32 v103, s56, v103\n\t" \
  "v_mfma_f32_16x16x128_f8f6f4 a[216:219], v[176:183], v[240:247], a[216:219]\n\t" \
  "buffer_load_dwordx4 v0, s[36:39], s53 offen offset:1024 sc0 sc1 lds\n\t" \
  "v_min_f32_e32 v20, s60, v100\n\t" \
  "v_min_f32_e32 v21, s60, v101\n\t" \
  "v_min_f32_e32 v22, s60, v102\n\t" \
  "v_min_f32_e32 v23, s60, v103\n\t" \
  "v_mfma_f32_16x16x128_f8f6f4 a[220:223], v[176:183], v[248:255], a[220:223]\n\t" \
  "v_mfma_f32_16x16x128_f8f6f4 a[252:255], v[184:191], v[248:255], a[252:255]\n\t" \
  "buffer_load_dwordx4 v0, s[36:39], s53 offen offset:2048 sc0 sc1 lds\n\t" \
  "v_exp_f32_e32 v20, v20\n\t" \
  "v_exp_f32_e32 v21, v21\n\t" \
  "v_mfma_f32_16x16x128_f8f6f4 a[248:251], v[184:191], v[240:247], a[248:251]\n\t" \
  "v_exp_f32_e32 v22, v22\n\t" \
  "v_exp_f32_e32 v23, v23\n\t" \
  "v_mfma_f32_16x16x128_f8f6f4 a[244:247], v[184:191], v[232:239], a[244:247]\n\t" \
  "buffer_load_dwordx4 v0, s[36:39], s53 offen offset:3072 sc0 sc1 lds\n\t" \
  "v_fma_f32 v20, v20, s58, s58\n\t" \
  "v_fma_f32 v21, v21, s58, s58\n\t" \
  "v_fma_f32 v22, v22, s58, s58\n\t" \
  "v_fma_f32 v23, v23, s58, s58\n\t" \
  "v_mfma_f32_16x16x128_f8f6f4 a[240:243], v[184:191], v[224:231], a[240:243]\n\t" \
  "s_waitcnt lgkmcnt(0)\n\t" \
  "v_mfma_f32_16x16x128_f8f6f4 a[16:19], v[128:135], v[192:199], a[16:19]\n\t" \
  "s_waitcnt vmcnt(33)\n\t" \
  "s_barrier\n\t" \
  "v_mfma_f32_16x16x128_f8f6f4 a[20:23], v[128:135], v[200:207], a[20:23]\n\t" \
  "ds_read_b128 v[160:163], v2 offset:49152\n\t" \
  "ds_read_b128 v[164:167], v2 offset:57344\n\t" \
  "v_rcp_f32_e32 v20, v20\n\t" \
  "v_rcp_f32_e32 v21, v21\n\t" \
  "v_mfma_f32_16x16x128_f8f6f4 a[24:27], v[128:135], v[208:215], a[24:27]\n\t" \
  "ds_read_b128 v[168:171], v2 offset:50176\n\t" \
  "ds_read_b128 v[172:175], v2 offset:58368\n\t" \
  "v_rcp_f32_e32 v22, v22\n\t" \
  "v_rcp_f32_e32 v23, v23\n\t" \
  "v_mfma_f32_16x16x128_f8f6f4 a[28:31], v[128:135], v[216:223], a[28:31]\n\t" \
  "ds_read_b128 v[176:179], v2 offset:51200\n\t" \
  "ds_read_b128 v[180:183], v2 offset:59392\n\t" \
  "v_mfma_f32_16x16x128_f8f6f4 a[60:63], v[136:143], v[216:223], a[60:63]\n\t" \
  "ds_read_b128 v[184:187], v2 offset:52224\n\t" \
  "ds_read_b128 v[188:191], v2 offset:60416\n\t" \
  "v_mul_f32_e32 v100, v100, v20\n\t" \
  "v_mul_f32_e32 v101, v101, v21\n\t" \
  "v_mul_f32_e32 v102, v102, v22\n\t" \
  "v_mul_f32_e32 v103, v103, v23\n\t" \
  "v_mfma_f32_16x16x128_f8f6f4 a[56:59], v[136:143], v[208:215], a[56:59]\n\t" \
  "ds_read_b128 v[224:227], v4 offset:16384\n\t" \
  "ds_read_b128 v[228:231], v4 offset:24576\n\t" \
  "s_xor_b32 s91, s68, 4259840\n\t" \
  "buffer_store_dwordx2 v[100:101], v5, s[40:43], s91 offen nt\n\t" \
  "v_mfma_f32_16x16x128_f8f6f4 a[52:55], v[136:143], v[200:207], a[52:55]\n\t" \
  "ds_read_b128 v[232:235], v4 offset:17408\n\t" \
  "ds_read_b128 v[236:239], v4 offset:25600\n\t" \
  "s_add_u32 s53, s52, 360448\n\t" \
  "s_mov_b32 m0, s48\n\t" \
  "buffer_store_dwordx2 v[102:103], v5, s[40:43], s91 offen offset:128 nt\n\t" \
  "v_mfma_f32_16x16x128_f8f6f4 a[48:51], v[136:143], v[192:199], a[48:51]\n\t" \
  "buffer_load_dwordx4 v0, s[36:39], s53 offen sc0 sc1 lds\n\t" \
  "v_mfma_f32_16x16x128_f8f6f4 a[80:83], v[144:151], v[192:199], a[80:83]\n\t" \
  "buffer_load_dwordx4 v0, s[36:39], s53 offen offset:1024 sc0 sc1 lds\n\t" \
  "v_mul_f32_e32 v104, s56, v104\n\t" \
  "v_mul_f32_e32 v105, s56, v105\n\t" \
  "v_mul_f32_e32 v106, s56, v106\n\t" \
  "v_mul_f32_e32 v107, s56, v107\n\t" \
  "v_mfma_f32_16x16x128_f8f6f4 a[84:87], v[144:151], v[200:207], a[84:87]\n\t" \
  "buffer_load_dwordx4 v0, s[36:39], s53 offen offset:2048 sc0 sc1 lds\n\t" \
  "v_min_f32_e32 v24, s60, v104\n\t" \
  "v_min_f32_e32 v25, s60, v105\n\t" \
  "v_min_f32_e32 v26, s60, v106\n\t" \
  "v_min_f32_e32 v27, s60, v107\n\t" \
  "v_mfma_f32_16x16x128_f8f6f4 a[88:91], v[144:151], v[208:215], a[88:91]\n\t" \
  "buffer_load_dwordx4 v0, s[36:39], s53 offen offset:3072 sc0 sc1 lds\n\t" \
  "s_add_u32 s53, s52, 376832\n\t" \
  "s_add_u32 m0, s48, 65536\n\t" \
  "v_exp_f32_e32 v24, v24\n\t" \
  "v_exp_f32_e32 v25, v25\n\t" \
  "v_mfma_f32_16x16x128_f8f6f4 a[92:95], v[144:151], v[216:223], a[92:95]\n\t" \
  "buffer_load_dwordx4 v0, s[36:39], s53 offen sc0 sc1 lds\n\t" \
  "v_mfma_f32_16x16x128_f8f6f4 a[124:127], v[152:159], v[216:223], a[124:127]\n\t" \
  "buffer_load_dwordx4 v0, s[36:39], s53 offen offset:1024 sc0 sc1 lds\n\t" \
  "v_exp_f32_e32 v26, v26\n\t" \
  "v_exp_f32_e32 v27, v27\n\t" \
  "v_mfma_f32_16x16x128_f8f6f4 a[120:123], v[152:159], v[208:215], a[120:123]\n\t" \
  "buffer_load_dwordx4 v0, s[36:39], s53 offen offset:2048 sc0 sc1 lds\n\t" \
  "v_fma_f32 v24, v24, s58, s58\n\t" \
  "v_fma_f32 v25, v25, s58, s58\n\t" \
  "v_fma_f32 v26, v26, s58, s58\n\t" \
  "v_fma_f32 v27, v27, s58, s58\n\t" \
  "v_mfma_f32_16x16x128_f8f6f4 a[116:119], v[152:159], v[200:207], a[116:119]\n\t" \
  "buffer_load_dwordx4 v0, s[36:39], s53 offen offset:3072 sc0 sc1 lds\n\t" \
  "v_mfma_f32_16x16x128_f8f6f4 a[112:115], v[152:159], v[192:199], a[112:115]\n\t" \
  "v_rcp_f32_e32 v24, v24\n\t" \
  "v_rcp_f32_e32 v25, v25\n\t" \
  "s_waitcnt lgkmcnt(0)\n\t" \
  "v_mfma_f32_16x16x128_f8f6f4 a[144:147], v[160:167], v[192:199], a[144:147]\n\t" \
  "s_waitcnt vmcnt(39)\n\t" \
  "s_barrier\n\t" \
  "v_mfma_f32_16x16x128_f8f6f4 a[148:151], v[160:167], v[200:207], a[148:151]\n\t" \
  "ds_read_b128 v[128:131], v1 offset:16384\n\t" \
  "ds_read_b128 v[132:135], v1 offset:24576\n\t" \
  "v_rcp_f32_e32 v26, v26\n\t" \
  "v_rcp_f32_e32 v27, v27\n\t" \
  "v_mfma_f32_16x16x128_f8f6f4 a[152:155], v[160:167], v[208:215], a[152:155]\n\t" \
  "ds_read_b128 v[136:139], v1 offset:17408\n\t" \
  "ds_read_b128 v[140:143], v1 offset:25600\n\t" \
  "v_mul_f32_e32 v104, v104, v24\n\t" \
  "v_mul_f32_e32 v105, v105, v25\n\t" \
  "v_mul_f32_e32 v106, v106, v26\n\t" \
  "v_mul_f32_e32 v107, v107, v27\n\t" \
  "v_mfma_f32_16x16x128_f8f6f4 a[156:159], v[160:167], v[216:223], a[156:159]\n\t" \
  "ds_read_b128 v[144:147], v1 offset:18432\n\t" \
  "ds_read_b128 v[148:151], v1 offset:26624\n\t" \
  "s_waitcnt vmcnt(32)\n\t" \
  "v_cmp_le_u32_e32 vcc, s75, v7\n\t" \
  "v_mfma_f32_16x16x128_f8f6f4 a[188:191], v[168:175], v[216:223], a[188:191]\n\t" \
  "ds_read_b128 v[152:155], v1 offset:19456\n\t" \
  "ds_read_b128 v[156:159], v1 offset:27648\n\t" \
  "v_mfma_f32_16x16x128_f8f6f4 a[184:187], v[168:175], v[208:215], a[184:187]\n\t" \
  "ds_read_b128 v[240:243], v4 offset:18432\n\t" \
  "ds_read_b128 v[244:247], v4 offset:26624\n\t" \
  "s_cmp_eq_u64 vcc, -1\n\t" \
  "s_cbranch_scc0 L_SLOWH1_%=\n\t" \
  "L_OKH1_%=:\n\t" \
  "s_add_u32 s76, s76, 1024\n\t" \
  "s_xor_b32 s91, s68, 4325376\n\t" \
  "buffer_store_dwordx2 v[104:105], v5, s[40:43], s91 offen nt\n\t" \
  "v_mfma_f32_16x16x128_f8f6f4 a[180:183], v[168:175], v[200:207], a[180:183]\n\t" \
  "ds_read_b128 v[248:251], v4 offset:19456\n\t" \
  "ds_read_b128 v[252:255], v4 offset:27648\n\t" \
  "buffer_store_dwordx2 v[106:107], v5, s[40:43], s91 offen offset:128 nt\n\t" \
  "v_mfma_f32_16x16x128_f8f6f4 a[176:179], v[168:175], v[192:199], a[176:179]\n\t" \
  "s_add_u32 s53, s50, 131072\n\t" \
  "s_add_u32 m0, s48, 114688\n\t" \
  "v_mul_f32_e32 v108, s56, v108\n\t" \
  "v_mul_f32_e32 v109, s56, v109\n\t" \
  "v_mul_f32_e32 v110, s56, v110\n\t" \
  "v_mul_f32_e32 v111, s56, v111\n\t" \
  "v_mfma_f32_16x16x128_f8f6f4 a[208:211], v[176:183], v[192:199], a[208:211]\n\t" \
  "buffer_load_dwordx4 v0, s[36:39], s53 offen sc0 sc1 lds\n\t" \
  "v_mfma_f32_16x16x128_f8f6f4 a[212:215], v[176:183], v[200:207], a[212:215]\n\t" \
  "v_min_f32_e32 v28, s60, v108\n\t" \
  "v_min_f32_e32 v29, s60, v109\n\t" \
  "v_min_f32_e32 v30, s60, v110\n\t" \
  "v_min_f32_e32 v31, s60, v111\n\t" \
  "v_mfma_f32_16x16x128_f8f6f4 a[216:219], v[176:183], v[208:215], a[216:219]\n\t" \
  "buffer_load_dwordx4 v0, s[36:39], s53 offen offset:1024 sc0 sc1 lds\n\t" \
  "v_exp_f32_e32 v28, v28\n\t" \
  "v_exp_f32_e32 v29, v29\n\t" \
  "v_mfma_f32_16x16x128_f8f6f4 a[220:223], v[176:183], v[216:223], a[220:223]\n\t" \
  "v_exp_f32_e32 v30, v30\n\t" \
  "v_exp_f32_e32 v31, v31\n\t" \
  "v_mfma_f32_16x16x128_f8f6f4 a[252:255], v[184:191], v[216:223], a[252:255]\n\t" \
  "buffer_load_dwordx4 v0, s[36:39], s53 offen offset:2048 sc0 sc1 lds\n\t" \
  "v_mfma_f32_16x16x128_f8f6f4 a[248:251], v[184:191], v[208:215], a[248:251]\n\t" \
  "v_fma_f32 v28, v28, s58, s58\n\t" \
  "v_fma_f32 v29, v29, s58, s58\n\t" \
  "v_fma_f32 v30, v30, s58, s58\n\t" \
  "v_fma_f32 v31, v31, s58, s58\n\t" \
  "v_mfma_f32_16x16x128_f8f6f4 a[244:247], v[184:191], v[200:207], a[244:247]\n\t" \
  "buffer_load_dwordx4 v0, s[36:39], s53 offen offset:3072 sc0 sc1 lds\n\t" \
  "v_rcp_f32_e32 v28, v28\n\t" \
  "v_rcp_f32_e32 v29, v29\n\t" \
  "v_mfma_f32_16x16x128_f8f6f4 a[240:243], v[184:191], v[192:199], a[240:243]\n\t" \
  "s_waitcnt lgkmcnt(0)\n\t" \
  "v_mfma_f32_16x16x128_f8f6f4 a[16:19], v[128:135], v[224:231], a[16:19]\n\t" \
  "s_waitcnt vmcnt(32)\n\t" \
  "s_barrier\n\t" \
  "v_mfma_f32_16x16x128_f8f6f4 a[20:23], v[128:135], v[232:239], a[20:23]\n\t" \
  "ds_read_b128 v[160:163], v6 offset:0\n\t" \
  "ds_read_b128 v[164:167], v6 offset:8192\n\t" \
  "v_rcp_f32_e32 v30, v30\n\t" \
  "v_rcp_f32_e32 v31, v31\n\t" \
  "v_mfma_f32_16x16x128_f8f6f4 a[24:27], v[128:135], v[240:247], a[24:27]\n\t" \
  "ds_read_b128 v[168:171], v6 offset:1024\n\t" \
  "ds_read_b128 v[172:175], v6 offset:9216\n\t" \
  "v_mul_f32_e32 v108, v108, v28\n\t" \
  "v_mul_f32_e32 v109, v109, v29\n\t" \
  "v_mul_f32_e32 v110, v110, v30\n\t" \
  "v_mul_f32_e32 v111, v111, v31\n\t" \
  "v_mfma_f32_16x16x128_f8f6f4 a[28:31], v[128:135], v[248:255], a[28:31]\n\t" \
  "ds_read_b128 v[176:179], v6 offset:2048\n\t" \
  "ds_read_b128 v[180:183], v6 offset:10240\n\t" \
  "s_xor_b32 s91, s68, 4390912\n\t" \
  "buffer_store_dwordx2 v[108:109], v5, s[40:43], s91 offen nt\n\t" \
  "v_mfma_f32_16x16x128_f8f6f4 a[60:63], v[136:143], v[248:255], a[60:63]\n\t" \
  "ds_read_b128 v[184:187], v6 offset:3072\n\t" \
  "ds_read_b128 v[188:191], v6 offset:11264\n\t" \
  "v_mfma_f32_16x16x128_f8f6f4 a[56:59], v[136:143], v[240:247], a[56:59]\n\t" \
  "ds_read_b128 v[192:195], v4 offset:32768\n\t" \
  "ds_read_b128 v[196:199], v4 offset:40960\n\t" \
  "buffer_store_dwordx2 v[110:111], v5, s[40:43], s91 offen offset:128 nt\n\t" \
  "v_mfma_f32_16x16x128_f8f6f4 a[52:55], v[136:143], v[232:239], a[52:55]\n\t" \
  "ds_read_b128 v[200:203], v4 offset:33792\n\t" \
  "ds_read_b128 v[204:207], v4 offset:41984\n\t" \
  "s_add_u32 s53, s52, 393216\n\t" \
  "s_add_u32 m0, s48, 16384\n\t" \
  "v_accvgpr_read_b32 v112, a160\n\t" \
  "v_accvgpr_read_b32 v113, a164\n\t" \
  "v_accvgpr_read_b32 v114, a168\n\t" \
  "v_accvgpr_read_b32 v115, a172\n\t" \
  "v_mfma_f32_16x16x128_f8f6f4 a[48:51], v[136:143], v[224:231], a[48:51]\n\t" \
  "buffer_load_dwordx4 v0, s[36:39], s53 offen sc0 sc1 lds\n\t" \
  "v_mul_f32_e32 v112, s56, v112\n\t" \
  "v_mul_f32_e32 v113, s56, v113\n\t" \
  "v_mul_f32_e32 v114, s56, v114\n\t" \
  "v_mul_f32_e32 v115, s56, v115\n\t" \
  "v_mfma_f32_16x16x128_f8f6f4 a[80:83], v[144:151], v[224:231], a[80:83]\n\t" \
  "buffer_load_dwordx4 v0, s[36:39], s53 offen offset:1024 sc0 sc1 lds\n\t" \
  "v_mfma_f32_16x16x128_f8f6f4 a[84:87], v[144:151], v[232:239], a[84:87]\n\t" \
  "buffer_load_dwordx4 v0, s[36:39], s53 offen offset:2048 sc0 sc1 lds\n\t" \
  "v_min_f32_e32 v12, s60, v112\n\t" \
  "v_min_f32_e32 v13, s60, v113\n\t" \
  "v_min_f32_e32 v14, s60, v114\n\t" \
  "v_min_f32_e32 v15, s60, v115\n\t" \
  "v_mfma_f32_16x16x128_f8f6f4 a[88:91], v[144:151], v[240:247], a[88:91]\n\t" \
  "buffer_load_dwordx4 v0, s[36:39], s53 offen offset:3072 sc0 sc1 lds\n\t" \
  "s_add_u32 s53, s52, 409600\n\t" \
  "s_add_u32 m0, s48, 81920\n\t" \
  "v_exp_f32_e32 v12, v12\n\t" \
  "v_exp_f32_e32 v13, v13\n\t" \
  "v_mfma_f32_16x16x128_f8f6f4 a[92:95], v[144:151], v[248:255], a[92:95]\n\t" \
  "buffer_load_dwordx4 v0, s[36:39], s53 offen sc0 sc1 lds\n\t" \
  "v_mfma_f32_16x16x128_f8f6f4 a[124:127], v[152:159], v[248:255], a[124:127]\n\t" \
  "buffer_load_dwordx4 v0, s[36:39], s53 offen offset:1024 sc0 sc1 lds\n\t" \
  "v_exp_f32_e32 v14, v14\n\t" \
  "v_exp_f32_e32 v15, v15\n\t" \
  "v_mfma_f32_16x16x128_f8f6f4 a[120:123], v[152:159], v[240:247], a[120:123]\n\t" \
  "buffer_load_dwordx4 v0, s[36:39], s53 offen offset:2048 sc0 sc1 lds\n\t" \
  "v_fma_f32 v12, v12, s58, s58\n\t" \
  "v_fma_f32 v13, v13, s58, s58\n\t" \
  "v_fma_f32 v14, v14, s58, s58\n\t" \
  "v_fma_f32 v15, v15, s58, s58\n\t" \
  "v_mfma_f32_16x16x128_f8f6f4 a[116:119], v[152:159], v[232:239], a[116:119]\n\t" \
  "buffer_load_dwordx4 v0, s[36:39], s53 offen offset:3072 sc0 sc1 lds\n\t" \
  "v_rcp_f32_e32 v12, v12\n\t" \
  "v_rcp_f32_e32 v13, v13\n\t" \
  "v_mfma_f32_16x16x128_f8f6f4 a[112:115], v[152:159], v[224:231], a[112:115]\n\t" \
  "s_waitcnt lgkmcnt(0)\n\t" \
  "v_mfma_f32_16x16x128_f8f6f4 a[144:147], v[160:167], v[224:231], a[144:147]\n\t" \
  "s_waitcnt vmcnt(36)\n\t" \
  "s_barrier\n\t" \
  "v_mfma_f32_16x16x128_f8f6f4 a[148:151], v[160:167], v[232:239], a[148:151]\n\t" \
  "ds_read_b128 v[128:131], v6 offset:16384\n\t" \
  "ds_read_b128 v[132:135], v6 offset:24576\n\t" \
  "v_rcp_f32_e32 v14, v14\n\t" \
  "v_rcp_f32_e32 v15, v15\n\t" \
  "v_mfma_f32_16x16x128_f8f6f4 a[152:155], v[160:167], v[240:247], a[152:155]\n\t" \
  "ds_read_b128 v[136:139], v6 offset:17408\n\t" \
  "ds_read_b128 v[140:143], v6 offset:25600\n\t" \
  "v_mul_f32_e32 v112, v112, v12\n\t" \
  "v_mul_f32_e32 v113, v113, v13\n\t" \
  "v_mul_f32_e32 v114, v114, v14\n\t" \
  "v_mul_f32_e32 v115, v115, v15\n\t" \
  "v_mfma_f32_16x16x128_f8f6f4 a[156:159], v[160:167], v[248:255], a[156:159]\n\t" \
  "ds_read_b128 v[144:147], v6 offset:18432\n\t" \
  "ds_read_b128 v[148:151], v6 offset:26624\n\t" \
  "s_xor_b32 s91, s68, 5242880\n\t" \
  "buffer_store_dwordx2 v[112:113], v5, s[40:43], s91 offen nt\n\t" \
  "v_mfma_f32_16x16x128_f8f6f4 a[188:191], v[168:175], v[248:255], a[188:191]\n\t" \
  "ds_read_b128 v[152:155], v6 offset:19456\n\t" \
  "ds_read_b128 v[156:159], v6 offset:27648\n\t" \
  "v_mfma_f32_16x16x128_f8f6f4 a[184:187], v[168:175], v[240:247], a[184:187]\n\t" \
  "ds_read_b128 v[208:211], v4 offset:34816\n\t" \
  "ds_read_b128 v[212:215], v4 offset:43008\n\t" \
  "buffer_store_dwordx2 v[114:115], v5, s[40:43], s91 offen offset:128 nt\n\t" \
  "v_mfma_f32_16x16x128_f8f6f4 a[180:183], v[168:175], v[232:239], a[180:183]\n\t" \
  "ds_read_b128 v[216:219], v4 offset:35840\n\t" \
  "ds_read_b128 v[220:223], v4 offset:44032\n\t" \
  "v_accvgpr_read_b32 v116, a161\n\t" \
  "v_accvgpr_read_b32 v117, a165\n\t" \
  "v_accvgpr_read_b32 v118, a169\n\t" \
  "v_accvgpr_read_b32 v119, a173\n\t" \
  "v_mfma_f32_16x16x128_f8f6f4 a[176:179], v[168:175], v[224:231], a[176:179]\n\t" \
  "s_add_u32 s53, s50, 147456\n\t" \
  "s_add_u32 m0, s48, 131072\n\t" \
  "v_mul_f32_e32 v116, s56, v116\n\t" \
  "v_mul_f32_e32 v117, s56, v117\n\t" \
  "v_mul_f32_e32 v118, s56, v118\n\t" \
  "v_mul_f32_e32 v119, s56, v119\n\t" \
  "v_mfma_f32_16x16x128_f8f6f4 a[208:211], v[176:183], v[224:231], a[208:211]\n\t" \
  "buffer_load_dwordx4 v0, s[36:39], s53 offen sc0 sc1 lds\n\t" \
  "v_mfma_f32_16x16x128_f8f6f4 a[212:215], v[176:183], v[232:239], a[212:215]\n\t" \
  "v_min_f32_e32 v16, s60, v116\n\t" \
  "v_min_f32_e32 v17, s60, v117\n\t" \
  "v_min_f32_e32 v18, s60, v118\n\t" \
  "v_min_f32_e32 v19, s60, v119\n\t" \
  "v_mfma_f32_16x16x128_f8f6f4 a[216:219], v[176:183], v[240:247], a[216:219]\n\t" \
  "buffer_load_dwordx4 v0, s[36:39], s53 offen offset:1024 sc0 sc1 lds\n\t" \
  "v_exp_f32_e32 v16, v16\n\t" \
  "v_exp_f32_e32 v17, v17\n\t" \
  "v_mfma_f32_16x16x128_f8f6f4 a[220:223], v[176:183], v[248:255], a[220:223]\n\t" \
  "v_mfma_f32_16x16x128_f8f6f4 a[252:255], v[184:191], v[248:255], a[252:255]\n\t" \
  "buffer_load_dwordx4 v0, s[36:39], s53 offen offset:2048 sc0 sc1 lds\n\t" \
  "v_exp_f32_e32 v18, v18\n\t" \
  "v_exp_f32_e32 v19, v19\n\t" \
  "v_mfma_f32_16x16x128_f8f6f4 a[248:251], v[184:191], v[240:247], a[248:251]\n\t" \
  "v_fma_f32 v16, v16, s58, s58\n\t" \
  "v_fma_f32 v17, v17, s58, s58\n\t" \
  "v_fma_f32 v18, v18, s58, s58\n\t" \
  "v_fma_f32 v19, v19, s58, s58\n\t" \
  "v_mfma_f32_16x16x128_f8f6f4 a[244:247], v[184:191], v[232:239], a[244:247]\n\t" \
  "buffer_load_dwordx4 v0, s[36:39], s53 offen offset:3072 sc0 sc1 lds\n\t" \
  "v_rcp_f32_e32 v16, v16\n\t" \
  "v_rcp_f32_e32 v17, v17\n\t" \
  "v_mfma_f32_16x16x128_f8f6f4 a[240:243], v[184:191], v[224:231], a[240:243]\n\t" \
  "s_waitcnt lgkmcnt(0)\n\t" \
  "v_mfma_f32_16x16x128_f8f6f4 a[16:19], v[128:135], v[192:199], a[16:19]\n\t" \
  "s_waitcnt vmcnt(32)\n\t" \
  "s_barrier\n\t" \
  "v_mfma_f32_16x16x128_f8f6f4 a[20:23], v[128:135], v[200:207], a[20:23]\n\t" \
  "ds_read_b128 v[160:163], v1 offset:32768\n\t" \
  "ds_read_b128 v[164:167], v1 offset:40960\n\t" \
  "v_rcp_f32_e32 v18, v18\n\t" \
  "v_rcp_f32_e32 v19, v19\n\t" \
  "v_mfma_f32_16x16x128_f8f6f4 a[24:27], v[128:135], v[208:215], a[24:27]\n\t" \
  "ds_read_b128 v[168:171], v1 offset:33792\n\t" \
  "ds_read_b128 v[172:175], v1 offset:41984\n\t" \
  "v_mul_f32_e32 v116, v116, v16\n\t" \
  "v_mul_f32_e32 v117, v117, v17\n\t" \
  "v_mul_f32_e32 v118, v118, v18\n\t" \
  "v_mul_f32_e32 v119, v119, v19\n\t" \
  "v_mfma_f32_16x16x128_f8f6f4 a[28:31], v[128:135], v[216:223], a[28:31]\n\t" \
  "ds_read_b128 v[176:179], v1 offset:34816\n\t" \
  "ds_read_b128 v[180:183], v1 offset:43008\n\t" \
  "s_xor_b32 s91, s68, 5308416\n\t" \
  "buffer_store_dwordx2 v[116:117], v5, s[40:43], s91 offen nt\n\t" \
  "v_mfma_f32_16x16x128_f8f6f4 a[60:63], v[136:143], v[216:223], a[60:63]\n\t" \
  "ds_read_b128 v[184:187], v1 offset:35840\n\t" \
  "ds_read_b128 v[188:191], v1 offset:44032\n\t" \
  "v_mfma_f32_16x16x128_f8f6f4 a[56:59], v[136:143], v[208:215], a[56:59]\n\t" \
  "ds_read_b128 v[224:227], v3 offset:49152\n\t" \
  "ds_read_b128 v[228:231], v3 offset:57344\n\t" \
  "buffer_store_dwordx2 v[118:119], v5, s[40:43], s91 offen offset:128 nt\n\t" \
  "v_mfma_f32_16x16x128_f8f6f4 a[52:55], v[136:143], v[200:207], a[52:55]\n\t" \
  "ds_read_b128 v[232:235], v3 offset:50176\n\t" \
  "ds_read_b128 v[236:239], v3 offset:58368\n\t" \
  "s_add_u32 s53, s52, 425984\n\t" \
  "s_add_u32 m0, s48, 147456\n\t" \
  "v_accvgpr_read_b32 v120, a162\n\t" \
  "v_accvgpr_read_b32 v121, a166\n\t" \
  "v_accvgpr_read_b32 v122, a170\n\t" \
  "v_accvgpr_read_b32 v123, a174\n\t" \
  "v_mfma_f32_16x16x128_f8f6f4 a[48:51], v[136:143], v[192:199], a[48:51]\n\t" \
  "buffer_load_dwordx4 v0, s[36:39], s53 offen sc0 sc1 lds\n\t" \
  "v_mfma_f32_16x16x128_f8f6f4 a[80:83], v[144:151], v[192:199], a[80:83]\n\t" \
  "buffer_load_dwordx4 v0, s[36:39], s53 offen offset:1024 sc0 sc1 lds\n\t" \
  "v_mul_f32_e32 v120, s56, v120\n\t" \
  "v_mul_f32_e32 v121, s56, v121\n\t" \
  "v_mul_f32_e32 v122, s56, v122\n\t" \
  "v_mul_f32_e32 v123, s56, v123\n\t" \
  "v_mfma_f32_16x16x128_f8f6f4 a[84:87], v[144:151], v[200:207], a[84:87]\n\t" \
  "buffer_load_dwordx4 v0, s[36:39], s53 offen offset:2048 sc0 sc1 lds\n\t" \
  "v_min_f32_e32 v20, s60, v120\n\t" \
  "v_min_f32_e32 v21, s60, v121\n\t" \
  "v_min_f32_e32 v22, s60, v122\n\t" \
  "v_min_f32_e32 v23, s60, v123\n\t" \
  "v_mfma_f32_16x16x128_f8f6f4 a[88:91], v[144:151], v[208:215], a[88:91]\n\t" \
  "buffer_load_dwordx4 v0, s[36:39], s53 offen offset:3072 sc0 sc1 lds\n\t" \
  "s_add_u32 s53, s52, 442368\n\t" \
  "s_add_u32 m0, s48, 98304\n\t" \
  "v_exp_f32_e32 v20, v20\n\t" \
  "v_exp_f32_e32 v21, v21\n\t" \
  "v_mfma_f32_16x16x128_f8f6f4 a[92:95], v[144:151], v[216:223], a[92:95]\n\t" \
  "buffer_load_dwordx4 v0, s[36:39], s53 offen sc0 sc1 lds\n\t" \
  "v_mfma_f32_16x16x128_f8f6f4 a[124:127], v[152:159], v[216:223], a[124:127]\n\t" \
  "buffer_load_dwordx4 v0, s[36:39], s53 offen offset:1024 sc0 sc1 lds\n\t" \
  "v_exp_f32_e32 v22, v22\n\t" \
  "v_exp_f32_e32 v23, v23\n\t" \
  "v_mfma_f32_16x16x128_f8f6f4 a[120:123], v[152:159], v[208:215], a[120:123]\n\t" \
  "buffer_load_dwordx4 v0, s[36:39], s53 offen offset:2048 sc0 sc1 lds\n\t" \
  "v_fma_f32 v20, v20, s58, s58\n\t" \
  "v_fma_f32 v21, v21, s58, s58\n\t" \
  "v_fma_f32 v22, v22, s58, s58\n\t" \
  "v_fma_f32 v23, v23, s58, s58\n\t" \
  "v_mfma_f32_16x16x128_f8f6f4 a[116:119], v[152:159], v[200:207], a[116:119]\n\t" \
  "buffer_load_dwordx4 v0, s[36:39], s53 offen offset:3072 sc0 sc1 lds\n\t" \
  "v_rcp_f32_e32 v20, v20\n\t" \
  "v_rcp_f32_e32 v21, v21\n\t" \
  "v_mfma_f32_16x16x128_f8f6f4 a[112:115], v[152:159], v[192:199], a[112:115]\n\t" \
  "s_waitcnt lgkmcnt(0)\n\t" \
  "v_mfma_f32_16x16x128_f8f6f4 a[144:147], v[160:167], v[192:199], a[144:147]\n\t" \
  "s_waitcnt vmcnt(36)\n\t" \
  "s_barrier\n\t" \
  "v_mfma_f32_16x16x128_f8f6f4 a[148:151], v[160:167], v[200:207], a[148:151]\n\t" \
  "ds_read_b128 v[128:131], v1 offset:0\n\t" \
  "ds_read_b128 v[132:135], v1 offset:8192\n\t" \
  "v_rcp_f32_e32 v22, v22\n\t" \
  "v_rcp_f32_e32 v23, v23\n\t" \
  "v_mfma_f32_16x16x128_f8f6f4 a[152:155], v[160:167], v[208:215], a[152:155]\n\t" \
  "ds_read_b128 v[136:139], v1 offset:1024\n\t" \
  "ds_read_b128 v[140:143], v1 offset:9216\n\t" \
  "v_mul_f32_e32 v120, v120, v20\n\t" \
  "v_mul_f32_e32 v121, v121, v21\n\t" \
  "v_mul_f32_e32 v122, v122, v22\n\t" \
  "v_mul_f32_e32 v123, v123, v23\n\t" \
  "v_mfma_f32_16x16x128_f8f6f4 a[156:159], v[160:167], v[216:223], a[156:159]\n\t" \
  "ds_read_b128 v[144:147], v1 offset:2048\n\t" \
  "ds_read_b128 v[148:151], v1 offset:10240\n\t" \
  "s_xor_b32 s91, s68, 5373952\n\t" \
  "buffer_store_dwordx2 v[120:121], v5, s[40:43], s91 offen nt\n\t" \
  "v_mfma_f32_16x16x128_f8f6f4 a[188:191], v[168:175], v[216:223], a[188:191]\n\t" \
  "ds_read_b128 v[152:155], v1 offset:3072\n\t" \
  "ds_read_b128 v[156:159], v1 offset:11264\n\t" \
  "v_mfma_f32_16x16x128_f8f6f4 a[184:187], v[168:175], v[208:215], a[184:187]\n\t" \
  "ds_read_b128 v[240:243], v3 offset:51200\n\t" \
  "ds_read_b128 v[244:247], v3 offset:59392\n\t" \
  "buffer_store_dwordx2 v[122:123], v5, s[40:43], s91 offen offset:128 nt\n\t" \
  "v_mfma_f32_16x16x128_f8f6f4 a[180:183], v[168:175], v[200:207], a[180:183]\n\t" \
  "ds_read_b128 v[248:251], v3 offset:52224\n\t" \
  "ds_read_b128 v[252:255], v3 offset:60416\n\t" \
  "v_accvgpr_read_b32 v112, a163\n\t" \
  "v_accvgpr_read_b32 v113, a167\n\t" \
  "v_accvgpr_read_b32 v114, a171\n\t" \
  "v_accvgpr_read_b32 v115, a175\n\t" \
  "v_mfma_f32_16x16x128_f8f6f4 a[176:179], v[168:175], v[192:199], a[176:179]\n\t" \
  "s_add_u32 s53, s50, 163840\n\t" \
  "s_add_u32 m0, s48, 32768\n\t" \
  "v_mfma_f32_16x16x128_f8f6f4 a[208:211], v[176:183], v[192:199], a[208:211]\n\t" \
  "buffer_load_dwordx4 v0, s[36:39], s53 offen sc0 sc1 lds\n\t" \
  "v_mul_f32_e32 v112, s56, v112\n\t" \
  "v_mul_f32_e32 v113, s56, v113\n\t" \
  "v_mul_f32_e32 v114, s56, v114\n\t" \
  "v_mul_f32_e32 v115, s56, v115\n\t" \
  "v_mfma_f32_16x16x128_f8f6f4 a[212:215], v[176:183], v[200:207], a[212:215]\n\t" \
  "v_min_f32_e32 v24, s60, v112\n\t" \
  "v_min_f32_e32 v25, s60, v113\n\t" \
  "v_min_f32_e32 v26, s60, v114\n\t" \
  "v_min_f32_e32 v27, s60, v115\n\t" \
  "v_mfma_f32_16x16x128_f8f6f4 a[216:219], v[176:183], v[208:215], a[216:219]\n\t" \
  "buffer_load_dwordx4 v0, s[36:39], s53 offen offset:1024 sc0 sc1 lds\n\t" \
  "v_exp_f32_e32 v24, v24\n\t" \
  "v_exp_f32_e32 v25, v25\n\t" \
  "v_mfma_f32_16x16x128_f8f6f4 a[220:223], v[176:183], v[216:223], a[220:223]\n\t" \
  "v_mfma_f32_16x16x128_f8f6f4 a[252:255], v[184:191], v[216:223], a[252:255]\n\t" \
  "buffer_load_dwordx4 v0, s[36:39], s53 offen offset:2048 sc0 sc1 lds\n\t" \
  "v_exp_f32_e32 v26, v26\n\t" \
  "v_exp_f32_e32 v27, v27\n\t" \
  "v_mfma_f32_16x16x128_f8f6f4 a[248:251], v[184:191], v[208:215], a[248:251]\n\t" \
  "v_fma_f32 v24, v24, s58, s58\n\t" \
  "v_fma_f32 v25, v25, s58, s58\n\t" \
  "v_fma_f32 v26, v26, s58, s58\n\t" \
  "v_fma_f32 v27, v27, s58, s58\n\t" \
  "v_mfma_f32_16x16x128_f8f6f4 a[244:247], v[184:191], v[200:207], a[244:247]\n\t" \
  "buffer_load_dwordx4 v0, s[36:39], s53 offen offset:3072 sc0 sc1 lds\n\t" \
  "v_rcp_f32_e32 v24, v24\n\t" \
  "v_rcp_f32_e32 v25, v25\n\t" \
  "v_mfma_f32_16x16x128_f8f6f4 a[240:243], v[184:191], v[192:199], a[240:243]\n\t" \
  "s_waitcnt lgkmcnt(0)\n\t" \
  "v_mfma_f32_16x16x128_f8f6f4 a[16:19], v[128:135], v[224:231], a[16:19]\n\t" \
  "s_waitcnt vmcnt(32)\n\t" \
  "s_barrier\n\t" \
  "v_mfma_f32_16x16x128_f8f6f4 a[20:23], v[128:135], v[232:239], a[20:23]\n\t" \
  "ds_read_b128 v[160:163], v2 offset:0\n\t" \
  "ds_read_b128 v[164:167], v2 offset:8192\n\t" \
  "v_rcp_f32_e32 v26, v26\n\t" \
  "v_rcp_f32_e32 v27, v27\n\t" \
  "v_mfma_f32_16x16x128_f8f6f4 a[24:27], v[128:135], v[240:247], a[24:27]\n\t" \
  "ds_read_b128 v[168:171], v2 offset:1024\n\t" \
  "ds_read_b128 v[172:175], v2 offset:9216\n\t" \
  "v_mul_f32_e32 v112, v112, v24\n\t" \
  "v_mul_f32_e32 v113, v113, v25\n\t" \
  "v_mul_f32_e32 v114, v114, v26\n\t" \
  "v_mul_f32_e32 v115, v115, v27\n\t" \
  "v_mfma_f32_16x16x128_f8f6f4 a[28:31], v[128:135], v[248:255], a[28:31]\n\t" \
  "ds_read_b128 v[176:179], v2 offset:2048\n\t" \
  "ds_read_b128 v[180:183], v2 offset:10240\n\t" \
  "v_mfma_f32_16x16x128_f8f6f4 a[60:63], v[136:143], v[248:255], a[60:63]\n\t" \
  "ds_read_b128 v[184:187], v2 offset:3072\n\t" \
  "ds_read_b128 v[188:191], v2 offset:11264\n\t" \
  "s_xor_b32 s91, s68, 5439488\n\t" \
  "buffer_store_dwordx2 v[112:113], v5, s[40:43], s91 offen nt\n\t" \
  "v_mfma_f32_16x16x128_f8f6f4 a[56:59], v[136:143], v[240:247], a[56:59]\n\t" \
  "ds_read_b128 v[192:195], v4 offset:49152\n\t" \
  "ds_read_b128 v[196:199], v4 offset:57344\n\t" \
  "buffer_store_dwordx2 v[114:115], v5, s[40:43], s91 offen offset:128 nt\n\t" \
  "v_mfma_f32_16x16x128_f8f6f4 a[52:55], v[136:143], v[232:239], a[52:55]\n\t" \
  "ds_read_b128 v[200:203], v4 offset:50176\n\t" \
  "ds_read_b128 v[204:207], v4 offset:58368\n\t" \
  "s_add_u32 s53, s52, 458752\n\t" \
  "s_mov_b32 m0, s48\n\t" \
  "v_accvgpr_read_b32 v116, a192\n\t" \
  "v_accvgpr_read_b32 v117, a196\n\t" \
  "v_accvgpr_read_b32 v118, a200\n\t" \
  "v_accvgpr_read_b32 v119, a204\n\t" \
  "v_mfma_f32_16x16x128_f8f6f4 a[48:51], v[136:143], v[224:231], a[48:51]\n\t" \
  "buffer_load_dwordx4 v0, s[36:39], s53 offen sc0 sc1 lds\n\t" \
  "v_mfma_f32_16x16x128_f8f6f4 a[80:83], v[144:151], v[224:231], a[80:83]\n\t" \
  "buffer_load_dwordx4 v0, s[36:39], s53 offen offset:1024 sc0 sc1 lds\n\t" \
  "v_mul_f32_e32 v116, s56, v116\n\t" \
  "v_mul_f32_e32 v117, s56, v117\n\t" \
  "v_mul_f32_e32 v118, s56, v118\n\t" \
  "v_mul_f32_e32 v119, s56, v119\n\t" \
  "v_mfma_f32_16x16x128_f8f6f4 a[84:87], v[144:151], v[232:239], a[84:87]\n\t" \
  "buffer_load_dwordx4 v0, s[36:39], s53 offen offset:2048 sc0 sc1 lds\n\t" \
  "v_min_f32_e32 v28, s60, v116\n\t" \
  "v_min_f32_e32 v29, s60, v117\n\t" \
  "v_min_f32_e32 v30, s60, v118\n\t" \
  "v_min_f32_e32 v31, s60, v119\n\t" \
  "v_mfma_f32_16x16x128_f8f6f4 a[88:91], v[144:151], v[240:247], a[88:91]\n\t" \
  "buffer_load_dwordx4 v0, s[36:39], s53 offen offset:3072 sc0 sc1 lds\n\t" \
  "s_add_u32 s53, s52, 475136\n\t" \
  "s_add_u32 m0, s48, 49152\n\t" \
  "v_exp_f32_e32 v28, v28\n\t" \
  "v_exp_f32_e32 v29, v29\n\t" \
  "v_mfma_f32_16x16x128_f8f6f4 a[92:95], v[144:151], v[248:255], a[92:95]\n\t" \
  "buffer_load_dwordx4 v0, s[36:39], s53 offen sc0 sc1 lds\n\t" \
  "v_mfma_f32_16x16x128_f8f6f4 a[124:127], v[152:159], v[248:255], a[124:127]\n\t" \
  "buffer_load_dwordx4 v0, s[36:39], s53 offen offset:1024 sc0 sc1 lds\n\t" \
  "v_exp_f32_e32 v30, v30\n\t" \
  "v_exp_f32_e32 v31, v31\n\t" \
  "v_mfma_f32_16x16x128_f8f6f4 a[120:123], v[152:159], v[240:247], a[120:123]\n\t" \
  "buffer_load_dwordx4 v0, s[36:39], s53 offen offset:2048 sc0 sc1 lds\n\t" \
  "v_fma_f32 v28, v28, s58, s58\n\t" \
  "v_fma_f32 v29, v29, s58, s58\n\t" \
  "v_fma_f32 v30, v30, s58, s58\n\t" \
  "v_fma_f32 v31, v31, s58, s58\n\t" \
  "v_mfma_f32_16x16x128_f8f6f4 a[116:119], v[152:159], v[232:239], a[116:119]\n\t" \
  "buffer_load_dwordx4 v0, s[36:39], s53 offen offset:3072 sc0 sc1 lds\n\t" \
  "v_rcp_f32_e32 v28, v28\n\t" \
  "v_rcp_f32_e32 v29, v29\n\t" \
  "v_mfma_f32_16x16x128_f8f6f4 a[112:115], v[152:159], v[224:231], a[112:115]\n\t" \
  "s_waitcnt lgkmcnt(0)\n\t" \
  "v_mfma_f32_16x16x128_f8f6f4 a[144:147], v[160:167], v[224:231], a[144:147]\n\t" \
  "s_waitcnt vmcnt(36)\n\t" \
  "s_barrier\n\t" \
  "v_mfma_f32_16x16x128_f8f6f4 a[148:151], v[160:167], v[232:239], a[148:151]\n\t" \
  "ds_read_b128 v[128:131], v1 offset:16384\n\t" \
  "ds_read_b128 v[132:135], v1 offset:24576\n\t" \
  "v_rcp_f32_e32 v30, v30\n\t" \
  "v_rcp_f32_e32 v31, v31\n\t" \
  "v_mfma_f32_16x16x128_f8f6f4 a[152:155], v[160:167], v[240:247], a[152:155]\n\t" \
  "ds_read_b128 v[136:139], v1 offset:17408\n\t" \
  "ds_read_b128 v[140:143], v1 offset:25600\n\t" \
  "v_mul_f32_e32 v116, v116, v28\n\t" \
  "v_mul_f32_e32 v117, v117, v29\n\t" \
  "v_mul_f32_e32 v118, v118, v30\n\t" \
  "v_mul_f32_e32 v119, v119, v31\n\t" \
  "v_mfma_f32_16x16x128_f8f6f4 a[156:159], v[160:167], v[248:255], a[156:159]\n\t" \
  "ds_read_b128 v[144:147], v1 offset:18432\n\t" \
  "ds_read_b128 v[148:151], v1 offset:26624\n\t" \
  "v_mfma_f32_16x16x128_f8f6f4 a[188:191], v[168:175], v[248:255], a[188:191]\n\t" \
  "ds_read_b128 v[152:155], v1 offset:19456\n\t" \
  "ds_read_b128 v[156:159], v1 offset:27648\n\t" \
  "s_xor_b32 s91, s68, 6291456\n\t" \
  "buffer_store_dwordx2 v[116:117], v5, s[40:43], s91 offen nt\n\t" \
  "v_mfma_f32_16x16x128_f8f6f4 a[184:187], v[168:175], v[240:247], a[184:187]\n\t" \
  "ds_read_b128 v[208:211], v4 offset:51200\n\t" \
  "ds_read_b128 v[212:215], v4 offset:59392\n\t" \
  "buffer_store_dwordx2 v[118:119], v5, s[40:43], s91 offen offset:128 nt\n\t" \
  "v_mfma_f32_16x16x128_f8f6f4 a[180:183], v[168:175], v[232:239], a[180:183]\n\t" \
  "ds_read_b128 v[216:219], v4 offset:52224\n\t" \
  "ds_read_b128 v[220:223], v4 offset:60416\n\t" \
  "v_accvgpr_read_b32 v120, a193\n\t" \
  "v_accvgpr_read_b32 v121, a197\n\t" \
  "v_accvgpr_read_b32 v122, a201\n\t" \
  "v_accvgpr_read_b32 v123, a205\n\t" \
  "v_mfma_f32_16x16x128_f8f6f4 a[176:179], v[168:175], v[224:231], a[176:179]\n\t" \
  "s_add_u32 s53, s50, 180224\n\t" \
  "s_add_u32 m0, s48, 65536\n\t" \
  "v_mfma_f32_16x16x128_f8f6f4 a[208:211], v[176:183], v[224:231], a[208:211]\n\t" \
  "buffer_load_dwordx4 v0, s[36:39], s53 offen sc0 sc1 lds\n\t" \
  "v_mul_f32_e32 v120, s56, v120\n\t" \
  "v_mul_f32_e32 v121, s56, v121\n\t" \
  "v_mul_f32_e32 v122, s56, v122\n\t" \
  "v_mul_f32_e32 v123, s56, v123\n\t" \
  "v_mfma_f32_16x16x128_f8f6f4 a[212:215], v[176:183], v[232:239], a[212:215]\n\t" \
  "v_min_f32_e32 v12, s60, v120\n\t" \
  "v_min_f32_e32 v13, s60, v121\n\t" \
  "v_min_f32_e32 v14, s60, v122\n\t" \
  "v_min_f32_e32 v15, s60, v123\n\t" \
  "v_mfma_f32_16x16x128_f8f6f4 a[216:219], v[176:183], v[240:247], a[216:219]\n\t" \
  "buffer_load_dwordx4 v0, s[36:39], s53 offen offset:1024 sc0 sc1 lds\n\t" \
  "v_exp_f32_e32 v12, v12\n\t" \
  "v_exp_f32_e32 v13, v13\n\t" \
  "v_mfma_f32_16x16x128_f8f6f4 a[220:223], v[176:183], v[248:255], a[220:223]\n\t" \
  "v_mfma_f32_16x16x128_f8f6f4 a[252:255], v[184:191], v[248:255], a[252:255]\n\t" \
  "buffer_load_dwordx4 v0, s[36:39], s53 offen offset:2048 sc0 sc1 lds\n\t" \
  "v_exp_f32_e32 v14, v14\n\t" \
  "v_exp_f32_e32 v15, v15\n\t" \
  "v_mfma_f32_16x16x128_f8f6f4 a[248:251], v[184:191], v[240:247], a[248:251]\n\t" \
  "v_fma_f32 v12, v12, s58, s58\n\t" \
  "v_fma_f32 v13, v13, s58, s58\n\t" \
  "v_fma_f32 v14, v14, s58, s58\n\t" \
  "v_fma_f32 v15, v15, s58, s58\n\t" \
  "v_mfma_f32_16x16x128_f8f6f4 a[244:247], v[184:191], v[232:239], a[244:247]\n\t" \
  "buffer_load_dwordx4 v0, s[36:39], s53 offen offset:3072 sc0 sc1 lds\n\t" \
  "v_rcp_f32_e32 v12, v12\n\t" \
  "v_rcp_f32_e32 v13, v13\n\t" \
  "v_mfma_f32_16x16x128_f8f6f4 a[240:243], v[184:191], v[224:231], a[240:243]\n\t" \
  "s_waitcnt lgkmcnt(0)\n\t" \
  "v_mfma_f32_16x16x128_f8f6f4 a[16:19], v[128:135], v[192:199], a[16:19]\n\t" \
  "s_waitcnt vmcnt(32)\n\t" \
  "s_barrier\n\t" \
  "v_mfma_f32_16x16x128_f8f6f4 a[20:23], v[128:135], v[200:207], a[20:23]\n\t" \
  "ds_read_b128 v[160:163], v2 offset:16384\n\t" \
  "ds_read_b128 v[164:167], v2 offset:24576\n\t" \
  "v_rcp_f32_e32 v14, v14\n\t" \
  "v_rcp_f32_e32 v15, v15\n\t" \
  "v_mfma_f32_16x16x128_f8f6f4 a[24:27], v[128:135], v[208:215], a[24:27]\n\t" \
  "ds_read_b128 v[168:171], v2 offset:17408\n\t" \
  "ds_read_b128 v[172:175], v2 offset:25600\n\t" \
  "v_mul_f32_e32 v120, v120, v12\n\t" \
  "v_mul_f32_e32 v121, v121, v13\n\t" \
  "v_mul_f32_e32 v122, v122, v14\n\t" \
  "v_mul_f32_e32 v123, v123, v15\n\t" \
  "v_mfma_f32_16x16x128_f8f6f4 a[28:31], v[128:135], v[216:223], a[28:31]\n\t" \
  "ds_read_b128 v[176:179], v2 offset:18432\n\t" \
  "ds_read_b128 v[180:183], v2 offset:26624\n\t" \
  "v_mfma_f32_16x16x128_f8f6f4 a[60:63], v[136:143], v[216:223], a[60:63]\n\t" \
  "ds_read_b128 v[184:187], v2 offset:19456\n\t" \
  "ds_read_b128 v[188:191], v2 offset:27648\n\t" \
  "s_xor_b32 s91, s68, 6356992\n\t" \
  "buffer_store_dwordx2 v[120:121], v5, s[40:43], s91 offen nt\n\t" \
  "v_mfma_f32_16x16x128_f8f6f4 a[56:59], v[136:143], v[208:215], a[56:59]\n\t" \
  "ds_read_b128 v[224:227], v127 offset:0\n\t" \
  "ds_read_b128 v[228:231], v127 offset:8192\n\t" \
  "buffer_store_dwordx2 v[122:123], v5, s[40:43], s91 offen offset:128 nt\n\t" \
  "v_mfma_f32_16x16x128_f8f6f4 a[52:55], v[136:143], v[200:207], a[52:55]\n\t" \
  "ds_read_b128 v[232:235], v127 offset:1024\n\t" \
  "ds_read_b128 v[236:239], v127 offset:9216\n\t" \
  "s_add_u32 s53, s52, 491520\n\t" \
  "s_add_u32 m0, s48, 16384\n\t" \
  "v_accvgpr_read_b32 v112, a194\n\t" \
  "v_accvgpr_read_b32 v113, a198\n\t" \
  "v_accvgpr_read_b32 v114, a202\n\t" \
  "v_accvgpr_read_b32 v115, a206\n\t" \
  "v_mfma_f32_16x16x128_f8f6f4 a[48:51], v[136:143], v[192:199], a[48:51]\n\t" \
  "buffer_load_dwordx4 v0, s[36:39], s53 offen sc0 sc1 lds\n\t" \
  "v_mfma_f32_16x16x128_f8f6f4 a[80:83], v[144:151], v[192:199], a[80:83]\n\t" \
  "buffer_load_dwordx4 v0, s[36:39], s53 offen offset:1024 sc0 sc1 lds\n\t" \
  "v_mul_f32_e32 v112, s56, v112\n\t" \
  "v_mul_f32_e32 v113, s56, v113\n\t" \
  "v_mul_f32_e32 v114, s56, v114\n\t" \
  "v_mul_f32_e32 v115, s56, v115\n\t" \
  "v_mfma_f32_16x16x128_f8f6f4 a[84:87], v[144:151], v[200:207], a[84:87]\n\t" \
  "buffer_load_dwordx4 v0, s[36:39], s53 offen offset:2048 sc0 sc1 lds\n\t" \
  "v_min_f32_e32 v16, s60, v112\n\t" \
  "v_min_f32_e32 v17, s60, v113\n\t" \
  "v_min_f32_e32 v18, s60, v114\n\t" \
  "v_min_f32_e32 v19, s60, v115\n\t" \
  "v_mfma_f32_16x16x128_f8f6f4 a[88:91], v[144:151], v[208:215], a[88:91]\n\t" \
  "buffer_load_dwordx4 v0, s[36:39], s53 offen offset:3072 sc0 sc1 lds\n\t" \
  "s_add_u32 s53, s52, 507904\n\t" \
  "s_add_u32 m0, s48, 114688\n\t" \
  "v_exp_f32_e32 v16, v16\n\t" \
  "v_exp_f32_e32 v17, v17\n\t" \
  "v_mfma_f32_16x16x128_f8f6f4 a[92:95], v[144:151], v[216:223], a[92:95]\n\t" \
  "buffer_load_dwordx4 v0, s[36:39], s53 offen sc0 sc1 lds\n\t" \
  "v_mfma_f32_16x16x128_f8f6f4 a[124:127], v[152:159], v[216:223], a[124:127]\n\t" \
  "buffer_load_dwordx4 v0, s[36:39], s53 offen offset:1024 sc0 sc1 lds\n\t" \
  "v_exp_f32_e32 v18, v18\n\t" \
  "v_exp_f32_e32 v19, v19\n\t" \
  "v_mfma_f32_16x16x128_f8f6f4 a[120:123], v[152:159], v[208:215], a[120:123]\n\t" \
  "buffer_load_dwordx4 v0, s[36:39], s53 offen offset:2048 sc0 sc1 lds\n\t" \
  "v_fma_f32 v16, v16, s58, s58\n\t" \
  "v_fma_f32 v17, v17, s58, s58\n\t" \
  "v_fma_f32 v18, v18, s58, s58\n\t" \
  "v_fma_f32 v19, v19, s58, s58\n\t" \
  "v_mfma_f32_16x16x128_f8f6f4 a[116:119], v[152:159], v[200:207], a[116:119]\n\t" \
  "buffer_load_dwordx4 v0, s[36:39], s53 offen offset:3072 sc0 sc1 lds\n\t" \
  "v_mfma_f32_16x16x128_f8f6f4 a[112:115], v[152:159], v[192:199], a[112:115]\n\t" \
  "v_rcp_f32_e32 v16, v16\n\t" \
  "v_rcp_f32_e32 v17, v17\n\t" \
  "s_waitcnt lgkmcnt(0)\n\t" \
  "v_mfma_f32_16x16x128_f8f6f4 a[144:147], v[160:167], v[192:199], a[144:147]\n\t" \
  "s_waitcnt vmcnt(36)\n\t" \
  "s_barrier\n\t" \
  "v_mfma_f32_16x16x128_f8f6f4 a[148:151], v[160:167], v[200:207], a[148:151]\n\t" \
  "ds_read_b128 v[128:131], v6 offset:16384\n\t" \
  "ds_read_b128 v[132:135], v6 offset:24576\n\t" \
  "v_rcp_f32_e32 v18, v18\n\t" \
  "v_rcp_f32_e32 v19, v19\n\t" \
  "v_mfma_f32_16x16x128_f8f6f4 a[152:155], v[160:167], v[208:215], a[152:155]\n\t" \
  "ds_read_b128 v[136:139], v6 offset:17408\n\t" \
  "ds_read_b128 v[140:143], v6 offset:25600\n\t" \
  "v_mul_f32_e32 v112, v112, v16\n\t" \
  "v_mul_f32_e32 v113, v113, v17\n\t" \
  "v_mul_f32_e32 v114, v114, v18\n\t" \
  "v_mul_f32_e32 v115, v115, v19\n\t" \
  "v_mfma_f32_16x16x128_f8f6f4 a[156:159], v[160:167], v[216:223], a[156:159]\n\t" \
  "ds_read_b128 v[144:147], v6 offset:18432\n\t" \
  "ds_read_b128 v[148:151], v6 offset:26624\n\t" \
  "v_mfma_f32_16x16x128_f8f6f4 a[188:191], v[168:175], v[216:223], a[188:191]\n\t" \
  "ds_read_b128 v[152:155], v6 offset:19456\n\t" \
  "ds_read_b128 v[156:159], v6 offset:27648\n\t" \
  "s_xor_b32 s91, s68, 6422528\n\t" \
  "buffer_store_dwordx2 v[112:113], v5, s[40:43], s91 offen nt\n\t" \
  "v_mfma_f32_16x16x128_f8f6f4 a[184:187], v[168:175], v[208:215], a[184:187]\n\t" \
  "ds_read_b128 v[240:243], v127 offset:2048\n\t" \
  "ds_read_b128 v[244:247], v127 offset:10240\n\t" \
  "buffer_store_dwordx2 v[114:115], v5, s[40:43], s91 offen offset:128 nt\n\t" \
  "v_mfma_f32_16x16x128_f8f6f4 a[180:183], v[168:175], v[200:207], a[180:183]\n\t" \
  "ds_read_b128 v[248:251], v127 offset:3072\n\t" \
  "ds_read_b128 v[252:255], v127 offset:11264\n\t" \
  "v_accvgpr_read_b32 v116, a195\n\t" \
  "v_accvgpr_read_b32 v117, a199\n\t" \
  "v_accvgpr_read_b32 v118, a203\n\t" \
  "v_accvgpr_read_b32 v119, a207\n\t" \
  "v_mfma_f32_16x16x128_f8f6f4 a[176:179], v[168:175], v[192:199], a[176:179]\n\t" \
  "v_mfma_f32_16x16x128_f8f6f4 a[208:211], v[176:183], v[192:199], a[208:211]\n\t" \
  "v_mul_f32_e32 v116, s56, v116\n\t" \
  "v_mul_f32_e32 v117, s56, v117\n\t" \
  "v_mul_f32_e32 v118, s56, v118\n\t" \
  "v_mul_f32_e32 v119, s56, v119\n\t" \
  "v_mfma_f32_16x16x128_f8f6f4 a[212:215], v[176:183], v[200:207], a[212:215]\n\t" \
  "v_min_f32_e32 v20, s60, v116\n\t" \
  "v_min_f32_e32 v21, s60, v117\n\t" \
  "v_min_f32_e32 v22, s60, v118\n\t" \
  "v_min_f32_e32 v23, s60, v119\n\t" \
  "v_mfma_f32_16x16x128_f8f6f4 a[216:219], v[176:183], v[208:215], a[216:219]\n\t" \
  "v_exp_f32_e32 v20, v20\n\t" \
  "v_exp_f32_e32 v21, v21\n\t" \
  "v_mfma_f32_16x16x128_f8f6f4 a[220:223], v[176:183], v[216:223], a[220:223]\n\t" \
  "v_mfma_f32_16x16x128_f8f6f4 a[252:255], v[184:191], v[216:223], a[252:255]\n\t" \
  "v_exp_f32_e32 v22, v22\n\t" \
  "v_exp_f32_e32 v23, v23\n\t" \
  "v_mfma_f32_16x16x128_f8f6f4 a[248:251], v[184:191], v[208:215], a[248:251]\n\t" \
  "v_fma_f32 v20, v20, s58, s58\n\t" \
  "v_fma_f32 v21, v21, s58, s58\n\t" \
  "v_fma_f32 v22, v22, s58, s58\n\t" \
  "v_fma_f32 v23, v23, s58, s58\n\t" \
  "v_mfma_f32_16x16x128_f8f6f4 a[244:247], v[184:191], v[200:207], a[244:247]\n\t" \
  "v_mfma_f32_16x16x128_f8f6f4 a[240:243], v[184:191], v[192:199], a[240:243]\n\t" \
  "v_rcp_f32_e32 v20, v20\n\t" \
  "v_rcp_f32_e32 v21, v21\n\t" \
  "s_waitcnt lgkmcnt(0)\n\t" \
  "v_mfma_f32_16x16x128_f8f6f4 a[16:19], v[128:135], v[224:231], a[16:19]\n\t" \
  "s_waitcnt vmcnt(28)\n\t" \
  "s_barrier\n\t" \
  "v_mfma_f32_16x16x128_f8f6f4 a[20:23], v[128:135], v[232:239], a[20:23]\n\t" \
  "ds_read_b128 v[160:163], v2 offset:32768\n\t" \
  "ds_read_b128 v[164:167], v2 offset:40960\n\t" \
  "v_rcp_f32_e32 v22, v22\n\t" \
  "v_rcp_f32_e32 v23, v23\n\t" \
  "v_mfma_f32_16x16x128_f8f6f4 a[24:27], v[128:135], v[240:247], a[24:27]\n\t" \
  "ds_read_b128 v[168:171], v2 offset:33792\n\t" \
  "ds_read_b128 v[172:175], v2 offset:41984\n\t" \
  "v_mul_f32_e32 v116, v116, v20\n\t" \
  "v_mul_f32_e32 v117, v117, v21\n\t" \
  "v_mul_f32_e32 v118, v118, v22\n\t" \
  "v_mul_f32_e32 v119, v119, v23\n\t" \
  "v_mfma_f32_16x16x128_f8f6f4 a[28:31], v[128:135], v[248:255], a[28:31]\n\t" \
  "ds_read_b128 v[176:179], v2 offset:34816\n\t" \
  "ds_read_b128 v[180:183], v2 offset:43008\n\t" \
  "v_mfma_f32_16x16x128_f8f6f4 a[60:63], v[136:143], v[248:255], a[60:63]\n\t" \
  "ds_read_b128 v[184:187], v2 offset:35840\n\t" \
  "ds_read_b128 v[188:191], v2 offset:44032\n\t" \
  "s_xor_b32 s91, s68, 6488064\n\t" \
  "buffer_store_dwordx2 v[116:117], v5, s[40:43], s91 offen nt\n\t" \
  "v_mfma_f32_16x16x128_f8f6f4 a[56:59], v[136:143], v[240:247], a[56:59]\n\t" \
  "ds_read_b128 v[192:195], v3 offset:32768\n\t" \
  "ds_read_b128 v[196:199], v3 offset:40960\n\t" \
  "buffer_store_dwordx2 v[118:119], v5, s[40:43], s91 offen offset:128 nt\n\t" \
  "v_mfma_f32_16x16x128_f8f6f4 a[52:55], v[136:143], v[232:239], a[52:55]\n\t" \
  "ds_read_b128 v[200:203], v3 offset:33792\n\t" \
  "ds_read_b128 v[204:207], v3 offset:41984\n\t" \
  "v_accvgpr_read_b32 v120, a224\n\t" \
  "v_accvgpr_read_b32 v121, a228\n\t" \
  "v_accvgpr_read_b32 v122, a232\n\t" \
  "v_accvgpr_read_b32 v123, a236\n\t" \
  "v_mfma_f32_16x16x128_f8f6f4 a[48:51], v[136:143], v[224:231], a[48:51]\n\t" \
  "v_mfma_f32_16x16x128_f8f6f4 a[80:83], v[144:151], v[224:231], a[80:83]\n\t" \
  "v_mul_f32_e32 v120, s56, v120\n\t" \
  "v_mul_f32_e32 v121, s56, v121\n\t" \
  "v_mul_f32_e32 v122, s56, v122\n\t" \
  "v_mul_f32_e32 v123, s56, v123\n\t" \
  "v_mfma_f32_16x16x128_f8f6f4 a[84:87], v[144:151], v[232:239], a[84:87]\n\t" \
  "v_min_f32_e32 v24, s60, v120\n\t" \
  "v_min_f32_e32 v25, s60, v121\n\t" \
  "v_min_f32_e32 v26, s60, v122\n\t" \
  "v_min_f32_e32 v27, s60, v123\n\t" \
  "v_mfma_f32_16x16x128_f8f6f4 a[88:91], v[144:151], v[240:247], a[88:91]\n\t" \
  "v_mfma_f32_16x16x128_f8f6f4 a[92:95], v[144:151], v[248:255], a[92:95]\n\t" \
  "v_exp_f32_e32 v24, v24\n\t" \
  "v_exp_f32_e32 v25, v25\n\t" \
  "v_mfma_f32_16x16x128_f8f6f4 a[124:127], v[152:159], v[248:255], a[124:127]\n\t" \
  "v_exp_f32_e32 v26, v26\n\t" \
  "v_exp_f32_e32 v27, v27\n\t" \
  "v_mfma_f32_16x16x128_f8f6f4 a[120:123], v[152:159], v[240:247], a[120:123]\n\t" \
  "v_fma_f32 v24, v24, s58, s58\n\t" \
  "v_fma_f32 v25, v25, s58, s58\n\t" \
  "v_fma_f32 v26, v26, s58, s58\n\t" \
  "v_fma_f32 v27, v27, s58, s58\n\t" \
  "v_mfma_f32_16x16x128_f8f6f4 a[116:119], v[152:159], v[232:239], a[116:119]\n\t" \
  "v_mfma_f32_16x16x128_f8f6f4 a[112:115], v[152:159], v[224:231], a[112:115]\n\t" \
  "v_rcp_f32_e32 v24, v24\n\t" \
  "v_rcp_f32_e32 v25, v25\n\t" \
  "s_waitcnt lgkmcnt(0)\n\t" \
  "v_mfma_f32_16x16x128_f8f6f4 a[144:147], v[160:167], v[224:231], a[144:147]\n\t" \
  "s_waitcnt vmcnt(24)\n\t" \
  "s_barrier\n\t" \
  "v_mfma_f32_16x16x128_f8f6f4 a[148:151], v[160:167], v[232:239], a[148:151]\n\t" \
  "ds_read_b128 v[128:131], v1 offset:0\n\t" \
  "ds_read_b128 v[132:135], v1 offset:8192\n\t" \
  "v_rcp_f32_e32 v26, v26\n\t" \
  "v_rcp_f32_e32 v27, v27\n\t" \
  "v_mfma_f32_16x16x128_f8f6f4 a[152:155], v[160:167], v[240:247], a[152:155]\n\t" \
  "ds_read_b128 v[136:139], v1 offset:1024\n\t" \
  "ds_read_b128 v[140:143], v1 offset:9216\n\t" \
  "v_mul_f32_e32 v120, v120, v24\n\t" \
  "v_mul_f32_e32 v121, v121, v25\n\t" \
  "v_mul_f32_e32 v122, v122, v26\n\t" \
  "v_mul_f32_e32 v123, v123, v27\n\t" \
  "v_mfma_f32_16x16x128_f8f6f4 a[156:159], v[160:167], v[248:255], a[156:159]\n\t" \
  "ds_read_b128 v[144:147], v1 offset:2048\n\t" \
  "ds_read_b128 v[148:151], v1 offset:10240\n\t" \
  "v_mfma_f32_16x16x128_f8f6f4 a[188:191], v[168:175], v[248:255], a[188:191]\n\t" \
  "ds_read_b128 v[152:155], v1 offset:3072\n\t" \
  "ds_read_b128 v[156:159], v1 offset:11264\n\t" \
  "s_xor_b32 s91, s68, 7340032\n\t" \
  "buffer_store_dwordx2 v[120:121], v5, s[40:43], s91 offen nt\n\t" \
  "v_mfma_f32_16x16x128_f8f6f4 a[184:187], v[168:175], v[240:247], a[184:187]\n\t" \
  "ds_read_b128 v[208:211], v3 offset:34816\n\t" \
  "ds_read_b128 v[212:215], v3 offset:43008\n\t" \
  "buffer_store_dwordx2 v[122:123], v5, s[40:43], s91 offen offset:128 nt\n\t" \
  "v_mfma_f32_16x16x128_f8f6f4 a[180:183], v[168:175], v[232:239], a[180:183]\n\t" \
  "ds_read_b128 v[216:219], v3 offset:35840\n\t" \
  "ds_read_b128 v[220:223], v3 offset:44032\n\t" \
  "v_accvgpr_read_b32 v112, a225\n\t" \
  "v_accvgpr_read_b32 v113, a229\n\t" \
  "v_accvgpr_read_b32 v114, a233\n\t" \
  "v_accvgpr_read_b32 v115, a237\n\t" \
  "v_mfma_f32_16x16x128_f8f6f4 a[176:179], v[168:175], v[224:231], a[176:179]\n\t" \
  "v_mfma_f32_16x16x128_f8f6f4 a[208:211], v[176:183], v[224:231], a[208:211]\n\t" \
  "v_mul_f32_e32 v112, s56, v112\n\t" \
  "v_mul_f32_e32 v113, s56, v113\n\t" \
  "v_mul_f32_e32 v114, s56, v114\n\t" \
  "v_mul_f32_e32 v115, s56, v115\n\t" \
  "v_mfma_f32_16x16x128_f8f6f4 a[212:215], v[176:183], v[232:239], a[212:215]\n\t" \
  "v_min_f32_e32 v28, s60, v112\n\t" \
  "v_min_f32_e32 v29, s60, v113\n\t" \
  "v_min_f32_e32 v30, s60, v114\n\t" \
  "v_min_f32_e32 v31, s60, v115\n\t" \
  "v_mfma_f32_16x16x128_f8f6f4 a[216:219], v[176:183], v[240:247], a[216:219]\n\t" \
  "v_mfma_f32_16x16x128_f8f6f4 a[220:223], v[176:183], v[248:255], a[220:223]\n\t" \
  "v_exp_f32_e32 v28, v28\n\t" \
  "v_exp_f32_e32 v29, v29\n\t" \
  "v_mfma_f32_16x16x128_f8f6f4 a[252:255], v[184:191], v[248:255], a[252:255]\n\t" \
  "v_exp_f32_e32 v30, v30\n\t" \
  "v_exp_f32_e32 v31, v31\n\t" \
  "v_mfma_f32_16x16x128_f8f6f4 a[248:251], v[184:191], v[240:247], a[248:251]\n\t" \
  "v_fma_f32 v28, v28, s58, s58\n\t" \
  "v_fma_f32 v29, v29, s58, s58\n\t" \
  "v_fma_f32 v30, v30, s58, s58\n\t" \
  "v_fma_f32 v31, v31, s58, s58\n\t" \
  "v_mfma_f32_16x16x128_f8f6f4 a[244:247], v[184:191], v[232:239], a[244:247]\n\t" \
  "v_mfma_f32_16x16x128_f8f6f4 a[240:243], v[184:191], v[224:231], a[240:243]\n\t" \
  "v_rcp_f32_e32 v28, v28\n\t" \
  "v_rcp_f32_e32 v29, v29\n\t" \
  "s_waitcnt lgkmcnt(0)\n\t" \
  "v_mfma_f32_16x16x128_f8f6f4 a[16:19], v[128:135], v[192:199], a[16:19]\n\t" \
  "s_waitcnt vmcnt(16)\n\t" \
  "s_barrier\n\t" \
  "v_mfma_f32_16x16x128_f8f6f4 a[20:23], v[128:135], v[200:207], a[20:23]\n\t" \
  "ds_read_b128 v[160:163], v1 offset:49152\n\t" \
  "ds_read_b128 v[164:167], v1 offset:57344\n\t" \
  "v_rcp_f32_e32 v30, v30\n\t" \
  "v_rcp_f32_e32 v31, v31\n\t" \
  "v_mfma_f32_16x16x128_f8f6f4 a[24:27], v[128:135], v[208:215], a[24:27]\n\t" \
  "ds_read_b128 v[168:171], v1 offset:50176\n\t" \
  "ds_read_b128 v[172:175], v1 offset:58368\n\t" \
  "v_mul_f32_e32 v112, v112, v28\n\t" \
  "v_mul_f32_e32 v113, v113, v29\n\t" \
  "v_mul_f32_e32 v114, v114, v30\n\t" \
  "v_mul_f32_e32 v115, v115, v31\n\t" \
  "v_mfma_f32_16x16x128_f8f6f4 a[28:31], v[128:135], v[216:223], a[28:31]\n\t" \
  "ds_read_b128 v[176:179], v1 offset:51200\n\t" \
  "ds_read_b128 v[180:183], v1 offset:59392\n\t" \
  "v_mfma_f32_16x16x128_f8f6f4 a[60:63], v[136:143], v[216:223], a[60:63]\n\t" \
  "ds_read_b128 v[184:187], v1 offset:52224\n\t" \
  "ds_read_b128 v[188:191], v1 offset:60416\n\t" \
  "s_xor_b32 s91, s68, 7405568\n\t" \
  "buffer_store_dwordx2 v[112:113], v5, s[40:43], s91 offen nt\n\t" \
  "v_mfma_f32_16x16x128_f8f6f4 a[56:59], v[136:143], v[208:215], a[56:59]\n\t" \
  "ds_read_b128 v[224:227], v4 offset:0\n\t" \
  "ds_read_b128 v[228:231], v4 offset:8192\n\t" \
  "buffer_store_dwordx2 v[114:115], v5, s[40:43], s91 offen offset:128 nt\n\t" \
  "v_mfma_f32_16x16x128_f8f6f4 a[52:55], v[136:143], v[200:207], a[52:55]\n\t" \
  "ds_read_b128 v[232:235], v4 offset:1024\n\t" \
  "ds_read_b128 v[236:239], v4 offset:9216\n\t" \
  "v_mfma_f32_16x16x128_f8f6f4 a[48:51], v[136:143], v[192:199], a[48:51]\n\t" \
  "v_accvgpr_read_b32 v116, a226\n\t" \
  "v_accvgpr_read_b32 v117, a230\n\t" \
  "v_accvgpr_read_b32 v118, a234\n\t" \
  "v_accvgpr_read_b32 v119, a238\n\t" \
  "v_mfma_f32_16x16x128_f8f6f4 a[80:83], v[144:151], v[192:199], a[80:83]\n\t" \
  "v_mul_f32_e32 v116, s56, v116\n\t" \
  "v_mul_f32_e32 v117, s56, v117\n\t" \
  "v_mul_f32_e32 v118, s56, v118\n\t" \
  "v_mul_f32_e32 v119, s56, v119\n\t" \
  "v_mfma_f32_16x16x128_f8f6f4 a[84:87], v[144:151], v[200:207], a[84:87]\n\t" \
  "v_min_f32_e32 v12, s60, v116\n\t" \
  "v_min_f32_e32 v13, s60, v117\n\t" \
  "v_min_f32_e32 v14, s60, v118\n\t" \
  "v_min_f32_e32 v15, s60, v119\n\t" \
  "v_mfma_f32_16x16x128_f8f6f4 a[88:91], v[144:151], v[208:215], a[88:91]\n\t" \
  "v_mfma_f32_16x16x128_f8f6f4 a[92:95], v[144:151], v[216:223], a[92:95]\n\t" \
  "v_exp_f32_e32 v12, v12\n\t" \
  "v_exp_f32_e32 v13, v13\n\t" \
  "v_mfma_f32_16x16x128_f8f6f4 a[124:127], v[152:159], v[216:223], a[124:127]\n\t" \
  "v_exp_f32_e32 v14, v14\n\t" \
  "v_exp_f32_e32 v15, v15\n\t" \
  "v_mfma_f32_16x16x128_f8f6f4 a[120:123], v[152:159], v[208:215], a[120:123]\n\t" \
  "v_fma_f32 v12, v12, s58, s58\n\t" \
  "v_fma_f32 v13, v13, s58, s58\n\t" \
  "v_fma_f32 v14, v14, s58, s58\n\t" \
  "v_fma_f32 v15, v15, s58, s58\n\t" \
  "v_mfma_f32_16x16x128_f8f6f4 a[116:119], v[152:159], v[200:207], a[116:119]\n\t" \
  "v_mfma_f32_16x16x128_f8f6f4 a[112:115], v[152:159], v[192:199], a[112:115]\n\t" \
  "v_rcp_f32_e32 v12, v12\n\t" \
  "v_rcp_f32_e32 v13, v13\n\t" \
  "s_waitcnt lgkmcnt(0)\n\t" \
  "v_mfma_f32_16x16x128_f8f6f4 a[144:147], v[160:167], v[192:199], a[144:147]\n\t" \
  "s_waitcnt vmcnt(12)\n\t" \
  "s_barrier\n\t" \
  "v_mfma_f32_16x16x128_f8f6f4 a[148:151], v[160:167], v[200:207], a[148:151]\n\t" \
  "ds_read_b128 v[128:131], v1 offset:16384\n\t" \
  "ds_read_b128 v[132:135], v1 offset:24576\n\t" \
  "v_rcp_f32_e32 v14, v14\n\t" \
  "v_rcp_f32_e32 v15, v15\n\t" \
  "v_mfma_f32_16x16x128_f8f6f4 a[152:155], v[160:167], v[208:215], a[152:155]\n\t" \
  "ds_read_b128 v[136:139], v1 offset:17408\n\t" \
  "ds_read_b128 v[140:143], v1 offset:25600\n\t" \
  "v_mul_f32_e32 v116, v116, v12\n\t" \
  "v_mul_f32_e32 v117, v117, v13\n\t" \
  "v_mul_f32_e32 v118, v118, v14\n\t" \
  "v_mul_f32_e32 v119, v119, v15\n\t" \
  "v_mfma_f32_16x16x128_f8f6f4 a[156:159], v[160:167], v[216:223], a[156:159]\n\t" \
  "ds_read_b128 v[144:147], v1 offset:18432\n\t" \
  "ds_read_b128 v[148:151], v1 offset:26624\n\t" \
  "v_mfma_f32_16x16x128_f8f6f4 a[188:191], v[168:175], v[216:223], a[188:191]\n\t" \
  "ds_read_b128 v[152:155], v1 offset:19456\n\t" \
  "ds_read_b128 v[156:159], v1 offset:27648\n\t" \
  "s_xor_b32 s91, s68, 7471104\n\t" \
  "buffer_store_dwordx2 v[116:117], v5, s[40:43], s91 offen nt\n\t" \
  "v_mfma_f32_16x16x128_f8f6f4 a[184:187], v[168:175], v[208:215], a[184:187]\n\t" \
  "ds_read_b128 v[240:243], v4 offset:2048\n\t" \
  "ds_read_b128 v[244:247], v4 offset:10240\n\t" \
  "buffer_store_dwordx2 v[118:119], v5, s[40:43], s91 offen offset:128 nt\n\t" \
  "v_mfma_f32_16x16x128_f8f6f4 a[180:183], v[168:175], v[200:207], a[180:183]\n\t" \
  "ds_read_b128 v[248:251], v4 offset:3072\n\t" \
  "ds_read_b128 v[252:255], v4 offset:11264\n\t" \
  "v_mfma_f32_16x16x128_f8f6f4 a[176:179], v[168:175], v[192:199], a[176:179]\n\t" \
  "v_accvgpr_read_b32 v120, a227\n\t" \
  "v_accvgpr_read_b32 v121, a231\n\t" \
  "v_accvgpr_read_b32 v122, a235\n\t" \
  "v_accvgpr_read_b32 v123, a239\n\t" \
  "v_mfma_f32_16x16x128_f8f6f4 a[208:211], v[176:183], v[192:199], a[208:211]\n\t" \
  "v_mul_f32_e32 v120, s56, v120\n\t" \
  "v_mul_f32_e32 v121, s56, v121\n\t" \
  "v_mul_f32_e32 v122, s56, v122\n\t" \
  "v_mul_f32_e32 v123, s56, v123\n\t" \
  "v_mfma_f32_16x16x128_f8f6f4 a[212:215], v[176:183], v[200:207], a[212:215]\n\t" \
  "v_min_f32_e32 v16, s60, v120\n\t" \
  "v_min_f32_e32 v17, s60, v121\n\t" \
  "v_min_f32_e32 v18, s60, v122\n\t" \
  "v_min_f32_e32 v19, s60, v123\n\t" \
  "v_mfma_f32_16x16x128_f8f6f4 a[216:219], v[176:183], v[208:215], a[216:219]\n\t" \
  "v_mfma_f32_16x16x128_f8f6f4 a[220:223], v[176:183], v[216:223], a[220:223]\n\t" \
  "v_exp_f32_e32 v16, v16\n\t" \
  "v_exp_f32_e32 v17, v17\n\t" \
  "v_mfma_f32_16x16x128_f8f6f4 a[252:255], v[184:191], v[216:223], a[252:255]\n\t" \
  "v_exp_f32_e32 v18, v18\n\t" \
  "v_exp_f32_e32 v19, v19\n\t" \
  "v_mfma_f32_16x16x128_f8f6f4 a[248:251], v[184:191], v[208:215], a[248:251]\n\t" \
  "v_fma_f32 v16, v16, s58, s58\n\t" \
  "v_fma_f32 v17, v17, s58, s58\n\t" \
  "v_fma_f32 v18, v18, s58, s58\n\t" \
  "v_fma_f32 v19, v19, s58, s58\n\t" \
  "v_mfma_f32_16x16x128_f8f6f4 a[244:247], v[184:191], v[200:207], a[244:247]\n\t" \
  "v_mfma_f32_16x16x128_f8f6f4 a[240:243], v[184:191], v[192:199], a[240:243]\n\t" \
  "v_rcp_f32_e32 v16, v16\n\t" \
  "v_rcp_f32_e32 v17, v17\n\t" \
  "s_waitcnt lgkmcnt(0)\n\t" \
  "v_mfma_f32_16x16x128_f8f6f4 a[16:19], v[128:135], v[224:231], a[16:19]\n\t" \
  "s_waitcnt vmcnt(10)\n\t" \
  "s_barrier\n\t" \
  "v_mfma_f32_16x16x128_f8f6f4 a[20:23], v[128:135], v[232:239], a[20:23]\n\t" \
  "ds_read_b128 v[160:163], v2 offset:49152\n\t" \
  "ds_read_b128 v[164:167], v2 offset:57344\n\t" \
  "v_rcp_f32_e32 v18, v18\n\t" \
  "v_rcp_f32_e32 v19, v19\n\t" \
  "v_mfma_f32_16x16x128_f8f6f4 a[24:27], v[128:135], v[240:247], a[24:27]\n\t" \
  "ds_read_b128 v[168:171], v2 offset:50176\n\t" \
  "ds_read_b128 v[172:175], v2 offset:58368\n\t" \
  "v_mul_f32_e32 v120, v120, v16\n\t" \
  "v_mul_f32_e32 v121, v121, v17\n\t" \
  "v_mul_f32_e32 v122, v122, v18\n\t" \
  "v_mul_f32_e32 v123, v123, v19\n\t" \
  "v_mfma_f32_16x16x128_f8f6f4 a[28:31], v[128:135], v[248:255], a[28:31]\n\t" \
  "ds_read_b128 v[176:179], v2 offset:51200\n\t" \
  "ds_read_b128 v[180:183], v2 offset:59392\n\t" \
  "v_mfma_f32_16x16x128_f8f6f4 a[60:63], v[136:143], v[248:255], a[60:63]\n\t" \
  "ds_read_b128 v[184:187], v2 offset:52224\n\t" \
  "ds_read_b128 v[188:191], v2 offset:60416\n\t" \
  "s_xor_b32 s91, s68, 7536640\n\t" \
  "buffer_store_dwordx2 v[120:121], v5, s[40:43], s91 offen nt\n\t" \
  "v_mfma_f32_16x16x128_f8f6f4 a[56:59], v[136:143], v[240:247], a[56:59]\n\t" \
  "buffer_store_dwordx2 v[122:123], v5, s[40:43], s91 offen offset:128 nt\n\t" \
  "v_mfma_f32_16x16x128_f8f6f4 a[52:55], v[136:143], v[232:239], a[52:55]\n\t" \
  "v_mfma_f32_16x16x128_f8f6f4 a[48:51], v[136:143], v[224:231], a[48:51]\n\t" \
  "v_mfma_f32_16x16x128_f8f6f4 a[80:83], v[144:151], v[224:231], a[80:83]\n\t" \
  "v_mfma_f32_16x16x128_f8f6f4 a[84:87], v[144:151], v[232:239], a[84:87]\n\t" \
  "v_mfma_f32_16x16x128_f8f6f4 a[88:91], v[144:151], v[240:247], a[88:91]\n\t" \
  "v_mfma_f32_16x16x128_f8f6f4 a[92:95], v[144:151], v[248:255], a[92:95]\n\t" \
  "v_mfma_f32_16x16x128_f8f6f4 a[124:127], v[152:159], v[248:255], a[124:127]\n\t" \
  "v_mfma_f32_16x16x128_f8f6f4 a[120:123], v[152:159], v[240:247], a[120:123]\n\t" \
  "v_mfma_f32_16x16x128_f8f6f4 a[116:119], v[152:159], v[232:239], a[116:119]\n\t" \
  "v_mfma_f32_16x16x128_f8f6f4 a[112:115], v[152:159], v[224:231], a[112:115]\n\t" \
  "s_waitcnt lgkmcnt(0)\n\t" \
  "v_mfma_f32_16x16x128_f8f6f4 a[144:147], v[160:167], v[224:231], a[144:147]\n\t" \
  "s_barrier\n\t" \
  "v_mfma_f32_16x16x128_f8f6f4 a[148:151], v[160:167], v[232:239], a[148:151]\n\t" \
  "v_mfma_f32_16x16x128_f8f6f4 a[152:155], v[160:167], v[240:247], a[152:155]\n\t" \
  "v_mfma_f32_16x16x128_f8f6f4 a[156:159], v[160:167], v[248:255], a[156:159]\n\t" \
  "v_mfma_f32_16x16x128_f8f6f4 a[188:191], v[168:175], v[248:255], a[188:191]\n\t" \
  "v_mfma_f32_16x16x128_f8f6f4 a[184:187], v[168:175], v[240:247], a[184:187]\n\t" \
  "v_mfma_f32_16x16x128_f8f6f4 a[180:183], v[168:175], v[232:239], a[180:183]\n\t" \
  "v_mfma_f32_16x16x128_f8f6f4 a[176:179], v[168:175], v[224:231], a[176:179]\n\t" \
  "v_mfma_f32_16x16x128_f8f6f4 a[208:211], v[176:183], v[224:231], a[208:211]\n\t" \
  "v_mfma_f32_16x16x128_f8f6f4 a[212:215], v[176:183], v[232:239], a[212:215]\n\t" \
  "v_mfma_f32_16x16x128_f8f6f4 a[216:219], v[176:183], v[240:247], a[216:219]\n\t" \
  "v_mfma_f32_16x16x128_f8f6f4 a[220:223], v[176:183], v[248:255], a[220:223]\n\t" \
  "v_mfma_f32_16x16x128_f8f6f4 a[252:255], v[184:191], v[248:255], a[252:255]\n\t" \
  "v_mfma_f32_16x16x128_f8f6f4 a[248:251], v[184:191], v[240:247], a[248:251]\n\t" \
  "v_mfma_f32_16x16x128_f8f6f4 a[244:247], v[184:191], v[232:239], a[244:247]\n\t" \
  "v_mfma_f32_16x16x128_f8f6f4 a[240:243], v[184:191], v[224:231], a[240:243]\n\t" \
  "s_waitcnt lgkmcnt(0)\n\t" \
  "global_load_dword v7, v10, s[86:87] sc1\n\t" \
  "s_nop 7\n\t" \
  "v_accvgpr_read_b32 v32, a16\n\t" \
  "v_accvgpr_read_b32 v33, a20\n\t" \
  "v_accvgpr_read_b32 v34, a24\n\t" \
  "v_accvgpr_read_b32 v35, a28\n\t" \
  "v_pk_mul_f32 v[32:33], v[32:33], s[56:57]\n\t" \
  "v_pk_mul_f32 v[34:35], v[34:35], s[56:57]\n\t" \
  "v_min_f32_e32 v12, s60, v32\n\t" \
  "v_min_f32_e32 v13, s60, v33\n\t" \
  "v_min_f32_e32 v14, s60, v34\n\t" \
  "v_min_f32_e32 v15, s60, v35\n\t" \
  "v_accvgpr_read_b32 v36, a17\n\t" \
  "v_accvgpr_read_b32 v37, a21\n\t" \
  "v_accvgpr_read_b32 v38, a25\n\t" \
  "v_accvgpr_read_b32 v39, a29\n\t" \
  "v_pk_mul_f32 v[36:37], v[36:37], s[56:57]\n\t" \
  "v_pk_mul_f32 v[38:39], v[38:39], s[56:57]\n\t" \
  "v_min_f32_e32 v16, s60, v36\n\t" \
  "v_min_f32_e32 v17, s60, v37\n\t" \
  "v_min_f32_e32 v18, s60, v38\n\t" \
  "v_min_f32_e32 v19, s60, v39\n\t" \
  "v_exp_f32_e32 v12, v12\n\t" \
  "v_exp_f32_e32 v13, v13\n\t" \
  "v_exp_f32_e32 v14, v14\n\t" \
  "v_exp_f32_e32 v15, v15\n\t" \
  "v_accvgpr_read_b32 v40, a18\n\t" \
  "v_accvgpr_read_b32 v41, a22\n\t" \
  "v_accvgpr_read_b32 v42, a26\n\t" \
  "v_accvgpr_read_b32 v43, a30\n\t" \
  "v_pk_mul_f32 v[40:41], v[40:41], s[56:57]\n\t" \
  "v_pk_mul_f32 v[42:43], v[42:43], s[56:57]\n\t" \
  "v_min_f32_e32 v20, s60, v40\n\t" \
  "v_min_f32_e32 v21, s60, v41\n\t" \
  "v_min_f32_e32 v22, s60, v42\n\t" \
  "v_min_f32_e32 v23, s60, v43\n\t" \
  "v_exp_f32_e32 v16, v16\n\t" \
  "v_exp_f32_e32 v17, v17\n\t" \
  "v_exp_f32_e32 v18, v18\n\t" \
  "v_exp_f32_e32 v19, v19\n\t" \
  "v_pk_fma_f32 v[12:13], v[12:13], s[58:59], s[58:59]\n\t" \
  "v_pk_fma_f32 v[14:15], v[14:15], s[58:59], s[58:59]\n\t" \
  "v_rcp_f32_e32 v12, v12\n\t" \
  "v_rcp_f32_e32 v13, v13\n\t" \
  "v_rcp_f32_e32 v14, v14\n\t" \
  "v_rcp_f32_e32 v15, v15\n\t" \
  "v_accvgpr_read_b32 v44, a19\n\t" \
  "v_accvgpr_read_b32 v45, a23\n\t" \
  "v_accvgpr_read_b32 v46, a27\n\t" \
  "v_accvgpr_read_b32 v47, a31\n\t" \
  "v_pk_mul_f32 v[44:45], v[44:45], s[56:57]\n\t" \
  "v_pk_mul_f32 v[46:47], v[46:47], s[56:57]\n\t" \
  "v_min_f32_e32 v24, s60, v44\n\t" \
  "v_min_f32_e32 v25, s60, v45\n\t" \
  "v_min_f32_e32 v26, s60, v46\n\t" \
  "v_min_f32_e32 v27, s60, v47\n\t" \
  "v_exp_f32_e32 v20, v20\n\t" \
  "v_exp_f32_e32 v21, v21\n\t" \
  "v_exp_f32_e32 v22, v22\n\t" \
  "v_exp_f32_e32 v23, v23\n\t" \
  "v_pk_fma_f32 v[16:17], v[16:17], s[58:59], s[58:59]\n\t" \
  "v_pk_fma_f32 v[18:19], v[18:19], s[58:59], s[58:59]\n\t" \
  "v_rcp_f32_e32 v16, v16\n\t" \
  "v_rcp_f32_e32 v17, v17\n\t" \
  "v_rcp_f32_e32 v18, v18\n\t" \
  "v_rcp_f32_e32 v19, v19\n\t" \
  "v_pk_mul_f32 v[32:33], v[32:33], v[12:13]\n\t" \
  "v_pk_mul_f32 v[34:35], v[34:35], v[14:15]\n\t" \
  "s_xor_b32 s55, s67, 0\n\t" \
  "buffer_store_dwordx2 v[32:33], v5, s[40:43], s55 offen nt\n\t" \
  "buffer_store_dwordx2 v[34:35], v5, s[40:43], s55 offen offset:128 nt\n\t" \
  "v_accvgpr_read_b32 v48, a48\n\t" \
  "v_accvgpr_read_b32 v49, a52\n\t" \
  "v_accvgpr_read_b32 v50, a56\n\t" \
  "v_accvgpr_read_b32 v51, a60\n\t" \
  "v_pk_mul_f32 v[48:49], v[48:49], s[56:57]\n\t" \
  "v_pk_mul_f32 v[50:51], v[50:51], s[56:57]\n\t" \
  "v_min_f32_e32 v28, s60, v48\n\t" \
  "v_min_f32_e32 v29, s60, v49\n\t" \
  "v_min_f32_e32 v30, s60, v50\n\t" \
  "v_min_f32_e32 v31, s60, v51\n\t" \
  "v_exp_f32_e32 v24, v24\n\t" \
  "v_exp_f32_e32 v25, v25\n\t" \
  "v_exp_f32_e32 v26, v26\n\t" \
  "v_exp_f32_e32 v27, v27\n\t" \
  "v_pk_fma_f32 v[20:21], v[20:21], s[58:59], s[58:59]\n\t" \
  "v_pk_fma_f32 v[22:23], v[22:23], s[58:59], s[58:59]\n\t" \
  "v_rcp_f32_e32 v20, v20\n\t" \
  "v_rcp_f32_e32 v21, v21\n\t" \
  "v_rcp_f32_e32 v22, v22\n\t" \
  "v_rcp_f32_e32 v23, v23\n\t" \
  "v_pk_mul_f32 v[36:37], v[36:37], v[16:17]\n\t" \
  "v_pk_mul_f32 v[38:39], v[38:39], v[18:19]\n\t" \
  "s_xor_b32 s55, s67, 65536\n\t" \
  "buffer_store_dwordx2 v[36:37], v5, s[40:43], s55 offen nt\n\t" \
  "buffer_store_dwordx2 v[38:39], v5, s[40:43], s55 offen offset:128 nt\n\t" \
  "v_accvgpr_read_b32 v52, a49\n\t" \
  "v_accvgpr_read_b32 v53, a53\n\t" \
  "v_accvgpr_read_b32 v54, a57\n\t" \
  "v_accvgpr_read_b32 v55, a61\n\t" \
  "v_pk_mul_f32 v[52:53], v[52:53], s[56:57]\n\t" \
  "v_pk_mul_f32 v[54:55], v[54:55], s[56:57]\n\t" \
  "v_min_f32_e32 v12, s60, v52\n\t" \
  "v_min_f32_e32 v13, s60, v53\n\t" \
  "v_min_f32_e32 v14, s60, v54\n\t" \
  "v_min_f32_e32 v15, s60, v55\n\t" \
  "v_exp_f32_e32 v28, v28\n\t" \
  "v_exp_f32_e32 v29, v29\n\t" \
  "v_exp_f32_e32 v30, v30\n\t" \
  "v_exp_f32_e32 v31, v31\n\t" \
  "v_pk_fma_f32 v[24:25], v[24:25], s[58:59], s[58:59]\n\t" \
  "v_pk_fma_f32 v[26:27], v[26:27], s[58:59], s[58:59]\n\t" \
  "v_rcp_f32_e32 v24, v24\n\t" \
  "v_rcp_f32_e32 v25, v25\n\t" \
  "v_rcp_f32_e32 v26, v26\n\t" \
  "v_rcp_f32_e32 v27, v27\n\t" \
  "v_pk_mul_f32 v[40:41], v[40:41], v[20:21]\n\t" \
  "v_pk_mul_f32 v[42:43], v[42:43], v[22:23]\n\t" \
  "s_xor_b32 s55, s67, 131072\n\t" \
  "buffer_store_dwordx2 v[40:41], v5, s[40:43], s55 offen nt\n\t" \
  "buffer_store_dwordx2 v[42:43], v5, s[40:43], s55 offen offset:128 nt\n\t" \
  "v_accvgpr_read_b32 v56, a50\n\t" \
  "v_accvgpr_read_b32 v57, a54\n\t" \
  "v_accvgpr_read_b32 v58, a58\n\t" \
  "v_accvgpr_read_b32 v59, a62\n\t" \
  "v_pk_mul_f32 v[56:57], v[56:57], s[56:57]\n\t" \
  "v_pk_mul_f32 v[58:59], v[58:59], s[56:57]\n\t" \
  "v_min_f32_e32 v16, s60, v56\n\t" \
  "v_min_f32_e32 v17, s60, v57\n\t" \
  "v_min_f32_e32 v18, s60, v58\n\t" \
  "v_min_f32_e32 v19, s60, v59\n\t" \
  "v_exp_f32_e32 v12, v12\n\t" \
  "v_exp_f32_e32 v13, v13\n\t" \
  "v_exp_f32_e32 v14, v14\n\t" \
  "v_exp_f32_e32 v15, v15\n\t" \
  "v_pk_fma_f32 v[28:29], v[28:29], s[58:59], s[58:59]\n\t" \
  "v_pk_fma_f32 v[30:31], v[30:31], s[58:59], s[58:59]\n\t" \
  "v_rcp_f32_e32 v28, v28\n\t" \
  "v_rcp_f32_e32 v29, v29\n\t" \
  "v_rcp_f32_e32 v30, v30\n\t" \
  "v_rcp_f32_e32 v31, v31\n\t" \
  "v_pk_mul_f32 v[44:45], v[44:45], v[24:25]\n\t" \
  "v_pk_mul_f32 v[46:47], v[46:47], v[26:27]\n\t" \
  "s_xor_b32 s55, s67, 196608\n\t" \
  "buffer_store_dwordx2 v[44:45], v5, s[40:43], s55 offen nt\n\t" \
  "buffer_store_dwordx2 v[46:47], v5, s[40:43], s55 offen offset:128 nt\n\t" \
  "v_accvgpr_read_b32 v60, a51\n\t" \
  "v_accvgpr_read_b32 v61, a55\n\t" \
  "v_accvgpr_read_b32 v62, a59\n\t" \
  "v_accvgpr_read_b32 v63, a63\n\t" \
  "v_pk_mul_f32 v[60:61], v[60:61], s[56:57]\n\t" \
  "v_pk_mul_f32 v[62:63], v[62:63], s[56:57]\n\t" \
  "v_min_f32_e32 v20, s60, v60\n\t" \
  "v_min_f32_e32 v21, s60, v61\n\t" \
  "v_min_f32_e32 v22, s60, v62\n\t" \
  "v_min_f32_e32 v23, s60, v63\n\t" \
  "v_exp_f32_e32 v16, v16\n\t" \
  "v_exp_f32_e32 v17, v17\n\t" \
  "v_exp_f32_e32 v18, v18\n\t" \
  "v_exp_f32_e32 v19, v19\n\t" \
  "v_pk_fma_f32 v[12:13], v[12:13], s[58:59], s[58:59]\n\t" \
  "v_pk_fma_f32 v[14:15], v[14:15], s[58:59], s[58:59]\n\t" \
  "v_rcp_f32_e32 v12, v12\n\t" \
  "v_rcp_f32_e32 v13, v13\n\t" \
  "v_rcp_f32_e32 v14, v14\n\t" \
  "v_rcp_f32_e32 v15, v15\n\t" \
  "v_pk_mul_f32 v[48:49], v[48:49], v[28:29]\n\t" \
  "v_pk_mul_f32 v[50:51], v[50:51], v[30:31]\n\t" \
  "s_xor_b32 s55, s67, 1048576\n\t" \
  "buffer_store_dwordx2 v[48:49], v5, s[40:43], s55 offen nt\n\t" \
  "buffer_store_dwordx2 v[50:51], v5, s[40:43], s55 offen offset:128 nt\n\t" \
  "v_accvgpr_read_b32 v32, a80\n\t" \
  "v_accvgpr_read_b32 v33, a84\n\t" \
  "v_accvgpr_read_b32 v34, a88\n\t" \
  "v_accvgpr_read_b32 v35, a92\n\t" \
  "v_pk_mul_f32 v[32:33], v[32:33], s[56:57]\n\t" \
  "v_pk_mul_f32 v[34:35], v[34:35], s[56:57]\n\t" \
  "v_min_f32_e32 v24, s60, v32\n\t" \
  "v_min_f32_e32 v25, s60, v33\n\t" \
  "v_min_f32_e32 v26, s60, v34\n\t" \
  "v_min_f32_e32 v27, s60, v35\n\t" \
  "v_exp_f32_e32 v20, v20\n\t" \
  "v_exp_f32_e32 v21, v21\n\t" \
  "v_exp_f32_e32 v22, v22\n\t" \
  "v_exp_f32_e32 v23, v23\n\t" \
  "v_pk_fma_f32 v[16:17], v[16:17], s[58:59], s[58:59]\n\t" \
  "v_pk_fma_f32 v[18:19], v[18:19], s[58:59], s[58:59]\n\t" \
  "v_rcp_f32_e32 v16, v16\n\t" \
  "v_rcp_f32_e32 v17, v17\n\t" \
  "v_rcp_f32_e32 v18, v18\n\t" \
  "v_rcp_f32_e32 v19, v19\n\t" \
  "v_pk_mul_f32 v[52:53], v[52:53], v[12:13]\n\t" \
  "v_pk_mul_f32 v[54:55], v[54:55], v[14:15]\n\t" \
  "s_xor_b32 s55, s67, 1114112\n\t" \
  "buffer_store_dwordx2 v[52:53], v5, s[40:43], s55 offen nt\n\t" \
  "buffer_store_dwordx2 v[54:55], v5, s[40:43], s55 offen offset:128 nt\n\t" \
  "v_accvgpr_read_b32 v36, a81\n\t" \
  "v_accvgpr_read_b32 v37, a85\n\t" \
  "v_accvgpr_read_b32 v38, a89\n\t" \
  "v_accvgpr_read_b32 v39, a93\n\t" \
  "v_pk_mul_f32 v[36:37], v[36:37], s[56:57]\n\t" \
  "v_pk_mul_f32 v[38:39], v[38:39], s[56:57]\n\t" \
  "v_min_f32_e32 v28, s60, v36\n\t" \
  "v_min_f32_e32 v29, s60, v37\n\t" \
  "v_min_f32_e32 v30, s60, v38\n\t" \
  "v_min_f32_e32 v31, s60, v39\n\t" \
  "v_exp_f32_e32 v24, v24\n\t" \
  "v_exp_f32_e32 v25, v25\n\t" \
  "v_exp_f32_e32 v26, v26\n\t" \
  "v_exp_f32_e32 v27, v27\n\t" \
  "v_pk_fma_f32 v[20:21], v[20:21], s[58:59], s[58:59]\n\t" \
  "v_pk_fma_f32 v[22:23], v[22:23], s[58:59], s[58:59]\n\t" \
  "v_rcp_f32_e32 v20, v20\n\t" \
  "v_rcp_f32_e32 v21, v21\n\t" \
  "v_rcp_f32_e32 v22, v22\n\t" \
  "v_rcp_f32_e32 v23, v23\n\t" \
  "v_pk_mul_f32 v[56:57], v[56:57], v[16:17]\n\t" \
  "v_pk_mul_f32 v[58:59], v[58:59], v[18:19]\n\t" \
  "s_xor_b32 s55, s67, 1179648\n\t" \
  "buffer_store_dwordx2 v[56:57], v5, s[40:43], s55 offen nt\n\t" \
  "buffer_store_dwordx2 v[58:59], v5, s[40:43], s55 offen offset:128 nt\n\t" \
  "v_accvgpr_read_b32 v40, a82\n\t" \
  "v_accvgpr_read_b32 v41, a86\n\t" \
  "v_accvgpr_read_b32 v42, a90\n\t" \
  "v_accvgpr_read_b32 v43, a94\n\t" \
  "v_pk_mul_f32 v[40:41], v[40:41], s[56:57]\n\t" \
  "v_pk_mul_f32 v[42:43], v[42:43], s[56:57]\n\t" \
  "v_min_f32_e32 v12, s60, v40\n\t" \
  "v_min_f32_e32 v13, s60, v41\n\t" \
  "v_min_f32_e32 v14, s60, v42\n\t" \
  "v_min_f32_e32 v15, s60, v43\n\t" \
  "v_exp_f32_e32 v28, v28\n\t" \
  "v_exp_f32_e32 v29, v29\n\t" \
  "v_exp_f32_e32 v30, v30\n\t" \
  "v_exp_f32_e32 v31, v31\n\t" \
  "v_pk_fma_f32 v[24:25], v[24:25], s[58:59], s[58:59]\n\t" \
  "v_pk_fma_f32 v[26:27], v[26:27], s[58:59], s[58:59]\n\t" \
  "v_rcp_f32_e32 v24, v24\n\t" \
  "v_rcp_f32_e32 v25, v25\n\t" \
  "v_rcp_f32_e32 v26, v26\n\t" \
  "v_rcp_f32_e32 v27, v27\n\t" \
  "v_pk_mul_f32 v[60:61], v[60:61], v[20:21]\n\t" \
  "v_pk_mul_f32 v[62:63], v[62:63], v[22:23]\n\t" \
  "s_xor_b32 s55, s67, 1245184\n\t" \
  "buffer_store_dwordx2 v[60:61], v5, s[40:43], s55 offen nt\n\t" \
  "buffer_store_dwordx2 v[62:63], v5, s[40:43], s55 offen offset:128 nt\n\t" \
  "v_accvgpr_read_b32 v44, a83\n\t" \
  "v_accvgpr_read_b32 v45, a87\n\t" \
  "v_accvgpr_read_b32 v46, a91\n\t" \
  "v_accvgpr_read_b32 v47, a95\n\t" \
  "v_pk_mul_f32 v[44:45], v[44:45], s[56:57]\n\t" \
  "v_pk_mul_f32 v[46:47], v[46:47], s[56:57]\n\t" \
  "v_min_f32_e32 v16, s60, v44\n\t" \
  "v_min_f32_e32 v17, s60, v45\n\t" \
  "v_min_f32_e32 v18, s60, v46\n\t" \
  "v_min_f32_e32 v19, s60, v47\n\t" \
  "v_exp_f32_e32 v12, v12\n\t" \
  "v_exp_f32_e32 v13, v13\n\t" \
  "v_exp_f32_e32 v14, v14\n\t" \
  "v_exp_f32_e32 v15, v15\n\t" \
  "v_pk_fma_f32 v[28:29], v[28:29], s[58:59], s[58:59]\n\t" \
  "v_pk_fma_f32 v[30:31], v[30:31], s[58:59], s[58:59]\n\t" \
  "v_rcp_f32_e32 v28, v28\n\t" \
  "v_rcp_f32_e32 v29, v29\n\t" \
  "v_rcp_f32_e32 v30, v30\n\t" \
  "v_rcp_f32_e32 v31, v31\n\t" \
  "v_pk_mul_f32 v[32:33], v[32:33], v[24:25]\n\t" \
  "v_pk_mul_f32 v[34:35], v[34:35], v[26:27]\n\t" \
  "s_xor_b32 s55, s67, 2097152\n\t" \
  "buffer_store_dwordx2 v[32:33], v5, s[40:43], s55 offen nt\n\t" \
  "buffer_store_dwordx2 v[34:35], v5, s[40:43], s55 offen offset:128 nt\n\t" \
  "v_accvgpr_read_b32 v48, a112\n\t" \
  "v_accvgpr_read_b32 v49, a116\n\t" \
  "v_accvgpr_read_b32 v50, a120\n\t" \
  "v_accvgpr_read_b32 v51, a124\n\t" \
  "v_pk_mul_f32 v[48:49], v[48:49], s[56:57]\n\t" \
  "v_pk_mul_f32 v[50:51], v[50:51], s[56:57]\n\t" \
  "v_min_f32_e32 v20, s60, v48\n\t" \
  "v_min_f32_e32 v21, s60, v49\n\t" \
  "v_min_f32_e32 v22, s60, v50\n\t" \
  "v_min_f32_e32 v23, s60, v51\n\t" \
  "v_exp_f32_e32 v16, v16\n\t" \
  "v_exp_f32_e32 v17, v17\n\t" \
  "v_exp_f32_e32 v18, v18\n\t" \
  "v_exp_f32_e32 v19, v19\n\t" \
  "v_pk_fma_f32 v[12:13], v[12:13], s[58:59], s[58:59]\n\t" \
  "v_pk_fma_f32 v[14:15], v[14:15], s[58:59], s[58:59]\n\t" \
  "v_rcp_f32_e32 v12, v12\n\t" \
  "v_rcp_f32_e32 v13, v13\n\t" \
  "v_rcp_f32_e32 v14, v14\n\t" \
  "v_rcp_f32_e32 v15, v15\n\t" \
  "v_pk_mul_f32 v[36:37], v[36:37], v[28:29]\n\t" \
  "v_pk_mul_f32 v[38:39], v[38:39], v[30:31]\n\t" \
  "s_xor_b32 s55, s67, 2162688\n\t" \
  "buffer_store_dwordx2 v[36:37], v5, s[40:43], s55 offen nt\n\t" \
  "buffer_store_dwordx2 v[38:39], v5, s[40:43], s55 offen offset:128 nt\n\t" \
  "v_accvgpr_read_b32 v52, a113\n\t" \
  "v_accvgpr_read_b32 v53, a117\n\t" \
  "v_accvgpr_read_b32 v54, a121\n\t" \
  "v_accvgpr_read_b32 v55, a125\n\t" \
  "v_pk_mul_f32 v[52:53], v[52:53], s[56:57]\n\t" \
  "v_pk_mul_f32 v[54:55], v[54:55], s[56:57]\n\t" \
  "v_min_f32_e32 v24, s60, v52\n\t" \
  "v_min_f32_e32 v25, s60, v53\n\t" \
  "v_min_f32_e32 v26, s60, v54\n\t" \
  "v_min_f32_e32 v27, s60, v55\n\t" \
  "v_exp_f32_e32 v20, v20\n\t" \
  "v_exp_f32_e32 v21, v21\n\t" \
  "v_exp_f32_e32 v22, v22\n\t" \
  "v_exp_f32_e32 v23, v23\n\t" \
  "v_pk_fma_f32 v[16:17], v[16:17], s[58:59], s[58:59]\n\t" \
  "v_pk_fma_f32 v[18:19], v[18:19], s[58:59], s[58:59]\n\t" \
  "v_rcp_f32_e32 v16, v16\n\t" \
  "v_rcp_f32_e32 v17, v17\n\t" \
  "v_rcp_f32_e32 v18, v18\n\t" \
  "v_rcp_f32_e32 v19, v19\n\t" \
  "v_pk_mul_f32 v[40:41], v[40:41], v[12:13]\n\t" \
  "v_pk_mul_f32 v[42:43], v[42:43], v[14:15]\n\t" \
  "s_xor_b32 s55, s67, 2228224\n\t" \
  "buffer_store_dwordx2 v[40:41], v5, s[40:43], s55 offen nt\n\t" \
  "buffer_store_dwordx2 v[42:43], v5, s[40:43], s55 offen offset:128 nt\n\t" \
  "v_accvgpr_read_b32 v56, a114\n\t" \
  "v_accvgpr_read_b32 v57, a118\n\t" \
  "v_accvgpr_read_b32 v58, a122\n\t" \
  "v_accvgpr_read_b32 v59, a126\n\t" \
  "v_pk_mul_f32 v[56:57], v[56:57], s[56:57]\n\t" \
  "v_pk_mul_f32 v[58:59], v[58:59], s[56:57]\n\t" \
  "v_min_f32_e32 v28, s60, v56\n\t" \
  "v_min_f32_e32 v29, s60, v57\n\t" \
  "v_min_f32_e32 v30, s60, v58\n\t" \
  "v_min_f32_e32 v31, s60, v59\n\t" \
  "v_exp_f32_e32 v24, v24\n\t" \
  "v_exp_f32_e32 v25, v25\n\t" \
  "v_exp_f32_e32 v26, v26\n\t" \
  "v_exp_f32_e32 v27, v27\n\t" \
  "v_pk_fma_f32 v[20:21], v[20:21], s[58:59], s[58:59]\n\t" \
  "v_pk_fma_f32 v[22:23], v[22:23], s[58:59], s[58:59]\n\t" \
  "v_rcp_f32_e32 v20, v20\n\t" \
  "v_rcp_f32_e32 v21, v21\n\t" \
  "v_rcp_f32_e32 v22, v22\n\t" \
  "v_rcp_f32_e32 v23, v23\n\t" \
  "v_pk_mul_f32 v[44:45], v[44:45], v[16:17]\n\t" \
  "v_pk_mul_f32 v[46:47], v[46:47], v[18:19]\n\t" \
  "s_xor_b32 s55, s67, 2293760\n\t" \
  "buffer_store_dwordx2 v[44:45], v5, s[40:43], s55 offen nt\n\t" \
  "buffer_store_dwordx2 v[46:47], v5, s[40:43], s55 offen offset:128 nt\n\t" \
  "v_accvgpr_read_b32 v60, a115\n\t" \
  "v_accvgpr_read_b32 v61, a119\n\t" \
  "v_accvgpr_read_b32 v62, a123\n\t" \
  "v_accvgpr_read_b32 v63, a127\n\t" \
  "v_pk_mul_f32 v[60:61], v[60:61], s[56:57]\n\t" \
  "v_pk_mul_f32 v[62:63], v[62:63], s[56:57]\n\t" \
  "v_min_f32_e32 v12, s60, v60\n\t" \
  "v_min_f32_e32 v13, s60, v61\n\t" \
  "v_min_f32_e32 v14, s60, v62\n\t" \
  "v_min_f32_e32 v15, s60, v63\n\t" \
  "v_exp_f32_e32 v28, v28\n\t" \
  "v_exp_f32_e32 v29, v29\n\t" \
  "v_exp_f32_e32 v30, v30\n\t" \
  "v_exp_f32_e32 v31, v31\n\t" \
  "v_pk_fma_f32 v[24:25], v[24:25], s[58:59], s[58:59]\n\t" \
  "v_pk_fma_f32 v[26:27], v[26:27], s[58:59], s[58:59]\n\t" \
  "v_rcp_f32_e32 v24, v24\n\t" \
  "v_rcp_f32_e32 v25, v25\n\t" \
  "v_rcp_f32_e32 v26, v26\n\t" \
  "v_rcp_f32_e32 v27, v27\n\t" \
  "v_pk_mul_f32 v[48:49], v[48:49], v[20:21]\n\t" \
  "v_pk_mul_f32 v[50:51], v[50:51], v[22:23]\n\t" \
  "s_xor_b32 s55, s67, 3145728\n\t" \
  "buffer_store_dwordx2 v[48:49], v5, s[40:43], s55 offen nt\n\t" \
  "buffer_store_dwordx2 v[50:51], v5, s[40:43], s55 offen offset:128 nt\n\t" \
  "v_accvgpr_read_b32 v32, a144\n\t" \
  "v_accvgpr_read_b32 v33, a148\n\t" \
  "v_accvgpr_read_b32 v34, a152\n\t" \
  "v_accvgpr_read_b32 v35, a156\n\t" \
  "v_pk_mul_f32 v[32:33], v[32:33], s[56:57]\n\t" \
  "v_pk_mul_f32 v[34:35], v[34:35], s[56:57]\n\t" \
  "v_min_f32_e32 v16, s60, v32\n\t" \
  "v_min_f32_e32 v17, s60, v33\n\t" \
  "v_min_f32_e32 v18, s60, v34\n\t" \
  "v_min_f32_e32 v19, s60, v35\n\t" \
  "v_exp_f32_e32 v12, v12\n\t" \
  "v_exp_f32_e32 v13, v13\n\t" \
  "v_exp_f32_e32 v14, v14\n\t" \
  "v_exp_f32_e32 v15, v15\n\t" \
  "v_pk_fma_f32 v[28:29], v[28:29], s[58:59], s[58:59]\n\t" \
  "v_pk_fma_f32 v[30:31], v[30:31], s[58:59], s[58:59]\n\t" \
  "v_rcp_f32_e32 v28, v28\n\t" \
  "v_rcp_f32_e32 v29, v29\n\t" \
  "v_rcp_f32_e32 v30, v30\n\t" \
  "v_rcp_f32_e32 v31, v31\n\t" \
  "v_pk_mul_f32 v[52:53], v[52:53], v[24:25]\n\t" \
  "v_pk_mul_f32 v[54:55], v[54:55], v[26:27]\n\t" \
  "s_xor_b32 s55, s67, 3211264\n\t" \
  "buffer_store_dwordx2 v[52:53], v5, s[40:43], s55 offen nt\n\t" \
  "buffer_store_dwordx2 v[54:55], v5, s[40:43], s55 offen offset:128 nt\n\t" \
  "v_accvgpr_read_b32 v36, a145\n\t" \
  "v_accvgpr_read_b32 v37, a149\n\t" \
  "v_accvgpr_read_b32 v38, a153\n\t" \
  "v_accvgpr_read_b32 v39, a157\n\t" \
  "v_pk_mul_f32 v[36:37], v[36:37], s[56:57]\n\t" \
  "v_pk_mul_f32 v[38:39], v[38:39], s[56:57]\n\t" \
  "v_min_f32_e32 v20, s60, v36\n\t" \
  "v_min_f32_e32 v21, s60, v37\n\t" \
  "v_min_f32_e32 v22, s60, v38\n\t" \
  "v_min_f32_e32 v23, s60, v39\n\t" \
  "v_exp_f32_e32 v16, v16\n\t" \
  "v_exp_f32_e32 v17, v17\n\t" \
  "v_exp_f32_e32 v18, v18\n\t" \
  "v_exp_f32_e32 v19, v19\n\t" \
  "v_pk_fma_f32 v[12:13], v[12:13], s[58:59], s[58:59]\n\t" \
  "v_pk_fma_f32 v[14:15], v[14:15], s[58:59], s[58:59]\n\t" \
  "v_rcp_f32_e32 v12, v12\n\t" \
  "v_rcp_f32_e32 v13, v13\n\t" \
  "v_rcp_f32_e32 v14, v14\n\t" \
  "v_rcp_f32_e32 v15, v15\n\t" \
  "v_pk_mul_f32 v[56:57], v[56:57], v[28:29]\n\t" \
  "v_pk_mul_f32 v[58:59], v[58:59], v[30:31]\n\t" \
  "s_xor_b32 s55, s67, 3276800\n\t" \
  "buffer_store_dwordx2 v[56:57], v5, s[40:43], s55 offen nt\n\t" \
  "buffer_store_dwordx2 v[58:59], v5, s[40:43], s55 offen offset:128 nt\n\t" \
  "v_accvgpr_read_b32 v40, a146\n\t" \
  "v_accvgpr_read_b32 v41, a150\n\t" \
  "v_accvgpr_read_b32 v42, a154\n\t" \
  "v_accvgpr_read_b32 v43, a158\n\t" \
  "v_pk_mul_f32 v[40:41], v[40:41], s[56:57]\n\t" \
  "v_pk_mul_f32 v[42:43], v[42:43], s[56:57]\n\t" \
  "v_min_f32_e32 v24, s60, v40\n\t" \
  "v_min_f32_e32 v25, s60, v41\n\t" \
  "v_min_f32_e32 v26, s60, v42\n\t" \
  "v_min_f32_e32 v27, s60, v43\n\t" \
  "v_exp_f32_e32 v20, v20\n\t" \
  "v_exp_f32_e32 v21, v21\n\t" \
  "v_exp_f32_e32 v22, v22\n\t" \
  "v_exp_f32_e32 v23, v23\n\t" \
  "v_pk_fma_f32 v[16:17], v[16:17], s[58:59], s[58:59]\n\t" \
  "v_pk_fma_f32 v[18:19], v[18:19], s[58:59], s[58:59]\n\t" \
  "v_rcp_f32_e32 v16, v16\n\t" \
  "v_rcp_f32_e32 v17, v17\n\t" \
  "v_rcp_f32_e32 v18, v18\n\t" \
  "v_rcp_f32_e32 v19, v19\n\t" \
  "v_pk_mul_f32 v[60:61], v[60:61], v[12:13]\n\t" \
  "v_pk_mul_f32 v[62:63], v[62:63], v[14:15]\n\t" \
  "s_xor_b32 s55, s67, 3342336\n\t" \
  "buffer_store_dwordx2 v[60:61], v5, s[40:43], s55 offen nt\n\t" \
  "buffer_store_dwordx2 v[62:63], v5, s[40:43], s55 offen offset:128 nt\n\t" \
  "v_accvgpr_read_b32 v44, a147\n\t" \
  "v_accvgpr_read_b32 v45, a151\n\t" \
  "v_accvgpr_read_b32 v46, a155\n\t" \
  "v_accvgpr_read_b32 v47, a159\n\t" \
  "v_pk_mul_f32 v[44:45], v[44:45], s[56:57]\n\t" \
  "v_pk_mul_f32 v[46:47], v[46:47], s[56:57]\n\t" \
  "v_min_f32_e32 v28, s60, v44\n\t" \
  "v_min_f32_e32 v29, s60, v45\n\t" \
  "v_min_f32_e32 v30, s60, v46\n\t" \
  "v_min_f32_e32 v31, s60, v47\n\t" \
  "v_exp_f32_e32 v24, v24\n\t" \
  "v_exp_f32_e32 v25, v25\n\t" \
  "v_exp_f32_e32 v26, v26\n\t" \
  "v_exp_f32_e32 v27, v27\n\t" \
  "v_pk_fma_f32 v[20:21], v[20:21], s[58:59], s[58:59]\n\t" \
  "v_pk_fma_f32 v[22:23], v[22:23], s[58:59], s[58:59]\n\t" \
  "v_rcp_f32_e32 v20, v20\n\t" \
  "v_rcp_f32_e32 v21, v21\n\t" \
  "v_rcp_f32_e32 v22, v22\n\t" \
  "v_rcp_f32_e32 v23, v23\n\t" \
  "v_pk_mul_f32 v[32:33], v[32:33], v[16:17]\n\t" \
  "v_pk_mul_f32 v[34:35], v[34:35], v[18:19]\n\t" \
  "s_xor_b32 s55, s67, 4194304\n\t" \
  "buffer_store_dwordx2 v[32:33], v5, s[40:43], s55 offen nt\n\t" \
  "buffer_store_dwordx2 v[34:35], v5, s[40:43], s55 offen offset:128 nt\n\t" \
  "v_accvgpr_read_b32 v48, a176\n\t" \
  "v_accvgpr_read_b32 v49, a180\n\t" \
  "v_accvgpr_read_b32 v50, a184\n\t" \
  "v_accvgpr_read_b32 v51, a188\n\t" \
  "v_pk_mul_f32 v[48:49], v[48:49], s[56:57]\n\t" \
  "v_pk_mul_f32 v[50:51], v[50:51], s[56:57]\n\t" \
  "v_min_f32_e32 v12, s60, v48\n\t" \
  "v_min_f32_e32 v13, s60, v49\n\t" \
  "v_min_f32_e32 v14, s60, v50\n\t" \
  "v_min_f32_e32 v15, s60, v51\n\t" \
  "v_exp_f32_e32 v28, v28\n\t" \
  "v_exp_f32_e32 v29, v29\n\t" \
  "v_exp_f32_e32 v30, v30\n\t" \
  "v_exp_f32_e32 v31, v31\n\t" \
  "v_pk_fma_f32 v[24:25], v[24:25], s[58:59], s[58:59]\n\t" \
  "v_pk_fma_f32 v[26:27], v[26:27], s[58:59], s[58:59]\n\t" \
  "v_rcp_f32_e32 v24, v24\n\t" \
  "v_rcp_f32_e32 v25, v25\n\t" \
  "v_rcp_f32_e32 v26, v26\n\t" \
  "v_rcp_f32_e32 v27, v27\n\t" \
  "v_pk_mul_f32 v[36:37], v[36:37], v[20:21]\n\t" \
  "v_pk_mul_f32 v[38:39], v[38:39], v[22:23]\n\t" \
  "s_xor_b32 s55, s67, 4259840\n\t" \
  "buffer_store_dwordx2 v[36:37], v5, s[40:43], s55 offen nt\n\t" \
  "buffer_store_dwordx2 v[38:39], v5, s[40:43], s55 offen offset:128 nt\n\t" \
  "v_accvgpr_read_b32 v52, a177\n\t" \
  "v_accvgpr_read_b32 v53, a181\n\t" \
  "v_accvgpr_read_b32 v54, a185\n\t" \
  "v_accvgpr_read_b32 v55, a189\n\t" \
  "v_pk_mul_f32 v[52:53], v[52:53], s[56:57]\n\t" \
  "v_pk_mul_f32 v[54:55], v[54:55], s[56:57]\n\t" \
  "v_min_f32_e32 v16, s60, v52\n\t" \
  "v_min_f32_e32 v17, s60, v53\n\t" \
  "v_min_f32_e32 v18, s60, v54\n\t" \
  "v_min_f32_e32 v19, s60, v55\n\t" \
  "v_exp_f32_e32 v12, v12\n\t" \
  "v_exp_f32_e32 v13, v13\n\t" \
  "v_exp_f32_e32 v14, v14\n\t" \
  "v_exp_f32_e32 v15, v15\n\t" \
  "v_pk_fma_f32 v[28:29], v[28:29], s[58:59], s[58:59]\n\t" \
  "v_pk_fma_f32 v[30:31], v[30:31], s[58:59], s[58:59]\n\t" \
  "v_rcp_f32_e32 v28, v28\n\t" \
  "v_rcp_f32_e32 v29, v29\n\t" \
  "v_rcp_f32_e32 v30, v30\n\t" \
  "v_rcp_f32_e32 v31, v31\n\t" \
  "v_pk_mul_f32 v[40:41], v[40:41], v[24:25]\n\t" \
  "v_pk_mul_f32 v[42:43], v[42:43], v[26:27]\n\t" \
  "s_xor_b32 s55, s67, 4325376\n\t" \
  "buffer_store_dwordx2 v[40:41], v5, s[40:43], s55 offen nt\n\t" \
  "buffer_store_dwordx2 v[42:43], v5, s[40:43], s55 offen offset:128 nt\n\t" \
  "v_accvgpr_read_b32 v56, a178\n\t" \
  "v_accvgpr_read_b32 v57, a182\n\t" \
  "v_accvgpr_read_b32 v58, a186\n\t" \
  "v_accvgpr_read_b32 v59, a190\n\t" \
  "v_pk_mul_f32 v[56:57], v[56:57], s[56:57]\n\t" \
  "v_pk_mul_f32 v[58:59], v[58:59], s[56:57]\n\t" \
  "v_min_f32_e32 v20, s60, v56\n\t" \
  "v_min_f32_e32 v21, s60, v57\n\t" \
  "v_min_f32_e32 v22, s60, v58\n\t" \
  "v_min_f32_e32 v23, s60, v59\n\t" \
  "v_exp_f32_e32 v16, v16\n\t" \
  "v_exp_f32_e32 v17, v17\n\t" \
  "v_exp_f32_e32 v18, v18\n\t" \
  "v_exp_f32_e32 v19, v19\n\t" \
  "v_pk_fma_f32 v[12:13], v[12:13], s[58:59], s[58:59]\n\t" \
  "v_pk_fma_f32 v[14:15], v[14:15], s[58:59], s[58:59]\n\t" \
  "v_rcp_f32_e32 v12, v12\n\t" \
  "v_rcp_f32_e32 v13, v13\n\t" \
  "v_rcp_f32_e32 v14, v14\n\t" \
  "v_rcp_f32_e32 v15, v15\n\t" \
  "v_pk_mul_f32 v[44:45], v[44:45], v[28:29]\n\t" \
  "v_pk_mul_f32 v[46:47], v[46:47], v[30:31]\n\t" \
  "s_xor_b32 s55, s67, 4390912\n\t" \
  "buffer_store_dwordx2 v[44:45], v5, s[40:43], s55 offen nt\n\t" \
  "buffer_store_dwordx2 v[46:47], v5, s[40:43], s55 offen offset:128 nt\n\t" \
  "v_accvgpr_read_b32 v60, a179\n\t" \
  "v_accvgpr_read_b32 v61, a183\n\t" \
  "v_accvgpr_read_b32 v62, a187\n\t" \
  "v_accvgpr_read_b32 v63, a191\n\t" \
  "v_pk_mul_f32 v[60:61], v[60:61], s[56:57]\n\t" \
  "v_pk_mul_f32 v[62:63], v[62:63], s[56:57]\n\t" \
  "v_min_f32_e32 v24, s60, v60\n\t" \
  "v_min_f32_e32 v25, s60, v61\n\t" \
  "v_min_f32_e32 v26, s60, v62\n\t" \
  "v_min_f32_e32 v27, s60, v63\n\t" \
  "v_exp_f32_e32 v20, v20\n\t" \
  "v_exp_f32_e32 v21, v21\n\t" \
  "v_exp_f32_e32 v22, v22\n\t" \
  "v_exp_f32_e32 v23, v23\n\t" \
  "v_pk_fma_f32 v[16:17], v[16:17], s[58:59], s[58:59]\n\t" \
  "v_pk_fma_f32 v[18:19], v[18:19], s[58:59], s[58:59]\n\t" \
  "v_rcp_f32_e32 v16, v16\n\t" \
  "v_rcp_f32_e32 v17, v17\n\t" \
  "v_rcp_f32_e32 v18, v18\n\t" \
  "v_rcp_f32_e32 v19, v19\n\t" \
  "v_pk_mul_f32 v[48:49], v[48:49], v[12:13]\n\t" \
  "v_pk_mul_f32 v[50:51], v[50:51], v[14:15]\n\t" \
  "s_xor_b32 s55, s67, 5242880\n\t" \
  "buffer_store_dwordx2 v[48:49], v5, s[40:43], s55 offen nt\n\t" \
  "buffer_store_dwordx2 v[50:51], v5, s[40:43], s55 offen offset:128 nt\n\t" \
  "v_accvgpr_read_b32 v32, a208\n\t" \
  "v_accvgpr_read_b32 v33, a212\n\t" \
  "v_accvgpr_read_b32 v34, a216\n\t" \
  "v_accvgpr_read_b32 v35, a220\n\t" \
  "v_pk_mul_f32 v[32:33], v[32:33], s[56:57]\n\t" \
  "v_pk_mul_f32 v[34:35], v[34:35], s[56:57]\n\t" \
  "v_min_f32_e32 v28, s60, v32\n\t" \
  "v_min_f32_e32 v29, s60, v33\n\t" \
  "v_min_f32_e32 v30, s60, v34\n\t" \
  "v_min_f32_e32 v31, s60, v35\n\t" \
  "v_exp_f32_e32 v24, v24\n\t" \
  "v_exp_f32_e32 v25, v25\n\t" \
  "v_exp_f32_e32 v26, v26\n\t" \
  "v_exp_f32_e32 v27, v27\n\t" \
  "v_pk_fma_f32 v[20:21], v[20:21], s[58:59], s[58:59]\n\t" \
  "v_pk_fma_f32 v[22:23], v[22:23], s[58:59], s[58:59]\n\t" \
  "v_rcp_f32_e32 v20, v20\n\t" \
  "v_rcp_f32_e32 v21, v21\n\t" \
  "v_rcp_f32_e32 v22, v22\n\t" \
  "v_rcp_f32_e32 v23, v23\n\t" \
  "v_pk_mul_f32 v[52:53], v[52:53], v[16:17]\n\t" \
  "v_pk_mul_f32 v[54:55], v[54:55], v[18:19]\n\t" \
  "s_xor_b32 s55, s67, 5308416\n\t" \
  "buffer_store_dwordx2 v[52:53], v5, s[40:43], s55 offen nt\n\t" \
  "buffer_store_dwordx2 v[54:55], v5, s[40:43], s55 offen offset:128 nt\n\t" \
  "v_accvgpr_read_b32 v36, a209\n\t" \
  "v_accvgpr_read_b32 v37, a213\n\t" \
  "v_accvgpr_read_b32 v38, a217\n\t" \
  "v_accvgpr_read_b32 v39, a221\n\t" \
  "v_pk_mul_f32 v[36:37], v[36:37], s[56:57]\n\t" \
  "v_pk_mul_f32 v[38:39], v[38:39], s[56:57]\n\t" \
  "v_min_f32_e32 v12, s60, v36\n\t" \
  "v_min_f32_e32 v13, s60, v37\n\t" \
  "v_min_f32_e32 v14, s60, v38\n\t" \
  "v_min_f32_e32 v15, s60, v39\n\t" \
  "v_exp_f32_e32 v28, v28\n\t" \
  "v_exp_f32_e32 v29, v29\n\t" \
  "v_exp_f32_e32 v30, v30\n\t" \
  "v_exp_f32_e32 v31, v31\n\t" \
  "v_pk_fma_f32 v[24:25], v[24:25], s[58:59], s[58:59]\n\t" \
  "v_pk_fma_f32 v[26:27], v[26:27], s[58:59], s[58:59]\n\t" \
  "v_rcp_f32_e32 v24, v24\n\t" \
  "v_rcp_f32_e32 v25, v25\n\t" \
  "v_rcp_f32_e32 v26, v26\n\t" \
  "v_rcp_f32_e32 v27, v27\n\t" \
  "v_pk_mul_f32 v[56:57], v[56:57], v[20:21]\n\t" \
  "v_pk_mul_f32 v[58:59], v[58:59], v[22:23]\n\t" \
  "s_xor_b32 s55, s67, 5373952\n\t" \
  "buffer_store_dwordx2 v[56:57], v5, s[40:43], s55 offen nt\n\t" \
  "buffer_store_dwordx2 v[58:59], v5, s[40:43], s55 offen offset:128 nt\n\t" \
  "v_accvgpr_read_b32 v40, a210\n\t" \
  "v_accvgpr_read_b32 v41, a214\n\t" \
  "v_accvgpr_read_b32 v42, a218\n\t" \
  "v_accvgpr_read_b32 v43, a222\n\t" \
  "v_pk_mul_f32 v[40:41], v[40:41], s[56:57]\n\t" \
  "v_pk_mul_f32 v[42:43], v[42:43], s[56:57]\n\t" \
  "v_min_f32_e32 v16, s60, v40\n\t" \
  "v_min_f32_e32 v17, s60, v41\n\t" \
  "v_min_f32_e32 v18, s60, v42\n\t" \
  "v_min_f32_e32 v19, s60, v43\n\t" \
  "v_exp_f32_e32 v12, v12\n\t" \
  "v_exp_f32_e32 v13, v13\n\t" \
  "v_exp_f32_e32 v14, v14\n\t" \
  "v_exp_f32_e32 v15, v15\n\t" \
  "v_pk_fma_f32 v[28:29], v[28:29], s[58:59], s[58:59]\n\t" \
  "v_pk_fma_f32 v[30:31], v[30:31], s[58:59], s[58:59]\n\t" \
  "v_rcp_f32_e32 v28, v28\n\t" \
  "v_rcp_f32_e32 v29, v29\n\t" \
  "v_rcp_f32_e32 v30, v30\n\t" \
  "v_rcp_f32_e32 v31, v31\n\t" \
  "v_pk_mul_f32 v[60:61], v[60:61], v[24:25]\n\t" \
  "v_pk_mul_f32 v[62:63], v[62:63], v[26:27]\n\t" \
  "s_xor_b32 s55, s67, 5439488\n\t" \
  "buffer_store_dwordx2 v[60:61], v5, s[40:43], s55 offen nt\n\t" \
  "buffer_store_dwordx2 v[62:63], v5, s[40:43], s55 offen offset:128 nt\n\t" \
  "v_accvgpr_read_b32 v44, a211\n\t" \
  "v_accvgpr_read_b32 v45, a215\n\t" \
  "v_accvgpr_read_b32 v46, a219\n\t" \
  "v_accvgpr_read_b32 v47, a223\n\t" \
  "v_pk_mul_f32 v[44:45], v[44:45], s[56:57]\n\t" \
  "v_pk_mul_f32 v[46:47], v[46:47], s[56:57]\n\t" \
  "v_min_f32_e32 v20, s60, v44\n\t" \
  "v_min_f32_e32 v21, s60, v45\n\t" \
  "v_min_f32_e32 v22, s60, v46\n\t" \
  "v_min_f32_e32 v23, s60, v47\n\t" \
  "v_exp_f32_e32 v16, v16\n\t" \
  "v_exp_f32_e32 v17, v17\n\t" \
  "v_exp_f32_e32 v18, v18\n\t" \
  "v_exp_f32_e32 v19, v19\n\t" \
  "v_pk_fma_f32 v[12:13], v[12:13], s[58:59], s[58:59]\n\t" \
  "v_pk_fma_f32 v[14:15], v[14:15], s[58:59], s[58:59]\n\t" \
  "v_rcp_f32_e32 v12, v12\n\t" \
  "v_rcp_f32_e32 v13, v13\n\t" \
  "v_rcp_f32_e32 v14, v14\n\t" \
  "v_rcp_f32_e32 v15, v15\n\t" \
  "v_pk_mul_f32 v[32:33], v[32:33], v[28:29]\n\t" \
  "v_pk_mul_f32 v[34:35], v[34:35], v[30:31]\n\t" \
  "s_xor_b32 s55, s67, 6291456\n\t" \
  "buffer_store_dwordx2 v[32:33], v5, s[40:43], s55 offen nt\n\t" \
  "buffer_store_dwordx2 v[34:35], v5, s[40:43], s55 offen offset:128 nt\n\t" \
  "v_accvgpr_read_b32 v48, a240\n\t" \
  "v_accvgpr_read_b32 v49, a244\n\t" \
  "v_accvgpr_read_b32 v50, a248\n\t" \
  "v_accvgpr_read_b32 v51, a252\n\t" \
  "v_pk_mul_f32 v[48:49], v[48:49], s[56:57]\n\t" \
  "v_pk_mul_f32 v[50:51], v[50:51], s[56:57]\n\t" \
  "v_min_f32_e32 v24, s60, v48\n\t" \
  "v_min_f32_e32 v25, s60, v49\n\t" \
  "v_min_f32_e32 v26, s60, v50\n\t" \
  "v_min_f32_e32 v27, s60, v51\n\t" \
  "v_exp_f32_e32 v20, v20\n\t" \
  "v_exp_f32_e32 v21, v21\n\t" \
  "v_exp_f32_e32 v22, v22\n\t" \
  "v_exp_f32_e32 v23, v23\n\t" \
  "v_pk_fma_f32 v[16:17], v[16:17], s[58:59], s[58:59]\n\t" \
  "v_pk_fma_f32 v[18:19], v[18:19], s[58:59], s[58:59]\n\t" \
  "v_rcp_f32_e32 v16, v16\n\t" \
  "v_rcp_f32_e32 v17, v17\n\t" \
  "v_rcp_f32_e32 v18, v18\n\t" \
  "v_rcp_f32_e32 v19, v19\n\t" \
  "v_pk_mul_f32 v[36:37], v[36:37], v[12:13]\n\t" \
  "v_pk_mul_f32 v[38:39], v[38:39], v[14:15]\n\t" \
  "s_xor_b32 s55, s67, 6356992\n\t" \
  "buffer_store_dwordx2 v[36:37], v5, s[40:43], s55 offen nt\n\t" \
  "buffer_store_dwordx2 v[38:39], v5, s[40:43], s55 offen offset:128 nt\n\t" \
  "v_accvgpr_read_b32 v52, a241\n\t" \
  "v_accvgpr_read_b32 v53, a245\n\t" \
  "v_accvgpr_read_b32 v54, a249\n\t" \
  "v_accvgpr_read_b32 v55, a253\n\t" \
  "v_pk_mul_f32 v[52:53], v[52:53], s[56:57]\n\t" \
  "v_pk_mul_f32 v[54:55], v[54:55], s[56:57]\n\t" \
  "v_min_f32_e32 v28, s60, v52\n\t" \
  "v_min_f32_e32 v29, s60, v53\n\t" \
  "v_min_f32_e32 v30, s60, v54\n\t" \
  "v_min_f32_e32 v31, s60, v55\n\t" \
  "v_exp_f32_e32 v24, v24\n\t" \
  "v_exp_f32_e32 v25, v25\n\t" \
  "v_exp_f32_e32 v26, v26\n\t" \
  "v_exp_f32_e32 v27, v27\n\t" \
  "v_pk_fma_f32 v[20:21], v[20:21], s[58:59], s[58:59]\n\t" \
  "v_pk_fma_f32 v[22:23], v[22:23], s[58:59], s[58:59]\n\t" \
  "v_rcp_f32_e32 v20, v20\n\t" \
  "v_rcp_f32_e32 v21, v21\n\t" \
  "v_rcp_f32_e32 v22, v22\n\t" \
  "v_rcp_f32_e32 v23, v23\n\t" \
  "v_pk_mul_f32 v[40:41], v[40:41], v[16:17]\n\t" \
  "v_pk_mul_f32 v[42:43], v[42:43], v[18:19]\n\t" \
  "s_xor_b32 s55, s67, 6422528\n\t" \
  "buffer_store_dwordx2 v[40:41], v5, s[40:43], s55 offen nt\n\t" \
  "buffer_store_dwordx2 v[42:43], v5, s[40:43], s55 offen offset:128 nt\n\t" \
  "v_accvgpr_read_b32 v56, a242\n\t" \
  "v_accvgpr_read_b32 v57, a246\n\t" \
  "v_accvgpr_read_b32 v58, a250\n\t" \
  "v_accvgpr_read_b32 v59, a254\n\t" \
  "v_pk_mul_f32 v[56:57], v[56:57], s[56:57]\n\t" \
  "v_pk_mul_f32 v[58:59], v[58:59], s[56:57]\n\t" \
  "v_min_f32_e32 v12, s60, v56\n\t" \
  "v_min_f32_e32 v13, s60, v57\n\t" \
  "v_min_f32_e32 v14, s60, v58\n\t" \
  "v_min_f32_e32 v15, s60, v59\n\t" \
  "v_exp_f32_e32 v28, v28\n\t" \
  "v_exp_f32_e32 v29, v29\n\t" \
  "v_exp_f32_e32 v30, v30\n\t" \
  "v_exp_f32_e32 v31, v31\n\t" \
  "v_pk_fma_f32 v[24:25], v[24:25], s[58:59], s[58:59]\n\t" \
  "v_pk_fma_f32 v[26:27], v[26:27], s[58:59], s[58:59]\n\t" \
  "v_rcp_f32_e32 v24, v24\n\t" \
  "v_rcp_f32_e32 v25, v25\n\t" \
  "v_rcp_f32_e32 v26, v26\n\t" \
  "v_rcp_f32_e32 v27, v27\n\t" \
  "v_pk_mul_f32 v[44:45], v[44:45], v[20:21]\n\t" \
  "v_pk_mul_f32 v[46:47], v[46:47], v[22:23]\n\t" \
  "s_xor_b32 s55, s67, 6488064\n\t" \
  "buffer_store_dwordx2 v[44:45], v5, s[40:43], s55 offen nt\n\t" \
  "buffer_store_dwordx2 v[46:47], v5, s[40:43], s55 offen offset:128 nt\n\t" \
  "v_accvgpr_read_b32 v60, a243\n\t" \
  "v_accvgpr_read_b32 v61, a247\n\t" \
  "v_accvgpr_read_b32 v62, a251\n\t" \
  "v_accvgpr_read_b32 v63, a255\n\t" \
  "v_pk_mul_f32 v[60:61], v[60:61], s[56:57]\n\t" \
  "v_pk_mul_f32 v[62:63], v[62:63], s[56:57]\n\t" \
  "v_min_f32_e32 v16, s60, v60\n\t" \
  "v_min_f32_e32 v17, s60, v61\n\t" \
  "v_min_f32_e32 v18, s60, v62\n\t" \
  "v_min_f32_e32 v19, s60, v63\n\t" \
  "v_exp_f32_e32 v12, v12\n\t" \
  "v_exp_f32_e32 v13, v13\n\t" \
  "v_exp_f32_e32 v14, v14\n\t" \
  "v_exp_f32_e32 v15, v15\n\t" \
  "v_pk_fma_f32 v[28:29], v[28:29], s[58:59], s[58:59]\n\t" \
  "v_pk_fma_f32 v[30:31], v[30:31], s[58:59], s[58:59]\n\t" \
  "v_rcp_f32_e32 v28, v28\n\t" \
  "v_rcp_f32_e32 v29, v29\n\t" \
  "v_rcp_f32_e32 v30, v30\n\t" \
  "v_rcp_f32_e32 v31, v31\n\t" \
  "v_pk_mul_f32 v[48:49], v[48:49], v[24:25]\n\t" \
  "v_pk_mul_f32 v[50:51], v[50:51], v[26:27]\n\t" \
  "s_xor_b32 s55, s67, 7340032\n\t" \
  "buffer_store_dwordx2 v[48:49], v5, s[40:43], s55 offen nt\n\t" \
  "buffer_store_dwordx2 v[50:51], v5, s[40:43], s55 offen offset:128 nt\n\t" \
  "v_exp_f32_e32 v16, v16\n\t" \
  "v_exp_f32_e32 v17, v17\n\t" \
  "v_exp_f32_e32 v18, v18\n\t" \
  "v_exp_f32_e32 v19, v19\n\t" \
  "v_pk_fma_f32 v[12:13], v[12:13], s[58:59], s[58:59]\n\t" \
  "v_pk_fma_f32 v[14:15], v[14:15], s[58:59], s[58:59]\n\t" \
  "v_rcp_f32_e32 v12, v12\n\t" \
  "v_rcp_f32_e32 v13, v13\n\t" \
  "v_rcp_f32_e32 v14, v14\n\t" \
  "v_rcp_f32_e32 v15, v15\n\t" \
  "v_pk_mul_f32 v[52:53], v[52:53], v[28:29]\n\t" \
  "v_pk_mul_f32 v[54:55], v[54:55], v[30:31]\n\t" \
  "s_xor_b32 s55, s67, 7405568\n\t" \
  "buffer_store_dwordx2 v[52:53], v5, s[40:43], s55 offen nt\n\t" \
  "buffer_store_dwordx2 v[54:55], v5, s[40:43], s55 offen offset:128 nt\n\t" \
  "v_pk_fma_f32 v[16:17], v[16:17], s[58:59], s[58:59]\n\t" \
  "v_pk_fma_f32 v[18:19], v[18:19], s[58:59], s[58:59]\n\t" \
  "v_rcp_f32_e32 v16, v16\n\t" \
  "v_rcp_f32_e32 v17, v17\n\t" \
  "v_rcp_f32_e32 v18, v18\n\t" \
  "v_rcp_f32_e32 v19, v19\n\t" \
  "v_pk_mul_f32 v[56:57], v[56:57], v[12:13]\n\t" \
  "v_pk_mul_f32 v[58:59], v[58:59], v[14:15]\n\t" \
  "s_xor_b32 s55, s67, 7471104\n\t" \
  "buffer_store_dwordx2 v[56:57], v5, s[40:43], s55 offen nt\n\t" \
  "buffer_store_dwordx2 v[58:59], v5, s[40:43], s55 offen offset:128 nt\n\t" \
  "v_pk_mul_f32 v[60:61], v[60:61], v[16:17]\n\t" \
  "v_pk_mul_f32 v[62:63], v[62:63], v[18:19]\n\t" \
  "s_xor_b32 s55, s67, 7536640\n\t" \
  "buffer_store_dwordx2 v[60:61], v5, s[40:43], s55 offen nt\n\t" \
  "buffer_store_dwordx2 v[62:63], v5, s[40:43], s55 offen offset:128 nt\n\t" \
  "s_waitcnt vmcnt(0)\n\t" \
  "s_branch L_END_%=\n\t" \
  "L_NS_%=:\n\t" \
  "s_mov_b32 s66, s69\n\t" \
  "s_add_u32 s77, s77, 1\n\t" \
  "s_and_b32 s54, s62, 0xff\n\t" \
  "s_lshr_b64 s[62:63], s[62:63], 8\n\t" \
  "s_lshl_b32 s55, s64, 24\n\t" \
  "s_or_b32 s63, s63, s55\n\t" \
  "s_lshr_b64 s[64:65], s[64:65], 8\n\t" \
  "s_and_b32 s54, s54, 0x7f\n\t" \
  "s_lshl_b32 s69, s54, 9\n\t" \
  "s_add_u32 s69, s69, s71\n\t" \
  "s_lshl_b32 s54, s54, 19\n\t" \
  "s_add_u32 s54, s54, s70\n\t" \
  "s_add_u32 s51, s54, 262144\n\t" \
  "v_add_u32_e32 v126, 0x20000, v1\n\t" \
  "v_add_u32_e32 v127, 0x20000, v3\n\t" \
  "s_mov_b32 s53, s50\n\t" \
  "s_add_u32 m0, s48, 114688\n\t" \
  "s_nop 0\n\t" \
  "buffer_load_dwordx4 v0, s[36:39], s53 offen sc0 sc1 lds\n\t" \
  "buffer_load_dwordx4 v0, s[36:39], s53 offen offset:1024 sc0 sc1 lds\n\t" \
  "buffer_load_dwordx4 v0, s[36:39], s53 offen offset:2048 sc0 sc1 lds\n\t" \
  "buffer_load_dwordx4 v0, s[36:39], s53 offen offset:3072 sc0 sc1 lds\n\t" \
  "s_mov_b32 s53, s52\n\t" \
  "s_add_u32 m0, s48, 16384\n\t" \
  "s_nop 0\n\t" \
  "buffer_load_dwordx4 v0, s[36:39], s53 offen sc0 sc1 lds\n\t" \
  "buffer_load_dwordx4 v0, s[36:39], s53 offen offset:1024 sc0 sc1 lds\n\t" \
  "buffer_load_dwordx4 v0, s[36:39], s53 offen offset:2048 sc0 sc1 lds\n\t" \
  "buffer_load_dwordx4 v0, s[36:39], s53 offen offset:3072 sc0 sc1 lds\n\t" \
  "s_add_u32 s53, s52, 16384\n\t" \
  "s_add_u32 m0, s48, 65536\n\t" \
  "s_nop 0\n\t" \
  "buffer_load_dwordx4 v0, s[36:39], s53 offen sc0 sc1 lds\n\t" \
  "buffer_load_dwordx4 v0, s[36:39], s53 offen offset:1024 sc0 sc1 lds\n\t" \
  "buffer_load_dwordx4 v0, s[36:39], s53 offen offset:2048 sc0 sc1 lds\n\t" \
  "buffer_load_dwordx4 v0, s[36:39], s53 offen offset:3072 sc0 sc1 lds\n\t" \
  "s_add_u32 s53, s50, 16384\n\t" \
  "s_add_u32 m0, s48, 81920\n\t" \
  "s_nop 0\n\t" \
  "buffer_load_dwordx4 v0, s[36:39], s53 offen sc0 sc1 lds\n\t" \
  "buffer_load_dwordx4 v0, s[36:39], s53 offen offset:1024 sc0 sc1 lds\n\t" \
  "buffer_load_dwordx4 v0, s[36:39], s53 offen offset:2048 sc0 sc1 lds\n\t" \
  "buffer_load_dwordx4 v0, s[36:39], s53 offen offset:3072 sc0 sc1 lds\n\t" \
  "s_add_u32 s53, s52, 32768\n\t" \
  "s_add_u32 m0, s48, 98304\n\t" \
  "s_nop 0\n\t" \
  "buffer_load_dwordx4 v0, s[36:39], s53 offen sc0 sc1 lds\n\t" \
  "buffer_load_dwordx4 v0, s[36:39], s53 offen offset:1024 sc0 sc1 lds\n\t" \
  "buffer_load_dwordx4 v0, s[36:39], s53 offen offset:2048 sc0 sc1 lds\n\t" \
  "buffer_load_dwordx4 v0, s[36:39], s53 offen offset:3072 sc0 sc1 lds\n\t" \
  "s_add_u32 s53, s52, 49152\n\t" \
  "s_add_u32 m0, s48, 131072\n\t" \
  "s_nop 0\n\t" \
  "buffer_load_dwordx4 v0, s[36:39], s53 offen sc0 sc1 lds\n\t" \
  "buffer_load_dwordx4 v0, s[36:39], s53 offen offset:1024 sc0 sc1 lds\n\t" \
  "buffer_load_dwordx4 v0, s[36:39], s53 offen offset:2048 sc0 sc1 lds\n\t" \
  "buffer_load_dwordx4 v0, s[36:39], s53 offen offset:3072 sc0 sc1 lds\n\t" \
  "s_add_u32 s53, s50, 32768\n\t" \
  "s_add_u32 m0, s48, 147456\n\t" \
  "s_nop 0\n\t" \
  "buffer_load_dwordx4 v0, s[36:39], s53 offen sc0 sc1 lds\n\t" \
  "buffer_load_dwordx4 v0, s[36:39], s53 offen offset:1024 sc0 sc1 lds\n\t" \
  "buffer_load_dwordx4 v0, s[36:39], s53 offen offset:2048 sc0 sc1 lds\n\t" \
  "buffer_load_dwordx4 v0, s[36:39], s53 offen offset:3072 sc0 sc1 lds\n\t" \
  "s_add_u32 s53, s52, 65536\n\t" \
  "s_mov_b32 m0, s48\n\t" \
  "s_nop 0\n\t" \
  "buffer_load_dwordx4 v0, s[36:39], s53 offen sc0 sc1 lds\n\t" \
  "buffer_load_dwordx4 v0, s[36:39], s53 offen offset:1024 sc0 sc1 lds\n\t" \
  "buffer_load_dwordx4 v0, s[36:39], s53 offen offset:2048 sc0 sc1 lds\n\t" \
  "buffer_load_dwordx4 v0, s[36:39], s53 offen offset:3072 sc0 sc1 lds\n\t" \
  "s_add_u32 s53, s52, 81920\n\t" \
  "s_add_u32 m0, s48, 32768\n\t" \
  "s_nop 0\n\t" \
  "buffer_load_dwordx4 v0, s[36:39], s53 offen sc0 sc1 lds\n\t" \
  "buffer_load_dwordx4 v0, s[36:39], s53 offen offset:1024 sc0 sc1 lds\n\t" \
  "buffer_load_dwordx4 v0, s[36:39], s53 offen offset:2048 sc0 sc1 lds\n\t" \
  "buffer_load_dwordx4 v0, s[36:39], s53 offen offset:3072 sc0 sc1 lds\n\t" \
  "s_add_u32 s53, s50, 49152\n\t" \
  "s_add_u32 m0, s48, 49152\n\t" \
  "s_nop 0\n\t" \
  "buffer_load_dwordx4 v0, s[36:39], s53 offen sc0 sc1 lds\n\t" \
  "buffer_load_dwordx4 v0, s[36:39], s53 offen offset:1024 sc0 sc1 lds\n\t" \
  "buffer_load_dwordx4 v0, s[36:39], s53 offen offset:2048 sc0 sc1 lds\n\t" \
  "buffer_load_dwordx4 v0, s[36:39], s53 offen offset:3072 sc0 sc1 lds\n\t" \
  "s_waitcnt vmcnt(0)\n\t" \
  "s_barrier\n\t" \
  "ds_read_b128 v[128:131], v1 offset:16384\n\t" \
  "ds_read_b128 v[132:135], v1 offset:24576\n\t" \
  "ds_read_b128 v[136:139], v1 offset:17408\n\t" \
  "ds_read_b128 v[140:143], v1 offset:25600\n\t" \
  "ds_read_b128 v[144:147], v1 offset:18432\n\t" \
  "ds_read_b128 v[148:151], v1 offset:26624\n\t" \
  "ds_read_b128 v[152:155], v1 offset:19456\n\t" \
  "ds_read_b128 v[156:159], v1 offset:27648\n\t" \
  "ds_read_b128 v[192:195], v4 offset:49152\n\t" \
  "ds_read_b128 v[196:199], v4 offset:57344\n\t" \
  "ds_read_b128 v[200:203], v4 offset:50176\n\t" \
  "ds_read_b128 v[204:207], v4 offset:58368\n\t" \
  "ds_read_b128 v[208:211], v4 offset:51200\n\t" \
  "ds_read_b128 v[212:215], v4 offset:59392\n\t" \
  "ds_read_b128 v[216:219], v4 offset:52224\n\t" \
  "ds_read_b128 v[220:223], v4 offset:60416\n\t" \
  "s_waitcnt lgkmcnt(0)\n\t" \
  "v_mfma_f32_16x16x128_f8f6f4 a[0:3], v[128:135], v[192:199], a[0:3]\n\t" \
  "s_barrier\n\t" \
  "v_mfma_f32_16x16x128_f8f6f4 a[4:7], v[128:135], v[200:207], a[4:7]\n\t" \
  "ds_read_b128 v[160:163], v2 offset:0\n\t" \
  "ds_read_b128 v[164:167], v2 offset:8192\n\t" \
  "v_mfma_f32_16x16x128_f8f6f4 a[8:11], v[128:135], v[208:215], a[8:11]\n\t" \
  "ds_read_b128 v[168:171], v2 offset:1024\n\t" \
  "ds_read_b128 v[172:175], v2 offset:9216\n\t" \
  "v_mfma_f32_16x16x128_f8f6f4 a[12:15], v[128:135], v[216:223], a[12:15]\n\t" \
  "ds_read_b128 v[176:179], v2 offset:2048\n\t" \
  "ds_read_b128 v[180:183], v2 offset:10240\n\t" \
  "v_mfma_f32_16x16x128_f8f6f4 a[44:47], v[136:143], v[216:223], a[44:47]\n\t" \
  "ds_read_b128 v[184:187], v2 offset:3072\n\t" \
  "ds_read_b128 v[188:191], v2 offset:11264\n\t" \
  "v_mfma_f32_16x16x128_f8f6f4 a[40:43], v[136:143], v[208:215], a[40:43]\n\t" \
  "ds_read_b128 v[224:227], v4 offset:16384\n\t" \
  "ds_read_b128 v[228:231], v4 offset:24576\n\t" \
  "v_mfma_f32_16x16x128_f8f6f4 a[36:39], v[136:143], v[200:207], a[36:39]\n\t" \
  "ds_read_b128 v[232:235], v4 offset:17408\n\t" \
  "ds_read_b128 v[236:239], v4 offset:25600\n\t" \
  "s_add_u32 s53, s52, 98304\n\t" \
  "s_add_u32 m0, s48, 114688\n\t" \
  "v_mfma_f32_16x16x128_f8f6f4 a[32:35], v[136:143], v[192:199], a[32:35]\n\t" \
  "buffer_load_dwordx4 v0, s[36:39], s53 offen sc0 sc1 lds\n\t" \
  "v_mfma_f32_16x16x128_f8f6f4 a[64:67], v[144:151], v[192:199], a[64:67]\n\t" \
  "buffer_load_dwordx4 v0, s[36:39], s53 offen offset:1024 sc0 sc1 lds\n\t" \
  "v_mfma_f32_16x16x128_f8f6f4 a[68:71], v[144:151], v[200:207], a[68:71]\n\t" \
  "buffer_load_dwordx4 v0, s[36:39], s53 offen offset:2048 sc0 sc1 lds\n\t" \
  "v_mfma_f32_16x16x128_f8f6f4 a[72:75], v[144:151], v[208:215], a[72:75]\n\t" \
  "buffer_load_dwordx4 v0, s[36:39], s53 offen offset:3072 sc0 sc1 lds\n\t" \
  "s_add_u32 s53, s52, 114688\n\t" \
  "s_add_u32 m0, s48, 16384\n\t" \
  "v_mfma_f32_16x16x128_f8f6f4 a[76:79], v[144:151], v[216:223], a[76:79]\n\t" \
  "buffer_load_dwordx4 v0, s[36:39], s53 offen sc0 sc1 lds\n\t" \
  "v_mfma_f32_16x16x128_f8f6f4 a[108:111], v[152:159], v[216:223], a[108:111]\n\t" \
  "buffer_load_dwordx4 v0, s[36:39], s53 offen offset:1024 sc0 sc1 lds\n\t" \
  "v_mfma_f32_16x16x128_f8f6f4 a[104:107], v[152:159], v[208:215], a[104:107]\n\t" \
  "buffer_load_dwordx4 v0, s[36:39], s53 offen offset:2048 sc0 sc1 lds\n\t" \
  "v_mfma_f32_16x16x128_f8f6f4 a[100:103], v[152:159], v[200:207], a[100:103]\n\t" \
  "buffer_load_dwordx4 v0, s[36:39], s53 offen offset:3072 sc0 sc1 lds\n\t" \
  "v_mfma_f32_16x16x128_f8f6f4 a[96:99], v[152:159], v[192:199], a[96:99]\n\t" \
  "buffer_load_dword v7, v8, s[44:47], s76 offen sc0 sc1\n\t" \
  "s_waitcnt vmcnt(0)\n\t" \
  "v_cmp_le_u32_e32 vcc, s75, v7\n\t" \
  "s_nop 1\n\t" \
  "s_cmp_eq_u64 vcc, -1\n\t" \
  "s_cbranch_scc1 L_OKN01_%=\n\t" \
  "s_swappc_b64 s[80:81], s[78:79]\n\t" \
  "L_OKN01_%=:\n\t" \
  "s_add_u32 s76, s76, 1024\n\t" \
  "s_waitcnt lgkmcnt(0)\n\t" \
  "v_mfma_f32_16x16x128_f8f6f4 a[128:131], v[160:167], v[192:199], a[128:131]\n\t" \
  "s_barrier\n\t" \
  "v_mfma_f32_16x16x128_f8f6f4 a[132:135], v[160:167], v[200:207], a[132:135]\n\t" \
  "ds_read_b128 v[128:131], v2 offset:32768\n\t" \
  "ds_read_b128 v[132:135], v2 offset:40960\n\t" \
  "v_mfma_f32_16x16x128_f8f6f4 a[136:139], v[160:167], v[208:215], a[136:139]\n\t" \
  "ds_read_b128 v[136:139], v2 offset:33792\n\t" \
  "ds_read_b128 v[140:143], v2 offset:41984\n\t" \
  "v_mfma_f32_16x16x128_f8f6f4 a[140:143], v[160:167], v[216:223], a[140:143]\n\t" \
  "ds_read_b128 v[144:147], v2 offset:34816\n\t" \
  "ds_read_b128 v[148:151], v2 offset:43008\n\t" \
  "v_mfma_f32_16x16x128_f8f6f4 a[172:175], v[168:175], v[216:223], a[172:175]\n\t" \
  "ds_read_b128 v[152:155], v2 offset:35840\n\t" \
  "ds_read_b128 v[156:159], v2 offset:44032\n\t" \
  "v_mfma_f32_16x16x128_f8f6f4 a[168:171], v[168:175], v[208:215], a[168:171]\n\t" \
  "ds_read_b128 v[240:243], v4 offset:18432\n\t" \
  "ds_read_b128 v[244:247], v4 offset:26624\n\t" \
  "v_mfma_f32_16x16x128_f8f6f4 a[164:167], v[168:175], v[200:207], a[164:167]\n\t" \
  "ds_read_b128 v[248:251], v4 offset:19456\n\t" \
  "ds_read_b128 v[252:255], v4 offset:27648\n\t" \
  "v_mfma_f32_16x16x128_f8f6f4 a[160:163], v[168:175], v[192:199], a[160:163]\n\t" \
  "s_add_u32 s53, s50, 65536\n\t" \
  "s_add_u32 m0, s48, 65536\n\t" \
  "v_mfma_f32_16x16x128_f8f6f4 a[192:195], v[176:183], v[192:199], a[192:195]\n\t" \
  "buffer_load_dwordx4 v0, s[36:39], s53 offen sc0 sc1 lds\n\t" \
  "v_mfma_f32_16x16x128_f8f6f4 a[196:199], v[176:183], v[200:207], a[196:199]\n\t" \
  "v_mfma_f32_16x16x128_f8f6f4 a[200:203], v[176:183], v[208:215], a[200:203]\n\t" \
  "buffer_load_dwordx4 v0, s[36:39], s53 offen offset:1024 sc0 sc1 lds\n\t" \
  "v_mfma_f32_16x16x128_f8f6f4 a[204:207], v[176:183], v[216:223], a[204:207]\n\t" \
  "v_mfma_f32_16x16x128_f8f6f4 a[236:239], v[184:191], v[216:223], a[236:239]\n\t" \
  "buffer_load_dwordx4 v0, s[36:39], s53 offen offset:2048 sc0 sc1 lds\n\t" \
  "v_mfma_f32_16x16x128_f8f6f4 a[232:235], v[184:191], v[208:215], a[232:235]\n\t" \
  "v_mfma_f32_16x16x128_f8f6f4 a[228:231], v[184:191], v[200:207], a[228:231]\n\t" \
  "buffer_load_dwordx4 v0, s[36:39], s53 offen offset:3072 sc0 sc1 lds\n\t" \
  "v_mfma_f32_16x16x128_f8f6f4 a[224:227], v[184:191], v[192:199], a[224:227]\n\t" \
  "s_waitcnt lgkmcnt(0)\n\t" \
  "v_mfma_f32_16x16x128_f8f6f4 a[0:3], v[128:135], v[224:231], a[0:3]\n\t" \
  "s_barrier\n\t" \
  "v_mfma_f32_16x16x128_f8f6f4 a[4:7], v[128:135], v[232:239], a[4:7]\n\t" \
  "ds_read_b128 v[160:163], v126 offset:0\n\t" \
  "ds_read_b128 v[164:167], v126 offset:8192\n\t" \
  "v_mfma_f32_16x16x128_f8f6f4 a[8:11], v[128:135], v[240:247], a[8:11]\n\t" \
  "ds_read_b128 v[168:171], v126 offset:1024\n\t" \
  "ds_read_b128 v[172:175], v126 offset:9216\n\t" \
  "v_mfma_f32_16x16x128_f8f6f4 a[12:15], v[128:135], v[248:255], a[12:15]\n\t" \
  "ds_read_b128 v[176:179], v126 offset:2048\n\t" \
  "ds_read_b128 v[180:183], v126 offset:10240\n\t" \
  "v_mfma_f32_16x16x128_f8f6f4 a[44:47], v[136:143], v[248:255], a[44:47]\n\t" \
  "ds_read_b128 v[184:187], v126 offset:3072\n\t" \
  "ds_read_b128 v[188:191], v126 offset:11264\n\t" \
  "v_mfma_f32_16x16x128_f8f6f4 a[40:43], v[136:143], v[240:247], a[40:43]\n\t" \
  "ds_read_b128 v[192:195], v127 offset:16384\n\t" \
  "ds_read_b128 v[196:199], v127 offset:24576\n\t" \
  "v_mfma_f32_16x16x128_f8f6f4 a[36:39], v[136:143], v[232:239], a[36:39]\n\t" \
  "ds_read_b128 v[200:203], v127 offset:17408\n\t" \
  "ds_read_b128 v[204:207], v127 offset:25600\n\t" \
  "s_add_u32 s53, s52, 131072\n\t" \
  "s_add_u32 m0, s48, 81920\n\t" \
  "v_mfma_f32_16x16x128_f8f6f4 a[32:35], v[136:143], v[224:231], a[32:35]\n\t" \
  "buffer_load_dwordx4 v0, s[36:39], s53 offen sc0 sc1 lds\n\t" \
  "v_mfma_f32_16x16x128_f8f6f4 a[64:67], v[144:151], v[224:231], a[64:67]\n\t" \
  "buffer_load_dwordx4 v0, s[36:39], s53 offen offset:1024 sc0 sc1 lds\n\t" \
  "v_mfma_f32_16x16x128_f8f6f4 a[68:71], v[144:151], v[232:239], a[68:71]\n\t" \
  "buffer_load_dwordx4 v0, s[36:39], s53 offen offset:2048 sc0 sc1 lds\n\t" \
  "v_mfma_f32_16x16x128_f8f6f4 a[72:75], v[144:151], v[240:247], a[72:75]\n\t" \
  "buffer_load_dwordx4 v0, s[36:39], s53 offen offset:3072 sc0 sc1 lds\n\t" \
  "s_add_u32 s53, s52, 147456\n\t" \
  "s_add_u32 m0, s48, 98304\n\t" \
  "v_mfma_f32_16x16x128_f8f6f4 a[76:79], v[144:151], v[248:255], a[76:79]\n\t" \
  "buffer_load_dwordx4 v0, s[36:39], s53 offen sc0 sc1 lds\n\t" \
  "v_mfma_f32_16x16x128_f8f6f4 a[108:111], v[152:159], v[248:255], a[108:111]\n\t" \
  "buffer_load_dwordx4 v0, s[36:39], s53 offen offset:1024 sc0 sc1 lds\n\t" \
  "v_mfma_f32_16x16x128_f8f6f4 a[104:107], v[152:159], v[240:247], a[104:107]\n\t" \
  "buffer_load_dwordx4 v0, s[36:39], s53 offen offset:2048 sc0 sc1 lds\n\t" \
  "v_mfma_f32_16x16x128_f8f6f4 a[100:103], v[152:159], v[232:239], a[100:103]\n\t" \
  "buffer_load_dwordx4 v0, s[36:39], s53 offen offset:3072 sc0 sc1 lds\n\t" \
  "v_mfma_f32_16x16x128_f8f6f4 a[96:99], v[152:159], v[224:231], a[96:99]\n\t" \
  "s_waitcnt lgkmcnt(0)\n\t" \
  "v_mfma_f32_16x16x128_f8f6f4 a[128:131], v[160:167], v[224:231], a[128:131]\n\t" \
  "s_barrier\n\t" \
  "v_mfma_f32_16x16x128_f8f6f4 a[132:135], v[160:167], v[232:239], a[132:135]\n\t" \
  "ds_read_b128 v[128:131], v1 offset:0\n\t" \
  "ds_read_b128 v[132:135], v1 offset:8192\n\t" \
  "v_mfma_f32_16x16x128_f8f6f4 a[136:139], v[160:167], v[240:247], a[136:139]\n\t" \
  "ds_read_b128 v[136:139], v1 offset:1024\n\t" \
  "ds_read_b128 v[140:143], v1 offset:9216\n\t" \
  "v_mfma_f32_16x16x128_f8f6f4 a[140:143], v[160:167], v[248:255], a[140:143]\n\t" \
  "ds_read_b128 v[144:147], v1 offset:2048\n\t" \
  "ds_read_b128 v[148:151], v1 offset:10240\n\t" \
  "v_mfma_f32_16x16x128_f8f6f4 a[172:175], v[168:175], v[248:255], a[172:175]\n\t" \
  "ds_read_b128 v[152:155], v1 offset:3072\n\t" \
  "ds_read_b128 v[156:159], v1 offset:11264\n\t" \
  "v_mfma_f32_16x16x128_f8f6f4 a[168:171], v[168:175], v[240:247], a[168:171]\n\t" \
  "ds_read_b128 v[208:211], v127 offset:18432\n\t" \
  "ds_read_b128 v[212:215], v127 offset:26624\n\t" \
  "v_mfma_f32_16x16x128_f8f6f4 a[164:167], v[168:175], v[232:239], a[164:167]\n\t" \
  "ds_read_b128 v[216:219], v127 offset:19456\n\t" \
  "ds_read_b128 v[220:223], v127 offset:27648\n\t" \
  "v_mfma_f32_16x16x128_f8f6f4 a[160:163], v[168:175], v[224:231], a[160:163]\n\t" \
  "s_add_u32 s53, s50, 81920\n\t" \
  "s_add_u32 m0, s48, 131072\n\t" \
  "v_mfma_f32_16x16x128_f8f6f4 a[192:195], v[176:183], v[224:231], a[192:195]\n\t" \
  "buffer_load_dwordx4 v0, s[36:39], s53 offen sc0 sc1 lds\n\t" \
  "v_mfma_f32_16x16x128_f8f6f4 a[196:199], v[176:183], v[232:239], a[196:199]\n\t" \
  "v_mfma_f32_16x16x128_f8f6f4 a[200:203], v[176:183], v[240:247], a[200:203]\n\t" \
  "buffer_load_dwordx4 v0, s[36:39], s53 offen offset:1024 sc0 sc1 lds\n\t" \
  "v_mfma_f32_16x16x128_f8f6f4 a[204:207], v[176:183], v[248:255], a[204:207]\n\t" \
  "v_mfma_f32_16x16x128_f8f6f4 a[236:239], v[184:191], v[248:255], a[236:239]\n\t" \
  "buffer_load_dwordx4 v0, s[36:39], s53 offen offset:2048 sc0 sc1 lds\n\t" \
  "v_mfma_f32_16x16x128_f8f6f4 a[232:235], v[184:191], v[240:247], a[232:235]\n\t" \
  "v_mfma_f32_16x16x128_f8f6f4 a[228:231], v[184:191], v[232:239], a[228:231]\n\t" \
  "buffer_load_dwordx4 v0, s[36:39], s53 offen offset:3072 sc0 sc1 lds\n\t" \
  "v_mfma_f32_16x16x128_f8f6f4 a[224:227], v[184:191], v[224:231], a[224:227]\n\t" \
  "s_waitcnt lgkmcnt(0)\n\t" \
  "v_mfma_f32_16x16x128_f8f6f4 a[0:3], v[128:135], v[192:199], a[0:3]\n\t" \
  "s_barrier\n\t" \
  "v_mfma_f32_16x16x128_f8f6f4 a[4:7], v[128:135], v[200:207], a[4:7]\n\t" \
  "ds_read_b128 v[160:163], v1 offset:32768\n\t" \
  "ds_read_b128 v[164:167], v1 offset:40960\n\t" \
  "v_mfma_f32_16x16x128_f8f6f4 a[8:11], v[128:135], v[208:215], a[8:11]\n\t" \
  "ds_read_b128 v[168:171], v1 offset:33792\n\t" \
  "ds_read_b128 v[172:175], v1 offset:41984\n\t" \
  "v_mfma_f32_16x16x128_f8f6f4 a[12:15], v[128:135], v[216:223], a[12:15]\n\t" \
  "ds_read_b128 v[176:179], v1 offset:34816\n\t" \
  "ds_read_b128 v[180:183], v1 offset:43008\n\t" \
  "v_mfma_f32_16x16x128_f8f6f4 a[44:47], v[136:143], v[216:223], a[44:47]\n\t" \
  "ds_read_b128 v[184:187], v1 offset:35840\n\t" \
  "ds_read_b128 v[188:191], v1 offset:44032\n\t" \
  "v_mfma_f32_16x16x128_f8f6f4 a[40:43], v[136:143], v[208:215], a[40:43]\n\t" \
  "ds_read_b128 v[224:227], v3 offset:49152\n\t" \
  "ds_read_b128 v[228:231], v3 offset:57344\n\t" \
  "v_mfma_f32_16x16x128_f8f6f4 a[36:39], v[136:143], v[200:207], a[36:39]\n\t" \
  "ds_read_b128 v[232:235], v3 offset:50176\n\t" \
  "ds_read_b128 v[236:239], v3 offset:58368\n\t" \
  "s_add_u32 s53, s52, 163840\n\t" \
  "s_add_u32 m0, s48, 147456\n\t" \
  "v_mfma_f32_16x16x128_f8f6f4 a[32:35], v[136:143], v[192:199], a[32:35]\n\t" \
  "buffer_load_dwordx4 v0, s[36:39], s53 offen sc0 sc1 lds\n\t" \
  "v_mfma_f32_16x16x128_f8f6f4 a[64:67], v[144:151], v[192:199], a[64:67]\n\t" \
  "buffer_load_dwordx4 v0, s[36:39], s53 offen offset:1024 sc0 sc1 lds\n\t" \
  "v_mfma_f32_16x16x128_f8f6f4 a[68:71], v[144:151], v[200:207], a[68:71]\n\t" \
  "buffer_load_dwordx4 v0, s[36:39], s53 offen offset:2048 sc0 sc1 lds\n\t" \
  "v_mfma_f32_16x16x128_f8f6f4 a[72:75], v[144:151], v[208:215], a[72:75]\n\t" \
  "buffer_load_dwordx4 v0, s[36:39], s53 offen offset:3072 sc0 sc1 lds\n\t" \
  "s_add_u32 s53, s52, 180224\n\t" \
  "s_mov_b32 m0, s48\n\t" \
  "v_mfma_f32_16x16x128_f8f6f4 a[76:79], v[144:151], v[216:223], a[76:79]\n\t" \
  "buffer_load_dwordx4 v0, s[36:39], s53 offen sc0 sc1 lds\n\t" \
  "v_mfma_f32_16x16x128_f8f6f4 a[108:111], v[152:159], v[216:223], a[108:111]\n\t" \
  "buffer_load_dwordx4 v0, s[36:39], s53 offen offset:1024 sc0 sc1 lds\n\t" \
  "v_mfma_f32_16x16x128_f8f6f4 a[104:107], v[152:159], v[208:215], a[104:107]\n\t" \
  "buffer_load_dwordx4 v0, s[36:39], s53 offen offset:2048 sc0 sc1 lds\n\t" \
  "v_mfma_f32_16x16x128_f8f6f4 a[100:103], v[152:159], v[200:207], a[100:103]\n\t" \
  "buffer_load_dwordx4 v0, s[36:39], s53 offen offset:3072 sc0 sc1 lds\n\t" \
  "v_mfma_f32_16x16x128_f8f6f4 a[96:99], v[152:159], v[192:199], a[96:99]\n\t" \
  "s_waitcnt lgkmcnt(0)\n\t" \
  "v_mfma_f32_16x16x128_f8f6f4 a[128:131], v[160:167], v[192:199], a[128:131]\n\t" \
  "s_waitcnt vmcnt(29)\n\t" \
  "s_barrier\n\t" \
  "v_mfma_f32_16x16x128_f8f6f4 a[132:135], v[160:167], v[200:207], a[132:135]\n\t" \
  "ds_read_b128 v[128:131], v2 offset:49152\n\t" \
  "ds_read_b128 v[132:135], v2 offset:57344\n\t" \
  "v_mfma_f32_16x16x128_f8f6f4 a[136:139], v[160:167], v[208:215], a[136:139]\n\t" \
  "ds_read_b128 v[136:139], v2 offset:50176\n\t" \
  "ds_read_b128 v[140:143], v2 offset:58368\n\t" \
  "v_mfma_f32_16x16x128_f8f6f4 a[140:143], v[160:167], v[216:223], a[140:143]\n\t" \
  "ds_read_b128 v[144:147], v2 offset:51200\n\t" \
  "ds_read_b128 v[148:151], v2 offset:59392\n\t" \
  "v_mfma_f32_16x16x128_f8f6f4 a[172:175], v[168:175], v[216:223], a[172:175]\n\t" \
  "ds_read_b128 v[152:155], v2 offset:52224\n\t" \
  "ds_read_b128 v[156:159], v2 offset:60416\n\t" \
  "v_mfma_f32_16x16x128_f8f6f4 a[168:171], v[168:175], v[208:215], a[168:171]\n\t" \
  "ds_read_b128 v[240:243], v3 offset:51200\n\t" \
  "ds_read_b128 v[244:247], v3 offset:59392\n\t" \
  "v_mfma_f32_16x16x128_f8f6f4 a[164:167], v[168:175], v[200:207], a[164:167]\n\t" \
  "ds_read_b128 v[248:251], v3 offset:52224\n\t" \
  "ds_read_b128 v[252:255], v3 offset:60416\n\t" \
  "v_mfma_f32_16x16x128_f8f6f4 a[160:163], v[168:175], v[192:199], a[160:163]\n\t" \
  "s_add_u32 s53, s50, 98304\n\t" \
  "s_add_u32 m0, s48, 32768\n\t" \
  "v_mfma_f32_16x16x128_f8f6f4 a[192:195], v[176:183], v[192:199], a[192:195]\n\t" \
  "buffer_load_dwordx4 v0, s[36:39], s53 offen sc0 sc1 lds\n\t" \
  "v_mfma_f32_16x16x128_f8f6f4 a[196:199], v[176:183], v[200:207], a[196:199]\n\t" \
  "v_mfma_f32_16x16x128_f8f6f4 a[200:203], v[176:183], v[208:215], a[200:203]\n\t" \
  "buffer_load_dwordx4 v0, s[36:39], s53 offen offset:1024 sc0 sc1 lds\n\t" \
  "v_mfma_f32_16x16x128_f8f6f4 a[204:207], v[176:183], v[216:223], a[204:207]\n\t" \
  "v_mfma_f32_16x16x128_f8f6f4 a[236:239], v[184:191], v[216:223], a[236:239]\n\t" \
  "buffer_load_dwordx4 v0, s[36:39], s53 offen offset:2048 sc0 sc1 lds\n\t" \
  "v_mfma_f32_16x16x128_f8f6f4 a[232:235], v[184:191], v[208:215], a[232:235]\n\t" \
  "v_mfma_f32_16x16x128_f8f6f4 a[228:231], v[184:191], v[200:207], a[228:231]\n\t" \
  "buffer_load_dwordx4 v0, s[36:39], s53 offen offset:3072 sc0 sc1 lds\n\t" \
  "v_mfma_f32_16x16x128_f8f6f4 a[224:227], v[184:191], v[192:199], a[224:227]\n\t" \
  "s_waitcnt lgkmcnt(0)\n\t" \
  "v_mfma_f32_16x16x128_f8f6f4 a[0:3], v[128:135], v[224:231], a[0:3]\n\t" \
  "s_waitcnt vmcnt(24)\n\t" \
  "s_barrier\n\t" \
  "v_mfma_f32_16x16x128_f8f6f4 a[4:7], v[128:135], v[232:239], a[4:7]\n\t" \
  "ds_read_b128 v[160:163], v1 offset:16384\n\t" \
  "ds_read_b128 v[164:167], v1 offset:24576\n\t" \
  "v_mfma_f32_16x16x128_f8f6f4 a[8:11], v[128:135], v[240:247], a[8:11]\n\t" \
  "ds_read_b128 v[168:171], v1 offset:17408\n\t" \
  "ds_read_b128 v[172:175], v1 offset:25600\n\t" \
  "v_mfma_f32_16x16x128_f8f6f4 a[12:15], v[128:135], v[248:255], a[12:15]\n\t" \
  "ds_read_b128 v[176:179], v1 offset:18432\n\t" \
  "ds_read_b128 v[180:183], v1 offset:26624\n\t" \
  "v_mfma_f32_16x16x128_f8f6f4 a[44:47], v[136:143], v[248:255], a[44:47]\n\t" \
  "ds_read_b128 v[184:187], v1 offset:19456\n\t" \
  "ds_read_b128 v[188:191], v1 offset:27648\n\t" \
  "v_mfma_f32_16x16x128_f8f6f4 a[40:43], v[136:143], v[240:247], a[40:43]\n\t" \
  "ds_read_b128 v[192:195], v4 offset:0\n\t" \
  "ds_read_b128 v[196:199], v4 offset:8192\n\t" \
  "v_mfma_f32_16x16x128_f8f6f4 a[36:39], v[136:143], v[232:239], a[36:39]\n\t" \
  "ds_read_b128 v[200:203], v4 offset:1024\n\t" \
  "ds_read_b128 v[204:207], v4 offset:9216\n\t" \
  "s_add_u32 s53, s52, 196608\n\t" \
  "s_add_u32 m0, s48, 114688\n\t" \
  "v_mfma_f32_16x16x128_f8f6f4 a[32:35], v[136:143], v[224:231], a[32:35]\n\t" \
  "buffer_load_dwordx4 v0, s[36:39], s53 offen sc0 sc1 lds\n\t" \
  "v_mfma_f32_16x16x128_f8f6f4 a[64:67], v[144:151], v[224:231], a[64:67]\n\t" \
  "buffer_load_dwordx4 v0, s[36:39], s53 offen offset:1024 sc0 sc1 lds\n\t" \
  "v_mfma_f32_16x16x128_f8f6f4 a[68:71], v[144:151], v[232:239], a[68:71]\n\t" \
  "buffer_load_dwordx4 v0, s[36:39], s53 offen offset:2048 sc0 sc1 lds\n\t" \
  "v_mfma_f32_16x16x128_f8f6f4 a[72:75], v[144:151], v[240:247], a[72:75]\n\t" \
  "buffer_load_dwordx4 v0, s[36:39], s53 offen offset:3072 sc0 sc1 lds\n\t" \
  "s_add_u32 s53, s52, 212992\n\t" \
  "s_add_u32 m0, s48, 49152\n\t" \
  "v_mfma_f32_16x16x128_f8f6f4 a[76:79], v[144:151], v[248:255], a[76:79]\n\t" \
  "buffer_load_dwordx4 v0, s[36:39], s53 offen sc0 sc1 lds\n\t" \
  "v_mfma_f32_16x16x128_f8f6f4 a[108:111], v[152:159], v[248:255], a[108:111]\n\t" \
  "buffer_load_dwordx4 v0, s[36:39], s53 offen offset:1024 sc0 sc1 lds\n\t" \
  "v_mfma_f32_16x16x128_f8f6f4 a[104:107], v[152:159], v[240:247], a[104:107]\n\t" \
  "buffer_load_dwordx4 v0, s[36:39], s53 offen offset:2048 sc0 sc1 lds\n\t" \
  "v_mfma_f32_16x16x128_f8f6f4 a[100:103], v[152:159], v[232:239], a[100:103]\n\t" \
  "buffer_load_dwordx4 v0, s[36:39], s53 offen offset:3072 sc0 sc1 lds\n\t" \
  "v_mfma_f32_16x16x128_f8f6f4 a[96:99], v[152:159], v[224:231], a[96:99]\n\t" \
  "buffer_load_dword v7, v8, s[44:47], s76 offen sc0 sc1\n\t" \
  "s_waitcnt lgkmcnt(0)\n\t" \
  "v_mfma_f32_16x16x128_f8f6f4 a[128:131], v[160:167], v[224:231], a[128:131]\n\t" \
  "s_waitcnt vmcnt(29)\n\t" \
  "s_barrier\n\t" \
  "v_mfma_f32_16x16x128_f8f6f4 a[132:135], v[160:167], v[232:239], a[132:135]\n\t" \
  "ds_read_b128 v[128:131], v2 offset:16384\n\t" \
  "ds_read_b128 v[132:135], v2 offset:24576\n\t" \
  "v_mfma_f32_16x16x128_f8f6f4 a[136:139], v[160:167], v[240:247], a[136:139]\n\t" \
  "ds_read_b128 v[136:139], v2 offset:17408\n\t" \
  "ds_read_b128 v[140:143], v2 offset:25600\n\t" \
  "v_mfma_f32_16x16x128_f8f6f4 a[140:143], v[160:167], v[248:255], a[140:143]\n\t" \
  "ds_read_b128 v[144:147], v2 offset:18432\n\t" \
  "ds_read_b128 v[148:151], v2 offset:26624\n\t" \
  "v_mfma_f32_16x16x128_f8f6f4 a[172:175], v[168:175], v[248:255], a[172:175]\n\t" \
  "ds_read_b128 v[152:155], v2 offset:19456\n\t" \
  "ds_read_b128 v[156:159], v2 offset:27648\n\t" \
  "v_mfma_f32_16x16x128_f8f6f4 a[168:171], v[168:175], v[240:247], a[168:171]\n\t" \
  "ds_read_b128 v[208:211], v4 offset:2048\n\t" \
  "ds_read_b128 v[212:215], v4 offset:10240\n\t" \
  "v_mfma_f32_16x16x128_f8f6f4 a[164:167], v[168:175], v[232:239], a[164:167]\n\t" \
  "ds_read_b128 v[216:219], v4 offset:3072\n\t" \
  "ds_read_b128 v[220:223], v4 offset:11264\n\t" \
  "v_mfma_f32_16x16x128_f8f6f4 a[160:163], v[168:175], v[224:231], a[160:163]\n\t" \
  "s_add_u32 s53, s50, 114688\n\t" \
  "s_add_u32 m0, s48, 16384\n\t" \
  "v_mfma_f32_16x16x128_f8f6f4 a[192:195], v[176:183], v[224:231], a[192:195]\n\t" \
  "buffer_load_dwordx4 v0, s[36:39], s53 offen sc0 sc1 lds\n\t" \
  "v_mfma_f32_16x16x128_f8f6f4 a[196:199], v[176:183], v[232:239], a[196:199]\n\t" \
  "v_mfma_f32_16x16x128_f8f6f4 a[200:203], v[176:183], v[240:247], a[200:203]\n\t" \
  "buffer_load_dwordx4 v0, s[36:39], s53 offen offset:1024 sc0 sc1 lds\n\t" \
  "v_mfma_f32_16x16x128_f8f6f4 a[204:207], v[176:183], v[248:255], a[204:207]\n\t" \
  "v_mfma_f32_16x16x128_f8f6f4 a[236:239], v[184:191], v[248:255], a[236:239]\n\t" \
  "buffer_load_dwordx4 v0, s[36:39], s53 offen offset:2048 sc0 sc1 lds\n\t" \
  "v_mfma_f32_16x16x128_f8f6f4 a[232:235], v[184:191], v[240:247], a[232:235]\n\t" \
  "v_mfma_f32_16x16x128_f8f6f4 a[228:231], v[184:191], v[232:239], a[228:231]\n\t" \
  "buffer_load_dwordx4 v0, s[36:39], s53 offen offset:3072 sc0 sc1 lds\n\t" \
  "v_mfma_f32_16x16x128_f8f6f4 a[224:227], v[184:191], v[224:231], a[224:227]\n\t" \
  "s_waitcnt lgkmcnt(0)\n\t" \
  "v_mfma_f32_16x16x128_f8f6f4 a[0:3], v[128:135], v[192:199], a[0:3]\n\t" \
  "s_waitcnt vmcnt(25)\n\t" \
  "s_barrier\n\t" \
  "v_mfma_f32_16x16x128_f8f6f4 a[4:7], v[128:135], v[200:207], a[4:7]\n\t" \
  "ds_read_b128 v[160:163], v2 offset:32768\n\t" \
  "ds_read_b128 v[164:167], v2 offset:40960\n\t" \
  "v_mfma_f32_16x16x128_f8f6f4 a[8:11], v[128:135], v[208:215], a[8:11]\n\t" \
  "ds_read_b128 v[168:171], v2 offset:33792\n\t" \
  "ds_read_b128 v[172:175], v2 offset:41984\n\t" \
  "v_mfma_f32_16x16x128_f8f6f4 a[12:15], v[128:135], v[216:223], a[12:15]\n\t" \
  "ds_read_b128 v[176:179], v2 offset:34816\n\t" \
  "ds_read_b128 v[180:183], v2 offset:43008\n\t" \
  "v_mfma_f32_16x16x128_f8f6f4 a[44:47], v[136:143], v[216:223], a[44:47]\n\t" \
  "ds_read_b128 v[184:187], v2 offset:35840\n\t" \
  "ds_read_b128 v[188:191], v2 offset:44032\n\t" \
  "v_mfma_f32_16x16x128_f8f6f4 a[40:43], v[136:143], v[208:215], a[40:43]\n\t" \
  "ds_read_b128 v[224:227], v127 offset:0\n\t" \
  "ds_read_b128 v[228:231], v127 offset:8192\n\t" \
  "v_mfma_f32_16x16x128_f8f6f4 a[36:39], v[136:143], v[200:207], a[36:39]\n\t" \
  "ds_read_b128 v[232:235], v127 offset:1024\n\t" \
  "ds_read_b128 v[236:239], v127 offset:9216\n\t" \
  "s_add_u32 s53, s52, 229376\n\t" \
  "s_add_u32 m0, s48, 65536\n\t" \
  "v_mfma_f32_16x16x128_f8f6f4 a[32:35], v[136:143], v[192:199], a[32:35]\n\t" \
  "buffer_load_dwordx4 v0, s[36:39], s53 offen sc0 sc1 lds\n\t" \
  "v_mfma_f32_16x16x128_f8f6f4 a[64:67], v[144:151], v[192:199], a[64:67]\n\t" \
  "buffer_load_dwordx4 v0, s[36:39], s53 offen offset:1024 sc0 sc1 lds\n\t" \
  "v_mfma_f32_16x16x128_f8f6f4 a[68:71], v[144:151], v[200:207], a[68:71]\n\t" \
  "buffer_load_dwordx4 v0, s[36:39], s53 offen offset:2048 sc0 sc1 lds\n\t" \
  "v_mfma_f32_16x16x128_f8f6f4 a[72:75], v[144:151], v[208:215], a[72:75]\n\t" \
  "buffer_load_dwordx4 v0, s[36:39], s53 offen offset:3072 sc0 sc1 lds\n\t" \
  "s_add_u32 s53, s52, 245760\n\t" \
  "s_add_u32 m0, s48, 81920\n\t" \
  "v_mfma_f32_16x16x128_f8f6f4 a[76:79], v[144:151], v[216:223], a[76:79]\n\t" \
  "buffer_load_dwordx4 v0, s[36:39], s53 offen sc0 sc1 lds\n\t" \
  "v_mfma_f32_16x16x128_f8f6f4 a[108:111], v[152:159], v[216:223], a[108:111]\n\t" \
  "buffer_load_dwordx4 v0, s[36:39], s53 offen offset:1024 sc0 sc1 lds\n\t" \
  "v_mfma_f32_16x16x128_f8f6f4 a[104:107], v[152:159], v[208:215], a[104:107]\n\t" \
  "buffer_load_dwordx4 v0, s[36:39], s53 offen offset:2048 sc0 sc1 lds\n\t" \
  "v_mfma_f32_16x16x128_f8f6f4 a[100:103], v[152:159], v[200:207], a[100:103]\n\t" \
  "buffer_load_dwordx4 v0, s[36:39], s53 offen offset:3072 sc0 sc1 lds\n\t" \
  "v_mfma_f32_16x16x128_f8f6f4 a[96:99], v[152:159], v[192:199], a[96:99]\n\t" \
  "s_waitcnt vmcnt(12)\n\t" \
  "v_cmp_le_u32_e32 vcc, s75, v7\n\t" \
  "s_nop 1\n\t" \
  "s_cmp_eq_u64 vcc, -1\n\t" \
  "s_cbranch_scc1 L_OKN02_%=\n\t" \
  "s_swappc_b64 s[80:81], s[78:79]\n\t" \
  "L_OKN02_%=:\n\t" \
  "s_add_u32 s76, s76, 1024\n\t" \
  "s_waitcnt lgkmcnt(0)\n\t" \
  "v_mfma_f32_16x16x128_f8f6f4 a[128:131], v[160:167], v[192:199], a[128:131]\n\t" \
  "s_waitcnt vmcnt(29)\n\t" \
  "s_barrier\n\t" \
  "v_mfma_f32_16x16x128_f8f6f4 a[132:135], v[160:167], v[200:207], a[132:135]\n\t" \
  "ds_read_b128 v[128:131], v126 offset:16384\n\t" \
  "ds_read_b128 v[132:135], v126 offset:24576\n\t" \
  "v_mfma_f32_16x16x128_f8f6f4 a[136:139], v[160:167], v[208:215], a[136:139]\n\t" \
  "ds_read_b128 v[136:139], v126 offset:17408\n\t" \
  "ds_read_b128 v[140:143], v126 offset:25600\n\t" \
  "v_mfma_f32_16x16x128_f8f6f4 a[140:143], v[160:167], v[216:223], a[140:143]\n\t" \
  "ds_read_b128 v[144:147], v126 offset:18432\n\t" \
  "ds_read_b128 v[148:151], v126 offset:26624\n\t" \
  "v_mfma_f32_16x16x128_f8f6f4 a[172:175], v[168:175], v[216:223], a[172:175]\n\t" \
  "ds_read_b128 v[152:155], v126 offset:19456\n\t" \
  "ds_read_b128 v[156:159], v126 offset:27648\n\t" \
  "v_mfma_f32_16x16x128_f8f6f4 a[168:171], v[168:175], v[208:215], a[168:171]\n\t" \
  "ds_read_b128 v[240:243], v127 offset:2048\n\t" \
  "ds_read_b128 v[244:247], v127 offset:10240\n\t" \
  "v_mfma_f32_16x16x128_f8f6f4 a[164:167], v[168:175], v[200:207], a[164:167]\n\t" \
  "ds_read_b128 v[248:251], v127 offset:3072\n\t" \
  "ds_read_b128 v[252:255], v127 offset:11264\n\t" \
  "v_mfma_f32_16x16x128_f8f6f4 a[160:163], v[168:175], v[192:199], a[160:163]\n\t" \
  "s_add_u32 s53, s50, 131072\n\t" \
  "s_add_u32 m0, s48, 98304\n\t" \
  "v_mfma_f32_16x16x128_f8f6f4 a[192:195], v[176:183], v[192:199], a[192:195]\n\t" \
  "buffer_load_dwordx4 v0, s[36:39], s53 offen sc0 sc1 lds\n\t" \
  "v_mfma_f32_16x16x128_f8f6f4 a[196:199], v[176:183], v[200:207], a[196:199]\n\t" \
  "v_mfma_f32_16x16x128_f8f6f4 a[200:203], v[176:183], v[208:215], a[200:203]\n\t" \
  "buffer_load_dwordx4 v0, s[36:39], s53 offen offset:1024 sc0 sc1 lds\n\t" \
  "v_mfma_f32_16x16x128_f8f6f4 a[204:207], v[176:183], v[216:223], a[204:207]\n\t" \
  "v_mfma_f32_16x16x128_f8f6f4 a[236:239], v[184:191], v[216:223], a[236:239]\n\t" \
  "buffer_load_dwordx4 v0, s[36:39], s53 offen offset:2048 sc0 sc1 lds\n\t" \
  "v_mfma_f32_16x16x128_f8f6f4 a[232:235], v[184:191], v[208:215], a[232:235]\n\t" \
  "v_mfma_f32_16x16x128_f8f6f4 a[228:231], v[184:191], v[200:207], a[228:231]\n\t" \
  "buffer_load_dwordx4 v0, s[36:39], s53 offen offset:3072 sc0 sc1 lds\n\t" \
  "v_mfma_f32_16x16x128_f8f6f4 a[224:227], v[184:191], v[192:199], a[224:227]\n\t" \
  "s_waitcnt lgkmcnt(0)\n\t" \
  "v_mfma_f32_16x16x128_f8f6f4 a[0:3], v[128:135], v[224:231], a[0:3]\n\t" \
  "s_waitcnt vmcnt(25)\n\t" \
  "s_barrier\n\t" \
  "v_mfma_f32_16x16x128_f8f6f4 a[4:7], v[128:135], v[232:239], a[4:7]\n\t" \
  "ds_read_b128 v[160:163], v1 offset:0\n\t" \
  "ds_read_b128 v[164:167], v1 offset:8192\n\t" \
  "v_mfma_f32_16x16x128_f8f6f4 a[8:11], v[128:135], v[240:247], a[8:11]\n\t" \
  "ds_read_b128 v[168:171], v1 offset:1024\n\t" \
  "ds_read_b128 v[172:175], v1 offset:9216\n\t" \
  "v_mfma_f32_16x16x128_f8f6f4 a[12:15], v[128:135], v[248:255], a[12:15]\n\t" \
  "ds_read_b128 v[176:179], v1 offset:2048\n\t" \
  "ds_read_b128 v[180:183], v1 offset:10240\n\t" \
  "v_mfma_f32_16x16x128_f8f6f4 a[44:47], v[136:143], v[248:255], a[44:47]\n\t" \
  "ds_read_b128 v[184:187], v1 offset:3072\n\t" \
  "ds_read_b128 v[188:191], v1 offset:11264\n\t" \
  "v_mfma_f32_16x16x128_f8f6f4 a[40:43], v[136:143], v[240:247], a[40:43]\n\t" \
  "ds_read_b128 v[192:195], v3 offset:32768\n\t" \
  "ds_read_b128 v[196:199], v3 offset:40960\n\t" \
  "v_mfma_f32_16x16x128_f8f6f4 a[36:39], v[136:143], v[232:239], a[36:39]\n\t" \
  "ds_read_b128 v[200:203], v3 offset:33792\n\t" \
  "ds_read_b128 v[204:207], v3 offset:41984\n\t" \
  "s_add_u32 s53, s52, 262144\n\t" \
  "s_add_u32 m0, s48, 131072\n\t" \
  "v_mfma_f32_16x16x128_f8f6f4 a[32:35], v[136:143], v[224:231], a[32:35]\n\t" \
  "buffer_load_dwordx4 v0, s[36:39], s53 offen sc0 sc1 lds\n\t" \
  "v_mfma_f32_16x16x128_f8f6f4 a[64:67], v[144:151], v[224:231], a[64:67]\n\t" \
  "buffer_load_dwordx4 v0, s[36:39], s53 offen offset:1024 sc0 sc1 lds\n\t" \
  "v_mfma_f32_16x16x128_f8f6f4 a[68:71], v[144:151], v[232:239], a[68:71]\n\t" \
  "buffer_load_dwordx4 v0, s[36:39], s53 offen offset:2048 sc0 sc1 lds\n\t" \
  "v_mfma_f32_16x16x128_f8f6f4 a[72:75], v[144:151], v[240:247], a[72:75]\n\t" \
  "buffer_load_dwordx4 v0, s[36:39], s53 offen offset:3072 sc0 sc1 lds\n\t" \
  "s_add_u32 s53, s52, 278528\n\t" \
  "s_add_u32 m0, s48, 147456\n\t" \
  "v_mfma_f32_16x16x128_f8f6f4 a[76:79], v[144:151], v[248:255], a[76:79]\n\t" \
  "buffer_load_dwordx4 v0, s[36:39], s53 offen sc0 sc1 lds\n\t" \
  "v_mfma_f32_16x16x128_f8f6f4 a[108:111], v[152:159], v[248:255], a[108:111]\n\t" \
  "buffer_load_dwordx4 v0, s[36:39], s53 offen offset:1024 sc0 sc1 lds\n\t" \
  "v_mfma_f32_16x16x128_f8f6f4 a[104:107], v[152:159], v[240:247], a[104:107]\n\t" \
  "buffer_load_dwordx4 v0, s[36:39], s53 offen offset:2048 sc0 sc1 lds\n\t" \
  "v_mfma_f32_16x16x128_f8f6f4 a[100:103], v[152:159], v[232:239], a[100:103]\n\t" \
  "buffer_load_dwordx4 v0, s[36:39], s53 offen offset:3072 sc0 sc1 lds\n\t" \
  "v_mfma_f32_16x16x128_f8f6f4 a[96:99], v[152:159], v[224:231], a[96:99]\n\t" \
  "s_waitcnt lgkmcnt(0)\n\t" \
  "v_mfma_f32_16x16x128_f8f6f4 a[128:131], v[160:167], v[224:231], a[128:131]\n\t" \
  "s_waitcnt vmcnt(29)\n\t" \
  "s_barrier\n\t" \
  "v_mfma_f32_16x16x128_f8f6f4 a[132:135], v[160:167], v[232:239], a[132:135]\n\t" \
  "ds_read_b128 v[128:131], v2 offset:49152\n\t" \
  "ds_read_b128 v[132:135], v2 offset:57344\n\t" \
  "v_mfma_f32_16x16x128_f8f6f4 a[136:139], v[160:167], v[240:247], a[136:139]\n\t" \
  "ds_read_b128 v[136:139], v2 offset:50176\n\t" \
  "ds_read_b128 v[140:143], v2 offset:58368\n\t" \
  "v_mfma_f32_16x16x128_f8f6f4 a[140:143], v[160:167], v[248:255], a[140:143]\n\t" \
  "ds_read_b128 v[144:147], v2 offset:51200\n\t" \
  "ds_read_b128 v[148:151], v2 offset:59392\n\t" \
  "v_mfma_f32_16x16x128_f8f6f4 a[172:175], v[168:175], v[248:255], a[172:175]\n\t" \
  "ds_read_b128 v[152:155], v2 offset:52224\n\t" \
  "ds_read_b128 v[156:159], v2 offset:60416\n\t" \
  "v_mfma_f32_16x16x128_f8f6f4 a[168:171], v[168:175], v[240:247], a[168:171]\n\t" \
  "ds_read_b128 v[208:211], v3 offset:34816\n\t" \
  "ds_read_b128 v[212:215], v3 offset:43008\n\t" \
  "v_mfma_f32_16x16x128_f8f6f4 a[164:167], v[168:175], v[232:239], a[164:167]\n\t" \
  "ds_read_b128 v[216:219], v3 offset:35840\n\t" \
  "ds_read_b128 v[220:223], v3 offset:44032\n\t" \
  "v_mfma_f32_16x16x128_f8f6f4 a[160:163], v[168:175], v[224:231], a[160:163]\n\t" \
  "s_add_u32 s53, s50, 147456\n\t" \
  "s_mov_b32 m0, s48\n\t" \
  "v_mfma_f32_16x16x128_f8f6f4 a[192:195], v[176:183], v[224:231], a[192:195]\n\t" \
  "buffer_load_dwordx4 v0, s[36:39], s53 offen sc0 sc1 lds\n\t" \
  "v_mfma_f32_16x16x128_f8f6f4 a[196:199], v[176:183], v[232:239], a[196:199]\n\t" \
  "v_mfma_f32_16x16x128_f8f6f4 a[200:203], v[176:183], v[240:247], a[200:203]\n\t" \
  "buffer_load_dwordx4 v0, s[36:39], s53 offen offset:1024 sc0 sc1 lds\n\t" \
  "v_mfma_f32_16x16x128_f8f6f4 a[204:207], v[176:183], v[248:255], a[204:207]\n\t" \
  "v_mfma_f32_16x16x128_f8f6f4 a[236:239], v[184:191], v[248:255], a[236:239]\n\t" \
  "buffer_load_dwordx4 v0, s[36:39], s53 offen offset:2048 sc0 sc1 lds\n\t" \
  "v_mfma_f32_16x16x128_f8f6f4 a[232:235], v[184:191], v[240:247], a[232:235]\n\t" \
  "v_mfma_f32_16x16x128_f8f6f4 a[228:231], v[184:191], v[232:239], a[228:231]\n\t" \
  "buffer_load_dwordx4 v0, s[36:39], s53 offen offset:3072 sc0 sc1 lds\n\t" \
  "v_mfma_f32_16x16x128_f8f6f4 a[224:227], v[184:191], v[224:231], a[224:227]\n\t" \
  "s_waitcnt lgkmcnt(0)\n\t" \
  "v_mfma_f32_16x16x128_f8f6f4 a[0:3], v[128:135], v[192:199], a[0:3]\n\t" \
  "s_waitcnt vmcnt(24)\n\t" \
  "s_barrier\n\t" \
  "v_mfma_f32_16x16x128_f8f6f4 a[4:7], v[128:135], v[200:207], a[4:7]\n\t" \
  "ds_read_b128 v[160:163], v1 offset:49152\n\t" \
  "ds_read_b128 v[164:167], v1 offset:57344\n\t" \
  "v_mfma_f32_16x16x128_f8f6f4 a[8:11], v[128:135], v[208:215], a[8:11]\n\t" \
  "ds_read_b128 v[168:171], v1 offset:50176\n\t" \
  "ds_read_b128 v[172:175], v1 offset:58368\n\t" \
  "v_mfma_f32_16x16x128_f8f6f4 a[12:15], v[128:135], v[216:223], a[12:15]\n\t" \
  "ds_read_b128 v[176:179], v1 offset:51200\n\t" \
  "ds_read_b128 v[180:183], v1 offset:59392\n\t" \
  "v_mfma_f32_16x16x128_f8f6f4 a[44:47], v[136:143], v[216:223], a[44:47]\n\t" \
  "ds_read_b128 v[184:187], v1 offset:52224\n\t" \
  "ds_read_b128 v[188:191], v1 offset:60416\n\t" \
  "v_mfma_f32_16x16x128_f8f6f4 a[40:43], v[136:143], v[208:215], a[40:43]\n\t" \
  "ds_read_b128 v[224:227], v3 offset:16384\n\t" \
  "ds_read_b128 v[228:231], v3 offset:24576\n\t" \
  "v_mfma_f32_16x16x128_f8f6f4 a[36:39], v[136:143], v[200:207], a[36:39]\n\t" \
  "ds_read_b128 v[232:235], v3 offset:17408\n\t" \
  "ds_read_b128 v[236:239], v3 offset:25600\n\t" \
  "s_add_u32 s53, s52, 294912\n\t" \
  "s_add_u32 m0, s48, 114688\n\t" \
  "v_mfma_f32_16x16x128_f8f6f4 a[32:35], v[136:143], v[192:199], a[32:35]\n\t" \
  "buffer_load_dwordx4 v0, s[36:39], s53 offen sc0 sc1 lds\n\t" \
  "v_mfma_f32_16x16x128_f8f6f4 a[64:67], v[144:151], v[192:199], a[64:67]\n\t" \
  "buffer_load_dwordx4 v0, s[36:39], s53 offen offset:1024 sc0 sc1 lds\n\t" \
  "v_mfma_f32_16x16x128_f8f6f4 a[68:71], v[144:151], v[200:207], a[68:71]\n\t" \
  "buffer_load_dwordx4 v0, s[36:39], s53 offen offset:2048 sc0 sc1 lds\n\t" \
  "v_mfma_f32_16x16x128_f8f6f4 a[72:75], v[144:151], v[208:215], a[72:75]\n\t" \
  "buffer_load_dwordx4 v0, s[36:39], s53 offen offset:3072 sc0 sc1 lds\n\t" \
  "s_add_u32 s53, s52, 311296\n\t" \
  "s_add_u32 m0, s48, 32768\n\t" \
  "v_mfma_f32_16x16x128_f8f6f4 a[76:79], v[144:151], v[216:223], a[76:79]\n\t" \
  "buffer_load_dwordx4 v0, s[36:39], s53 offen sc0 sc1 lds\n\t" \
  "v_mfma_f32_16x16x128_f8f6f4 a[108:111], v[152:159], v[216:223], a[108:111]\n\t" \
  "buffer_load_dwordx4 v0, s[36:39], s53 offen offset:1024 sc0 sc1 lds\n\t" \
  "v_mfma_f32_16x16x128_f8f6f4 a[104:107], v[152:159], v[208:215], a[104:107]\n\t" \
  "buffer_load_dwordx4 v0, s[36:39], s53 offen offset:2048 sc0 sc1 lds\n\t" \
  "v_mfma_f32_16x16x128_f8f6f4 a[100:103], v[152:159], v[200:207], a[100:103]\n\t" \
  "buffer_load_dwordx4 v0, s[36:39], s53 offen offset:3072 sc0 sc1 lds\n\t" \
  "v_mfma_f32_16x16x128_f8f6f4 a[96:99], v[152:159], v[192:199], a[96:99]\n\t" \
  "s_waitcnt lgkmcnt(0)\n\t" \
  "v_mfma_f32_16x16x128_f8f6f4 a[128:131], v[160:167], v[192:199], a[128:131]\n\t" \
  "s_waitcnt vmcnt(28)\n\t" \
  "s_barrier\n\t" \
  "v_mfma_f32_16x16x128_f8f6f4 a[132:135], v[160:167], v[200:207], a[132:135]\n\t" \
  "ds_read_b128 v[128:131], v2 offset:0\n\t" \
  "ds_read_b128 v[132:135], v2 offset:8192\n\t" \
  "v_mfma_f32_16x16x128_f8f6f4 a[136:139], v[160:167], v[208:215], a[136:139]\n\t" \
  "ds_read_b128 v[136:139], v2 offset:1024\n\t" \
  "ds_read_b128 v[140:143], v2 offset:9216\n\t" \
  "v_mfma_f32_16x16x128_f8f6f4 a[140:143], v[160:167], v[216:223], a[140:143]\n\t" \
  "ds_read_b128 v[144:147], v2 offset:2048\n\t" \
  "ds_read_b128 v[148:151], v2 offset:10240\n\t" \
  "v_mfma_f32_16x16x128_f8f6f4 a[172:175], v[168:175], v[216:223], a[172:175]\n\t" \
  "ds_read_b128 v[152:155], v2 offset:3072\n\t" \
  "ds_read_b128 v[156:159], v2 offset:11264\n\t" \
  "v_mfma_f32_16x16x128_f8f6f4 a[168:171], v[168:175], v[208:215], a[168:171]\n\t" \
  "ds_read_b128 v[240:243], v3 offset:18432\n\t" \
  "ds_read_b128 v[244:247], v3 offset:26624\n\t" \
  "v_mfma_f32_16x16x128_f8f6f4 a[164:167], v[168:175], v[200:207], a[164:167]\n\t" \
  "ds_read_b128 v[248:251], v3 offset:19456\n\t" \
  "ds_read_b128 v[252:255], v3 offset:27648\n\t" \
  "v_mfma_f32_16x16x128_f8f6f4 a[160:163], v[168:175], v[192:199], a[160:163]\n\t" \
  "s_add_u32 s53, s50, 163840\n\t" \
  "s_add_u32 m0, s48, 49152\n\t" \
  "v_mfma_f32_16x16x128_f8f6f4 a[192:195], v[176:183], v[192:199], a[192:195]\n\t" \
  "buffer_load_dwordx4 v0, s[36:39], s53 offen sc0 sc1 lds\n\t" \
  "v_mfma_f32_16x16x128_f8f6f4 a[196:199], v[176:183], v[200:207], a[196:199]\n\t" \
  "v_mfma_f32_16x16x128_f8f6f4 a[200:203], v[176:183], v[208:215], a[200:203]\n\t" \
  "buffer_load_dwordx4 v0, s[36:39], s53 offen offset:1024 sc0 sc1 lds\n\t" \
  "v_mfma_f32_16x16x128_f8f6f4 a[204:207], v[176:183], v[216:223], a[204:207]\n\t" \
  "v_mfma_f32_16x16x128_f8f6f4 a[236:239], v[184:191], v[216:223], a[236:239]\n\t" \
  "buffer_load_dwordx4 v0, s[36:39], s53 offen offset:2048 sc0 sc1 lds\n\t" \
  "v_mfma_f32_16x16x128_f8f6f4 a[232:235], v[184:191], v[208:215], a[232:235]\n\t" \
  "v_mfma_f32_16x16x128_f8f6f4 a[228:231], v[184:191], v[200:207], a[228:231]\n\t" \
  "buffer_load_dwordx4 v0, s[36:39], s53 offen offset:3072 sc0 sc1 lds\n\t" \
  "v_mfma_f32_16x16x128_f8f6f4 a[224:227], v[184:191], v[192:199], a[224:227]\n\t" \
  "s_waitcnt lgkmcnt(0)\n\t" \
  "v_mfma_f32_16x16x128_f8f6f4 a[0:3], v[128:135], v[224:231], a[0:3]\n\t" \
  "s_waitcnt vmcnt(24)\n\t" \
  "s_barrier\n\t" \
  "v_mfma_f32_16x16x128_f8f6f4 a[4:7], v[128:135], v[232:239], a[4:7]\n\t" \
  "ds_read_b128 v[160:163], v2 offset:16384\n\t" \
  "ds_read_b128 v[164:167], v2 offset:24576\n\t" \
  "v_mfma_f32_16x16x128_f8f6f4 a[8:11], v[128:135], v[240:247], a[8:11]\n\t" \
  "ds_read_b128 v[168:171], v2 offset:17408\n\t" \
  "ds_read_b128 v[172:175], v2 offset:25600\n\t" \
  "v_mfma_f32_16x16x128_f8f6f4 a[12:15], v[128:135], v[248:255], a[12:15]\n\t" \
  "ds_read_b128 v[176:179], v2 offset:18432\n\t" \
  "ds_read_b128 v[180:183], v2 offset:26624\n\t" \
  "v_mfma_f32_16x16x128_f8f6f4 a[44:47], v[136:143], v[248:255], a[44:47]\n\t" \
  "ds_read_b128 v[184:187], v2 offset:19456\n\t" \
  "ds_read_b128 v[188:191], v2 offset:27648\n\t" \
  "v_mfma_f32_16x16x128_f8f6f4 a[40:43], v[136:143], v[240:247], a[40:43]\n\t" \
  "ds_read_b128 v[192:195], v4 offset:32768\n\t" \
  "ds_read_b128 v[196:199], v4 offset:40960\n\t" \
  "v_mfma_f32_16x16x128_f8f6f4 a[36:39], v[136:143], v[232:239], a[36:39]\n\t" \
  "ds_read_b128 v[200:203], v4 offset:33792\n\t" \
  "ds_read_b128 v[204:207], v4 offset:41984\n\t" \
  "s_add_u32 s53, s52, 327680\n\t" \
  "s_add_u32 m0, s48, 16384\n\t" \
  "v_mfma_f32_16x16x128_f8f6f4 a[32:35], v[136:143], v[224:231], a[32:35]\n\t" \
  "buffer_load_dwordx4 v0, s[36:39], s53 offen sc0 sc1 lds\n\t" \
  "v_mfma_f32_16x16x128_f8f6f4 a[64:67], v[144:151], v[224:231], a[64:67]\n\t" \
  "buffer_load_dwordx4 v0, s[36:39], s53 offen offset:1024 sc0 sc1 lds\n\t" \
  "v_mfma_f32_16x16x128_f8f6f4 a[68:71], v[144:151], v[232:239], a[68:71]\n\t" \
  "buffer_load_dwordx4 v0, s[36:39], s53 offen offset:2048 sc0 sc1 lds\n\t" \
  "v_mfma_f32_16x16x128_f8f6f4 a[72:75], v[144:151], v[240:247], a[72:75]\n\t" \
  "buffer_load_dwordx4 v0, s[36:39], s53 offen offset:3072 sc0 sc1 lds\n\t" \
  "s_add_u32 s53, s52, 344064\n\t" \
  "s_add_u32 m0, s48, 65536\n\t" \
  "v_mfma_f32_16x16x128_f8f6f4 a[76:79], v[144:151], v[248:255], a[76:79]\n\t" \
  "buffer_load_dwordx4 v0, s[36:39], s53 offen sc0 sc1 lds\n\t" \
  "v_mfma_f32_16x16x128_f8f6f4 a[108:111], v[152:159], v[248:255], a[108:111]\n\t" \
  "buffer_load_dwordx4 v0, s[36:39], s53 offen offset:1024 sc0 sc1 lds\n\t" \
  "v_mfma_f32_16x16x128_f8f6f4 a[104:107], v[152:159], v[240:247], a[104:107]\n\t" \
  "buffer_load_dwordx4 v0, s[36:39], s53 offen offset:2048 sc0 sc1 lds\n\t" \
  "v_mfma_f32_16x16x128_f8f6f4 a[100:103], v[152:159], v[232:239], a[100:103]\n\t" \
  "buffer_load_dwordx4 v0, s[36:39], s53 offen offset:3072 sc0 sc1 lds\n\t" \
  "v_mfma_f32_16x16x128_f8f6f4 a[96:99], v[152:159], v[224:231], a[96:99]\n\t" \
  "buffer_load_dword v7, v8, s[44:47], s76 offen sc0 sc1\n\t" \
  "s_waitcnt lgkmcnt(0)\n\t" \
  "v_mfma_f32_16x16x128_f8f6f4 a[128:131], v[160:167], v[224:231], a[128:131]\n\t" \
  "s_waitcnt vmcnt(29)\n\t" \
  "s_barrier\n\t" \
  "v_mfma_f32_16x16x128_f8f6f4 a[132:135], v[160:167], v[232:239], a[132:135]\n\t" \
  "ds_read_b128 v[128:131], v126 offset:0\n\t" \
  "ds_read_b128 v[132:135], v126 offset:8192\n\t" \
  "v_mfma_f32_16x16x128_f8f6f4 a[136:139], v[160:167], v[240:247], a[136:139]\n\t" \
  "ds_read_b128 v[136:139], v126 offset:1024\n\t" \
  "ds_read_b128 v[140:143], v126 offset:9216\n\t" \
  "v_mfma_f32_16x16x128_f8f6f4 a[140:143], v[160:167], v[248:255], a[140:143]\n\t" \
  "ds_read_b128 v[144:147], v126 offset:2048\n\t" \
  "ds_read_b128 v[148:151], v126 offset:10240\n\t" \
  "v_mfma_f32_16x16x128_f8f6f4 a[172:175], v[168:175], v[248:255], a[172:175]\n\t" \
  "ds_read_b128 v[152:155], v126 offset:3072\n\t" \
  "ds_read_b128 v[156:159], v126 offset:11264\n\t" \
  "v_mfma_f32_16x16x128_f8f6f4 a[168:171], v[168:175], v[240:247], a[168:171]\n\t" \
  "ds_read_b128 v[208:211], v4 offset:34816\n\t" \
  "ds_read_b128 v[212:215], v4 offset:43008\n\t" \
  "v_mfma_f32_16x16x128_f8f6f4 a[164:167], v[168:175], v[232:239], a[164:167]\n\t" \
  "ds_read_b128 v[216:219], v4 offset:35840\n\t" \
  "ds_read_b128 v[220:223], v4 offset:44032\n\t" \
  "v_mfma_f32_16x16x128_f8f6f4 a[160:163], v[168:175], v[224:231], a[160:163]\n\t" \
  "s_add_u32 s53, s50, 180224\n\t" \
  "s_add_u32 m0, s48, 81920\n\t" \
  "v_mfma_f32_16x16x128_f8f6f4 a[192:195], v[176:183], v[224:231], a[192:195]\n\t" \
  "buffer_load_dwordx4 v0, s[36:39], s53 offen sc0 sc1 lds\n\t" \
  "v_mfma_f32_16x16x128_f8f6f4 a[196:199], v[176:183], v[232:239], a[196:199]\n\t" \
  "v_mfma_f32_16x16x128_f8f6f4 a[200:203], v[176:183], v[240:247], a[200:203]\n\t" \
  "buffer_load_dwordx4 v0, s[36:39], s53 offen offset:1024 sc0 sc1 lds\n\t" \
  "v_mfma_f32_16x16x128_f8f6f4 a[204:207], v[176:183], v[248:255], a[204:207]\n\t" \
  "v_mfma_f32_16x16x128_f8f6f4 a[236:239], v[184:191], v[248:255], a[236:239]\n\t" \
  "buffer_load_dwordx4 v0, s[36:39], s53 offen offset:2048 sc0 sc1 lds\n\t" \
  "v_mfma_f32_16x16x128_f8f6f4 a[232:235], v[184:191], v[240:247], a[232:235]\n\t" \
  "v_mfma_f32_16x16x128_f8f6f4 a[228:231], v[184:191], v[232:239], a[228:231]\n\t" \
  "buffer_load_dwordx4 v0, s[36:39], s53 offen offset:3072 sc0 sc1 lds\n\t" \
  "v_mfma_f32_16x16x128_f8f6f4 a[224:227], v[184:191], v[224:231], a[224:227]\n\t" \
  "s_waitcnt lgkmcnt(0)\n\t" \
  "v_mfma_f32_16x16x128_f8f6f4 a[0:3], v[128:135], v[192:199], a[0:3]\n\t" \
  "s_waitcnt vmcnt(25)\n\t" \
  "s_barrier\n\t" \
  "v_mfma_f32_16x16x128_f8f6f4 a[4:7], v[128:135], v[200:207], a[4:7]\n\t" \
  "ds_read_b128 v[160:163], v126 offset:16384\n\t" \
  "ds_read_b128 v[164:167], v126 offset:24576\n\t" \
  "v_mfma_f32_16x16x128_f8f6f4 a[8:11], v[128:135], v[208:215], a[8:11]\n\t" \
  "ds_read_b128 v[168:171], v126 offset:17408\n\t" \
  "ds_read_b128 v[172:175], v126 offset:25600\n\t" \
  "v_mfma_f32_16x16x128_f8f6f4 a[12:15], v[128:135], v[216:223], a[12:15]\n\t" \
  "ds_read_b128 v[176:179], v126 offset:18432\n\t" \
  "ds_read_b128 v[180:183], v126 offset:26624\n\t" \
  "v_mfma_f32_16x16x128_f8f6f4 a[44:47], v[136:143], v[216:223], a[44:47]\n\t" \
  "ds_read_b128 v[184:187], v126 offset:19456\n\t" \
  "ds_read_b128 v[188:191], v126 offset:27648\n\t" \
  "v_mfma_f32_16x16x128_f8f6f4 a[40:43], v[136:143], v[208:215], a[40:43]\n\t" \
  "ds_read_b128 v[224:227], v3 offset:0\n\t" \
  "ds_read_b128 v[228:231], v3 offset:8192\n\t" \
  "v_mfma_f32_16x16x128_f8f6f4 a[36:39], v[136:143], v[200:207], a[36:39]\n\t" \
  "ds_read_b128 v[232:235], v3 offset:1024\n\t" \
  "ds_read_b128 v[236:239], v3 offset:9216\n\t" \
  "s_add_u32 s53, s52, 360448\n\t" \
  "s_add_u32 m0, s48, 98304\n\t" \
  "v_mfma_f32_16x16x128_f8f6f4 a[32:35], v[136:143], v[192:199], a[32:35]\n\t" \
  "buffer_load_dwordx4 v0, s[36:39], s53 offen sc0 sc1 lds\n\t" \
  "v_mfma_f32_16x16x128_f8f6f4 a[64:67], v[144:151], v[192:199], a[64:67]\n\t" \
  "buffer_load_dwordx4 v0, s[36:39], s53 offen offset:1024 sc0 sc1 lds\n\t" \
  "v_mfma_f32_16x16x128_f8f6f4 a[68:71], v[144:151], v[200:207], a[68:71]\n\t" \
  "buffer_load_dwordx4 v0, s[36:39], s53 offen offset:2048 sc0 sc1 lds\n\t" \
  "v_mfma_f32_16x16x128_f8f6f4 a[72:75], v[144:151], v[208:215], a[72:75]\n\t" \
  "buffer_load_dwordx4 v0, s[36:39], s53 offen offset:3072 sc0 sc1 lds\n\t" \
  "s_add_u32 s53, s52, 376832\n\t" \
  "s_add_u32 m0, s48, 131072\n\t" \
  "v_mfma_f32_16x16x128_f8f6f4 a[76:79], v[144:151], v[216:223], a[76:79]\n\t" \
  "buffer_load_dwordx4 v0, s[36:39], s53 offen sc0 sc1 lds\n\t" \
  "v_mfma_f32_16x16x128_f8f6f4 a[108:111], v[152:159], v[216:223], a[108:111]\n\t" \
  "buffer_load_dwordx4 v0, s[36:39], s53 offen offset:1024 sc0 sc1 lds\n\t" \
  "v_mfma_f32_16x16x128_f8f6f4 a[104:107], v[152:159], v[208:215], a[104:107]\n\t" \
  "buffer_load_dwordx4 v0, s[36:39], s53 offen offset:2048 sc0 sc1 lds\n\t" \
  "v_mfma_f32_16x16x128_f8f6f4 a[100:103], v[152:159], v[200:207], a[100:103]\n\t" \
  "buffer_load_dwordx4 v0, s[36:39], s53 offen offset:3072 sc0 sc1 lds\n\t" \
  "v_mfma_f32_16x16x128_f8f6f4 a[96:99], v[152:159], v[192:199], a[96:99]\n\t" \
  "s_waitcnt vmcnt(12)\n\t" \
  "v_cmp_le_u32_e32 vcc, s75, v7\n\t" \
  "s_nop 1\n\t" \
  "s_cmp_eq_u64 vcc, -1\n\t" \
  "s_cbranch_scc1 L_OKN03_%=\n\t" \
  "s_swappc_b64 s[80:81], s[78:79]\n\t" \
  "L_OKN03_%=:\n\t" \
  "s_add_u32 s76, s76, 1024\n\t" \
  "s_waitcnt lgkmcnt(0)\n\t" \
  "v_mfma_f32_16x16x128_f8f6f4 a[128:131], v[160:167], v[192:199], a[128:131]\n\t" \
  "s_waitcnt vmcnt(29)\n\t" \
  "s_barrier\n\t" \
  "v_mfma_f32_16x16x128_f8f6f4 a[132:135], v[160:167], v[200:207], a[132:135]\n\t" \
  "ds_read_b128 v[128:131], v2 offset:49152\n\t" \
  "ds_read_b128 v[132:135], v2 offset:57344\n\t" \
  "v_mfma_f32_16x16x128_f8f6f4 a[136:139], v[160:167], v[208:215], a[136:139]\n\t" \
  "ds_read_b128 v[136:139], v2 offset:50176\n\t" \
  "ds_read_b128 v[140:143], v2 offset:58368\n\t" \
  "v_mfma_f32_16x16x128_f8f6f4 a[140:143], v[160:167], v[216:223], a[140:143]\n\t" \
  "ds_read_b128 v[144:147], v2 offset:51200\n\t" \
  "ds_read_b128 v[148:151], v2 offset:59392\n\t" \
  "v_mfma_f32_16x16x128_f8f6f4 a[172:175], v[168:175], v[216:223], a[172:175]\n\t" \
  "ds_read_b128 v[152:155], v2 offset:52224\n\t" \
  "ds_read_b128 v[156:159], v2 offset:60416\n\t" \
  "v_mfma_f32_16x16x128_f8f6f4 a[168:171], v[168:175], v[208:215], a[168:171]\n\t" \
  "ds_read_b128 v[240:243], v3 offset:2048\n\t" \
  "ds_read_b128 v[244:247], v3 offset:10240\n\t" \
  "v_mfma_f32_16x16x128_f8f6f4 a[164:167], v[168:175], v[200:207], a[164:167]\n\t" \
  "ds_read_b128 v[248:251], v3 offset:3072\n\t" \
  "ds_read_b128 v[252:255], v3 offset:11264\n\t" \
  "v_mfma_f32_16x16x128_f8f6f4 a[160:163], v[168:175], v[192:199], a[160:163]\n\t" \
  "s_add_u32 s53, s50, 196608\n\t" \
  "s_add_u32 m0, s48, 147456\n\t" \
  "v_mfma_f32_16x16x128_f8f6f4 a[192:195], v[176:183], v[192:199], a[192:195]\n\t" \
  "buffer_load_dwordx4 v0, s[36:39], s53 offen sc0 sc1 lds\n\t" \
  "v_mfma_f32_16x16x128_f8f6f4 a[196:199], v[176:183], v[200:207], a[196:199]\n\t" \
  "v_mfma_f32_16x16x128_f8f6f4 a[200:203], v[176:183], v[208:215], a[200:203]\n\t" \
  "buffer_load_dwordx4 v0, s[36:39], s53 offen offset:1024 sc0 sc1 lds\n\t" \
  "v_mfma_f32_16x16x128_f8f6f4 a[204:207], v[176:183], v[216:223], a[204:207]\n\t" \
  "v_mfma_f32_16x16x128_f8f6f4 a[236:239], v[184:191], v[216:223], a[236:239]\n\t" \
  "buffer_load_dwordx4 v0, s[36:39], s53 offen offset:2048 sc0 sc1 lds\n\t" \
  "v_mfma_f32_16x16x128_f8f6f4 a[232:235], v[184:191], v[208:215], a[232:235]\n\t" \
  "v_mfma_f32_16x16x128_f8f6f4 a[228:231], v[184:191], v[200:207], a[228:231]\n\t" \
  "buffer_load_dwordx4 v0, s[36:39], s53 offen offset:3072 sc0 sc1 lds\n\t" \
  "v_mfma_f32_16x16x128_f8f6f4 a[224:227], v[184:191], v[192:199], a[224:227]\n\t" \
  "s_waitcnt lgkmcnt(0)\n\t" \
  "v_mfma_f32_16x16x128_f8f6f4 a[0:3], v[128:135], v[224:231], a[0:3]\n\t" \
  "s_waitcnt vmcnt(25)\n\t" \
  "s_barrier\n\t" \
  "v_mfma_f32_16x16x128_f8f6f4 a[4:7], v[128:135], v[232:239], a[4:7]\n\t" \
  "ds_read_b128 v[160:163], v1 offset:32768\n\t" \
  "ds_read_b128 v[164:167], v1 offset:40960\n\t" \
  "v_mfma_f32_16x16x128_f8f6f4 a[8:11], v[128:135], v[240:247], a[8:11]\n\t" \
  "ds_read_b128 v[168:171], v1 offset:33792\n\t" \
  "ds_read_b128 v[172:175], v1 offset:41984\n\t" \
  "v_mfma_f32_16x16x128_f8f6f4 a[12:15], v[128:135], v[248:255], a[12:15]\n\t" \
  "ds_read_b128 v[176:179], v1 offset:34816\n\t" \
  "ds_read_b128 v[180:183], v1 offset:43008\n\t" \
  "v_mfma_f32_16x16x128_f8f6f4 a[44:47], v[136:143], v[248:255], a[44:47]\n\t" \
  "ds_read_b128 v[184:187], v1 offset:35840\n\t" \
  "ds_read_b128 v[188:191], v1 offset:44032\n\t" \
  "v_mfma_f32_16x16x128_f8f6f4 a[40:43], v[136:143], v[240:247], a[40:43]\n\t" \
  "ds_read_b128 v[192:195], v3 offset:49152\n\t" \
  "ds_read_b128 v[196:199], v3 offset:57344\n\t" \
  "v_mfma_f32_16x16x128_f8f6f4 a[36:39], v[136:143], v[232:239], a[36:39]\n\t" \
  "ds_read_b128 v[200:203], v3 offset:50176\n\t" \
  "ds_read_b128 v[204:207], v3 offset:58368\n\t" \
  "s_add_u32 s53, s52, 393216\n\t" \
  "s_add_u32 m0, s48, 114688\n\t" \
  "v_mfma_f32_16x16x128_f8f6f4 a[32:35], v[136:143], v[224:231], a[32:35]\n\t" \
  "buffer_load_dwordx4 v0, s[36:39], s53 offen sc0 sc1 lds\n\t" \
  "v_mfma_f32_16x16x128_f8f6f4 a[64:67], v[144:151], v[224:231], a[64:67]\n\t" \
  "buffer_load_dwordx4 v0, s[36:39], s53 offen offset:1024 sc0 sc1 lds\n\t" \
  "v_mfma_f32_16x16x128_f8f6f4 a[68:71], v[144:151], v[232:239], a[68:71]\n\t" \
  "buffer_load_dwordx4 v0, s[36:39], s53 offen offset:2048 sc0 sc1 lds\n\t" \
  "v_mfma_f32_16x16x128_f8f6f4 a[72:75], v[144:151], v[240:247], a[72:75]\n\t" \
  "buffer_load_dwordx4 v0, s[36:39], s53 offen offset:3072 sc0 sc1 lds\n\t" \
  "s_add_u32 s53, s52, 409600\n\t" \
  "s_mov_b32 m0, s48\n\t" \
  "v_mfma_f32_16x16x128_f8f6f4 a[76:79], v[144:151], v[248:255], a[76:79]\n\t" \
  "buffer_load_dwordx4 v0, s[36:39], s53 offen sc0 sc1 lds\n\t" \
  "v_mfma_f32_16x16x128_f8f6f4 a[108:111], v[152:159], v[248:255], a[108:111]\n\t" \
  "buffer_load_dwordx4 v0, s[36:39], s53 offen offset:1024 sc0 sc1 lds\n\t" \
  "v_mfma_f32_16x16x128_f8f6f4 a[104:107], v[152:159], v[240:247], a[104:107]\n\t" \
  "buffer_load_dwordx4 v0, s[36:39], s53 offen offset:2048 sc0 sc1 lds\n\t" \
  "v_mfma_f32_16x16x128_f8f6f4 a[100:103], v[152:159], v[232:239], a[100:103]\n\t" \
  "buffer_load_dwordx4 v0, s[36:39], s53 offen offset:3072 sc0 sc1 lds\n\t" \
  "v_mfma_f32_16x16x128_f8f6f4 a[96:99], v[152:159], v[224:231], a[96:99]\n\t" \
  "s_waitcnt lgkmcnt(0)\n\t" \
  "v_mfma_f32_16x16x128_f8f6f4 a[128:131], v[160:167], v[224:231], a[128:131]\n\t" \
  "s_waitcnt vmcnt(29)\n\t" \
  "s_barrier\n\t" \
  "v_mfma_f32_16x16x128_f8f6f4 a[132:135], v[160:167], v[232:239], a[132:135]\n\t" \
  "ds_read_b128 v[128:131], v1 offset:16384\n\t" \
  "ds_read_b128 v[132:135], v1 offset:24576\n\t" \
  "v_mfma_f32_16x16x128_f8f6f4 a[136:139], v[160:167], v[240:247], a[136:139]\n\t" \
  "ds_read_b128 v[136:139], v1 offset:17408\n\t" \
  "ds_read_b128 v[140:143], v1 offset:25600\n\t" \
  "v_mfma_f32_16x16x128_f8f6f4 a[140:143], v[160:167], v[248:255], a[140:143]\n\t" \
  "ds_read_b128 v[144:147], v1 offset:18432\n\t" \
  "ds_read_b128 v[148:151], v1 offset:26624\n\t" \
  "v_mfma_f32_16x16x128_f8f6f4 a[172:175], v[168:175], v[248:255], a[172:175]\n\t" \
  "ds_read_b128 v[152:155], v1 offset:19456\n\t" \
  "ds_read_b128 v[156:159], v1 offset:27648\n\t" \
  "v_mfma_f32_16x16x128_f8f6f4 a[168:171], v[168:175], v[240:247], a[168:171]\n\t" \
  "ds_read_b128 v[208:211], v3 offset:51200\n\t" \
  "ds_read_b128 v[212:215], v3 offset:59392\n\t" \
  "v_mfma_f32_16x16x128_f8f6f4 a[164:167], v[168:175], v[232:239], a[164:167]\n\t" \
  "ds_read_b128 v[216:219], v3 offset:52224\n\t" \
  "ds_read_b128 v[220:223], v3 offset:60416\n\t" \
  "v_mfma_f32_16x16x128_f8f6f4 a[160:163], v[168:175], v[224:231], a[160:163]\n\t" \
  "s_add_u32 s53, s50, 212992\n\t" \
  "s_add_u32 m0, s48, 32768\n\t" \
  "v_mfma_f32_16x16x128_f8f6f4 a[192:195], v[176:183], v[224:231], a[192:195]\n\t" \
  "buffer_load_dwordx4 v0, s[36:39], s53 offen sc0 sc1 lds\n\t" \
  "v_mfma_f32_16x16x128_f8f6f4 a[196:199], v[176:183], v[232:239], a[196:199]\n\t" \
  "v_mfma_f32_16x16x128_f8f6f4 a[200:203], v[176:183], v[240:247], a[200:203]\n\t" \
  "buffer_load_dwordx4 v0, s[36:39], s53 offen offset:1024 sc0 sc1 lds\n\t" \
  "v_mfma_f32_16x16x128_f8f6f4 a[204:207], v[176:183], v[248:255], a[204:207]\n\t" \
  "v_mfma_f32_16x16x128_f8f6f4 a[236:239], v[184:191], v[248:255], a[236:239]\n\t" \
  "buffer_load_dwordx4 v0, s[36:39], s53 offen offset:2048 sc0 sc1 lds\n\t" \
  "v_mfma_f32_16x16x128_f8f6f4 a[232:235], v[184:191], v[240:247], a[232:235]\n\t" \
  "v_mfma_f32_16x16x128_f8f6f4 a[228:231], v[184:191], v[232:239], a[228:231]\n\t" \
  "buffer_load_dwordx4 v0, s[36:39], s53 offen offset:3072 sc0 sc1 lds\n\t" \
  "v_mfma_f32_16x16x128_f8f6f4 a[224:227], v[184:191], v[224:231], a[224:227]\n\t" \
  "s_waitcnt lgkmcnt(0)\n\t" \
  "v_mfma_f32_16x16x128_f8f6f4 a[0:3], v[128:135], v[192:199], a[0:3]\n\t" \
  "s_waitcnt vmcnt(24)\n\t" \
  "s_barrier\n\t" \
  "v_mfma_f32_16x16x128_f8f6f4 a[4:7], v[128:135], v[200:207], a[4:7]\n\t" \
  "ds_read_b128 v[160:163], v2 offset:0\n\t" \
  "ds_read_b128 v[164:167], v2 offset:8192\n\t" \
  "v_mfma_f32_16x16x128_f8f6f4 a[8:11], v[128:135], v[208:215], a[8:11]\n\t" \
  "ds_read_b128 v[168:171], v2 offset:1024\n\t" \
  "ds_read_b128 v[172:175], v2 offset:9216\n\t" \
  "v_mfma_f32_16x16x128_f8f6f4 a[12:15], v[128:135], v[216:223], a[12:15]\n\t" \
  "ds_read_b128 v[176:179], v2 offset:2048\n\t" \
  "ds_read_b128 v[180:183], v2 offset:10240\n\t" \
  "v_mfma_f32_16x16x128_f8f6f4 a[44:47], v[136:143], v[216:223], a[44:47]\n\t" \
  "ds_read_b128 v[184:187], v2 offset:3072\n\t" \
  "ds_read_b128 v[188:191], v2 offset:11264\n\t" \
  "v_mfma_f32_16x16x128_f8f6f4 a[40:43], v[136:143], v[208:215], a[40:43]\n\t" \
  "ds_read_b128 v[224:227], v4 offset:16384\n\t" \
  "ds_read_b128 v[228:231], v4 offset:24576\n\t" \
  "v_mfma_f32_16x16x128_f8f6f4 a[36:39], v[136:143], v[200:207], a[36:39]\n\t" \
  "ds_read_b128 v[232:235], v4 offset:17408\n\t" \
  "ds_read_b128 v[236:239], v4 offset:25600\n\t" \
  "s_add_u32 s53, s52, 425984\n\t" \
  "s_add_u32 m0, s48, 16384\n\t" \
  "v_mfma_f32_16x16x128_f8f6f4 a[32:35], v[136:143], v[192:199], a[32:35]\n\t" \
  "buffer_load_dwordx4 v0, s[36:39], s53 offen sc0 sc1 lds\n\t" \
  "v_mfma_f32_16x16x128_f8f6f4 a[64:67], v[144:151], v[192:199], a[64:67]\n\t" \
  "buffer_load_dwordx4 v0, s[36:39], s53 offen offset:1024 sc0 sc1 lds\n\t" \
  "v_mfma_f32_16x16x128_f8f6f4 a[68:71], v[144:151], v[200:207], a[68:71]\n\t" \
  "buffer_load_dwordx4 v0, s[36:39], s53 offen offset:2048 sc0 sc1 lds\n\t" \
  "v_mfma_f32_16x16x128_f8f6f4 a[72:75], v[144:151], v[208:215], a[72:75]\n\t" \
  "buffer_load_dwordx4 v0, s[36:39], s53 offen offset:3072 sc0 sc1 lds\n\t" \
  "s_add_u32 s53, s52, 442368\n\t" \
  "s_add_u32 m0, s48, 49152\n\t" \
  "v_mfma_f32_16x16x128_f8f6f4 a[76:79], v[144:151], v[216:223], a[76:79]\n\t" \
  "buffer_load_dwordx4 v0, s[36:39], s53 offen sc0 sc1 lds\n\t" \
  "v_mfma_f32_16x16x128_f8f6f4 a[108:111], v[152:159], v[216:223], a[108:111]\n\t" \
  "buffer_load_dwordx4 v0, s[36:39], s53 offen offset:1024 sc0 sc1 lds\n\t" \
  "v_mfma_f32_16x16x128_f8f6f4 a[104:107], v[152:159], v[208:215], a[104:107]\n\t" \
  "buffer_load_dwordx4 v0, s[36:39], s53 offen offset:2048 sc0 sc1 lds\n\t" \
  "v_mfma_f32_16x16x128_f8f6f4 a[100:103], v[152:159], v[200:207], a[100:103]\n\t" \
  "buffer_load_dwordx4 v0, s[36:39], s53 offen offset:3072 sc0 sc1 lds\n\t" \
  "v_mfma_f32_16x16x128_f8f6f4 a[96:99], v[152:159], v[192:199], a[96:99]\n\t" \
  "s_waitcnt lgkmcnt(0)\n\t" \
  "v_mfma_f32_16x16x128_f8f6f4 a[128:131], v[160:167], v[192:199], a[128:131]\n\t" \
  "s_waitcnt vmcnt(28)\n\t" \
  "s_barrier\n\t" \
  "v_mfma_f32_16x16x128_f8f6f4 a[132:135], v[160:167], v[200:207], a[132:135]\n\t" \
  "ds_read_b128 v[128:131], v2 offset:32768\n\t" \
  "ds_read_b128 v[132:135], v2 offset:40960\n\t" \
  "v_mfma_f32_16x16x128_f8f6f4 a[136:139], v[160:167], v[208:215], a[136:139]\n\t" \
  "ds_read_b128 v[136:139], v2 offset:33792\n\t" \
  "ds_read_b128 v[140:143], v2 offset:41984\n\t" \
  "v_mfma_f32_16x16x128_f8f6f4 a[140:143], v[160:167], v[216:223], a[140:143]\n\t" \
  "ds_read_b128 v[144:147], v2 offset:34816\n\t" \
  "ds_read_b128 v[148:151], v2 offset:43008\n\t" \
  "v_mfma_f32_16x16x128_f8f6f4 a[172:175], v[168:175], v[216:223], a[172:175]\n\t" \
  "ds_read_b128 v[152:155], v2 offset:35840\n\t" \
  "ds_read_b128 v[156:159], v2 offset:44032\n\t" \
  "v_mfma_f32_16x16x128_f8f6f4 a[168:171], v[168:175], v[208:215], a[168:171]\n\t" \
  "ds_read_b128 v[240:243], v4 offset:18432\n\t" \
  "ds_read_b128 v[244:247], v4 offset:26624\n\t" \
  "v_mfma_f32_16x16x128_f8f6f4 a[164:167], v[168:175], v[200:207], a[164:167]\n\t" \
  "ds_read_b128 v[248:251], v4 offset:19456\n\t" \
  "ds_read_b128 v[252:255], v4 offset:27648\n\t" \
  "v_mfma_f32_16x16x128_f8f6f4 a[160:163], v[168:175], v[192:199], a[160:163]\n\t" \
  "s_add_u32 s53, s50, 229376\n\t" \
  "s_add_u32 m0, s48, 65536\n\t" \
  "v_mfma_f32_16x16x128_f8f6f4 a[192:195], v[176:183], v[192:199], a[192:195]\n\t" \
  "buffer_load_dwordx4 v0, s[36:39], s53 offen sc0 sc1 lds\n\t" \
  "v_mfma_f32_16x16x128_f8f6f4 a[196:199], v[176:183], v[200:207], a[196:199]\n\t" \
  "v_mfma_f32_16x16x128_f8f6f4 a[200:203], v[176:183], v[208:215], a[200:203]\n\t" \
  "buffer_load_dwordx4 v0, s[36:39], s53 offen offset:1024 sc0 sc1 lds\n\t" \
  "v_mfma_f32_16x16x128_f8f6f4 a[204:207], v[176:183], v[216:223], a[204:207]\n\t" \
  "v_mfma_f32_16x16x128_f8f6f4 a[236:239], v[184:191], v[216:223], a[236:239]\n\t" \
  "buffer_load_dwordx4 v0, s[36:39], s53 offen offset:2048 sc0 sc1 lds\n\t" \
  "v_mfma_f32_16x16x128_f8f6f4 a[232:235], v[184:191], v[208:215], a[232:235]\n\t" \
  "v_mfma_f32_16x16x128_f8f6f4 a[228:231], v[184:191], v[200:207], a[228:231]\n\t" \
  "buffer_load_dwordx4 v0, s[36:39], s53 offen offset:3072 sc0 sc1 lds\n\t" \
  "v_mfma_f32_16x16x128_f8f6f4 a[224:227], v[184:191], v[192:199], a[224:227]\n\t" \
  "s_waitcnt lgkmcnt(0)\n\t" \
  "v_mfma_f32_16x16x128_f8f6f4 a[0:3], v[128:135], v[224:231], a[0:3]\n\t" \
  "s_waitcnt vmcnt(24)\n\t" \
  "s_barrier\n\t" \
  "v_mfma_f32_16x16x128_f8f6f4 a[4:7], v[128:135], v[232:239], a[4:7]\n\t" \
  "ds_read_b128 v[160:163], v126 offset:0\n\t" \
  "ds_read_b128 v[164:167], v126 offset:8192\n\t" \
  "v_mfma_f32_16x16x128_f8f6f4 a[8:11], v[128:135], v[240:247], a[8:11]\n\t" \
  "ds_read_b128 v[168:171], v126 offset:1024\n\t" \
  "ds_read_b128 v[172:175], v126 offset:9216\n\t" \
  "v_mfma_f32_16x16x128_f8f6f4 a[12:15], v[128:135], v[248:255], a[12:15]\n\t" \
  "ds_read_b128 v[176:179], v126 offset:2048\n\t" \
  "ds_read_b128 v[180:183], v126 offset:10240\n\t" \
  "v_mfma_f32_16x16x128_f8f6f4 a[44:47], v[136:143], v[248:255], a[44:47]\n\t" \
  "ds_read_b128 v[184:187], v126 offset:3072\n\t" \
  "ds_read_b128 v[188:191], v126 offset:11264\n\t" \
  "v_mfma_f32_16x16x128_f8f6f4 a[40:43], v[136:143], v[240:247], a[40:43]\n\t" \
  "ds_read_b128 v[192:195], v127 offset:16384\n\t" \
  "ds_read_b128 v[196:199], v127 offset:24576\n\t" \
  "v_mfma_f32_16x16x128_f8f6f4 a[36:39], v[136:143], v[232:239], a[36:39]\n\t" \
  "ds_read_b128 v[200:203], v127 offset:17408\n\t" \
  "ds_read_b128 v[204:207], v127 offset:25600\n\t" \
  "s_add_u32 s53, s52, 458752\n\t" \
  "s_add_u32 m0, s48, 81920\n\t" \
  "v_mfma_f32_16x16x128_f8f6f4 a[32:35], v[136:143], v[224:231], a[32:35]\n\t" \
  "buffer_load_dwordx4 v0, s[36:39], s53 offen sc0 sc1 lds\n\t" \
  "v_mfma_f32_16x16x128_f8f6f4 a[64:67], v[144:151], v[224:231], a[64:67]\n\t" \
  "buffer_load_dwordx4 v0, s[36:39], s53 offen offset:1024 sc0 sc1 lds\n\t" \
  "v_mfma_f32_16x16x128_f8f6f4 a[68:71], v[144:151], v[232:239], a[68:71]\n\t" \
  "buffer_load_dwordx4 v0, s[36:39], s53 offen offset:2048 sc0 sc1 lds\n\t" \
  "v_mfma_f32_16x16x128_f8f6f4 a[72:75], v[144:151], v[240:247], a[72:75]\n\t" \
  "buffer_load_dwordx4 v0, s[36:39], s53 offen offset:3072 sc0 sc1 lds\n\t" \
  "s_add_u32 s53, s52, 475136\n\t" \
  "s_add_u32 m0, s48, 98304\n\t" \
  "v_mfma_f32_16x16x128_f8f6f4 a[76:79], v[144:151], v[248:255], a[76:79]\n\t" \
  "buffer_load_dwordx4 v0, s[36:39], s53 offen sc0 sc1 lds\n\t" \
  "v_mfma_f32_16x16x128_f8f6f4 a[108:111], v[152:159], v[248:255], a[108:111]\n\t" \
  "buffer_load_dwordx4 v0, s[36:39], s53 offen offset:1024 sc0 sc1 lds\n\t" \
  "v_mfma_f32_16x16x128_f8f6f4 a[104:107], v[152:159], v[240:247], a[104:107]\n\t" \
  "buffer_load_dwordx4 v0, s[36:39], s53 offen offset:2048 sc0 sc1 lds\n\t" \
  "v_mfma_f32_16x16x128_f8f6f4 a[100:103], v[152:159], v[232:239], a[100:103]\n\t" \
  "buffer_load_dwordx4 v0, s[36:39], s53 offen offset:3072 sc0 sc1 lds\n\t" \
  "v_and_b32_e32 v8, 0x3ff, v8\n\t" \
  "v_add_u32_e32 v8, s92, v8\n\t" \
  "v_mfma_f32_16x16x128_f8f6f4 a[96:99], v[152:159], v[224:231], a[96:99]\n\t" \
  "buffer_load_dword v7, v8, s[44:47], s76 offen sc0 sc1\n\t" \
  "s_waitcnt lgkmcnt(0)\n\t" \
  "v_mfma_f32_16x16x128_f8f6f4 a[128:131], v[160:167], v[224:231], a[128:131]\n\t" \
  "s_waitcnt vmcnt(29)\n\t" \
  "s_barrier\n\t" \
  "v_mfma_f32_16x16x128_f8f6f4 a[132:135], v[160:167], v[232:239], a[132:135]\n\t" \
  "ds_read_b128 v[128:131], v2 offset:49152\n\t" \
  "ds_read_b128 v[132:135], v2 offset:57344\n\t" \
  "v_mfma_f32_16x16x128_f8f6f4 a[136:139], v[160:167], v[240:247], a[136:139]\n\t" \
  "ds_read_b128 v[136:139], v2 offset:50176\n\t" \
  "ds_read_b128 v[140:143], v2 offset:58368\n\t" \
  "v_mfma_f32_16x16x128_f8f6f4 a[140:143], v[160:167], v[248:255], a[140:143]\n\t" \
  "ds_read_b128 v[144:147], v2 offset:51200\n\t" \
  "ds_read_b128 v[148:151], v2 offset:59392\n\t" \
  "v_mfma_f32_16x16x128_f8f6f4 a[172:175], v[168:175], v[248:255], a[172:175]\n\t" \
  "ds_read_b128 v[152:155], v2 offset:52224\n\t" \
  "ds_read_b128 v[156:159], v2 offset:60416\n\t" \
  "v_mfma_f32_16x16x128_f8f6f4 a[168:171], v[168:175], v[240:247], a[168:171]\n\t" \
  "ds_read_b128 v[208:211], v127 offset:18432\n\t" \
  "ds_read_b128 v[212:215], v127 offset:26624\n\t" \
  "v_mfma_f32_16x16x128_f8f6f4 a[164:167], v[168:175], v[232:239], a[164:167]\n\t" \
  "ds_read_b128 v[216:219], v127 offset:19456\n\t" \
  "ds_read_b128 v[220:223], v127 offset:27648\n\t" \
  "v_mfma_f32_16x16x128_f8f6f4 a[160:163], v[168:175], v[224:231], a[160:163]\n\t" \
  "s_add_u32 s53, s50, 245760\n\t" \
  "s_add_u32 m0, s48, 131072\n\t" \
  "v_mfma_f32_16x16x128_f8f6f4 a[192:195], v[176:183], v[224:231], a[192:195]\n\t" \
  "buffer_load_dwordx4 v0, s[36:39], s53 offen sc0 sc1 lds\n\t" \
  "v_mfma_f32_16x16x128_f8f6f4 a[196:199], v[176:183], v[232:239], a[196:199]\n\t" \
  "v_mfma_f32_16x16x128_f8f6f4 a[200:203], v[176:183], v[240:247], a[200:203]\n\t" \
  "buffer_load_dwordx4 v0, s[36:39], s53 offen offset:1024 sc0 sc1 lds\n\t" \
  "v_mfma_f32_16x16x128_f8f6f4 a[204:207], v[176:183], v[248:255], a[204:207]\n\t" \
  "v_mfma_f32_16x16x128_f8f6f4 a[236:239], v[184:191], v[248:255], a[236:239]\n\t" \
  "buffer_load_dwordx4 v0, s[36:39], s53 offen offset:2048 sc0 sc1 lds\n\t" \
  "v_mfma_f32_16x16x128_f8f6f4 a[232:235], v[184:191], v[240:247], a[232:235]\n\t" \
  "v_mfma_f32_16x16x128_f8f6f4 a[228:231], v[184:191], v[232:239], a[228:231]\n\t" \
  "buffer_load_dwordx4 v0, s[36:39], s53 offen offset:3072 sc0 sc1 lds\n\t" \
  "v_mfma_f32_16x16x128_f8f6f4 a[224:227], v[184:191], v[224:231], a[224:227]\n\t" \
  "s_waitcnt lgkmcnt(0)\n\t" \
  "v_mfma_f32_16x16x128_f8f6f4 a[0:3], v[128:135], v[192:199], a[0:3]\n\t" \
  "s_waitcnt vmcnt(25)\n\t" \
  "s_barrier\n\t" \
  "v_mfma_f32_16x16x128_f8f6f4 a[4:7], v[128:135], v[200:207], a[4:7]\n\t" \
  "ds_read_b128 v[160:163], v1 offset:0\n\t" \
  "ds_read_b128 v[164:167], v1 offset:8192\n\t" \
  "v_mfma_f32_16x16x128_f8f6f4 a[8:11], v[128:135], v[208:215], a[8:11]\n\t" \
  "ds_read_b128 v[168:171], v1 offset:1024\n\t" \
  "ds_read_b128 v[172:175], v1 offset:9216\n\t" \
  "v_mfma_f32_16x16x128_f8f6f4 a[12:15], v[128:135], v[216:223], a[12:15]\n\t" \
  "ds_read_b128 v[176:179], v1 offset:2048\n\t" \
  "ds_read_b128 v[180:183], v1 offset:10240\n\t" \
  "v_mfma_f32_16x16x128_f8f6f4 a[44:47], v[136:143], v[216:223], a[44:47]\n\t" \
  "ds_read_b128 v[184:187], v1 offset:3072\n\t" \
  "ds_read_b128 v[188:191], v1 offset:11264\n\t" \
  "v_mfma_f32_16x16x128_f8f6f4 a[40:43], v[136:143], v[208:215], a[40:43]\n\t" \
  "ds_read_b128 v[224:227], v3 offset:32768\n\t" \
  "ds_read_b128 v[228:231], v3 offset:40960\n\t" \
  "v_mfma_f32_16x16x128_f8f6f4 a[36:39], v[136:143], v[200:207], a[36:39]\n\t" \
  "ds_read_b128 v[232:235], v3 offset:33792\n\t" \
  "ds_read_b128 v[236:239], v3 offset:41984\n\t" \
  "s_add_u32 s53, s52, 491520\n\t" \
  "s_add_u32 m0, s48, 114688\n\t" \
  "v_mfma_f32_16x16x128_f8f6f4 a[32:35], v[136:143], v[192:199], a[32:35]\n\t" \
  "buffer_load_dwordx4 v0, s[36:39], s53 offen sc0 sc1 lds\n\t" \
  "v_mfma_f32_16x16x128_f8f6f4 a[64:67], v[144:151], v[192:199], a[64:67]\n\t" \
  "buffer_load_dwordx4 v0, s[36:39], s53 offen offset:1024 sc0 sc1 lds\n\t" \
  "v_mfma_f32_16x16x128_f8f6f4 a[68:71], v[144:151], v[200:207], a[68:71]\n\t" \
  "buffer_load_dwordx4 v0, s[36:39], s53 offen offset:2048 sc0 sc1 lds\n\t" \
  "v_mfma_f32_16x16x128_f8f6f4 a[72:75], v[144:151], v[208:215], a[72:75]\n\t" \
  "buffer_load_dwordx4 v0, s[36:39], s53 offen offset:3072 sc0 sc1 lds\n\t" \
  "s_add_u32 s53, s52, 507904\n\t" \
  "s_add_u32 m0, s48, 147456\n\t" \
  "v_mfma_f32_16x16x128_f8f6f4 a[76:79], v[144:151], v[216:223], a[76:79]\n\t" \
  "buffer_load_dwordx4 v0, s[36:39], s53 offen sc0 sc1 lds\n\t" \
  "v_mfma_f32_16x16x128_f8f6f4 a[108:111], v[152:159], v[216:223], a[108:111]\n\t" \
  "buffer_load_dwordx4 v0, s[36:39], s53 offen offset:1024 sc0 sc1 lds\n\t" \
  "v_mfma_f32_16x16x128_f8f6f4 a[104:107], v[152:159], v[208:215], a[104:107]\n\t" \
  "buffer_load_dwordx4 v0, s[36:39], s53 offen offset:2048 sc0 sc1 lds\n\t" \
  "v_mfma_f32_16x16x128_f8f6f4 a[100:103], v[152:159], v[200:207], a[100:103]\n\t" \
  "buffer_load_dwordx4 v0, s[36:39], s53 offen offset:3072 sc0 sc1 lds\n\t" \
  "v_mfma_f32_16x16x128_f8f6f4 a[96:99], v[152:159], v[192:199], a[96:99]\n\t" \
  "s_waitcnt vmcnt(12)\n\t" \
  "v_cmp_le_u32_e32 vcc, s75, v7\n\t" \
  "s_nop 1\n\t" \
  "s_cmp_eq_u64 vcc, -1\n\t" \
  "s_cbranch_scc1 L_OKN04_%=\n\t" \
  "s_swappc_b64 s[80:81], s[78:79]\n\t" \
  "L_OKN04_%=:\n\t" \
  "s_add_u32 s76, s76, 1024\n\t" \
  "s_waitcnt lgkmcnt(0)\n\t" \
  "v_mfma_f32_16x16x128_f8f6f4 a[128:131], v[160:167], v[192:199], a[128:131]\n\t" \
  "s_waitcnt vmcnt(29)\n\t" \
  "s_barrier\n\t" \
  "v_mfma_f32_16x16x128_f8f6f4 a[132:135], v[160:167], v[200:207], a[132:135]\n\t" \
  "ds_read_b128 v[128:131], v1 offset:16384\n\t" \
  "ds_read_b128 v[132:135], v1 offset:24576\n\t" \
  "v_mfma_f32_16x16x128_f8f6f4 a[136:139], v[160:167], v[208:215], a[136:139]\n\t" \
  "ds_read_b128 v[136:139], v1 offset:17408\n\t" \
  "ds_read_b128 v[140:143], v1 offset:25600\n\t" \
  "v_mfma_f32_16x16x128_f8f6f4 a[140:143], v[160:167], v[216:223], a[140:143]\n\t" \
  "ds_read_b128 v[144:147], v1 offset:18432\n\t" \
  "ds_read_b128 v[148:151], v1 offset:26624\n\t" \
  "v_mfma_f32_16x16x128_f8f6f4 a[172:175], v[168:175], v[216:223], a[172:175]\n\t" \
  "ds_read_b128 v[152:155], v1 offset:19456\n\t" \
  "ds_read_b128 v[156:159], v1 offset:27648\n\t" \
  "v_mfma_f32_16x16x128_f8f6f4 a[168:171], v[168:175], v[208:215], a[168:171]\n\t" \
  "ds_read_b128 v[240:243], v3 offset:34816\n\t" \
  "ds_read_b128 v[244:247], v3 offset:43008\n\t" \
  "v_mfma_f32_16x16x128_f8f6f4 a[164:167], v[168:175], v[200:207], a[164:167]\n\t" \
  "ds_read_b128 v[248:251], v3 offset:35840\n\t" \
  "ds_read_b128 v[252:255], v3 offset:44032\n\t" \
  "v_mfma_f32_16x16x128_f8f6f4 a[160:163], v[168:175], v[192:199], a[160:163]\n\t" \
  "s_add_u32 s53, s52, 524288\n\t" \
  "s_mov_b32 m0, s48\n\t" \
  "v_mfma_f32_16x16x128_f8f6f4 a[192:195], v[176:183], v[192:199], a[192:195]\n\t" \
  "buffer_load_dwordx4 v0, s[36:39], s53 offen sc0 sc1 lds\n\t" \
  "v_mfma_f32_16x16x128_f8f6f4 a[196:199], v[176:183], v[200:207], a[196:199]\n\t" \
  "v_mfma_f32_16x16x128_f8f6f4 a[200:203], v[176:183], v[208:215], a[200:203]\n\t" \
  "buffer_load_dwordx4 v0, s[36:39], s53 offen offset:1024 sc0 sc1 lds\n\t" \
  "v_mfma_f32_16x16x128_f8f6f4 a[204:207], v[176:183], v[216:223], a[204:207]\n\t" \
  "v_mfma_f32_16x16x128_f8f6f4 a[236:239], v[184:191], v[216:223], a[236:239]\n\t" \
  "buffer_load_dwordx4 v0, s[36:39], s53 offen offset:2048 sc0 sc1 lds\n\t" \
  "v_mfma_f32_16x16x128_f8f6f4 a[232:235], v[184:191], v[208:215], a[232:235]\n\t" \
  "v_mfma_f32_16x16x128_f8f6f4 a[228:231], v[184:191], v[200:207], a[228:231]\n\t" \
  "buffer_load_dwordx4 v0, s[36:39], s53 offen offset:3072 sc0 sc1 lds\n\t" \
  "v_mfma_f32_16x16x128_f8f6f4 a[224:227], v[184:191], v[192:199], a[224:227]\n\t" \
  "s_waitcnt lgkmcnt(0)\n\t" \
  "v_mfma_f32_16x16x128_f8f6f4 a[0:3], v[128:135], v[224:231], a[0:3]\n\t" \
  "s_waitcnt vmcnt(25)\n\t" \
  "s_barrier\n\t" \
  "v_mfma_f32_16x16x128_f8f6f4 a[4:7], v[128:135], v[232:239], a[4:7]\n\t" \
  "ds_read_b128 v[160:163], v1 offset:49152\n\t" \
  "ds_read_b128 v[164:167], v1 offset:57344\n\t" \
  "v_mfma_f32_16x16x128_f8f6f4 a[8:11], v[128:135], v[240:247], a[8:11]\n\t" \
  "ds_read_b128 v[168:171], v1 offset:50176\n\t" \
  "ds_read_b128 v[172:175], v1 offset:58368\n\t" \
  "v_mfma_f32_16x16x128_f8f6f4 a[12:15], v[128:135], v[248:255], a[12:15]\n\t" \
  "ds_read_b128 v[176:179], v1 offset:51200\n\t" \
  "ds_read_b128 v[180:183], v1 offset:59392\n\t" \
  "v_mfma_f32_16x16x128_f8f6f4 a[44:47], v[136:143], v[248:255], a[44:47]\n\t" \
  "ds_read_b128 v[184:187], v1 offset:52224\n\t" \
  "ds_read_b128 v[188:191], v1 offset:60416\n\t" \
  "v_mfma_f32_16x16x128_f8f6f4 a[40:43], v[136:143], v[240:247], a[40:43]\n\t" \
  "ds_read_b128 v[192:195], v4 offset:0\n\t" \
  "ds_read_b128 v[196:199], v4 offset:8192\n\t" \
  "v_mfma_f32_16x16x128_f8f6f4 a[36:39], v[136:143], v[232:239], a[36:39]\n\t" \
  "ds_read_b128 v[200:203], v4 offset:1024\n\t" \
  "ds_read_b128 v[204:207], v4 offset:9216\n\t" \
  "s_add_u32 s53, s50, 262144\n\t" \
  "s_add_u32 m0, s48, 16384\n\t" \
  "v_mfma_f32_16x16x128_f8f6f4 a[32:35], v[136:143], v[224:231], a[32:35]\n\t" \
  "buffer_load_dwordx4 v0, s[36:39], s53 offen sc0 sc1 lds\n\t" \
  "v_mfma_f32_16x16x128_f8f6f4 a[64:67], v[144:151], v[224:231], a[64:67]\n\t" \
  "buffer_load_dwordx4 v0, s[36:39], s53 offen offset:1024 sc0 sc1 lds\n\t" \
  "v_mfma_f32_16x16x128_f8f6f4 a[68:71], v[144:151], v[232:239], a[68:71]\n\t" \
  "buffer_load_dwordx4 v0, s[36:39], s53 offen offset:2048 sc0 sc1 lds\n\t" \
  "v_mfma_f32_16x16x128_f8f6f4 a[72:75], v[144:151], v[240:247], a[72:75]\n\t" \
  "buffer_load_dwordx4 v0, s[36:39], s53 offen offset:3072 sc0 sc1 lds\n\t" \
  "s_mov_b32 s53, s51\n\t" \
  "s_add_u32 m0, s48, 32768\n\t" \
  "v_mfma_f32_16x16x128_f8f6f4 a[76:79], v[144:151], v[248:255], a[76:79]\n\t" \
  "buffer_load_dwordx4 v0, s[36:39], s53 offen sc0 sc1 lds\n\t" \
  "v_mfma_f32_16x16x128_f8f6f4 a[108:111], v[152:159], v[248:255], a[108:111]\n\t" \
  "buffer_load_dwordx4 v0, s[36:39], s53 offen offset:1024 sc0 sc1 lds\n\t" \
  "v_mfma_f32_16x16x128_f8f6f4 a[104:107], v[152:159], v[240:247], a[104:107]\n\t" \
  "buffer_load_dwordx4 v0, s[36:39], s53 offen offset:2048 sc0 sc1 lds\n\t" \
  "v_mfma_f32_16x16x128_f8f6f4 a[100:103], v[152:159], v[232:239], a[100:103]\n\t" \
  "buffer_load_dwordx4 v0, s[36:39], s53 offen offset:3072 sc0 sc1 lds\n\t" \
  "v_mfma_f32_16x16x128_f8f6f4 a[96:99], v[152:159], v[224:231], a[96:99]\n\t" \
  "s_waitcnt lgkmcnt(0)\n\t" \
  "v_mfma_f32_16x16x128_f8f6f4 a[128:131], v[160:167], v[224:231], a[128:131]\n\t" \
  "s_waitcnt vmcnt(29)\n\t" \
  "s_barrier\n\t" \
  "v_mfma_f32_16x16x128_f8f6f4 a[132:135], v[160:167], v[232:239], a[132:135]\n\t" \
  "ds_read_b128 v[128:131], v2 offset:16384\n\t" \
  "ds_read_b128 v[132:135], v2 offset:24576\n\t" \
  "v_mfma_f32_16x16x128_f8f6f4 a[136:139], v[160:167], v[240:247], a[136:139]\n\t" \
  "ds_read_b128 v[136:139], v2 offset:17408\n\t" \
  "ds_read_b128 v[140:143], v2 offset:25600\n\t" \
  "v_mfma_f32_16x16x128_f8f6f4 a[140:143], v[160:167], v[248:255], a[140:143]\n\t" \
  "ds_read_b128 v[144:147], v2 offset:18432\n\t" \
  "ds_read_b128 v[148:151], v2 offset:26624\n\t" \
  "v_mfma_f32_16x16x128_f8f6f4 a[172:175], v[168:175], v[248:255], a[172:175]\n\t" \
  "ds_read_b128 v[152:155], v2 offset:19456\n\t" \
  "ds_read_b128 v[156:159], v2 offset:27648\n\t" \
  "v_mfma_f32_16x16x128_f8f6f4 a[168:171], v[168:175], v[240:247], a[168:171]\n\t" \
  "ds_read_b128 v[208:211], v4 offset:2048\n\t" \
  "ds_read_b128 v[212:215], v4 offset:10240\n\t" \
  "v_mfma_f32_16x16x128_f8f6f4 a[164:167], v[168:175], v[232:239], a[164:167]\n\t" \
  "ds_read_b128 v[216:219], v4 offset:3072\n\t" \
  "ds_read_b128 v[220:223], v4 offset:11264\n\t" \
  "v_mfma_f32_16x16x128_f8f6f4 a[160:163], v[168:175], v[224:231], a[160:163]\n\t" \
  "s_add_u32 s53, s52, 540672\n\t" \
  "s_add_u32 m0, s48, 49152\n\t" \
  "v_mfma_f32_16x16x128_f8f6f4 a[192:195], v[176:183], v[224:231], a[192:195]\n\t" \
  "buffer_load_dwordx4 v0, s[36:39], s53 offen sc0 sc1 lds\n\t" \
  "v_mfma_f32_16x16x128_f8f6f4 a[196:199], v[176:183], v[232:239], a[196:199]\n\t" \
  "v_mfma_f32_16x16x128_f8f6f4 a[200:203], v[176:183], v[240:247], a[200:203]\n\t" \
  "buffer_load_dwordx4 v0, s[36:39], s53 offen offset:1024 sc0 sc1 lds\n\t" \
  "v_mfma_f32_16x16x128_f8f6f4 a[204:207], v[176:183], v[248:255], a[204:207]\n\t" \
  "v_mfma_f32_16x16x128_f8f6f4 a[236:239], v[184:191], v[248:255], a[236:239]\n\t" \
  "buffer_load_dwordx4 v0, s[36:39], s53 offen offset:2048 sc0 sc1 lds\n\t" \
  "v_mfma_f32_16x16x128_f8f6f4 a[232:235], v[184:191], v[240:247], a[232:235]\n\t" \
  "v_mfma_f32_16x16x128_f8f6f4 a[228:231], v[184:191], v[232:239], a[228:231]\n\t" \
  "buffer_load_dwordx4 v0, s[36:39], s53 offen offset:3072 sc0 sc1 lds\n\t" \
  "v_mfma_f32_16x16x128_f8f6f4 a[224:227], v[184:191], v[224:231], a[224:227]\n\t" \
  "s_waitcnt lgkmcnt(0)\n\t" \
  "v_mfma_f32_16x16x128_f8f6f4 a[0:3], v[128:135], v[192:199], a[0:3]\n\t" \
  "s_waitcnt vmcnt(24)\n\t" \
  "s_barrier\n\t" \
  "v_mfma_f32_16x16x128_f8f6f4 a[4:7], v[128:135], v[200:207], a[4:7]\n\t" \
  "ds_read_b128 v[160:163], v2 offset:32768\n\t" \
  "ds_read_b128 v[164:167], v2 offset:40960\n\t" \
  "v_mfma_f32_16x16x128_f8f6f4 a[8:11], v[128:135], v[208:215], a[8:11]\n\t" \
  "ds_read_b128 v[168:171], v2 offset:33792\n\t" \
  "ds_read_b128 v[172:175], v2 offset:41984\n\t" \
  "v_mfma_f32_16x16x128_f8f6f4 a[12:15], v[128:135], v[216:223], a[12:15]\n\t" \
  "ds_read_b128 v[176:179], v2 offset:34816\n\t" \
  "ds_read_b128 v[180:183], v2 offset:43008\n\t" \
  "v_mfma_f32_16x16x128_f8f6f4 a[44:47], v[136:143], v[216:223], a[44:47]\n\t" \
  "ds_read_b128 v[184:187], v2 offset:35840\n\t" \
  "ds_read_b128 v[188:191], v2 offset:44032\n\t" \
  "v_mfma_f32_16x16x128_f8f6f4 a[40:43], v[136:143], v[208:215], a[40:43]\n\t" \
  "ds_read_b128 v[224:227], v127 offset:0\n\t" \
  "ds_read_b128 v[228:231], v127 offset:8192\n\t" \
  "v_mfma_f32_16x16x128_f8f6f4 a[36:39], v[136:143], v[200:207], a[36:39]\n\t" \
  "ds_read_b128 v[232:235], v127 offset:1024\n\t" \
  "ds_read_b128 v[236:239], v127 offset:9216\n\t" \
  "s_add_u32 s53, s52, 557056\n\t" \
  "s_add_u32 m0, s48, 65536\n\t" \
  "v_mfma_f32_16x16x128_f8f6f4 a[32:35], v[136:143], v[192:199], a[32:35]\n\t" \
  "buffer_load_dwordx4 v0, s[36:39], s53 offen sc0 sc1 lds\n\t" \
  "v_mfma_f32_16x16x128_f8f6f4 a[64:67], v[144:151], v[192:199], a[64:67]\n\t" \
  "buffer_load_dwordx4 v0, s[36:39], s53 offen offset:1024 sc0 sc1 lds\n\t" \
  "v_mfma_f32_16x16x128_f8f6f4 a[68:71], v[144:151], v[200:207], a[68:71]\n\t" \
  "buffer_load_dwordx4 v0, s[36:39], s53 offen offset:2048 sc0 sc1 lds\n\t" \
  "v_mfma_f32_16x16x128_f8f6f4 a[72:75], v[144:151], v[208:215], a[72:75]\n\t" \
  "buffer_load_dwordx4 v0, s[36:39], s53 offen offset:3072 sc0 sc1 lds\n\t" \
  "s_add_u32 s53, s51, 16384\n\t" \
  "s_add_u32 m0, s48, 81920\n\t" \
  "v_mfma_f32_16x16x128_f8f6f4 a[76:79], v[144:151], v[216:223], a[76:79]\n\t" \
  "buffer_load_dwordx4 v0, s[36:39], s53 offen sc0 sc1 lds\n\t" \
  "v_mfma_f32_16x16x128_f8f6f4 a[108:111], v[152:159], v[216:223], a[108:111]\n\t" \
  "buffer_load_dwordx4 v0, s[36:39], s53 offen offset:1024 sc0 sc1 lds\n\t" \
  "v_mfma_f32_16x16x128_f8f6f4 a[104:107], v[152:159], v[208:215], a[104:107]\n\t" \
  "buffer_load_dwordx4 v0, s[36:39], s53 offen offset:2048 sc0 sc1 lds\n\t" \
  "v_mfma_f32_16x16x128_f8f6f4 a[100:103], v[152:159], v[200:207], a[100:103]\n\t" \
  "buffer_load_dwordx4 v0, s[36:39], s53 offen offset:3072 sc0 sc1 lds\n\t" \
  "v_mfma_f32_16x16x128_f8f6f4 a[96:99], v[152:159], v[192:199], a[96:99]\n\t" \
  "s_waitcnt lgkmcnt(0)\n\t" \
  "v_mfma_f32_16x16x128_f8f6f4 a[128:131], v[160:167], v[192:199], a[128:131]\n\t" \
  "s_waitcnt vmcnt(28)\n\t" \
  "s_barrier\n\t" \
  "v_mfma_f32_16x16x128_f8f6f4 a[132:135], v[160:167], v[200:207], a[132:135]\n\t" \
  "ds_read_b128 v[128:131], v2 offset:49152\n\t" \
  "ds_read_b128 v[132:135], v2 offset:57344\n\t" \
  "v_mfma_f32_16x16x128_f8f6f4 a[136:139], v[160:167], v[208:215], a[136:139]\n\t" \
  "ds_read_b128 v[136:139], v2 offset:50176\n\t" \
  "ds_read_b128 v[140:143], v2 offset:58368\n\t" \
  "v_mfma_f32_16x16x128_f8f6f4 a[140:143], v[160:167], v[216:223], a[140:143]\n\t" \
  "ds_read_b128 v[144:147], v2 offset:51200\n\t" \
  "ds_read_b128 v[148:151], v2 offset:59392\n\t" \
  "v_mfma_f32_16x16x128_f8f6f4 a[172:175], v[168:175], v[216:223], a[172:175]\n\t" \
  "ds_read_b128 v[152:155], v2 offset:52224\n\t" \
  "ds_read_b128 v[156:159], v2 offset:60416\n\t" \
  "v_mfma_f32_16x16x128_f8f6f4 a[168:171], v[168:175], v[208:215], a[168:171]\n\t" \
  "ds_read_b128 v[240:243], v127 offset:2048\n\t" \
  "ds_read_b128 v[244:247], v127 offset:10240\n\t" \
  "v_mfma_f32_16x16x128_f8f6f4 a[164:167], v[168:175], v[200:207], a[164:167]\n\t" \
  "ds_read_b128 v[248:251], v127 offset:3072\n\t" \
  "ds_read_b128 v[252:255], v127 offset:11264\n\t" \
  "v_mfma_f32_16x16x128_f8f6f4 a[160:163], v[168:175], v[192:199], a[160:163]\n\t" \
  "s_add_u32 s53, s50, 278528\n\t" \
  "s_add_u32 m0, s48, 98304\n\t" \
  "v_mfma_f32_16x16x128_f8f6f4 a[192:195], v[176:183], v[192:199], a[192:195]\n\t" \
  "buffer_load_dwordx4 v0, s[36:39], s53 offen sc0 sc1 lds\n\t" \
  "v_mfma_f32_16x16x128_f8f6f4 a[196:199], v[176:183], v[200:207], a[196:199]\n\t" \
  "v_mfma_f32_16x16x128_f8f6f4 a[200:203], v[176:183], v[208:215], a[200:203]\n\t" \
  "buffer_load_dwordx4 v0, s[36:39], s53 offen offset:1024 sc0 sc1 lds\n\t" \
  "v_mfma_f32_16x16x128_f8f6f4 a[204:207], v[176:183], v[216:223], a[204:207]\n\t" \
  "v_mfma_f32_16x16x128_f8f6f4 a[236:239], v[184:191], v[216:223], a[236:239]\n\t" \
  "buffer_load_dwordx4 v0, s[36:39], s53 offen offset:2048 sc0 sc1 lds\n\t" \
  "v_mfma_f32_16x16x128_f8f6f4 a[232:235], v[184:191], v[208:215], a[232:235]\n\t" \
  "v_mfma_f32_16x16x128_f8f6f4 a[228:231], v[184:191], v[200:207], a[228:231]\n\t" \
  "buffer_load_dwordx4 v0, s[36:39], s53 offen offset:3072 sc0 sc1 lds\n\t" \
  "v_mfma_f32_16x16x128_f8f6f4 a[224:227], v[184:191], v[192:199], a[224:227]\n\t" \
  "s_waitcnt lgkmcnt(0)\n\t" \
  "v_mfma_f32_16x16x128_f8f6f4 a[0:3], v[128:135], v[224:231], a[0:3]\n\t" \
  "s_waitcnt vmcnt(20)\n\t" \
  "s_barrier\n\t" \
  "v_mfma_f32_16x16x128_f8f6f4 a[4:7], v[128:135], v[232:239], a[4:7]\n\t" \
  "ds_read_b128 v[160:163], v126 offset:16384\n\t" \
  "ds_read_b128 v[164:167], v126 offset:24576\n\t" \
  "v_mfma_f32_16x16x128_f8f6f4 a[8:11], v[128:135], v[240:247], a[8:11]\n\t" \
  "ds_read_b128 v[168:171], v126 offset:17408\n\t" \
  "ds_read_b128 v[172:175], v126 offset:25600\n\t" \
  "v_mfma_f32_16x16x128_f8f6f4 a[12:15], v[128:135], v[248:255], a[12:15]\n\t" \
  "ds_read_b128 v[176:179], v126 offset:18432\n\t" \
  "ds_read_b128 v[180:183], v126 offset:26624\n\t" \
  "v_mfma_f32_16x16x128_f8f6f4 a[44:47], v[136:143], v[248:255], a[44:47]\n\t" \
  "ds_read_b128 v[184:187], v126 offset:19456\n\t" \
  "ds_read_b128 v[188:191], v126 offset:27648\n\t" \
  "v_mfma_f32_16x16x128_f8f6f4 a[40:43], v[136:143], v[240:247], a[40:43]\n\t" \
  "ds_read_b128 v[192:195], v3 offset:16384\n\t" \
  "ds_read_b128 v[196:199], v3 offset:24576\n\t" \
  "v_mfma_f32_16x16x128_f8f6f4 a[36:39], v[136:143], v[232:239], a[36:39]\n\t" \
  "ds_read_b128 v[200:203], v3 offset:17408\n\t" \
  "ds_read_b128 v[204:207], v3 offset:25600\n\t" \
  "v_mfma_f32_16x16x128_f8f6f4 a[32:35], v[136:143], v[224:231], a[32:35]\n\t" \
  "s_add_u32 s53, s52, 573440\n\t" \
  "s_add_u32 m0, s48, 114688\n\t" \
  "v_mfma_f32_16x16x128_f8f6f4 a[64:67], v[144:151], v[224:231], a[64:67]\n\t" \
  "buffer_load_dwordx4 v0, s[36:39], s53 offen sc0 sc1 lds\n\t" \
  "v_mfma_f32_16x16x128_f8f6f4 a[68:71], v[144:151], v[232:239], a[68:71]\n\t" \
  "v_mfma_f32_16x16x128_f8f6f4 a[72:75], v[144:151], v[240:247], a[72:75]\n\t" \
  "buffer_load_dwordx4 v0, s[36:39], s53 offen offset:1024 sc0 sc1 lds\n\t" \
  "v_mfma_f32_16x16x128_f8f6f4 a[76:79], v[144:151], v[248:255], a[76:79]\n\t" \
  "v_mfma_f32_16x16x128_f8f6f4 a[108:111], v[152:159], v[248:255], a[108:111]\n\t" \
  "buffer_load_dwordx4 v0, s[36:39], s53 offen offset:2048 sc0 sc1 lds\n\t" \
  "v_mfma_f32_16x16x128_f8f6f4 a[104:107], v[152:159], v[240:247], a[104:107]\n\t" \
  "v_mfma_f32_16x16x128_f8f6f4 a[100:103], v[152:159], v[232:239], a[100:103]\n\t" \
  "buffer_load_dwordx4 v0, s[36:39], s53 offen offset:3072 sc0 sc1 lds\n\t" \
  "v_mfma_f32_16x16x128_f8f6f4 a[96:99], v[152:159], v[224:231], a[96:99]\n\t" \
  "s_waitcnt lgkmcnt(0)\n\t" \
  "v_mfma_f32_16x16x128_f8f6f4 a[128:131], v[160:167], v[224:231], a[128:131]\n\t" \
  "s_waitcnt vmcnt(28)\n\t" \
  "s_barrier\n\t" \
  "v_mfma_f32_16x16x128_f8f6f4 a[132:135], v[160:167], v[232:239], a[132:135]\n\t" \
  "ds_read_b128 v[128:131], v1 offset:0\n\t" \
  "ds_read_b128 v[132:135], v1 offset:8192\n\t" \
  "v_mfma_f32_16x16x128_f8f6f4 a[136:139], v[160:167], v[240:247], a[136:139]\n\t" \
  "ds_read_b128 v[136:139], v1 offset:1024\n\t" \
  "ds_read_b128 v[140:143], v1 offset:9216\n\t" \
  "v_mfma_f32_16x16x128_f8f6f4 a[140:143], v[160:167], v[248:255], a[140:143]\n\t" \
  "ds_read_b128 v[144:147], v1 offset:2048\n\t" \
  "ds_read_b128 v[148:151], v1 offset:10240\n\t" \
  "v_mfma_f32_16x16x128_f8f6f4 a[172:175], v[168:175], v[248:255], a[172:175]\n\t" \
  "ds_read_b128 v[152:155], v1 offset:3072\n\t" \
  "ds_read_b128 v[156:159], v1 offset:11264\n\t" \
  "v_mfma_f32_16x16x128_f8f6f4 a[168:171], v[168:175], v[240:247], a[168:171]\n\t" \
  "ds_read_b128 v[208:211], v3 offset:18432\n\t" \
  "ds_read_b128 v[212:215], v3 offset:26624\n\t" \
  "v_mfma_f32_16x16x128_f8f6f4 a[164:167], v[168:175], v[232:239], a[164:167]\n\t" \
  "ds_read_b128 v[216:219], v3 offset:19456\n\t" \
  "ds_read_b128 v[220:223], v3 offset:27648\n\t" \
  "v_mfma_f32_16x16x128_f8f6f4 a[160:163], v[168:175], v[224:231], a[160:163]\n\t" \
  "v_mfma_f32_16x16x128_f8f6f4 a[192:195], v[176:183], v[224:231], a[192:195]\n\t" \
  "v_mfma_f32_16x16x128_f8f6f4 a[196:199], v[176:183], v[232:239], a[196:199]\n\t" \
  "v_mfma_f32_16x16x128_f8f6f4 a[200:203], v[176:183], v[240:247], a[200:203]\n\t" \
  "v_mfma_f32_16x16x128_f8f6f4 a[204:207], v[176:183], v[248:255], a[204:207]\n\t" \
  "v_mfma_f32_16x16x128_f8f6f4 a[236:239], v[184:191], v[248:255], a[236:239]\n\t" \
  "v_mfma_f32_16x16x128_f8f6f4 a[232:235], v[184:191], v[240:247], a[232:235]\n\t" \
  "v_mfma_f32_16x16x128_f8f6f4 a[228:231], v[184:191], v[232:239], a[228:231]\n\t" \
  "v_mfma_f32_16x16x128_f8f6f4 a[224:227], v[184:191], v[224:231], a[224:227]\n\t" \
  "s_add_u32 s49, s52, 589824\n\t" \
  "s_add_u32 s50, s50, 294912\n\t" \
  "s_add_u32 s51, s51, 32768\n\t" \
  "s_sub_u32 s72, s72, 1\n\t" \
  "s_branch L_WO_%=\n\t" \
  "L_NP_%=:\n\t" \
  "s_mov_b32 s67, s69\n\t" \
  "s_add_u32 s77, s77, 1\n\t" \
  "s_and_b32 s54, s62, 0xff\n\t" \
  "s_lshr_b64 s[62:63], s[62:63], 8\n\t" \
  "s_lshl_b32 s55, s64, 24\n\t" \
  "s_or_b32 s63, s63, s55\n\t" \
  "s_lshr_b64 s[64:65], s[64:65], 8\n\t" \
  "s_and_b32 s54, s54, 0x7f\n\t" \
  "s_lshl_b32 s69, s54, 9\n\t" \
  "s_add_u32 s69, s69, s71\n\t" \
  "s_lshl_b32 s54, s54, 19\n\t" \
  "s_add_u32 s54, s54, s70\n\t" \
  "s_mov_b32 s50, s54\n\t" \
  "s_sub_u32 s91, s51, 262144\n\t" \
  "v_add_u32_e32 v126, 0x20000, v1\n\t" \
  "v_add_u32_e32 v127, 0x20000, v3\n\t" \
  "s_mov_b32 s53, s51\n\t" \
  "s_add_u32 m0, s48, 114688\n\t" \
  "s_nop 0\n\t" \
  "buffer_load_dwordx4 v0, s[36:39], s53 offen sc0 sc1 lds\n\t" \
  "buffer_load_dwordx4 v0, s[36:39], s53 offen offset:1024 sc0 sc1 lds\n\t" \
  "buffer_load_dwordx4 v0, s[36:39], s53 offen offset:2048 sc0 sc1 lds\n\t" \
  "buffer_load_dwordx4 v0, s[36:39], s53 offen offset:3072 sc0 sc1 lds\n\t" \
  "s_add_u32 s53, s52, 524288\n\t" \
  "s_add_u32 m0, s48, 16384\n\t" \
  "s_nop 0\n\t" \
  "buffer_load_dwordx4 v0, s[36:39], s53 offen sc0 sc1 lds\n\t" \
  "buffer_load_dwordx4 v0, s[36:39], s53 offen offset:1024 sc0 sc1 lds\n\t" \
  "buffer_load_dwordx4 v0, s[36:39], s53 offen offset:2048 sc0 sc1 lds\n\t" \
  "buffer_load_dwordx4 v0, s[36:39], s53 offen offset:3072 sc0 sc1 lds\n\t" \
  "s_add_u32 s53, s52, 540672\n\t" \
  "s_add_u32 m0, s48, 65536\n\t" \
  "s_nop 0\n\t" \
  "buffer_load_dwordx4 v0, s[36:39], s53 offen sc0 sc1 lds\n\t" \
  "buffer_load_dwordx4 v0, s[36:39], s53 offen offset:1024 sc0 sc1 lds\n\t" \
  "buffer_load_dwordx4 v0, s[36:39], s53 offen offset:2048 sc0 sc1 lds\n\t" \
  "buffer_load_dwordx4 v0, s[36:39], s53 offen offset:3072 sc0 sc1 lds\n\t" \
  "s_add_u32 s53, s51, 16384\n\t" \
  "s_add_u32 m0, s48, 81920\n\t" \
  "s_nop 0\n\t" \
  "buffer_load_dwordx4 v0, s[36:39], s53 offen sc0 sc1 lds\n\t" \
  "buffer_load_dwordx4 v0, s[36:39], s53 offen offset:1024 sc0 sc1 lds\n\t" \
  "buffer_load_dwordx4 v0, s[36:39], s53 offen offset:2048 sc0 sc1 lds\n\t" \
  "buffer_load_dwordx4 v0, s[36:39], s53 offen offset:3072 sc0 sc1 lds\n\t" \
  "s_add_u32 s53, s52, 557056\n\t" \
  "s_add_u32 m0, s48, 98304\n\t" \
  "s_nop 0\n\t" \
  "buffer_load_dwordx4 v0, s[36:39], s53 offen sc0 sc1 lds\n\t" \
  "buffer_load_dwordx4 v0, s[36:39], s53 offen offset:1024 sc0 sc1 lds\n\t" \
  "buffer_load_dwordx4 v0, s[36:39], s53 offen offset:2048 sc0 sc1 lds\n\t" \
  "buffer_load_dwordx4 v0, s[36:39], s53 offen offset:3072 sc0 sc1 lds\n\t" \
  "s_add_u32 s53, s52, 573440\n\t" \
  "s_add_u32 m0, s48, 131072\n\t" \
  "s_nop 0\n\t" \
  "buffer_load_dwordx4 v0, s[36:39], s53 offen sc0 sc1 lds\n\t" \
  "buffer_load_dwordx4 v0, s[36:39], s53 offen offset:1024 sc0 sc1 lds\n\t" \
  "buffer_load_dwordx4 v0, s[36:39], s53 offen offset:2048 sc0 sc1 lds\n\t" \
  "buffer_load_dwordx4 v0, s[36:39], s53 offen offset:3072 sc0 sc1 lds\n\t" \
  "s_add_u32 s53, s51, 32768\n\t" \
  "s_add_u32 m0, s48, 147456\n\t" \
  "s_nop 0\n\t" \
  "buffer_load_dwordx4 v0, s[36:39], s53 offen sc0 sc1 lds\n\t" \
  "buffer_load_dwordx4 v0, s[36:39], s53 offen offset:1024 sc0 sc1 lds\n\t" \
  "buffer_load_dwordx4 v0, s[36:39], s53 offen offset:2048 sc0 sc1 lds\n\t" \
  "buffer_load_dwordx4 v0, s[36:39], s53 offen offset:3072 sc0 sc1 lds\n\t" \
  "s_add_u32 s53, s52, 589824\n\t" \
  "s_mov_b32 m0, s48\n\t" \
  "s_nop 0\n\t" \
  "buffer_load_dwordx4 v0, s[36:39], s53 offen sc0 sc1 lds\n\t" \
  "buffer_load_dwordx4 v0, s[36:39], s53 offen offset:1024 sc0 sc1 lds\n\t" \
  "buffer_load_dwordx4 v0, s[36:39], s53 offen offset:2048 sc0 sc1 lds\n\t" \
  "buffer_load_dwordx4 v0, s[36:39], s53 offen offset:3072 sc0 sc1 lds\n\t" \
  "s_add_u32 s53, s52, 606208\n\t" \
  "s_add_u32 m0, s48, 32768\n\t" \
  "s_nop 0\n\t" \
  "buffer_load_dwordx4 v0, s[36:39], s53 offen sc0 sc1 lds\n\t" \
  "buffer_load_dwordx4 v0, s[36:39], s53 offen offset:1024 sc0 sc1 lds\n\t" \
  "buffer_load_dwordx4 v0, s[36:39], s53 offen offset:2048 sc0 sc1 lds\n\t" \
  "buffer_load_dwordx4 v0, s[36:39], s53 offen offset:3072 sc0 sc1 lds\n\t" \
  "s_add_u32 s53, s51, 49152\n\t" \
  "s_add_u32 m0, s48, 49152\n\t" \
  "s_nop 0\n\t" \
  "buffer_load_dwordx4 v0, s[36:39], s53 offen sc0 sc1 lds\n\t" \
  "buffer_load_dwordx4 v0, s[36:39], s53 offen offset:1024 sc0 sc1 lds\n\t" \
  "buffer_load_dwordx4 v0, s[36:39], s53 offen offset:2048 sc0 sc1 lds\n\t" \
  "buffer_load_dwordx4 v0, s[36:39], s53 offen offset:3072 sc0 sc1 lds\n\t" \
  "s_waitcnt vmcnt(0)\n\t" \
  "s_barrier\n\t" \
  "ds_read_b128 v[128:131], v1 offset:16384\n\t" \
  "ds_read_b128 v[132:135], v1 offset:24576\n\t" \
  "ds_read_b128 v[136:139], v1 offset:17408\n\t" \
  "ds_read_b128 v[140:143], v1 offset:25600\n\t" \
  "ds_read_b128 v[144:147], v1 offset:18432\n\t" \
  "ds_read_b128 v[148:151], v1 offset:26624\n\t" \
  "ds_read_b128 v[152:155], v1 offset:19456\n\t" \
  "ds_read_b128 v[156:159], v1 offset:27648\n\t" \
  "ds_read_b128 v[192:195], v4 offset:49152\n\t" \
  "ds_read_b128 v[196:199], v4 offset:57344\n\t" \
  "ds_read_b128 v[200:203], v4 offset:50176\n\t" \
  "ds_read_b128 v[204:207], v4 offset:58368\n\t" \
  "ds_read_b128 v[208:211], v4 offset:51200\n\t" \
  "ds_read_b128 v[212:215], v4 offset:59392\n\t" \
  "ds_read_b128 v[216:219], v4 offset:52224\n\t" \
  "ds_read_b128 v[220:223], v4 offset:60416\n\t" \
  "s_waitcnt lgkmcnt(0)\n\t" \
  "v_mfma_f32_16x16x128_f8f6f4 a[16:19], v[128:135], v[192:199], a[16:19]\n\t" \
  "s_barrier\n\t" \
  "v_mfma_f32_16x16x128_f8f6f4 a[20:23], v[128:135], v[200:207], a[20:23]\n\t" \
  "ds_read_b128 v[160:163], v2 offset:0\n\t" \
  "ds_read_b128 v[164:167], v2 offset:8192\n\t" \
  "v_mfma_f32_16x16x128_f8f6f4 a[24:27], v[128:135], v[208:215], a[24:27]\n\t" \
  "ds_read_b128 v[168:171], v2 offset:1024\n\t" \
  "ds_read_b128 v[172:175], v2 offset:9216\n\t" \
  "v_mfma_f32_16x16x128_f8f6f4 a[28:31], v[128:135], v[216:223], a[28:31]\n\t" \
  "ds_read_b128 v[176:179], v2 offset:2048\n\t" \
  "ds_read_b128 v[180:183], v2 offset:10240\n\t" \
  "v_mfma_f32_16x16x128_f8f6f4 a[60:63], v[136:143], v[216:223], a[60:63]\n\t" \
  "ds_read_b128 v[184:187], v2 offset:3072\n\t" \
  "ds_read_b128 v[188:191], v2 offset:11264\n\t" \
  "v_mfma_f32_16x16x128_f8f6f4 a[56:59], v[136:143], v[208:215], a[56:59]\n\t" \
  "ds_read_b128 v[224:227], v4 offset:16384\n\t" \
  "ds_read_b128 v[228:231], v4 offset:24576\n\t" \
  "v_mfma_f32_16x16x128_f8f6f4 a[52:55], v[136:143], v[200:207], a[52:55]\n\t" \
  "ds_read_b128 v[232:235], v4 offset:17408\n\t" \
  "ds_read_b128 v[236:239], v4 offset:25600\n\t" \
  "s_add_u32 s53, s52, 622592\n\t" \
  "s_add_u32 m0, s48, 114688\n\t" \
  "v_mfma_f32_16x16x128_f8f6f4 a[48:51], v[136:143], v[192:199], a[48:51]\n\t" \
  "buffer_load_dwordx4 v0, s[36:39], s53 offen sc0 sc1 lds\n\t" \
  "v_mfma_f32_16x16x128_f8f6f4 a[80:83], v[144:151], v[192:199], a[80:83]\n\t" \
  "buffer_load_dwordx4 v0, s[36:39], s53 offen offset:1024 sc0 sc1 lds\n\t" \
  "v_mfma_f32_16x16x128_f8f6f4 a[84:87], v[144:151], v[200:207], a[84:87]\n\t" \
  "buffer_load_dwordx4 v0, s[36:39], s53 offen offset:2048 sc0 sc1 lds\n\t" \
  "v_mfma_f32_16x16x128_f8f6f4 a[88:91], v[144:151], v[208:215], a[88:91]\n\t" \
  "buffer_load_dwordx4 v0, s[36:39], s53 offen offset:3072 sc0 sc1 lds\n\t" \
  "s_add_u32 s53, s52, 638976\n\t" \
  "s_add_u32 m0, s48, 16384\n\t" \
  "v_mfma_f32_16x16x128_f8f6f4 a[92:95], v[144:151], v[216:223], a[92:95]\n\t" \
  "buffer_load_dwordx4 v0, s[36:39], s53 offen sc0 sc1 lds\n\t" \
  "v_mfma_f32_16x16x128_f8f6f4 a[124:127], v[152:159], v[216:223], a[124:127]\n\t" \
  "buffer_load_dwordx4 v0, s[36:39], s53 offen offset:1024 sc0 sc1 lds\n\t" \
  "v_mfma_f32_16x16x128_f8f6f4 a[120:123], v[152:159], v[208:215], a[120:123]\n\t" \
  "buffer_load_dwordx4 v0, s[36:39], s53 offen offset:2048 sc0 sc1 lds\n\t" \
  "v_mfma_f32_16x16x128_f8f6f4 a[116:119], v[152:159], v[200:207], a[116:119]\n\t" \
  "buffer_load_dwordx4 v0, s[36:39], s53 offen offset:3072 sc0 sc1 lds\n\t" \
  "v_mfma_f32_16x16x128_f8f6f4 a[112:115], v[152:159], v[192:199], a[112:115]\n\t" \
  "buffer_load_dword v7, v8, s[44:47], s76 offen sc0 sc1\n\t" \
  "s_waitcnt vmcnt(0)\n\t" \
  "v_cmp_le_u32_e32 vcc, s75, v7\n\t" \
  "s_nop 1\n\t" \
  "s_cmp_eq_u64 vcc, -1\n\t" \
  "s_cbranch_scc1 L_OKN11_%=\n\t" \
  "s_swappc_b64 s[80:81], s[78:79]\n\t" \
  "L_OKN11_%=:\n\t" \
  "s_add_u32 s76, s76, 1024\n\t" \
  "s_waitcnt lgkmcnt(0)\n\t" \
  "v_mfma_f32_16x16x128_f8f6f4 a[144:147], v[160:167], v[192:199], a[144:147]\n\t" \
  "s_barrier\n\t" \
  "v_mfma_f32_16x16x128_f8f6f4 a[148:151], v[160:167], v[200:207], a[148:151]\n\t" \
  "ds_read_b128 v[128:131], v2 offset:32768\n\t" \
  "ds_read_b128 v[132:135], v2 offset:40960\n\t" \
  "v_mfma_f32_16x16x128_f8f6f4 a[152:155], v[160:167], v[208:215], a[152:155]\n\t" \
  "ds_read_b128 v[136:139], v2 offset:33792\n\t" \
  "ds_read_b128 v[140:143], v2 offset:41984\n\t" \
  "v_mfma_f32_16x16x128_f8f6f4 a[156:159], v[160:167], v[216:223], a[156:159]\n\t" \
  "ds_read_b128 v[144:147], v2 offset:34816\n\t" \
  "ds_read_b128 v[148:151], v2 offset:43008\n\t" \
  "v_mfma_f32_16x16x128_f8f6f4 a[188:191], v[168:175], v[216:223], a[188:191]\n\t" \
  "ds_read_b128 v[152:155], v2 offset:35840\n\t" \
  "ds_read_b128 v[156:159], v2 offset:44032\n\t" \
  "v_mfma_f32_16x16x128_f8f6f4 a[184:187], v[168:175], v[208:215], a[184:187]\n\t" \
  "ds_read_b128 v[240:243], v4 offset:18432\n\t" \
  "ds_read_b128 v[244:247], v4 offset:26624\n\t" \
  "v_mfma_f32_16x16x128_f8f6f4 a[180:183], v[168:175], v[200:207], a[180:183]\n\t" \
  "ds_read_b128 v[248:251], v4 offset:19456\n\t" \
  "ds_read_b128 v[252:255], v4 offset:27648\n\t" \
  "v_mfma_f32_16x16x128_f8f6f4 a[176:179], v[168:175], v[192:199], a[176:179]\n\t" \
  "s_add_u32 s53, s51, 65536\n\t" \
  "s_add_u32 m0, s48, 65536\n\t" \
  "v_mfma_f32_16x16x128_f8f6f4 a[208:211], v[176:183], v[192:199], a[208:211]\n\t" \
  "buffer_load_dwordx4 v0, s[36:39], s53 offen sc0 sc1 lds\n\t" \
  "v_mfma_f32_16x16x128_f8f6f4 a[212:215], v[176:183], v[200:207], a[212:215]\n\t" \
  "v_mfma_f32_16x16x128_f8f6f4 a[216:219], v[176:183], v[208:215], a[216:219]\n\t" \
  "buffer_load_dwordx4 v0, s[36:39], s53 offen offset:1024 sc0 sc1 lds\n\t" \
  "v_mfma_f32_16x16x128_f8f6f4 a[220:223], v[176:183], v[216:223], a[220:223]\n\t" \
  "v_mfma_f32_16x16x128_f8f6f4 a[252:255], v[184:191], v[216:223], a[252:255]\n\t" \
  "buffer_load_dwordx4 v0, s[36:39], s53 offen offset:2048 sc0 sc1 lds\n\t" \
  "v_mfma_f32_16x16x128_f8f6f4 a[248:251], v[184:191], v[208:215], a[248:251]\n\t" \
  "v_mfma_f32_16x16x128_f8f6f4 a[244:247], v[184:191], v[200:207], a[244:247]\n\t" \
  "buffer_load_dwordx4 v0, s[36:39], s53 offen offset:3072 sc0 sc1 lds\n\t" \
  "v_mfma_f32_16x16x128_f8f6f4 a[240:243], v[184:191], v[192:199], a[240:243]\n\t" \
  "s_waitcnt lgkmcnt(0)\n\t" \
  "v_mfma_f32_16x16x128_f8f6f4 a[16:19], v[128:135], v[224:231], a[16:19]\n\t" \
  "s_barrier\n\t" \
  "v_mfma_f32_16x16x128_f8f6f4 a[20:23], v[128:135], v[232:239], a[20:23]\n\t" \
  "ds_read_b128 v[160:163], v126 offset:0\n\t" \
  "ds_read_b128 v[164:167], v126 offset:8192\n\t" \
  "v_mfma_f32_16x16x128_f8f6f4 a[24:27], v[128:135], v[240:247], a[24:27]\n\t" \
  "ds_read_b128 v[168:171], v126 offset:1024\n\t" \
  "ds_read_b128 v[172:175], v126 offset:9216\n\t" \
  "v_mfma_f32_16x16x128_f8f6f4 a[28:31], v[128:135], v[248:255], a[28:31]\n\t" \
  "ds_read_b128 v[176:179], v126 offset:2048\n\t" \
  "ds_read_b128 v[180:183], v126 offset:10240\n\t" \
  "v_mfma_f32_16x16x128_f8f6f4 a[60:63], v[136:143], v[248:255], a[60:63]\n\t" \
  "ds_read_b128 v[184:187], v126 offset:3072\n\t" \
  "ds_read_b128 v[188:191], v126 offset:11264\n\t" \
  "v_mfma_f32_16x16x128_f8f6f4 a[56:59], v[136:143], v[240:247], a[56:59]\n\t" \
  "ds_read_b128 v[192:195], v127 offset:16384\n\t" \
  "ds_read_b128 v[196:199], v127 offset:24576\n\t" \
  "v_mfma_f32_16x16x128_f8f6f4 a[52:55], v[136:143], v[232:239], a[52:55]\n\t" \
  "ds_read_b128 v[200:203], v127 offset:17408\n\t" \
  "ds_read_b128 v[204:207], v127 offset:25600\n\t" \
  "s_add_u32 s53, s52, 655360\n\t" \
  "s_add_u32 m0, s48, 81920\n\t" \
  "v_mfma_f32_16x16x128_f8f6f4 a[48:51], v[136:143], v[224:231], a[48:51]\n\t" \
  "buffer_load_dwordx4 v0, s[36:39], s53 offen sc0 sc1 lds\n\t" \
  "v_mfma_f32_16x16x128_f8f6f4 a[80:83], v[144:151], v[224:231], a[80:83]\n\t" \
  "buffer_load_dwordx4 v0, s[36:39], s53 offen offset:1024 sc0 sc1 lds\n\t" \
  "v_mfma_f32_16x16x128_f8f6f4 a[84:87], v[144:151], v[232:239], a[84:87]\n\t" \
  "buffer_load_dwordx4 v0, s[36:39], s53 offen offset:2048 sc0 sc1 lds\n\t" \
  "v_mfma_f32_16x16x128_f8f6f4 a[88:91], v[144:151], v[240:247], a[88:91]\n\t" \
  "buffer_load_dwordx4 v0, s[36:39], s53 offen offset:3072 sc0 sc1 lds\n\t" \
  "s_add_u32 s53, s52, 671744\n\t" \
  "s_add_u32 m0, s48, 98304\n\t" \
  "v_mfma_f32_16x16x128_f8f6f4 a[92:95], v[144:151], v[248:255], a[92:95]\n\t" \
  "buffer_load_dwordx4 v0, s[36:39], s53 offen sc0 sc1 lds\n\t" \
  "v_mfma_f32_16x16x128_f8f6f4 a[124:127], v[152:159], v[248:255], a[124:127]\n\t" \
  "buffer_load_dwordx4 v0, s[36:39], s53 offen offset:1024 sc0 sc1 lds\n\t" \
  "v_mfma_f32_16x16x128_f8f6f4 a[120:123], v[152:159], v[240:247], a[120:123]\n\t" \
  "buffer_load_dwordx4 v0, s[36:39], s53 offen offset:2048 sc0 sc1 lds\n\t" \
  "v_mfma_f32_16x16x128_f8f6f4 a[116:119], v[152:159], v[232:239], a[116:119]\n\t" \
  "buffer_load_dwordx4 v0, s[36:39], s53 offen offset:3072 sc0 sc1 lds\n\t" \
  "v_mfma_f32_16x16x128_f8f6f4 a[112:115], v[152:159], v[224:231], a[112:115]\n\t" \
  "s_waitcnt lgkmcnt(0)\n\t" \
  "v_mfma_f32_16x16x128_f8f6f4 a[144:147], v[160:167], v[224:231], a[144:147]\n\t" \
  "s_barrier\n\t" \
  "v_mfma_f32_16x16x128_f8f6f4 a[148:151], v[160:167], v[232:239], a[148:151]\n\t" \
  "ds_read_b128 v[128:131], v1 offset:0\n\t" \
  "ds_read_b128 v[132:135], v1 offset:8192\n\t" \
  "v_mfma_f32_16x16x128_f8f6f4 a[152:155], v[160:167], v[240:247], a[152:155]\n\t" \
  "ds_read_b128 v[136:139], v1 offset:1024\n\t" \
  "ds_read_b128 v[140:143], v1 offset:9216\n\t" \
  "v_mfma_f32_16x16x128_f8f6f4 a[156:159], v[160:167], v[248:255], a[156:159]\n\t" \
  "ds_read_b128 v[144:147], v1 offset:2048\n\t" \
  "ds_read_b128 v[148:151], v1 offset:10240\n\t" \
  "v_mfma_f32_16x16x128_f8f6f4 a[188:191], v[168:175], v[248:255], a[188:191]\n\t" \
  "ds_read_b128 v[152:155], v1 offset:3072\n\t" \
  "ds_read_b128 v[156:159], v1 offset:11264\n\t" \
  "v_mfma_f32_16x16x128_f8f6f4 a[184:187], v[168:175], v[240:247], a[184:187]\n\t" \
  "ds_read_b128 v[208:211], v127 offset:18432\n\t" \
  "ds_read_b128 v[212:215], v127 offset:26624\n\t" \
  "v_mfma_f32_16x16x128_f8f6f4 a[180:183], v[168:175], v[232:239], a[180:183]\n\t" \
  "ds_read_b128 v[216:219], v127 offset:19456\n\t" \
  "ds_read_b128 v[220:223], v127 offset:27648\n\t" \
  "v_mfma_f32_16x16x128_f8f6f4 a[176:179], v[168:175], v[224:231], a[176:179]\n\t" \
  "s_add_u32 s53, s51, 81920\n\t" \
  "s_add_u32 m0, s48, 131072\n\t" \
  "v_mfma_f32_16x16x128_f8f6f4 a[208:211], v[176:183], v[224:231], a[208:211]\n\t" \
  "buffer_load_dwordx4 v0, s[36:39], s53 offen sc0 sc1 lds\n\t" \
  "v_mfma_f32_16x16x128_f8f6f4 a[212:215], v[176:183], v[232:239], a[212:215]\n\t" \
  "v_mfma_f32_16x16x128_f8f6f4 a[216:219], v[176:183], v[240:247], a[216:219]\n\t" \
  "buffer_load_dwordx4 v0, s[36:39], s53 offen offset:1024 sc0 sc1 lds\n\t" \
  "v_mfma_f32_16x16x128_f8f6f4 a[220:223], v[176:183], v[248:255], a[220:223]\n\t" \
  "v_mfma_f32_16x16x128_f8f6f4 a[252:255], v[184:191], v[248:255], a[252:255]\n\t" \
  "buffer_load_dwordx4 v0, s[36:39], s53 offen offset:2048 sc0 sc1 lds\n\t" \
  "v_mfma_f32_16x16x128_f8f6f4 a[248:251], v[184:191], v[240:247], a[248:251]\n\t" \
  "v_mfma_f32_16x16x128_f8f6f4 a[244:247], v[184:191], v[232:239], a[244:247]\n\t" \
  "buffer_load_dwordx4 v0, s[36:39], s53 offen offset:3072 sc0 sc1 lds\n\t" \
  "v_mfma_f32_16x16x128_f8f6f4 a[240:243], v[184:191], v[224:231], a[240:243]\n\t" \
  "s_waitcnt lgkmcnt(0)\n\t" \
  "v_mfma_f32_16x16x128_f8f6f4 a[16:19], v[128:135], v[192:199], a[16:19]\n\t" \
  "s_barrier\n\t" \
  "v_mfma_f32_16x16x128_f8f6f4 a[20:23], v[128:135], v[200:207], a[20:23]\n\t" \
  "ds_read_b128 v[160:163], v1 offset:32768\n\t" \
  "ds_read_b128 v[164:167], v1 offset:40960\n\t" \
  "v_mfma_f32_16x16x128_f8f6f4 a[24:27], v[128:135], v[208:215], a[24:27]\n\t" \
  "ds_read_b128 v[168:171], v1 offset:33792\n\t" \
  "ds_read_b128 v[172:175], v1 offset:41984\n\t" \
  "v_mfma_f32_16x16x128_f8f6f4 a[28:31], v[128:135], v[216:223], a[28:31]\n\t" \
  "ds_read_b128 v[176:179], v1 offset:34816\n\t" \
  "ds_read_b128 v[180:183], v1 offset:43008\n\t" \
  "v_mfma_f32_16x16x128_f8f6f4 a[60:63], v[136:143], v[216:223], a[60:63]\n\t" \
  "ds_read_b128 v[184:187], v1 offset:35840\n\t" \
  "ds_read_b128 v[188:191], v1 offset:44032\n\t" \
  "v_mfma_f32_16x16x128_f8f6f4 a[56:59], v[136:143], v[208:215], a[56:59]\n\t" \
  "ds_read_b128 v[224:227], v3 offset:49152\n\t" \
  "ds_read_b128 v[228:231], v3 offset:57344\n\t" \
  "v_mfma_f32_16x16x128_f8f6f4 a[52:55], v[136:143], v[200:207], a[52:55]\n\t" \
  "ds_read_b128 v[232:235], v3 offset:50176\n\t" \
  "ds_read_b128 v[236:239], v3 offset:58368\n\t" \
  "s_add_u32 s53, s52, 688128\n\t" \
  "s_add_u32 m0, s48, 147456\n\t" \
  "v_mfma_f32_16x16x128_f8f6f4 a[48:51], v[136:143], v[192:199], a[48:51]\n\t" \
  "buffer_load_dwordx4 v0, s[36:39], s53 offen sc0 sc1 lds\n\t" \
  "v_mfma_f32_16x16x128_f8f6f4 a[80:83], v[144:151], v[192:199], a[80:83]\n\t" \
  "buffer_load_dwordx4 v0, s[36:39], s53 offen offset:1024 sc0 sc1 lds\n\t" \
  "v_mfma_f32_16x16x128_f8f6f4 a[84:87], v[144:151], v[200:207], a[84:87]\n\t" \
  "buffer_load_dwordx4 v0, s[36:39], s53 offen offset:2048 sc0 sc1 lds\n\t" \
  "v_mfma_f32_16x16x128_f8f6f4 a[88:91], v[144:151], v[208:215], a[88:91]\n\t" \
  "buffer_load_dwordx4 v0, s[36:39], s53 offen offset:3072 sc0 sc1 lds\n\t" \
  "s_add_u32 s53, s52, 704512\n\t" \
  "s_mov_b32 m0, s48\n\t" \
  "v_mfma_f32_16x16x128_f8f6f4 a[92:95], v[144:151], v[216:223], a[92:95]\n\t" \
  "buffer_load_dwordx4 v0, s[36:39], s53 offen sc0 sc1 lds\n\t" \
  "v_mfma_f32_16x16x128_f8f6f4 a[124:127], v[152:159], v[216:223], a[124:127]\n\t" \
  "buffer_load_dwordx4 v0, s[36:39], s53 offen offset:1024 sc0 sc1 lds\n\t" \
  "v_mfma_f32_16x16x128_f8f6f4 a[120:123], v[152:159], v[208:215], a[120:123]\n\t" \
  "buffer_load_dwordx4 v0, s[36:39], s53 offen offset:2048 sc0 sc1 lds\n\t" \
  "v_mfma_f32_16x16x128_f8f6f4 a[116:119], v[152:159], v[200:207], a[116:119]\n\t" \
  "buffer_load_dwordx4 v0, s[36:39], s53 offen offset:3072 sc0 sc1 lds\n\t" \
  "v_mfma_f32_16x16x128_f8f6f4 a[112:115], v[152:159], v[192:199], a[112:115]\n\t" \
  "s_waitcnt lgkmcnt(0)\n\t" \
  "v_mfma_f32_16x16x128_f8f6f4 a[144:147], v[160:167], v[192:199], a[144:147]\n\t" \
  "s_waitcnt vmcnt(29)\n\t" \
  "s_barrier\n\t" \
  "v_mfma_f32_16x16x128_f8f6f4 a[148:151], v[160:167], v[200:207], a[148:151]\n\t" \
  "ds_read_b128 v[128:131], v2 offset:49152\n\t" \
  "ds_read_b128 v[132:135], v2 offset:57344\n\t" \
  "v_mfma_f32_16x16x128_f8f6f4 a[152:155], v[160:167], v[208:215], a[152:155]\n\t" \
  "ds_read_b128 v[136:139], v2 offset:50176\n\t" \
  "ds_read_b128 v[140:143], v2 offset:58368\n\t" \
  "v_mfma_f32_16x16x128_f8f6f4 a[156:159], v[160:167], v[216:223], a[156:159]\n\t" \
  "ds_read_b128 v[144:147], v2 offset:51200\n\t" \
  "ds_read_b128 v[148:151], v2 offset:59392\n\t" \
  "v_mfma_f32_16x16x128_f8f6f4 a[188:191], v[168:175], v[216:223], a[188:191]\n\t" \
  "ds_read_b128 v[152:155], v2 offset:52224\n\t" \
  "ds_read_b128 v[156:159], v2 offset:60416\n\t" \
  "v_mfma_f32_16x16x128_f8f6f4 a[184:187], v[168:175], v[208:215], a[184:187]\n\t" \
  "ds_read_b128 v[240:243], v3 offset:51200\n\t" \
  "ds_read_b128 v[244:247], v3 offset:59392\n\t" \
  "v_mfma_f32_16x16x128_f8f6f4 a[180:183], v[168:175], v[200:207], a[180:183]\n\t" \
  "ds_read_b128 v[248:251], v3 offset:52224\n\t" \
  "ds_read_b128 v[252:255], v3 offset:60416\n\t" \
  "v_mfma_f32_16x16x128_f8f6f4 a[176:179], v[168:175], v[192:199], a[176:179]\n\t" \
  "s_add_u32 s53, s51, 98304\n\t" \
  "s_add_u32 m0, s48, 32768\n\t" \
  "v_mfma_f32_16x16x128_f8f6f4 a[208:211], v[176:183], v[192:199], a[208:211]\n\t" \
  "buffer_load_dwordx4 v0, s[36:39], s53 offen sc0 sc1 lds\n\t" \
  "v_mfma_f32_16x16x128_f8f6f4 a[212:215], v[176:183], v[200:207], a[212:215]\n\t" \
  "v_mfma_f32_16x16x128_f8f6f4 a[216:219], v[176:183], v[208:215], a[216:219]\n\t" \
  "buffer_load_dwordx4 v0, s[36:39], s53 offen offset:1024 sc0 sc1 lds\n\t" \
  "v_mfma_f32_16x16x128_f8f6f4 a[220:223], v[176:183], v[216:223], a[220:223]\n\t" \
  "v_mfma_f32_16x16x128_f8f6f4 a[252:255], v[184:191], v[216:223], a[252:255]\n\t" \
  "buffer_load_dwordx4 v0, s[36:39], s53 offen offset:2048 sc0 sc1 lds\n\t" \
  "v_mfma_f32_16x16x128_f8f6f4 a[248:251], v[184:191], v[208:215], a[248:251]\n\t" \
  "v_mfma_f32_16x16x128_f8f6f4 a[244:247], v[184:191], v[200:207], a[244:247]\n\t" \
  "buffer_load_dwordx4 v0, s[36:39], s53 offen offset:3072 sc0 sc1 lds\n\t" \
  "v_mfma_f32_16x16x128_f8f6f4 a[240:243], v[184:191], v[192:199], a[240:243]\n\t" \
  "s_waitcnt lgkmcnt(0)\n\t" \
  "v_mfma_f32_16x16x128_f8f6f4 a[16:19], v[128:135], v[224:231], a[16:19]\n\t" \
  "s_waitcnt vmcnt(24)\n\t" \
  "s_barrier\n\t" \
  "v_mfma_f32_16x16x128_f8f6f4 a[20:23], v[128:135], v[232:239], a[20:23]\n\t" \
  "ds_read_b128 v[160:163], v1 offset:16384\n\t" \
  "ds_read_b128 v[164:167], v1 offset:24576\n\t" \
  "v_mfma_f32_16x16x128_f8f6f4 a[24:27], v[128:135], v[240:247], a[24:27]\n\t" \
  "ds_read_b128 v[168:171], v1 offset:17408\n\t" \
  "ds_read_b128 v[172:175], v1 offset:25600\n\t" \
  "v_mfma_f32_16x16x128_f8f6f4 a[28:31], v[128:135], v[248:255], a[28:31]\n\t" \
  "ds_read_b128 v[176:179], v1 offset:18432\n\t" \
  "ds_read_b128 v[180:183], v1 offset:26624\n\t" \
  "v_mfma_f32_16x16x128_f8f6f4 a[60:63], v[136:143], v[248:255], a[60:63]\n\t" \
  "ds_read_b128 v[184:187], v1 offset:19456\n\t" \
  "ds_read_b128 v[188:191], v1 offset:27648\n\t" \
  "v_mfma_f32_16x16x128_f8f6f4 a[56:59], v[136:143], v[240:247], a[56:59]\n\t" \
  "ds_read_b128 v[192:195], v4 offset:0\n\t" \
  "ds_read_b128 v[196:199], v4 offset:8192\n\t" \
  "v_mfma_f32_16x16x128_f8f6f4 a[52:55], v[136:143], v[232:239], a[52:55]\n\t" \
  "ds_read_b128 v[200:203], v4 offset:1024\n\t" \
  "ds_read_b128 v[204:207], v4 offset:9216\n\t" \
  "s_add_u32 s53, s52, 720896\n\t" \
  "s_add_u32 m0, s48, 114688\n\t" \
  "v_mfma_f32_16x16x128_f8f6f4 a[48:51], v[136:143], v[224:231], a[48:51]\n\t" \
  "buffer_load_dwordx4 v0, s[36:39], s53 offen sc0 sc1 lds\n\t" \
  "v_mfma_f32_16x16x128_f8f6f4 a[80:83], v[144:151], v[224:231], a[80:83]\n\t" \
  "buffer_load_dwordx4 v0, s[36:39], s53 offen offset:1024 sc0 sc1 lds\n\t" \
  "v_mfma_f32_16x16x128_f8f6f4 a[84:87], v[144:151], v[232:239], a[84:87]\n\t" \
  "buffer_load_dwordx4 v0, s[36:39], s53 offen offset:2048 sc0 sc1 lds\n\t" \
  "v_mfma_f32_16x16x128_f8f6f4 a[88:91], v[144:151], v[240:247], a[88:91]\n\t" \
  "buffer_load_dwordx4 v0, s[36:39], s53 offen offset:3072 sc0 sc1 lds\n\t" \
  "s_add_u32 s53, s52, 737280\n\t" \
  "s_add_u32 m0, s48, 49152\n\t" \
  "v_mfma_f32_16x16x128_f8f6f4 a[92:95], v[144:151], v[248:255], a[92:95]\n\t" \
  "buffer_load_dwordx4 v0, s[36:39], s53 offen sc0 sc1 lds\n\t" \
  "v_mfma_f32_16x16x128_f8f6f4 a[124:127], v[152:159], v[248:255], a[124:127]\n\t" \
  "buffer_load_dwordx4 v0, s[36:39], s53 offen offset:1024 sc0 sc1 lds\n\t" \
  "v_mfma_f32_16x16x128_f8f6f4 a[120:123], v[152:159], v[240:247], a[120:123]\n\t" \
  "buffer_load_dwordx4 v0, s[36:39], s53 offen offset:2048 sc0 sc1 lds\n\t" \
  "v_mfma_f32_16x16x128_f8f6f4 a[116:119], v[152:159], v[232:239], a[116:119]\n\t" \
  "buffer_load_dwordx4 v0, s[36:39], s53 offen offset:3072 sc0 sc1 lds\n\t" \
  "v_mfma_f32_16x16x128_f8f6f4 a[112:115], v[152:159], v[224:231], a[112:115]\n\t" \
  "buffer_load_dword v7, v8, s[44:47], s76 offen sc0 sc1\n\t" \
  "s_waitcnt lgkmcnt(0)\n\t" \
  "v_mfma_f32_16x16x128_f8f6f4 a[144:147], v[160:167], v[224:231], a[144:147]\n\t" \
  "s_waitcnt vmcnt(29)\n\t" \
  "s_barrier\n\t" \
  "v_mfma_f32_16x16x128_f8f6f4 a[148:151], v[160:167], v[232:239], a[148:151]\n\t" \
  "ds_read_b128 v[128:131], v2 offset:16384\n\t" \
  "ds_read_b128 v[132:135], v2 offset:24576\n\t" \
  "v_mfma_f32_16x16x128_f8f6f4 a[152:155], v[160:167], v[240:247], a[152:155]\n\t" \
  "ds_read_b128 v[136:139], v2 offset:17408\n\t" \
  "ds_read_b128 v[140:143], v2 offset:25600\n\t" \
  "v_mfma_f32_16x16x128_f8f6f4 a[156:159], v[160:167], v[248:255], a[156:159]\n\t" \
  "ds_read_b128 v[144:147], v2 offset:18432\n\t" \
  "ds_read_b128 v[148:151], v2 offset:26624\n\t" \
  "v_mfma_f32_16x16x128_f8f6f4 a[188:191], v[168:175], v[248:255], a[188:191]\n\t" \
  "ds_read_b128 v[152:155], v2 offset:19456\n\t" \
  "ds_read_b128 v[156:159], v2 offset:27648\n\t" \
  "v_mfma_f32_16x16x128_f8f6f4 a[184:187], v[168:175], v[240:247], a[184:187]\n\t" \
  "ds_read_b128 v[208:211], v4 offset:2048\n\t" \
  "ds_read_b128 v[212:215], v4 offset:10240\n\t" \
  "v_mfma_f32_16x16x128_f8f6f4 a[180:183], v[168:175], v[232:239], a[180:183]\n\t" \
  "ds_read_b128 v[216:219], v4 offset:3072\n\t" \
  "ds_read_b128 v[220:223], v4 offset:11264\n\t" \
  "v_mfma_f32_16x16x128_f8f6f4 a[176:179], v[168:175], v[224:231], a[176:179]\n\t" \
  "s_add_u32 s53, s51, 114688\n\t" \
  "s_add_u32 m0, s48, 16384\n\t" \
  "v_mfma_f32_16x16x128_f8f6f4 a[208:211], v[176:183], v[224:231], a[208:211]\n\t" \
  "buffer_load_dwordx4 v0, s[36:39], s53 offen sc0 sc1 lds\n\t" \
  "v_mfma_f32_16x16x128_f8f6f4 a[212:215], v[176:183], v[232:239], a[212:215]\n\t" \
  "v_mfma_f32_16x16x128_f8f6f4 a[216:219], v[176:183], v[240:247], a[216:219]\n\t" \
  "buffer_load_dwordx4 v0, s[36:39], s53 offen offset:1024 sc0 sc1 lds\n\t" \
  "v_mfma_f32_16x16x128_f8f6f4 a[220:223], v[176:183], v[248:255], a[220:223]\n\t" \
  "v_mfma_f32_16x16x128_f8f6f4 a[252:255], v[184:191], v[248:255], a[252:255]\n\t" \
  "buffer_load_dwordx4 v0, s[36:39], s53 offen offset:2048 sc0 sc1 lds\n\t" \
  "v_mfma_f32_16x16x128_f8f6f4 a[248:251], v[184:191], v[240:247], a[248:251]\n\t" \
  "v_mfma_f32_16x16x128_f8f6f4 a[244:247], v[184:191], v[232:239], a[244:247]\n\t" \
  "buffer_load_dwordx4 v0, s[36:39], s53 offen offset:3072 sc0 sc1 lds\n\t" \
  "v_mfma_f32_16x16x128_f8f6f4 a[240:243], v[184:191], v[224:231], a[240:243]\n\t" \
  "s_waitcnt lgkmcnt(0)\n\t" \
  "v_mfma_f32_16x16x128_f8f6f4 a[16:19], v[128:135], v[192:199], a[16:19]\n\t" \
  "s_waitcnt vmcnt(25)\n\t" \
  "s_barrier\n\t" \
  "v_mfma_f32_16x16x128_f8f6f4 a[20:23], v[128:135], v[200:207], a[20:23]\n\t" \
  "ds_read_b128 v[160:163], v2 offset:32768\n\t" \
  "ds_read_b128 v[164:167], v2 offset:40960\n\t" \
  "v_mfma_f32_16x16x128_f8f6f4 a[24:27], v[128:135], v[208:215], a[24:27]\n\t" \
  "ds_read_b128 v[168:171], v2 offset:33792\n\t" \
  "ds_read_b128 v[172:175], v2 offset:41984\n\t" \
  "v_mfma_f32_16x16x128_f8f6f4 a[28:31], v[128:135], v[216:223], a[28:31]\n\t" \
  "ds_read_b128 v[176:179], v2 offset:34816\n\t" \
  "ds_read_b128 v[180:183], v2 offset:43008\n\t" \
  "v_mfma_f32_16x16x128_f8f6f4 a[60:63], v[136:143], v[216:223], a[60:63]\n\t" \
  "ds_read_b128 v[184:187], v2 offset:35840\n\t" \
  "ds_read_b128 v[188:191], v2 offset:44032\n\t" \
  "v_mfma_f32_16x16x128_f8f6f4 a[56:59], v[136:143], v[208:215], a[56:59]\n\t" \
  "ds_read_b128 v[224:227], v127 offset:0\n\t" \
  "ds_read_b128 v[228:231], v127 offset:8192\n\t" \
  "v_mfma_f32_16x16x128_f8f6f4 a[52:55], v[136:143], v[200:207], a[52:55]\n\t" \
  "ds_read_b128 v[232:235], v127 offset:1024\n\t" \
  "ds_read_b128 v[236:239], v127 offset:9216\n\t" \
  "s_add_u32 s53, s52, 753664\n\t" \
  "s_add_u32 m0, s48, 65536\n\t" \
  "v_mfma_f32_16x16x128_f8f6f4 a[48:51], v[136:143], v[192:199], a[48:51]\n\t" \
  "buffer_load_dwordx4 v0, s[36:39], s53 offen sc0 sc1 lds\n\t" \
  "v_mfma_f32_16x16x128_f8f6f4 a[80:83], v[144:151], v[192:199], a[80:83]\n\t" \
  "buffer_load_dwordx4 v0, s[36:39], s53 offen offset:1024 sc0 sc1 lds\n\t" \
  "v_mfma_f32_16x16x128_f8f6f4 a[84:87], v[144:151], v[200:207], a[84:87]\n\t" \
  "buffer_load_dwordx4 v0, s[36:39], s53 offen offset:2048 sc0 sc1 lds\n\t" \
  "v_mfma_f32_16x16x128_f8f6f4 a[88:91], v[144:151], v[208:215], a[88:91]\n\t" \
  "buffer_load_dwordx4 v0, s[36:39], s53 offen offset:3072 sc0 sc1 lds\n\t" \
  "s_add_u32 s53, s52, 770048\n\t" \
  "s_add_u32 m0, s48, 81920\n\t" \
  "v_mfma_f32_16x16x128_f8f6f4 a[92:95], v[144:151], v[216:223], a[92:95]\n\t" \
  "buffer_load_dwordx4 v0, s[36:39], s53 offen sc0 sc1 lds\n\t" \
  "v_mfma_f32_16x16x128_f8f6f4 a[124:127], v[152:159], v[216:223], a[124:127]\n\t" \
  "buffer_load_dwordx4 v0, s[36:39], s53 offen offset:1024 sc0 sc1 lds\n\t" \
  "v_mfma_f32_16x16x128_f8f6f4 a[120:123], v[152:159], v[208:215], a[120:123]\n\t" \
  "buffer_load_dwordx4 v0, s[36:39], s53 offen offset:2048 sc0 sc1 lds\n\t" \
  "v_mfma_f32_16x16x128_f8f6f4 a[116:119], v[152:159], v[200:207], a[116:119]\n\t" \
  "buffer_load_dwordx4 v0, s[36:39], s53 offen offset:3072 sc0 sc1 lds\n\t" \
  "v_mfma_f32_16x16x128_f8f6f4 a[112:115], v[152:159], v[192:199], a[112:115]\n\t" \
  "s_waitcnt vmcnt(12)\n\t" \
  "v_cmp_le_u32_e32 vcc, s75, v7\n\t" \
  "s_nop 1\n\t" \
  "s_cmp_eq_u64 vcc, -1\n\t" \
  "s_cbranch_scc1 L_OKN12_%=\n\t" \
  "s_swappc_b64 s[80:81], s[78:79]\n\t" \
  "L_OKN12_%=:\n\t" \
  "s_add_u32 s76, s76, 1024\n\t" \
  "s_waitcnt lgkmcnt(0)\n\t" \
  "v_mfma_f32_16x16x128_f8f6f4 a[144:147], v[160:167], v[192:199], a[144:147]\n\t" \
  "s_waitcnt vmcnt(29)\n\t" \
  "s_barrier\n\t" \
  "v_mfma_f32_16x16x128_f8f6f4 a[148:151], v[160:167], v[200:207], a[148:151]\n\t" \
  "ds_read_b128 v[128:131], v126 offset:16384\n\t" \
  "ds_read_b128 v[132:135], v126 offset:24576\n\t" \
  "v_mfma_f32_16x16x128_f8f6f4 a[152:155], v[160:167], v[208:215], a[152:155]\n\t" \
  "ds_read_b128 v[136:139], v126 offset:17408\n\t" \
  "ds_read_b128 v[140:143], v126 offset:25600\n\t" \
  "v_mfma_f32_16x16x128_f8f6f4 a[156:159], v[160:167], v[216:223], a[156:159]\n\t" \
  "ds_read_b128 v[144:147], v126 offset:18432\n\t" \
  "ds_read_b128 v[148:151], v126 offset:26624\n\t" \
  "v_mfma_f32_16x16x128_f8f6f4 a[188:191], v[168:175], v[216:223], a[188:191]\n\t" \
  "ds_read_b128 v[152:155], v126 offset:19456\n\t" \
  "ds_read_b128 v[156:159], v126 offset:27648\n\t" \
  "v_mfma_f32_16x16x128_f8f6f4 a[184:187], v[168:175], v[208:215], a[184:187]\n\t" \
  "ds_read_b128 v[240:243], v127 offset:2048\n\t" \
  "ds_read_b128 v[244:247], v127 offset:10240\n\t" \
  "v_mfma_f32_16x16x128_f8f6f4 a[180:183], v[168:175], v[200:207], a[180:183]\n\t" \
  "ds_read_b128 v[248:251], v127 offset:3072\n\t" \
  "ds_read_b128 v[252:255], v127 offset:11264\n\t" \
  "v_mfma_f32_16x16x128_f8f6f4 a[176:179], v[168:175], v[192:199], a[176:179]\n\t" \
  "s_add_u32 s53, s51, 131072\n\t" \
  "s_add_u32 m0, s48, 98304\n\t" \
  "v_mfma_f32_16x16x128_f8f6f4 a[208:211], v[176:183], v[192:199], a[208:211]\n\t" \
  "buffer_load_dwordx4 v0, s[36:39], s53 offen sc0 sc1 lds\n\t" \
  "v_mfma_f32_16x16x128_f8f6f4 a[212:215], v[176:183], v[200:207], a[212:215]\n\t" \
  "v_mfma_f32_16x16x128_f8f6f4 a[216:219], v[176:183], v[208:215], a[216:219]\n\t" \
  "buffer_load_dwordx4 v0, s[36:39], s53 offen offset:1024 sc0 sc1 lds\n\t" \
  "v_mfma_f32_16x16x128_f8f6f4 a[220:223], v[176:183], v[216:223], a[220:223]\n\t" \
  "v_mfma_f32_16x16x128_f8f6f4 a[252:255], v[184:191], v[216:223], a[252:255]\n\t" \
  "buffer_load_dwordx4 v0, s[36:39], s53 offen offset:2048 sc0 sc1 lds\n\t" \
  "v_mfma_f32_16x16x128_f8f6f4 a[248:251], v[184:191], v[208:215], a[248:251]\n\t" \
  "v_mfma_f32_16x16x128_f8f6f4 a[244:247], v[184:191], v[200:207], a[244:247]\n\t" \
  "buffer_load_dwordx4 v0, s[36:39], s53 offen offset:3072 sc0 sc1 lds\n\t" \
  "v_mfma_f32_16x16x128_f8f6f4 a[240:243], v[184:191], v[192:199], a[240:243]\n\t" \
  "s_waitcnt lgkmcnt(0)\n\t" \
  "v_mfma_f32_16x16x128_f8f6f4 a[16:19], v[128:135], v[224:231], a[16:19]\n\t" \
  "s_waitcnt vmcnt(25)\n\t" \
  "s_barrier\n\t" \
  "v_mfma_f32_16x16x128_f8f6f4 a[20:23], v[128:135], v[232:239], a[20:23]\n\t" \
  "ds_read_b128 v[160:163], v1 offset:0\n\t" \
  "ds_read_b128 v[164:167], v1 offset:8192\n\t" \
  "v_mfma_f32_16x16x128_f8f6f4 a[24:27], v[128:135], v[240:247], a[24:27]\n\t" \
  "ds_read_b128 v[168:171], v1 offset:1024\n\t" \
  "ds_read_b128 v[172:175], v1 offset:9216\n\t" \
  "v_mfma_f32_16x16x128_f8f6f4 a[28:31], v[128:135], v[248:255], a[28:31]\n\t" \
  "ds_read_b128 v[176:179], v1 offset:2048\n\t" \
  "ds_read_b128 v[180:183], v1 offset:10240\n\t" \
  "v_mfma_f32_16x16x128_f8f6f4 a[60:63], v[136:143], v[248:255], a[60:63]\n\t" \
  "ds_read_b128 v[184:187], v1 offset:3072\n\t" \
  "ds_read_b128 v[188:191], v1 offset:11264\n\t" \
  "v_mfma_f32_16x16x128_f8f6f4 a[56:59], v[136:143], v[240:247], a[56:59]\n\t" \
  "ds_read_b128 v[192:195], v3 offset:32768\n\t" \
  "ds_read_b128 v[196:199], v3 offset:40960\n\t" \
  "v_mfma_f32_16x16x128_f8f6f4 a[52:55], v[136:143], v[232:239], a[52:55]\n\t" \
  "ds_read_b128 v[200:203], v3 offset:33792\n\t" \
  "ds_read_b128 v[204:207], v3 offset:41984\n\t" \
  "s_add_u32 s53, s52, 786432\n\t" \
  "s_add_u32 m0, s48, 131072\n\t" \
  "v_mfma_f32_16x16x128_f8f6f4 a[48:51], v[136:143], v[224:231], a[48:51]\n\t" \
  "buffer_load_dwordx4 v0, s[36:39], s53 offen sc0 sc1 lds\n\t" \
  "v_mfma_f32_16x16x128_f8f6f4 a[80:83], v[144:151], v[224:231], a[80:83]\n\t" \
  "buffer_load_dwordx4 v0, s[36:39], s53 offen offset:1024 sc0 sc1 lds\n\t" \
  "v_mfma_f32_16x16x128_f8f6f4 a[84:87], v[144:151], v[232:239], a[84:87]\n\t" \
  "buffer_load_dwordx4 v0, s[36:39], s53 offen offset:2048 sc0 sc1 lds\n\t" \
  "v_mfma_f32_16x16x128_f8f6f4 a[88:91], v[144:151], v[240:247], a[88:91]\n\t" \
  "buffer_load_dwordx4 v0, s[36:39], s53 offen offset:3072 sc0 sc1 lds\n\t" \
  "s_add_u32 s53, s52, 802816\n\t" \
  "s_add_u32 m0, s48, 147456\n\t" \
  "v_mfma_f32_16x16x128_f8f6f4 a[92:95], v[144:151], v[248:255], a[92:95]\n\t" \
  "buffer_load_dwordx4 v0, s[36:39], s53 offen sc0 sc1 lds\n\t" \
  "v_mfma_f32_16x16x128_f8f6f4 a[124:127], v[152:159], v[248:255], a[124:127]\n\t" \
  "buffer_load_dwordx4 v0, s[36:39], s53 offen offset:1024 sc0 sc1 lds\n\t" \
  "v_mfma_f32_16x16x128_f8f6f4 a[120:123], v[152:159], v[240:247], a[120:123]\n\t" \
  "buffer_load_dwordx4 v0, s[36:39], s53 offen offset:2048 sc0 sc1 lds\n\t" \
  "v_mfma_f32_16x16x128_f8f6f4 a[116:119], v[152:159], v[232:239], a[116:119]\n\t" \
  "buffer_load_dwordx4 v0, s[36:39], s53 offen offset:3072 sc0 sc1 lds\n\t" \
  "v_mfma_f32_16x16x128_f8f6f4 a[112:115], v[152:159], v[224:231], a[112:115]\n\t" \
  "s_waitcnt lgkmcnt(0)\n\t" \
  "v_mfma_f32_16x16x128_f8f6f4 a[144:147], v[160:167], v[224:231], a[144:147]\n\t" \
  "s_waitcnt vmcnt(29)\n\t" \
  "s_barrier\n\t" \
  "v_mfma_f32_16x16x128_f8f6f4 a[148:151], v[160:167], v[232:239], a[148:151]\n\t" \
  "ds_read_b128 v[128:131], v2 offset:49152\n\t" \
  "ds_read_b128 v[132:135], v2 offset:57344\n\t" \
  "v_mfma_f32_16x16x128_f8f6f4 a[152:155], v[160:167], v[240:247], a[152:155]\n\t" \
  "ds_read_b128 v[136:139], v2 offset:50176\n\t" \
  "ds_read_b128 v[140:143], v2 offset:58368\n\t" \
  "v_mfma_f32_16x16x128_f8f6f4 a[156:159], v[160:167], v[248:255], a[156:159]\n\t" \
  "ds_read_b128 v[144:147], v2 offset:51200\n\t" \
  "ds_read_b128 v[148:151], v2 offset:59392\n\t" \
  "v_mfma_f32_16x16x128_f8f6f4 a[188:191], v[168:175], v[248:255], a[188:191]\n\t" \
  "ds_read_b128 v[152:155], v2 offset:52224\n\t" \
  "ds_read_b128 v[156:159], v2 offset:60416\n\t" \
  "v_mfma_f32_16x16x128_f8f6f4 a[184:187], v[168:175], v[240:247], a[184:187]\n\t" \
  "ds_read_b128 v[208:211], v3 offset:34816\n\t" \
  "ds_read_b128 v[212:215], v3 offset:43008\n\t" \
  "v_mfma_f32_16x16x128_f8f6f4 a[180:183], v[168:175], v[232:239], a[180:183]\n\t" \
  "ds_read_b128 v[216:219], v3 offset:35840\n\t" \
  "ds_read_b128 v[220:223], v3 offset:44032\n\t" \
  "v_mfma_f32_16x16x128_f8f6f4 a[176:179], v[168:175], v[224:231], a[176:179]\n\t" \
  "s_add_u32 s53, s51, 147456\n\t" \
  "s_mov_b32 m0, s48\n\t" \
  "v_mfma_f32_16x16x128_f8f6f4 a[208:211], v[176:183], v[224:231], a[208:211]\n\t" \
  "buffer_load_dwordx4 v0, s[36:39], s53 offen sc0 sc1 lds\n\t" \
  "v_mfma_f32_16x16x128_f8f6f4 a[212:215], v[176:183], v[232:239], a[212:215]\n\t" \
  "v_mfma_f32_16x16x128_f8f6f4 a[216:219], v[176:183], v[240:247], a[216:219]\n\t" \
  "buffer_load_dwordx4 v0, s[36:39], s53 offen offset:1024 sc0 sc1 lds\n\t" \
  "v_mfma_f32_16x16x128_f8f6f4 a[220:223], v[176:183], v[248:255], a[220:223]\n\t" \
  "v_mfma_f32_16x16x128_f8f6f4 a[252:255], v[184:191], v[248:255], a[252:255]\n\t" \
  "buffer_load_dwordx4 v0, s[36:39], s53 offen offset:2048 sc0 sc1 lds\n\t" \
  "v_mfma_f32_16x16x128_f8f6f4 a[248:251], v[184:191], v[240:247], a[248:251]\n\t" \
  "v_mfma_f32_16x16x128_f8f6f4 a[244:247], v[184:191], v[232:239], a[244:247]\n\t" \
  "buffer_load_dwordx4 v0, s[36:39], s53 offen offset:3072 sc0 sc1 lds\n\t" \
  "v_mfma_f32_16x16x128_f8f6f4 a[240:243], v[184:191], v[224:231], a[240:243]\n\t" \
  "s_waitcnt lgkmcnt(0)\n\t" \
  "v_mfma_f32_16x16x128_f8f6f4 a[16:19], v[128:135], v[192:199], a[16:19]\n\t" \
  "s_waitcnt vmcnt(24)\n\t" \
  "s_barrier\n\t" \
  "v_mfma_f32_16x16x128_f8f6f4 a[20:23], v[128:135], v[200:207], a[20:23]\n\t" \
  "ds_read_b128 v[160:163], v1 offset:49152\n\t" \
  "ds_read_b128 v[164:167], v1 offset:57344\n\t" \
  "v_mfma_f32_16x16x128_f8f6f4 a[24:27], v[128:135], v[208:215], a[24:27]\n\t" \
  "ds_read_b128 v[168:171], v1 offset:50176\n\t" \
  "ds_read_b128 v[172:175], v1 offset:58368\n\t" \
  "v_mfma_f32_16x16x128_f8f6f4 a[28:31], v[128:135], v[216:223], a[28:31]\n\t" \
  "ds_read_b128 v[176:179], v1 offset:51200\n\t" \
  "ds_read_b128 v[180:183], v1 offset:59392\n\t" \
  "v_mfma_f32_16x16x128_f8f6f4 a[60:63], v[136:143], v[216:223], a[60:63]\n\t" \
  "ds_read_b128 v[184:187], v1 offset:52224\n\t" \
  "ds_read_b128 v[188:191], v1 offset:60416\n\t" \
  "v_mfma_f32_16x16x128_f8f6f4 a[56:59], v[136:143], v[208:215], a[56:59]\n\t" \
  "ds_read_b128 v[224:227], v3 offset:16384\n\t" \
  "ds_read_b128 v[228:231], v3 offset:24576\n\t" \
  "v_mfma_f32_16x16x128_f8f6f4 a[52:55], v[136:143], v[200:207], a[52:55]\n\t" \
  "ds_read_b128 v[232:235], v3 offset:17408\n\t" \
  "ds_read_b128 v[236:239], v3 offset:25600\n\t" \
  "s_add_u32 s53, s52, 819200\n\t" \
  "s_add_u32 m0, s48, 114688\n\t" \
  "v_mfma_f32_16x16x128_f8f6f4 a[48:51], v[136:143], v[192:199], a[48:51]\n\t" \
  "buffer_load_dwordx4 v0, s[36:39], s53 offen sc0 sc1 lds\n\t" \
  "v_mfma_f32_16x16x128_f8f6f4 a[80:83], v[144:151], v[192:199], a[80:83]\n\t" \
  "buffer_load_dwordx4 v0, s[36:39], s53 offen offset:1024 sc0 sc1 lds\n\t" \
  "v_mfma_f32_16x16x128_f8f6f4 a[84:87], v[144:151], v[200:207], a[84:87]\n\t" \
  "buffer_load_dwordx4 v0, s[36:39], s53 offen offset:2048 sc0 sc1 lds\n\t" \
  "v_mfma_f32_16x16x128_f8f6f4 a[88:91], v[144:151], v[208:215], a[88:91]\n\t" \
  "buffer_load_dwordx4 v0, s[36:39], s53 offen offset:3072 sc0 sc1 lds\n\t" \
  "s_add_u32 s53, s52, 835584\n\t" \
  "s_add_u32 m0, s48, 32768\n\t" \
  "v_mfma_f32_16x16x128_f8f6f4 a[92:95], v[144:151], v[216:223], a[92:95]\n\t" \
  "buffer_load_dwordx4 v0, s[36:39], s53 offen sc0 sc1 lds\n\t" \
  "v_mfma_f32_16x16x128_f8f6f4 a[124:127], v[152:159], v[216:223], a[124:127]\n\t" \
  "buffer_load_dwordx4 v0, s[36:39], s53 offen offset:1024 sc0 sc1 lds\n\t" \
  "v_mfma_f32_16x16x128_f8f6f4 a[120:123], v[152:159], v[208:215], a[120:123]\n\t" \
  "buffer_load_dwordx4 v0, s[36:39], s53 offen offset:2048 sc0 sc1 lds\n\t" \
  "v_mfma_f32_16x16x128_f8f6f4 a[116:119], v[152:159], v[200:207], a[116:119]\n\t" \
  "buffer_load_dwordx4 v0, s[36:39], s53 offen offset:3072 sc0 sc1 lds\n\t" \
  "v_mfma_f32_16x16x128_f8f6f4 a[112:115], v[152:159], v[192:199], a[112:115]\n\t" \
  "s_waitcnt lgkmcnt(0)\n\t" \
  "v_mfma_f32_16x16x128_f8f6f4 a[144:147], v[160:167], v[192:199], a[144:147]\n\t" \
  "s_waitcnt vmcnt(28)\n\t" \
  "s_barrier\n\t" \
  "v_mfma_f32_16x16x128_f8f6f4 a[148:151], v[160:167], v[200:207], a[148:151]\n\t" \
  "ds_read_b128 v[128:131], v2 offset:0\n\t" \
  "ds_read_b128 v[132:135], v2 offset:8192\n\t" \
  "v_mfma_f32_16x16x128_f8f6f4 a[152:155], v[160:167], v[208:215], a[152:155]\n\t" \
  "ds_read_b128 v[136:139], v2 offset:1024\n\t" \
  "ds_read_b128 v[140:143], v2 offset:9216\n\t" \
  "v_mfma_f32_16x16x128_f8f6f4 a[156:159], v[160:167], v[216:223], a[156:159]\n\t" \
  "ds_read_b128 v[144:147], v2 offset:2048\n\t" \
  "ds_read_b128 v[148:151], v2 offset:10240\n\t" \
  "v_mfma_f32_16x16x128_f8f6f4 a[188:191], v[168:175], v[216:223], a[188:191]\n\t" \
  "ds_read_b128 v[152:155], v2 offset:3072\n\t" \
  "ds_read_b128 v[156:159], v2 offset:11264\n\t" \
  "v_mfma_f32_16x16x128_f8f6f4 a[184:187], v[168:175], v[208:215], a[184:187]\n\t" \
  "ds_read_b128 v[240:243], v3 offset:18432\n\t" \
  "ds_read_b128 v[244:247], v3 offset:26624\n\t" \
  "v_mfma_f32_16x16x128_f8f6f4 a[180:183], v[168:175], v[200:207], a[180:183]\n\t" \
  "ds_read_b128 v[248:251], v3 offset:19456\n\t" \
  "ds_read_b128 v[252:255], v3 offset:27648\n\t" \
  "v_mfma_f32_16x16x128_f8f6f4 a[176:179], v[168:175], v[192:199], a[176:179]\n\t" \
  "s_add_u32 s53, s51, 163840\n\t" \
  "s_add_u32 m0, s48, 49152\n\t" \
  "v_mfma_f32_16x16x128_f8f6f4 a[208:211], v[176:183], v[192:199], a[208:211]\n\t" \
  "buffer_load_dwordx4 v0, s[36:39], s53 offen sc0 sc1 lds\n\t" \
  "v_mfma_f32_16x16x128_f8f6f4 a[212:215], v[176:183], v[200:207], a[212:215]\n\t" \
  "v_mfma_f32_16x16x128_f8f6f4 a[216:219], v[176:183], v[208:215], a[216:219]\n\t" \
  "buffer_load_dwordx4 v0, s[36:39], s53 offen offset:1024 sc0 sc1 lds\n\t" \
  "v_mfma_f32_16x16x128_f8f6f4 a[220:223], v[176:183], v[216:223], a[220:223]\n\t" \
  "v_mfma_f32_16x16x128_f8f6f4 a[252:255], v[184:191], v[216:223], a[252:255]\n\t" \
  "buffer_load_dwordx4 v0, s[36:39], s53 offen offset:2048 sc0 sc1 lds\n\t" \
  "v_mfma_f32_16x16x128_f8f6f4 a[248:251], v[184:191], v[208:215], a[248:251]\n\t" \
  "v_mfma_f32_16x16x128_f8f6f4 a[244:247], v[184:191], v[200:207], a[244:247]\n\t" \
  "buffer_load_dwordx4 v0, s[36:39], s53 offen offset:3072 sc0 sc1 lds\n\t" \
  "v_mfma_f32_16x16x128_f8f6f4 a[240:243], v[184:191], v[192:199], a[240:243]\n\t" \
  "s_waitcnt lgkmcnt(0)\n\t" \
  "v_mfma_f32_16x16x128_f8f6f4 a[16:19], v[128:135], v[224:231], a[16:19]\n\t" \
  "s_waitcnt vmcnt(24)\n\t" \
  "s_barrier\n\t" \
  "v_mfma_f32_16x16x128_f8f6f4 a[20:23], v[128:135], v[232:239], a[20:23]\n\t" \
  "ds_read_b128 v[160:163], v2 offset:16384\n\t" \
  "ds_read_b128 v[164:167], v2 offset:24576\n\t" \
  "v_mfma_f32_16x16x128_f8f6f4 a[24:27], v[128:135], v[240:247], a[24:27]\n\t" \
  "ds_read_b128 v[168:171], v2 offset:17408\n\t" \
  "ds_read_b128 v[172:175], v2 offset:25600\n\t" \
  "v_mfma_f32_16x16x128_f8f6f4 a[28:31], v[128:135], v[248:255], a[28:31]\n\t" \
  "ds_read_b128 v[176:179], v2 offset:18432\n\t" \
  "ds_read_b128 v[180:183], v2 offset:26624\n\t" \
  "v_mfma_f32_16x16x128_f8f6f4 a[60:63], v[136:143], v[248:255], a[60:63]\n\t" \
  "ds_read_b128 v[184:187], v2 offset:19456\n\t" \
  "ds_read_b128 v[188:191], v2 offset:27648\n\t" \
  "v_mfma_f32_16x16x128_f8f6f4 a[56:59], v[136:143], v[240:247], a[56:59]\n\t" \
  "ds_read_b128 v[192:195], v4 offset:32768\n\t" \
  "ds_read_b128 v[196:199], v4 offset:40960\n\t" \
  "v_mfma_f32_16x16x128_f8f6f4 a[52:55], v[136:143], v[232:239], a[52:55]\n\t" \
  "ds_read_b128 v[200:203], v4 offset:33792\n\t" \
  "ds_read_b128 v[204:207], v4 offset:41984\n\t" \
  "s_add_u32 s53, s52, 851968\n\t" \
  "s_add_u32 m0, s48, 16384\n\t" \
  "v_mfma_f32_16x16x128_f8f6f4 a[48:51], v[136:143], v[224:231], a[48:51]\n\t" \
  "buffer_load_dwordx4 v0, s[36:39], s53 offen sc0 sc1 lds\n\t" \
  "v_mfma_f32_16x16x128_f8f6f4 a[80:83], v[144:151], v[224:231], a[80:83]\n\t" \
  "buffer_load_dwordx4 v0, s[36:39], s53 offen offset:1024 sc0 sc1 lds\n\t" \
  "v_mfma_f32_16x16x128_f8f6f4 a[84:87], v[144:151], v[232:239], a[84:87]\n\t" \
  "buffer_load_dwordx4 v0, s[36:39], s53 offen offset:2048 sc0 sc1 lds\n\t" \
  "v_mfma_f32_16x16x128_f8f6f4 a[88:91], v[144:151], v[240:247], a[88:91]\n\t" \
  "buffer_load_dwordx4 v0, s[36:39], s53 offen offset:3072 sc0 sc1 lds\n\t" \
  "s_add_u32 s53, s52, 868352\n\t" \
  "s_add_u32 m0, s48, 65536\n\t" \
  "v_mfma_f32_16x16x128_f8f6f4 a[92:95], v[144:151], v[248:255], a[92:95]\n\t" \
  "buffer_load_dwordx4 v0, s[36:39], s53 offen sc0 sc1 lds\n\t" \
  "v_mfma_f32_16x16x128_f8f6f4 a[124:127], v[152:159], v[248:255], a[124:127]\n\t" \
  "buffer_load_dwordx4 v0, s[36:39], s53 offen offset:1024 sc0 sc1 lds\n\t" \
  "v_mfma_f32_16x16x128_f8f6f4 a[120:123], v[152:159], v[240:247], a[120:123]\n\t" \
  "buffer_load_dwordx4 v0, s[36:39], s53 offen offset:2048 sc0 sc1 lds\n\t" \
  "v_mfma_f32_16x16x128_f8f6f4 a[116:119], v[152:159], v[232:239], a[116:119]\n\t" \
  "buffer_load_dwordx4 v0, s[36:39], s53 offen offset:3072 sc0 sc1 lds\n\t" \
  "v_mfma_f32_16x16x128_f8f6f4 a[112:115], v[152:159], v[224:231], a[112:115]\n\t" \
  "buffer_load_dword v7, v8, s[44:47], s76 offen sc0 sc1\n\t" \
  "s_waitcnt lgkmcnt(0)\n\t" \
  "v_mfma_f32_16x16x128_f8f6f4 a[144:147], v[160:167], v[224:231], a[144:147]\n\t" \
  "s_waitcnt vmcnt(29)\n\t" \
  "s_barrier\n\t" \
  "v_mfma_f32_16x16x128_f8f6f4 a[148:151], v[160:167], v[232:239], a[148:151]\n\t" \
  "ds_read_b128 v[128:131], v126 offset:0\n\t" \
  "ds_read_b128 v[132:135], v126 offset:8192\n\t" \
  "v_mfma_f32_16x16x128_f8f6f4 a[152:155], v[160:167], v[240:247], a[152:155]\n\t" \
  "ds_read_b128 v[136:139], v126 offset:1024\n\t" \
  "ds_read_b128 v[140:143], v126 offset:9216\n\t" \
  "v_mfma_f32_16x16x128_f8f6f4 a[156:159], v[160:167], v[248:255], a[156:159]\n\t" \
  "ds_read_b128 v[144:147], v126 offset:2048\n\t" \
  "ds_read_b128 v[148:151], v126 offset:10240\n\t" \
  "v_mfma_f32_16x16x128_f8f6f4 a[188:191], v[168:175], v[248:255], a[188:191]\n\t" \
  "ds_read_b128 v[152:155], v126 offset:3072\n\t" \
  "ds_read_b128 v[156:159], v126 offset:11264\n\t" \
  "v_mfma_f32_16x16x128_f8f6f4 a[184:187], v[168:175], v[240:247], a[184:187]\n\t" \
  "ds_read_b128 v[208:211], v4 offset:34816\n\t" \
  "ds_read_b128 v[212:215], v4 offset:43008\n\t" \
  "v_mfma_f32_16x16x128_f8f6f4 a[180:183], v[168:175], v[232:239], a[180:183]\n\t" \
  "ds_read_b128 v[216:219], v4 offset:35840\n\t" \
  "ds_read_b128 v[220:223], v4 offset:44032\n\t" \
  "v_mfma_f32_16x16x128_f8f6f4 a[176:179], v[168:175], v[224:231], a[176:179]\n\t" \
  "s_add_u32 s53, s51, 180224\n\t" \
  "s_add_u32 m0, s48, 81920\n\t" \
  "v_mfma_f32_16x16x128_f8f6f4 a[208:211], v[176:183], v[224:231], a[208:211]\n\t" \
  "buffer_load_dwordx4 v0, s[36:39], s53 offen sc0 sc1 lds\n\t" \
  "v_mfma_f32_16x16x128_f8f6f4 a[212:215], v[176:183], v[232:239], a[212:215]\n\t" \
  "v_mfma_f32_16x16x128_f8f6f4 a[216:219], v[176:183], v[240:247], a[216:219]\n\t" \
  "buffer_load_dwordx4 v0, s[36:39], s53 offen offset:1024 sc0 sc1 lds\n\t" \
  "v_mfma_f32_16x16x128_f8f6f4 a[220:223], v[176:183], v[248:255], a[220:223]\n\t" \
  "v_mfma_f32_16x16x128_f8f6f4 a[252:255], v[184:191], v[248:255], a[252:255]\n\t" \
  "buffer_load_dwordx4 v0, s[36:39], s53 offen offset:2048 sc0 sc1 lds\n\t" \
  "v_mfma_f32_16x16x128_f8f6f4 a[248:251], v[184:191], v[240:247], a[248:251]\n\t" \
  "v_mfma_f32_16x16x128_f8f6f4 a[244:247], v[184:191], v[232:239], a[244:247]\n\t" \
  "buffer_load_dwordx4 v0, s[36:39], s53 offen offset:3072 sc0 sc1 lds\n\t" \
  "v_mfma_f32_16x16x128_f8f6f4 a[240:243], v[184:191], v[224:231], a[240:243]\n\t" \
  "s_waitcnt lgkmcnt(0)\n\t" \
  "v_mfma_f32_16x16x128_f8f6f4 a[16:19], v[128:135], v[192:199], a[16:19]\n\t" \
  "s_waitcnt vmcnt(25)\n\t" \
  "s_barrier\n\t" \
  "v_mfma_f32_16x16x128_f8f6f4 a[20:23], v[128:135], v[200:207], a[20:23]\n\t" \
  "ds_read_b128 v[160:163], v126 offset:16384\n\t" \
  "ds_read_b128 v[164:167], v126 offset:24576\n\t" \
  "v_mfma_f32_16x16x128_f8f6f4 a[24:27], v[128:135], v[208:215], a[24:27]\n\t" \
  "ds_read_b128 v[168:171], v126 offset:17408\n\t" \
  "ds_read_b128 v[172:175], v126 offset:25600\n\t" \
  "v_mfma_f32_16x16x128_f8f6f4 a[28:31], v[128:135], v[216:223], a[28:31]\n\t" \
  "ds_read_b128 v[176:179], v126 offset:18432\n\t" \
  "ds_read_b128 v[180:183], v126 offset:26624\n\t" \
  "v_mfma_f32_16x16x128_f8f6f4 a[60:63], v[136:143], v[216:223], a[60:63]\n\t" \
  "ds_read_b128 v[184:187], v126 offset:19456\n\t" \
  "ds_read_b128 v[188:191], v126 offset:27648\n\t" \
  "v_mfma_f32_16x16x128_f8f6f4 a[56:59], v[136:143], v[208:215], a[56:59]\n\t" \
  "ds_read_b128 v[224:227], v3 offset:0\n\t" \
  "ds_read_b128 v[228:231], v3 offset:8192\n\t" \
  "v_mfma_f32_16x16x128_f8f6f4 a[52:55], v[136:143], v[200:207], a[52:55]\n\t" \
  "ds_read_b128 v[232:235], v3 offset:1024\n\t" \
  "ds_read_b128 v[236:239], v3 offset:9216\n\t" \
  "s_add_u32 s53, s52, 884736\n\t" \
  "s_add_u32 m0, s48, 98304\n\t" \
  "v_mfma_f32_16x16x128_f8f6f4 a[48:51], v[136:143], v[192:199], a[48:51]\n\t" \
  "buffer_load_dwordx4 v0, s[36:39], s53 offen sc0 sc1 lds\n\t" \
  "v_mfma_f32_16x16x128_f8f6f4 a[80:83], v[144:151], v[192:199], a[80:83]\n\t" \
  "buffer_load_dwordx4 v0, s[36:39], s53 offen offset:1024 sc0 sc1 lds\n\t" \
  "v_mfma_f32_16x16x128_f8f6f4 a[84:87], v[144:151], v[200:207], a[84:87]\n\t" \
  "buffer_load_dwordx4 v0, s[36:39], s53 offen offset:2048 sc0 sc1 lds\n\t" \
  "v_mfma_f32_16x16x128_f8f6f4 a[88:91], v[144:151], v[208:215], a[88:91]\n\t" \
  "buffer_load_dwordx4 v0, s[36:39], s53 offen offset:3072 sc0 sc1 lds\n\t" \
  "s_add_u32 s53, s52, 901120\n\t" \
  "s_add_u32 m0, s48, 131072\n\t" \
  "v_mfma_f32_16x16x128_f8f6f4 a[92:95], v[144:151], v[216:223], a[92:95]\n\t" \
  "buffer_load_dwordx4 v0, s[36:39], s53 offen sc0 sc1 lds\n\t" \
  "v_mfma_f32_16x16x128_f8f6f4 a[124:127], v[152:159], v[216:223], a[124:127]\n\t" \
  "buffer_load_dwordx4 v0, s[36:39], s53 offen offset:1024 sc0 sc1 lds\n\t" \
  "v_mfma_f32_16x16x128_f8f6f4 a[120:123], v[152:159], v[208:215], a[120:123]\n\t" \
  "buffer_load_dwordx4 v0, s[36:39], s53 offen offset:2048 sc0 sc1 lds\n\t" \
  "v_mfma_f32_16x16x128_f8f6f4 a[116:119], v[152:159], v[200:207], a[116:119]\n\t" \
  "buffer_load_dwordx4 v0, s[36:39], s53 offen offset:3072 sc0 sc1 lds\n\t" \
  "v_mfma_f32_16x16x128_f8f6f4 a[112:115], v[152:159], v[192:199], a[112:115]\n\t" \
  "s_waitcnt vmcnt(12)\n\t" \
  "v_cmp_le_u32_e32 vcc, s75, v7\n\t" \
  "s_nop 1\n\t" \
  "s_cmp_eq_u64 vcc, -1\n\t" \
  "s_cbranch_scc1 L_OKN13_%=\n\t" \
  "s_swappc_b64 s[80:81], s[78:79]\n\t" \
  "L_OKN13_%=:\n\t" \
  "s_add_u32 s76, s76, 1024\n\t" \
  "s_waitcnt lgkmcnt(0)\n\t" \
  "v_mfma_f32_16x16x128_f8f6f4 a[144:147], v[160:167], v[192:199], a[144:147]\n\t" \
  "s_waitcnt vmcnt(29)\n\t" \
  "s_barrier\n\t" \
  "v_mfma_f32_16x16x128_f8f6f4 a[148:151], v[160:167], v[200:207], a[148:151]\n\t" \
  "ds_read_b128 v[128:131], v2 offset:49152\n\t" \
  "ds_read_b128 v[132:135], v2 offset:57344\n\t" \
  "v_mfma_f32_16x16x128_f8f6f4 a[152:155], v[160:167], v[208:215], a[152:155]\n\t" \
  "ds_read_b128 v[136:139], v2 offset:50176\n\t" \
  "ds_read_b128 v[140:143], v2 offset:58368\n\t" \
  "v_mfma_f32_16x16x128_f8f6f4 a[156:159], v[160:167], v[216:223], a[156:159]\n\t" \
  "ds_read_b128 v[144:147], v2 offset:51200\n\t" \
  "ds_read_b128 v[148:151], v2 offset:59392\n\t" \
  "v_mfma_f32_16x16x128_f8f6f4 a[188:191], v[168:175], v[216:223], a[188:191]\n\t" \
  "ds_read_b128 v[152:155], v2 offset:52224\n\t" \
  "ds_read_b128 v[156:159], v2 offset:60416\n\t" \
  "v_mfma_f32_16x16x128_f8f6f4 a[184:187], v[168:175], v[208:215], a[184:187]\n\t" \
  "ds_read_b128 v[240:243], v3 offset:2048\n\t" \
  "ds_read_b128 v[244:247], v3 offset:10240\n\t" \
  "v_mfma_f32_16x16x128_f8f6f4 a[180:183], v[168:175], v[200:207], a[180:183]\n\t" \
  "ds_read_b128 v[248:251], v3 offset:3072\n\t" \
  "ds_read_b128 v[252:255], v3 offset:11264\n\t" \
  "v_mfma_f32_16x16x128_f8f6f4 a[176:179], v[168:175], v[192:199], a[176:179]\n\t" \
  "s_add_u32 s53, s51, 196608\n\t" \
  "s_add_u32 m0, s48, 147456\n\t" \
  "v_mfma_f32_16x16x128_f8f6f4 a[208:211], v[176:183], v[192:199], a[208:211]\n\t" \
  "buffer_load_dwordx4 v0, s[36:39], s53 offen sc0 sc1 lds\n\t" \
  "v_mfma_f32_16x16x128_f8f6f4 a[212:215], v[176:183], v[200:207], a[212:215]\n\t" \
  "v_mfma_f32_16x16x128_f8f6f4 a[216:219], v[176:183], v[208:215], a[216:219]\n\t" \
  "buffer_load_dwordx4 v0, s[36:39], s53 offen offset:1024 sc0 sc1 lds\n\t" \
  "v_mfma_f32_16x16x128_f8f6f4 a[220:223], v[176:183], v[216:223], a[220:223]\n\t" \
  "v_mfma_f32_16x16x128_f8f6f4 a[252:255], v[184:191], v[216:223], a[252:255]\n\t" \
  "buffer_load_dwordx4 v0, s[36:39], s53 offen offset:2048 sc0 sc1 lds\n\t" \
  "v_mfma_f32_16x16x128_f8f6f4 a[248:251], v[184:191], v[208:215], a[248:251]\n\t" \
  "v_mfma_f32_16x16x128_f8f6f4 a[244:247], v[184:191], v[200:207], a[244:247]\n\t" \
  "buffer_load_dwordx4 v0, s[36:39], s53 offen offset:3072 sc0 sc1 lds\n\t" \
  "v_mfma_f32_16x16x128_f8f6f4 a[240:243], v[184:191], v[192:199], a[240:243]\n\t" \
  "s_waitcnt lgkmcnt(0)\n\t" \
  "v_mfma_f32_16x16x128_f8f6f4 a[16:19], v[128:135], v[224:231], a[16:19]\n\t" \
  "s_waitcnt vmcnt(25)\n\t" \
  "s_barrier\n\t" \
  "v_mfma_f32_16x16x128_f8f6f4 a[20:23], v[128:135], v[232:239], a[20:23]\n\t" \
  "ds_read_b128 v[160:163], v1 offset:32768\n\t" \
  "ds_read_b128 v[164:167], v1 offset:40960\n\t" \
  "v_mfma_f32_16x16x128_f8f6f4 a[24:27], v[128:135], v[240:247], a[24:27]\n\t" \
  "ds_read_b128 v[168:171], v1 offset:33792\n\t" \
  "ds_read_b128 v[172:175], v1 offset:41984\n\t" \
  "v_mfma_f32_16x16x128_f8f6f4 a[28:31], v[128:135], v[248:255], a[28:31]\n\t" \
  "ds_read_b128 v[176:179], v1 offset:34816\n\t" \
  "ds_read_b128 v[180:183], v1 offset:43008\n\t" \
  "v_mfma_f32_16x16x128_f8f6f4 a[60:63], v[136:143], v[248:255], a[60:63]\n\t" \
  "ds_read_b128 v[184:187], v1 offset:35840\n\t" \
  "ds_read_b128 v[188:191], v1 offset:44032\n\t" \
  "v_mfma_f32_16x16x128_f8f6f4 a[56:59], v[136:143], v[240:247], a[56:59]\n\t" \
  "ds_read_b128 v[192:195], v3 offset:49152\n\t" \
  "ds_read_b128 v[196:199], v3 offset:57344\n\t" \
  "v_mfma_f32_16x16x128_f8f6f4 a[52:55], v[136:143], v[232:239], a[52:55]\n\t" \
  "ds_read_b128 v[200:203], v3 offset:50176\n\t" \
  "ds_read_b128 v[204:207], v3 offset:58368\n\t" \
  "s_add_u32 s53, s52, 917504\n\t" \
  "s_add_u32 m0, s48, 114688\n\t" \
  "v_mfma_f32_16x16x128_f8f6f4 a[48:51], v[136:143], v[224:231], a[48:51]\n\t" \
  "buffer_load_dwordx4 v0, s[36:39], s53 offen sc0 sc1 lds\n\t" \
  "v_mfma_f32_16x16x128_f8f6f4 a[80:83], v[144:151], v[224:231], a[80:83]\n\t" \
  "buffer_load_dwordx4 v0, s[36:39], s53 offen offset:1024 sc0 sc1 lds\n\t" \
  "v_mfma_f32_16x16x128_f8f6f4 a[84:87], v[144:151], v[232:239], a[84:87]\n\t" \
  "buffer_load_dwordx4 v0, s[36:39], s53 offen offset:2048 sc0 sc1 lds\n\t" \
  "v_mfma_f32_16x16x128_f8f6f4 a[88:91], v[144:151], v[240:247], a[88:91]\n\t" \
  "buffer_load_dwordx4 v0, s[36:39], s53 offen offset:3072 sc0 sc1 lds\n\t" \
  "s_add_u32 s53, s52, 933888\n\t" \
  "s_mov_b32 m0, s48\n\t" \
  "v_mfma_f32_16x16x128_f8f6f4 a[92:95], v[144:151], v[248:255], a[92:95]\n\t" \
  "buffer_load_dwordx4 v0, s[36:39], s53 offen sc0 sc1 lds\n\t" \
  "v_mfma_f32_16x16x128_f8f6f4 a[124:127], v[152:159], v[248:255], a[124:127]\n\t" \
  "buffer_load_dwordx4 v0, s[36:39], s53 offen offset:1024 sc0 sc1 lds\n\t" \
  "v_mfma_f32_16x16x128_f8f6f4 a[120:123], v[152:159], v[240:247], a[120:123]\n\t" \
  "buffer_load_dwordx4 v0, s[36:39], s53 offen offset:2048 sc0 sc1 lds\n\t" \
  "v_mfma_f32_16x16x128_f8f6f4 a[116:119], v[152:159], v[232:239], a[116:119]\n\t" \
  "buffer_load_dwordx4 v0, s[36:39], s53 offen offset:3072 sc0 sc1 lds\n\t" \
  "v_mfma_f32_16x16x128_f8f6f4 a[112:115], v[152:159], v[224:231], a[112:115]\n\t" \
  "s_waitcnt lgkmcnt(0)\n\t" \
  "v_mfma_f32_16x16x128_f8f6f4 a[144:147], v[160:167], v[224:231], a[144:147]\n\t" \
  "s_waitcnt vmcnt(29)\n\t" \
  "s_barrier\n\t" \
  "v_mfma_f32_16x16x128_f8f6f4 a[148:151], v[160:167], v[232:239], a[148:151]\n\t" \
  "ds_read_b128 v[128:131], v1 offset:16384\n\t" \
  "ds_read_b128 v[132:135], v1 offset:24576\n\t" \
  "v_mfma_f32_16x16x128_f8f6f4 a[152:155], v[160:167], v[240:247], a[152:155]\n\t" \
  "ds_read_b128 v[136:139], v1 offset:17408\n\t" \
  "ds_read_b128 v[140:143], v1 offset:25600\n\t" \
  "v_mfma_f32_16x16x128_f8f6f4 a[156:159], v[160:167], v[248:255], a[156:159]\n\t" \
  "ds_read_b128 v[144:147], v1 offset:18432\n\t" \
  "ds_read_b128 v[148:151], v1 offset:26624\n\t" \
  "v_mfma_f32_16x16x128_f8f6f4 a[188:191], v[168:175], v[248:255], a[188:191]\n\t" \
  "ds_read_b128 v[152:155], v1 offset:19456\n\t" \
  "ds_read_b128 v[156:159], v1 offset:27648\n\t" \
  "v_mfma_f32_16x16x128_f8f6f4 a[184:187], v[168:175], v[240:247], a[184:187]\n\t" \
  "ds_read_b128 v[208:211], v3 offset:51200\n\t" \
  "ds_read_b128 v[212:215], v3 offset:59392\n\t" \
  "v_mfma_f32_16x16x128_f8f6f4 a[180:183], v[168:175], v[232:239], a[180:183]\n\t" \
  "ds_read_b128 v[216:219], v3 offset:52224\n\t" \
  "ds_read_b128 v[220:223], v3 offset:60416\n\t" \
  "v_mfma_f32_16x16x128_f8f6f4 a[176:179], v[168:175], v[224:231], a[176:179]\n\t" \
  "s_add_u32 s53, s51, 212992\n\t" \
  "s_add_u32 m0, s48, 32768\n\t" \
  "v_mfma_f32_16x16x128_f8f6f4 a[208:211], v[176:183], v[224:231], a[208:211]\n\t" \
  "buffer_load_dwordx4 v0, s[36:39], s53 offen sc0 sc1 lds\n\t" \
  "v_mfma_f32_16x16x128_f8f6f4 a[212:215], v[176:183], v[232:239], a[212:215]\n\t" \
  "v_mfma_f32_16x16x128_f8f6f4 a[216:219], v[176:183], v[240:247], a[216:219]\n\t" \
  "buffer_load_dwordx4 v0, s[36:39], s53 offen offset:1024 sc0 sc1 lds\n\t" \
  "v_mfma_f32_16x16x128_f8f6f4 a[220:223], v[176:183], v[248:255], a[220:223]\n\t" \
  "v_mfma_f32_16x16x128_f8f6f4 a[252:255], v[184:191], v[248:255], a[252:255]\n\t" \
  "buffer_load_dwordx4 v0, s[36:39], s53 offen offset:2048 sc0 sc1 lds\n\t" \
  "v_mfma_f32_16x16x128_f8f6f4 a[248:251], v[184:191], v[240:247], a[248:251]\n\t" \
  "v_mfma_f32_16x16x128_f8f6f4 a[244:247], v[184:191], v[232:239], a[244:247]\n\t" \
  "buffer_load_dwordx4 v0, s[36:39], s53 offen offset:3072 sc0 sc1 lds\n\t" \
  "v_mfma_f32_16x16x128_f8f6f4 a[240:243], v[184:191], v[224:231], a[240:243]\n\t" \
  "s_waitcnt lgkmcnt(0)\n\t" \
  "v_mfma_f32_16x16x128_f8f6f4 a[16:19], v[128:135], v[192:199], a[16:19]\n\t" \
  "s_waitcnt vmcnt(24)\n\t" \
  "s_barrier\n\t" \
  "v_mfma_f32_16x16x128_f8f6f4 a[20:23], v[128:135], v[200:207], a[20:23]\n\t" \
  "ds_read_b128 v[160:163], v2 offset:0\n\t" \
  "ds_read_b128 v[164:167], v2 offset:8192\n\t" \
  "v_mfma_f32_16x16x128_f8f6f4 a[24:27], v[128:135], v[208:215], a[24:27]\n\t" \
  "ds_read_b128 v[168:171], v2 offset:1024\n\t" \
  "ds_read_b128 v[172:175], v2 offset:9216\n\t" \
  "v_mfma_f32_16x16x128_f8f6f4 a[28:31], v[128:135], v[216:223], a[28:31]\n\t" \
  "ds_read_b128 v[176:179], v2 offset:2048\n\t" \
  "ds_read_b128 v[180:183], v2 offset:10240\n\t" \
  "v_mfma_f32_16x16x128_f8f6f4 a[60:63], v[136:143], v[216:223], a[60:63]\n\t" \
  "ds_read_b128 v[184:187], v2 offset:3072\n\t" \
  "ds_read_b128 v[188:191], v2 offset:11264\n\t" \
  "v_mfma_f32_16x16x128_f8f6f4 a[56:59], v[136:143], v[208:215], a[56:59]\n\t" \
  "ds_read_b128 v[224:227], v4 offset:16384\n\t" \
  "ds_read_b128 v[228:231], v4 offset:24576\n\t" \
  "v_mfma_f32_16x16x128_f8f6f4 a[52:55], v[136:143], v[200:207], a[52:55]\n\t" \
  "ds_read_b128 v[232:235], v4 offset:17408\n\t" \
  "ds_read_b128 v[236:239], v4 offset:25600\n\t" \
  "s_add_u32 s53, s52, 950272\n\t" \
  "s_add_u32 m0, s48, 16384\n\t" \
  "v_mfma_f32_16x16x128_f8f6f4 a[48:51], v[136:143], v[192:199], a[48:51]\n\t" \
  "buffer_load_dwordx4 v0, s[36:39], s53 offen sc0 sc1 lds\n\t" \
  "v_mfma_f32_16x16x128_f8f6f4 a[80:83], v[144:151], v[192:199], a[80:83]\n\t" \
  "buffer_load_dwordx4 v0, s[36:39], s53 offen offset:1024 sc0 sc1 lds\n\t" \
  "v_mfma_f32_16x16x128_f8f6f4 a[84:87], v[144:151], v[200:207], a[84:87]\n\t" \
  "buffer_load_dwordx4 v0, s[36:39], s53 offen offset:2048 sc0 sc1 lds\n\t" \
  "v_mfma_f32_16x16x128_f8f6f4 a[88:91], v[144:151], v[208:215], a[88:91]\n\t" \
  "buffer_load_dwordx4 v0, s[36:39], s53 offen offset:3072 sc0 sc1 lds\n\t" \
  "s_add_u32 s53, s52, 966656\n\t" \
  "s_add_u32 m0, s48, 49152\n\t" \
  "v_mfma_f32_16x16x128_f8f6f4 a[92:95], v[144:151], v[216:223], a[92:95]\n\t" \
  "buffer_load_dwordx4 v0, s[36:39], s53 offen sc0 sc1 lds\n\t" \
  "v_mfma_f32_16x16x128_f8f6f4 a[124:127], v[152:159], v[216:223], a[124:127]\n\t" \
  "buffer_load_dwordx4 v0, s[36:39], s53 offen offset:1024 sc0 sc1 lds\n\t" \
  "v_mfma_f32_16x16x128_f8f6f4 a[120:123], v[152:159], v[208:215], a[120:123]\n\t" \
  "buffer_load_dwordx4 v0, s[36:39], s53 offen offset:2048 sc0 sc1 lds\n\t" \
  "v_mfma_f32_16x16x128_f8f6f4 a[116:119], v[152:159], v[200:207], a[116:119]\n\t" \
  "buffer_load_dwordx4 v0, s[36:39], s53 offen offset:3072 sc0 sc1 lds\n\t" \
  "v_mfma_f32_16x16x128_f8f6f4 a[112:115], v[152:159], v[192:199], a[112:115]\n\t" \
  "s_waitcnt lgkmcnt(0)\n\t" \
  "v_mfma_f32_16x16x128_f8f6f4 a[144:147], v[160:167], v[192:199], a[144:147]\n\t" \
  "s_waitcnt vmcnt(28)\n\t" \
  "s_barrier\n\t" \
  "v_mfma_f32_16x16x128_f8f6f4 a[148:151], v[160:167], v[200:207], a[148:151]\n\t" \
  "ds_read_b128 v[128:131], v2 offset:32768\n\t" \
  "ds_read_b128 v[132:135], v2 offset:40960\n\t" \
  "v_mfma_f32_16x16x128_f8f6f4 a[152:155], v[160:167], v[208:215], a[152:155]\n\t" \
  "ds_read_b128 v[136:139], v2 offset:33792\n\t" \
  "ds_read_b128 v[140:143], v2 offset:41984\n\t" \
  "v_mfma_f32_16x16x128_f8f6f4 a[156:159], v[160:167], v[216:223], a[156:159]\n\t" \
  "ds_read_b128 v[144:147], v2 offset:34816\n\t" \
  "ds_read_b128 v[148:151], v2 offset:43008\n\t" \
  "v_mfma_f32_16x16x128_f8f6f4 a[188:191], v[168:175], v[216:223], a[188:191]\n\t" \
  "ds_read_b128 v[152:155], v2 offset:35840\n\t" \
  "ds_read_b128 v[156:159], v2 offset:44032\n\t" \
  "v_mfma_f32_16x16x128_f8f6f4 a[184:187], v[168:175], v[208:215], a[184:187]\n\t" \
  "ds_read_b128 v[240:243], v4 offset:18432\n\t" \
  "ds_read_b128 v[244:247], v4 offset:26624\n\t" \
  "v_mfma_f32_16x16x128_f8f6f4 a[180:183], v[168:175], v[200:207], a[180:183]\n\t" \
  "ds_read_b128 v[248:251], v4 offset:19456\n\t" \
  "ds_read_b128 v[252:255], v4 offset:27648\n\t" \
  "v_mfma_f32_16x16x128_f8f6f4 a[176:179], v[168:175], v[192:199], a[176:179]\n\t" \
  "s_add_u32 s53, s51, 229376\n\t" \
  "s_add_u32 m0, s48, 65536\n\t" \
  "v_mfma_f32_16x16x128_f8f6f4 a[208:211], v[176:183], v[192:199], a[208:211]\n\t" \
  "buffer_load_dwordx4 v0, s[36:39], s53 offen sc0 sc1 lds\n\t" \
  "v_mfma_f32_16x16x128_f8f6f4 a[212:215], v[176:183], v[200:207], a[212:215]\n\t" \
  "v_mfma_f32_16x16x128_f8f6f4 a[216:219], v[176:183], v[208:215], a[216:219]\n\t" \
  "buffer_load_dwordx4 v0, s[36:39], s53 offen offset:1024 sc0 sc1 lds\n\t" \
  "v_mfma_f32_16x16x128_f8f6f4 a[220:223], v[176:183], v[216:223], a[220:223]\n\t" \
  "v_mfma_f32_16x16x128_f8f6f4 a[252:255], v[184:191], v[216:223], a[252:255]\n\t" \
  "buffer_load_dwordx4 v0, s[36:39], s53 offen offset:2048 sc0 sc1 lds\n\t" \
  "v_mfma_f32_16x16x128_f8f6f4 a[248:251], v[184:191], v[208:215], a[248:251]\n\t" \
  "v_mfma_f32_16x16x128_f8f6f4 a[244:247], v[184:191], v[200:207], a[244:247]\n\t" \
  "buffer_load_dwordx4 v0, s[36:39], s53 offen offset:3072 sc0 sc1 lds\n\t" \
  "v_mfma_f32_16x16x128_f8f6f4 a[240:243], v[184:191], v[192:199], a[240:243]\n\t" \
  "s_waitcnt lgkmcnt(0)\n\t" \
  "v_mfma_f32_16x16x128_f8f6f4 a[16:19], v[128:135], v[224:231], a[16:19]\n\t" \
  "s_waitcnt vmcnt(24)\n\t" \
  "s_barrier\n\t" \
  "v_mfma_f32_16x16x128_f8f6f4 a[20:23], v[128:135], v[232:239], a[20:23]\n\t" \
  "ds_read_b128 v[160:163], v126 offset:0\n\t" \
  "ds_read_b128 v[164:167], v126 offset:8192\n\t" \
  "v_mfma_f32_16x16x128_f8f6f4 a[24:27], v[128:135], v[240:247], a[24:27]\n\t" \
  "ds_read_b128 v[168:171], v126 offset:1024\n\t" \
  "ds_read_b128 v[172:175], v126 offset:9216\n\t" \
  "v_mfma_f32_16x16x128_f8f6f4 a[28:31], v[128:135], v[248:255], a[28:31]\n\t" \
  "ds_read_b128 v[176:179], v126 offset:2048\n\t" \
  "ds_read_b128 v[180:183], v126 offset:10240\n\t" \
  "v_mfma_f32_16x16x128_f8f6f4 a[60:63], v[136:143], v[248:255], a[60:63]\n\t" \
  "ds_read_b128 v[184:187], v126 offset:3072\n\t" \
  "ds_read_b128 v[188:191], v126 offset:11264\n\t" \
  "v_mfma_f32_16x16x128_f8f6f4 a[56:59], v[136:143], v[240:247], a[56:59]\n\t" \
  "ds_read_b128 v[192:195], v127 offset:16384\n\t" \
  "ds_read_b128 v[196:199], v127 offset:24576\n\t" \
  "v_mfma_f32_16x16x128_f8f6f4 a[52:55], v[136:143], v[232:239], a[52:55]\n\t" \
  "ds_read_b128 v[200:203], v127 offset:17408\n\t" \
  "ds_read_b128 v[204:207], v127 offset:25600\n\t" \
  "s_add_u32 s53, s52, 983040\n\t" \
  "s_add_u32 m0, s48, 81920\n\t" \
  "v_mfma_f32_16x16x128_f8f6f4 a[48:51], v[136:143], v[224:231], a[48:51]\n\t" \
  "buffer_load_dwordx4 v0, s[36:39], s53 offen sc0 sc1 lds\n\t" \
  "v_mfma_f32_16x16x128_f8f6f4 a[80:83], v[144:151], v[224:231], a[80:83]\n\t" \
  "buffer_load_dwordx4 v0, s[36:39], s53 offen offset:1024 sc0 sc1 lds\n\t" \
  "v_mfma_f32_16x16x128_f8f6f4 a[84:87], v[144:151], v[232:239], a[84:87]\n\t" \
  "buffer_load_dwordx4 v0, s[36:39], s53 offen offset:2048 sc0 sc1 lds\n\t" \
  "v_mfma_f32_16x16x128_f8f6f4 a[88:91], v[144:151], v[240:247], a[88:91]\n\t" \
  "buffer_load_dwordx4 v0, s[36:39], s53 offen offset:3072 sc0 sc1 lds\n\t" \
  "s_add_u32 s53, s52, 999424\n\t" \
  "s_add_u32 m0, s48, 98304\n\t" \
  "v_mfma_f32_16x16x128_f8f6f4 a[92:95], v[144:151], v[248:255], a[92:95]\n\t" \
  "buffer_load_dwordx4 v0, s[36:39], s53 offen sc0 sc1 lds\n\t" \
  "v_mfma_f32_16x16x128_f8f6f4 a[124:127], v[152:159], v[248:255], a[124:127]\n\t" \
  "buffer_load_dwordx4 v0, s[36:39], s53 offen offset:1024 sc0 sc1 lds\n\t" \
  "v_mfma_f32_16x16x128_f8f6f4 a[120:123], v[152:159], v[240:247], a[120:123]\n\t" \
  "buffer_load_dwordx4 v0, s[36:39], s53 offen offset:2048 sc0 sc1 lds\n\t" \
  "v_mfma_f32_16x16x128_f8f6f4 a[116:119], v[152:159], v[232:239], a[116:119]\n\t" \
  "buffer_load_dwordx4 v0, s[36:39], s53 offen offset:3072 sc0 sc1 lds\n\t" \
  "v_and_b32_e32 v8, 0x3ff, v8\n\t" \
  "v_add_u32_e32 v8, s92, v8\n\t" \
  "v_mfma_f32_16x16x128_f8f6f4 a[112:115], v[152:159], v[224:231], a[112:115]\n\t" \
  "buffer_load_dword v7, v8, s[44:47], s76 offen sc0 sc1\n\t" \
  "s_waitcnt lgkmcnt(0)\n\t" \
  "v_mfma_f32_16x16x128_f8f6f4 a[144:147], v[160:167], v[224:231], a[144:147]\n\t" \
  "s_waitcnt vmcnt(29)\n\t" \
  "s_barrier\n\t" \
  "v_mfma_f32_16x16x128_f8f6f4 a[148:151], v[160:167], v[232:239], a[148:151]\n\t" \
  "ds_read_b128 v[128:131], v2 offset:49152\n\t" \
  "ds_read_b128 v[132:135], v2 offset:57344\n\t" \
  "v_mfma_f32_16x16x128_f8f6f4 a[152:155], v[160:167], v[240:247], a[152:155]\n\t" \
  "ds_read_b128 v[136:139], v2 offset:50176\n\t" \
  "ds_read_b128 v[140:143], v2 offset:58368\n\t" \
  "v_mfma_f32_16x16x128_f8f6f4 a[156:159], v[160:167], v[248:255], a[156:159]\n\t" \
  "ds_read_b128 v[144:147], v2 offset:51200\n\t" \
  "ds_read_b128 v[148:151], v2 offset:59392\n\t" \
  "v_mfma_f32_16x16x128_f8f6f4 a[188:191], v[168:175], v[248:255], a[188:191]\n\t" \
  "ds_read_b128 v[152:155], v2 offset:52224\n\t" \
  "ds_read_b128 v[156:159], v2 offset:60416\n\t" \
  "v_mfma_f32_16x16x128_f8f6f4 a[184:187], v[168:175], v[240:247], a[184:187]\n\t" \
  "ds_read_b128 v[208:211], v127 offset:18432\n\t" \
  "ds_read_b128 v[212:215], v127 offset:26624\n\t" \
  "v_mfma_f32_16x16x128_f8f6f4 a[180:183], v[168:175], v[232:239], a[180:183]\n\t" \
  "ds_read_b128 v[216:219], v127 offset:19456\n\t" \
  "ds_read_b128 v[220:223], v127 offset:27648\n\t" \
  "v_mfma_f32_16x16x128_f8f6f4 a[176:179], v[168:175], v[224:231], a[176:179]\n\t" \
  "s_add_u32 s53, s51, 245760\n\t" \
  "s_add_u32 m0, s48, 131072\n\t" \
  "v_mfma_f32_16x16x128_f8f6f4 a[208:211], v[176:183], v[224:231], a[208:211]\n\t" \
  "buffer_load_dwordx4 v0, s[36:39], s53 offen sc0 sc1 lds\n\t" \
  "v_mfma_f32_16x16x128_f8f6f4 a[212:215], v[176:183], v[232:239], a[212:215]\n\t" \
  "v_mfma_f32_16x16x128_f8f6f4 a[216:219], v[176:183], v[240:247], a[216:219]\n\t" \
  "buffer_load_dwordx4 v0, s[36:39], s53 offen offset:1024 sc0 sc1 lds\n\t" \
  "v_mfma_f32_16x16x128_f8f6f4 a[220:223], v[176:183], v[248:255], a[220:223]\n\t" \
  "v_mfma_f32_16x16x128_f8f6f4 a[252:255], v[184:191], v[248:255], a[252:255]\n\t" \
  "buffer_load_dwordx4 v0, s[36:39], s53 offen offset:2048 sc0 sc1 lds\n\t" \
  "v_mfma_f32_16x16x128_f8f6f4 a[248:251], v[184:191], v[240:247], a[248:251]\n\t" \
  "v_mfma_f32_16x16x128_f8f6f4 a[244:247], v[184:191], v[232:239], a[244:247]\n\t" \
  "buffer_load_dwordx4 v0, s[36:39], s53 offen offset:3072 sc0 sc1 lds\n\t" \
  "v_mfma_f32_16x16x128_f8f6f4 a[240:243], v[184:191], v[224:231], a[240:243]\n\t" \
  "s_waitcnt lgkmcnt(0)\n\t" \
  "v_mfma_f32_16x16x128_f8f6f4 a[16:19], v[128:135], v[192:199], a[16:19]\n\t" \
  "s_waitcnt vmcnt(25)\n\t" \
  "s_barrier\n\t" \
  "v_mfma_f32_16x16x128_f8f6f4 a[20:23], v[128:135], v[200:207], a[20:23]\n\t" \
  "ds_read_b128 v[160:163], v1 offset:0\n\t" \
  "ds_read_b128 v[164:167], v1 offset:8192\n\t" \
  "v_mfma_f32_16x16x128_f8f6f4 a[24:27], v[128:135], v[208:215], a[24:27]\n\t" \
  "ds_read_b128 v[168:171], v1 offset:1024\n\t" \
  "ds_read_b128 v[172:175], v1 offset:9216\n\t" \
  "v_mfma_f32_16x16x128_f8f6f4 a[28:31], v[128:135], v[216:223], a[28:31]\n\t" \
  "ds_read_b128 v[176:179], v1 offset:2048\n\t" \
  "ds_read_b128 v[180:183], v1 offset:10240\n\t" \
  "v_mfma_f32_16x16x128_f8f6f4 a[60:63], v[136:143], v[216:223], a[60:63]\n\t" \
  "ds_read_b128 v[184:187], v1 offset:3072\n\t" \
  "ds_read_b128 v[188:191], v1 offset:11264\n\t" \
  "v_mfma_f32_16x16x128_f8f6f4 a[56:59], v[136:143], v[208:215], a[56:59]\n\t" \
  "ds_read_b128 v[224:227], v3 offset:32768\n\t" \
  "ds_read_b128 v[228:231], v3 offset:40960\n\t" \
  "v_mfma_f32_16x16x128_f8f6f4 a[52:55], v[136:143], v[200:207], a[52:55]\n\t" \
  "ds_read_b128 v[232:235], v3 offset:33792\n\t" \
  "ds_read_b128 v[236:239], v3 offset:41984\n\t" \
  "s_add_u32 s53, s52, 1015808\n\t" \
  "s_add_u32 m0, s48, 114688\n\t" \
  "v_mfma_f32_16x16x128_f8f6f4 a[48:51], v[136:143], v[192:199], a[48:51]\n\t" \
  "buffer_load_dwordx4 v0, s[36:39], s53 offen sc0 sc1 lds\n\t" \
  "v_mfma_f32_16x16x128_f8f6f4 a[80:83], v[144:151], v[192:199], a[80:83]\n\t" \
  "buffer_load_dwordx4 v0, s[36:39], s53 offen offset:1024 sc0 sc1 lds\n\t" \
  "v_mfma_f32_16x16x128_f8f6f4 a[84:87], v[144:151], v[200:207], a[84:87]\n\t" \
  "buffer_load_dwordx4 v0, s[36:39], s53 offen offset:2048 sc0 sc1 lds\n\t" \
  "v_mfma_f32_16x16x128_f8f6f4 a[88:91], v[144:151], v[208:215], a[88:91]\n\t" \
  "buffer_load_dwordx4 v0, s[36:39], s53 offen offset:3072 sc0 sc1 lds\n\t" \
  "s_add_u32 s53, s52, 1032192\n\t" \
  "s_add_u32 m0, s48, 147456\n\t" \
  "v_mfma_f32_16x16x128_f8f6f4 a[92:95], v[144:151], v[216:223], a[92:95]\n\t" \
  "buffer_load_dwordx4 v0, s[36:39], s53 offen sc0 sc1 lds\n\t" \
  "v_mfma_f32_16x16x128_f8f6f4 a[124:127], v[152:159], v[216:223], a[124:127]\n\t" \
  "buffer_load_dwordx4 v0, s[36:39], s53 offen offset:1024 sc0 sc1 lds\n\t" \
  "v_mfma_f32_16x16x128_f8f6f4 a[120:123], v[152:159], v[208:215], a[120:123]\n\t" \
  "buffer_load_dwordx4 v0, s[36:39], s53 offen offset:2048 sc0 sc1 lds\n\t" \
  "v_mfma_f32_16x16x128_f8f6f4 a[116:119], v[152:159], v[200:207], a[116:119]\n\t" \
  "buffer_load_dwordx4 v0, s[36:39], s53 offen offset:3072 sc0 sc1 lds\n\t" \
  "v_mfma_f32_16x16x128_f8f6f4 a[112:115], v[152:159], v[192:199], a[112:115]\n\t" \
  "s_waitcnt vmcnt(12)\n\t" \
  "v_cmp_le_u32_e32 vcc, s75, v7\n\t" \
  "s_nop 1\n\t" \
  "s_cmp_eq_u64 vcc, -1\n\t" \
  "s_cbranch_scc1 L_OKN14_%=\n\t" \
  "s_swappc_b64 s[80:81], s[78:79]\n\t" \
  "L_OKN14_%=:\n\t" \
  "s_add_u32 s76, s76, 1024\n\t" \
  "s_waitcnt lgkmcnt(0)\n\t" \
  "v_mfma_f32_16x16x128_f8f6f4 a[144:147], v[160:167], v[192:199], a[144:147]\n\t" \
  "s_waitcnt vmcnt(29)\n\t" \
  "s_barrier\n\t" \
  "v_mfma_f32_16x16x128_f8f6f4 a[148:151], v[160:167], v[200:207], a[148:151]\n\t" \
  "ds_read_b128 v[128:131], v1 offset:16384\n\t" \
  "ds_read_b128 v[132:135], v1 offset:24576\n\t" \
  "v_mfma_f32_16x16x128_f8f6f4 a[152:155], v[160:167], v[208:215], a[152:155]\n\t" \
  "ds_read_b128 v[136:139], v1 offset:17408\n\t" \
  "ds_read_b128 v[140:143], v1 offset:25600\n\t" \
  "v_mfma_f32_16x16x128_f8f6f4 a[156:159], v[160:167], v[216:223], a[156:159]\n\t" \
  "ds_read_b128 v[144:147], v1 offset:18432\n\t" \
  "ds_read_b128 v[148:151], v1 offset:26624\n\t" \
  "v_mfma_f32_16x16x128_f8f6f4 a[188:191], v[168:175], v[216:223], a[188:191]\n\t" \
  "ds_read_b128 v[152:155], v1 offset:19456\n\t" \
  "ds_read_b128 v[156:159], v1 offset:27648\n\t" \
  "v_mfma_f32_16x16x128_f8f6f4 a[184:187], v[168:175], v[208:215], a[184:187]\n\t" \
  "ds_read_b128 v[240:243], v3 offset:34816\n\t" \
  "ds_read_b128 v[244:247], v3 offset:43008\n\t" \
  "v_mfma_f32_16x16x128_f8f6f4 a[180:183], v[168:175], v[200:207], a[180:183]\n\t" \
  "ds_read_b128 v[248:251], v3 offset:35840\n\t" \
  "ds_read_b128 v[252:255], v3 offset:44032\n\t" \
  "v_mfma_f32_16x16x128_f8f6f4 a[176:179], v[168:175], v[192:199], a[176:179]\n\t" \
  "s_mov_b32 s53, s52\n\t" \
  "s_mov_b32 m0, s48\n\t" \
  "v_mfma_f32_16x16x128_f8f6f4 a[208:211], v[176:183], v[192:199], a[208:211]\n\t" \
  "buffer_load_dwordx4 v0, s[36:39], s53 offen sc0 sc1 lds\n\t" \
  "v_mfma_f32_16x16x128_f8f6f4 a[212:215], v[176:183], v[200:207], a[212:215]\n\t" \
  "v_mfma_f32_16x16x128_f8f6f4 a[216:219], v[176:183], v[208:215], a[216:219]\n\t" \
  "buffer_load_dwordx4 v0, s[36:39], s53 offen offset:1024 sc0 sc1 lds\n\t" \
  "v_mfma_f32_16x16x128_f8f6f4 a[220:223], v[176:183], v[216:223], a[220:223]\n\t" \
  "v_mfma_f32_16x16x128_f8f6f4 a[252:255], v[184:191], v[216:223], a[252:255]\n\t" \
  "buffer_load_dwordx4 v0, s[36:39], s53 offen offset:2048 sc0 sc1 lds\n\t" \
  "v_mfma_f32_16x16x128_f8f6f4 a[248:251], v[184:191], v[208:215], a[248:251]\n\t" \
  "v_mfma_f32_16x16x128_f8f6f4 a[244:247], v[184:191], v[200:207], a[244:247]\n\t" \
  "buffer_load_dwordx4 v0, s[36:39], s53 offen offset:3072 sc0 sc1 lds\n\t" \
  "v_mfma_f32_16x16x128_f8f6f4 a[240:243], v[184:191], v[192:199], a[240:243]\n\t" \
  "s_waitcnt lgkmcnt(0)\n\t" \
  "v_mfma_f32_16x16x128_f8f6f4 a[16:19], v[128:135], v[224:231], a[16:19]\n\t" \
  "s_waitcnt vmcnt(25)\n\t" \
  "s_barrier\n\t" \
  "v_mfma_f32_16x16x128_f8f6f4 a[20:23], v[128:135], v[232:239], a[20:23]\n\t" \
  "ds_read_b128 v[160:163], v1 offset:49152\n\t" \
  "ds_read_b128 v[164:167], v1 offset:57344\n\t" \
  "v_mfma_f32_16x16x128_f8f6f4 a[24:27], v[128:135], v[240:247], a[24:27]\n\t" \
  "ds_read_b128 v[168:171], v1 offset:50176\n\t" \
  "ds_read_b128 v[172:175], v1 offset:58368\n\t" \
  "v_mfma_f32_16x16x128_f8f6f4 a[28:31], v[128:135], v[248:255], a[28:31]\n\t" \
  "ds_read_b128 v[176:179], v1 offset:51200\n\t" \
  "ds_read_b128 v[180:183], v1 offset:59392\n\t" \
  "v_mfma_f32_16x16x128_f8f6f4 a[60:63], v[136:143], v[248:255], a[60:63]\n\t" \
  "ds_read_b128 v[184:187], v1 offset:52224\n\t" \
  "ds_read_b128 v[188:191], v1 offset:60416\n\t" \
  "v_mfma_f32_16x16x128_f8f6f4 a[56:59], v[136:143], v[240:247], a[56:59]\n\t" \
  "ds_read_b128 v[192:195], v4 offset:0\n\t" \
  "ds_read_b128 v[196:199], v4 offset:8192\n\t" \
  "v_mfma_f32_16x16x128_f8f6f4 a[52:55], v[136:143], v[232:239], a[52:55]\n\t" \
  "ds_read_b128 v[200:203], v4 offset:1024\n\t" \
  "ds_read_b128 v[204:207], v4 offset:9216\n\t" \
  "s_mov_b32 s53, s50\n\t" \
  "s_add_u32 m0, s48, 16384\n\t" \
  "v_mfma_f32_16x16x128_f8f6f4 a[48:51], v[136:143], v[224:231], a[48:51]\n\t" \
  "buffer_load_dwordx4 v0, s[36:39], s53 offen sc0 sc1 lds\n\t" \
  "v_mfma_f32_16x16x128_f8f6f4 a[80:83], v[144:151], v[224:231], a[80:83]\n\t" \
  "buffer_load_dwordx4 v0, s[36:39], s53 offen offset:1024 sc0 sc1 lds\n\t" \
  "v_mfma_f32_16x16x128_f8f6f4 a[84:87], v[144:151], v[232:239], a[84:87]\n\t" \
  "buffer_load_dwordx4 v0, s[36:39], s53 offen offset:2048 sc0 sc1 lds\n\t" \
  "v_mfma_f32_16x16x128_f8f6f4 a[88:91], v[144:151], v[240:247], a[88:91]\n\t" \
  "buffer_load_dwordx4 v0, s[36:39], s53 offen offset:3072 sc0 sc1 lds\n\t" \
  "s_mov_b32 s53, s91\n\t" \
  "s_add_u32 m0, s48, 32768\n\t" \
  "v_mfma_f32_16x16x128_f8f6f4 a[92:95], v[144:151], v[248:255], a[92:95]\n\t" \
  "buffer_load_dwordx4 v0, s[36:39], s53 offen sc0 sc1 lds\n\t" \
  "v_mfma_f32_16x16x128_f8f6f4 a[124:127], v[152:159], v[248:255], a[124:127]\n\t" \
  "buffer_load_dwordx4 v0, s[36:39], s53 offen offset:1024 sc0 sc1 lds\n\t" \
  "v_mfma_f32_16x16x128_f8f6f4 a[120:123], v[152:159], v[240:247], a[120:123]\n\t" \
  "buffer_load_dwordx4 v0, s[36:39], s53 offen offset:2048 sc0 sc1 lds\n\t" \
  "v_mfma_f32_16x16x128_f8f6f4 a[116:119], v[152:159], v[232:239], a[116:119]\n\t" \
  "buffer_load_dwordx4 v0, s[36:39], s53 offen offset:3072 sc0 sc1 lds\n\t" \
  "v_mfma_f32_16x16x128_f8f6f4 a[112:115], v[152:159], v[224:231], a[112:115]\n\t" \
  "s_waitcnt lgkmcnt(0)\n\t" \
  "v_mfma_f32_16x16x128_f8f6f4 a[144:147], v[160:167], v[224:231], a[144:147]\n\t" \
  "s_waitcnt vmcnt(29)\n\t" \
  "s_barrier\n\t" \
  "v_mfma_f32_16x16x128_f8f6f4 a[148:151], v[160:167], v[232:239], a[148:151]\n\t" \
  "ds_read_b128 v[128:131], v2 offset:16384\n\t" \
  "ds_read_b128 v[132:135], v2 offset:24576\n\t" \
  "v_mfma_f32_16x16x128_f8f6f4 a[152:155], v[160:167], v[240:247], a[152:155]\n\t" \
  "ds_read_b128 v[136:139], v2 offset:17408\n\t" \
  "ds_read_b128 v[140:143], v2 offset:25600\n\t" \
  "v_mfma_f32_16x16x128_f8f6f4 a[156:159], v[160:167], v[248:255], a[156:159]\n\t" \
  "ds_read_b128 v[144:147], v2 offset:18432\n\t" \
  "ds_read_b128 v[148:151], v2 offset:26624\n\t" \
  "v_mfma_f32_16x16x128_f8f6f4 a[188:191], v[168:175], v[248:255], a[188:191]\n\t" \
  "ds_read_b128 v[152:155], v2 offset:19456\n\t" \
  "ds_read_b128 v[156:159], v2 offset:27648\n\t" \
  "v_mfma_f32_16x16x128_f8f6f4 a[184:187], v[168:175], v[240:247], a[184:187]\n\t" \
  "ds_read_b128 v[208:211], v4 offset:2048\n\t" \
  "ds_read_b128 v[212:215], v4 offset:10240\n\t" \
  "v_mfma_f32_16x16x128_f8f6f4 a[180:183], v[168:175], v[232:239], a[180:183]\n\t" \
  "ds_read_b128 v[216:219], v4 offset:3072\n\t" \
  "ds_read_b128 v[220:223], v4 offset:11264\n\t" \
  "v_mfma_f32_16x16x128_f8f6f4 a[176:179], v[168:175], v[224:231], a[176:179]\n\t" \
  "s_add_u32 s53, s52, 16384\n\t" \
  "s_add_u32 m0, s48, 49152\n\t" \
  "v_mfma_f32_16x16x128_f8f6f4 a[208:211], v[176:183], v[224:231], a[208:211]\n\t" \
  "buffer_load_dwordx4 v0, s[36:39], s53 offen sc0 sc1 lds\n\t" \
  "v_mfma_f32_16x16x128_f8f6f4 a[212:215], v[176:183], v[232:239], a[212:215]\n\t" \
  "v_mfma_f32_16x16x128_f8f6f4 a[216:219], v[176:183], v[240:247], a[216:219]\n\t" \
  "buffer_load_dwordx4 v0, s[36:39], s53 offen offset:1024 sc0 sc1 lds\n\t" \
  "v_mfma_f32_16x16x128_f8f6f4 a[220:223], v[176:183], v[248:255], a[220:223]\n\t" \
  "v_mfma_f32_16x16x128_f8f6f4 a[252:255], v[184:191], v[248:255], a[252:255]\n\t" \
  "buffer_load_dwordx4 v0, s[36:39], s53 offen offset:2048 sc0 sc1 lds\n\t" \
  "v_mfma_f32_16x16x128_f8f6f4 a[248:251], v[184:191], v[240:247], a[248:251]\n\t" \
  "v_mfma_f32_16x16x128_f8f6f4 a[244:247], v[184:191], v[232:239], a[244:247]\n\t" \
  "buffer_load_dwordx4 v0, s[36:39], s53 offen offset:3072 sc0 sc1 lds\n\t" \
  "v_mfma_f32_16x16x128_f8f6f4 a[240:243], v[184:191], v[224:231], a[240:243]\n\t" \
  "s_waitcnt lgkmcnt(0)\n\t" \
  "v_mfma_f32_16x16x128_f8f6f4 a[16:19], v[128:135], v[192:199], a[16:19]\n\t" \
  "s_waitcnt vmcnt(24)\n\t" \
  "s_barrier\n\t" \
  "v_mfma_f32_16x16x128_f8f6f4 a[20:23], v[128:135], v[200:207], a[20:23]\n\t" \
  "ds_read_b128 v[160:163], v2 offset:32768\n\t" \
  "ds_read_b128 v[164:167], v2 offset:40960\n\t" \
  "v_mfma_f32_16x16x128_f8f6f4 a[24:27], v[128:135], v[208:215], a[24:27]\n\t" \
  "ds_read_b128 v[168:171], v2 offset:33792\n\t" \
  "ds_read_b128 v[172:175], v2 offset:41984\n\t" \
  "v_mfma_f32_16x16x128_f8f6f4 a[28:31], v[128:135], v[216:223], a[28:31]\n\t" \
  "ds_read_b128 v[176:179], v2 offset:34816\n\t" \
  "ds_read_b128 v[180:183], v2 offset:43008\n\t" \
  "v_mfma_f32_16x16x128_f8f6f4 a[60:63], v[136:143], v[216:223], a[60:63]\n\t" \
  "ds_read_b128 v[184:187], v2 offset:35840\n\t" \
  "ds_read_b128 v[188:191], v2 offset:44032\n\t" \
  "v_mfma_f32_16x16x128_f8f6f4 a[56:59], v[136:143], v[208:215], a[56:59]\n\t" \
  "ds_read_b128 v[224:227], v127 offset:0\n\t" \
  "ds_read_b128 v[228:231], v127 offset:8192\n\t" \
  "v_mfma_f32_16x16x128_f8f6f4 a[52:55], v[136:143], v[200:207], a[52:55]\n\t" \
  "ds_read_b128 v[232:235], v127 offset:1024\n\t" \
  "ds_read_b128 v[236:239], v127 offset:9216\n\t" \
  "s_add_u32 s53, s52, 32768\n\t" \
  "s_add_u32 m0, s48, 65536\n\t" \
  "v_mfma_f32_16x16x128_f8f6f4 a[48:51], v[136:143], v[192:199], a[48:51]\n\t" \
  "buffer_load_dwordx4 v0, s[36:39], s53 offen sc0 sc1 lds\n\t" \
  "v_mfma_f32_16x16x128_f8f6f4 a[80:83], v[144:151], v[192:199], a[80:83]\n\t" \
  "buffer_load_dwordx4 v0, s[36:39], s53 offen offset:1024 sc0 sc1 lds\n\t" \
  "v_mfma_f32_16x16x128_f8f6f4 a[84:87], v[144:151], v[200:207], a[84:87]\n\t" \
  "buffer_load_dwordx4 v0, s[36:39], s53 offen offset:2048 sc0 sc1 lds\n\t" \
  "v_mfma_f32_16x16x128_f8f6f4 a[88:91], v[144:151], v[208:215], a[88:91]\n\t" \
  "buffer_load_dwordx4 v0, s[36:39], s53 offen offset:3072 sc0 sc1 lds\n\t" \
  "s_add_u32 s53, s91, 16384\n\t" \
  "s_add_u32 m0, s48, 81920\n\t" \
  "v_mfma_f32_16x16x128_f8f6f4 a[92:95], v[144:151], v[216:223], a[92:95]\n\t" \
  "buffer_load_dwordx4 v0, s[36:39], s53 offen sc0 sc1 lds\n\t" \
  "v_mfma_f32_16x16x128_f8f6f4 a[124:127], v[152:159], v[216:223], a[124:127]\n\t" \
  "buffer_load_dwordx4 v0, s[36:39], s53 offen offset:1024 sc0 sc1 lds\n\t" \
  "v_mfma_f32_16x16x128_f8f6f4 a[120:123], v[152:159], v[208:215], a[120:123]\n\t" \
  "buffer_load_dwordx4 v0, s[36:39], s53 offen offset:2048 sc0 sc1 lds\n\t" \
  "v_mfma_f32_16x16x128_f8f6f4 a[116:119], v[152:159], v[200:207], a[116:119]\n\t" \
  "buffer_load_dwordx4 v0, s[36:39], s53 offen offset:3072 sc0 sc1 lds\n\t" \
  "v_mfma_f32_16x16x128_f8f6f4 a[112:115], v[152:159], v[192:199], a[112:115]\n\t" \
  "s_waitcnt lgkmcnt(0)\n\t" \
  "v_mfma_f32_16x16x128_f8f6f4 a[144:147], v[160:167], v[192:199], a[144:147]\n\t" \
  "s_waitcnt vmcnt(28)\n\t" \
  "s_barrier\n\t" \
  "v_mfma_f32_16x16x128_f8f6f4 a[148:151], v[160:167], v[200:207], a[148:151]\n\t" \
  "ds_read_b128 v[128:131], v2 offset:49152\n\t" \
  "ds_read_b128 v[132:135], v2 offset:57344\n\t" \
  "v_mfma_f32_16x16x128_f8f6f4 a[152:155], v[160:167], v[208:215], a[152:155]\n\t" \
  "ds_read_b128 v[136:139], v2 offset:50176\n\t" \
  "ds_read_b128 v[140:143], v2 offset:58368\n\t" \
  "v_mfma_f32_16x16x128_f8f6f4 a[156:159], v[160:167], v[216:223], a[156:159]\n\t" \
  "ds_read_b128 v[144:147], v2 offset:51200\n\t" \
  "ds_read_b128 v[148:151], v2 offset:59392\n\t" \
  "v_mfma_f32_16x16x128_f8f6f4 a[188:191], v[168:175], v[216:223], a[188:191]\n\t" \
  "ds_read_b128 v[152:155], v2 offset:52224\n\t" \
  "ds_read_b128 v[156:159], v2 offset:60416\n\t" \
  "v_mfma_f32_16x16x128_f8f6f4 a[184:187], v[168:175], v[208:215], a[184:187]\n\t" \
  "ds_read_b128 v[240:243], v127 offset:2048\n\t" \
  "ds_read_b128 v[244:247], v127 offset:10240\n\t" \
  "v_mfma_f32_16x16x128_f8f6f4 a[180:183], v[168:175], v[200:207], a[180:183]\n\t" \
  "ds_read_b128 v[248:251], v127 offset:3072\n\t" \
  "ds_read_b128 v[252:255], v127 offset:11264\n\t" \
  "v_mfma_f32_16x16x128_f8f6f4 a[176:179], v[168:175], v[192:199], a[176:179]\n\t" \
  "s_add_u32 s53, s50, 16384\n\t" \
  "s_add_u32 m0, s48, 98304\n\t" \
  "v_mfma_f32_16x16x128_f8f6f4 a[208:211], v[176:183], v[192:199], a[208:211]\n\t" \
  "buffer_load_dwordx4 v0, s[36:39], s53 offen sc0 sc1 lds\n\t" \
  "v_mfma_f32_16x16x128_f8f6f4 a[212:215], v[176:183], v[200:207], a[212:215]\n\t" \
  "v_mfma_f32_16x16x128_f8f6f4 a[216:219], v[176:183], v[208:215], a[216:219]\n\t" \
  "buffer_load_dwordx4 v0, s[36:39], s53 offen offset:1024 sc0 sc1 lds\n\t" \
  "v_mfma_f32_16x16x128_f8f6f4 a[220:223], v[176:183], v[216:223], a[220:223]\n\t" \
  "v_mfma_f32_16x16x128_f8f6f4 a[252:255], v[184:191], v[216:223], a[252:255]\n\t" \
  "buffer_load_dwordx4 v0, s[36:39], s53 offen offset:2048 sc0 sc1 lds\n\t" \
  "v_mfma_f32_16x16x128_f8f6f4 a[248:251], v[184:191], v[208:215], a[248:251]\n\t" \
  "v_mfma_f32_16x16x128_f8f6f4 a[244:247], v[184:191], v[200:207], a[244:247]\n\t" \
  "buffer_load_dwordx4 v0, s[36:39], s53 offen offset:3072 sc0 sc1 lds\n\t" \
  "v_mfma_f32_16x16x128_f8f6f4 a[240:243], v[184:191], v[192:199], a[240:243]\n\t" \
  "s_waitcnt lgkmcnt(0)\n\t" \
  "v_mfma_f32_16x16x128_f8f6f4 a[16:19], v[128:135], v[224:231], a[16:19]\n\t" \
  "s_waitcnt vmcnt(20)\n\t" \
  "s_barrier\n\t" \
  "v_mfma_f32_16x16x128_f8f6f4 a[20:23], v[128:135], v[232:239], a[20:23]\n\t" \
  "ds_read_b128 v[160:163], v126 offset:16384\n\t" \
  "ds_read_b128 v[164:167], v126 offset:24576\n\t" \
  "v_mfma_f32_16x16x128_f8f6f4 a[24:27], v[128:135], v[240:247], a[24:27]\n\t" \
  "ds_read_b128 v[168:171], v126 offset:17408\n\t" \
  "ds_read_b128 v[172:175], v126 offset:25600\n\t" \
  "v_mfma_f32_16x16x128_f8f6f4 a[28:31], v[128:135], v[248:255], a[28:31]\n\t" \
  "ds_read_b128 v[176:179], v126 offset:18432\n\t" \
  "ds_read_b128 v[180:183], v126 offset:26624\n\t" \
  "v_mfma_f32_16x16x128_f8f6f4 a[60:63], v[136:143], v[248:255], a[60:63]\n\t" \
  "ds_read_b128 v[184:187], v126 offset:19456\n\t" \
  "ds_read_b128 v[188:191], v126 offset:27648\n\t" \
  "v_mfma_f32_16x16x128_f8f6f4 a[56:59], v[136:143], v[240:247], a[56:59]\n\t" \
  "ds_read_b128 v[192:195], v3 offset:16384\n\t" \
  "ds_read_b128 v[196:199], v3 offset:24576\n\t" \
  "v_mfma_f32_16x16x128_f8f6f4 a[52:55], v[136:143], v[232:239], a[52:55]\n\t" \
  "ds_read_b128 v[200:203], v3 offset:17408\n\t" \
  "ds_read_b128 v[204:207], v3 offset:25600\n\t" \
  "v_mfma_f32_16x16x128_f8f6f4 a[48:51], v[136:143], v[224:231], a[48:51]\n\t" \
  "s_add_u32 s53, s52, 49152\n\t" \
  "s_add_u32 m0, s48, 114688\n\t" \
  "v_mfma_f32_16x16x128_f8f6f4 a[80:83], v[144:151], v[224:231], a[80:83]\n\t" \
  "buffer_load_dwordx4 v0, s[36:39], s53 offen sc0 sc1 lds\n\t" \
  "v_mfma_f32_16x16x128_f8f6f4 a[84:87], v[144:151], v[232:239], a[84:87]\n\t" \
  "v_mfma_f32_16x16x128_f8f6f4 a[88:91], v[144:151], v[240:247], a[88:91]\n\t" \
  "buffer_load_dwordx4 v0, s[36:39], s53 offen offset:1024 sc0 sc1 lds\n\t" \
  "v_mfma_f32_16x16x128_f8f6f4 a[92:95], v[144:151], v[248:255], a[92:95]\n\t" \
  "v_mfma_f32_16x16x128_f8f6f4 a[124:127], v[152:159], v[248:255], a[124:127]\n\t" \
  "buffer_load_dwordx4 v0, s[36:39], s53 offen offset:2048 sc0 sc1 lds\n\t" \
  "v_mfma_f32_16x16x128_f8f6f4 a[120:123], v[152:159], v[240:247], a[120:123]\n\t" \
  "v_mfma_f32_16x16x128_f8f6f4 a[116:119], v[152:159], v[232:239], a[116:119]\n\t" \
  "buffer_load_dwordx4 v0, s[36:39], s53 offen offset:3072 sc0 sc1 lds\n\t" \
  "v_mfma_f32_16x16x128_f8f6f4 a[112:115], v[152:159], v[224:231], a[112:115]\n\t" \
  "s_waitcnt lgkmcnt(0)\n\t" \
  "v_mfma_f32_16x16x128_f8f6f4 a[144:147], v[160:167], v[224:231], a[144:147]\n\t" \
  "s_waitcnt vmcnt(28)\n\t" \
  "s_barrier\n\t" \
  "v_mfma_f32_16x16x128_f8f6f4 a[148:151], v[160:167], v[232:239], a[148:151]\n\t" \
  "ds_read_b128 v[128:131], v1 offset:0\n\t" \
  "ds_read_b128 v[132:135], v1 offset:8192\n\t" \
  "v_mfma_f32_16x16x128_f8f6f4 a[152:155], v[160:167], v[240:247], a[152:155]\n\t" \
  "ds_read_b128 v[136:139], v1 offset:1024\n\t" \
  "ds_read_b128 v[140:143], v1 offset:9216\n\t" \
  "v_mfma_f32_16x16x128_f8f6f4 a[156:159], v[160:167], v[248:255], a[156:159]\n\t" \
  "ds_read_b128 v[144:147], v1 offset:2048\n\t" \
  "ds_read_b128 v[148:151], v1 offset:10240\n\t" \
  "v_mfma_f32_16x16x128_f8f6f4 a[188:191], v[168:175], v[248:255], a[188:191]\n\t" \
  "ds_read_b128 v[152:155], v1 offset:3072\n\t" \
  "ds_read_b128 v[156:159], v1 offset:11264\n\t" \
  "v_mfma_f32_16x16x128_f8f6f4 a[184:187], v[168:175], v[240:247], a[184:187]\n\t" \
  "ds_read_b128 v[208:211], v3 offset:18432\n\t" \
  "ds_read_b128 v[212:215], v3 offset:26624\n\t" \
  "v_mfma_f32_16x16x128_f8f6f4 a[180:183], v[168:175], v[232:239], a[180:183]\n\t" \
  "ds_read_b128 v[216:219], v3 offset:19456\n\t" \
  "ds_read_b128 v[220:223], v3 offset:27648\n\t" \
  "v_mfma_f32_16x16x128_f8f6f4 a[176:179], v[168:175], v[224:231], a[176:179]\n\t" \
  "v_mfma_f32_16x16x128_f8f6f4 a[208:211], v[176:183], v[224:231], a[208:211]\n\t" \
  "v_mfma_f32_16x16x128_f8f6f4 a[212:215], v[176:183], v[232:239], a[212:215]\n\t" \
  "v_mfma_f32_16x16x128_f8f6f4 a[216:219], v[176:183], v[240:247], a[216:219]\n\t" \
  "v_mfma_f32_16x16x128_f8f6f4 a[220:223], v[176:183], v[248:255], a[220:223]\n\t" \
  "v_mfma_f32_16x16x128_f8f6f4 a[252:255], v[184:191], v[248:255], a[252:255]\n\t" \
  "v_mfma_f32_16x16x128_f8f6f4 a[248:251], v[184:191], v[240:247], a[248:251]\n\t" \
  "v_mfma_f32_16x16x128_f8f6f4 a[244:247], v[184:191], v[232:239], a[244:247]\n\t" \
  "v_mfma_f32_16x16x128_f8f6f4 a[240:243], v[184:191], v[224:231], a[240:243]\n\t" \
  "s_add_u32 s49, s52, 65536\n\t" \
  "s_add_u32 s50, s50, 32768\n\t" \
  "s_add_u32 s51, s91, 32768\n\t" \
  "s_sub_u32 s72, s72, 1\n\t" \
  "s_branch L_WE_%=\n\t" \
  "L_POLL_%=:\n\t" \
  "s_memrealtime s[82:83]\n\t" \
  "s_mov_b32 s55, 0\n\t" \
  "s_waitcnt lgkmcnt(0)\n\t" \
  "L_POLL1_%=:\n\t" \
  "buffer_load_dword v7, v8, s[44:47], s76 offen sc0 sc1\n\t" \
  "s_waitcnt vmcnt(0)\n\t" \
  "v_cmp_le_u32_e32 vcc, s75, v7\n\t" \
  "s_nop 1\n\t" \
  "s_cmp_eq_u64 vcc, -1\n\t" \
  "s_cbranch_scc1 L_POLLOK_%=\n\t" \
  "s_sleep 16\n\t" \
  "s_add_u32 s55, s55, 1\n\t" \
  "s_and_b32 s54, s55, 15\n\t" \
  "s_cmp_lg_u32 s54, 0\n\t" \
  "s_cbranch_scc1 L_POLL1_%=\n\t" \
  "global_load_dword v9, v10, s[86:87] sc1\n\t" \
  "s_memrealtime s[84:85]\n\t" \
  "s_waitcnt vmcnt(0) lgkmcnt(0)\n\t" \
  "v_readfirstlane_b32 s54, v9\n\t" \
  "s_nop 3\n\t" \
  "s_cmp_eq_u32 s54, s88\n\t" \
  "s_cbranch_scc1 L_POLLBAD_%=\n\t" \
  "s_sub_u32 s84, s84, s82\n\t" \
  "s_subb_u32 s85, s85, s83\n\t" \
  "s_cmp_lg_u32 s85, 0\n\t" \
  "s_cbranch_scc1 L_POLLBAD_%=\n\t" \
  "s_cmp_gt_u32 s84, 100000000\n\t" \
  "s_cbranch_scc0 L_POLL1_%=\n\t" \
  "L_POLLBAD_%=:\n\t" \
  "v_mov_b32 v9, s88\n\t" \
  "global_store_dword v10, v9, s[86:87] sc1\n\t" \
  "s_mov_b32 s56, 0x7fc00000\n\t" \
  "s_mov_b32 s57, 0x7fc00000\n\t" \
  "s_mov_b32 s75, 0\n\t" \
  "L_POLLOK_%=:\n\t" \
  "s_setpc_b64 s[80:81]\n\t" \
  "L_SLOW01_%=:\n\t" \
  "s_swappc_b64 s[80:81], s[78:79]\n\t" \
  "s_branch L_OK01_%=\n\t" \
  "L_SLOW03_%=:\n\t" \
  "s_swappc_b64 s[80:81], s[78:79]\n\t" \
  "s_branch L_OK03_%=\n\t" \
  "L_SLOW05_%=:\n\t" \
  "s_swappc_b64 s[80:81], s[78:79]\n\t" \
  "s_branch L_OK05_%=\n\t" \
  "L_SLOW07_%=:\n\t" \
  "s_swappc_b64 s[80:81], s[78:79]\n\t" \
  "s_branch L_OK07_%=\n\t" \
  "L_SLOW11_%=:\n\t" \
  "s_swappc_b64 s[80:81], s[78:79]\n\t" \
  "s_branch L_OK11_%=\n\t" \
  "L_SLOW13_%=:\n\t" \
  "s_swappc_b64 s[80:81], s[78:79]\n\t" \
  "s_branch L_OK13_%=\n\t" \
  "L_SLOW15_%=:\n\t" \
  "s_swappc_b64 s[80:81], s[78:79]\n\t" \
  "s_branch L_OK15_%=\n\t" \
  "L_SLOW17_%=:\n\t" \
  "s_swappc_b64 s[80:81], s[78:79]\n\t" \
  "s_branch L_OK17_%=\n\t" \
  "L_SLOWH0_%=:\n\t" \
  "s_swappc_b64 s[80:81], s[78:79]\n\t" \
  "s_branch L_OKH0_%=\n\t" \
  "L_SLOWH1_%=:\n\t" \
  "s_swappc_b64 s[80:81], s[78:79]\n\t" \
  "s_branch L_OKH1_%=\n\t" \
  "L_END_%=:\n\t" \
  "v_readfirstlane_b32 s54, v7\n\t" \
  "s_nop 3\n\t" \
  "s_cmp_lg_u32 s54, s88\n\t" \
  "s_cbranch_scc1 L_FIN_%=\n\t" \
  "v_mov_b32 v9, 0x7fc00000\n\t" \
  "buffer_store_dword v9, v5, s[40:43], s66 offen\n\t" \
  "L_FIN_%=:\n\t" \
  "s_waitcnt vmcnt(0) lgkmcnt(0)\n\t" \
  ""
#define GEMM_CLOB "v0", "v1", "v2", "v3", "v4", "v5", "v6", "v7", "v8", "v9", "v10", "v11", "v12", "v13", "v14", "v15", "v16", "v17", "v18", "v19", "v20", "v21", "v22", "v23", "v24", "v25", "v26", "v27", "v28", "v29", "v30", "v31", "v32", "v33", "v34", "v35", "v36", "v37", "v38", "v39", "v40", "v41", "v42", "v43", "v44", "v45", "v46", "v47", "v48", "v49", "v50", "v51", "v52", "v53", "v54", "v55", "v56", "v57", "v58", "v59", "v60", "v61", "v62", "v63", "v64", "v65", "v66", "v67", "v68", "v69", "v70", "v71", "v72", "v73", "v74", "v75", "v76", "v77", "v78", "v79", "v80", "v81", "v82", "v83", "v84", "v85", "v86", "v87", "v88", "v89", "v90", "v91", "v92", "v93", "v94", "v95", "v96", "v97", "v98", "v99", "v100", "v101", "v102", "v103", "v104", "v105", "v106", "v107", "v108", "v109", "v110", "v111", "v112", "v113", "v114", "v115", "v116", "v117", "v118", "v119", "v120", "v121", "v122", "v123", "v124", "v125", "v126", "v127", "v128", "v129", "v130", "v131", "v132", "v133", "v134", "v135", "v136", "v137", "v138", "v139", "v140", "v141", "v142", "v143", "v144", "v145", "v146", "v147", "v148", "v149", "v150", "v151", "v152", "v153", "v154", "v155", "v156", "v157", "v158", "v159", "v160", "v161", "v162", "v163", "v164", "v165", "v166", "v167", "v168", "v169", "v170", "v171", "v172", "v173", "v174", "v175", "v176", "v177", "v178", "v179", "v180", "v181", "v182", "v183", "v184", "v185", "v186", "v187", "v188", "v189", "v190", "v191", "v192", "v193", "v194", "v195", "v196", "v197", "v198", "v199", "v200", "v201", "v202", "v203", "v204", "v205", "v206", "v207", "v208", "v209", "v210", "v211", "v212", "v213", "v214", "v215", "v216", "v217", "v218", "v219", "v220", "v221", "v222", "v223", "v224", "v225", "v226", "v227", "v228", "v229", "v230", "v231", "v232", "v233", "v234", "v235", "v236", "v237", "v238", "v239", "v240", "v241", "v242", "v243", "v244", "v245", "v246", "v247", "v248", "v249", "v250", "v251", "v252", "v253", "v254", "v255", "a0", "a1", "a2", "a3", "a4", "a5", "a6", "a7", "a8", "a9", "a10", "a11", "a12", "a13", "a14", "a15", "a16", "a17", "a18", "a19", "a20", "a21", "a22", "a23", "a24", "a25", "a26", "a27", "a28", "a29", "a30", "a31", "a32", "a33", "a34", "a35", "a36", "a37", "a38", "a39", "a40", "a41", "a42", "a43", "a44", "a45", "a46", "a47", "a48", "a49", "a50", "a51", "a52", "a53", "a54", "a55", "a56", "a57", "a58", "a59", "a60", "a61", "a62", "a63", "a64", "a65", "a66", "a67", "a68", "a69", "a70", "a71", "a72", "a73", "a74", "a75", "a76", "a77", "a78", "a79", "a80", "a81", "a82", "a83", "a84", "a85", "a86", "a87", "a88", "a89", "a90", "a91", "a92", "a93", "a94", "a95", "a96", "a97", "a98", "a99", "a100", "a101", "a102", "a103", "a104", "a105", "a106", "a107", "a108", "a109", "a110", "a111", "a112", "a113", "a114", "a115", "a116", "a117", "a118", "a119", "a120", "a121", "a122", "a123", "a124", "a125", "a126", "a127", "a128", "a129", "a130", "a131", "a132", "a133", "a134", "a135", "a136", "a137", "a138", "a139", "a140", "a141", "a142", "a143", "a144", "a145", "a146", "a147", "a148", "a149", "a150", "a151", "a152", "a153", "a154", "a155", "a156", "a157", "a158", "a159", "a160", "a161", "a162", "a163", "a164", "a165", "a166", "a167", "a168", "a169", "a170", "a171", "a172", "a173", "a174", "a175", "a176", "a177", "a178", "a179", "a180", "a181", "a182", "a183", "a184", "a185", "a186", "a187", "a188", "a189", "a190", "a191", "a192", "a193", "a194", "a195", "a196", "a197", "a198", "a199", "a200", "a201", "a202", "a203", "a204", "a205", "a206", "a207", "a208", "a209", "a210", "a211", "a212", "a213", "a214", "a215", "a216", "a217", "a218", "a219", "a220", "a221", "a222", "a223", "a224", "a225", "a226", "a227", "a228", "a229", "a230", "a231", "a232", "a233", "a234", "a235", "a236", "a237", "a238", "a239", "a240", "a241", "a242", "a243", "a244", "a245", "a246", "a247", "a248", "a249", "a250", "a251", "a252", "a253", "a254", "a255", "s36", "s37", "s38", "s39", "s40", "s41", "s42", "s43", "s44", "s45", "s46", "s47", "s48", "s49", "s50", "s51", "s52", "s53", "s54", "s55", "s56", "s57", "s58", "s59", "s60", "s61", "s62", "s63", "s64", "s65", "s66", "s67", "s68", "s69", "s70", "s71", "s72", "s73", "s74", "s75", "s76", "s77", "s78", "s79", "s80", "s81", "s82", "s83", "s84", "s85", "s86", "s87", "s88", "s89", "s90", "s91", "s92", "s93", "s94", "s95", "s96", "vcc", "memory"

namespace {
constexpr int NR = 8;
constexpr int M = 4096, N = 16384, KS = 512;
constexpr int NCOMM = 16;
constexpr int GRID = 256, THREADS = 256;
constexpr int LDS_BYTES = 163840;
constexpr size_t WS_TICKET = 81920  , WS_TMO = 128, WS_PCTR = 512, WS_FLAGS = 4096;
constexpr size_t WS_STAMP = 262144;
constexpr size_t WS_STG = 1u << 20, A_BYTES = 16u << 20, STG_BYTES = 80u << 20;
constexpr size_t WS_NEED = WS_STG + 2 * STG_BYTES;
constexpr int NUNITS = 640;
#define PB 2
#define PW 6
#define RET_D 1
#define TA 5
#define ORD1 0x73625140u
#define SFIRST 1

typedef unsigned u32x4 __attribute__((ext_vector_type(4)));
typedef unsigned u32x4s __attribute__((ext_vector_type(4)));
typedef __attribute__((address_space(3))) u32x4 lds_u32x4;
typedef volatile __attribute__((address_space(3))) unsigned lds_u32;

struct Params {
    unsigned char* self;
    unsigned char* nws[NR];
    float* out;
    const unsigned char* x;
    const unsigned char* w;
    const float* sx;
    const float* sw;
    unsigned nidx;
    int pad;
};

#define RFL(x) __builtin_amdgcn_readfirstlane(x)

__device__ __forceinline__ int blk(int g, int i) { return g < 8 ? (i == 0 ? g : 15 * i - 7 + g) : 15 * i + 8 + g; }

inline int near_dest(int rank, int x) {
    const unsigned S1 = 0x4d, mine = ((S1 >> rank) & 1) ? S1 : (~S1 & 0xffu);
    if (x == 0) return rank;
    unsigned m = x < 4 ? (mine & ~(1u << rank)) : (~mine & 0xffu);
    for (int k = x < 4 ? x - 1 : x - 4; k > 0; --k) m &= m - 1;
    return __builtin_ctz(m);
}

struct Unit { unsigned src, delta, dst0, dst1, cw; };
__device__ __forceinline__ Unit decode_unit(int U) {
    int st, qq, u, n; bool isx = false, cls_s = false; int pm = 0;
    if (U < 80) {
        st = 0; n = 20;
        if (U < 32) { qq = U >> 4; u = U & 15; }
        else if (U < 40) { qq = (U - 32) >> 2; u = 16 + (U & 3); }
        else { qq = U < 60 ? 2 : 3; u = U - (U < 60 ? 40 : 60); }
        isx = u < 16;
    } else if (U < 190) {
        st = 1;
        int v = U - 80;
        qq = 0; n = 0;
        for (int k = 0; k < 8; ++k) {
            const int seg = (ORD1 >> (4 * k)) & 7;
            cls_s = seg >= 4; qq = seg & 3; n = cls_s ? 4 - (qq & 1) : 24;
            if (v < n) break;
            v -= n;
        }
        u = v;
        isx = !cls_s && u < 16;
    } else if (U < 610) {
        const int v = U - 190, k = v / 15, r = v - 15 * k;
        st = 2 + (k >> 2); qq = k & 3;
        cls_s = SFIRST ? r < 7 : r >= 8;
        u = SFIRST ? (cls_s ? r : r - 7) : (cls_s ? r - 8 : r);
        n = cls_s ? 7 : 8;
    } else {
        st = 9;
        const int v = U - 610, h = v / 15, r = v - 15 * h;
        qq = 2 * h + (r >= 8 ? 1 : 0); u = r >= 8 ? r - 8 : r;
        cls_s = u >= 4; if (cls_s) u -= 4;
        n = cls_s ? 4 - (qq & 1) : 4;
    }
    pm = u & 15;
    const int jj = ((st & 1) ? 0 : 4) + qq;
    const int s = cls_s ? st - 1 : st;
    Unit r;
    r.cw = (unsigned)((cls_s ? 40 : 0) + 4 * s + qq) | ((unsigned)(u == 0 ? 128 - n + 1 : 1) << 8);
    if (isx) {
        r.src = 0x80000000u | (pm << 17) | ((pm & 3) << 14) | (jj << 6);
        r.delta = 0;
        r.dst0 = ((unsigned)pm << 20) + jj * 4 * 32768 + 1u;
        r.dst1 = ((unsigned)pm << 20) + jj * 4 * 32768 + 16384;
    } else {
        int a0, a1, j1 = jj;
        const int last = cls_s ? 8 : 9;
        if (s == 0 || s == last) {
            const int k = st < 2 && !cls_s ? u - 16 : u, i = s == 0 ? 0 : s - 1;
            const int g = (cls_s ? 8 : 0) + 2 * k;
            a0 = blk(g, i);
            if (g == 14) { a1 = a0; j1 = jj + 1; } else a1 = blk(g + 1, i);
        } else {
            const int g = cls_s ? 8 + u : (st == 1 ? u - 16 : u);
            a0 = blk(g, s - 1); a1 = blk(g, s);
        }
        const int o0 = jj * 64 * N + a0 * 128, o1 = j1 * 64 * N + a1 * 128;
        r.src = (unsigned)(o0 < o1 ? o0 : o1);
        r.delta = (unsigned)(o1 - o0);
        r.dst0 = (unsigned)A_BYTES + a0 * (32 * 16384) + jj * 4 * 16384;
        r.dst1 = (unsigned)A_BYTES + a1 * (32 * 16384) + j1 * 4 * 16384;
    }
    return r;
}

__device__ __forceinline__ u32x4s make_srd(const void* base, unsigned bytes) {
    const unsigned long long a = (unsigned long long)base;
    u32x4s r; r.x = (unsigned)a; r.y = (unsigned)(a >> 32) & 0xffffu; r.z = bytes; r.w = 0x20000u;
    return r;
}

__device__ __forceinline__ void pin16(u32x4 (&d)[16]) {
    asm volatile("" : "+v"(d[0]), "+v"(d[1]), "+v"(d[2]), "+v"(d[3]), "+v"(d[4]), "+v"(d[5]), "+v"(d[6]), "+v"(d[7]),
                      "+v"(d[8]), "+v"(d[9]), "+v"(d[10]), "+v"(d[11]), "+v"(d[12]), "+v"(d[13]), "+v"(d[14]), "+v"(d[15]));
}

__device__ __forceinline__ void retile_w(lds_u32x4* mine, int lane) {
    u32x4 d[16];
#pragma unroll
    for (int i = 0; i < 16; ++i) d[i] = mine[i * 64 + lane];
    asm volatile("s_waitcnt lgkmcnt(0)" ::: "memory");
    pin16(d);
    const int kg = lane >> 3, cp = lane & 7;
    lds_u32x4* t = mine + (kg >> 2) * 512 + (cp >> 2) * 256 + ((cp >> 1) & 1) * 128 + (kg & 3) * 16 + (cp & 1) * 8;
#pragma unroll
    for (int g = 0; g < 4; ++g) {
        u32x4 q4[4];
#pragma unroll
        for (int pq = 0; pq < 4; ++pq) {
            const unsigned r0 = d[pq * 4][g], r1 = d[pq * 4 + 1][g], r2 = d[pq * 4 + 2][g], r3 = d[pq * 4 + 3][g];
            const unsigned t0 = __builtin_amdgcn_perm(r1, r0, 0x05010400), t1 = __builtin_amdgcn_perm(r1, r0, 0x07030602);
            const unsigned t2 = __builtin_amdgcn_perm(r3, r2, 0x05010400), t3 = __builtin_amdgcn_perm(r3, r2, 0x07030602);
            q4[0][pq] = __builtin_amdgcn_perm(t2, t0, 0x05040100); q4[1][pq] = __builtin_amdgcn_perm(t2, t0, 0x07060302);
            q4[2][pq] = __builtin_amdgcn_perm(t3, t1, 0x05040100); q4[3][pq] = __builtin_amdgcn_perm(t3, t1, 0x07060302);
        }
#pragma unroll
        for (int i = 0; i < 4; ++i) t[(i & 1) * 64 + 2 * g + (i >> 1)] = q4[i];
    }
}

__device__ __forceinline__ void issue_dma(const u32x4s& srdx, const u32x4s& srdw, unsigned src, unsigned delta, unsigned ldsb, int lane) {
    const unsigned soff = src & 0x7fffffffu;
    unsigned tmp;
    if ((int)src < 0) {
        const unsigned vo = (lane >> 2) * KS + ((lane & 3) ^ ((0 - (lane >> 4)) & 3)) * 16;
#pragma unroll
        for (int i = 0; i < 16; ++i) {
            const int hi = i >> 3, wr = (i >> 2) & 1, f = i & 3;
            asm volatile("s_add_u32 m0, %4, %6\n\ts_xor_b32 %0, %3, %5\n\tbuffer_load_dwordx4 %1, %2, %0 offen lds"
                         : "=&s"(tmp) : "v"(vo), "s"(srdx), "s"(soff), "s"(ldsb), "n"((wr << 16) | (((4 * hi) ^ (f ^ wr)) << 13)), "n"(i * 1024) : "memory", "scc");
        }
    } else {
        const int d = (int)delta;
        const unsigned vo = ((lane >> 3) & 3) * 16 * N + (lane & 7) * 16 + (lane < 32 ? (d < 0 ? -d : 0) : (d > 0 ? d : 0));
#pragma unroll
        for (int i = 0; i < 16; ++i)
            asm volatile("s_add_u32 m0, %4, %6\n\ts_add_u32 %0, %3, %5\n\tbuffer_load_dwordx4 %1, %2, %0 offen lds"
                         : "=&s"(tmp) : "v"(vo), "s"(srdw), "s"(soff), "s"(ldsb), "n"(i * N), "n"(i * 1024) : "memory", "scc");
    }
}

template <int NN> __device__ __forceinline__ void vm_wait() { asm volatile("s_waitcnt vmcnt(%0)" :: "n"(NN) : "memory"); }
template <int B> __device__ __forceinline__ void store_batch(u32x4 (&o)[16], int g, const unsigned char* bs0, const unsigned char* bs1, unsigned voff) {
    const int i0 = g * B;
    const unsigned char* bb = (i0 < 8 ? bs0 : bs1) + ((i0 & 4) ? 0 : -4096);
    if (B == 4) {
        asm volatile("global_store_dwordx4 %0, %1, %5 sc0 sc1\n\t" "global_store_dwordx4 %0, %2, %5 offset:1024 sc0 sc1\n\t"
                     "global_store_dwordx4 %0, %3, %5 offset:2048 sc0 sc1\n\t" "global_store_dwordx4 %0, %4, %5 offset:3072 sc0 sc1\n\ts_nop 1"
                     :: "v"(voff), "v"(o[i0]), "v"(o[i0 + 1]), "v"(o[i0 + 2]), "v"(o[i0 + 3]), "s"(bb) : "memory");
    } else if (B == 1) {
        if ((i0 & 3) == 0) asm volatile("global_store_dwordx4 %0, %1, %2 sc0 sc1\n\ts_nop 1" :: "v"(voff), "v"(o[i0]), "s"(bb) : "memory");
        else if ((i0 & 3) == 1) asm volatile("global_store_dwordx4 %0, %1, %2 offset:1024 sc0 sc1\n\ts_nop 1" :: "v"(voff), "v"(o[i0]), "s"(bb) : "memory");
        else if ((i0 & 3) == 2) asm volatile("global_store_dwordx4 %0, %1, %2 offset:2048 sc0 sc1\n\ts_nop 1" :: "v"(voff), "v"(o[i0]), "s"(bb) : "memory");
        else asm volatile("global_store_dwordx4 %0, %1, %2 offset:3072 sc0 sc1\n\ts_nop 1" :: "v"(voff), "v"(o[i0]), "s"(bb) : "memory");
    } else if (i0 & 2) {
        asm volatile("global_store_dwordx4 %0, %1, %3 offset:2048 sc0 sc1\n\t" "global_store_dwordx4 %0, %2, %3 offset:3072 sc0 sc1\n\ts_nop 1"
                     :: "v"(voff), "v"(o[i0]), "v"(o[i0 + 1]), "s"(bb) : "memory");
    } else {
        asm volatile("global_store_dwordx4 %0, %1, %3 sc0 sc1\n\t" "global_store_dwordx4 %0, %2, %3 offset:1024 sc0 sc1\n\ts_nop 1"
                     :: "v"(voff), "v"(o[i0]), "v"(o[i0 + 1]), "s"(bb) : "memory");
    }
}
#define SIGNAL(cw) asm volatile("global_atomic_add %0, %1, %2 sc1" :: "v"(((cw) & 0xffu) * 1024u), "v"((cw) >> 8), "s"(dflag) : "memory")

template <int W> __device__ __forceinline__ void push_role(const Params& p, int b, int wave, lds_u32x4* lds) {
    constexpr int NS = 6;
    lds_u32* const tsrc = (lds_u32*)(lds + NS * 1024);
    lds_u32* const tdel = tsrc + NUNITS;
    lds_u32* const tds0 = tdel + NUNITS;
    lds_u32* const tds1 = tds0 + NUNITS;
    lds_u32* const tcw = tds1 + NUNITS;
    lds_u32x4* const meta = lds + NS * 1024 + 800;
    lds_u32* const tkr = (lds_u32*)(meta + 8);
    const int lane = __builtin_amdgcn_mbcnt_hi(~0u, __builtin_amdgcn_mbcnt_lo(~0u, 0u));
    unsigned char* const ws = p.self;
    const int q = (b & 3) + (b >> 3) * 4;
    const int par = (b >> 2) & 1;
    const u32x4s srdx = make_srd(p.x, M * KS), srdw = make_srd(p.w, KS * N);
    if (wave == 0) {
        unsigned long long sv;
        asm volatile("s_mov_b64 %0, exec\n\ts_mov_b64 exec, 1\n\tv_mov_b32 v254, 0\n\tv_mov_b32 v253, 1\n\tglobal_atomic_add v255, v254, v253, %1 sc0\n\ts_mov_b64 exec, %0"
                     : "=&s"(sv) : "s"(ws + WS_TICKET + b * 128) : "v253", "v254", "v255", "memory");
    }
    unsigned uid = 8 * wave + q;
    unsigned src = 0x80000000u | ((uid & 15) << 17) | ((uid & 3) << 14) | ((4 + (uid >> 4)) << 6);
    unsigned ud0 = ((uid & 15) << 20) + (4 + (uid >> 4)) * (4 * 32768) + 1u, ud1 = ud0 - 1u + 16384u, ucw = (uid >> 4) | (((uid & 15) == 0 ? 128u - 20u + 1u : 1u) << 8);
    issue_dma(srdx, srdw, src, 0, wave * 16384, lane);
    const int dsel = par * 4 + ((wave + 1) & 3);
    const bool nostore = dsel == 0;
    unsigned char* const dws = p.nws[dsel];
    const int idx = (p.nidx >> (4 * dsel)) & 7;
    if (lane == 0) { u32x4 m; m.x = ud0; m.y = ud1; m.z = ucw; m.w = 1; meta[wave] = m; }
    for (int U = wave * 64 + lane; U < NUNITS; U += THREADS) {
        const Unit un = decode_unit(U);
        tsrc[U] = un.src; tdel[U] = un.delta; tds0[U] = un.dst0; tds1[U] = un.dst1; tcw[U] = un.cw;
    }
    asm volatile("s_waitcnt vmcnt(0)" ::: "memory");
    if (wave == 0) { unsigned tk; asm volatile("v_readfirstlane_b32 %0, v255" : "=s"(tk) :: "memory"); if (lane == 0) tkr[16] = tk + 1; }
    asm volatile("s_waitcnt lgkmcnt(0)" ::: "memory");
    __builtin_amdgcn_s_barrier();
    asm volatile("" ::: "memory");
    const unsigned call = RFL(tkr[16]);
    u32x4 oa[16], ob[16];
    u32x4 mt = meta[0], mtn = meta[1];
    const int xlane = (lane & 15) * 4 + ((lane >> 4) ^ ((0 - ((lane & 15) >> 2)) & 3));
#pragma unroll
    for (int i = 0; i < 16; ++i) oa[i] = lds[i * 64 + xlane];
    unsigned char* const dstg = dws + WS_STG + (size_t)(call & 1) * STG_BYTES + 4096 + (idx & 1) * 8192;
    unsigned char* const dflag = dws + WS_FLAGS + idx * 128;
    const unsigned char* const ctr = ws + WS_PCTR + par * 128;
    const unsigned base = (call - 1) * NUNITS;
    const unsigned voff = lane * 16;
    unsigned pa = 0xffffffffu;
    int sl_dma = 4, sl_pub = 2, sl_rd = 1;
    for (int t2 = 0;; t2 += 2) {
        bool done = false;
#pragma unroll
        for (int hh = 0; hh < 2; ++hh) {
            const int t = t2 + hh;
            const unsigned m0 = RFL(mt.x), m1 = RFL(mt.y), cw = RFL(mt.z);
            const unsigned po = (unsigned)(idx >> 1) << (14 + (m0 & 1));
            const unsigned char* bs0 = dstg + ((m0 & ~1u) + po);
            const unsigned char* bs1 = dstg + (m1 + po);
#pragma unroll
            for (int g = 0; g < 16 / PB; ++g) {
                constexpr int GS = (W + PB - 1) / PB;
                if (nostore) { if (g == 0) vm_wait<0>(); }
                else if (g <= GS) vm_wait<W>();
                else if ((t & 3) == wave) vm_wait<W + 18>();
                else vm_wait<W + 2>();
                if (g == GS) {
                    if (pa != 0xffffffffu && lane == 0 && !nostore) SIGNAL(pa);
                    pa = cw;
                    if (wave == 0) {
                        if (t >= 1) { unsigned tk; asm volatile("v_readfirstlane_b32 %0, v255" : "=s"(tk) :: "memory"); if (lane == 0) tkr[(t + TA) & 15] = 8 * (TA + 1) + (tk - base); }
                        unsigned long long sv;
                        asm volatile("s_mov_b64 %0, exec\n\ts_mov_b64 exec, 1\n\tv_mov_b32 v254, 0\n\tv_mov_b32 v253, 1\n\tglobal_atomic_add v255, v254, v253, %1 sc0\n\ts_mov_b64 exec, %0"
                                     : "=&s"(sv) : "s"(ctr) : "v253", "v254", "v255", "memory");
                    }
                    if ((t & 3) == wave) {
                        unsigned del = 0;
                        uid = t + 4 <= TA ? 8 * (t + 4) + q : RFL(tkr[(t + 4) & 15]);
                        if (uid < NUNITS) { src = RFL(tsrc[uid]); del = RFL(tdel[uid]); ud0 = RFL(tds0[uid]); ud1 = RFL(tds1[uid]); ucw = RFL(tcw[uid]); }
                        if (uid < NUNITS) issue_dma(srdx, srdw, src, del, sl_dma * 16384, lane);
                    }
                }
                if (!nostore) { if (hh) store_batch<PB>(ob, g, bs0, bs1, voff); else store_batch<PB>(oa, g, bs0, bs1, voff); }
                if (g == GS + RET_D) {
                    if ((t & 3) == ((wave + 2) & 3) && t >= 2) {
                        u32x4 m; m.x = 0; m.y = 0; m.z = 0; m.w = 0;
                        if (uid < NUNITS) {
                            if ((int)src >= 0) retile_w(lds + sl_pub * 1024, lane);
                            m.x = ud0; m.y = ud1; m.z = ucw; m.w = 1;
                        }
                        if (lane == 0) meta[(t + 2) & 7] = m;
                    }
                }
            }
            asm volatile("s_waitcnt lgkmcnt(0)" ::: "memory");
            __builtin_amdgcn_s_barrier();
            asm volatile("" ::: "memory");
            mt = mtn;
            mtn = meta[(t + 2) & 7];
            const int rl = sl_rd * 1024 + ((RFL(mt.x) & 1) ? xlane : lane);
            if (hh) {
#pragma unroll
                for (int i = 0; i < 16; ++i) oa[i] = lds[rl + i * 64];
            } else {
#pragma unroll
                for (int i = 0; i < 16; ++i) ob[i] = lds[rl + i * 64];
            }
            sl_dma = sl_dma == NS - 1 ? 0 : sl_dma + 1; sl_pub = sl_pub == NS - 1 ? 0 : sl_pub + 1; sl_rd = sl_rd == NS - 1 ? 0 : sl_rd + 1;
            if (RFL(mt.w) == 0) { done = true; break; }
        }
        if (done) break;
    }
    asm volatile("s_waitcnt vmcnt(0)" ::: "memory");
    pin16(oa); pin16(ob);
    if (lane == 0) {
        if (!nostore) SIGNAL(pa);
    }
    asm volatile("s_waitcnt vmcnt(0)" ::: "memory");
}

__global__ __launch_bounds__(THREADS, 1) void fused_gemm_ag(Params p) {
    extern __shared__ __attribute__((aligned(16))) unsigned char smem[];
    const int b = blockIdx.x;
    const unsigned wave = RFL(threadIdx.x >> 6);
    if (b < NCOMM) {
        push_role<PW>(p, b, (int)wave, (lds_u32x4*)smem);
        return;
    }
    unsigned char* const ws = p.self;
    if (threadIdx.x == 0) {
        const unsigned t = __hip_atomic_fetch_add((unsigned*)(ws + WS_TICKET + b * 128), 1u, __ATOMIC_RELAXED, __HIP_MEMORY_SCOPE_AGENT);
        *(lds_u32*)smem = t + 1;
    }
    __syncthreads();
    const unsigned call = RFL(*(lds_u32*)smem);
    __syncthreads();
    const int x = b & 7, s = (b - NCOMM) >> 3;
    int pm, g;
    if (s < 28) { pm = (x >> 1) * 4 + (s & 3); g = (x & 1) * 8 + (s >> 2); }
    else { pm = (x >> 1) * 4 + (x & 1) * 2 + (s - 28); g = 7; }
    const int cls = RFL(g >> 3);
    const unsigned tl0 = blk(g, 0) | (blk(g, 1) << 8) | (blk(g, 2) << 16) | (blk(g, 3) << 24);
    const unsigned tl1 = blk(g, 4) | (blk(g, 5) << 8) | (blk(g, 6) << 16) | (blk(g, 7) << 24);
    const unsigned tl2 = cls == 0 ? blk(g, 8) : 0;
    {
        const int U = RFL((b - NCOMM) * 4 + (int)wave);
        __builtin_amdgcn_s_sleep(64);
        if (U < NUNITS) {
            lds_u32x4* const lds = (lds_u32x4*)smem;
            const int lane = __builtin_amdgcn_mbcnt_hi(~0u, __builtin_amdgcn_mbcnt_lo(~0u, 0u));
            const Unit un = decode_unit(U);
            const unsigned src = RFL(un.src), del = RFL(un.delta), m0 = RFL(un.dst0), m1 = RFL(un.dst1), cw = RFL(un.cw);
            const u32x4s srdx = make_srd(p.x, M * KS), srdw = make_srd(p.w, KS * N);
            issue_dma(srdx, srdw, src, del, wave * 16384, lane);
            vm_wait<0>();
            if ((int)src >= 0) retile_w(lds + wave * 1024, lane);
            asm volatile("s_waitcnt lgkmcnt(0)" ::: "memory");
            const int xlane = (lane & 15) * 4 + ((lane >> 4) ^ ((0 - ((lane & 15) >> 2)) & 3));
            const int rl = wave * 1024 + ((m0 & 1) ? xlane : lane);
            u32x4 o[16];
#pragma unroll
            for (int i = 0; i < 16; ++i) o[i] = lds[rl + i * 64];
            const unsigned char* const dstg = ws + WS_STG + (size_t)(call & 1) * STG_BYTES + 4096;
            const unsigned char* bs0 = dstg + (m0 & ~1u);
            const unsigned char* bs1 = dstg + m1;
#pragma unroll
            for (int g4 = 0; g4 < 4; ++g4) store_batch<4>(o, g4, bs0, bs1, lane * 16);
            vm_wait<0>();
            const unsigned char* const dflag = ws + WS_FLAGS;
            if (lane == 0) SIGNAL(cw);
            vm_wait<0>();
        }
        __syncthreads();
    }
    const unsigned long long wp = (unsigned long long)ws, op = (unsigned long long)p.out;
    asm volatile(GEMM_ASM
        :
        : [ws_lo] "s"((unsigned)wp), [ws_hi] "s"((unsigned)(wp >> 32)), [out_lo] "s"((unsigned)op), [out_hi] "s"((unsigned)(op >> 32)),
          [wave] "s"(wave), [tl0] "s"(tl0), [tl1] "s"(tl1), [tl2] "s"(tl2), [pm] "s"(pm), [call] "s"(call),
          [nwin] "s"(9 - cls), [start_odd] "s"(1 - cls), [fbase] "s"(cls * 40 * 1024),
          [psx] "s"(p.sx), [psw] "s"(p.sw)
        : GEMM_CLOB);
    __builtin_amdgcn_endpgm();
}
}

extern "C" void kernel_launch(int rank, int nranks, ncclComm_t comm,
                              hipStream_t stream,
                              void* const* d_in, const int* in_sizes, int n_in,
                              void* d_out, int out_size,
                              const int* in_dtypes, int out_dtype,
                              void* const* d_out_all,
                              void* const* d_ws_all, size_t ws_bytes) {
    (void)comm; (void)in_dtypes; (void)out_dtype; (void)d_out_all;
    if (nranks != NR || rank < 0 || rank >= NR || n_in != 4 || ws_bytes < WS_NEED || out_size != M * N ||
        in_sizes[0] != M * KS || in_sizes[1] != KS * N) {
        fprintf(stderr, "kernel_launch: unexpected problem shape\n");
        return;
    }
    static bool init = false;
    if (!init) {
        HIP_CHECK(hipFuncSetAttribute((const void*)fused_gemm_ag, hipFuncAttributeMaxDynamicSharedMemorySize, LDS_BYTES));
        init = true;
    }
    const size_t base = (ws_bytes - WS_NEED) & ~(size_t)((2u << 20) - 1);
    Params p;
    p.self = (unsigned char*)d_ws_all[rank] + base;
    p.nidx = 0;
    for (int i = 0; i < NR; ++i) {
        const int dest = near_dest(rank, i);
        p.nws[i] = (unsigned char*)d_ws_all[dest] + base;
        p.nidx |= (unsigned)((rank - dest) & (NR - 1)) << (4 * i);
    }
    p.out = (float*)d_out;
    p.x = (const unsigned char*)d_in[0];
    p.w = (const unsigned char*)d_in[1];
    p.sx = (const float*)d_in[2];
    p.sw = (const float*)d_in[3];
    p.pad = 0;
    hipLaunchKernelGGL(fused_gemm_ag, dim3(GRID), dim3(THREADS), LDS_BYTES, stream, p);
    HIP_CHECK(hipGetLastError());
}
```
